```python
import numpy as np

kernel_name = "gpu_ag_gemm_m8192_k8192_n8192_f32_gelu"


ABSMAX_RTOL = 0.001

COMPUTE_DTYPE = "float32"
EPILOGUE = "gelu"
IN_QUANT = None
OUT_QUANT = None
IN_DTYPES = {"x": "float32", "w_mat": "float32"}
WORLD_SIZE = 8

def setup_inputs():

    x = np.random.randn(8192, 8192).astype(np.float32)
    w_mat = (np.random.randn(8192, 8192) / np.sqrt(8192)).astype(np.float32)
    return {"x": x, "w_mat": w_mat}

def _epilogue(y):
    c = 0.7978845608028654
    return (0.5 * y * (1.0 + np.tanh(c * (y + 0.044715 * y ** 3)))).astype(np.float32)

def reference(x, w_mat):
    y = (x @ w_mat).astype(np.float32)
    return _epilogue(y)
```

```cpp
#include <hip/hip_runtime.h>
#include <rccl/rccl.h>
#include <cstdio>
#include <cstdint>

typedef unsigned u32;
typedef unsigned long long u64;
typedef u32 u32x4 __attribute__((ext_vector_type(4)));
typedef float f32x4 __attribute__((ext_vector_type(4)));
typedef _Float16 f16x8 __attribute__((ext_vector_type(8)));
typedef __amdgpu_buffer_rsrc_t rsrc_t;
#define LAS __attribute__((address_space(3)))
#define GAS __attribute__((address_space(1)))

namespace {

constexpr int NR = 8;
constexpr int KD = 8192;
constexpr int NS = 1024;
constexpr int NBLK = 64;
#define P_FI 47
constexpr int FI = P_FI;
constexpr int F2 = 16;
constexpr int FI1 = FI - F2;
constexpr int FX = 64 - FI;
constexpr int RF1 = 208, RF2 = 48, CF = 128;
static_assert(FI + 3 * FI1 + 4 * FX == RF1, "stage-1 rows");

#define P_NWI 20
#define P_CWAVES 4
constexpr int NCOMM = 48;
constexpr int NWG_I = P_NWI;
constexpr int NWG_C = NCOMM - NWG_I;
constexpr int CWAVES = P_CWAVES;
constexpr int GRID = 256, BLOCK = 512;

constexpr u32 OFF_SEQ  = 0;
constexpr u32 OFF_TMO  = 16384;
constexpr u32 OFF_DONE = 20480;
constexpr u32 OFF_OCNT = 32768;
constexpr u32 OFF_CNT  = 131072;
constexpr u32 CNT_BLK  = 24 * 128;
constexpr u32 OFF_DIAG = 524288;
constexpr u32 A1_BYTES = (u32)NBLK * RF1 * 4096;
constexpr u32 A2_BYTES = (u32)NBLK * RF2 * 4096;
constexpr u32 B_BYTES  = (u32)NBLK * CF * 4096;
constexpr u32 PAR_BYTES = A1_BYTES + A2_BYTES + B_BYTES;
constexpr u32 OFF_PAR  = 1u << 20;
constexpr u32 OFF_STG  = OFF_PAR + 2 * PAR_BYTES;
constexpr u32 STG_BYTES = 104u * 131072u;
constexpr u32 WS_NEED  = OFF_STG + STG_BYTES;
constexpr u32 OUT_BYTES = (u32)KD * NS * 4;
static_assert(OFF_CNT + NBLK * CNT_BLK <= OFF_DIAG && OFF_OCNT + 416 * 128 <= OFF_CNT, "control area");

constexpr u32 DONE_PER_CALL = 26 * 16 + 24 * 8;
constexpr u64 DEADLINE = 40000000ull;

constexpr int BUF_BYTES = 49152;
constexpr int LDS_BYTES = 3 * BUF_BYTES;

struct Params {
    const float* x;
    const float* w;
    char* ws[NR];
    char* out[NR];
    int rank;
    int pad;
};

__device__ __forceinline__ u32 rfl(u32 v) { return __builtin_amdgcn_readfirstlane(v); }
__device__ __forceinline__ rsrc_t mk_rsrc(const void* p, u32 bytes) {
    const u64 a = (u64)p;
    const u64 b = ((u64)rfl((u32)(a >> 32)) << 32) | rfl((u32)a);
    return __builtin_amdgcn_make_buffer_rsrc((void*)b, (short)0, (int)bytes, 0x00020000);
}
__device__ __forceinline__ u32 ld_sys(const char* p) {
    return __hip_atomic_load((const GAS u32*)p, __ATOMIC_RELAXED, __HIP_MEMORY_SCOPE_SYSTEM);
}
__device__ __forceinline__ u32 ld_agent(const char* p) {
    return __hip_atomic_load((const GAS u32*)p, __ATOMIC_RELAXED, __HIP_MEMORY_SCOPE_AGENT);
}
__device__ __forceinline__ void add_sys(char* p) {
    __hip_atomic_fetch_add((GAS u32*)p, 1u, __ATOMIC_RELAXED, __HIP_MEMORY_SCOPE_SYSTEM);
}
#define WAIT_VM0() asm volatile("s_waitcnt vmcnt(0)" ::: "memory")

__device__ __forceinline__ void spin_ge(const char* p, u32 need, char* tmo, u32 call) {
    const u64 t0 = __builtin_amdgcn_s_memrealtime();
    for (;;) {
        const u32 v = ld_sys(p);
        if (__builtin_amdgcn_ballot_w64(v < need) == 0) break;
        if (rfl(ld_agent(tmo)) == call || __builtin_amdgcn_s_memrealtime() - t0 > DEADLINE) {
            __hip_atomic_store((GAS u32*)tmo, call, __ATOMIC_RELAXED, __HIP_MEMORY_SCOPE_AGENT);
            break;
        }
        __builtin_amdgcn_s_sleep(2);
    }
    asm volatile("" ::: "memory");
}

#define STAMP(slot) do {} while (0)

__device__ __forceinline__ u32x4 cvt8(f32x4 a, f32x4 b) {
    f16x8 h;
    h[0] = (_Float16)a[0]; h[1] = (_Float16)a[1]; h[2] = (_Float16)a[2]; h[3] = (_Float16)a[3];
    h[4] = (_Float16)b[0]; h[5] = (_Float16)b[1]; h[6] = (_Float16)b[2]; h[7] = (_Float16)b[3];
    return __builtin_bit_cast(u32x4, h);
}
__device__ __forceinline__ void st_sys(u32x4 v, rsrc_t r, u32 voff, u32 soff) {
    __builtin_amdgcn_raw_buffer_store_b128(v, r, voff + soff, 0, 17);
}
__device__ __forceinline__ float gelu(float y) {
    const float u = y * (0.7978845608028654f + 0.7978845608028654f * 0.044715f * y * y);
    const float e = __builtin_amdgcn_exp2f(u * -2.8853900817779268f);
    return y * __builtin_amdgcn_rcpf(1.0f + e);
}

__device__ __forceinline__ int row_of_rf1(int rf, int g, int j) {
    if (rf < FI) return (g * 4 + j) * NS + rf * 16;
    rf -= FI;
    if (rf < 3 * FI1) { const int i = rf / FI1; return (g * 4 + ((j + 1 + i) & 3)) * NS + (rf - i * FI1) * 16; }
    rf -= 3 * FI1;
    const int q = rf / FX;
    return ((g ^ 1) * 4 + q) * NS + (FI + rf - q * FX) * 16;
}
__device__ __forceinline__ int src_of_rf1(int rf, int g, int j) { return row_of_rf1(rf, g, j) / NS; }

__device__ __forceinline__ void load_x(f32x4 (&v)[8], const float* x, int frag, int blk, int lane) {
    const float* s = x + (size_t)(frag * 16 + (lane & 15)) * KD + blk * 128 + (lane >> 4) * 8;
#pragma unroll
    for (int i = 0; i < 8; ++i) v[i] = *(const f32x4*)(s + (i >> 1) * 32 + (i & 1) * 4);
}
__device__ __forceinline__ void load_w(f32x4 (&v)[8], const float* w, int cg, int k0, int lane) {
    const float* s = w + (size_t)(k0 + (lane >> 4) * 8) * NS + cg * 64 + (lane & 15) * 4;
#pragma unroll
    for (int i = 0; i < 8; ++i) v[i] = *(const f32x4*)(s + i * NS);
}

__device__ __forceinline__ void comm_intra(const Params& p, int wv, int lane, u32 call) {
    constexpr int W = NWG_I * CWAVES;
    constexpr int N1 = NBLK * FI, NT = N1 + NBLK * F2;
    const int g = p.rank >> 2, j = p.rank & 3;
    const u32 par = OFF_PAR + (call & 1) * PAR_BYTES;
    const rsrc_t r0 = mk_rsrc(p.ws[p.rank], WS_NEED);
    const rsrc_t r1 = mk_rsrc(p.ws[g * 4 + ((j + 3) & 3)], WS_NEED);
    const rsrc_t r2 = mk_rsrc(p.ws[g * 4 + ((j + 2) & 3)], WS_NEED);
    const rsrc_t r3 = mk_rsrc(p.ws[g * 4 + ((j + 1) & 3)], WS_NEED);
    char* sig = p.ws[g * 4 + ((j - lane) & 3)] + OFF_CNT + p.rank * 128;
    const u32 vo = lane * 16;
    f32x4 v[8];
    int e = wv;
    auto decode = [&](int it, int& blk, int& f) {
        if (it < N1) { blk = it / FI; f = it - blk * FI; }
        else { const int q = it - N1; blk = q / F2; f = FI1 + (q - blk * F2); }
    };
    int blk, f;
    if (e < NT) { decode(e, blk, f); load_x(v, p.x, f, blk, lane); }
    for (; e < NT; e += W) {
        u32x4 h[4];
#pragma unroll
        for (int i = 0; i < 4; ++i) h[i] = cvt8(v[2 * i], v[2 * i + 1]);
        const int cb = blk, cf = f;
        if (e + W < NT) { decode(e + W, blk, f); load_x(v, p.x, f, blk, lane); }
        if (e < N1) {
            const u32 base = par + (u32)cb * (RF1 * 4096);
            if (cf < FI1) {
#pragma unroll
                for (int i = 0; i < 4; ++i) {
                    st_sys(h[i], r1, vo + i * 1024, base + (FI + 0 * FI1 + cf) * 4096);
                    st_sys(h[i], r2, vo + i * 1024, base + (FI + 1 * FI1 + cf) * 4096);
                    st_sys(h[i], r3, vo + i * 1024, base + (FI + 2 * FI1 + cf) * 4096);
                }
            }
#pragma unroll
            for (int i = 0; i < 4; ++i) st_sys(h[i], r0, vo + i * 1024, base + cf * 4096);
            WAIT_VM0();
            if (lane < (cf < FI1 ? 4 : 1)) add_sys(sig + cb * CNT_BLK);
        } else {
            const u32 base = par + A1_BYTES + (u32)cb * (RF2 * 4096) + (cf - FI1) * 4096;
#pragma unroll
            for (int i = 0; i < 4; ++i) {
                st_sys(h[i], r1, vo + i * 1024, base + 0 * F2 * 4096);
                st_sys(h[i], r2, vo + i * 1024, base + 1 * F2 * 4096);
                st_sys(h[i], r3, vo + i * 1024, base + 2 * F2 * 4096);
            }
            WAIT_VM0();
            if (lane >= 1 && lane < 4) add_sys(sig + cb * CNT_BLK + 2 * 1024);
        }
    }
    STAMP(2);
}

__device__ __forceinline__ void comm_cross(const Params& p, int wv, int lane, u32 call) {
    constexpr int W = NWG_C * CWAVES;
    constexpr int PER = FX + 16, N1 = NBLK * PER;
    const int g = p.rank >> 2, j = p.rank & 3, og = (g ^ 1) * 4;
    const u32 par = OFF_PAR + (call & 1) * PAR_BYTES;
    const rsrc_t r0 = mk_rsrc(p.ws[p.rank], WS_NEED);
    const rsrc_t c0 = mk_rsrc(p.ws[og + 0], WS_NEED), c1 = mk_rsrc(p.ws[og + 1], WS_NEED);
    const rsrc_t c2 = mk_rsrc(p.ws[og + 2], WS_NEED), c3 = mk_rsrc(p.ws[og + 3], WS_NEED);
    char* sigx = p.ws[og + (lane & 3)] + OFF_CNT + p.rank * 128;
    const u32 vo = lane * 16;
    f32x4 v[8];
    for (int e = wv; e < N1; e += W) {
        const int blk = e / PER, s = e - blk * PER;
        if (s < 16) {
            const int q = (s + j) & 3, cg = q * 4 + (s >> 2);
            const rsrc_t rq = mk_rsrc(p.ws[og + q], WS_NEED);
            const u32 base = par + A1_BYTES + A2_BYTES + (u32)blk * (CF * 4096);
            load_w(v, p.w, cg, blk * 128, lane);
#pragma unroll
            for (int u = 0; u < 4; ++u) {
                u32x4 h[4];
#pragma unroll
                for (int c = 0; c < 4; ++c)
                    h[c] = cvt8((f32x4){v[0][c], v[1][c], v[2][c], v[3][c]}, (f32x4){v[4][c], v[5][c], v[6][c], v[7][c]});
                if (u < 3) load_w(v, p.w, cg, blk * 128 + (u + 1) * 32, lane);
#pragma unroll
                for (int c = 0; c < 4; ++c) {
                    st_sys(h[c], rq, vo + u * 1024, base + (64 + j * 16 + (s >> 2) * 4 + c) * 4096);
                    st_sys(h[c], r0, vo + u * 1024, base + (cg * 4 + c) * 4096);
                }
            }
            WAIT_VM0();
            if (lane < 2) add_sys((lane == 0 ? p.ws[p.rank] : p.ws[og + q]) + OFF_CNT + p.rank * 128 + blk * CNT_BLK + 1024);
        } else {
            const int f = s - 16;
            load_x(v, p.x, FI + f, blk, lane);
            const u32 base = par + (u32)blk * (RF1 * 4096) + (FI + 3 * FI1 + j * FX + f) * 4096;
#pragma unroll
            for (int i = 0; i < 4; ++i) {
                const u32x4 h = cvt8(v[2 * i], v[2 * i + 1]);
                st_sys(h, c0, vo + i * 1024, base); st_sys(h, c1, vo + i * 1024, base);
                st_sys(h, c2, vo + i * 1024, base); st_sys(h, c3, vo + i * 1024, base);
            }
            WAIT_VM0();
            if (lane < 4) add_sys(sigx + blk * CNT_BLK);
        }
    }
    STAMP(1);
    char* tmo = p.ws[p.rank] + OFF_TMO;
    for (int e = wv; e < 104 * 16; e += W) {
        const int q = (e + j) & 3, t = e >> 2;
        const int half = t & 1, m = (t >> 1) & 3, wm = (t >> 3) & 3, rt = t >> 5;
        const int ft = rt * 8 + q * 2 + half;
        spin_ge(p.ws[p.rank] + OFF_OCNT + (ft * 4 + wm) * 128, 2 * call, tmo, call);
        const int rf = wm * 4 + m;
        u32x4 d[8];
        const u32 sb = OFF_STG + (u32)ft * 131072 + rf * 8192;
#pragma unroll
        for (int i = 0; i < 8; ++i) d[i] = __builtin_amdgcn_raw_buffer_load_b128(r0, vo + i * 1024, sb, 17);
        const rsrc_t ro = mk_rsrc(p.out[og + q], OUT_BYTES);
        const u32 ob = (u32)row_of_rf1(rt * 16 + rf, g, j) * 4096 + (j * 256 + half * 128) * 4;
        const u32 vo2 = (lane >> 5) * 4096 + (lane & 31) * 16;
#pragma unroll
        for (int i = 0; i < 8; ++i) st_sys(d[i], ro, vo2, ob + i * 8192);
        WAIT_VM0();
        if (lane == 0) add_sys(p.ws[og + q] + OFF_DONE + p.rank * 128);
    }
    STAMP(2);
}

#define GLDS2(m0v, voff, rs, soff, imm0, imm1) \
    asm volatile("s_mov_b32 m0, %0\n\ts_nop 0\n\tbuffer_load_dwordx4 %1, %2, %3 offen offset:" #imm0 " sc0 sc1 lds\n\t" \
                 "buffer_load_dwordx4 %1, %2, %3 offen offset:" #imm1 " sc0 sc1 lds" \
                 :: "s"(m0v), "v"(voff), "s"(rs), "s"(soff) : "memory")
#define BARRIER() do { __builtin_amdgcn_sched_barrier(0); __builtin_amdgcn_s_barrier(); __builtin_amdgcn_sched_barrier(0); } while (0)

#define SCHED() __builtin_amdgcn_sched_barrier(0)
#define WAIT_LGKM0() __builtin_amdgcn_s_waitcnt(0xC07F)

template <int STAGE>
struct Gemm {
    static constexpr int NT = STAGE == 1 ? 128 : 64;
    static constexpr int NA = STAGE == 1 ? 4 : 2;
    static constexpr int NB = STAGE == 1 ? 4 : 8;
    static constexpr int NM = STAGE == 1 ? 4 : 1;
    static constexpr int NMM = STAGE == 1 ? 16 : 8;
    static constexpr int NRD = NA + NB;
    struct Frags { f16x8 a[NA], b[NB]; };

    rsrc_t rs;
    u32 sa, sb;
    u32 ldsw;
    int wid, lane;

    template <int I> __device__ __forceinline__ void piece(int t, int buf) const {
        const u32 vo = lane * 16;
        const u32 ma = ldsw + buf * BUF_BYTES + wid * 4096, mb = ldsw + buf * BUF_BYTES + 32768 + wid * 2048;
        const u32 a = STAGE == 1 ? sa + (u32)(t >> 1) * (RF1 * 4096) + (t & 1) * 2048 : sa + (u32)t * (RF2 * 4096);
        const u32 b = STAGE == 1 ? sb + (u32)(t >> 1) * (CF * 4096) + (t & 1) * 2048 : sb + (u32)t * (CF * 4096);
        const u32 a2 = STAGE == 1 ? a + 2048 : a;
        if (I == 0) GLDS2(ma, vo, rs, a, 0, 1024);
        if (I == 1) GLDS2(ma, vo, rs, a2, 2048, 3072);
        if (I == 2) GLDS2(mb, vo, rs, b, 0, 1024);
    }
    __device__ __forceinline__ void stage(int t, int buf) const { piece<0>(t, buf); piece<1>(t, buf); piece<2>(t, buf); }

    template <int I> __device__ __forceinline__ void rd(Frags& f, const LAS char* lds, int buf, int half) const {
        const LAS char* base = lds + buf * BUF_BYTES + lane * 16;
        if constexpr (STAGE == 1) {
            const LAS char* pa = base + (wid >> 1) * 8192 + half * 1024;
            const LAS char* pb = base + 32768 + (wid & 1) * 8192 + half * 1024;
            if constexpr (I == 0) f.a[0] = *(const LAS f16x8*)pa;
            else if constexpr (I <= 4) f.b[I - 1] = *(const LAS f16x8*)(pb + (I - 1) * 2048);
            else f.a[I - 4] = *(const LAS f16x8*)(pa + (I - 4) * 2048);
        } else {
            const LAS char* pa = base + wid * 4096 + half * 2048;
            const LAS char* pb = base + 32768 + half * 2048;
            constexpr int h = I / 5, n = I % 5;
            if constexpr (n == 0) f.a[h] = *(const LAS f16x8*)(pa + h * 1024);
            else f.b[h * 4 + n - 1] = *(const LAS f16x8*)(pb + (n - 1) * 4096 + h * 1024);
        }
    }
    template <int I> __device__ __forceinline__ void mm(f32x4 (&acc)[NM][4], const Frags& f) const {
        if constexpr (STAGE == 1) acc[I >> 2][I & 3] = __builtin_amdgcn_mfma_f32_16x16x32_f16(f.a[I >> 2], f.b[I & 3], acc[I >> 2][I & 3], 0, 0, 0);
        else            acc[0][I & 3] = __builtin_amdgcn_mfma_f32_16x16x32_f16(f.a[I >> 2], f.b[I], acc[0][I & 3], 0, 0, 0);
    }
    template <int I, int N> __device__ __forceinline__ void rds(Frags& f, const LAS char* lds, int buf, int half) const {
        if constexpr (I < N) { rd<I>(f, lds, buf, half); rds<I + 1, N>(f, lds, buf, half); }
    }
    static __device__ __forceinline__ int blk_of(int t) { return STAGE == 1 ? (t >> 1) : t; }

    template <bool RD, bool ST>
    __device__ __forceinline__ void half_step(f32x4 (&acc)[NM][4], const Frags& cur, Frags& nxt, const LAS char* lds, int rbuf, int rhalf,
                                              int ts, int bs, u32& fl, u32 cvo, u32 fso) const {
        if constexpr (STAGE == 1) {
            mm<0>(acc, cur);  mm<1>(acc, cur);  if (RD) rds<0, 2>(nxt, lds, rbuf, rhalf); SCHED();
            mm<2>(acc, cur);  mm<3>(acc, cur);  if (RD) rds<2, 4>(nxt, lds, rbuf, rhalf); SCHED();
            mm<4>(acc, cur);  mm<5>(acc, cur);  if (RD) rds<4, 6>(nxt, lds, rbuf, rhalf); SCHED();
            mm<6>(acc, cur);  mm<7>(acc, cur);  if (RD) rds<6, 8>(nxt, lds, rbuf, rhalf); SCHED();
            mm<8>(acc, cur);  mm<9>(acc, cur);  SCHED();
            if (ST) { asm volatile("buffer_load_dword %0, %1, %2, %3 offen sc0 sc1" : "=v"(fl) : "v"(cvo), "s"(rs), "s"(fso) : "memory"); piece<0>(ts, bs); SCHED(); }
            mm<10>(acc, cur); mm<11>(acc, cur); SCHED();
            if (ST) { piece<1>(ts, bs); SCHED(); }
            mm<12>(acc, cur); mm<13>(acc, cur); SCHED();
            if (ST) { piece<2>(ts, bs); SCHED(); }
            mm<14>(acc, cur); mm<15>(acc, cur); SCHED();
        } else {
            mm<0>(acc, cur); if (RD) rds<0, 3>(nxt, lds, rbuf, rhalf); SCHED();
            mm<1>(acc, cur); if (RD) rds<3, 5>(nxt, lds, rbuf, rhalf); SCHED();
            mm<2>(acc, cur); if (RD) rds<5, 8>(nxt, lds, rbuf, rhalf); SCHED();
            mm<3>(acc, cur); if (RD) rds<8, 10>(nxt, lds, rbuf, rhalf); SCHED();
            mm<4>(acc, cur); SCHED();
            if (ST) { asm volatile("buffer_load_dword %0, %1, %2, %3 offen sc0 sc1" : "=v"(fl) : "v"(cvo), "s"(rs), "s"(fso) : "memory"); piece<0>(ts, bs); SCHED(); }
            mm<5>(acc, cur); SCHED();
            if (ST) { piece<1>(ts, bs); SCHED(); }
            mm<6>(acc, cur); SCHED();
            if (ST) { piece<2>(ts, bs); SCHED(); }
            mm<7>(acc, cur); SCHED();
        }
    }

    __device__ __forceinline__ void run(f32x4 (&acc)[NM][4], const LAS char* lds, char* ws, u32 cvo, u32 need, u32 call) const {
        char* tmo = ws + OFF_TMO;
        Frags f0, f1;
        if (wid == 0) spin_ge(ws + cvo + blk_of(0) * CNT_BLK, need, tmo, call);
        BARRIER();
        stage(0, 0);
        if (wid == 0) { spin_ge(ws + cvo + blk_of(1) * CNT_BLK, need, tmo, call); spin_ge(ws + cvo + blk_of(2) * CNT_BLK, need, tmo, call); }
        BARRIER();
        stage(1, 1); stage(2, 2);
        asm volatile("s_waitcnt vmcnt(12)" ::: "memory");
        BARRIER();
        rds<0, NRD>(f0, lds, 0, 0);
        const u32 fvo = wid == 0 ? cvo : WS_NEED, fneed = wid == 0 ? need : 0u;
        u32 fl = 0;
        int buf = 0;
        for (int t = 0; t < NT - 3; ++t) {
            const int nb = buf == 2 ? 0 : buf + 1;
            half_step<true, false>(acc, f0, f1, lds, buf, 1, 0, 0, fl, fvo, 0);
            asm volatile("s_waitcnt vmcnt(6)" : "+v"(fl) :: "memory");
            WAIT_LGKM0();
            if (__builtin_amdgcn_ballot_w64(fl < fneed) != 0) spin_ge(ws + cvo + blk_of(t + 3) * CNT_BLK, need, tmo, call);
            BARRIER();
            half_step<true, true>(acc, f1, f0, lds, nb, 0, t + 3, buf, fl, fvo, blk_of(t + 4 < NT ? t + 4 : t + 3) * CNT_BLK);
            buf = nb;
        }
#pragma unroll
        for (int t = NT - 3; t < NT; ++t) {
            const int nb = buf == 2 ? 0 : buf + 1;
            half_step<true, false>(acc, f0, f1, lds, buf, 1, 0, 0, fl, fvo, 0);
            if (t == NT - 3) asm volatile("s_waitcnt vmcnt(6)" : "+v"(fl) :: "memory");
            else             asm volatile("s_waitcnt vmcnt(0)" : "+v"(fl) :: "memory");
            WAIT_LGKM0();
            BARRIER();
            if (t < NT - 1) half_step<true, false>(acc, f1, f0, lds, nb, 0, 0, 0, fl, fvo, 0);
            else            half_step<false, false>(acc, f1, f0, lds, nb, 0, 0, 0, fl, fvo, 0);
            buf = nb;
        }
    }
};

__device__ __forceinline__ void gemm_role(const Params& p, const LAS char* lds, int xcd, int s, int wid, int lane, u32 call) {
    const int g = p.rank >> 2, j = p.rank & 3, og = (g ^ 1) * 4;
    char* ws = p.ws[p.rank];
    char* tmo = ws + OFF_TMO;
    const u32 par = OFF_PAR + (call & 1) * PAR_BYTES;
    const rsrc_t rs = mk_rsrc(ws, WS_NEED);
    const u32 ldsw = rfl((u32)(size_t)lds);
    const float nanv = __builtin_nanf("");
    {
        const int u = (xcd & 1) * 26 + s, rt = u >> 2, ct = (xcd >> 1) * 4 + (u & 3);
        Gemm<1> G;
        G.rs = rs; G.ldsw = ldsw; G.wid = wid; G.lane = lane;
        G.sa = par + (rt * 16 + wid * 2) * 4096;
        G.sb = par + A1_BYTES + A2_BYTES + (ct * 8 + wid) * 4096;
        const int bsrc = ct < 8 ? p.rank : og + ((ct - 8) >> 1);
        u32 cvo = OFF_CNT + 1024 + bsrc * 128, need = (ct < 8 ? 16u : 4u) * call;
        u32 mask = 0;
        for (int i = 0; i < 16; ++i) mask |= 1u << src_of_rf1(rt * 16 + i, g, j);
        if (lane < 8 && ((mask >> lane) & 1)) {
            cvo = OFF_CNT + lane * 128;
            need = (lane == p.rank ? FI : ((lane >> 2) == g ? FI1 : FX)) * call;
        }
        f32x4 acc[4][4] = {};
        G.run(acc, lds, ws, cvo, need, call);
        STAMP(1);
        const bool bad = rfl(ld_agent(tmo)) == call;
        const int wm = wid >> 1, wn = wid & 1;
        if (ct < 8) {
            const rsrc_t ro = mk_rsrc(p.out[p.rank], OUT_BYTES);
            const u32 vo = (lane >> 4) * 4 * 4096 + (lane & 15) * 16;
#pragma unroll
            for (int m = 0; m < 4; ++m) {
                const u32 ob = (u32)row_of_rf1(rt * 16 + wm * 4 + m, g, j) * 4096 + (ct * 128 + wn * 64) * 4;
#pragma unroll
                for (int r = 0; r < 4; ++r) {
                    f32x4 o = {gelu(acc[m][0][r]), gelu(acc[m][1][r]), gelu(acc[m][2][r]), gelu(acc[m][3][r])};
                    if (bad) o = (f32x4){nanv, nanv, nanv, nanv};
                    __builtin_amdgcn_raw_buffer_store_b128(__builtin_bit_cast(u32x4, o), ro, vo + ob + r * 4096, 0, 0);
                }
            }
        } else {
            const int ft = rt * 8 + (ct - 8);
            const u32 vo = (lane >> 4) * 4 * 512 + (lane & 15) * 16;
#pragma unroll
            for (int m = 0; m < 4; ++m) {
                const u32 ob = OFF_STG + (u32)ft * 131072 + (wm * 64 + m * 16) * 512 + wn * 256;
#pragma unroll
                for (int r = 0; r < 4; ++r) {
                    f32x4 o = {gelu(acc[m][0][r]), gelu(acc[m][1][r]), gelu(acc[m][2][r]), gelu(acc[m][3][r])};
                    if (bad) o = (f32x4){nanv, nanv, nanv, nanv};
                    st_sys(__builtin_bit_cast(u32x4, o), rs, vo, ob + r * 512);
                }
            }
            WAIT_VM0();
            if (lane == 0) add_sys(ws + OFF_OCNT + (ft * 4 + wm) * 128);
        }
        STAMP(2);
    }
    if (s < 24) {
        const int rt = s >> 2, ct = ((xcd & 1) * 16) + (xcd >> 1) * 4 + (s & 3);
        Gemm<2> G;
        G.rs = rs; G.ldsw = ldsw; G.wid = wid; G.lane = lane;
        G.sa = par + A1_BYTES + (rt * 8 + wid) * 4096;
        G.sb = par + A1_BYTES + A2_BYTES + ct * 4 * 4096 + wid * 2048;
        const int asrc = g * 4 + ((j + 1 + (rt >> 1)) & 3);
        const int bsrc = ct < 16 ? p.rank : og + ((ct - 16) >> 2);
        u32 cvo = OFF_CNT + 1024 + bsrc * 128, need = (ct < 16 ? 16u : 4u) * call;
        if (lane == 0) { cvo = OFF_CNT + 2048 + asrc * 128; need = F2 * call; }
        f32x4 acc[1][4] = {};
        G.run(acc, lds, ws, cvo, need, call);
        STAMP(3);
        const bool bad = rfl(ld_agent(tmo)) == call;
        const int dst = ct < 16 ? p.rank : bsrc;
        const rsrc_t ro = mk_rsrc(p.out[dst], OUT_BYTES);
        const u32 vo = (lane >> 4) * 4 * 4096 + (lane & 15) * 16;
        const int row = asrc * NS + (FI1 + (rt & 1) * 8 + wid) * 16;
        const int col = ct < 16 ? ct * 64 : j * 256 + ((ct - 16) & 3) * 64;
        const u32 ob = (u32)row * 4096 + col * 4;
#pragma unroll
        for (int r = 0; r < 4; ++r) {
            f32x4 o = {gelu(acc[0][0][r]), gelu(acc[0][1][r]), gelu(acc[0][2][r]), gelu(acc[0][3][r])};
            if (bad) o = (f32x4){nanv, nanv, nanv, nanv};
            st_sys(__builtin_bit_cast(u32x4, o), ro, vo, ob + r * 4096);
        }
        WAIT_VM0();
        if (ct >= 16) {
            if (lane == 0) add_sys(p.ws[dst] + OFF_DONE + p.rank * 128);
        } else if (xcd == 0 && s == 0 && wid == 0) {
            spin_ge(ws + OFF_DONE + (og + (lane & 3)) * 128, DONE_PER_CALL * call, tmo, call);
            if (rfl(ld_agent(tmo)) == call)
                st_sys(__builtin_bit_cast(u32x4, (f32x4){nanv, nanv, nanv, nanv}), ro, vo, ob);
        }
        STAMP(4);
    }
}

__global__ __launch_bounds__(BLOCK, 2) void ag_gemm_kernel(Params p) {
    __shared__ __attribute__((aligned(1024))) char smem[LDS_BYTES];
    const int lane = threadIdx.x & 63;
    const int wid = rfl(threadIdx.x >> 6);
    const int b = blockIdx.x, xcd = b & 7, slot = b >> 3;
    if (slot < NCOMM / 8 && wid >= CWAVES) return;
    STAMP(0);
    char* seq = p.ws[p.rank] + OFF_SEQ + (b * 8 + wid) * 4;
    const u32 call = rfl(ld_agent(seq)) + 1;
    if (lane == 0) __hip_atomic_store((GAS u32*)seq, call, __ATOMIC_RELAXED, __HIP_MEMORY_SCOPE_AGENT);
    if (slot < NCOMM / 8) {
        if (b < NWG_I) comm_intra(p, wid * NWG_I + b, lane, call);
        else comm_cross(p, wid * NWG_C + (b - NWG_I), lane, call);
    } else {
        gemm_role(p, (const LAS char*)smem, xcd, slot - NCOMM / 8, wid, lane, call);
    }
}

}

extern "C" void kernel_launch(int rank, int nranks, ncclComm_t comm, hipStream_t stream,
                              void* const* d_in, const int* in_sizes, int n_in,
                              void* d_out, int out_size, const int* in_dtypes, int out_dtype,
                              void* const* d_out_all, void* const* d_ws_all, size_t ws_bytes) {
    (void)comm; (void)in_sizes; (void)n_in; (void)d_out; (void)out_size; (void)in_dtypes; (void)out_dtype;
    if (nranks != NR || (unsigned)rank >= (unsigned)NR || ws_bytes < WS_NEED) {
        fprintf(stderr, "unsupported configuration\n");
        return;
    }
    Params p;
    p.x = (const float*)d_in[0];
    p.w = (const float*)d_in[1];
    for (int i = 0; i < NR; ++i) { p.ws[i] = (char*)d_ws_all[i]; p.out[i] = (char*)d_out_all[i]; }
    p.rank = rank;
    p.pad = 0;
    ag_gemm_kernel<<<GRID, BLOCK, 0, stream>>>(p);
    hipError_t e = hipGetLastError();
    if (e != hipSuccess) fprintf(stderr, "launch failed: %s\n", hipGetErrorString(e));
}
```

```cpp
#include <hip/hip_runtime.h>
#include <rccl/rccl.h>
#include <cstdio>
#include <cstdint>

typedef unsigned u32;
typedef unsigned long long u64;
typedef u32 u32x4 __attribute__((ext_vector_type(4)));
typedef float f32x4 __attribute__((ext_vector_type(4)));
typedef _Float16 f16x8 __attribute__((ext_vector_type(8)));
typedef __amdgpu_buffer_rsrc_t rsrc_t;
#define LAS __attribute__((address_space(3)))
#define GAS __attribute__((address_space(1)))

namespace {

constexpr int NR = 8;
constexpr int KD = 8192;
constexpr int NS = 1024;
constexpr int NBLK = 64;
#define P_FI 46
constexpr int FI = P_FI;
constexpr int F2 = 16;
constexpr int FI1 = FI - F2;
constexpr int FX = 64 - FI;
constexpr int RF1 = 208, RF2 = 48, CF = 128;
static_assert(FI + 3 * FI1 + 4 * FX == RF1, "stage-1 rows");

#define P_WI 23
#define P_WC 23
#define P_XA 15
constexpr int NCOMM = 48;
constexpr int WI = P_WI, WC = P_WC;
constexpr int NPULL = 3 * WI + 4 * WC;
constexpr int XA = P_XA;
static_assert(NPULL <= NCOMM * 8 && XA <= FX && XA <= 16, "wave roles");
constexpr int GRID = 256, BLOCK = 512, NGEMM = GRID - NCOMM;
constexpr int RS1 = 64 + 3 * FI1 + 4 * FX;

constexpr u32 OFF_SEQ  = 0;
constexpr u32 OFF_TMO  = 16384;
constexpr u32 OFF_QUE  = 16640;
constexpr u32 OFF_RDY  = 32768;
constexpr u32 OFF_CNT  = 131072;
constexpr u32 CNT_BLK  = 24 * 128;
constexpr u32 OFF_DIAG = 524288;
constexpr u32 A1_BYTES = (u32)NBLK * RS1 * 4096;
constexpr u32 A2_BYTES = (u32)NBLK * RF2 * 4096;
constexpr u32 B_BYTES  = (u32)NBLK * CF * 4096;
constexpr u32 STG_BYTES = 4u * 1024 * 4096;
constexpr u32 PK_BYTES = (u32)NBLK * 128 * 4096;
constexpr u32 REL_A2 = A1_BYTES, REL_B = REL_A2 + A2_BYTES, REL_PK = REL_B + B_BYTES, REL_STG = REL_PK + PK_BYTES;
constexpr u32 PAR_BYTES = REL_STG + STG_BYTES;
constexpr u32 OFF_PAR  = 1u << 20;
constexpr u32 WS_NEED  = OFF_PAR + 2 * PAR_BYTES;
constexpr u32 OUT_BYTES = (u32)KD * NS * 4;
static_assert(OFF_CNT + NBLK * CNT_BLK <= OFF_DIAG && OFF_RDY + 8 * 8192 <= OFF_CNT, "control area");

constexpr u64 DEADLINE = 40000000ull;

constexpr int BUF_BYTES = 49152;
constexpr int LDS_BYTES = 3 * BUF_BYTES;

struct Params {
    const float* x;
    const float* w;
    char* ws[NR];
    char* out[NR];
    int rank;
    int pad;
};

__device__ __forceinline__ u32 rfl(u32 v) { return __builtin_amdgcn_readfirstlane(v); }
__device__ __forceinline__ char* uni(const void* p) {
    const u64 a = (u64)p;
    return (char*)(((u64)rfl((u32)(a >> 32)) << 32) | rfl((u32)a));
}
__device__ __forceinline__ rsrc_t rsrc_u(const void* p, u32 bytes) {
    return __builtin_amdgcn_make_buffer_rsrc((void*)p, (short)0, (int)bytes, 0x00020000);
}
__device__ __forceinline__ rsrc_t mk_rsrc(const void* p, u32 bytes) { return rsrc_u(uni(p), bytes); }
__device__ __forceinline__ u32 ld_sys(const char* p) {
    return __hip_atomic_load((const GAS u32*)p, __ATOMIC_RELAXED, __HIP_MEMORY_SCOPE_SYSTEM);
}
__device__ __forceinline__ u32 ld_agent(const char* p) {
    return __hip_atomic_load((const GAS u32*)p, __ATOMIC_RELAXED, __HIP_MEMORY_SCOPE_AGENT);
}
__device__ __forceinline__ void st_flag(char* p, u32 v) {
    __hip_atomic_store((GAS u32*)p, v, __ATOMIC_RELAXED, __HIP_MEMORY_SCOPE_SYSTEM);
}
__device__ __forceinline__ void add_sys(char* p) {
    __hip_atomic_fetch_add((GAS u32*)p, 1u, __ATOMIC_RELAXED, __HIP_MEMORY_SCOPE_SYSTEM);
}
__device__ __forceinline__ u32 draw(rsrc_t r, u32 voff) {
    u32 t = 1;
    asm volatile("s_nop 0\n\tbuffer_atomic_add %0, %1, %2, 0 offen sc0" : "+v"(t) : "v"(voff), "s"(r) : "memory");
    return t;
}
#define WAIT_VM0() do { asm volatile("s_waitcnt vmcnt(0)" ::: "memory"); __builtin_amdgcn_s_waitcnt(0x0F70); } while (0)
#define WAIT_VM0_T(t) do { asm volatile("s_waitcnt vmcnt(0)" : "+v"(t) :: "memory"); __builtin_amdgcn_s_waitcnt(0x0F70); } while (0)

template <int SLEEP = 2>
__device__ __forceinline__ void spin_ge(const char* p, u32 need, char* tmo, u32 call) {
    const u64 t0 = __builtin_amdgcn_s_memrealtime();
    for (;;) {
        const u32 v = ld_sys(p);
        if (__builtin_amdgcn_ballot_w64(v < need) == 0) break;
        if (rfl(ld_agent(tmo)) == call || __builtin_amdgcn_s_memrealtime() - t0 > DEADLINE) {
            __hip_atomic_store((GAS u32*)tmo, call, __ATOMIC_RELAXED, __HIP_MEMORY_SCOPE_AGENT);
            break;
        }
        __builtin_amdgcn_s_sleep(SLEEP);
    }
    asm volatile("" ::: "memory");
}

#define STAMP_IF(c, slot) do {} while (0)
#define STAMP(slot) do {} while (0)

__device__ __forceinline__ u32x4 cvt8(f32x4 a, f32x4 b) {
    f16x8 h;
    h[0] = (_Float16)a[0]; h[1] = (_Float16)a[1]; h[2] = (_Float16)a[2]; h[3] = (_Float16)a[3];
    h[4] = (_Float16)b[0]; h[5] = (_Float16)b[1]; h[6] = (_Float16)b[2]; h[7] = (_Float16)b[3];
    return __builtin_bit_cast(u32x4, h);
}
__device__ __forceinline__ void st_sys(u32x4 v, rsrc_t r, u32 voff, u32 soff) {
    __builtin_amdgcn_raw_buffer_store_b128(v, r, voff + soff, 0, 17);
}
__device__ __forceinline__ float gelu(float y) {
    const float u = y * (0.7978845608028654f + 0.7978845608028654f * 0.044715f * y * y);
    const float e = __builtin_amdgcn_exp2f(u * -2.8853900817779268f);
    return y * __builtin_amdgcn_rcpf(1.0f + e);
}

__device__ __forceinline__ int row_of_rf1(int rf, int g, int j) {
    if (rf < FI) return (g * 4 + j) * NS + rf * 16;
    rf -= FI;
    if (rf < 3 * FI1) { const int i = rf / FI1; return (g * 4 + ((j + 1 + i) & 3)) * NS + (rf - i * FI1) * 16; }
    rf -= 3 * FI1;
    const int q = rf / FX;
    return ((g ^ 1) * 4 + q) * NS + (FI + rf - q * FX) * 16;
}
__device__ __forceinline__ int src_of_rf1(int rf, int g, int j) { return row_of_rf1(rf, g, j) / NS; }

__device__ __forceinline__ void load_x(f32x4* v, const float* x, int frag, int blk, int lane) {
    const float* s = x + (size_t)(frag * 16 + (lane & 15)) * KD + blk * 128 + (lane >> 4) * 4;
#pragma unroll
    for (int i = 0; i < 8; ++i) v[i] = *(const f32x4*)(s + i * 16);
}
__device__ __forceinline__ void load_w(f32x4* v, const float* w, int cg, int k0, int lane) {
    const float* s = w + (size_t)(k0 + (lane >> 4) * 4) * NS + cg * 64 + (lane & 15) * 4;
#pragma unroll
    for (int i = 0; i < 8; ++i) v[i] = *(const f32x4*)(s + ((i & 3) + (i >> 2) * 16) * NS);
}
__device__ __forceinline__ u32x4 cvt_w(const f32x4* v, int c) {
    return cvt8((f32x4){v[0][c], v[1][c], v[2][c], v[3][c]}, (f32x4){v[4][c], v[5][c], v[6][c], v[7][c]});
}

#define DPP_STEP(op, v, ctrl, rmask) v = op(v, (u32)__builtin_amdgcn_update_dpp(0, (int)v, ctrl, rmask, 0xf, false))
__device__ __forceinline__ u32 umax2(u32 a, u32 b) { return a > b ? a : b; }
__device__ __forceinline__ u32 uadd2(u32 a, u32 b) { return a + b; }
__device__ __forceinline__ u32 wave_max(u32 v) {
    DPP_STEP(umax2, v, 0x111, 0xf); DPP_STEP(umax2, v, 0x112, 0xf); DPP_STEP(umax2, v, 0x114, 0xf); DPP_STEP(umax2, v, 0x118, 0xf);
    DPP_STEP(umax2, v, 0x142, 0xa); DPP_STEP(umax2, v, 0x143, 0xc);
    return __builtin_amdgcn_readlane(v, 63);
}
__device__ __forceinline__ u32 wave_sum(u32 v) {
    DPP_STEP(uadd2, v, 0x111, 0xf); DPP_STEP(uadd2, v, 0x112, 0xf); DPP_STEP(uadd2, v, 0x114, 0xf); DPP_STEP(uadd2, v, 0x118, 0xf);
    DPP_STEP(uadd2, v, 0x142, 0xa); DPP_STEP(uadd2, v, 0x143, 0xc);
    return __builtin_amdgcn_readlane(v, 63);
}

__device__ __forceinline__ void pack_item(const u32x4 (&h)[4], float amax, LAS char* scr, rsrc_t r0, u32 dst, int lane) {
    u32 L[8], H[8];
#pragma unroll
    for (int t = 0; t < 8; ++t) {
        const u32 d0 = h[t >> 1][(t & 1) * 2], d1 = h[t >> 1][(t & 1) * 2 + 1];
        L[t] = __builtin_amdgcn_perm(d1, d0, 0x06040200u);
        H[t] = __builtin_amdgcn_perm(d1, d0, 0x07050301u);
    }
    *(LAS u32x4*)(scr + lane * 32) = (u32x4){H[0], H[1], H[2], H[3]};
    *(LAS u32x4*)(scr + lane * 32 + 16) = (u32x4){H[4], H[5], H[6], H[7]};
    const float wmax = __builtin_bit_cast(float, wave_max(__builtin_bit_cast(u32, amax)));
    const u32 emax = rfl((u32)__builtin_bit_cast(unsigned short, (_Float16)wmax) >> 10) & 31;
    u32 ntop = 0;
#pragma unroll
    for (int t = 0; t < 8; ++t) ntop += __builtin_popcount(((H[t] & 0x7f7f7f7fu) + (0x80u - emax * 4) * 0x01010101u) & 0x80808080u);
    const int top = (int)emax - (wave_sum(ntop) <= 16 ? 1 : 0);
    const u32 lo = top > 7 ? top - 7 : 0, lo4 = lo * 0x04040404u;
    LAS unsigned short* list = (LAS unsigned short*)(scr + 2048);
    list[lane] = (unsigned short)lo;
    u32 F[8], em = 0;
#pragma unroll
    for (int t = 0; t < 8; ++t) {
        const u32 u = ((H[t] | 0x80808080u) - lo4) ^ 0x80808080u;
        const u32 x = u & 0xe0e0e0e0u;
        const u32 nz = (x | (x << 1) | (x << 2)) & 0x80808080u;
        const u32 m = (nz >> 2) - (nz >> 5);
        F[t] = (u & 0x1f1f1f1fu & ~m) | ((H[t] >> 2) & 0x20202020u);
        em = (em >> 1) | nz;
    }
    u32 used = 1;
    for (u64 act = __builtin_amdgcn_ballot_w64(em != 0); act != 0; act = __builtin_amdgcn_ballot_w64(em != 0)) {
        const u32 k = used + __builtin_amdgcn_mbcnt_hi((u32)(act >> 32), __builtin_amdgcn_mbcnt_lo((u32)act, 0));
        const u32 b = __builtin_ctz(em | 0x80000000u), t = b & 7, y = b >> 3;
        const u32 e5 = (*(const LAS unsigned char*)(scr + lane * 32 + t * 4 + y) >> 2) & 31;
        const u32 pos = (t >> 1) * 512 + lane * 8 + (t & 1) * 4 + y;
        list[em != 0 && k < 64 ? k : 64] = (unsigned short)(pos * 32 + e5);
        em &= em - 1;
        used += __builtin_popcountll(act);
        if (used > 64) break;
    }
    asm volatile("" ::: "memory");
    u32 ent = list[lane];
    if (lane == 0 && used > 64) ent = 0xffffu;
    st_sys((u32x4){L[0], L[1], L[2], L[3]}, r0, lane * 16, dst);
    st_sys((u32x4){L[4], L[5], L[6], L[7]}, r0, lane * 16, dst + 1024);
    st_sys((u32x4){F[0] | ((F[3] & 0x03030303u) << 6), F[1] | ((F[3] & 0x0c0c0c0cu) << 4), F[2] | ((F[3] & 0x30303030u) << 2),
                   F[4] | ((F[7] & 0x03030303u) << 6)}, r0, lane * 16, dst + 2048);
    typedef u32 u32x2v __attribute__((ext_vector_type(2)));
    __builtin_amdgcn_raw_buffer_store_b64((u32x2v){F[5] | ((F[7] & 0x0c0c0c0cu) << 4), F[6] | ((F[7] & 0x30303030u) << 2)},
                                          r0, lane * 8 + dst + 3072, 0, 17);
    __builtin_amdgcn_raw_buffer_store_b16((unsigned short)ent, r0, lane * 2 + dst + 3584, 0, 17);
}
__device__ __forceinline__ void unpack_item(u32x4 (&h)[4], u32 p4, u32 p5, u32 ent, u32 lo, LAS char* scr, int lane) {
    const u32 L[8] = {h[0][0], h[0][1], h[0][2], h[0][3], h[1][0], h[1][1], h[1][2], h[1][3]};
    const u32 P[6] = {h[2][0], h[2][1], h[2][2], h[2][3], p4, p5};
    u32 F[8];
#pragma unroll
    for (int z = 0; z < 2; ++z) {
        F[4 * z] = P[3 * z] & 0x3f3f3f3fu; F[4 * z + 1] = P[3 * z + 1] & 0x3f3f3f3fu; F[4 * z + 2] = P[3 * z + 2] & 0x3f3f3f3fu;
        F[4 * z + 3] = ((P[3 * z] >> 6) & 0x03030303u) | ((P[3 * z + 1] >> 4) & 0x0c0c0c0cu) | ((P[3 * z + 2] >> 2) & 0x30303030u);
    }
    const u32 lo4 = lo * 0x04040404u;
#pragma unroll
    for (int t = 0; t < 8; ++t) {
        const u32 sg = F[t] & 0x20202020u, H = F[t] + ((sg << 1) + sg) + lo4;
        h[t >> 1][(t & 1) * 2] = __builtin_amdgcn_perm(H, L[t], 0x05010400u);
        h[t >> 1][(t & 1) * 2 + 1] = __builtin_amdgcn_perm(H, L[t], 0x07030602u);
    }
#pragma unroll
    for (int i = 0; i < 4; ++i) *(LAS u32x4*)(scr + i * 1024 + lane * 16) = h[i];
    __hip_atomic_fetch_add((LAS u32*)(scr + (ent >> 6) * 4), ((ent & 31) - lo) << (10 + ((ent >> 1) & 16)),
                           __ATOMIC_RELAXED, __HIP_MEMORY_SCOPE_WORKGROUP);
#pragma unroll
    for (int i = 0; i < 4; ++i) h[i] = *(const LAS u32x4*)(scr + i * 1024 + lane * 16);
}

__device__ __forceinline__ void convert_local(const Params& p, LAS char* scr, int wg, int wid, int lane, u32 call) {
    const int rank = rfl(p.rank);
    const u32 par = OFF_PAR + (call & 1) * PAR_BYTES;
    const rsrc_t r0 = mk_rsrc(p.ws[rank], WS_NEED);
    char* flag = p.ws[lane & 7] + OFF_RDY + rank * 8192;
    for (int u = wg; u < NBLK * 16; u += NGEMM) {
        const int blk = u >> 4, grp = u & 15, it = grp * 8 + wid;
        f32x4 v[8];
        u32x4 h[4];
        u32 base, step;
        if (grp < 8) {
            load_x(v, p.x, it, blk, lane);
            base = par + (u32)blk * (RS1 * 4096) + it * 4096; step = 1024;
#pragma unroll
            for (int i = 0; i < 4; ++i) h[i] = cvt8(v[2 * i], v[2 * i + 1]);
        } else {
            const int cg = (it - 64) >> 2, ks = wid & 3;
            load_w(v, p.w, cg, blk * 128 + ks * 32, lane);
            base = par + REL_B + (u32)blk * (CF * 4096) + cg * 4 * 4096 + ks * 1024; step = 4096;
#pragma unroll
            for (int c = 0; c < 4; ++c) h[c] = cvt_w(v, c);
        }
        float amax = 0.0f;
#pragma unroll
        for (int i = 0; i < 8; ++i) amax = fmaxf(fmaxf(fmaxf(amax, fabsf(v[i][0])), fmaxf(fabsf(v[i][1]), fabsf(v[i][2]))), fabsf(v[i][3]));
#pragma unroll
        for (int i = 0; i < 4; ++i) st_sys(h[i], r0, lane * 16, base + i * step);
        pack_item(h, amax, scr, r0, par + REL_PK + (u32)(blk * 128 + it) * 4096, lane);
        WAIT_VM0();
        __builtin_amdgcn_s_barrier();
        STAMP_IF(u == wg, 6);
        if (wid == 0) {
            if (lane < 8 && lane != rank) st_flag(flag + u * 4, call);
            if (lane == 8) add_sys(p.ws[rank] + OFF_CNT + blk * CNT_BLK + (grp < 8 ? 0 : 1024) + rank * 128);
        }
    }
}

__device__ __forceinline__ void pull_role(const Params& p, LAS char* scr, int slot, int wv, int lane, u32 call) {
    const int rank = rfl(p.rank);
    const int g = rank >> 2, j = rank & 3;
    const bool cross = slot >= 3;
    const int q = slot - 3, peer = cross ? (g ^ 1) * 4 + q : g * 4 + ((j + 1 + slot) & 3);
    constexpr int N1 = NBLK * FI1, NIN = N1 + NBLK * F2;
    constexpr int PER = 16 + XA, NP = NBLK * PER, NX = NBLK * (16 + FX);
    const int W = cross ? WC : WI, NT = cross ? NX + 1024 : NIN;
    const u32 par = OFF_PAR + (call & 1) * PAR_BYTES;
    char* ws = uni(p.ws[rank]);
    char* wp = uni(p.ws[peer]);
    char* outp = uni(p.out[rank]);
    const rsrc_t r0 = rsrc_u(ws, WS_NEED);
    const char* rdy = ws + OFF_RDY + peer * 8192;
    const u32 vo = lane * 16, vout = (lane >> 4) * 4 * 4096 + (lane & 15) * 16;
    const u32 head = lane == 0 ? OFF_QUE + slot * 256 : WS_NEED, tbase = 3 * W - (call - 1) * NT;
    const bool nowait = (p.pad & 4) != 0;
    struct Item { u32 src, raw, dst, step, fi, cnt; };
    auto decode = [&](int it) {
        Item m;
        if (it >= NT) it = NT - 1;
        if (cross && it >= NX) {
            const int o = it - NX;
            m.src = m.raw = par + REL_STG + (u32)(j * 1024 + o) * 4096; m.cnt = 0; m.step = 4096;
            int row, col;
            if (o < 832) {
                row = row_of_rf1((o >> 6) * 16 + ((o >> 3) & 3) * 4 + (o & 3), g ^ 1, q);
                col = ((o >> 5) & 1) * 128 + ((o >> 2) & 1) * 64; m.fi = 1024 + (o >> 2);
            } else {
                const int z = o - 832, rt = z >> 5;
                row = ((g ^ 1) * 4 + ((q + 1 + (rt >> 1)) & 3)) * NS + (FI1 + (rt & 1) * 8 + (z & 7)) * 16;
                col = ((z >> 3) & 3) * 64; m.fi = 1024 + 208 + z;
            }
            m.dst = (u32)row * 4096 + (q * 256 + col) * 4;
            return m;
        }
        int blk, s;
        if (!cross) {
            if (it < N1) { blk = it / FI1; s = it - blk * FI1; }
            else { blk = (it - N1) / F2; s = FI1 + (it - N1) - blk * F2; }
        } else {
            int xi = NBLK * XA + (it - NP), c = -1;
            blk = 0;
            if (it < NP) {
                blk = it / PER; c = it - blk * PER;
                if (c < 2 * XA) { if (c & 1) { xi = blk * XA + (c >> 1); c = -1; } else c >>= 1; }
                else c -= XA;
            }
            if (c >= 0) s = 64 + 16 * j + c;
            else { blk = xi / FX; s = FI + xi - blk * FX; }
        }
        m.src = par + REL_PK + (u32)(blk * 128 + s) * 4096;
        m.fi = blk * 16 + (s >> 3);
        m.cnt = OFF_CNT + blk * CNT_BLK + peer * 128;
        if (s >= 64) {
            const u32 in = (u32)blk * (CF * 4096) + ((s >> 2) & 3) * 16384 + (s & 3) * 1024;
            m.raw = par + REL_B + in + j * 65536; m.dst = par + REL_B + in + (4 + q) * 65536; m.step = 4096; m.cnt += 1024;
        } else {
            m.raw = par + (u32)blk * (RS1 * 4096) + s * 4096; m.step = 1024;
            if (cross) m.dst = m.raw + (64 + 3 * FI1 + q * FX - FI) * 4096;
            else if (s < FI1) m.dst = m.raw + (64 + slot * FI1) * 4096;
            else { m.dst = par + REL_A2 + (u32)blk * (RF2 * 4096) + (slot * F2 + s - FI1) * 4096; m.cnt += 2048; }
        }
        return m;
    };
    u32 okupto = 0, wbase = 0; u64 wmask = 0;
    auto ready = [&](u32 fi) {
        if (fi < okupto || (fi - wbase < 64u && ((wmask >> (fi - wbase)) & 1))) return true;
        for (;;) {
            wbase = okupto;
            wmask = __builtin_amdgcn_ballot_w64(ld_sys(rdy + (wbase + lane) * 4) >= call);
            if (wmask != ~0ull) break;
            okupto += 64;
            if (fi < okupto) return true;
        }
        okupto += __builtin_ctzll(~wmask);
        if (fi - wbase < 64u) return ((wmask >> (fi - wbase)) & 1) != 0;
        return rfl(ld_sys(rdy + fi * 4)) >= call;
    };
    u32x4 v[4];
    typedef u32 u32x2v __attribute__((ext_vector_type(2)));
    u32x2v vp; u32 ve;
    auto fetch = [&](const Item& m, bool on) {
        const rsrc_t ra = rsrc_u(wp, on ? WS_NEED : 0u);
        const rsrc_t rb = rsrc_u(wp, on && m.cnt == 0 ? WS_NEED : 0u), rc = rsrc_u(wp, on && m.cnt != 0 ? WS_NEED : 0u);
#pragma unroll
        for (int i = 0; i < 3; ++i) v[i] = __builtin_amdgcn_raw_buffer_load_b128(ra, vo + i * 1024, m.src, 17);
        v[3] = __builtin_amdgcn_raw_buffer_load_b128(rb, vo + 3072, m.src, 17);
        vp = __builtin_amdgcn_raw_buffer_load_b64(rc, lane * 8 + 3072, m.src, 17);
        ve = __builtin_amdgcn_raw_buffer_load_b16(rc, lane * 2 + 3584, m.src, 17);
    };
    int e = wv, e1 = wv + W, e2 = wv + 2 * W;
    Item cu = decode(e), nx = decode(e1);
    spin_ge<4>(rdy + cu.fi * 4, call, ws + OFF_TMO, call);
    STAMP(5);
    bool bad = rfl(ld_agent(ws + OFF_TMO)) == call;
    fetch(cu, true);
    bool pre = ready(nx.fi);
    WAIT_VM0();
    STAMP(6);
    while (e < NT) {
        u32x4 h[4] = {v[0], v[1], v[2], v[3]};
        const u32 p4 = vp[0], p5 = vp[1], ent = ve;
        fetch(nx, pre);
        u32 tk = draw(r0, head);
        const bool isout = cu.cnt == 0;
        if (!isout) {
            const u32 hd = rfl(ent);
            if (hd < 32) unpack_item(h, p4, p5, ent, hd, scr, lane);
            else {
                const rsrc_t rr = rsrc_u(wp, WS_NEED);
#pragma unroll
                for (int i = 0; i < 4; ++i) h[i] = __builtin_amdgcn_raw_buffer_load_b128(rr, vo, cu.raw + i * cu.step, 17);
                WAIT_VM0();
            }
        }
        if (bad) {
#pragma unroll
            for (int i = 0; i < 4; ++i) h[i] = (u32x4){0x7fc07fc0u, 0x7fc07fc0u, 0x7fc07fc0u, 0x7fc07fc0u};
        }
        const rsrc_t rd = rsrc_u(isout ? outp : ws, isout ? OUT_BYTES : WS_NEED);
        const u32 vd = (isout ? vout : vo) + cu.dst;
#pragma unroll
        for (int i = 0; i < 4; ++i) st_sys(h[i], rd, vd, i * cu.step);
        const Item nn = decode(e2);
        bool pre2 = e2 < NT;
        if (pre2 && !(nowait && nn.cnt == 0)) pre2 = ready(nn.fi);
        WAIT_VM0_T(tk);
        if (lane == 0 && !isout) add_sys(ws + cu.cnt);
        if (!pre && e1 < NT) {
            spin_ge<4>(rdy + nx.fi * 4, call, ws + OFF_TMO, call);
            bad = rfl(ld_agent(ws + OFF_TMO)) == call;
            fetch(nx, true); WAIT_VM0();
        }
        STAMP_IF(cross ? (e < NP && e1 >= NP) : (e < N1 && e1 >= N1), 1);
        STAMP_IF(cross && e < NX && e1 >= NX, 2);
        STAMP_IF(cross && e < NX + 832 && e1 >= NX + 832, 3);
        cu = nx; nx = nn; pre = pre2;
        e = e1; e1 = e2; e2 = (int)(rfl(tk) + tbase);
    }
    STAMP(4);
}

#define LDPOL " sc0 sc1"
#define GLDS2(m0v, voff, rs, soff, imm0, imm1) \
    asm volatile("s_mov_b32 m0, %0\n\ts_nop 0\n\tbuffer_load_dwordx4 %1, %2, %3 offen offset:" #imm0 LDPOL " lds\n\t" \
                 "buffer_load_dwordx4 %1, %2, %3 offen offset:" #imm1 LDPOL " lds" \
                 :: "s"(m0v), "v"(voff), "s"(rs), "s"(soff) : "memory")
#define BARRIER() do { __builtin_amdgcn_sched_barrier(0); __builtin_amdgcn_s_barrier(); __builtin_amdgcn_sched_barrier(0); } while (0)

#define SCHED() __builtin_amdgcn_sched_barrier(0)
#define WAIT_LGKM0() __builtin_amdgcn_s_waitcnt(0xC07F)

template <int STAGE>
struct Gemm {
    static constexpr int NT = STAGE == 1 ? 128 : 64;
    static constexpr int NA = STAGE == 1 ? 4 : 2;
    static constexpr int NB = STAGE == 1 ? 4 : 8;
    static constexpr int NM = STAGE == 1 ? 4 : 1;
    static constexpr int NMM = STAGE == 1 ? 16 : 8;
    static constexpr int NRD = NA + NB;
    struct Frags { f16x8 a[NA], b[NB]; };

    rsrc_t rs;
    u32 sa, sa2, sb;
    u32 ldsw;
    int wid, lane;

    template <int I> __device__ __forceinline__ void piece(int t, int buf) const {
        const u32 vo = lane * 16;
        const u32 ma = ldsw + buf * BUF_BYTES + wid * 4096, mb = ldsw + buf * BUF_BYTES + 32768 + wid * 2048;
        const u32 ka = STAGE == 1 ? (u32)(t >> 1) * (RS1 * 4096) + (t & 1) * 2048 : (u32)t * (RF2 * 4096);
        const u32 a = sa + ka, a2 = sa2 + ka;
        const u32 b = STAGE == 1 ? sb + (u32)(t >> 1) * (CF * 4096) + (t & 1) * 2048 : sb + (u32)t * (CF * 4096);
        if (I == 0) GLDS2(ma, vo, rs, a, 0, 1024);
        if (I == 1) GLDS2(ma, vo, rs, a2, 2048, 3072);
        if (I == 2) GLDS2(mb, vo, rs, b, 0, 1024);
    }
    __device__ __forceinline__ void stage(int t, int buf) const { piece<0>(t, buf); piece<1>(t, buf); piece<2>(t, buf); }

    template <int I> __device__ __forceinline__ void rd(Frags& f, const LAS char* lds, int buf, int half) const {
        const LAS char* base = lds + buf * BUF_BYTES + lane * 16;
        if constexpr (STAGE == 1) {
            const LAS char* pa = base + (wid >> 1) * 8192 + half * 1024;
            const LAS char* pb = base + 32768 + (wid & 1) * 8192 + half * 1024;
            if constexpr (I == 0) f.a[0] = *(const LAS f16x8*)pa;
            else if constexpr (I <= 4) f.b[I - 1] = *(const LAS f16x8*)(pb + (I - 1) * 2048);
            else f.a[I - 4] = *(const LAS f16x8*)(pa + (I - 4) * 2048);
        } else {
            const LAS char* pa = base + wid * 4096 + half * 2048;
            const LAS char* pb = base + 32768 + half * 2048;
            constexpr int h = I / 5, n = I % 5;
            if constexpr (n == 0) f.a[h] = *(const LAS f16x8*)(pa + h * 1024);
            else f.b[h * 4 + n - 1] = *(const LAS f16x8*)(pb + (n - 1) * 4096 + h * 1024);
        }
    }
    template <int I> __device__ __forceinline__ void mm(f32x4 (&acc)[NM][4], const Frags& f) const {
        if constexpr (STAGE == 1) acc[I >> 2][I & 3] = __builtin_amdgcn_mfma_f32_16x16x32_f16(f.a[I >> 2], f.b[I & 3], acc[I >> 2][I & 3], 0, 0, 0);
        else            acc[0][I & 3] = __builtin_amdgcn_mfma_f32_16x16x32_f16(f.a[I >> 2], f.b[I], acc[0][I & 3], 0, 0, 0);
    }
    template <int I, int N> __device__ __forceinline__ void rds(Frags& f, const LAS char* lds, int buf, int half) const {
        if constexpr (I < N) { rd<I>(f, lds, buf, half); rds<I + 1, N>(f, lds, buf, half); }
    }
    static __device__ __forceinline__ int blk_of(int t) { return STAGE == 1 ? (t >> 1) : t; }

    template <bool RD, int ST>
    __device__ __forceinline__ void half_step(f32x4 (&acc)[NM][4], const Frags& cur, Frags& nxt, const LAS char* lds, int rbuf, int rhalf,
                                              int ts, int bs, u32& fl, u32 cvo, u32 fso) const {
        if constexpr (STAGE == 1) {
            mm<0>(acc, cur);  mm<1>(acc, cur);  if (RD) rds<0, 2>(nxt, lds, rbuf, rhalf); SCHED();
            mm<2>(acc, cur);  mm<3>(acc, cur);  if (RD) rds<2, 4>(nxt, lds, rbuf, rhalf); SCHED();
            mm<4>(acc, cur);  mm<5>(acc, cur);  if (RD) rds<4, 6>(nxt, lds, rbuf, rhalf); SCHED();
            mm<6>(acc, cur);  mm<7>(acc, cur);  if (RD) rds<6, 8>(nxt, lds, rbuf, rhalf); SCHED();
            mm<8>(acc, cur);  mm<9>(acc, cur);  SCHED();
            if (ST == 1) asm volatile("buffer_load_dword %0, %1, %2, %3 offen sc0 sc1" : "=v"(fl) : "v"(cvo), "s"(rs), "s"(fso) : "memory");
            if (ST) { piece<0>(ts, bs); SCHED(); }
            mm<10>(acc, cur); mm<11>(acc, cur); SCHED();
            if (ST) { piece<1>(ts, bs); SCHED(); }
            mm<12>(acc, cur); mm<13>(acc, cur); SCHED();
            if (ST) { piece<2>(ts, bs); SCHED(); }
            mm<14>(acc, cur); mm<15>(acc, cur); SCHED();
        } else {
            mm<0>(acc, cur); if (RD) rds<0, 3>(nxt, lds, rbuf, rhalf); SCHED();
            mm<1>(acc, cur); if (RD) rds<3, 5>(nxt, lds, rbuf, rhalf); SCHED();
            mm<2>(acc, cur); if (RD) rds<5, 8>(nxt, lds, rbuf, rhalf); SCHED();
            mm<3>(acc, cur); if (RD) rds<8, 10>(nxt, lds, rbuf, rhalf); SCHED();
            mm<4>(acc, cur); SCHED();
            if (ST == 1) asm volatile("buffer_load_dword %0, %1, %2, %3 offen sc0 sc1" : "=v"(fl) : "v"(cvo), "s"(rs), "s"(fso) : "memory");
            if (ST) { piece<0>(ts, bs); SCHED(); }
            mm<5>(acc, cur); SCHED();
            if (ST) { piece<1>(ts, bs); SCHED(); }
            mm<6>(acc, cur); SCHED();
            if (ST) { piece<2>(ts, bs); SCHED(); }
            mm<7>(acc, cur); SCHED();
        }
    }

    template <bool LATE>
    __device__ __forceinline__ void run(f32x4 (&acc)[NM][4], const LAS char* lds, char* ws, u32 cvo, u32 need, u32 call) const {
        char* tmo = ws + OFF_TMO;
        Frags f0, f1;
        if (!LATE && wid == 0) spin_ge(ws + cvo + blk_of(0) * CNT_BLK, need, tmo, call);
        BARRIER();
        stage(0, 0);
        if (!LATE && wid == 0) { spin_ge(ws + cvo + blk_of(1) * CNT_BLK, need, tmo, call); spin_ge(ws + cvo + blk_of(2) * CNT_BLK, need, tmo, call); }
        BARRIER();
        stage(1, 1);
        if (!LATE) { stage(2, 2); asm volatile("s_waitcnt vmcnt(12)" ::: "memory"); }
        else asm volatile("s_waitcnt vmcnt(6)" ::: "memory");
        BARRIER();
        rds<0, NRD>(f0, lds, 0, 0);
        const u32 fvo = wid == 0 ? cvo : WS_NEED, fneed = wid == 0 ? need : 0u;
        u32 fl = 0;
        int buf = 0;
        for (int t = 0; t < NT - 3; ++t) {
            const int nb = buf == 2 ? 0 : buf + 1, pb = buf == 0 ? 2 : buf - 1;
            half_step<true, LATE ? 2 : 0>(acc, f0, f1, lds, buf, 1, t + 2, pb, fl, fvo, 0);
            if (!LATE) {
                asm volatile("s_waitcnt vmcnt(6)" : "+v"(fl) :: "memory");
                WAIT_LGKM0();
                if (__builtin_amdgcn_ballot_w64(fl < fneed) != 0) spin_ge(ws + cvo + blk_of(t + 3) * CNT_BLK, need, tmo, call);
            } else {
                asm volatile("s_waitcnt vmcnt(6)" ::: "memory");
                WAIT_LGKM0();
            }
            BARRIER();
            half_step<true, LATE ? 0 : 1>(acc, f1, f0, lds, nb, 0, t + 3, buf, fl, fvo, blk_of(t + 4 < NT ? t + 4 : t + 3) * CNT_BLK);
            buf = nb;
        }
#pragma unroll
        for (int t = NT - 3; t < NT; ++t) {
            const int nb = buf == 2 ? 0 : buf + 1, pb = buf == 0 ? 2 : buf - 1;
            if (t == NT - 3) half_step<true, LATE ? 2 : 0>(acc, f0, f1, lds, buf, 1, t + 2, pb, fl, fvo, 0);
            else             half_step<true, 0>(acc, f0, f1, lds, buf, 1, 0, 0, fl, fvo, 0);
            if (t == NT - 3) asm volatile("s_waitcnt vmcnt(6)" : "+v"(fl) :: "memory");
            else             asm volatile("s_waitcnt vmcnt(0)" : "+v"(fl) :: "memory");
            WAIT_LGKM0();
            BARRIER();
            if (t < NT - 1) half_step<true, 0>(acc, f1, f0, lds, nb, 0, 0, 0, fl, fvo, 0);
            else            half_step<false, 0>(acc, f1, f0, lds, nb, 0, 0, 0, fl, fvo, 0);
            buf = nb;
        }
    }
};

__device__ __forceinline__ u32x4 gelu4(const f32x4 (&c)[4], int r, bool bad) {
    const float nanv = __builtin_nanf("");
    f32x4 o = {gelu(c[0][r]), gelu(c[1][r]), gelu(c[2][r]), gelu(c[3][r])};
    if (bad) o = (f32x4){nanv, nanv, nanv, nanv};
    return __builtin_bit_cast(u32x4, o);
}

__device__ __forceinline__ void gemm_role(const Params& p, const LAS char* lds, int xcd, int s, int wid, int lane, u32 call) {
    const int rank = rfl(p.rank);
    const int g = rank >> 2, j = rank & 3, og = (g ^ 1) * 4;
    char* ws = p.ws[rank];
    char* tmo = ws + OFF_TMO;
    const u32 par = OFF_PAR + (call & 1) * PAR_BYTES;
    const rsrc_t rs = mk_rsrc(ws, WS_NEED), ro = mk_rsrc(p.out[rank], OUT_BYTES);
    const u32 ldsw = rfl((u32)(size_t)lds);
    const u32 vout = (lane >> 4) * 4 * 4096 + (lane & 15) * 16;
    {
        const int u = (xcd & 1) * 26 + s, rt = u >> 2, ct = (xcd >> 1) * 4 + (u & 3);
        Gemm<1> G;
        G.rs = rs; G.ldsw = ldsw; G.wid = wid; G.lane = lane;
        const int rf = rt * 16 + wid * 2;
        G.sa = par + (rf < FI ? rf : rf + FX) * 4096;
        G.sa2 = par + (rf + 1 < FI ? rf + 1 : rf + 1 + FX) * 4096 - 2048;
        G.sb = par + REL_B + (ct * 8 + wid) * 4096;
        const int bsrc = ct < 8 ? rank : og + ((ct - 8) >> 1);
        u32 cvo = OFF_CNT + 1024 + bsrc * 128, need = (ct < 8 ? 8u : 16u) * call;
        u32 mask = 0;
        for (int i = 0; i < 16; ++i) mask |= 1u << src_of_rf1(rt * 16 + i, g, j);
        if (lane < 8 && ((mask >> lane) & 1)) {
            cvo = OFF_CNT + lane * 128;
            need = (lane == rank ? 8 : ((lane >> 2) == g ? FI1 : FX)) * call;
        }
        f32x4 acc[4][4] = {};
        if (wid < 4) G.template run<false>(acc, lds, ws, cvo, need, call);
        else         G.template run<true>(acc, lds, ws, cvo, need, call);
        STAMP(1);
        const bool bad = rfl(ld_agent(tmo)) == call;
        const int wm = wid >> 1, wn = wid & 1;
        if (ct < 8) {
#pragma unroll
            for (int m = 0; m < 4; ++m) {
                const u32 ob = (u32)row_of_rf1(rt * 16 + wm * 4 + m, g, j) * 4096 + (ct * 128 + wn * 64) * 4;
#pragma unroll
                for (int r = 0; r < 4; ++r) __builtin_amdgcn_raw_buffer_store_b128(gelu4(acc[m], r, bad), ro, vout + ob + r * 4096, 0, 0);
            }
        } else {
            const int fo = (rt * 2 + (ct & 1)) * 8 + wid;
            const u32 sb = par + REL_STG + (u32)(((ct - 8) >> 1) * 1024 + fo * 4) * 4096;
#pragma unroll
            for (int m = 0; m < 4; ++m)
#pragma unroll
                for (int r = 0; r < 4; ++r) st_sys(gelu4(acc[m], r, bad), rs, lane * 16 + r * 1024, sb + m * 4096);
            WAIT_VM0();
            if (lane == 0) st_flag(p.ws[bsrc] + OFF_RDY + rank * 8192 + (1024 + fo) * 4, call);
        }
        STAMP(2);
    }
    if (s < 24) {
        const int rt = s >> 2, ct = ((xcd & 1) * 16) + (xcd >> 1) * 4 + (s & 3);
        Gemm<2> G;
        G.rs = rs; G.ldsw = ldsw; G.wid = wid; G.lane = lane;
        G.sa = G.sa2 = par + REL_A2 + (rt * 8 + wid) * 4096;
        G.sb = par + REL_B + ct * 4 * 4096 + wid * 2048;
        const int asrc = g * 4 + ((j + 1 + (rt >> 1)) & 3);
        const int bsrc = ct < 16 ? rank : og + ((ct - 16) >> 2);
        u32 cvo = OFF_CNT + 1024 + bsrc * 128, need = (ct < 16 ? 8u : 16u) * call;
        if (lane == 0) { cvo = OFF_CNT + 2048 + asrc * 128; need = F2 * call; }
        f32x4 acc[1][4] = {};
        if (wid < 4) G.template run<false>(acc, lds, ws, cvo, need, call);
        else         G.template run<true>(acc, lds, ws, cvo, need, call);
        STAMP(3);
        const bool bad = rfl(ld_agent(tmo)) == call;
        if (ct < 16) {
            const u32 ob = (u32)(asrc * NS + (FI1 + (rt & 1) * 8 + wid) * 16) * 4096 + ct * 256;
#pragma unroll
            for (int r = 0; r < 4; ++r) __builtin_amdgcn_raw_buffer_store_b128(gelu4(acc[0], r, bad), ro, vout + ob + r * 4096, 0, 0);
        } else {
            const int z = (rt * 4 + (ct & 3)) * 8 + wid;
            const u32 sb = par + REL_STG + (u32)(((ct - 16) >> 2) * 1024 + 832 + z) * 4096;
#pragma unroll
            for (int r = 0; r < 4; ++r) st_sys(gelu4(acc[0], r, bad), rs, lane * 16 + r * 1024, sb);
            WAIT_VM0();
            if (lane == 0) st_flag(p.ws[bsrc] + OFF_RDY + rank * 8192 + (1024 + 208 + z) * 4, call);
        }
        STAMP(4);
    }
}

__global__ __launch_bounds__(BLOCK, 2) void ag_gemm_kernel(Params p) {
    __shared__ __attribute__((aligned(1024))) char smem[LDS_BYTES];
    const int rank = rfl(p.rank);
    const int lane = threadIdx.x & 63;
    const int wid = rfl(threadIdx.x >> 6);
    const int b = blockIdx.x, xcd = b & 7, slot = b >> 3;
    const int gw = wid * NCOMM + b;
    if (b < NCOMM && gw >= NPULL) return;
    STAMP(0);
    char* seq = p.ws[rank] + OFF_SEQ + (b * 8 + wid) * 4;
    const u32 call = rfl(ld_agent(seq)) + 1;
    if (lane == 0) __hip_atomic_store((GAS u32*)seq, call, __ATOMIC_RELAXED, __HIP_MEMORY_SCOPE_AGENT);
    STAMP(7);
    if (b < NCOMM) {
        constexpr int WM = WI < WC ? WI : WC;
        int ps, wv;
        if (gw < 7 * WM) { ps = gw % 7; wv = gw / 7; }
        else if (WI > WC) { ps = (gw - 7 * WM) % 3; wv = WM + (gw - 7 * WM) / 3; }
        else { ps = 3 + (gw - 7 * WM) % 4; wv = WM + (gw - 7 * WM) / 4; }
        pull_role(p, (LAS char*)smem + wid * 4096, rfl(ps), rfl(wv), lane, call);
    } else {
        convert_local(p, (LAS char*)smem + wid * 4096, b - NCOMM, wid, lane, call);
        STAMP(5);
        gemm_role(p, (const LAS char*)smem, xcd, slot - NCOMM / 8, wid, lane, call);
    }
}

}

constexpr int g_mode = 0;

extern "C" void kernel_launch(int rank, int nranks, ncclComm_t comm, hipStream_t stream,
                              void* const* d_in, const int* in_sizes, int n_in,
                              void* d_out, int out_size, const int* in_dtypes, int out_dtype,
                              void* const* d_out_all, void* const* d_ws_all, size_t ws_bytes) {
    (void)comm; (void)in_sizes; (void)n_in; (void)d_out; (void)out_size; (void)in_dtypes; (void)out_dtype;
    if (nranks != NR || (unsigned)rank >= (unsigned)NR || ws_bytes < WS_NEED) {
        fprintf(stderr, "unsupported configuration\n");
        return;
    }
    Params p;
    p.x = (const float*)d_in[0];
    p.w = (const float*)d_in[1];
    for (int i = 0; i < NR; ++i) { p.ws[i] = (char*)d_ws_all[i]; p.out[i] = (char*)d_out_all[i]; }
    p.rank = rank;
    p.pad = g_mode;
    ag_gemm_kernel<<<GRID, BLOCK, 0, stream>>>(p);
    hipError_t e = hipGetLastError();
    if (e != hipSuccess) fprintf(stderr, "launch failed: %s\n", hipGetErrorString(e));
}
```

```cpp
#include <hip/hip_runtime.h>
#include <rccl/rccl.h>
#include <cstdio>
#include <cstdint>

typedef unsigned u32;
typedef unsigned long long u64;
typedef u32 u32x4 __attribute__((ext_vector_type(4)));
typedef float f32x4 __attribute__((ext_vector_type(4)));
typedef _Float16 f16x8 __attribute__((ext_vector_type(8)));
typedef __amdgpu_buffer_rsrc_t rsrc_t;
#define LAS __attribute__((address_space(3)))
#define GAS __attribute__((address_space(1)))

namespace {

constexpr int NR = 8;
constexpr int KD = 8192;
constexpr int NS = 1024;
constexpr int NBLK = 64;
#define P_FI 46
constexpr int FI = P_FI;
constexpr int F2 = 16;
constexpr int FI1 = FI - F2;
constexpr int FX = 64 - FI;
constexpr int RF1 = 208, RF2 = 48, CF = 128;
static_assert(FI + 3 * FI1 + 4 * FX == RF1, "stage-1 rows");

#define P_WI 21
#define P_WC 21
#define P_XA 15
#define P_EARLY 3
#define P_SLEEP 64
#define P_PB 2
constexpr int EARLY = P_EARLY, LATE_SLEEP = P_SLEEP, PB = P_PB;
constexpr int NCOMM = 48;
constexpr int WI = P_WI, WC = P_WC;
constexpr int NPULL = 3 * WI + 4 * WC;
constexpr int XA = P_XA;
static_assert(NPULL <= NCOMM * 8 && XA <= FX && XA <= 16, "wave roles");
constexpr int GRID = 256, BLOCK = 512, NGEMM = GRID - NCOMM;
constexpr int RS1 = 64 + 3 * FI1 + 4 * FX;

constexpr u32 OFF_SEQ  = 0;
constexpr u32 OFF_TMO  = 16384;
constexpr u32 OFF_QUE  = 16640;
constexpr u32 OFF_RDY  = 32768;
constexpr u32 OFF_CNT  = 131072;
constexpr u32 CNT_BLK  = 24 * 128;
constexpr u32 OFF_DIAG = 524288;
constexpr u32 A1_BYTES = (u32)NBLK * RS1 * 4096;
constexpr u32 A2_BYTES = (u32)NBLK * RF2 * 4096;
constexpr u32 B_BYTES  = (u32)NBLK * CF * 4096;
constexpr u32 STG_BYTES = 4u * 1024 * 4096;
constexpr u32 PK_BYTES = (u32)NBLK * 128 * 4096;
constexpr u32 REL_A2 = A1_BYTES, REL_B = REL_A2 + A2_BYTES, REL_PK = REL_B + B_BYTES, REL_STG = REL_PK + PK_BYTES;
constexpr u32 PAR_BYTES = REL_STG + 2 * STG_BYTES;
constexpr u32 OFF_PAR  = 1u << 20;
constexpr u32 WS_NEED  = OFF_PAR + 2 * PAR_BYTES;
constexpr u32 OUT_BYTES = (u32)KD * NS * 4;
static_assert(PK_BYTES == 8192u * 4096u, "staged outputs follow the packed image");
static_assert(OFF_CNT + NBLK * CNT_BLK <= OFF_DIAG && OFF_RDY + 8 * 8192 <= OFF_CNT, "control area");

constexpr u64 DEADLINE = 40000000ull;

constexpr int BUF_BYTES = 49152;
constexpr int LDS_BYTES = 3 * BUF_BYTES;

struct Params {
    const float* x;
    const float* w;
    char* ws[NR];
    char* out[NR];
    int rank;
    int pad;
};

__device__ __forceinline__ u32 rfl(u32 v) { return __builtin_amdgcn_readfirstlane(v); }
__device__ __forceinline__ char* uni(const void* p) {
    const u64 a = (u64)p;
    return (char*)(((u64)rfl((u32)(a >> 32)) << 32) | rfl((u32)a));
}
__device__ __forceinline__ rsrc_t rsrc_u(const void* p, u32 bytes) {
    return __builtin_amdgcn_make_buffer_rsrc((void*)p, (short)0, (int)bytes, 0x00020000);
}
__device__ __forceinline__ rsrc_t mk_rsrc(const void* p, u32 bytes) { return rsrc_u(uni(p), bytes); }
__device__ __forceinline__ u32 ld_sys(const char* p) {
    return __hip_atomic_load((const GAS u32*)p, __ATOMIC_RELAXED, __HIP_MEMORY_SCOPE_SYSTEM);
}
__device__ __forceinline__ u32 ld_agent(const char* p) {
    return __hip_atomic_load((const GAS u32*)p, __ATOMIC_RELAXED, __HIP_MEMORY_SCOPE_AGENT);
}
__device__ __forceinline__ void st_flag(char* p, u32 v) {
    __hip_atomic_store((GAS u32*)p, v, __ATOMIC_RELAXED, __HIP_MEMORY_SCOPE_SYSTEM);
}
__device__ __forceinline__ void add_sys(char* p) {
    __hip_atomic_fetch_add((GAS u32*)p, 1u, __ATOMIC_RELAXED, __HIP_MEMORY_SCOPE_SYSTEM);
}
__device__ __forceinline__ u32 draw(rsrc_t r, u32 voff) {
    u32 t = 1;
    asm volatile("s_nop 0\n\tbuffer_atomic_add %0, %1, %2, 0 offen sc0" : "+v"(t) : "v"(voff), "s"(r) : "memory");
    return t;
}
#define WAIT_VM0() do { asm volatile("s_waitcnt vmcnt(0)" ::: "memory"); __builtin_amdgcn_s_waitcnt(0x0F70); } while (0)
#define WAIT_VM0_T(t) do { asm volatile("s_waitcnt vmcnt(0)" : "+v"(t) :: "memory"); __builtin_amdgcn_s_waitcnt(0x0F70); } while (0)

template <int SLEEP = 2>
__device__ __forceinline__ void spin_ge(const char* p, u32 need, char* tmo, u32 call) {
    const u64 t0 = __builtin_amdgcn_s_memrealtime();
    for (;;) {
        const u32 v = ld_sys(p);
        if (__builtin_amdgcn_ballot_w64(v < need) == 0) break;
        if (rfl(ld_agent(tmo)) == call || __builtin_amdgcn_s_memrealtime() - t0 > DEADLINE) {
            __hip_atomic_store((GAS u32*)tmo, call, __ATOMIC_RELAXED, __HIP_MEMORY_SCOPE_AGENT);
            break;
        }
        __builtin_amdgcn_s_sleep(SLEEP);
    }
    asm volatile("" ::: "memory");
}

#define STAMP_IF(c, slot) do {} while (0)
#define STAMP(slot) do {} while (0)
#define STAMPB_IF(c, slot) do {} while (0)

__device__ __forceinline__ u32x4 cvt8(f32x4 a, f32x4 b) {
    f16x8 h;
    h[0] = (_Float16)a[0]; h[1] = (_Float16)a[1]; h[2] = (_Float16)a[2]; h[3] = (_Float16)a[3];
    h[4] = (_Float16)b[0]; h[5] = (_Float16)b[1]; h[6] = (_Float16)b[2]; h[7] = (_Float16)b[3];
    return __builtin_bit_cast(u32x4, h);
}
__device__ __forceinline__ void st_sys(u32x4 v, rsrc_t r, u32 voff, u32 soff) {
    __builtin_amdgcn_raw_buffer_store_b128(v, r, voff + soff, 0, 17);
}
__device__ __forceinline__ float gelu(float y) {
    const float u = y * (0.7978845608028654f + 0.7978845608028654f * 0.044715f * y * y);
    const float e = __builtin_amdgcn_exp2f(u * -2.8853900817779268f);
    return y * __builtin_amdgcn_rcpf(1.0f + e);
}

__device__ __forceinline__ int row_of_rf1(int rf, int g, int j) {
    if (rf < FI) return (g * 4 + j) * NS + rf * 16;
    rf -= FI;
    if (rf < 3 * FI1) { const int i = rf / FI1; return (g * 4 + ((j + 1 + i) & 3)) * NS + (rf - i * FI1) * 16; }
    rf -= 3 * FI1;
    const int q = rf / FX;
    return ((g ^ 1) * 4 + q) * NS + (FI + rf - q * FX) * 16;
}
__device__ __forceinline__ int src_of_rf1(int rf, int g, int j) { return row_of_rf1(rf, g, j) / NS; }

#define LDIN(p) __builtin_nontemporal_load(p)
constexpr int OUT_AUX = 2;
__device__ __forceinline__ void load_x(f32x4* v, const float* x, int frag, int blk, int lane) {
    const float* s = x + (size_t)(frag * 16 + (lane & 15)) * KD + blk * 128 + (lane >> 4) * 4;
#pragma unroll
    for (int i = 0; i < 8; ++i) v[i] = LDIN((const f32x4*)(s + i * 16));
}
__device__ __forceinline__ void load_w(f32x4* v, const float* w, int cg, int k0, int lane) {
    const float* s = w + (size_t)(k0 + (lane >> 4) * 4) * NS + cg * 64 + (lane & 15) * 4;
#pragma unroll
    for (int i = 0; i < 8; ++i) v[i] = LDIN((const f32x4*)(s + ((i & 3) + (i >> 2) * 16) * NS));
}
__device__ __forceinline__ u32x4 cvt_w(const f32x4* v, int c) {
    return cvt8((f32x4){v[0][c], v[1][c], v[2][c], v[3][c]}, (f32x4){v[4][c], v[5][c], v[6][c], v[7][c]});
}

#define DPP_STEP(op, v, ctrl, rmask) v = op(v, (u32)__builtin_amdgcn_update_dpp(0, (int)v, ctrl, rmask, 0xf, false))
__device__ __forceinline__ u32 umax2(u32 a, u32 b) { return a > b ? a : b; }
__device__ __forceinline__ u32 uadd2(u32 a, u32 b) { return a + b; }
__device__ __forceinline__ u32 wave_max(u32 v) {
    DPP_STEP(umax2, v, 0x111, 0xf); DPP_STEP(umax2, v, 0x112, 0xf); DPP_STEP(umax2, v, 0x114, 0xf); DPP_STEP(umax2, v, 0x118, 0xf);
    DPP_STEP(umax2, v, 0x142, 0xa); DPP_STEP(umax2, v, 0x143, 0xc);
    return __builtin_amdgcn_readlane(v, 63);
}
__device__ __forceinline__ u32 wave_sum(u32 v) {
    DPP_STEP(uadd2, v, 0x111, 0xf); DPP_STEP(uadd2, v, 0x112, 0xf); DPP_STEP(uadd2, v, 0x114, 0xf); DPP_STEP(uadd2, v, 0x118, 0xf);
    DPP_STEP(uadd2, v, 0x142, 0xa); DPP_STEP(uadd2, v, 0x143, 0xc);
    return __builtin_amdgcn_readlane(v, 63);
}

__device__ __forceinline__ void pack_item(const u32x4 (&h)[4], float amax, LAS char* scr, rsrc_t r0, u32 dst, int lane) {
    u32 L[8], H[8];
#pragma unroll
    for (int t = 0; t < 8; ++t) {
        const u32 d0 = h[t >> 1][(t & 1) * 2], d1 = h[t >> 1][(t & 1) * 2 + 1];
        L[t] = __builtin_amdgcn_perm(d1, d0, 0x06040200u);
        H[t] = __builtin_amdgcn_perm(d1, d0, 0x07050301u);
    }
    *(LAS u32x4*)(scr + lane * 16) = (u32x4){H[0], H[1], H[2], H[3]};
    *(LAS u32x4*)(scr + lane * 16 + 1024) = (u32x4){H[4], H[5], H[6], H[7]};
    const float wmax = __builtin_bit_cast(float, wave_max(__builtin_bit_cast(u32, amax)));
    const u32 emax = rfl((u32)__builtin_bit_cast(unsigned short, (_Float16)wmax) >> 10) & 31;
    u32 ntop = 0;
#pragma unroll
    for (int t = 0; t < 8; ++t) ntop += __builtin_popcount(((H[t] & 0x7f7f7f7fu) + (0x80u - emax * 4) * 0x01010101u) & 0x80808080u);
    const int top = (int)emax - (wave_sum(ntop) <= 16 ? 1 : 0);
    const u32 lo = top > 7 ? top - 7 : 0, lo4 = lo * 0x04040404u;
    LAS unsigned short* list = (LAS unsigned short*)(scr + 2048);
    list[lane] = (unsigned short)lo;
    u32 F[8], em = 0;
#pragma unroll
    for (int t = 0; t < 8; ++t) {
        const u32 u = ((H[t] | 0x80808080u) - lo4) ^ 0x80808080u;
        const u32 x = u & 0xe0e0e0e0u;
        const u32 nz = (x | (x << 1) | (x << 2)) & 0x80808080u;
        const u32 m = (nz >> 2) - (nz >> 5);
        F[t] = (u & 0x1f1f1f1fu & ~m) | ((H[t] >> 2) & 0x20202020u);
        em = (em >> 1) | nz;
    }
    u32 used = 1;
    for (u64 act = __builtin_amdgcn_ballot_w64(em != 0); act != 0; act = __builtin_amdgcn_ballot_w64(em != 0)) {
        const u32 k = used + __builtin_amdgcn_mbcnt_hi((u32)(act >> 32), __builtin_amdgcn_mbcnt_lo((u32)act, 0));
        const u32 b = __builtin_ctz(em | 0x80000000u), t = b & 7, y = b >> 3;
        const u32 pos = (t >> 2) * 1024 + lane * 16 + (t & 3) * 4 + y;
        const u32 e5 = (*(const LAS unsigned char*)(scr + pos) >> 2) & 31;
        list[em != 0 && k < 64 ? k : 64] = (unsigned short)(pos * 32 + e5);
        em &= em - 1;
        used += __builtin_popcountll(act);
        if (used > 64) break;
    }
    asm volatile("" ::: "memory");
    u32 ent = list[lane];
    if (lane == 0 && used > 64) ent = 0xffffu;
    st_sys((u32x4){L[0], L[1], L[2], L[3]}, r0, lane * 16, dst);
    st_sys((u32x4){L[4], L[5], L[6], L[7]}, r0, lane * 16, dst + 1024);
    st_sys((u32x4){F[0] | ((F[3] & 0x03030303u) << 6), F[1] | ((F[3] & 0x0c0c0c0cu) << 4), F[2] | ((F[3] & 0x30303030u) << 2),
                   F[4] | ((F[7] & 0x03030303u) << 6)}, r0, lane * 16, dst + 2048);
    typedef u32 u32x2v __attribute__((ext_vector_type(2)));
    __builtin_amdgcn_raw_buffer_store_b64((u32x2v){F[5] | ((F[7] & 0x0c0c0c0cu) << 4), F[6] | ((F[7] & 0x30303030u) << 2)},
                                          r0, lane * 8 + dst + 3072, 0, 17);
    __builtin_amdgcn_raw_buffer_store_b16((unsigned short)ent, r0, lane * 2 + dst + 3584, 0, 17);
}
__device__ __forceinline__ void unpack_item(u32x4 (&h)[4], const u32x4& c0, const u32x4& c1, const u32x4& c2, u32 p4, u32 p5,
                                            u32 ent, u32 lo, LAS char* scr, int lane) {
    const u32 P[6] = {c2[0], c2[1], c2[2], c2[3], p4, p5};
    const u32 lo4 = lo * 0x04040404u;
    u32x4 H[2];
#pragma unroll
    for (int z = 0; z < 2; ++z) {
        u32 F[4] = {P[3 * z] & 0x3f3f3f3fu, P[3 * z + 1] & 0x3f3f3f3fu, P[3 * z + 2] & 0x3f3f3f3fu,
                    ((P[3 * z] >> 6) & 0x03030303u) | ((P[3 * z + 1] >> 4) & 0x0c0c0c0cu) | ((P[3 * z + 2] >> 2) & 0x30303030u)};
#pragma unroll
        for (int t = 0; t < 4; ++t) { const u32 sg = F[t] & 0x20202020u; H[z][t] = F[t] + ((sg << 1) + sg) + lo4; }
        *(LAS u32x4*)(scr + z * 1024 + lane * 16) = H[z];
    }
    __hip_atomic_fetch_add((LAS u32*)(scr + ((ent >> 5) & 0x7fc)), ((ent & 31) - lo) << (2 + ((ent >> 2) & 24)),
                           __ATOMIC_RELAXED, __HIP_MEMORY_SCOPE_WORKGROUP);
    asm volatile("" ::: "memory");
#pragma unroll
    for (int z = 0; z < 2; ++z) {
        H[z] = *(const LAS u32x4*)(scr + z * 1024 + lane * 16);
        const u32x4& L = z ? c1 : c0;
#pragma unroll
        for (int t = 0; t < 4; ++t) {
            h[z * 2 + (t >> 1)][(t & 1) * 2] = __builtin_amdgcn_perm(H[z][t], L[t], 0x05010400u);
            h[z * 2 + (t >> 1)][(t & 1) * 2 + 1] = __builtin_amdgcn_perm(H[z][t], L[t], 0x07030602u);
        }
    }
}

__device__ __forceinline__ void pack_out(const u32x4 (&o)[4], LAS char* scr, rsrc_t r0, u32 dst, int lane) {
    u32 M[12], mx = 0;
    u32x4 E;
#pragma unroll
    for (int r = 0; r < 4; ++r) {
        u32 R[4];
#pragma unroll
        for (int c = 0; c < 4; ++c) { R[c] = __builtin_amdgcn_alignbit(o[r][c], o[r][c], 31); mx = umax2(mx, R[c]); }
#pragma unroll
        for (int i = 0; i < 3; ++i) M[r * 3 + i] = __builtin_amdgcn_perm(R[3], R[i], 0x04020100u + (i << 24));
        E[r] = __builtin_amdgcn_perm(R[1], R[0], 0x0c0c0703u) | __builtin_amdgcn_perm(R[3], R[2], 0x07030c0cu);
    }
    *(LAS u32x4*)(scr + lane * 16) = E;
    const u32 emax = wave_max(mx) >> 24, lo = emax > 15 ? emax - 15 : 0, lo1 = lo * 0x01010101u;
    LAS u32* list = (LAS u32*)(scr + 1024);
    list[lane] = lo;
    u32 C[4], em = 0;
#pragma unroll
    for (int r = 0; r < 4; ++r) {
        const u32 d = ((E[r] | 0x80808080u) - (lo1 & 0x7f7f7f7fu)) ^ ((E[r] ^ ~lo1) & 0x80808080u);
        const u32 x = d & 0xf0f0f0f0u, y = x | (x << 1), nz = (y | (y << 2)) & 0x80808080u;
        C[r] = d & 0x0f0f0f0fu & ~((nz >> 3) - (nz >> 7));
        em = (em >> 1) | nz;
    }
    u32 used = 1;
    for (u64 act = __builtin_amdgcn_ballot_w64(em != 0); act != 0; act = __builtin_amdgcn_ballot_w64(em != 0)) {
        const u32 k = used + __builtin_amdgcn_mbcnt_hi((u32)(act >> 32), __builtin_amdgcn_mbcnt_lo((u32)act, 0));
        const u32 b = __builtin_ctz(em | 0x80000000u);
        const u32 pos = lane * 16 + (b & 3) * 4 + (b >> 3);
        list[em != 0 && k < 16 ? k : 64] = pos * 256 + *(const LAS unsigned char*)(scr + pos);
        em &= em - 1;
        used += __builtin_popcountll(act);
        if (used > 16) break;
    }
    asm volatile("" ::: "memory");
    u32 ent = list[lane];
    if (used > 16) {
        if (lane == 0) ent = 0xffffffffu;
#pragma unroll
        for (int r = 0; r < 4; ++r) st_sys(o[r], r0, lane * 16 + r * 1024, dst + STG_BYTES);
    }
#pragma unroll
    for (int i = 0; i < 3; ++i) st_sys((u32x4){M[4 * i], M[4 * i + 1], M[4 * i + 2], M[4 * i + 3]}, r0, lane * 16 + i * 1024, dst);
    typedef u32 u32x2v __attribute__((ext_vector_type(2)));
    __builtin_amdgcn_raw_buffer_store_b64((u32x2v){C[0] | (C[1] << 4), C[2] | (C[3] << 4)}, r0, lane * 8 + dst + 3072, 0, 17);
    __builtin_amdgcn_raw_buffer_store_b32(ent, r0, (lane < 16 ? lane * 4 : 0x80000000u) + dst + 3584, 0, 17);
}
__device__ __forceinline__ void unpack_out(u32x4 (&o)[4], const u32x4& c0, const u32x4& c1, const u32x4& c2, u32 p4, u32 p5,
                                           u32 ent, u32 lo, LAS char* scr, int lane) {
    const u32 M[12] = {c0[0], c0[1], c0[2], c0[3], c1[0], c1[1], c1[2], c1[3], c2[0], c2[1], c2[2], c2[3]};
    const u32 lo1 = lo * 0x01010101u;
    u32x4 E = {(p4 & 0x0f0f0f0fu) + lo1, ((p4 >> 4) & 0x0f0f0f0fu) + lo1, (p5 & 0x0f0f0f0fu) + lo1, ((p5 >> 4) & 0x0f0f0f0fu) + lo1};
    *(LAS u32x4*)(scr + lane * 16) = E;
    __hip_atomic_fetch_add((LAS u32*)(scr + ((ent >> 8) & 0x3fc)), ((ent & 255) - lo) << ((ent >> 5) & 24),
                           __ATOMIC_RELAXED, __HIP_MEMORY_SCOPE_WORKGROUP);
    asm volatile("" ::: "memory");
    E = *(const LAS u32x4*)(scr + lane * 16);
#pragma unroll
    for (int r = 0; r < 4; ++r) {
        u32 R[4];
#pragma unroll
        for (int i = 0; i < 3; ++i) R[i] = __builtin_amdgcn_perm(E[r], M[r * 3 + i], 0x04020100u + (i << 24));
        R[3] = __builtin_amdgcn_perm(M[r * 3 + 1], M[r * 3], 0x0c0c0703u) | __builtin_amdgcn_perm(E[r], M[r * 3 + 2], 0x07030c0cu);
#pragma unroll
        for (int c = 0; c < 4; ++c) o[r][c] = __builtin_amdgcn_alignbit(R[c], R[c], 1);
    }
}

__device__ __forceinline__ void load_item(f32x4 (&v)[8], const Params& p, int u, int wid, int lane) {
    const int blk = u >> 4, it = (u & 15) * 8 + wid;
    if (it < 64) load_x(v, p.x, it, blk, lane);
    else load_w(v, p.w, (it - 64) >> 2, blk * 128 + (wid & 3) * 32, lane);
}
__device__ __forceinline__ void convert_local(const Params& p, LAS char* scr, f32x4 (&v)[8], char* wsl, int wg, int wid, int lane, u32 call) {
    const int rank = rfl(p.rank);
    const u32 par = OFF_PAR + (call & 1) * PAR_BYTES;
    const rsrc_t r0 = mk_rsrc(p.ws[rank], WS_NEED);
    char* flag = wsl + OFF_RDY + rank * 8192;
    for (int u = wg; u < NBLK * 16; u += NGEMM) {
        const int blk = u >> 4, grp = u & 15, it = grp * 8 + wid;
        u32x4 h[4];
        u32 base, step;
        if (grp < 8) {
            base = par + (u32)blk * (RS1 * 4096) + it * 4096; step = 1024;
#pragma unroll
            for (int i = 0; i < 4; ++i) h[i] = cvt8(v[2 * i], v[2 * i + 1]);
        } else {
            base = par + REL_B + (u32)blk * (CF * 4096) + ((it - 64) & 60) * 4096 + (wid & 3) * 1024; step = 4096;
#pragma unroll
            for (int c = 0; c < 4; ++c) h[c] = cvt_w(v, c);
        }
        float amax = 0.0f;
#pragma unroll
        for (int i = 0; i < 8; ++i) amax = fmaxf(fmaxf(fmaxf(amax, fabsf(v[i][0])), fmaxf(fabsf(v[i][1]), fabsf(v[i][2]))), fabsf(v[i][3]));
        STAMPB_IF(u == wg, 0);
#pragma unroll
        for (int i = 0; i < 4; ++i) st_sys(h[i], r0, lane * 16, base + i * step);
        pack_item(h, amax, scr, r0, par + REL_PK + (u32)(blk * 128 + it) * 4096, lane);
        STAMPB_IF(u == wg, 1);
        WAIT_VM0();
        STAMPB_IF(u == wg, 2);
        __builtin_amdgcn_s_barrier();
        STAMP_IF(u == wg, 6);
        if (wid == 0) {
            if (lane < 8 && lane != rank) st_flag(flag + u * 4, call);
            if (lane == 8) add_sys(p.ws[rank] + OFF_CNT + blk * CNT_BLK + (grp < 8 ? 0 : 1024) + rank * 128);
        }
        if (u + NGEMM < NBLK * 16) load_item(v, p, u + NGEMM, wid, lane);
    }
}

constexpr int PUSH_PER = FI1 + 64 + XA, NPUSH = PB * PUSH_PER, NSPARE = NCOMM * 8 - NPULL;
__device__ __forceinline__ void push_decode(int n, int& blk, int& it) {
    blk = n / PUSH_PER; it = n - blk * PUSH_PER;
    if (it >= FI1 + 64) { const int xi = blk * XA + it - FI1 - 64; blk = xi / FX; it = FI + xi - blk * FX; }
    else if (it >= FI1) it += 64 - FI1;
}
__device__ __forceinline__ void push_role(const Params& p, f32x4 (&v)[8], char* wsl, int sw, int lane, u32 call) {
    const int rank = rfl(p.rank);
    const int g = rank >> 2, j = rank & 3, og = (g ^ 1) * 4;
    const u32 par = OFF_PAR + (call & 1) * PAR_BYTES;
    for (int n = sw; n < NPUSH; n += NSPARE) {
        int blk, it;
        push_decode(n, blk, it);
        u32x4 h[4];
        int d0 = 0, nd = 4, dstep = 1; u32 base, step = 1024, ostep = 0;
        if (it < 64) {
#pragma unroll
            for (int i = 0; i < 4; ++i) h[i] = cvt8(v[2 * i], v[2 * i + 1]);
            if (it < FI1) { d0 = j + 3; nd = 3; dstep = 3; ostep = FI1 * 4096; base = par + (blk * RS1 + 64 + it) * 4096; }
            else base = par + (blk * RS1 + 64 + 3 * FI1 + j * FX + it - FI) * 4096;
        } else {
#pragma unroll
            for (int c = 0; c < 4; ++c) h[c] = cvt_w(v, c);
            d0 = (it - 64) >> 4; nd = 1; step = 4096;
            base = par + REL_B + (blk * CF + 64 + 16 * j + (it & 12)) * 4096 + (it & 3) * 1024;
        }
        if (n + NSPARE < NPUSH) { int b2, i2; push_decode(n + NSPARE, b2, i2); load_item(v, p, b2 * 16 + (i2 >> 3), i2 & 7, lane); }
        u32 dmask = 0;
        for (int d = 0; d < nd; ++d) {
            const int dr = (it < FI1 ? g * 4 : og) + ((d0 + d * dstep) & 3);
            const rsrc_t rd = mk_rsrc(p.ws[dr], WS_NEED);
#pragma unroll
            for (int i = 0; i < 4; ++i) st_sys(h[i], rd, lane * 16, base + d * ostep + i * step);
            dmask |= 1u << dr;
        }
        WAIT_VM0();
        if (lane < 8 && ((dmask >> lane) & 1)) add_sys(wsl + OFF_CNT + blk * CNT_BLK + (it < 64 ? 0 : 1024) + rank * 128);
    }
    STAMP(4);
}

template <bool CROSS>
__device__ __forceinline__ void pull_role(const Params& p, LAS char* scr, int slot, int wv, int lane, u32 call) {
    const int rank = rfl(p.rank);
    const int g = rank >> 2, j = rank & 3;
    const int q = slot - 3, peer = CROSS ? (g ^ 1) * 4 + q : g * 4 + ((j + 1 + slot) & 3);
    constexpr int N1 = NBLK * FI1;
    constexpr int PER = 16 + XA, NP = NBLK * PER, NX = NBLK * (16 + FX);
    constexpr int W = CROSS ? WC : WI, NT = CROSS ? NX + 1024 : N1 + NBLK * F2;
    const u32 par = OFF_PAR + (call & 1) * PAR_BYTES;
    char* ws = uni(p.ws[rank]);
    char* wp = uni(p.ws[peer]);
    const rsrc_t r0 = rsrc_u(ws, WS_NEED), ro = mk_rsrc(p.out[rank], OUT_BYTES);
    const char* rdy = ws + OFF_RDY + peer * 8192;
    const u32 vo = lane * 16, vout = (lane >> 4) * 4 * 4096 + (lane & 15) * 16;
    constexpr int SKIP = PB * (CROSS ? PER : FI1);
    const u32 head = lane == 0 ? OFF_QUE + slot * 256 : WS_NEED, tbase = SKIP + 3 * W - (call - 1) * (NT - SKIP);
    const bool nowait = (p.pad & 4) != 0;
    struct Item { u32 idx, dst; };
    auto decode = [&](int it) {
        Item m;
        it = it < NT ? it : NT - 1;
        if constexpr (!CROSS) {
            const bool s2 = it >= N1;
            const int a = it - N1, b1 = it / FI1;
            const int blk = s2 ? a >> 4 : b1, s = s2 ? FI1 + (a & 15) : it - b1 * FI1;
            m.idx = blk * 128 + s;
            m.dst = par + (s2 ? REL_A2 + (slot * F2 - FI1) * 4096 : (64 + slot * FI1) * 4096) + (blk * (s2 ? RF2 : RS1) + s) * 4096;
        } else if (it < NX) {
            const int blk = it / PER, c = it - blk * PER;
            const bool tail = it >= NP, isx = tail || (c < 2 * XA && (c & 1));
            const int cw = c < 2 * XA ? c >> 1 : c - XA;
            const int xi = tail ? it - NP + NBLK * XA : blk * XA + (c >> 1), xb = xi / FX, xs = FI + xi - xb * FX;
            m.idx = isx ? xb * 128 + xs : blk * 128 + 64 + 16 * j + cw;
            m.dst = par + (isx ? (xb * RS1 + xs + 64 + 3 * FI1 + q * FX - FI) * 4096
                               : REL_B + (blk * CF + 64 + 16 * q + (cw & 12)) * 4096 + (cw & 3) * 1024);
        } else {
            const int o = it - NX;
            m.idx = 8192 + o;
            int row, col;
            if (o < 832) {
                row = row_of_rf1((o >> 6) * 16 + ((o >> 3) & 3) * 4 + (o & 3), g ^ 1, q);
                col = ((o >> 5) & 1) * 128 + ((o >> 2) & 1) * 64;
            } else {
                const int z = o - 832, rt = z >> 5;
                row = ((g ^ 1) * 4 + ((q + 1 + (rt >> 1)) & 3)) * NS + (FI1 + (rt & 1) * 8 + (z & 7)) * 16;
                col = ((z >> 3) & 3) * 64;
            }
            m.dst = (u32)row * 4096 + (q * 256 + col) * 4;
        }
        return m;
    };
    auto flag_of = [&](u32 idx) { return idx < 8192 ? idx >> 3 : (idx < 8192 + 832 ? 1024 + ((idx - 8192) >> 2) : idx - (8192 + 832 - 1024 - 208)); };
    u32 okupto = 0, wbase = 0; u64 wmask = 0;
    auto ready = [&](u32 fi) {
        if (fi < okupto) return true;
        if (fi - wbase < 64u && ((wmask >> (fi - wbase)) & 1)) return true;
        for (;;) {
            wbase = okupto;
            wmask = __builtin_amdgcn_ballot_w64(ld_sys(rdy + (wbase + lane) * 4) >= call);
            if (wmask != ~0ull) break;
            okupto += 64;
            if (fi < okupto) return true;
        }
        okupto += __builtin_ctzll(~wmask);
        if (fi - wbase < 64u) return ((wmask >> (fi - wbase)) & 1) != 0;
        return rfl(ld_sys(rdy + fi * 4)) >= call;
    };
    u32x4 v[4];
    const u32 vo40 = lane < 40 ? vo : 0x80000000u, vo36 = lane < 36 ? vo : 0x80000000u;
    auto fetch = [&](u32 idx, bool on) {
        const rsrc_t ra = rsrc_u(wp, on ? WS_NEED : 0u);
        const bool out = CROSS && idx >= 8192;
        const u32 src = par + REL_PK + (idx + (out ? j * 1024 : 0)) * 4096;
#pragma unroll
        for (int i = 0; i < 3; ++i) v[i] = __builtin_amdgcn_raw_buffer_load_b128(ra, vo + i * 1024, src, 17);
        v[3] = __builtin_amdgcn_raw_buffer_load_b128(ra, (out ? vo36 : vo40) + 3072, src, 17);
    };
    const u32 vsig = lane == 0 ? 0u : 0x80000000u;
    auto signal = [&](u32 c) {
        const u32 one = 1;
        asm volatile("buffer_atomic_add %0, %1, %2, 0 offen sc1" :: "v"(one), "v"(vsig + c), "s"(r0) : "memory");
    };
    const u32x4 nan4 = {0x7fc07fc0u, 0x7fc07fc0u, 0x7fc07fc0u, 0x7fc07fc0u};
    int e = SKIP + wv, e1 = e + W, e2 = e + 2 * W;
    Item cu = decode(e), nx = decode(e1);
    spin_ge<4>(rdy + flag_of(cu.idx) * 4, call, ws + OFF_TMO, call);
    STAMP(5);
    bool bad = rfl(ld_agent(ws + OFF_TMO)) == call;
    fetch(cu.idx, true);
    bool pre = ready(flag_of(nx.idx));
    WAIT_VM0();
    STAMP(6);
    u32 owed = WS_NEED;
    while (e < NT) {
        u32 tk;
        if (!CROSS || cu.idx < 8192) {
            typedef u32 u32x2v __attribute__((ext_vector_type(2)));
            *(LAS u32x4*)(scr + 2048 + lane * 16) = v[3];
            asm volatile("" ::: "memory");
            const u32x2v pp = *(const LAS u32x2v*)(scr + 2048 + lane * 8);
            const u32 ent = *(const LAS unsigned short*)(scr + 2048 + 512 + lane * 2);
            const u32 hd = __builtin_amdgcn_readlane(v[3][0], 32) & 0xffffu;
            const u32x4 c0 = v[0], c1 = v[1], c2 = v[2];
            fetch(nx.idx, pre);
            tk = draw(r0, head);
            const bool isw = CROSS && (cu.idx & 64);
            const u32 step = isw ? 4096 : 1024;
            u32x4 h[4];
            if (hd < 32 && !bad) unpack_item(h, c0, c1, c2, pp[0], pp[1], ent, hd, scr, lane);
            else {
                const u32 blk = cu.idx >> 7, s = cu.idx & 127;
                const u32 raw = par + (isw ? REL_B + (blk * CF + (s & 60)) * 4096 + (s & 3) * 1024 : (blk * RS1 + s) * 4096);
                const rsrc_t rr = rsrc_u(wp, WS_NEED);
#pragma unroll
                for (int i = 0; i < 4; ++i) h[i] = __builtin_amdgcn_raw_buffer_load_b128(rr, vo, raw + i * step, 17);
                WAIT_VM0();
                if (bad) { h[0] = nan4; h[1] = nan4; h[2] = nan4; h[3] = nan4; }
            }
            asm volatile("s_waitcnt vmcnt(5)" ::: "memory");
            signal(owed);
#pragma unroll
            for (int i = 0; i < 4; ++i) st_sys(h[i], r0, vo + cu.dst, i * step);
            owed = OFF_CNT + (cu.idx >> 7) * CNT_BLK + peer * 128 + (CROSS ? (isw ? 1024 : 0) : ((cu.idx & 127) >= FI1 ? 2048 : 0));
        } else {
            typedef u32 u32x2v __attribute__((ext_vector_type(2)));
            *(LAS u32x4*)(scr + 2048 + lane * 16) = v[3];
            asm volatile("" ::: "memory");
            const u32x2v pp = *(const LAS u32x2v*)(scr + 2048 + lane * 8);
            const u32 ent = *(const LAS u32*)(scr + 2048 + 512 + (lane & 15) * 4);
            const u32 hd = __builtin_amdgcn_readlane(v[3][0], 32);
            const u32x4 c0 = v[0], c1 = v[1], c2 = v[2];
            fetch(nx.idx, pre);
            tk = draw(r0, head);
            u32x4 h[4];
            if (hd < 256 && !bad) unpack_out(h, c0, c1, c2, pp[0], pp[1], lane < 16 ? ent : hd, hd, scr, lane);
            else {
                const rsrc_t rr = rsrc_u(wp, WS_NEED);
#pragma unroll
                for (int i = 0; i < 4; ++i)
                    h[i] = __builtin_amdgcn_raw_buffer_load_b128(rr, vo + i * 1024, par + REL_PK + STG_BYTES + (cu.idx + j * 1024) * 4096, 17);
                WAIT_VM0();
                if (bad) { h[0] = nan4; h[1] = nan4; h[2] = nan4; h[3] = nan4; }
            }
            asm volatile("s_waitcnt vmcnt(5)" ::: "memory");
            signal(owed);
#pragma unroll
            for (int i = 0; i < 4; ++i) __builtin_amdgcn_raw_buffer_store_b128(h[i], ro, vout + cu.dst + i * 4096, 0, OUT_AUX);
            owed = WS_NEED;
        }
        const Item nn = decode(e2);
        bool pre2 = e2 < NT;
        if (pre2 && !(nowait && nn.idx >= 8192)) pre2 = ready(flag_of(nn.idx));
        asm volatile("s_waitcnt vmcnt(5)" : "+v"(tk) :: "memory");
        __builtin_amdgcn_s_waitcnt(0x0F74);
        if (!pre && e1 < NT) {
            WAIT_VM0();
            signal(owed); owed = WS_NEED;
            spin_ge<4>(rdy + flag_of(nx.idx) * 4, call, ws + OFF_TMO, call);
            bad = rfl(ld_agent(ws + OFF_TMO)) == call;
            fetch(nx.idx, true); WAIT_VM0();
        }
        STAMP_IF(CROSS ? (e < NP && e1 >= NP) : (e < N1 && e1 >= N1), 1);
        STAMP_IF(CROSS && e < NX && e1 >= NX, 2);
        STAMP_IF(CROSS && e < NX + 832 && e1 >= NX + 832, 3);
        cu = nx; nx = nn; pre = pre2;
        e = e1; e1 = e2; e2 = (int)(rfl(tk) + tbase);
    }
    WAIT_VM0();
    signal(owed);
    STAMP(4);
}

#define LDPOL " sc0 sc1"
#define GLDS2(m0v, voff, rs, soff, imm0, imm1) \
    asm volatile("s_mov_b32 m0, %0\n\ts_nop 0\n\tbuffer_load_dwordx4 %1, %2, %3 offen offset:" #imm0 LDPOL " lds\n\t" \
                 "buffer_load_dwordx4 %1, %2, %3 offen offset:" #imm1 LDPOL " lds" \
                 :: "s"(m0v), "v"(voff), "s"(rs), "s"(soff) : "memory")
#define BARRIER() do { __builtin_amdgcn_sched_barrier(0); __builtin_amdgcn_s_barrier(); __builtin_amdgcn_sched_barrier(0); } while (0)

#define SCHED() __builtin_amdgcn_sched_barrier(0)
#define WAIT_LGKM0() __builtin_amdgcn_s_waitcnt(0xC07F)

template <int STAGE>
struct Gemm {
    static constexpr int NT = STAGE == 1 ? 128 : 64;
    static constexpr int NA = STAGE == 1 ? 4 : 2;
    static constexpr int NB = STAGE == 1 ? 4 : 8;
    static constexpr int NM = STAGE == 1 ? 4 : 1;
    static constexpr int NMM = STAGE == 1 ? 16 : 8;
    static constexpr int NRD = NA + NB;
    struct Frags { f16x8 a[NA], b[NB]; };

    rsrc_t rs;
    u32 sa, sa2, sb;
    u32 ldsw;
    int wid, lane;

    template <int I> __device__ __forceinline__ void piece(int t, int buf) const {
        const u32 vo = lane * 16;
        const u32 ma = ldsw + buf * BUF_BYTES + wid * 4096, mb = ldsw + buf * BUF_BYTES + 32768 + wid * 2048;
        const u32 ka = STAGE == 1 ? (u32)(t >> 1) * (RS1 * 4096) + (t & 1) * 2048 : (u32)t * (RF2 * 4096);
        const u32 a = sa + ka, a2 = sa2 + ka;
        const u32 b = STAGE == 1 ? sb + (u32)(t >> 1) * (CF * 4096) + (t & 1) * 2048 : sb + (u32)t * (CF * 4096);
        if (I == 0) GLDS2(ma, vo, rs, a, 0, 1024);
        if (I == 1) GLDS2(ma, vo, rs, a2, 2048, 3072);
        if (I == 2) GLDS2(mb, vo, rs, b, 0, 1024);
    }
    __device__ __forceinline__ void stage(int t, int buf) const { piece<0>(t, buf); piece<1>(t, buf); piece<2>(t, buf); }

    template <int I> __device__ __forceinline__ void rd(Frags& f, const LAS char* lds, int buf, int half) const {
        const LAS char* base = lds + buf * BUF_BYTES + lane * 16;
        if constexpr (STAGE == 1) {
            const LAS char* pa = base + (wid >> 1) * 8192 + half * 1024;
            const LAS char* pb = base + 32768 + (wid & 1) * 8192 + half * 1024;
            if constexpr (I == 0) f.a[0] = *(const LAS f16x8*)pa;
            else if constexpr (I <= 4) f.b[I - 1] = *(const LAS f16x8*)(pb + (I - 1) * 2048);
            else f.a[I - 4] = *(const LAS f16x8*)(pa + (I - 4) * 2048);
        } else {
            const LAS char* pa = base + wid * 4096 + half * 2048;
            const LAS char* pb = base + 32768 + half * 2048;
            constexpr int h = I / 5, n = I % 5;
            if constexpr (n == 0) f.a[h] = *(const LAS f16x8*)(pa + h * 1024);
            else f.b[h * 4 + n - 1] = *(const LAS f16x8*)(pb + (n - 1) * 4096 + h * 1024);
        }
    }
    template <int I> __device__ __forceinline__ void mm(f32x4 (&acc)[NM][4], const Frags& f) const {
        if constexpr (STAGE == 1) acc[I >> 2][I & 3] = __builtin_amdgcn_mfma_f32_16x16x32_f16(f.a[I >> 2], f.b[I & 3], acc[I >> 2][I & 3], 0, 0, 0);
        else            acc[0][I & 3] = __builtin_amdgcn_mfma_f32_16x16x32_f16(f.a[I >> 2], f.b[I], acc[0][I & 3], 0, 0, 0);
    }
    template <int I, int N> __device__ __forceinline__ void rds(Frags& f, const LAS char* lds, int buf, int half) const {
        if constexpr (I < N) { rd<I>(f, lds, buf, half); rds<I + 1, N>(f, lds, buf, half); }
    }
    static __device__ __forceinline__ int blk_of(int t) { return STAGE == 1 ? (t >> 1) : t; }

    template <bool RD, int ST>
    __device__ __forceinline__ void half_step(f32x4 (&acc)[NM][4], const Frags& cur, Frags& nxt, const LAS char* lds, int rbuf, int rhalf,
                                              int ts, int bs, u32& fl, u32 cvo, u32 fso) const {
        if constexpr (STAGE == 1) {
            mm<0>(acc, cur);  mm<1>(acc, cur);  if (RD) rds<0, 2>(nxt, lds, rbuf, rhalf); SCHED();
            mm<2>(acc, cur);  mm<3>(acc, cur);  if (RD) rds<2, 4>(nxt, lds, rbuf, rhalf); SCHED();
            mm<4>(acc, cur);  mm<5>(acc, cur);  if (RD) rds<4, 6>(nxt, lds, rbuf, rhalf); SCHED();
            mm<6>(acc, cur);  mm<7>(acc, cur);  if (RD) rds<6, 8>(nxt, lds, rbuf, rhalf); SCHED();
            mm<8>(acc, cur);  mm<9>(acc, cur);  SCHED();
            if (ST == 1) asm volatile("buffer_load_dword %0, %1, %2, %3 offen sc0 sc1" : "=v"(fl) : "v"(cvo), "s"(rs), "s"(fso) : "memory");
            if (ST) { piece<0>(ts, bs); SCHED(); }
            mm<10>(acc, cur); mm<11>(acc, cur); SCHED();
            if (ST) { piece<1>(ts, bs); SCHED(); }
            mm<12>(acc, cur); mm<13>(acc, cur); SCHED();
            if (ST) { piece<2>(ts, bs); SCHED(); }
            mm<14>(acc, cur); mm<15>(acc, cur); SCHED();
        } else {
            mm<0>(acc, cur); if (RD) rds<0, 3>(nxt, lds, rbuf, rhalf); SCHED();
            mm<1>(acc, cur); if (RD) rds<3, 5>(nxt, lds, rbuf, rhalf); SCHED();
            mm<2>(acc, cur); if (RD) rds<5, 8>(nxt, lds, rbuf, rhalf); SCHED();
            mm<3>(acc, cur); if (RD) rds<8, 10>(nxt, lds, rbuf, rhalf); SCHED();
            mm<4>(acc, cur); SCHED();
            if (ST == 1) asm volatile("buffer_load_dword %0, %1, %2, %3 offen sc0 sc1" : "=v"(fl) : "v"(cvo), "s"(rs), "s"(fso) : "memory");
            if (ST) { piece<0>(ts, bs); SCHED(); }
            mm<5>(acc, cur); SCHED();
            if (ST) { piece<1>(ts, bs); SCHED(); }
            mm<6>(acc, cur); SCHED();
            if (ST) { piece<2>(ts, bs); SCHED(); }
            mm<7>(acc, cur); SCHED();
        }
    }

    template <bool LATE>
    __device__ __forceinline__ void run(f32x4 (&acc)[NM][4], const LAS char* lds, char* ws, u32 cvo, u32 need, u32 call) const {
        char* tmo = ws + OFF_TMO;
        Frags f0, f1;
        if (!LATE && wid == 0) spin_ge(ws + cvo + blk_of(0) * CNT_BLK, need, tmo, call);
        BARRIER();
        stage(0, 0);
        if (!LATE && wid == 0) { spin_ge(ws + cvo + blk_of(1) * CNT_BLK, need, tmo, call); spin_ge(ws + cvo + blk_of(2) * CNT_BLK, need, tmo, call); }
        BARRIER();
        stage(1, 1);
        if (!LATE) { stage(2, 2); asm volatile("s_waitcnt vmcnt(12)" ::: "memory"); }
        else asm volatile("s_waitcnt vmcnt(6)" ::: "memory");
        BARRIER();
        rds<0, NRD>(f0, lds, 0, 0);
        const u32 fvo = wid == 0 ? cvo : WS_NEED, fneed = wid == 0 ? need : 0u;
        u32 fl = 0;
        int buf = 0;
        for (int t = 0; t < NT - 3; ++t) {
            const int nb = buf == 2 ? 0 : buf + 1, pb = buf == 0 ? 2 : buf - 1;
            half_step<true, LATE ? 2 : 0>(acc, f0, f1, lds, buf, 1, t + 2, pb, fl, fvo, 0);
            if (!LATE) {
                asm volatile("s_waitcnt vmcnt(6)" : "+v"(fl) :: "memory");
                WAIT_LGKM0();
                if (__builtin_amdgcn_ballot_w64(fl < fneed) != 0) spin_ge(ws + cvo + blk_of(t + 3) * CNT_BLK, need, tmo, call);
            } else {
                asm volatile("s_waitcnt vmcnt(6)" ::: "memory");
                WAIT_LGKM0();
            }
            BARRIER();
            half_step<true, LATE ? 0 : 1>(acc, f1, f0, lds, nb, 0, t + 3, buf, fl, fvo, blk_of(t + 4 < NT ? t + 4 : t + 3) * CNT_BLK);
            buf = nb;
        }
#pragma unroll
        for (int t = NT - 3; t < NT; ++t) {
            const int nb = buf == 2 ? 0 : buf + 1, pb = buf == 0 ? 2 : buf - 1;
            if (t == NT - 3) half_step<true, LATE ? 2 : 0>(acc, f0, f1, lds, buf, 1, t + 2, pb, fl, fvo, 0);
            else             half_step<true, 0>(acc, f0, f1, lds, buf, 1, 0, 0, fl, fvo, 0);
            if (t == NT - 3) asm volatile("s_waitcnt vmcnt(6)" : "+v"(fl) :: "memory");
            else             asm volatile("s_waitcnt vmcnt(0)" : "+v"(fl) :: "memory");
            WAIT_LGKM0();
            BARRIER();
            if (t < NT - 1) half_step<true, 0>(acc, f1, f0, lds, nb, 0, 0, 0, fl, fvo, 0);
            else            half_step<false, 0>(acc, f1, f0, lds, nb, 0, 0, 0, fl, fvo, 0);
            buf = nb;
        }
    }
};

__device__ __forceinline__ u32x4 gelu4(const f32x4 (&c)[4], int r, bool bad) {
    const float nanv = __builtin_nanf("");
    f32x4 o = {gelu(c[0][r]), gelu(c[1][r]), gelu(c[2][r]), gelu(c[3][r])};
    if (bad) o = (f32x4){nanv, nanv, nanv, nanv};
    return __builtin_bit_cast(u32x4, o);
}

__device__ __forceinline__ void gemm_role(const Params& p, const LAS char* lds, int xcd, int s, int wid, int lane, u32 call) {
    const int rank = rfl(p.rank);
    const int g = rank >> 2, j = rank & 3, og = (g ^ 1) * 4;
    char* ws = p.ws[rank];
    char* tmo = ws + OFF_TMO;
    const u32 par = OFF_PAR + (call & 1) * PAR_BYTES;
    const rsrc_t rs = mk_rsrc(ws, WS_NEED), ro = mk_rsrc(p.out[rank], OUT_BYTES);
    const u32 ldsw = rfl((u32)(size_t)lds);
    const u32 vout = (lane >> 4) * 4 * 4096 + (lane & 15) * 16;
    {
        const int u = (xcd & 1) * 26 + s, rt = u >> 2, ct = (xcd >> 1) * 4 + (u & 3);
        Gemm<1> G;
        G.rs = rs; G.ldsw = ldsw; G.wid = wid; G.lane = lane;
        const int rf = rt * 16 + wid * 2;
        G.sa = par + (rf < FI ? rf : rf + FX) * 4096;
        G.sa2 = par + (rf + 1 < FI ? rf + 1 : rf + 1 + FX) * 4096 - 2048;
        G.sb = par + REL_B + (ct * 8 + wid) * 4096;
        const int bsrc = ct < 8 ? rank : og + ((ct - 8) >> 1);
        u32 cvo = OFF_CNT + 1024 + bsrc * 128, need = (ct < 8 ? 8u : 16u) * call;
        u32 mask = 0;
        for (int i = 0; i < 16; ++i) mask |= 1u << src_of_rf1(rt * 16 + i, g, j);
        if (lane < 8 && ((mask >> lane) & 1)) {
            cvo = OFF_CNT + lane * 128;
            need = (lane == rank ? 8 : ((lane >> 2) == g ? FI1 : FX)) * call;
        }
        f32x4 acc[4][4] = {};
        if (wid < 4) G.template run<false>(acc, lds, ws, cvo, need, call);
        else         G.template run<true>(acc, lds, ws, cvo, need, call);
        STAMP(1);
        const bool bad = rfl(ld_agent(tmo)) == call;
        const int wm = wid >> 1, wn = wid & 1;
        if (ct < 8) {
#pragma unroll
            for (int m = 0; m < 4; ++m) {
                const u32 ob = (u32)row_of_rf1(rt * 16 + wm * 4 + m, g, j) * 4096 + (ct * 128 + wn * 64) * 4;
#pragma unroll
                for (int r = 0; r < 4; ++r) __builtin_amdgcn_raw_buffer_store_b128(gelu4(acc[m], r, bad), ro, vout + ob + r * 4096, 0, OUT_AUX);
            }
        } else {
            const int fo = (rt * 2 + (ct & 1)) * 8 + wid;
            const u32 sb = par + REL_STG + (u32)(((ct - 8) >> 1) * 1024 + fo * 4) * 4096;
#pragma unroll
            for (int m = 0; m < 4; ++m) {
                const u32x4 o[4] = {gelu4(acc[m], 0, bad), gelu4(acc[m], 1, bad), gelu4(acc[m], 2, bad), gelu4(acc[m], 3, bad)};
                pack_out(o, (LAS char*)lds + wid * 2048, rs, sb + m * 4096, lane);
            }
            WAIT_VM0();
            if (lane == 0) st_flag(p.ws[bsrc] + OFF_RDY + rank * 8192 + (1024 + fo) * 4, call);
        }
        STAMP(2);
    }
    if (s < 24) {
        const int rt = s >> 2, ct = ((xcd & 1) * 16) + (xcd >> 1) * 4 + (s & 3);
        Gemm<2> G;
        G.rs = rs; G.ldsw = ldsw; G.wid = wid; G.lane = lane;
        G.sa = G.sa2 = par + REL_A2 + (rt * 8 + wid) * 4096;
        G.sb = par + REL_B + ct * 4 * 4096 + wid * 2048;
        const int asrc = g * 4 + ((j + 1 + (rt >> 1)) & 3);
        const int bsrc = ct < 16 ? rank : og + ((ct - 16) >> 2);
        u32 cvo = OFF_CNT + 1024 + bsrc * 128, need = (ct < 16 ? 8u : 16u) * call;
        if (lane == 0) { cvo = OFF_CNT + 2048 + asrc * 128; need = F2 * call; }
        f32x4 acc[1][4] = {};
        if (wid < 4) G.template run<false>(acc, lds, ws, cvo, need, call);
        else         G.template run<true>(acc, lds, ws, cvo, need, call);
        STAMP(3);
        const bool bad = rfl(ld_agent(tmo)) == call;
        if (ct < 16) {
            const u32 ob = (u32)(asrc * NS + (FI1 + (rt & 1) * 8 + wid) * 16) * 4096 + ct * 256;
#pragma unroll
            for (int r = 0; r < 4; ++r) __builtin_amdgcn_raw_buffer_store_b128(gelu4(acc[0], r, bad), ro, vout + ob + r * 4096, 0, OUT_AUX);
        } else {
            const int z = (rt * 4 + (ct & 3)) * 8 + wid;
            const u32 sb = par + REL_STG + (u32)(((ct - 16) >> 2) * 1024 + 832 + z) * 4096;
            const u32x4 o[4] = {gelu4(acc[0], 0, bad), gelu4(acc[0], 1, bad), gelu4(acc[0], 2, bad), gelu4(acc[0], 3, bad)};
            pack_out(o, (LAS char*)lds + wid * 2048, rs, sb, lane);
            WAIT_VM0();
            if (lane == 0) st_flag(p.ws[bsrc] + OFF_RDY + rank * 8192 + (1024 + 208 + z) * 4, call);
        }
        STAMP(4);
    }
}

__global__ __launch_bounds__(BLOCK, 2) void ag_gemm_kernel(Params p) {
    __shared__ __attribute__((aligned(1024))) char smem[LDS_BYTES];
    const int rank = rfl(p.rank);
    const int lane = threadIdx.x & 63;
    const int wid = rfl(threadIdx.x >> 6);
    const int b = blockIdx.x, xcd = b & 7, slot = b >> 3;
    const int gw = wid * NCOMM + b;
    if (b < NCOMM && gw >= NPULL + NPUSH) return;
    STAMP(0);
    char* seq = p.ws[rank] + OFF_SEQ + (b * 8 + wid) * 4;
    const u32 prev = ld_agent(seq);
    char* wsl = p.ws[lane & 7];
    f32x4 v0[8];
    if (b >= NCOMM) {
        constexpr int B0 = PB > 0 ? PB - 1 : 0;
        if ((u32)(b - NCOMM - B0 * 16) >= EARLY * 16) __builtin_amdgcn_s_sleep(LATE_SLEEP);
        load_item(v0, p, b - NCOMM, wid, lane);
    } else if (gw >= NPULL) {
        int blk, it;
        push_decode(gw - NPULL, blk, it);
        load_item(v0, p, blk * 16 + (it >> 3), it & 7, lane);
    }
    const u32 call = rfl(prev) + 1;
    if (lane == 0) __hip_atomic_store((GAS u32*)seq, call, __ATOMIC_RELAXED, __HIP_MEMORY_SCOPE_AGENT);
    STAMP(7);
    if (b < NCOMM) {
        if (gw >= NPULL) { push_role(p, v0, wsl, gw - NPULL, lane, call); return; }
        constexpr int WM = WI < WC ? WI : WC;
        int ps, wv;
        if (gw < 7 * WM) { ps = gw % 7; wv = gw / 7; }
        else if (WI > WC) { ps = (gw - 7 * WM) % 3; wv = WM + (gw - 7 * WM) / 3; }
        else { ps = 3 + (gw - 7 * WM) % 4; wv = WM + (gw - 7 * WM) / 4; }
        if (ps < 3) pull_role<false>(p, (LAS char*)smem + wid * 4096, rfl(ps), rfl(wv), lane, call);
        else        pull_role<true>(p, (LAS char*)smem + wid * 4096, rfl(ps), rfl(wv), lane, call);
    } else {
        convert_local(p, (LAS char*)smem + wid * 4096, v0, wsl, b - NCOMM, wid, lane, call);
        STAMP(5);
        gemm_role(p, (const LAS char*)smem, xcd, slot - NCOMM / 8, wid, lane, call);
    }
}

}

constexpr int g_mode = 0;

extern "C" void kernel_launch(int rank, int nranks, ncclComm_t comm, hipStream_t stream,
                              void* const* d_in, const int* in_sizes, int n_in,
                              void* d_out, int out_size, const int* in_dtypes, int out_dtype,
                              void* const* d_out_all, void* const* d_ws_all, size_t ws_bytes) {
    (void)comm; (void)in_sizes; (void)n_in; (void)d_out; (void)out_size; (void)in_dtypes; (void)out_dtype;
    if (nranks != NR || (unsigned)rank >= (unsigned)NR || ws_bytes < WS_NEED) {
        fprintf(stderr, "unsupported configuration\n");
        return;
    }
    Params p;
    p.x = (const float*)d_in[0];
    p.w = (const float*)d_in[1];
    for (int i = 0; i < NR; ++i) { p.ws[i] = (char*)d_ws_all[i]; p.out[i] = (char*)d_out_all[i]; }
    p.rank = rank;
    p.pad = g_mode;
    ag_gemm_kernel<<<GRID, BLOCK, 0, stream>>>(p);
    hipError_t e = hipGetLastError();
    if (e != hipSuccess) fprintf(stderr, "launch failed: %s\n", hipGetErrorString(e));
}
```

```cpp
#include <hip/hip_runtime.h>
#include <rccl/rccl.h>
#include <cstdio>
#include <cstdint>

typedef unsigned u32;
typedef unsigned long long u64;
typedef u32 u32x4 __attribute__((ext_vector_type(4)));
typedef u32 u32x2v __attribute__((ext_vector_type(2)));
typedef float f32x4 __attribute__((ext_vector_type(4)));
typedef _Float16 f16x8 __attribute__((ext_vector_type(8)));
typedef __amdgpu_buffer_rsrc_t rsrc_t;
#define LAS __attribute__((address_space(3)))
#define GAS __attribute__((address_space(1)))

namespace {

constexpr int NR = 8;
constexpr int KD = 8192;
constexpr int NS = 1024;
constexpr int NBLK = 64;
#define P_FI 46
constexpr int FI = P_FI;
constexpr int F2 = 16;
constexpr int FI1 = FI - F2;
constexpr int FX = 64 - FI;
constexpr int RF1 = 208, RF2 = 48, CF = 128;
static_assert(FI + 3 * FI1 + 4 * FX == RF1, "stage-1 rows");

#define P_WI 19
#define P_WC 19
#define P_XA 14
#define P_EARLY 4
#define P_SLEEP 100
#define P_PF 1
#define P_PB 1
constexpr int PB = P_PB;
constexpr int EARLY = P_EARLY, LATE_SLEEP = P_SLEEP, PF_FROM = P_PF;
constexpr int NCOMM = 48;
constexpr int WI = P_WI, WC = P_WC;
constexpr int NPULL = 3 * WI + 4 * WC;
constexpr int XA = P_XA;
static_assert(NPULL <= NCOMM * 4 && XA <= FX && XA <= 16, "wave roles");
constexpr int GRID = 256, BLOCK = 512, NGEMM = GRID - NCOMM;
constexpr int RS1 = 64 + 3 * FI1 + 4 * FX;

constexpr u32 OFF_SEQ  = 0;
constexpr u32 OFF_TMO  = 16384;
constexpr u32 OFF_QUE  = 16640;
constexpr u32 OFF_RDY  = 32768;
constexpr u32 OFF_CNT  = 131072;
constexpr u32 CNT_BLK  = 24 * 128;
constexpr u32 OFF_DIAG = 524288;
constexpr u32 A1_BYTES = (u32)NBLK * RS1 * 4096;
constexpr u32 A2_BYTES = (u32)NBLK * RF2 * 4096;
constexpr u32 B_BYTES  = (u32)NBLK * CF * 4096;
constexpr u32 STG_BYTES = 4u * 1024 * 4096;
constexpr u32 PK_BYTES = (u32)NBLK * 128 * 4096;
constexpr u32 REL_A2 = A1_BYTES, REL_B = REL_A2 + A2_BYTES, REL_PK = REL_B + B_BYTES, REL_STG = REL_PK + PK_BYTES;
constexpr u32 PAR_BYTES = REL_STG + 2 * STG_BYTES;
constexpr u32 OFF_PAR  = 1u << 20;
constexpr u32 WS_NEED  = OFF_PAR + 2 * PAR_BYTES;
constexpr u32 OUT_BYTES = (u32)KD * NS * 4;
static_assert(PK_BYTES == 8192u * 4096u, "staged outputs follow the packed image");
static_assert(OFF_CNT + NBLK * CNT_BLK <= OFF_DIAG && OFF_RDY + 8 * 8192 <= OFF_CNT, "control area");

constexpr u64 DEADLINE = 40000000ull;

constexpr int BUF_BYTES = 49152;
constexpr int LDS_BYTES = 3 * BUF_BYTES;

struct Params {
    const float* x;
    const float* w;
    char* ws[NR];
    char* out[NR];
    int rank;
    int pad;
};

__device__ __forceinline__ u32 rfl(u32 v) { return __builtin_amdgcn_readfirstlane(v); }
__device__ __forceinline__ char* uni(const void* p) {
    const u64 a = (u64)p;
    return (char*)(((u64)rfl((u32)(a >> 32)) << 32) | rfl((u32)a));
}
__device__ __forceinline__ rsrc_t rsrc_u(const void* p, u32 bytes) {
    return __builtin_amdgcn_make_buffer_rsrc((void*)p, (short)0, (int)bytes, 0x00020000);
}
__device__ __forceinline__ rsrc_t mk_rsrc(const void* p, u32 bytes) { return rsrc_u(uni(p), bytes); }
__device__ __forceinline__ u32 ld_sys(const char* p) {
    return __hip_atomic_load((const GAS u32*)p, __ATOMIC_RELAXED, __HIP_MEMORY_SCOPE_SYSTEM);
}
__device__ __forceinline__ u32 ld_agent(const char* p) {
    return __hip_atomic_load((const GAS u32*)p, __ATOMIC_RELAXED, __HIP_MEMORY_SCOPE_AGENT);
}
__device__ __forceinline__ void st_flag(char* p, u32 v) {
    __hip_atomic_store((GAS u32*)p, v, __ATOMIC_RELAXED, __HIP_MEMORY_SCOPE_SYSTEM);
}
__device__ __forceinline__ void add_sys(char* p) {
    __hip_atomic_fetch_add((GAS u32*)p, 1u, __ATOMIC_RELAXED, __HIP_MEMORY_SCOPE_SYSTEM);
}
__device__ __forceinline__ u32 draw(rsrc_t r, u32 voff) {
    u32 t = 1;
    asm volatile("s_nop 0\n\tbuffer_atomic_add %0, %1, %2, 0 offen sc0" : "+v"(t) : "v"(voff), "s"(r) : "memory");
    return t;
}
#define WAIT_VM0() do { asm volatile("s_waitcnt vmcnt(0)" ::: "memory"); __builtin_amdgcn_s_waitcnt(0x0F70); } while (0)
#define WAIT_VM0_T(t) do { asm volatile("s_waitcnt vmcnt(0)" : "+v"(t) :: "memory"); __builtin_amdgcn_s_waitcnt(0x0F70); } while (0)

template <int SLEEP = 2>
__device__ __forceinline__ bool spin_ge(const char* p, u32 need, char* tmo, u32 call) {
    const u64 t0 = __builtin_amdgcn_s_memrealtime();
    bool gave_up = false;
    for (;;) {
        const u32 v = ld_sys(p);
        if (__builtin_amdgcn_ballot_w64(v < need) == 0) break;
        if (rfl(ld_agent(tmo)) == call || __builtin_amdgcn_s_memrealtime() - t0 > DEADLINE) {
            __hip_atomic_store((GAS u32*)tmo, call, __ATOMIC_RELAXED, __HIP_MEMORY_SCOPE_AGENT);
            gave_up = true;
            break;
        }
        __builtin_amdgcn_s_sleep(SLEEP);
    }
    asm volatile("" ::: "memory");
    return gave_up;
}

#define TICK(v) do {} while (0)
#define ACC(a, x) do {} while (0)
#define STAMP_IF(c, slot) do {} while (0)
#define STAMP(slot) do {} while (0)
#define STAMPB_IF(c, slot) do {} while (0)

__device__ __forceinline__ u32x4 cvt8(f32x4 a, f32x4 b) {
    f16x8 h;
    h[0] = (_Float16)a[0]; h[1] = (_Float16)a[1]; h[2] = (_Float16)a[2]; h[3] = (_Float16)a[3];
    h[4] = (_Float16)b[0]; h[5] = (_Float16)b[1]; h[6] = (_Float16)b[2]; h[7] = (_Float16)b[3];
    return __builtin_bit_cast(u32x4, h);
}
__device__ __forceinline__ void st_sys(u32x4 v, rsrc_t r, u32 voff, u32 soff) {
    __builtin_amdgcn_raw_buffer_store_b128(v, r, voff + soff, 0, 17);
}
__device__ __forceinline__ float gelu(float y) {
    const float u = y * (0.7978845608028654f + 0.7978845608028654f * 0.044715f * y * y);
    const float e = __builtin_amdgcn_exp2f(u * -2.8853900817779268f);
    return y * __builtin_amdgcn_rcpf(1.0f + e);
}

__device__ __forceinline__ int row_of_rf1(int rf, int g, int j) {
    if (rf < FI) return (g * 4 + j) * NS + rf * 16;
    rf -= FI;
    if (rf < 3 * FI1) { const int i = rf / FI1; return (g * 4 + ((j + 1 + i) & 3)) * NS + (rf - i * FI1) * 16; }
    rf -= 3 * FI1;
    const int q = rf / FX;
    return ((g ^ 1) * 4 + q) * NS + (FI + rf - q * FX) * 16;
}
__device__ __forceinline__ int src_of_rf1(int rf, int g, int j) { return row_of_rf1(rf, g, j) / NS; }

#define LDIN(p) __builtin_nontemporal_load(p)
#define P_OUT_AUX 17
constexpr int OUT_AUX = P_OUT_AUX;
__device__ __forceinline__ void load_x(f32x4* v, const float* x, int frag, int blk, int lane) {
    const float* s = x + (size_t)(frag * 16 + (lane & 15)) * KD + blk * 128 + (lane >> 4) * 4;
#pragma unroll
    for (int i = 0; i < 8; ++i) v[i] = LDIN((const f32x4*)(s + i * 16));
}
__device__ __forceinline__ void load_w(f32x4* v, const float* w, int cg, int k0, int lane) {
    const float* s = w + (size_t)(k0 + (lane >> 4) * 4) * NS + cg * 64 + (lane & 15) * 4;
#pragma unroll
    for (int i = 0; i < 8; ++i) v[i] = LDIN((const f32x4*)(s + ((i & 3) + (i >> 2) * 16) * NS));
}
__device__ __forceinline__ u32x4 cvt_w(const f32x4* v, int c) {
    return cvt8((f32x4){v[0][c], v[1][c], v[2][c], v[3][c]}, (f32x4){v[4][c], v[5][c], v[6][c], v[7][c]});
}

#define DPP_STEP(op, v, ctrl, rmask) v = op(v, (u32)__builtin_amdgcn_update_dpp(0, (int)v, ctrl, rmask, 0xf, false))
__device__ __forceinline__ u32 umax2(u32 a, u32 b) { return a > b ? a : b; }
__device__ __forceinline__ u32 uadd2(u32 a, u32 b) { return a + b; }
__device__ __forceinline__ u32 wave_max(u32 v) {
    DPP_STEP(umax2, v, 0x111, 0xf); DPP_STEP(umax2, v, 0x112, 0xf); DPP_STEP(umax2, v, 0x114, 0xf); DPP_STEP(umax2, v, 0x118, 0xf);
    DPP_STEP(umax2, v, 0x142, 0xa); DPP_STEP(umax2, v, 0x143, 0xc);
    return __builtin_amdgcn_readlane(v, 63);
}
__device__ __forceinline__ u32 wave_sum(u32 v) {
    DPP_STEP(uadd2, v, 0x111, 0xf); DPP_STEP(uadd2, v, 0x112, 0xf); DPP_STEP(uadd2, v, 0x114, 0xf); DPP_STEP(uadd2, v, 0x118, 0xf);
    DPP_STEP(uadd2, v, 0x142, 0xa); DPP_STEP(uadd2, v, 0x143, 0xc);
    return __builtin_amdgcn_readlane(v, 63);
}

__device__ __forceinline__ u32 h7_of(float a) { return (u32)__builtin_bit_cast(unsigned short, (_Float16)a) >> 8; }
__device__ __forceinline__ void pack_item(const u32x4 (&h)[4], float amax, LAS char* scr, rsrc_t r0, u32 dst, int lane) {
    u32 L[8], H[8];
#pragma unroll
    for (int t = 0; t < 8; ++t) {
        const u32 d0 = h[t >> 1][(t & 1) * 2], d1 = h[t >> 1][(t & 1) * 2 + 1];
        L[t] = __builtin_amdgcn_perm(d1, d0, 0x06040200u);
        H[t] = __builtin_amdgcn_perm(d1, d0, 0x07050301u);
    }
    *(LAS u32x4*)(scr + lane * 16) = (u32x4){H[0], H[1], H[2], H[3]};
    *(LAS u32x4*)(scr + lane * 16 + 1024) = (u32x4){H[4], H[5], H[6], H[7]};
    const u32 lm = h7_of(amax) & 127, tmax = wave_max(lm), tavg = (wave_sum(lm) + 192) >> 6;
    const u32 top = tmax < tavg ? tmax : tavg, lo = top > 31 ? top - 31 : 0, lo1 = lo * 0x01010101u;
    u32 F[8], em = 0;
#pragma unroll
    for (int t = 0; t < 8; ++t) {
        const u32 u = ((H[t] | 0x80808080u) - lo1) ^ 0x80808080u;
        const u32 x = u & 0xe0e0e0e0u;
        const u32 nz = (x | (x << 1) | (x << 2)) & 0x80808080u;
        const u32 g = u & 0x1f1f1f1fu & ~((nz >> 2) - (nz >> 7));
        F[t] = g | ((H[t] >> 2) & 0x20202020u);
        em = (em >> 1) | (~(g + 0x7f7f7f7fu) & 0x80808080u);
    }
    LAS unsigned char* list = (LAS unsigned char*)(scr + 2048);
    u32 used = 1;
    for (u64 act = __builtin_amdgcn_ballot_w64(em != 0); act != 0; act = __builtin_amdgcn_ballot_w64(em != 0)) {
        const u32 k = used + __builtin_amdgcn_mbcnt_hi((u32)(act >> 32), __builtin_amdgcn_mbcnt_lo((u32)act, 0));
        const u32 b = __builtin_ctz(em | 0x80000000u);
        list[em != 0 && k < 64 ? k : 64] = *(const LAS unsigned char*)(scr + (b & 4) * 256 + lane * 16 + (b & 3) * 4 + (b >> 3)) & 127;
        em &= em - 1;
        used += __builtin_popcountll(act);
        if (used > 64) break;
    }
    list[0] = used > 64 ? 255 : lo;
    asm volatile("" ::: "memory");
    const u32 ent = *(const LAS u32*)(list + (lane & 15) * 4);
    st_sys((u32x4){L[0], L[1], L[2], L[3]}, r0, lane * 16, dst);
    st_sys((u32x4){L[4], L[5], L[6], L[7]}, r0, lane * 16, dst + 1024);
    st_sys((u32x4){F[0] | ((F[3] & 0x03030303u) << 6), F[1] | ((F[3] & 0x0c0c0c0cu) << 4), F[2] | ((F[3] & 0x30303030u) << 2),
                   F[4] | ((F[7] & 0x03030303u) << 6)}, r0, lane * 16, dst + 2048);
    __builtin_amdgcn_raw_buffer_store_b64((u32x2v){F[5] | ((F[7] & 0x0c0c0c0cu) << 4), F[6] | ((F[7] & 0x30303030u) << 2)},
                                          r0, lane * 8 + dst + 3072, 0, 17);
    __builtin_amdgcn_raw_buffer_store_b32(ent, r0, (lane < 16 ? lane * 4 : 0x80000000u) + dst + 3584, 0, 17);
}
__device__ __forceinline__ void unpack_item(u32x4 (&h)[4], const u32x4& c0, const u32x4& c1, const u32x4& c2, u32 p4, u32 p5,
                                            const LAS unsigned char* list, u32 lo, LAS char* scr, int lane) {
    const u32 P[6] = {c2[0], c2[1], c2[2], c2[3], p4, p5};
    const u32 lo1 = lo * 0x01010101u;
    u32x4 H[2];
    u32 em = 0;
#pragma unroll
    for (int z = 0; z < 2; ++z) {
        u32 F[4] = {P[3 * z] & 0x3f3f3f3fu, P[3 * z + 1] & 0x3f3f3f3fu, P[3 * z + 2] & 0x3f3f3f3fu,
                    ((P[3 * z] >> 6) & 0x03030303u) | ((P[3 * z + 1] >> 4) & 0x0c0c0c0cu) | ((P[3 * z + 2] >> 2) & 0x30303030u)};
#pragma unroll
        for (int t = 0; t < 4; ++t) {
            const u32 sg = F[t] & 0x20202020u;
            H[z][t] = F[t] + ((sg << 1) + sg) + lo1;
            em = (em >> 1) | (~((F[t] & 0x1f1f1f1fu) + 0x7f7f7f7fu) & 0x80808080u);
        }
        *(LAS u32x4*)(scr + z * 1024 + lane * 16) = H[z];
    }
    u32 used = 1;
    auto step = [&](u32& b, u32& val) {
        const u64 act = __builtin_amdgcn_ballot_w64(em != 0);
        const u32 k = used + __builtin_amdgcn_mbcnt_hi((u32)(act >> 32), __builtin_amdgcn_mbcnt_lo((u32)act, 0));
        b = __builtin_ctz(em | 0x80000000u);
        val = list[em != 0 ? k : 0];
        em &= em - 1;
        used += __builtin_popcountll(act);
    };
    auto patch = [&](u32 b, u32 val) {
        __hip_atomic_fetch_add((LAS u32*)(scr + (b & 4) * 256 + lane * 16 + (b & 3) * 4), (val - lo) << (b & 24),
                               __ATOMIC_RELAXED, __HIP_MEMORY_SCOPE_WORKGROUP);
    };
    u32 b0, b1, b2, x0, x1, x2;
    step(b0, x0); step(b1, x1); step(b2, x2);
    patch(b0, x0); patch(b1, x1); patch(b2, x2);
    while (__builtin_amdgcn_ballot_w64(em != 0) != 0) { step(b0, x0); patch(b0, x0); }
#pragma unroll
    for (int z = 0; z < 2; ++z) {
        H[z] = *(const LAS u32x4*)(scr + z * 1024 + lane * 16);
        const u32x4& L = z ? c1 : c0;
#pragma unroll
        for (int t = 0; t < 4; ++t) {
            h[z * 2 + (t >> 1)][(t & 1) * 2] = __builtin_amdgcn_perm(H[z][t], L[t], 0x05010400u);
            h[z * 2 + (t >> 1)][(t & 1) * 2 + 1] = __builtin_amdgcn_perm(H[z][t], L[t], 0x07030602u);
        }
    }
}

__device__ __forceinline__ void pack_out(const u32x4 (&o)[4], LAS char* scr, rsrc_t r0, u32 dst, int lane) {
    u32 M[12], mx = 0;
    u32x4 E;
#pragma unroll
    for (int r = 0; r < 4; ++r) {
        u32 R[4];
#pragma unroll
        for (int c = 0; c < 4; ++c) { R[c] = __builtin_amdgcn_alignbit(o[r][c], o[r][c], 31); mx = umax2(mx, R[c]); }
#pragma unroll
        for (int i = 0; i < 3; ++i) M[r * 3 + i] = __builtin_amdgcn_perm(R[3], R[i], 0x04020100u + (i << 24));
        E[r] = __builtin_amdgcn_perm(R[1], R[0], 0x0c0c0703u) | __builtin_amdgcn_perm(R[3], R[2], 0x07030c0cu);
    }
    *(LAS u32x4*)(scr + lane * 16) = E;
    const u32 emax = wave_max(mx) >> 24, lo = emax > 15 ? emax - 15 : 0, lo1 = lo * 0x01010101u;
    LAS u32* list = (LAS u32*)(scr + 1024);
    list[lane] = lo;
    u32 C[4], em = 0;
#pragma unroll
    for (int r = 0; r < 4; ++r) {
        const u32 d = ((E[r] | 0x80808080u) - (lo1 & 0x7f7f7f7fu)) ^ ((E[r] ^ ~lo1) & 0x80808080u);
        const u32 x = d & 0xf0f0f0f0u, y = x | (x << 1), nz = (y | (y << 2)) & 0x80808080u;
        C[r] = d & 0x0f0f0f0fu & ~((nz >> 3) - (nz >> 7));
        em = (em >> 1) | nz;
    }
    u32 used = 1;
    for (u64 act = __builtin_amdgcn_ballot_w64(em != 0); act != 0; act = __builtin_amdgcn_ballot_w64(em != 0)) {
        const u32 k = used + __builtin_amdgcn_mbcnt_hi((u32)(act >> 32), __builtin_amdgcn_mbcnt_lo((u32)act, 0));
        const u32 b = __builtin_ctz(em | 0x80000000u);
        const u32 pos = lane * 16 + (b & 3) * 4 + (b >> 3);
        list[em != 0 && k < 16 ? k : 64] = pos * 256 + *(const LAS unsigned char*)(scr + pos);
        em &= em - 1;
        used += __builtin_popcountll(act);
        if (used > 16) break;
    }
    asm volatile("" ::: "memory");
    u32 ent = list[lane];
    if (used > 16) {
        if (lane == 0) ent = 0xffffffffu;
#pragma unroll
        for (int r = 0; r < 4; ++r) st_sys(o[r], r0, lane * 16 + r * 1024, dst + STG_BYTES);
    }
#pragma unroll
    for (int i = 0; i < 3; ++i) st_sys((u32x4){M[4 * i], M[4 * i + 1], M[4 * i + 2], M[4 * i + 3]}, r0, lane * 16 + i * 1024, dst);
    __builtin_amdgcn_raw_buffer_store_b64((u32x2v){C[0] | (C[1] << 4), C[2] | (C[3] << 4)}, r0, lane * 8 + dst + 3072, 0, 17);
    __builtin_amdgcn_raw_buffer_store_b32(ent, r0, (lane < 16 ? lane * 4 : 0x80000000u) + dst + 3584, 0, 17);
}
__device__ __forceinline__ void unpack_out(u32x4 (&o)[4], const u32x4& c0, const u32x4& c1, const u32x4& c2, u32 p4, u32 p5,
                                           u32 ent, u32 lo, LAS char* scr, int lane) {
    const u32 M[12] = {c0[0], c0[1], c0[2], c0[3], c1[0], c1[1], c1[2], c1[3], c2[0], c2[1], c2[2], c2[3]};
    const u32 lo1 = lo * 0x01010101u;
    u32x4 E = {(p4 & 0x0f0f0f0fu) + lo1, ((p4 >> 4) & 0x0f0f0f0fu) + lo1, (p5 & 0x0f0f0f0fu) + lo1, ((p5 >> 4) & 0x0f0f0f0fu) + lo1};
    *(LAS u32x4*)(scr + lane * 16) = E;
    __hip_atomic_fetch_add((LAS u32*)(scr + ((ent >> 8) & 0x3fc)), ((ent & 255) - lo) << ((ent >> 5) & 24),
                           __ATOMIC_RELAXED, __HIP_MEMORY_SCOPE_WORKGROUP);
    asm volatile("" ::: "memory");
    E = *(const LAS u32x4*)(scr + lane * 16);
#pragma unroll
    for (int r = 0; r < 4; ++r) {
        u32 R[4];
#pragma unroll
        for (int i = 0; i < 3; ++i) R[i] = __builtin_amdgcn_perm(E[r], M[r * 3 + i], 0x04020100u + (i << 24));
        R[3] = __builtin_amdgcn_perm(M[r * 3 + 1], M[r * 3], 0x0c0c0703u) | __builtin_amdgcn_perm(E[r], M[r * 3 + 2], 0x07030c0cu);
#pragma unroll
        for (int c = 0; c < 4; ++c) o[r][c] = __builtin_amdgcn_alignbit(R[c], R[c], 1);
    }
}

__device__ __forceinline__ void load_item(f32x4 (&v)[8], const Params& p, int u, int wid, int lane) {
    const int blk = u >> 4, it = (u & 15) * 8 + wid;
    if (it < 64) load_x(v, p.x, it, blk, lane);
    else load_w(v, p.w, (it - 64) >> 2, blk * 128 + (wid & 3) * 32, lane);
}
__device__ __forceinline__ void convert_local(const Params& p, LAS char* scr, f32x4 (&v)[8], char* wsl, int wg, int wid, int lane, u32 call) {
    const int rank = rfl(p.rank);
    const u32 par = OFF_PAR + (call & 1) * PAR_BYTES;
    const rsrc_t r0 = mk_rsrc(p.ws[rank], WS_NEED);
    char* flag = wsl + OFF_RDY + rank * 8192;
    for (int u = wg; u < NBLK * 16; u += NGEMM) {
        const int blk = u >> 4, grp = u & 15, it = grp * 8 + wid;
        u32x4 h[4];
        u32 base, step;
        if (grp < 8) {
            base = par + (u32)blk * (RS1 * 4096) + it * 4096; step = 1024;
#pragma unroll
            for (int i = 0; i < 4; ++i) h[i] = cvt8(v[2 * i], v[2 * i + 1]);
        } else {
            base = par + REL_B + (u32)blk * (CF * 4096) + ((it - 64) & 60) * 4096 + (wid & 3) * 1024; step = 4096;
#pragma unroll
            for (int c = 0; c < 4; ++c) h[c] = cvt_w(v, c);
        }
        float amax = 0.0f;
#pragma unroll
        for (int i = 0; i < 8; ++i) amax = fmaxf(fmaxf(fmaxf(amax, fabsf(v[i][0])), fmaxf(fabsf(v[i][1]), fabsf(v[i][2]))), fabsf(v[i][3]));
        STAMPB_IF(u == wg, 0);
        const bool late = u < PF_FROM * NGEMM;
        if (!late && u + NGEMM < NBLK * 16) load_item(v, p, u + NGEMM, wid, lane);
#pragma unroll
        for (int i = 0; i < 4; ++i) st_sys(h[i], r0, lane * 16, base + i * step);
        pack_item(h, amax, scr, r0, par + REL_PK + (u32)(blk * 128 + it) * 4096, lane);
        STAMPB_IF(u == wg, 1);
        WAIT_VM0();
        STAMPB_IF(u == wg, 2);
        __builtin_amdgcn_s_barrier();
        STAMP_IF(u == wg, 6);
        if (wid == 0) {
            if (lane < 8 && lane != rank) st_flag(flag + u * 4, call);
            if (lane == 8) add_sys(p.ws[rank] + OFF_CNT + blk * CNT_BLK + (grp < 8 ? 0 : 1024) + rank * 128);
        }
        if (late && u + NGEMM < NBLK * 16) load_item(v, p, u + NGEMM, wid, lane);
    }
}

constexpr int PUSH_PER = FI1 + 64 + XA, NPUSH = PB * PUSH_PER, NSPARE = 2 * (NCOMM * 4 - NPULL);
__device__ __forceinline__ void push_decode(int n, int& blk, int& it) {
    blk = n / PUSH_PER; it = n - blk * PUSH_PER;
    if (it >= FI1 + 64) { const int xi = blk * XA + it - FI1 - 64; blk = xi / FX; it = FI + xi - blk * FX; }
    else if (it >= FI1) it += 64 - FI1;
}
__device__ __forceinline__ void push_role(const Params& p, f32x4 (&v)[8], char* wsl, int sw, int lane, u32 call) {
    const int rank = rfl(p.rank);
    const int g = rank >> 2, j = rank & 3, og = (g ^ 1) * 4;
    const u32 par = OFF_PAR + (call & 1) * PAR_BYTES;
    for (int n = sw; n < NPUSH; n += NSPARE) {
        int blk, it;
        push_decode(n, blk, it);
        u32x4 h[4];
        int d0 = 0, nd = 4, dstep = 1; u32 base, step = 1024, ostep = 0;
        if (it < 64) {
#pragma unroll
            for (int i = 0; i < 4; ++i) h[i] = cvt8(v[2 * i], v[2 * i + 1]);
            if (it < FI1) { d0 = j + 3; nd = 3; dstep = 3; ostep = FI1 * 4096; base = par + (blk * RS1 + 64 + it) * 4096; }
            else base = par + (blk * RS1 + 64 + 3 * FI1 + j * FX + it - FI) * 4096;
        } else {
#pragma unroll
            for (int c = 0; c < 4; ++c) h[c] = cvt_w(v, c);
            d0 = (it - 64) >> 4; nd = 1; step = 4096;
            base = par + REL_B + (blk * CF + 64 + 16 * j + (it & 12)) * 4096 + (it & 3) * 1024;
        }
        if (n + NSPARE < NPUSH) { int b2, i2; push_decode(n + NSPARE, b2, i2); load_item(v, p, b2 * 16 + (i2 >> 3), i2 & 7, lane); }
        u32 dmask = 0;
        for (int d = 0; d < nd; ++d) {
            const int dr = (it < FI1 ? g * 4 : og) + ((d0 + d * dstep) & 3);
            const rsrc_t rd = mk_rsrc(p.ws[dr], WS_NEED);
#pragma unroll
            for (int i = 0; i < 4; ++i) st_sys(h[i], rd, lane * 16, base + d * ostep + i * step);
            dmask |= 1u << dr;
        }
        WAIT_VM0();
        if (lane < 8 && ((dmask >> lane) & 1)) add_sys(wsl + OFF_CNT + blk * CNT_BLK + (it < 64 ? 0 : 1024) + rank * 128);
    }
    STAMP(4);
}

#define P_FLUSH 10
#define P_DSLEEP 1
constexpr int RING = 4, PAIR_BYTES = RING * 4096 + 2048 + 256;
constexpr u32 IDX_END = 0xffffffffu, IDX_BAD = 0x80000000u;
__device__ __forceinline__ u32 lds_get(const LAS char* a) {
    u32 v;
    asm volatile("ds_read_b32 %0, %1\n\ts_waitcnt lgkmcnt(0)" : "=v"(v) : "v"((u32)(size_t)a) : "memory");
    return rfl(v);
}
__device__ __forceinline__ void lds_put(LAS char* a, u32 v) {
    asm volatile("" ::: "memory");
    *(volatile LAS u32*)a = v;
    asm volatile("" ::: "memory");
}

template <bool CROSS>
__device__ __forceinline__ void load_role(const Params& p, LAS char* pr, int slot, int wv, int lane, u32 call) {
    const int rank = rfl(p.rank);
    const int g = rank >> 2, j = rank & 3;
    const int q = slot - 3, peer = CROSS ? (g ^ 1) * 4 + q : g * 4 + ((j + 1 + slot) & 3);
    constexpr int N1 = NBLK * FI1;
    constexpr int PER = 16 + XA, NP = NBLK * PER, NX = NBLK * (16 + FX);
    constexpr int W = CROSS ? WC : WI, NT = CROSS ? NX + 1024 : N1 + NBLK * F2;
    const u32 par = OFF_PAR + (call & 1) * PAR_BYTES;
    char* ws = uni(p.ws[rank]);
    const rsrc_t r0 = rsrc_u(ws, WS_NEED), rp = mk_rsrc(p.ws[peer], WS_NEED);
    const char* rdy = ws + OFF_RDY + peer * 8192;
    const u32 vo = lane * 16, vo36 = lane < 36 ? vo : 0x80000000u;
    constexpr int SKIP = PB * (CROSS ? PER : FI1);
    const u32 head = lane == 0 ? OFF_QUE + slot * 256 : WS_NEED, tbase = SKIP + 4 * W - (call - 1) * (NT - SKIP);
    const bool nowait = (p.pad & 4) != 0;
    LAS char* ctl = pr + RING * 4096 + 2048;
    const u32 ldsb = rfl((u32)(size_t)pr);
    struct Item { u32 idx, dst; };
    auto decode = [&](int it) {
        Item m;
        it = it < NT ? it : NT - 1;
        if constexpr (!CROSS) {
            const bool s2 = it >= N1;
            const int a = it - N1, b1 = it / FI1;
            const int blk = s2 ? a >> 4 : b1, s = s2 ? FI1 + (a & 15) : it - b1 * FI1;
            m.idx = blk * 128 + s;
            m.dst = par + (s2 ? REL_A2 + (slot * F2 - FI1) * 4096 : (64 + slot * FI1) * 4096) + (blk * (s2 ? RF2 : RS1) + s) * 4096;
        } else if (it < NX) {
            const int blk = it / PER, c = it - blk * PER;
            const bool tail = it >= NP, isx = tail || (c < 2 * XA && (c & 1));
            const int cw = c < 2 * XA ? c >> 1 : c - XA;
            const int xi = tail ? it - NP + NBLK * XA : blk * XA + (c >> 1), xb = xi / FX, xs = FI + xi - xb * FX;
            m.idx = isx ? xb * 128 + xs : blk * 128 + 64 + 16 * j + cw;
            m.dst = par + (isx ? (xb * RS1 + xs + 64 + 3 * FI1 + q * FX - FI) * 4096
                               : REL_B + (blk * CF + 64 + 16 * q + (cw & 12)) * 4096 + (cw & 3) * 1024);
        } else {
            const int o = it - NX;
            m.idx = 8192 + o;
            int row, col;
            if (o < 832) {
                row = row_of_rf1((o >> 6) * 16 + ((o >> 3) & 3) * 4 + (o & 3), g ^ 1, q);
                col = ((o >> 5) & 1) * 128 + ((o >> 2) & 1) * 64;
            } else {
                const int z = o - 832, rt = z >> 5;
                row = ((g ^ 1) * 4 + ((q + 1 + (rt >> 1)) & 3)) * NS + (FI1 + (rt & 1) * 8 + (z & 7)) * 16;
                col = ((z >> 3) & 3) * 64;
            }
            m.dst = (u32)row * 4096 + (q * 256 + col) * 4;
        }
        return m;
    };
    auto flag_of = [&](u32 idx) { return idx < 8192 ? idx >> 3 : (idx < 8192 + 832 ? 1024 + ((idx - 8192) >> 2) : idx - (8192 + 832 - 1024 - 208)); };
    u32 okupto = 0, wbase = 0; u64 wmask = 0;
    auto ready = [&](u32 fi) {
        if (fi < okupto) return true;
        if (fi - wbase < 64u && ((wmask >> (fi - wbase)) & 1)) return true;
        for (;;) {
            wbase = okupto;
            wmask = __builtin_amdgcn_ballot_w64(ld_sys(rdy + (wbase + lane) * 4) >= call);
            if (wmask != ~0ull) break;
            okupto += 64;
            if (fi < okupto) return true;
        }
        okupto += __builtin_ctzll(~wmask);
        if (fi - wbase < 64u) return ((wmask >> (fi - wbase)) & 1) != 0;
        return rfl(ld_sys(rdy + fi * 4)) >= call;
    };
    auto issue = [&](const Item& m, u32 n, bool bad) {
        const u32 src = par + REL_PK + (m.idx + (CROSS && m.idx >= 8192 ? j * 1024 : 0)) * 4096;
        const u32 m0v = ldsb + (n % RING) * 4096;
        asm volatile("s_mov_b32 m0, %0\n\ts_nop 0\n\t"
                     "buffer_load_dwordx4 %1, %3, %4 offen sc0 sc1 lds\n\t"
                     "buffer_load_dwordx4 %1, %3, %4 offen offset:1024 sc0 sc1 lds\n\t"
                     "buffer_load_dwordx4 %1, %3, %4 offen offset:2048 sc0 sc1 lds\n\t"
                     "buffer_load_dwordx4 %2, %3, %4 offen offset:3072 sc0 sc1 lds"
                     :: "s"(m0v), "v"(vo), "v"(vo36), "s"(rp), "s"(src) : "memory");
        *(volatile LAS u32x2v*)(ctl + 16 + (n % RING) * 8) = (u32x2v){m.idx | (bad ? IDX_BAD : 0), m.dst};
    };
    int e = SKIP + wv, e1 = e + W, e2 = e + 2 * W;
    Item nx = decode(e1);
    bool bad;
    {
        const Item cu = decode(e);
        bad = spin_ge<2>(rdy + flag_of(cu.idx) * 4, call, ws + OFF_TMO, call);
        STAMP(5);
        issue(cu, 0, bad);
    }
    bool pre = ready(flag_of(nx.idx));
    u32 n = 0;
    int e3 = e + 3 * W;
    auto body = [&](u32& tnew, u32& told) {
        tnew = 1;
        asm volatile("s_nop 0\n\tbuffer_atomic_add %0, %1, %2, 0 offen sc0" : "+v"(tnew) : "v"(head), "s"(r0) : "memory");
        const Item nn = decode(e2);
        bool pre2 = e2 < NT;
        if (pre2 && !(nowait && nn.idx >= 8192)) pre2 = ready(flag_of(nn.idx));
        TICK(t1);
        while (lds_get(ctl + 4) + RING < n + 2) __builtin_amdgcn_s_sleep(1);
        TICK(t2);
        asm volatile("s_waitcnt vmcnt(1)" : "+v"(told) :: "memory");
        TICK(t3);
        if (pre) issue(nx, n + 1, bad);
        lds_put(ctl, n + 1);
        if (!pre && e1 < NT) {
            bad |= spin_ge<2>(rdy + flag_of(nx.idx) * 4, call, ws + OFF_TMO, call);
            issue(nx, n + 1, bad);
        }
        STAMP_IF(n == 0, 6);
        STAMP_IF(CROSS ? (e < NP && e1 >= NP) : (e < N1 && e1 >= N1), 1);
        STAMP_IF(CROSS && e < NX && e1 >= NX, 2);
        STAMP_IF(CROSS && e < NX + 832 && e1 >= NX + 832, 3);
        nx = nn; pre = pre2; ++n;
        e = e1; e1 = e2; e2 = n == 1 ? e3 : (int)(rfl(told) + tbase);
    };
    u32 ta = 0, tb = 0;
    for (;;) {
        body(ta, tb);
        if (e >= NT) break;
        body(tb, ta);
        if (e >= NT) break;
    }
    asm volatile("s_waitcnt vmcnt(0)" : "+v"(ta), "+v"(tb) :: "memory");
    *(volatile LAS u32x2v*)(ctl + 16 + (n % RING) * 8) = (u32x2v){IDX_END, 0u};
    lds_put(ctl, n + 1);
}

template <bool CROSS>
__device__ __forceinline__ void decode_role(const Params& p, LAS char* pr, int slot, int lane, u32 call) {
    const int rank = rfl(p.rank);
    const int g = rank >> 2, j = rank & 3;
    const int peer = CROSS ? (g ^ 1) * 4 + slot - 3 : g * 4 + ((j + 1 + slot) & 3);
    const u32 par = OFF_PAR + (call & 1) * PAR_BYTES;
    const rsrc_t r0 = mk_rsrc(p.ws[rank], WS_NEED), ro = mk_rsrc(p.out[rank], OUT_BYTES), rp = mk_rsrc(p.ws[peer], WS_NEED);
    const u32 vo = lane * 16, vout = (lane >> 4) * 4 * 4096 + (lane & 15) * 16;
    LAS char* scr = pr + RING * 4096;
    LAS char* ctl = scr + 2048;
    const u32 vsig = lane == 0 ? 0u : 0x80000000u;
    u32 one = 1;
    asm volatile("" : "+v"(one));
    auto signal = [&](u32 c) {
        asm volatile("s_nop 4\n\tbuffer_atomic_add %0, %1, %2, %3 offen sc1" :: "v"(one), "v"(vsig), "s"(r0), "s"(rfl(c)) : "memory");
    };
    const u32x4 nan4 = {0x7fc07fc0u, 0x7fc07fc0u, 0x7fc07fc0u, 0x7fc07fc0u};
    u32 owed = WS_NEED, owed2 = WS_NEED;
    for (u32 n = 0;; ++n) {
        TICK(t0);
        u32 full = lds_get(ctl);
        for (int polls = 0; full <= n && polls < P_FLUSH; ++polls) { __builtin_amdgcn_s_sleep(P_DSLEEP); full = lds_get(ctl); }
        if (full <= n) {
            asm volatile("s_waitcnt vmcnt(0)" ::: "memory");
            signal(owed2); signal(owed); owed = owed2 = WS_NEED;
            while (lds_get(ctl) <= n) __builtin_amdgcn_s_sleep(P_DSLEEP);
        }
        TICK(t1);
        const LAS char* sl = pr + (n % RING) * 4096;
        const u32x2v ds = *(const volatile LAS u32x2v*)(ctl + 16 + (n % RING) * 8);
        const u32x4 c0 = *(const LAS u32x4*)(sl + vo), c1 = *(const LAS u32x4*)(sl + 1024 + vo), c2 = *(const LAS u32x4*)(sl + 2048 + vo);
        const u32x2v pp = *(const LAS u32x2v*)(sl + 3072 + lane * 8);
        const u32 ent = *(const LAS u32*)(sl + 3584 + (lane & 15) * 4);
        const u32 fidx = rfl(ds[0]), dst = rfl(ds[1]), hd = rfl(ent);
        if (fidx == IDX_END) break;
        const bool bad = (fidx & IDX_BAD) != 0;
        const u32 idx = fidx & ~IDX_BAD;
        const bool out = CROSS && idx >= 8192, isw = CROSS && !out && (idx & 64);
        const u32 step = (out || isw) ? 4096 : 1024;
        u32x4 h[4];
        if (!bad && !out && (hd & 255) != 255) unpack_item(h, c0, c1, c2, pp[0], pp[1], (const LAS unsigned char*)(sl + 3584), hd & 255, scr, lane);
        else if (!bad && out && hd < 256) unpack_out(h, c0, c1, c2, pp[0], pp[1], lane < 16 ? ent : hd, hd, scr, lane);
        else {
            const u32 blk = idx >> 7, s = idx & 127;
            const u32 raw = par + (out ? REL_PK + STG_BYTES + (idx + j * 1024) * 4096
                                       : isw ? REL_B + (blk * CF + (s & 60)) * 4096 + (s & 3) * 1024 : (blk * RS1 + s) * 4096);
            const u32 rs = isw ? 4096 : 1024;
            asm volatile("s_nop 4\n\tbuffer_load_dwordx4 %0, %4, %5, %6 offen sc0 sc1\n\tbuffer_load_dwordx4 %1, %4, %5, %7 offen sc0 sc1\n\t"
                         "buffer_load_dwordx4 %2, %4, %5, %8 offen sc0 sc1\n\tbuffer_load_dwordx4 %3, %4, %5, %9 offen sc0 sc1\n\ts_waitcnt vmcnt(0)"
                         : "=&v"(h[0]), "=&v"(h[1]), "=&v"(h[2]), "=&v"(h[3])
                         : "v"(vo), "s"(rp), "s"(rfl(raw)), "s"(rfl(raw + rs)), "s"(rfl(raw + 2 * rs)), "s"(rfl(raw + 3 * rs)) : "memory");
            if (bad) { h[0] = nan4; h[1] = nan4; h[2] = nan4; h[3] = nan4; }
        }
        lds_put(ctl + 4, n + 1);
        const rsrc_t rd = out ? ro : r0;
        const u32 vd = (out ? vout : vo) + dst;
#pragma unroll
        for (int i = 0; i < 4; ++i) st_sys(h[i], rd, vd, i * step);
        TICK(t2);
        asm volatile("s_waitcnt vmcnt(9)" ::: "memory");
        TICK(t3);
        signal(owed2);
        owed2 = owed;
        owed = rfl(out ? WS_NEED : OFF_CNT + (idx >> 7) * CNT_BLK + peer * 128 + (CROSS ? (isw ? 1024 : 0) : ((idx & 127) >= FI1 ? 2048 : 0)));
    }
    WAIT_VM0();
    signal(owed2); signal(owed);
    STAMP(4);
}

#define LDPOL " sc0 sc1"
#define GLDS2(m0v, voff, rs, soff, imm0, imm1) \
    asm volatile("s_mov_b32 m0, %0\n\ts_nop 0\n\tbuffer_load_dwordx4 %1, %2, %3 offen offset:" #imm0 LDPOL " lds\n\t" \
                 "buffer_load_dwordx4 %1, %2, %3 offen offset:" #imm1 LDPOL " lds" \
                 :: "s"(m0v), "v"(voff), "s"(rs), "s"(soff) : "memory")
#define BARRIER() do { __builtin_amdgcn_sched_barrier(0); __builtin_amdgcn_s_barrier(); __builtin_amdgcn_sched_barrier(0); } while (0)

#define SCHED() __builtin_amdgcn_sched_barrier(0)
#define WAIT_LGKM0() __builtin_amdgcn_s_waitcnt(0xC07F)

template <int STAGE>
struct Gemm {
    static constexpr int NT = STAGE == 1 ? 128 : 64;
    static constexpr int NA = STAGE == 1 ? 4 : 2;
    static constexpr int NB = STAGE == 1 ? 4 : 8;
    static constexpr int NM = STAGE == 1 ? 4 : 1;
    static constexpr int NMM = STAGE == 1 ? 16 : 8;
    static constexpr int NRD = NA + NB;
    struct Frags { f16x8 a[NA], b[NB]; };

    rsrc_t rs;
    u32 sa, sa2, sb;
    u32 ldsw;
    int wid, lane;

    template <int I> __device__ __forceinline__ void piece(int t, int buf) const {
        const u32 vo = lane * 16;
        const u32 ma = ldsw + buf * BUF_BYTES + wid * 4096, mb = ldsw + buf * BUF_BYTES + 32768 + wid * 2048;
        const u32 ka = STAGE == 1 ? (u32)(t >> 1) * (RS1 * 4096) + (t & 1) * 2048 : (u32)t * (RF2 * 4096);
        const u32 a = sa + ka, a2 = sa2 + ka;
        const u32 b = STAGE == 1 ? sb + (u32)(t >> 1) * (CF * 4096) + (t & 1) * 2048 : sb + (u32)t * (CF * 4096);
        if (I == 0) GLDS2(ma, vo, rs, a, 0, 1024);
        if (I == 1) GLDS2(ma, vo, rs, a2, 2048, 3072);
        if (I == 2) GLDS2(mb, vo, rs, b, 0, 1024);
    }
    __device__ __forceinline__ void stage(int t, int buf) const { piece<0>(t, buf); piece<1>(t, buf); piece<2>(t, buf); }

    template <int I> __device__ __forceinline__ void rd(Frags& f, const LAS char* lds, int buf, int half) const {
        const LAS char* base = lds + buf * BUF_BYTES + lane * 16;
        if constexpr (STAGE == 1) {
            const LAS char* pa = base + (wid >> 1) * 8192 + half * 1024;
            const LAS char* pb = base + 32768 + (wid & 1) * 8192 + half * 1024;
            if constexpr (I == 0) f.a[0] = *(const LAS f16x8*)pa;
            else if constexpr (I <= 4) f.b[I - 1] = *(const LAS f16x8*)(pb + (I - 1) * 2048);
            else f.a[I - 4] = *(const LAS f16x8*)(pa + (I - 4) * 2048);
        } else {
            const LAS char* pa = base + wid * 4096 + half * 2048;
            const LAS char* pb = base + 32768 + half * 2048;
            constexpr int h = I / 5, n = I % 5;
            if constexpr (n == 0) f.a[h] = *(const LAS f16x8*)(pa + h * 1024);
            else f.b[h * 4 + n - 1] = *(const LAS f16x8*)(pb + (n - 1) * 4096 + h * 1024);
        }
    }
    template <int I> __device__ __forceinline__ void mm(f32x4 (&acc)[NM][4], const Frags& f) const {
        if constexpr (STAGE == 1) acc[I >> 2][I & 3] = __builtin_amdgcn_mfma_f32_16x16x32_f16(f.a[I >> 2], f.b[I & 3], acc[I >> 2][I & 3], 0, 0, 0);
        else            acc[0][I & 3] = __builtin_amdgcn_mfma_f32_16x16x32_f16(f.a[I >> 2], f.b[I], acc[0][I & 3], 0, 0, 0);
    }
    template <int I, int N> __device__ __forceinline__ void rds(Frags& f, const LAS char* lds, int buf, int half) const {
        if constexpr (I < N) { rd<I>(f, lds, buf, half); rds<I + 1, N>(f, lds, buf, half); }
    }
    static __device__ __forceinline__ int blk_of(int t) { return STAGE == 1 ? (t >> 1) : t; }

    template <bool RD, int ST>
    __device__ __forceinline__ void half_step(f32x4 (&acc)[NM][4], const Frags& cur, Frags& nxt, const LAS char* lds, int rbuf, int rhalf,
                                              int ts, int bs, u32& fl, u32 cvo, u32 fso) const {
        auto first = [&] {
            if (ST == 1) asm volatile("buffer_load_dword %0, %1, %2, %3 offen sc0 sc1" : "=v"(fl) : "v"(cvo), "s"(rs), "s"(fso) : "memory");
            if (ST) { piece<0>(ts, bs); SCHED(); }
        };
        if constexpr (STAGE == 1) {
            mm<0>(acc, cur);  mm<1>(acc, cur);  if (RD) rds<0, 2>(nxt, lds, rbuf, rhalf); SCHED();
            mm<2>(acc, cur);  mm<3>(acc, cur);  if (RD) rds<2, 4>(nxt, lds, rbuf, rhalf); SCHED();
            mm<4>(acc, cur);  mm<5>(acc, cur);  if (RD) rds<4, 6>(nxt, lds, rbuf, rhalf); SCHED();
            mm<6>(acc, cur);  mm<7>(acc, cur);  if (RD) rds<6, 8>(nxt, lds, rbuf, rhalf); SCHED();
            mm<8>(acc, cur);  mm<9>(acc, cur);  SCHED();
            first();
            mm<10>(acc, cur); mm<11>(acc, cur); SCHED();
            if (ST) { piece<1>(ts, bs); SCHED(); }
            mm<12>(acc, cur); mm<13>(acc, cur); SCHED();
            if (ST) { piece<2>(ts, bs); SCHED(); }
            mm<14>(acc, cur); mm<15>(acc, cur); SCHED();
        } else {
            mm<0>(acc, cur); if (RD) rds<0, 3>(nxt, lds, rbuf, rhalf); SCHED();
            mm<1>(acc, cur); if (RD) rds<3, 5>(nxt, lds, rbuf, rhalf); SCHED();
            mm<2>(acc, cur); if (RD) rds<5, 8>(nxt, lds, rbuf, rhalf); SCHED();
            mm<3>(acc, cur); if (RD) rds<8, 10>(nxt, lds, rbuf, rhalf); SCHED();
            mm<4>(acc, cur); SCHED();
            first();
            mm<5>(acc, cur); SCHED();
            if (ST) { piece<1>(ts, bs); SCHED(); }
            mm<6>(acc, cur); SCHED();
            if (ST) { piece<2>(ts, bs); SCHED(); }
            mm<7>(acc, cur); SCHED();
        }
    }

    template <int ROLE>
    __device__ __forceinline__ u32 run(f32x4 (&acc)[NM][4], const LAS char* lds, char* ws, u32 cvo, u32 need, u32 call) const {
        constexpr bool LATE = ROLE == 2;
        char* tmo = ws + OFF_TMO;
        Frags f0, f1;
        if (ROLE == 0) spin_ge(ws + cvo + blk_of(0) * CNT_BLK, need, tmo, call);
        BARRIER();
        stage(0, 0);
        if (ROLE == 0) { spin_ge(ws + cvo + blk_of(1) * CNT_BLK, need, tmo, call); spin_ge(ws + cvo + blk_of(2) * CNT_BLK, need, tmo, call); }
        BARRIER();
        stage(1, 1);
        if (!LATE) { stage(2, 2); asm volatile("s_waitcnt vmcnt(12)" ::: "memory"); }
        else asm volatile("s_waitcnt vmcnt(6)" ::: "memory");
        BARRIER();
        rds<0, NRD>(f0, lds, 0, 0);
        WAIT_LGKM0();
        u32 fl = 0;
        int buf = 0;
        for (int t = 0; t < NT - 3; ++t) {
            const int nb = buf == 2 ? 0 : buf + 1, pb = buf == 0 ? 2 : buf - 1;
            half_step<true, LATE ? 2 : 0>(acc, f0, f1, lds, buf, 1, t + 2, pb, fl, 0, 0);
            if (ROLE == 0) {
                asm volatile("s_waitcnt vmcnt(6)" : "+v"(fl) :: "memory");
                WAIT_LGKM0();
                if (__builtin_amdgcn_ballot_w64(fl < need) != 0) spin_ge(ws + cvo + blk_of(t + 3) * CNT_BLK, need, tmo, call);
            } else {
                asm volatile("s_waitcnt vmcnt(6)" ::: "memory");
                WAIT_LGKM0();
            }
            BARRIER();
            half_step<true, LATE ? 0 : (ROLE == 0 ? 1 : 2)>(acc, f1, f0, lds, nb, 0, t + 3, buf, fl, cvo, blk_of(t + 4 < NT ? t + 4 : t + 3) * CNT_BLK);
            buf = nb;
        }
        u32 tw = 0;
#pragma unroll
        for (int t = NT - 3; t < NT; ++t) {
            const int nb = buf == 2 ? 0 : buf + 1, pb = buf == 0 ? 2 : buf - 1;
            if (t == NT - 3) {
                half_step<true, LATE ? 2 : 0>(acc, f0, f1, lds, buf, 1, t + 2, pb, fl, 0, 0);
                asm volatile("s_waitcnt vmcnt(6)" : "+v"(fl) :: "memory");
            } else {
                if (t == NT - 2) {
                    const u32 to = OFF_TMO;
                    asm volatile("buffer_load_dword %0, %1, %2, 0 offen sc1" : "=v"(tw) : "v"(to), "s"(rs) : "memory");
                }
                half_step<true, 0>(acc, f0, f1, lds, buf, 1, 0, 0, fl, 0, 0);
                asm volatile("s_waitcnt vmcnt(0)" : "+v"(tw) :: "memory");
            }
            WAIT_LGKM0();
            BARRIER();
            if (t < NT - 1) half_step<true, 0>(acc, f1, f0, lds, nb, 0, 0, 0, fl, 0, 0);
            else            half_step<false, 0>(acc, f1, f0, lds, nb, 0, 0, 0, fl, 0, 0);
            buf = nb;
        }
        return rfl(tw);
    }
};

__device__ __forceinline__ u32x4 gelu4(const f32x4 (&c)[4], int r, bool bad) {
    const float nanv = __builtin_nanf("");
    f32x4 o = {gelu(c[0][r]), gelu(c[1][r]), gelu(c[2][r]), gelu(c[3][r])};
    if (bad) o = (f32x4){nanv, nanv, nanv, nanv};
    return __builtin_bit_cast(u32x4, o);
}

__device__ __forceinline__ void gemm_role(const Params& p, const LAS char* lds, int xcd, int s, int wid, int lane, u32 call) {
    const int rank = rfl(p.rank);
    const int g = rank >> 2, j = rank & 3, og = (g ^ 1) * 4;
    char* ws = p.ws[rank];
    const u32 par = OFF_PAR + (call & 1) * PAR_BYTES;
    const rsrc_t rs = mk_rsrc(ws, WS_NEED), ro = mk_rsrc(p.out[rank], OUT_BYTES);
    const u32 ldsw = rfl((u32)(size_t)lds);
    const u32 vout = (lane >> 4) * 4 * 4096 + (lane & 15) * 16;
    {
        const int u = (xcd & 1) * 26 + s, rt = u >> 2, ct = (xcd >> 1) * 4 + (u & 3);
        Gemm<1> G;
        G.rs = rs; G.ldsw = ldsw; G.wid = wid; G.lane = lane;
        const int rf = rt * 16 + wid * 2;
        G.sa = par + (rf < FI ? rf : rf + FX) * 4096;
        G.sa2 = par + (rf + 1 < FI ? rf + 1 : rf + 1 + FX) * 4096 - 2048;
        G.sb = par + REL_B + (ct * 8 + wid) * 4096;
        const int bsrc = ct < 8 ? rank : og + ((ct - 8) >> 1);
        u32 cvo = OFF_CNT + 1024 + bsrc * 128, need = (ct < 8 ? 8u : 16u) * call;
        u32 mask = 0;
        for (int i = 0; i < 16; ++i) mask |= 1u << src_of_rf1(rt * 16 + i, g, j);
        if (lane < 8 && ((mask >> lane) & 1)) {
            cvo = OFF_CNT + lane * 128;
            need = (lane == rank ? 8 : ((lane >> 2) == g ? FI1 : FX)) * call;
        }
        f32x4 acc[4][4] = {};
        const bool bad = call == (wid == 0 ? G.template run<0>(acc, lds, ws, cvo, need, call)
                                  : wid < 4 ? G.template run<1>(acc, lds, ws, cvo, need, call)
                                            : G.template run<2>(acc, lds, ws, cvo, need, call));
        STAMP(1);
        const int wm = wid >> 1, wn = wid & 1;
        if (ct < 8) {
#pragma unroll
            for (int m = 0; m < 4; ++m) {
                const u32 ob = (u32)row_of_rf1(rt * 16 + wm * 4 + m, g, j) * 4096 + (ct * 128 + wn * 64) * 4;
#pragma unroll
                for (int r = 0; r < 4; ++r) __builtin_amdgcn_raw_buffer_store_b128(gelu4(acc[m], r, bad), ro, vout + ob + r * 4096, 0, OUT_AUX);
            }
        } else {
            const int fo = (rt * 2 + (ct & 1)) * 8 + wid;
            const u32 sb = par + REL_STG + (u32)(((ct - 8) >> 1) * 1024 + fo * 4) * 4096;
#pragma unroll
            for (int m = 0; m < 4; ++m) {
                const u32x4 o[4] = {gelu4(acc[m], 0, bad), gelu4(acc[m], 1, bad), gelu4(acc[m], 2, bad), gelu4(acc[m], 3, bad)};
                pack_out(o, (LAS char*)lds + wid * 2048, rs, sb + m * 4096, lane);
            }
            WAIT_VM0();
            if (lane == 0) st_flag(p.ws[bsrc] + OFF_RDY + rank * 8192 + (1024 + fo) * 4, call);
        }
        STAMP(2);
    }
    if (s < 24) {
        const int rt = s >> 2, ct = ((~xcd & 1) * 16) + (xcd >> 1) * 4 + (s & 3);
        Gemm<2> G;
        G.rs = rs; G.ldsw = ldsw; G.wid = wid; G.lane = lane;
        G.sa = G.sa2 = par + REL_A2 + (rt * 8 + wid) * 4096;
        G.sb = par + REL_B + ct * 4 * 4096 + wid * 2048;
        const int asrc = g * 4 + ((j + 1 + (rt >> 1)) & 3);
        const int bsrc = ct < 16 ? rank : og + ((ct - 16) >> 2);
        u32 cvo = OFF_CNT + 1024 + bsrc * 128, need = (ct < 16 ? 8u : 16u) * call;
        if (lane == 0) { cvo = OFF_CNT + 2048 + asrc * 128; need = F2 * call; }
        f32x4 acc[1][4] = {};
        const bool bad = call == (wid == 0 ? G.template run<0>(acc, lds, ws, cvo, need, call)
                                  : wid < 4 ? G.template run<1>(acc, lds, ws, cvo, need, call)
                                            : G.template run<2>(acc, lds, ws, cvo, need, call));
        STAMP(3);
        if (ct < 16) {
            const u32 ob = (u32)(asrc * NS + (FI1 + (rt & 1) * 8 + wid) * 16) * 4096 + ct * 256;
#pragma unroll
            for (int r = 0; r < 4; ++r) __builtin_amdgcn_raw_buffer_store_b128(gelu4(acc[0], r, bad), ro, vout + ob + r * 4096, 0, OUT_AUX);
        } else {
            const int z = (rt * 4 + (ct & 3)) * 8 + wid;
            const u32 sb = par + REL_STG + (u32)(((ct - 16) >> 2) * 1024 + 832 + z) * 4096;
            const u32x4 o[4] = {gelu4(acc[0], 0, bad), gelu4(acc[0], 1, bad), gelu4(acc[0], 2, bad), gelu4(acc[0], 3, bad)};
            pack_out(o, (LAS char*)lds + wid * 2048, rs, sb, lane);
            WAIT_VM0();
            if (lane == 0) st_flag(p.ws[bsrc] + OFF_RDY + rank * 8192 + (1024 + 208 + z) * 4, call);
        }
        STAMP(4);
    }
}

__global__ __launch_bounds__(BLOCK, 2) void ag_gemm_kernel(Params p) {
    __shared__ __attribute__((aligned(1024))) char smem[LDS_BYTES];
    const int rank = rfl(p.rank);
    const int lane = threadIdx.x & 63;
    const int wid = rfl(threadIdx.x >> 6);
    const int b = blockIdx.x, xcd = b & 7, slot = b >> 3;
    const int gw = (wid & 3) * NCOMM + b;
    const int sw = (gw - NPULL) * 2 + (wid >> 2);
    if (b < NCOMM && gw >= NPULL && sw >= NPUSH) return;
    STAMP(0);
    char* seq = p.ws[rank] + OFF_SEQ + (b * 8 + wid) * 4;
    const u32 prev = ld_agent(seq);
    char* wsl = p.ws[lane & 7];
    LAS char* pr = (LAS char*)smem + (wid & 3) * PAIR_BYTES;
    if (b < NCOMM) {
        if (wid < 4 && lane < 2) *(LAS u32*)(pr + RING * 4096 + 2048 + lane * 4) = 0;
        asm volatile("s_waitcnt lgkmcnt(0)\n\ts_barrier" ::: "memory");
    }
    f32x4 v0[8];
    if (b >= NCOMM) {
        constexpr int B0 = PB > 0 ? PB - 1 : 0;
        if ((u32)(b - NCOMM - B0 * 16) >= EARLY * 16) __builtin_amdgcn_s_sleep(LATE_SLEEP);
        load_item(v0, p, b - NCOMM, wid, lane);
    } else if (gw >= NPULL) {
        int blk, it;
        push_decode(sw, blk, it);
        load_item(v0, p, blk * 16 + (it >> 3), it & 7, lane);
    }
    const u32 call = rfl(prev) + 1;
    if (lane == 0) __hip_atomic_store((GAS u32*)seq, call, __ATOMIC_RELAXED, __HIP_MEMORY_SCOPE_AGENT);
    STAMP(7);
    if (b < NCOMM) {
        if (gw >= NPULL) { push_role(p, v0, wsl, sw, lane, call); return; }
        constexpr int WM = WI < WC ? WI : WC;
        int ps, wv;
        if (gw < 7 * WM) { ps = gw % 7; wv = gw / 7; }
        else if (WI > WC) { ps = (gw - 7 * WM) % 3; wv = WM + (gw - 7 * WM) / 3; }
        else { ps = 3 + (gw - 7 * WM) % 4; wv = WM + (gw - 7 * WM) / 4; }
        if (wid < 4) {
            if (ps < 3) load_role<false>(p, pr, rfl(ps), rfl(wv), lane, call);
            else        load_role<true>(p, pr, rfl(ps), rfl(wv), lane, call);
        } else {
            if (ps < 3) decode_role<false>(p, pr, rfl(ps), lane, call);
            else        decode_role<true>(p, pr, rfl(ps), lane, call);
        }
    } else {
        convert_local(p, (LAS char*)smem + wid * 4096, v0, wsl, b - NCOMM, wid, lane, call);
        STAMP(5);
        gemm_role(p, (const LAS char*)smem, xcd, slot - NCOMM / 8, wid, lane, call);
    }
}

}

constexpr int g_mode = 0;

extern "C" void kernel_launch(int rank, int nranks, ncclComm_t comm, hipStream_t stream,
                              void* const* d_in, const int* in_sizes, int n_in,
                              void* d_out, int out_size, const int* in_dtypes, int out_dtype,
                              void* const* d_out_all, void* const* d_ws_all, size_t ws_bytes) {
    (void)comm; (void)in_sizes; (void)n_in; (void)d_out; (void)out_size; (void)in_dtypes; (void)out_dtype;
    if (nranks != NR || (unsigned)rank >= (unsigned)NR || ws_bytes < WS_NEED) {
        fprintf(stderr, "unsupported configuration\n");
        return;
    }
    Params p;
    p.x = (const float*)d_in[0];
    p.w = (const float*)d_in[1];
    for (int i = 0; i < NR; ++i) { p.ws[i] = (char*)d_ws_all[i]; p.out[i] = (char*)d_out_all[i]; }
    p.rank = rank;
    p.pad = g_mode;
    ag_gemm_kernel<<<GRID, BLOCK, 0, stream>>>(p);
    hipError_t e = hipGetLastError();
    if (e != hipSuccess) fprintf(stderr, "launch failed: %s\n", hipGetErrorString(e));
}
```

```cpp
#include <hip/hip_runtime.h>
#include <rccl/rccl.h>
#include <cstdio>
#include <cstdint>
#include <type_traits>

typedef unsigned u32;
typedef unsigned long long u64;
typedef u32 u32x4 __attribute__((ext_vector_type(4)));
typedef u32 u32x2v __attribute__((ext_vector_type(2)));
typedef float f32x4 __attribute__((ext_vector_type(4)));
typedef _Float16 f16x8 __attribute__((ext_vector_type(8)));
typedef __amdgpu_buffer_rsrc_t rsrc_t;
#define LAS __attribute__((address_space(3)))
#define GAS __attribute__((address_space(1)))

namespace {

constexpr int NR = 8;
constexpr int KD = 8192;
constexpr int NS = 1024;
constexpr int NBLK = 64;
#define P_FI 46
constexpr int FI = P_FI;
constexpr int F2 = 16;
constexpr int FI1 = FI - F2;
constexpr int FX = 64 - FI;
constexpr int RF1 = 208, RF2 = 48, CF = 128;
static_assert(FI + 3 * FI1 + 4 * FX == RF1, "stage-1 rows");

#define P_W 19
#define P_XA 12
#define P_EARLY 4
#define P_SLEEP 100
#define P_PF 1
#define P_PB 1
constexpr int PB = P_PB;
constexpr int EARLY = P_EARLY, LATE_SLEEP = P_SLEEP, PF_FROM = P_PF;
constexpr int NCOMM = 48;
constexpr int WL = P_W;
constexpr int XA = P_XA;
static_assert(WL >= 14 && WL <= 20 && XA <= FX && XA <= 16, "wave roles");
constexpr int GRID = 256, BLOCK = 768, NGEMM = GRID - NCOMM;
constexpr int RS1 = 64 + 3 * FI1 + 4 * FX;

constexpr u32 OFF_SEQ  = 0;
constexpr u32 OFF_TMO  = 16384;
constexpr u32 OFF_QUE  = 16640;
constexpr u32 OFF_RDY  = 32768;
constexpr u32 OFF_CNT  = 131072;
constexpr u32 CNT_BLK  = 24 * 128;
constexpr u32 OFF_DIAG = 524288;
constexpr u32 A1_BYTES = (u32)NBLK * RS1 * 4096;
constexpr u32 A2_BYTES = (u32)NBLK * RF2 * 4096;
constexpr u32 B_BYTES  = (u32)NBLK * CF * 4096;
constexpr u32 STG_BYTES = 4u * 1024 * 4096;
constexpr u32 PK_BYTES = (u32)NBLK * 128 * 4096;
constexpr u32 REL_A2 = A1_BYTES, REL_B = REL_A2 + A2_BYTES, REL_PK = REL_B + B_BYTES, REL_STG = REL_PK + PK_BYTES;
constexpr u32 PAR_BYTES = REL_STG + 2 * STG_BYTES;
constexpr u32 OFF_PAR  = 1u << 20;
constexpr u32 WS_NEED  = OFF_PAR + 2 * PAR_BYTES;
constexpr u32 OUT_BYTES = (u32)KD * NS * 4;
static_assert(REL_PK < (256u << 20), "table entries hold 18 bits of dst / 1024");
static_assert(PK_BYTES == 8192u * 4096u, "staged outputs follow the packed image");
static_assert(OFF_CNT + NBLK * CNT_BLK <= OFF_DIAG && OFF_RDY + 8 * 8192 <= OFF_CNT, "control area");

constexpr u64 DEADLINE = 40000000ull;

constexpr int BUF_BYTES = 49152;
constexpr int LDS_BYTES = 3 * BUF_BYTES;

struct Params {
    const float* x;
    const float* w;
    char* ws[NR];
    char* out[NR];
    int rank;
    int pad;
};

__device__ __forceinline__ u32 rfl(u32 v) { return __builtin_amdgcn_readfirstlane(v); }
__device__ __forceinline__ char* uni(const void* p) {
    const u64 a = (u64)p;
    return (char*)(((u64)rfl((u32)(a >> 32)) << 32) | rfl((u32)a));
}
__device__ __forceinline__ rsrc_t rsrc_u(const void* p, u32 bytes) {
    return __builtin_amdgcn_make_buffer_rsrc((void*)p, (short)0, (int)bytes, 0x00020000);
}
__device__ __forceinline__ rsrc_t mk_rsrc(const void* p, u32 bytes) { return rsrc_u(uni(p), bytes); }
__device__ __forceinline__ u32 ld_sys(const char* p) {
    return __hip_atomic_load((const GAS u32*)p, __ATOMIC_RELAXED, __HIP_MEMORY_SCOPE_SYSTEM);
}
__device__ __forceinline__ u32 ld_agent(const char* p) {
    return __hip_atomic_load((const GAS u32*)p, __ATOMIC_RELAXED, __HIP_MEMORY_SCOPE_AGENT);
}
__device__ __forceinline__ void st_flag(char* p, u32 v) {
    __hip_atomic_store((GAS u32*)p, v, __ATOMIC_RELAXED, __HIP_MEMORY_SCOPE_SYSTEM);
}
__device__ __forceinline__ void add_sys(char* p) {
    __hip_atomic_fetch_add((GAS u32*)p, 1u, __ATOMIC_RELAXED, __HIP_MEMORY_SCOPE_SYSTEM);
}
#define WAIT_VM0() do { asm volatile("s_waitcnt vmcnt(0)" ::: "memory"); __builtin_amdgcn_s_waitcnt(0x0F70); } while (0)

template <int SLEEP = 2>
__device__ __forceinline__ bool spin_ge(const char* p, u32 need, char* tmo, u32 call) {
    const u64 t0 = __builtin_amdgcn_s_memrealtime();
    bool gave_up = false;
    for (u32 polls = 0;; ++polls) {
        const u32 v = ld_sys(p);
        if (__builtin_amdgcn_ballot_w64(v < need) == 0) break;
        if ((polls & 7) == 0 && (rfl(ld_agent(tmo)) == call || __builtin_amdgcn_s_memrealtime() - t0 > DEADLINE)) {
            __hip_atomic_store((GAS u32*)tmo, call, __ATOMIC_RELAXED, __HIP_MEMORY_SCOPE_AGENT);
            gave_up = true;
            break;
        }
        __builtin_amdgcn_s_sleep(SLEEP);
    }
    asm volatile("" ::: "memory");
    return gave_up;
}

__device__ __forceinline__ int dwave() { int w = threadIdx.x >> 6; return w < 8 ? w : w - 5; }
#define TICK(v) do {} while (0)
#define ACC(a, x) do {} while (0)
#define STAMP_IF(c, slot) do {} while (0)
#define STAMP(slot) do {} while (0)
#define STAMPB_IF(c, slot) do {} while (0)

__device__ __forceinline__ u32x4 cvt8(f32x4 a, f32x4 b) {
    f16x8 h;
    h[0] = (_Float16)a[0]; h[1] = (_Float16)a[1]; h[2] = (_Float16)a[2]; h[3] = (_Float16)a[3];
    h[4] = (_Float16)b[0]; h[5] = (_Float16)b[1]; h[6] = (_Float16)b[2]; h[7] = (_Float16)b[3];
    return __builtin_bit_cast(u32x4, h);
}
__device__ __forceinline__ void st_sys(u32x4 v, rsrc_t r, u32 voff, u32 soff) {
    __builtin_amdgcn_raw_buffer_store_b128(v, r, voff + soff, 0, 17);
}
__device__ __forceinline__ float gelu(float y) {
    const float u = y * (0.7978845608028654f + 0.7978845608028654f * 0.044715f * y * y);
    const float e = __builtin_amdgcn_exp2f(u * -2.8853900817779268f);
    return y * __builtin_amdgcn_rcpf(1.0f + e);
}

__device__ __forceinline__ int row_of_rf1(int rf, int g, int j) {
    if (rf < FI) return (g * 4 + j) * NS + rf * 16;
    rf -= FI;
    if (rf < 3 * FI1) { const int i = rf / FI1; return (g * 4 + ((j + 1 + i) & 3)) * NS + (rf - i * FI1) * 16; }
    rf -= 3 * FI1;
    const int q = rf / FX;
    return ((g ^ 1) * 4 + q) * NS + (FI + rf - q * FX) * 16;
}
__device__ __forceinline__ int src_of_rf1(int rf, int g, int j) { return row_of_rf1(rf, g, j) / NS; }

constexpr int IN_AUX = 2;
#define P_OUT_AUX 17
constexpr int OUT_AUX = P_OUT_AUX;
__device__ __forceinline__ u32x4 cvt_w(const f32x4* v, int c) {
    return cvt8((f32x4){v[0][c], v[1][c], v[2][c], v[3][c]}, (f32x4){v[4][c], v[5][c], v[6][c], v[7][c]});
}

#define DPP_STEP(op, v, ctrl, rmask) v = op(v, (u32)__builtin_amdgcn_update_dpp(0, (int)v, ctrl, rmask, 0xf, false))
__device__ __forceinline__ u32 umax2(u32 a, u32 b) { return a > b ? a : b; }
__device__ __forceinline__ u32 uadd2(u32 a, u32 b) { return a + b; }
__device__ __forceinline__ u32 wave_max(u32 v) {
    DPP_STEP(umax2, v, 0x111, 0xf); DPP_STEP(umax2, v, 0x112, 0xf); DPP_STEP(umax2, v, 0x114, 0xf); DPP_STEP(umax2, v, 0x118, 0xf);
    DPP_STEP(umax2, v, 0x142, 0xa); DPP_STEP(umax2, v, 0x143, 0xc);
    return __builtin_amdgcn_readlane(v, 63);
}
__device__ __forceinline__ u32 wave_sum(u32 v) {
    DPP_STEP(uadd2, v, 0x111, 0xf); DPP_STEP(uadd2, v, 0x112, 0xf); DPP_STEP(uadd2, v, 0x114, 0xf); DPP_STEP(uadd2, v, 0x118, 0xf);
    DPP_STEP(uadd2, v, 0x142, 0xa); DPP_STEP(uadd2, v, 0x143, 0xc);
    return __builtin_amdgcn_readlane(v, 63);
}

__device__ __forceinline__ u32 wave_scan(u32 v) {
    DPP_STEP(uadd2, v, 0x111, 0xf); DPP_STEP(uadd2, v, 0x112, 0xf); DPP_STEP(uadd2, v, 0x114, 0xf); DPP_STEP(uadd2, v, 0x118, 0xf);
    DPP_STEP(uadd2, v, 0x142, 0xa); DPP_STEP(uadd2, v, 0x143, 0xc);
    return v;
}
__device__ __forceinline__ u32 byte_scan(u32 m) {
    u32 a, r;
    asm("v_lshl_add_u32 %0, %1, 8, %1" : "=v"(a) : "v"(m));
    asm("v_lshl_add_u32 %0, %1, 16, %1" : "=v"(r) : "v"(a));
    return r;
}
__device__ __forceinline__ u32 times7(u32 m) {
    u32 a, r;
    asm("v_lshl_add_u32 %0, %1, 1, %1" : "=v"(a) : "v"(m));
    asm("v_lshl_add_u32 %0, %1, 2, %2" : "=v"(r) : "v"(m), "v"(a));
    return r;
}
__device__ __forceinline__ u32 rank_in(u64 m) { return __builtin_amdgcn_mbcnt_hi((u32)(m >> 32), __builtin_amdgcn_mbcnt_lo((u32)m, 0)); }

constexpr int VAR_X = 192, VAR_W = 256;
__device__ __forceinline__ u32 h7_of(float a) { return (u32)__builtin_bit_cast(unsigned short, (_Float16)a) >> 8; }
__device__ __forceinline__ void lds_or(LAS char* a, u32 v) {
    __hip_atomic_fetch_or((LAS u32*)a, v, __ATOMIC_RELAXED, __HIP_MEMORY_SCOPE_WORKGROUP);
}
__device__ __forceinline__ void put_bits(LAS char* area, u32 bo, u64 v) {
    const u32 sh = bo & 31;
    const u64 w = v << sh;
    LAS char* a = area + (bo >> 5) * 4;
    lds_or(a, (u32)w); lds_or(a + 4, (u32)(w >> 32)); lds_or(a + 8, (u32)((v >> 1) >> (63 - sh)));
}
__device__ __forceinline__ u64 get_bits(const LAS char* area, u32 bo) {
    const LAS u32* w = (const LAS u32*)(area + (bo >> 5) * 4);
    const u32 w0 = w[0], w1 = w[1], w2 = w[2];
    return (u64)__builtin_amdgcn_alignbit(w2, w1, bo) << 32 | __builtin_amdgcn_alignbit(w1, w0, bo);
}
template <class F>
__device__ __forceinline__ void pack_item(const u32x4 (&h)[4], float amax, u32 var, LAS char* scr, rsrc_t r0, u32 dst, int lane, F before_stores) {
    u32 L[8], H[8];
#pragma unroll
    for (int t = 0; t < 8; ++t) {
        const u32 d0 = h[t >> 1][(t & 1) * 2], d1 = h[t >> 1][(t & 1) * 2 + 1];
        L[t] = __builtin_amdgcn_perm(d1, d0, 0x06040200u);
        H[t] = __builtin_amdgcn_perm(d1, d0, 0x07050301u);
    }
    *(LAS u32x4*)(scr + lane * 16) = (u32x4){H[0], H[1], H[2], H[3]};
    *(LAS u32x4*)(scr + lane * 16 + 1024) = (u32x4){H[4], H[5], H[6], H[7]};
    LAS char* area = scr + 2048;
    *(LAS u32x2v*)(area + lane * 8) = (u32x2v){0u, 0u};
    u32 E1 = (wave_sum(h7_of(amax) & 127) + 122) >> 8;
    E1 = E1 < 31 ? E1 : 31;
    const u32 eb = E1 * 0x01010101u | 0x80808080u;
    u32 K[2] = {0, 0}, e4 = 0, pos = 0;
    u64 b2 = 0, b3 = 0;
#pragma unroll
    for (int t = 0; t < 8; ++t) {
        const u32 o = (eb - ((H[t] >> 2) & 0x1f1f1f1fu)) ^ 0x80808080u, o7 = o & 0x7f7f7f7fu;
        const u32 big = (o | (o7 + 0x7a7a7a7au)) & 0x80808080u, huge = (o | (o7 + 0x73737373u)) & 0x80808080u;
        const u32 M2 = big >> 7, m7 = (big >> 4) - M2;
        const u32 om = (o & ~m7) | (m7 & 0x06060606u);
        const u32 sel = om & 0x07070707u;
        const u32 c = __builtin_amdgcn_perm(0x00030303u, 0x02010003u, sel);
        const u32 q = __builtin_amdgcn_perm(0x00030100u, 0x00000002u, sel);
        const u32 M = __builtin_amdgcn_perm(0x00010101u, 0x00000001u, sel);
        const u32 v = ((o7 + 0x02020202u) | ((huge >> 4) - (huge >> 7))) & m7;
        const u32 P = byte_scan(M), P2 = byte_scan(M2), X2 = P2 - M2;
        const u32 bits2 = __builtin_amdgcn_udot4(q, __builtin_amdgcn_perm(0u, 0x40100401u, P - M), 0u, false);
        const u32 bits3 = __builtin_amdgcn_udot4(v, __builtin_amdgcn_perm(0u, 0x00400801u, X2), (X2 >> 24) == 3 ? (v >> 24) << 9 : 0u, false);
        b2 |= (u64)bits2 << (pos & 63);
        b3 |= (u64)bits3 << ((pos >> 16) & 63);
        pos += (P >> 23) + ((P2 >> 24) * 3 << 16);
        K[t >> 2] |= c << (2 * (t & 3));
        e4 = (e4 >> 1) | huge;
        if (t & 1) __builtin_amdgcn_sched_barrier(0);
    }
    const u32 incl = wave_scan(pos), tot = __builtin_amdgcn_readlane(incl, 63), ex = incl - pos;
    const u32 used = 8 + (tot & 0xffff) + (tot >> 16);
    u32 n4 = 0;
    bool fits = used <= 8 * var && __builtin_amdgcn_ballot_w64(pos >= (65u << 16)) == 0;
    if (fits) {
        put_bits(area, 8 + (ex & 0xffff), b2);
        put_bits(area, 8 + (tot & 0xffff) + (ex >> 16), b3);
        while (__builtin_amdgcn_ballot_w64(e4 != 0) != 0) {
            const u64 act = __builtin_amdgcn_ballot_w64(e4 != 0);
            const u32 any = (e4 | (e4 >> 8) | (e4 >> 16) | (e4 >> 24)) & 255, t = __builtin_ctz(any | 128);
            const u32 b = __builtin_ctz(((e4 >> t) & 0x01010101u) | 0x80000000u) + t;
            const u32 e = (*(const LAS unsigned char*)(scr + (t & 4) * 256 + lane * 16 + (t & 3) * 4 + ((b >> 3) & 3)) >> 2) & 31;
            const u32 k = var - 1 - n4 - rank_in(act);
            if (e4 != 0 && k < var) lds_or(area + (k & ~3u), e << ((k & 3) * 8));
            e4 &= ~(1u << (b & 31));
            n4 += __builtin_popcountll(act);
            if (used + 8 * n4 > 8 * var) { fits = false; break; }
        }
    }
    asm volatile("" ::: "memory");
    u32 vw = *(const LAS u32*)(area + lane * 4);
    if (lane == 0) vw |= fits ? E1 : 255;
    const u32 P0 = (H[0] & 0x83838383u) | ((H[1] & 0x03030303u) << 2) | ((H[1] >> 1) & 0x40404040u) | ((H[6] & 0x03030303u) << 4);
    const u32 P1 = (H[2] & 0x83838383u) | ((H[3] & 0x03030303u) << 2) | ((H[3] >> 1) & 0x40404040u) | ((H[7] & 0x03030303u) << 4);
    const u32 P2 = (H[4] & 0x83838383u) | ((H[5] & 0x03030303u) << 2) | ((H[5] >> 1) & 0x40404040u)
                 | ((H[6] >> 2) & 0x20202020u) | ((H[7] >> 3) & 0x10101010u);
    before_stores(fits);
    st_sys((u32x4){L[0], L[1], L[2], L[3]}, r0, lane * 16, dst);
    st_sys((u32x4){L[4], L[5], L[6], L[7]}, r0, lane * 16, dst + 1024);
    st_sys((u32x4){P0, P1, P2, K[0]}, r0, lane * 16, dst + 2048);
    __builtin_amdgcn_raw_buffer_store_b32(K[1], r0, lane * 4 + dst + 3072, 0, 17);
    __builtin_amdgcn_raw_buffer_store_b32(vw, r0, lane * 4 + dst + 3328, 0, 17);
}
__device__ __forceinline__ void unpack_item(u32x4 (&h)[4], const u32x4& c0, const u32x4& c1, const u32x4& c2, u32 k1,
                                            const LAS char* area, u32 E1, u32 var, LAS char* scr, int lane) {
    const u32 K[2] = {c2[3], k1};
    const u32 X[2] = {K[0] & (K[0] >> 1) & 0x55555555u, K[1] & (K[1] >> 1) & 0x55555555u};
    const u32 n2 = 2 * (__builtin_popcount(X[0]) + __builtin_popcount(X[1])), i2 = wave_scan(n2);
    u64 b2 = get_bits(area, 8 + i2 - n2);
    b2 &= n2 < 64 ? ~(~0ull << n2) : ~0ull;
    const u64 y2 = b2 & (b2 >> 1) & 0x5555555555555555ull;
    const u32 n3 = 3 * __builtin_popcountll(y2), i3 = wave_scan(n3);
    u64 b3 = get_bits(area, 8 + __builtin_amdgcn_readlane(i2, 63) + i3 - n3);
    const u32 e4x = E1 * 0x04040404u | 0x80808080u;
    const u32 t1 = (e4x - 0x000c0804u) & 0x007c7c7cu, t2 = (e4x - 0x00001410u) & 0x007c7c7cu;
    const u32 t3lo = (e4x - 0x24201c18u) & 0x7c7c7c7cu, t3hi = (e4x - 0x00302c28u) & 0x007c7c7cu;
    LAS char* row = scr + lane * 36;
    u32 e4 = 0;
    const u32 nch = (wave_max(n2) + 7) >> 3;
#pragma unroll
    for (int k = 0; k < 8; ++k) {
        if (k >= 4 && k >= nch) break;
        const u32 b8 = (u32)(b2 >> (8 * k)) & 255, y = b8 | (b8 << 12), q = (y | (y << 6)) & 0x03030303u;
        const u32 M2 = q & (q >> 1), P2 = byte_scan(M2), m7 = times7(M2);
        const u32 lo = (u32)b3, z = (lo & 0x3f) | ((lo & 0xfc0) << 10), f = (z | (z << 5)) & 0x07070707u;
        const u32 v = __builtin_amdgcn_perm(0u, f, P2 - M2) & m7;
        b3 >>= (P2 >> 24) * 3;
        *(LAS u32*)(row + 4 * k) = __builtin_amdgcn_perm(0u, t2, q) | __builtin_amdgcn_perm(t3hi, t3lo, v | (0x0c0c0c0cu - (M2 << 2) - (M2 << 3)));
        e4 |= (((v + M2) >> 3) & 0x01010101u) << k;
    }
    u32 n4 = 0;
    while (__builtin_amdgcn_ballot_w64(e4 != 0) != 0) {
        const u64 act = __builtin_amdgcn_ballot_w64(e4 != 0);
        const u32 any = (e4 | (e4 >> 8) | (e4 >> 16) | (e4 >> 24)) & 255, t = __builtin_ctz(any | 128);
        const u32 b = __builtin_ctz(((e4 >> t) & 0x01010101u) | 0x80000000u) + t;
        const u32 x = *(const LAS unsigned char*)(area + ((var - 1 - n4 - rank_in(act)) & 255));
        if (e4 != 0) *(LAS unsigned char*)(row + t * 4 + (b >> 3)) = x << 2;
        e4 &= ~(1u << (b & 31));
        n4 += __builtin_popcountll(act);
    }
    const u32 SM[8] = {c2[0] & 0x83838383u, ((c2[0] << 1) & 0x80808080u) | ((c2[0] >> 2) & 0x03030303u),
                       c2[1] & 0x83838383u, ((c2[1] << 1) & 0x80808080u) | ((c2[1] >> 2) & 0x03030303u),
                       c2[2] & 0x83838383u, ((c2[2] << 1) & 0x80808080u) | ((c2[2] >> 2) & 0x03030303u),
                       ((c2[2] << 2) & 0x80808080u) | ((c2[0] >> 4) & 0x03030303u),
                       ((c2[2] << 3) & 0x80808080u) | ((c2[1] >> 4) & 0x03030303u)};
    u32 M[8], P[8], g[8], at = 0;
#pragma unroll
    for (int t = 0; t < 8; ++t) {
        M[t] = (X[t >> 2] >> (2 * (t & 3))) & 0x01010101u;
        P[t] = byte_scan(M[t]);
        const LAS u32* w = (const LAS u32*)(row + (at & ~3u));
        g[t] = __builtin_amdgcn_alignbyte(w[1], w[0], at);
        at += P[t] >> 24;
    }
#pragma unroll
    for (int t = 0; t < 8; ++t) {
        const u32 c = (K[t >> 2] >> (2 * (t & 3))) & 0x03030303u;
        const u32 m = times7(M[t]);
        const u32 H = SM[t] | __builtin_amdgcn_perm(t1, g[t], ((P[t] - M[t]) & m) | ((c | 0x04040404u) & ~m));
        const u32 L = t < 4 ? c0[t & 3] : c1[t & 3];
        h[t >> 1][(t & 1) * 2] = __builtin_amdgcn_perm(H, L, 0x05010400u);
        h[t >> 1][(t & 1) * 2 + 1] = __builtin_amdgcn_perm(H, L, 0x07030602u);
    }
}

__device__ __forceinline__ void pack_out(const u32x4 (&o)[4], LAS char* scr, rsrc_t r0, u32 dst, int lane) {
    u32 M[12], mx = 0;
    u32x4 E;
#pragma unroll
    for (int r = 0; r < 4; ++r) {
        u32 R[4];
#pragma unroll
        for (int c = 0; c < 4; ++c) { R[c] = __builtin_amdgcn_alignbit(o[r][c], o[r][c], 31); mx = umax2(mx, R[c]); }
#pragma unroll
        for (int i = 0; i < 3; ++i) M[r * 3 + i] = __builtin_amdgcn_perm(R[3], R[i], 0x04020100u + (i << 24));
        E[r] = __builtin_amdgcn_perm(R[1], R[0], 0x0c0c0703u) | __builtin_amdgcn_perm(R[3], R[2], 0x07030c0cu);
    }
    *(LAS u32x4*)(scr + lane * 16) = E;
    const u32 emax = wave_max(mx) >> 24, lo = emax > 15 ? emax - 15 : 0, lo1 = lo * 0x01010101u;
    LAS u32* list = (LAS u32*)(scr + 1024);
    list[lane] = lo;
    u32 C[4], em = 0;
#pragma unroll
    for (int r = 0; r < 4; ++r) {
        const u32 d = ((E[r] | 0x80808080u) - (lo1 & 0x7f7f7f7fu)) ^ ((E[r] ^ ~lo1) & 0x80808080u);
        const u32 x = d & 0xf0f0f0f0u, y = x | (x << 1), nz = (y | (y << 2)) & 0x80808080u;
        C[r] = d & 0x0f0f0f0fu & ~((nz >> 3) - (nz >> 7));
        em = (em >> 1) | nz;
    }
    u32 used = 1;
    for (u64 act = __builtin_amdgcn_ballot_w64(em != 0); act != 0; act = __builtin_amdgcn_ballot_w64(em != 0)) {
        const u32 k = used + __builtin_amdgcn_mbcnt_hi((u32)(act >> 32), __builtin_amdgcn_mbcnt_lo((u32)act, 0));
        const u32 b = __builtin_ctz(em | 0x80000000u);
        const u32 pos = lane * 16 + (b & 3) * 4 + (b >> 3);
        list[em != 0 && k < 16 ? k : 64] = pos * 256 + *(const LAS unsigned char*)(scr + pos);
        em &= em - 1;
        used += __builtin_popcountll(act);
        if (used > 16) break;
    }
    asm volatile("" ::: "memory");
    u32 ent = list[lane];
    if (used > 16) {
        if (lane == 0) ent = 0xffffffffu;
#pragma unroll
        for (int r = 0; r < 4; ++r) st_sys(o[r], r0, lane * 16 + r * 1024, dst + STG_BYTES);
    }
#pragma unroll
    for (int i = 0; i < 3; ++i) st_sys((u32x4){M[4 * i], M[4 * i + 1], M[4 * i + 2], M[4 * i + 3]}, r0, lane * 16 + i * 1024, dst);
    __builtin_amdgcn_raw_buffer_store_b64((u32x2v){C[0] | (C[1] << 4), C[2] | (C[3] << 4)}, r0, lane * 8 + dst + 3072, 0, 17);
    __builtin_amdgcn_raw_buffer_store_b32(ent, r0, (lane < 16 ? lane * 4 : 0x80000000u) + dst + 3584, 0, 17);
}
__device__ __forceinline__ void unpack_out(u32x4 (&o)[4], const u32x4& c0, const u32x4& c1, const u32x4& c2, u32 p4, u32 p5,
                                           u32 ent, u32 lo, LAS char* scr, int lane) {
    const u32 M[12] = {c0[0], c0[1], c0[2], c0[3], c1[0], c1[1], c1[2], c1[3], c2[0], c2[1], c2[2], c2[3]};
    const u32 lo1 = lo * 0x01010101u;
    u32x4 E = {(p4 & 0x0f0f0f0fu) + lo1, ((p4 >> 4) & 0x0f0f0f0fu) + lo1, (p5 & 0x0f0f0f0fu) + lo1, ((p5 >> 4) & 0x0f0f0f0fu) + lo1};
    *(LAS u32x4*)(scr + lane * 16) = E;
    __hip_atomic_fetch_add((LAS u32*)(scr + ((ent >> 8) & 0x3fc)), ((ent & 255) - lo) << ((ent >> 5) & 24),
                           __ATOMIC_RELAXED, __HIP_MEMORY_SCOPE_WORKGROUP);
    asm volatile("" ::: "memory");
    E = *(const LAS u32x4*)(scr + lane * 16);
#pragma unroll
    for (int r = 0; r < 4; ++r) {
        u32 R[4];
#pragma unroll
        for (int i = 0; i < 3; ++i) R[i] = __builtin_amdgcn_perm(E[r], M[r * 3 + i], 0x04020100u + (i << 24));
        R[3] = __builtin_amdgcn_perm(M[r * 3 + 1], M[r * 3], 0x0c0c0703u) | __builtin_amdgcn_perm(E[r], M[r * 3 + 2], 0x07030c0cu);
#pragma unroll
        for (int c = 0; c < 4; ++c) o[r][c] = __builtin_amdgcn_alignbit(R[c], R[c], 1);
    }
}

__device__ __forceinline__ void load_item(f32x4 (&v)[8], const Params& p, int u, int wid, int lane) {
    const int blk = u >> 4, it = (u & 15) * 8 + wid;
    const bool isx = it < 64;
    const rsrc_t r = mk_rsrc(isx ? p.x : p.w, OUT_BYTES);
    const u32 vf = isx ? (lane & 15) * 32768 + (lane >> 4) * 16 : (lane >> 4) * 16384 + (lane & 15) * 16;
    const u32 base = isx ? it * 524288 + blk * 512 : (blk * 128 + (wid & 3) * 32) * 4096 + ((it - 64) >> 2) * 256;
    const u32 sa = isx ? 64 : 4096, sb = isx ? 256 : 65536;
#pragma unroll
    for (int i = 0; i < 8; ++i)
        v[i] = __builtin_bit_cast(f32x4, __builtin_amdgcn_raw_buffer_load_b128(r, vf, base + (i & 3) * sa + (i >> 2) * sb, IN_AUX));
}
__device__ __forceinline__ void convert_local(const Params& p, LAS char* lds, f32x4 (&v)[8], char* wsl, int wg, int wid, int lane, u32 call) {
    const int rank = rfl(p.rank);
    constexpr int NU = NBLK * 16;
    const u32 par = OFF_PAR + (call & 1) * PAR_BYTES;
    const rsrc_t r0 = mk_rsrc(p.ws[rank], WS_NEED);
    LAS char* scr = lds + wid * 4096;
    auto publish = [&](int u) {
        if (wid == 0) {
            if (lane < 8 && lane != rank) st_flag(wsl + OFF_RDY + rank * 8192 + u * 4, call);
            if (lane == 8) add_sys(p.ws[rank] + OFF_CNT + (u >> 4) * CNT_BLK + ((u & 8) ? 1024 : 0) + rank * 128);
        }
    };
    int u = wg, un = wg + NGEMM, up = -1;
    WAIT_VM0();
    for (int r = 0; u < NU; ++r) {
        const int blk = u >> 4, grp = u & 15, it = grp * 8 + wid;
        u32x4 h[4];
        u32 base, step;
        if (grp < 8) {
            base = par + (u32)blk * (RS1 * 4096) + it * 4096; step = 1024;
#pragma unroll
            for (int i = 0; i < 4; ++i) h[i] = cvt8(v[2 * i], v[2 * i + 1]);
        } else {
            base = par + REL_B + (u32)blk * (CF * 4096) + ((it - 64) & 60) * 4096 + (wid & 3) * 1024; step = 4096;
#pragma unroll
            for (int c = 0; c < 4; ++c) h[c] = cvt_w(v, c);
        }
        float amax = 0.0f;
#pragma unroll
        for (int i = 0; i < 8; ++i) amax = fmaxf(fmaxf(fmaxf(amax, fabsf(v[i][0])), fmaxf(fabsf(v[i][1]), fabsf(v[i][2]))), fabsf(v[i][3]));
        STAMPB_IF(r == 0, 0);
        if (r > 0) load_item(v, p, (u32)un < (u32)NU ? un : NU - 1, wid, lane);
        pack_item(h, amax, grp < 8 ? VAR_X : VAR_W, scr, r0, par + REL_PK + (u32)(blk * 128 + it) * 4096, lane, [&](bool fits) {
            WAIT_VM0();
            if (r > 0) {
                __builtin_amdgcn_s_barrier();
                if (up >= 0) publish(up);
            }
            if (!fits || it < FI || it >= 64) {
#pragma unroll
                for (int i = 0; i < 4; ++i) st_sys(h[i], r0, lane * 16, base + i * step);
            }
        });
        up = u;
        if (r == 0) {
            STAMPB_IF(true, 1);
            WAIT_VM0();
            STAMPB_IF(true, 2);
            __builtin_amdgcn_s_barrier();
            STAMP(6);
            publish(u);
            up = -1;
            load_item(v, p, un, wid, lane);
            WAIT_VM0();
        }
        u = un;
        un += NGEMM;
    }
    WAIT_VM0();
    __builtin_amdgcn_s_barrier();
    if (up >= 0) publish(up);
}

constexpr int PUSH_PER = FI1 + 64 + XA, NPUSH = PB * PUSH_PER, NSPARE = NCOMM * 5;
__device__ __forceinline__ void push_decode(int n, int& blk, int& it) {
    blk = n / PUSH_PER; it = n - blk * PUSH_PER;
    if (it >= FI1 + 64) { const int xi = blk * XA + it - FI1 - 64; blk = xi / FX; it = FI + xi - blk * FX; }
    else if (it >= FI1) it += 64 - FI1;
}
__device__ __forceinline__ void push_role(const Params& p, f32x4 (&v)[8], char* wsl, int sw, int lane, u32 call) {
    const int rank = rfl(p.rank);
    const int g = rank >> 2, j = rank & 3, og = (g ^ 1) * 4;
    const u32 par = OFF_PAR + (call & 1) * PAR_BYTES;
    for (int n = sw; n < NPUSH; n += NSPARE) {
        int blk, it;
        push_decode(n, blk, it);
        u32x4 h[4];
        int d0 = 0, nd = 4, dstep = 1; u32 base, step = 1024, ostep = 0;
        if (it < 64) {
#pragma unroll
            for (int i = 0; i < 4; ++i) h[i] = cvt8(v[2 * i], v[2 * i + 1]);
            if (it < FI1) { d0 = j + 3; nd = 3; dstep = 3; ostep = FI1 * 4096; base = par + (blk * RS1 + 64 + it) * 4096; }
            else base = par + (blk * RS1 + 64 + 3 * FI1 + j * FX + it - FI) * 4096;
        } else {
#pragma unroll
            for (int c = 0; c < 4; ++c) h[c] = cvt_w(v, c);
            d0 = (it - 64) >> 4; nd = 1; step = 4096;
            base = par + REL_B + (blk * CF + 64 + 16 * j + (it & 12)) * 4096 + (it & 3) * 1024;
        }
        if (n + NSPARE < NPUSH) { int b2, i2; push_decode(n + NSPARE, b2, i2); load_item(v, p, b2 * 16 + (i2 >> 3), i2 & 7, lane); }
        u32 dmask = 0;
        for (int d = 0; d < nd; ++d) {
            const int dr = (it < FI1 ? g * 4 : og) + ((d0 + d * dstep) & 3);
            const rsrc_t rd = mk_rsrc(p.ws[dr], WS_NEED);
#pragma unroll
            for (int i = 0; i < 4; ++i) st_sys(h[i], rd, lane * 16, base + d * ostep + i * step);
            dmask |= 1u << dr;
        }
        WAIT_VM0();
        if (lane < 8 && ((dmask >> lane) & 1)) add_sys(wsl + OFF_CNT + blk * CNT_BLK + (it < 64 ? 0 : 1024) + rank * 128);
    }
}

#define P_NDEC 9
#define P_FLUSH 4
#define P_DSLEEP 3
constexpr int SCR = 2560;
constexpr int NLW = 3, RING = 6, QLEN = 32, NDEC = P_NDEC;
constexpr int LDS_CTL = NLW * RING * 4096;
constexpr int LDS_DONE = LDS_CTL + QLEN * 16, LDS_TAIL = LDS_DONE + NLW * RING * 4, LDS_SCR = LDS_CTL + 1024;
static_assert(7 * WL <= NLW * NCOMM, "loaders");
constexpr u32 IDX_END = 0xffffffffu, IDX_BAD = 0x80000000u, IDX_CROSS = 0x40000000u;
__device__ __forceinline__ u32 lds_get(const LAS char* a) {
    u32 v;
    asm volatile("ds_read_b32 %0, %1\n\ts_waitcnt lgkmcnt(0)" : "=v"(v) : "v"((u32)(size_t)a) : "memory");
    return rfl(v);
}
__device__ __forceinline__ void lds_put(LAS char* a, u32 v) {
    asm volatile("" ::: "memory");
    *(volatile LAS u32*)a = v;
    asm volatile("" ::: "memory");
}

struct Item { u32 idx, dst; };
constexpr int N1 = NBLK * FI1, NT_I = N1 + NBLK * F2;
constexpr int PER = 16 + XA, NP = NBLK * PER, NX = NBLK * (16 + FX), NT_C = NX + 1024;
constexpr int TAB_BYTES = (NT_I > NT_C ? NT_I : NT_C) * 4, LDS_TAB = LDS_SCR + NDEC * SCR;
static_assert(LDS_TAB + NLW * TAB_BYTES <= LDS_BYTES, "communication LDS");
__device__ __forceinline__ void build_table(LAS char* tab, int rank, int slot, int first, int stride) {
    const int g = rank >> 2, j = rank & 3, q = slot - 3;
    for (int it = first; it < (slot < 3 ? NT_I : NT_C); it += stride) {
        Item m;
        if (slot < 3) {
            const bool s2 = it >= N1;
            const int a = it - N1, b1 = it / FI1;
            const int blk = s2 ? a >> 4 : b1, s = s2 ? FI1 + (a & 15) : it - b1 * FI1;
            m.idx = blk * 128 + s;
            m.dst = (s2 ? REL_A2 + (slot * F2 - FI1) * 4096 : (64 + slot * FI1) * 4096) + (blk * (s2 ? RF2 : RS1) + s) * 4096;
        } else if (it < NX) {
            const int blk = it / PER, c = it - blk * PER;
            const bool tail = it >= NP, isx = tail || (c < 2 * XA && (c & 1));
            const int cw = c < 2 * XA ? c >> 1 : c - XA;
            const int xi = tail ? it - NP + NBLK * XA : blk * XA + (c >> 1), xb = xi / FX, xs = FI + xi - xb * FX;
            m.idx = isx ? xb * 128 + xs : blk * 128 + 64 + 16 * j + cw;
            m.dst = isx ? (xb * RS1 + xs + 64 + 3 * FI1 + q * FX - FI) * 4096
                        : REL_B + (blk * CF + 64 + 16 * q + (cw & 12)) * 4096 + (cw & 3) * 1024;
        } else {
            const int o = it - NX;
            m.idx = 8192 + o;
            int row, col;
            if (o < 832) {
                row = row_of_rf1((o >> 6) * 16 + ((o >> 3) & 3) * 4 + (o & 3), g ^ 1, q);
                col = ((o >> 5) & 1) * 128 + ((o >> 2) & 1) * 64;
            } else {
                const int z = o - 832, rt = z >> 5;
                row = ((g ^ 1) * 4 + ((q + 1 + (rt >> 1)) & 3)) * NS + (FI1 + (rt & 1) * 8 + (z & 7)) * 16;
                col = ((z >> 3) & 3) * 64;
            }
            m.dst = (u32)row * 4096 + (q * 256 + col) * 4;
        }
        *(LAS u32*)(tab + it * 4) = m.idx | m.dst << (m.idx < 8192 ? 4 : 6);
    }
}

template <bool CROSS>
__device__ __forceinline__ void load_role(const Params& p, LAS char* lds, int ld, int nld, int slot, int wv, int lane, u32 call) {
    const int rank = rfl(p.rank);
    const int g = rank >> 2, j = rank & 3;
    const int peer = CROSS ? (g ^ 1) * 4 + slot - 3 : g * 4 + ((j + 1 + slot) & 3);
    constexpr int NT = CROSS ? NT_C : NT_I;
    const u32 par = OFF_PAR + (call & 1) * PAR_BYTES;
    char* ws = uni(p.ws[rank]);
    const rsrc_t r0 = rsrc_u(ws, WS_NEED), rp = mk_rsrc(p.ws[peer], WS_NEED);
    const char* rdy = ws + OFF_RDY + peer * 8192;
    const u32 vo = lane * 16;
    constexpr int SKIP = PB * (CROSS ? PER : FI1);
    const u32 head = lane == 0 ? OFF_QUE + slot * 256 : WS_NEED, tbase = SKIP + 4 * WL - (call - 1) * (NT - SKIP);
    const bool nowait = (p.pad & 4) != 0;
    const LAS char* tab = lds + LDS_TAB + ld * TAB_BYTES;
    const LAS char* done = lds + LDS_DONE + ld * RING * 4;
    const u32 ldsb = rfl((u32)(size_t)lds) + ld * RING * 4096;
    auto table = [&](int it) {
        asm volatile("ds_read_b32 v160, %0" :: "v"((u32)(size_t)tab + (it < NT ? it : NT - 1) * 4) : "memory", "v160");
    };
    auto entry = [&] {
        u32 t;
        asm volatile("s_waitcnt lgkmcnt(0)\n\tv_readfirstlane_b32 %0, v160" : "=s"(t) :: "memory");
        Item m;
        m.idx = t & 16383;
        m.dst = (t - m.idx) >> (CROSS && m.idx >= 8192 ? 6 : 4);
        return m;
    };
    auto decode = [&](int it) { table(it); return entry(); };
    auto flag_of = [&](u32 idx) { return idx < 8192 ? idx >> 3 : (idx < 8192 + 832 ? 1024 + ((idx - 8192) >> 2) : idx - (8192 + 832 - 1024 - 208)); };
    u32 okupto = 0, wbase = 0; u64 wmask = 0;
    auto ready = [&](u32 fi) {
        if (fi < okupto) return true;
        if (fi - wbase < 64u && ((wmask >> (fi - wbase)) & 1)) return true;
        for (;;) {
            wbase = okupto;
            wmask = __builtin_amdgcn_ballot_w64(ld_sys(rdy + (wbase + lane) * 4) >= call);
            if (wmask != ~0ull) break;
            okupto += 64;
            if (fi < okupto) return true;
        }
        okupto += __builtin_ctzll(~wmask);
        if (fi - wbase < 64u) return ((wmask >> (fi - wbase)) & 1) != 0;
        return rfl(ld_sys(rdy + fi * 4)) >= call;
    };
    struct Prep { u32 src, m0v, vl; };
    auto prep = [&](const Item& m, u32 n) {
        Prep r;
        r.src = par + REL_PK + (m.idx + (CROSS && m.idx >= 8192 ? j * 1024 : 0)) * 4096;
        r.m0v = ldsb + (n % RING) * 4096;
        const u32 rest = CROSS && m.idx >= 8192 ? 576 : (CROSS && (m.idx & 64)) ? 256 + VAR_W : 256 + VAR_X;
        r.vl = vo < rest ? vo : 0x80000000u;
        asm volatile("" : "+s"(r.src), "+s"(r.m0v), "+v"(r.vl));
        return r;
    };
    auto issue = [&](const Prep& r) {
        asm volatile("s_mov_b32 m0, %0\n\ts_nop 0\n\t"
                     "buffer_load_dwordx4 %1, %3, %4 offen sc0 sc1 lds\n\t"
                     "buffer_load_dwordx4 %1, %3, %4 offen offset:1024 sc0 sc1 lds\n\t"
                     "buffer_load_dwordx4 %1, %3, %4 offen offset:2048 sc0 sc1 lds\n\t"
                     "buffer_load_dwordx4 %2, %3, %4 offen offset:3072 sc0 sc1 lds"
                     :: "s"(r.m0v), "v"(vo), "v"(r.vl), "s"(rp), "s"(r.src) : "memory");
    };
    u32 vtail = (u32)(size_t)lds + LDS_TAIL, vone = 1;
    asm volatile("" : "+v"(vtail), "+v"(vone));
    auto enqueue = [&](u32 idx, u32 dst, u32 n) {
        asm volatile("s_mov_b64 exec, 1\n\tds_add_rtn_u32 v162, %0, %1\n\tv_mov_b32 v159, %2\n\tv_mov_b32 v160, %3\n\tv_mov_b32 v161, %4\n\t"
                     "s_waitcnt lgkmcnt(0)\n\tv_add_u32 v158, 1, v162\n\tv_and_b32 v162, %6, v162\n\tv_lshl_add_u32 v162, v162, 4, %5\n\t"
                     "ds_write_b128 v162, v[158:161]\n\ts_mov_b64 exec, -1"
                     :: "v"(vtail), "v"(vone), "s"(idx), "s"(dst), "s"(n << 8 | ld << 4 | peer), "s"(rfl((u32)(size_t)lds) + LDS_CTL), "n"(QLEN - 1)
                     : "memory", "v158", "v159", "v160", "v161", "v162");
    };
    __builtin_amdgcn_s_setprio(3);
    int e = SKIP + wv, e1 = e + WL, e2 = e + 2 * WL;
    const int e3 = e + 3 * WL;
    Item nx = decode(e1), ia = decode(e);
    bool bad = spin_ge<2>(rdy + flag_of(ia.idx) * 4, call, ws + OFF_TMO, call);
    STAMP(5);
    issue(prep(ia, 0));
    bool pre = ready(flag_of(nx.idx));
    u32 n = 0;
    const u32 cls = CROSS ? IDX_CROSS : 0;
    auto wait_free = [&](u32 m) { while (lds_get(done + (m % RING) * 4) + RING <= m) __builtin_amdgcn_s_sleep(1); };
    auto body = [&](auto odd) {
        if constexpr (odd.value) asm volatile("v_mov_b32 v164, 1\n\ts_nop 0\n\tbuffer_atomic_add v164, %0, %1, 0 offen sc0" :: "v"(head), "s"(r0) : "memory", "v164");
        else                     asm volatile("v_mov_b32 v165, 1\n\ts_nop 0\n\tbuffer_atomic_add v165, %0, %1, 0 offen sc0" :: "v"(head), "s"(r0) : "memory", "v165");
        table(e2);
        u32 dn;
        asm volatile("ds_read_b32 v162, %1\n\ts_waitcnt lgkmcnt(0)\n\tv_readfirstlane_b32 %0, v162"
                     : "=s"(dn) : "v"((u32)(size_t)done + ((n + 1) % RING) * 4) : "memory", "v162");
        const Item nn = entry();
        bool pre2 = e2 < NT;
        if (pre2 && !(nowait && nn.idx >= 8192)) pre2 = ready(flag_of(nn.idx));
        TICK(t1);
        if (dn + RING <= n + 1) wait_free(n + 1);
        TICK(t2);
        const Prep px = prep(nx, n + 1);
        u32 told;
        if constexpr (odd.value) asm volatile("s_waitcnt vmcnt(1)\n\tv_readfirstlane_b32 %0, v165" : "=s"(told) :: "memory");
        else                     asm volatile("s_waitcnt vmcnt(1)\n\tv_readfirstlane_b32 %0, v164" : "=s"(told) :: "memory");
        TICK(t3);
        if (pre) issue(px);
        enqueue(ia.idx | cls | (bad ? IDX_BAD : 0), ia.dst, n);
        if (!pre && e1 < NT) {
            bad |= spin_ge<2>(rdy + flag_of(nx.idx) * 4, call, ws + OFF_TMO, call);
            issue(px);
        }
        STAMP_IF(n == 0, 6);
        STAMP_IF(CROSS ? (e < NP && e1 >= NP) : (e < N1 && e1 >= N1), 1);
        STAMP_IF(CROSS && e < NX && e1 >= NX, 2);
        STAMP_IF(CROSS && e < NX + 832 && e1 >= NX + 832, 3);
        ia = nx; nx = nn; pre = pre2; ++n;
        e = e1; e1 = e2; e2 = n == 1 ? e3 : (int)(told + tbase);
    };
    for (;;) {
        body(std::false_type{});
        if (e >= NT) break;
        body(std::true_type{});
        if (e >= NT) break;
    }
    u32 fin;
    asm volatile("ds_add_rtn_u32 v162, %1, %2\n\ts_waitcnt lgkmcnt(0)\n\tv_readfirstlane_b32 %0, v162"
                 : "=s"(fin) : "v"((u32)(size_t)lds + LDS_TAIL + 4), "v"(lane == 0 ? 1u : 0u) : "memory", "v162");
    if (fin + 1 == (u32)nld)
        for (int k = 0; k < NDEC; ++k) enqueue(IDX_END, 0, 0);
}

__device__ __forceinline__ void decode_role(const Params& p, LAS char* lds, u32 me, int lane, u32 call) {
    const int rank = rfl(p.rank);
    const int j = rank & 3;
    const u32 par = OFF_PAR + (call & 1) * PAR_BYTES;
    const rsrc_t r0 = mk_rsrc(p.ws[rank], WS_NEED), ro = mk_rsrc(p.out[rank], OUT_BYTES);
    const u32 vo = lane * 16, vout = (lane >> 4) * 4 * 4096 + (lane & 15) * 16;
    LAS char* scr = lds + LDS_SCR + me * SCR;
    u32 q0, q1, q2, q3;
    auto peek = [&](u32 n) {
        asm volatile("ds_read_b128 v[158:161], %4\n\ts_waitcnt lgkmcnt(0)\n\tv_readfirstlane_b32 %0, v158\n\tv_readfirstlane_b32 %1, v159\n\t"
                     "v_readfirstlane_b32 %2, v160\n\tv_readfirstlane_b32 %3, v161"
                     : "=s"(q0), "=s"(q1), "=s"(q2), "=s"(q3) : "v"((u32)(size_t)lds + LDS_CTL + (n % QLEN) * 16) : "memory", "v158", "v159", "v160", "v161");
        return q0 == n + 1;
    };
    const u32 vsig = lane == 0 ? 0u : 0x80000000u;
    u32 one = 1;
    asm volatile("" : "+v"(one));
    auto signal = [&](u32 c) {
        asm volatile("s_nop 4\n\tbuffer_atomic_add %0, %1, %2, %3 offen sc1" :: "v"(one), "v"(vsig), "s"(r0), "s"(rfl(c)) : "memory");
    };
    const u32x4 nan4 = {0x7fc07fc0u, 0x7fc07fc0u, 0x7fc07fc0u, 0x7fc07fc0u};
    u32 owed = WS_NEED, owed2 = WS_NEED;
    for (u32 n = me;; n += NDEC) {
        TICK(t0);
        bool here = peek(n);
        for (int polls = 0; !here && polls < P_FLUSH; ++polls) { __builtin_amdgcn_s_sleep(P_DSLEEP); here = peek(n); }
        if (!here) {
            asm volatile("s_waitcnt vmcnt(0)" ::: "memory");
            signal(owed2); signal(owed); owed = owed2 = WS_NEED;
            while (!peek(n)) __builtin_amdgcn_s_sleep(P_DSLEEP);
        }
        TICK(t1);
        if (q1 == IDX_END) break;
        const u32 dst = q2, item = q3 >> 8, ring = (((q3 >> 4) & 3) * RING + item % RING), peer = q3 & 7;
        const LAS char* sl = lds + ring * 4096;
        const u32x4 c0 = *(const LAS u32x4*)(sl + vo), c1 = *(const LAS u32x4*)(sl + 1024 + vo), c2 = *(const LAS u32x4*)(sl + 2048 + vo);
        const u32 k1 = *(const LAS u32*)(sl + 3072 + lane * 4);
        const u32 hx = *(const LAS unsigned char*)(sl + 3328);
        const bool bad = (q1 & IDX_BAD) != 0;
        const u32 idx = q1 & 0xffff;
        const bool out = idx >= 8192, isw = !out && (idx & 64);
        const u32 step = (out || isw) ? 4096 : 1024;
        u32x4 h[4];
        bool done = false;
        if (!bad && !out) {
            const u32 hd = rfl(hx);
            if (hd != 255) { unpack_item(h, c0, c1, c2, k1, sl + 3328, hd, isw ? VAR_W : VAR_X, scr, lane); done = true; }
        } else if (!bad) {
            const u32x2v pp = *(const LAS u32x2v*)(sl + 3072 + lane * 8);
            const u32 ent = *(const LAS u32*)(sl + 3584 + (lane & 15) * 4), hd = rfl(ent);
            if (hd < 256) { unpack_out(h, c0, c1, c2, pp[0], pp[1], lane < 16 ? ent : hd, hd, scr, lane); done = true; }
        }
        if (!done) {
            const u32 blk = idx >> 7, s = idx & 127;
            const u32 raw = par + (out ? REL_PK + STG_BYTES + (idx + j * 1024) * 4096
                                       : isw ? REL_B + (blk * CF + (s & 60)) * 4096 + (s & 3) * 1024 : (blk * RS1 + s) * 4096);
            const u32 rs = isw ? 4096 : 1024;
            const rsrc_t rp = mk_rsrc(p.ws[peer], WS_NEED);
            asm volatile("s_nop 4\n\tbuffer_load_dwordx4 %0, %4, %5, %6 offen sc0 sc1\n\tbuffer_load_dwordx4 %1, %4, %5, %7 offen sc0 sc1\n\t"
                         "buffer_load_dwordx4 %2, %4, %5, %8 offen sc0 sc1\n\tbuffer_load_dwordx4 %3, %4, %5, %9 offen sc0 sc1\n\ts_waitcnt vmcnt(0)"
                         : "=&v"(h[0]), "=&v"(h[1]), "=&v"(h[2]), "=&v"(h[3])
                         : "v"(vo), "s"(rp), "s"(rfl(raw)), "s"(rfl(raw + rs)), "s"(rfl(raw + 2 * rs)), "s"(rfl(raw + 3 * rs)) : "memory");
            if (bad) { h[0] = nan4; h[1] = nan4; h[2] = nan4; h[3] = nan4; }
        }
        lds_put(lds + LDS_DONE + ring * 4, item + 1);
        const rsrc_t rd = out ? ro : r0;
        const u32 vd = (out ? vout : vo + par) + dst;
#pragma unroll
        for (int i = 0; i < 4; ++i) st_sys(h[i], rd, vd, i * step);
        TICK(t2);
        asm volatile("s_waitcnt vmcnt(9)" ::: "memory");
        TICK(t3);
        signal(owed2);
        owed2 = owed;
        owed = rfl(out ? WS_NEED : OFF_CNT + (idx >> 7) * CNT_BLK + peer * 128 + (isw ? 1024 : (!(q1 & IDX_CROSS) && (idx & 127) >= FI1) ? 2048 : 0));
    }
    WAIT_VM0();
    signal(owed2); signal(owed);
    STAMP(4);
}

#define LDPOL " sc0 sc1"
#define GLDS2(m0v, voff, rs, soff, imm0, imm1) \
    asm volatile("s_mov_b32 m0, %0\n\ts_nop 0\n\tbuffer_load_dwordx4 %1, %2, %3 offen offset:" #imm0 LDPOL " lds\n\t" \
                 "buffer_load_dwordx4 %1, %2, %3 offen offset:" #imm1 LDPOL " lds" \
                 :: "s"(m0v), "v"(voff), "s"(rs), "s"(soff) : "memory")
#define BARRIER() do { __builtin_amdgcn_sched_barrier(0); __builtin_amdgcn_s_barrier(); __builtin_amdgcn_sched_barrier(0); } while (0)

#define SCHED() __builtin_amdgcn_sched_barrier(0)
#define WAIT_LGKM0() __builtin_amdgcn_s_waitcnt(0xC07F)

template <int STAGE>
struct Gemm {
    static constexpr int NT = STAGE == 1 ? 128 : 64;
    static constexpr int NA = STAGE == 1 ? 4 : 2;
    static constexpr int NB = STAGE == 1 ? 4 : 8;
    static constexpr int NM = STAGE == 1 ? 4 : 1;
    static constexpr int NMM = STAGE == 1 ? 16 : 8;
    static constexpr int NRD = NA + NB;
    struct Frags { f16x8 a[NA], b[NB]; };

    rsrc_t rs;
    u32 sa, sa2, sb;
    u32 ldsw;
    int wid, lane;

    template <int I> __device__ __forceinline__ void piece(int t, int buf) const {
        const u32 vo = lane * 16;
        const u32 ma = ldsw + buf * BUF_BYTES + wid * 4096, mb = ldsw + buf * BUF_BYTES + 32768 + wid * 2048;
        const u32 ka = STAGE == 1 ? (u32)(t >> 1) * (RS1 * 4096) + (t & 1) * 2048 : (u32)t * (RF2 * 4096);
        const u32 a = sa + ka, a2 = sa2 + ka;
        const u32 b = STAGE == 1 ? sb + (u32)(t >> 1) * (CF * 4096) + (t & 1) * 2048 : sb + (u32)t * (CF * 4096);
        if (I == 0) GLDS2(ma, vo, rs, a, 0, 1024);
        if (I == 1) GLDS2(ma, vo, rs, a2, 2048, 3072);
        if (I == 2) GLDS2(mb, vo, rs, b, 0, 1024);
    }
    __device__ __forceinline__ void stage(int t, int buf) const { piece<0>(t, buf); piece<1>(t, buf); piece<2>(t, buf); }

    template <int I> __device__ __forceinline__ void rd(Frags& f, const LAS char* lds, int buf, int half) const {
        const LAS char* base = lds + buf * BUF_BYTES + lane * 16;
        if constexpr (STAGE == 1) {
            const LAS char* pa = base + (wid >> 1) * 8192 + half * 1024;
            const LAS char* pb = base + 32768 + (wid & 1) * 8192 + half * 1024;
            if constexpr (I == 0) f.a[0] = *(const LAS f16x8*)pa;
            else if constexpr (I <= 4) f.b[I - 1] = *(const LAS f16x8*)(pb + (I - 1) * 2048);
            else f.a[I - 4] = *(const LAS f16x8*)(pa + (I - 4) * 2048);
        } else {
            const LAS char* pa = base + wid * 4096 + half * 2048;
            const LAS char* pb = base + 32768 + half * 2048;
            constexpr int h = I / 5, n = I % 5;
            if constexpr (n == 0) f.a[h] = *(const LAS f16x8*)(pa + h * 1024);
            else f.b[h * 4 + n - 1] = *(const LAS f16x8*)(pb + (n - 1) * 4096 + h * 1024);
        }
    }
    template <int I> __device__ __forceinline__ void mm(f32x4 (&acc)[NM][4], const Frags& f) const {
        if constexpr (STAGE == 1) acc[I >> 2][I & 3] = __builtin_amdgcn_mfma_f32_16x16x32_f16(f.a[I >> 2], f.b[I & 3], acc[I >> 2][I & 3], 0, 0, 0);
        else            acc[0][I & 3] = __builtin_amdgcn_mfma_f32_16x16x32_f16(f.a[I >> 2], f.b[I], acc[0][I & 3], 0, 0, 0);
    }
    template <int I, int N> __device__ __forceinline__ void rds(Frags& f, const LAS char* lds, int buf, int half) const {
        if constexpr (I < N) { rd<I>(f, lds, buf, half); rds<I + 1, N>(f, lds, buf, half); }
    }
    static __device__ __forceinline__ int blk_of(int t) { return STAGE == 1 ? (t >> 1) : t; }

    template <bool RD, int ST>
    __device__ __forceinline__ void half_step(f32x4 (&acc)[NM][4], const Frags& cur, Frags& nxt, const LAS char* lds, int rbuf, int rhalf,
                                              int ts, int bs, u32 cvo, u32 fso) const {
        auto first = [&] {
            if (ST == 1) asm volatile("buffer_load_dword v166, %0, %1, %2 offen sc0 sc1" :: "v"(cvo), "s"(rs), "s"(fso) : "memory", "v166");
            if (ST) { piece<0>(ts, bs); SCHED(); }
        };
        if constexpr (STAGE == 1) {
            mm<0>(acc, cur);  mm<1>(acc, cur);  if (RD) rds<0, 2>(nxt, lds, rbuf, rhalf); SCHED();
            mm<2>(acc, cur);  mm<3>(acc, cur);  if (RD) rds<2, 4>(nxt, lds, rbuf, rhalf); SCHED();
            mm<4>(acc, cur);  mm<5>(acc, cur);  if (RD) rds<4, 6>(nxt, lds, rbuf, rhalf); SCHED();
            mm<6>(acc, cur);  mm<7>(acc, cur);  if (RD) rds<6, 8>(nxt, lds, rbuf, rhalf); SCHED();
            mm<8>(acc, cur);  mm<9>(acc, cur);  SCHED();
            first();
            mm<10>(acc, cur); mm<11>(acc, cur); SCHED();
            if (ST) { piece<1>(ts, bs); SCHED(); }
            mm<12>(acc, cur); mm<13>(acc, cur); SCHED();
            if (ST) { piece<2>(ts, bs); SCHED(); }
            mm<14>(acc, cur); mm<15>(acc, cur); SCHED();
        } else {
            mm<0>(acc, cur); if (RD) rds<0, 3>(nxt, lds, rbuf, rhalf); SCHED();
            mm<1>(acc, cur); if (RD) rds<3, 5>(nxt, lds, rbuf, rhalf); SCHED();
            mm<2>(acc, cur); if (RD) rds<5, 8>(nxt, lds, rbuf, rhalf); SCHED();
            mm<3>(acc, cur); if (RD) rds<8, 10>(nxt, lds, rbuf, rhalf); SCHED();
            mm<4>(acc, cur); SCHED();
            first();
            mm<5>(acc, cur); SCHED();
            if (ST) { piece<1>(ts, bs); SCHED(); }
            mm<6>(acc, cur); SCHED();
            if (ST) { piece<2>(ts, bs); SCHED(); }
            mm<7>(acc, cur); SCHED();
        }
    }

    template <int ROLE>
    __device__ __forceinline__ u32 run(f32x4 (&acc)[NM][4], const LAS char* lds, char* ws, u32 cvo, u32 need, u32 call) const {
        constexpr bool LATE = ROLE == 2;
        char* tmo = ws + OFF_TMO;
        Frags f0, f1;
        if (ROLE == 0) spin_ge(ws + cvo + blk_of(0) * CNT_BLK, need, tmo, call);
        BARRIER();
        stage(0, 0);
        if (ROLE == 0) { spin_ge(ws + cvo + blk_of(1) * CNT_BLK, need, tmo, call); spin_ge(ws + cvo + blk_of(2) * CNT_BLK, need, tmo, call); }
        BARRIER();
        stage(1, 1);
        if (!LATE) { stage(2, 2); asm volatile("s_waitcnt vmcnt(12)" ::: "memory"); }
        else asm volatile("s_waitcnt vmcnt(6)" ::: "memory");
        BARRIER();
        rds<0, NRD>(f0, lds, 0, 0);
        WAIT_LGKM0();
        if (ROLE == 0) asm volatile("v_mov_b32 v166, 0" ::: "v166");
        int buf = 0;
        for (int t = 0; t < NT - 3; ++t) {
            const int nb = buf == 2 ? 0 : buf + 1, pb = buf == 0 ? 2 : buf - 1;
            half_step<true, LATE ? 2 : 0>(acc, f0, f1, lds, buf, 1, t + 2, pb, 0, 0);
            if (ROLE == 0) {
                u64 miss;
                asm volatile("s_waitcnt vmcnt(6)\n\tv_cmp_lt_u32_e64 %0, v166, %1\n\ts_nop 0" : "=s"(miss) : "v"(need) : "memory");
                WAIT_LGKM0();
                if (miss != 0) spin_ge(ws + cvo + blk_of(t + 3) * CNT_BLK, need, tmo, call);
            } else {
                asm volatile("s_waitcnt vmcnt(6)" ::: "memory");
                WAIT_LGKM0();
            }
            BARRIER();
            half_step<true, LATE ? 0 : (ROLE == 0 ? 1 : 2)>(acc, f1, f0, lds, nb, 0, t + 3, buf, cvo, blk_of(t + 4 < NT ? t + 4 : t + 3) * CNT_BLK);
            buf = nb;
        }
#pragma unroll
        for (int t = NT - 3; t < NT; ++t) {
            const int nb = buf == 2 ? 0 : buf + 1, pb = buf == 0 ? 2 : buf - 1;
            if (t == NT - 3) {
                half_step<true, LATE ? 2 : 0>(acc, f0, f1, lds, buf, 1, t + 2, pb, 0, 0);
                asm volatile("s_waitcnt vmcnt(6)" ::: "memory");
            } else {
                if (t == NT - 2) {
                    const u32 to = OFF_TMO;
                    asm volatile("buffer_load_dword v167, %0, %1, 0 offen sc1" :: "v"(to), "s"(rs) : "memory", "v167");
                }
                half_step<true, 0>(acc, f0, f1, lds, buf, 1, 0, 0, 0, 0);
                asm volatile("s_waitcnt vmcnt(0)" ::: "memory");
            }
            WAIT_LGKM0();
            BARRIER();
            if (t < NT - 1) half_step<true, 0>(acc, f1, f0, lds, nb, 0, 0, 0, 0, 0);
            else            half_step<false, 0>(acc, f1, f0, lds, nb, 0, 0, 0, 0, 0);
            buf = nb;
        }
        u32 tw;
        asm volatile("v_readfirstlane_b32 %0, v167\n\ts_nop 1" : "=s"(tw) :: "memory");
        return tw;
    }
};

__device__ __forceinline__ u32x4 gelu4(const f32x4 (&c)[4], int r, bool bad) {
    const float nanv = __builtin_nanf("");
    f32x4 o = {gelu(c[0][r]), gelu(c[1][r]), gelu(c[2][r]), gelu(c[3][r])};
    if (bad) o = (f32x4){nanv, nanv, nanv, nanv};
    return __builtin_bit_cast(u32x4, o);
}

__device__ __forceinline__ void gemm_role(const Params& p, const LAS char* lds, int xcd, int s, int wid, int lane, u32 call) {
    const int rank = rfl(p.rank);
    const int g = rank >> 2, j = rank & 3, og = (g ^ 1) * 4;
    char* ws = p.ws[rank];
    const u32 par = OFF_PAR + (call & 1) * PAR_BYTES;
    const rsrc_t rs = mk_rsrc(ws, WS_NEED), ro = mk_rsrc(p.out[rank], OUT_BYTES);
    const u32 ldsw = rfl((u32)(size_t)lds);
    const u32 vout = (lane >> 4) * 4 * 4096 + (lane & 15) * 16;
    {
        const int u = (xcd & 1) * 26 + s, rt = u >> 2, ct = (xcd >> 1) * 4 + (u & 3);
        Gemm<1> G;
        G.rs = rs; G.ldsw = ldsw; G.wid = wid; G.lane = lane;
        const int rf = rt * 16 + wid * 2;
        G.sa = par + (rf < FI ? rf : rf + FX) * 4096;
        G.sa2 = par + (rf + 1 < FI ? rf + 1 : rf + 1 + FX) * 4096 - 2048;
        G.sb = par + REL_B + (ct * 8 + wid) * 4096;
        const int bsrc = ct < 8 ? rank : og + ((ct - 8) >> 1);
        u32 cvo = OFF_CNT + 1024 + bsrc * 128, need = (ct < 8 ? 8u : 16u) * call;
        u32 mask = 0;
        for (int i = 0; i < 16; ++i) mask |= 1u << src_of_rf1(rt * 16 + i, g, j);
        if (lane < 8 && ((mask >> lane) & 1)) {
            cvo = OFF_CNT + lane * 128;
            need = (lane == rank ? 8 : ((lane >> 2) == g ? FI1 : FX)) * call;
        }
        f32x4 acc[4][4] = {};
        const bool bad = call == (wid == 0 ? G.template run<0>(acc, lds, ws, cvo, need, call)
                                  : wid < 4 ? G.template run<1>(acc, lds, ws, cvo, need, call)
                                            : G.template run<2>(acc, lds, ws, cvo, need, call));
        STAMP(1);
        const int wm = wid >> 1, wn = wid & 1;
        if (ct < 8) {
#pragma unroll
            for (int m = 0; m < 4; ++m) {
                const u32 ob = (u32)row_of_rf1(rt * 16 + wm * 4 + m, g, j) * 4096 + (ct * 128 + wn * 64) * 4;
#pragma unroll
                for (int r = 0; r < 4; ++r) __builtin_amdgcn_raw_buffer_store_b128(gelu4(acc[m], r, bad), ro, vout + ob + r * 4096, 0, OUT_AUX);
            }
        } else {
            const int fo = (rt * 2 + (ct & 1)) * 8 + wid;
            const u32 sb = par + REL_STG + (u32)(((ct - 8) >> 1) * 1024 + fo * 4) * 4096;
#pragma unroll
            for (int m = 0; m < 4; ++m) {
                const u32x4 o[4] = {gelu4(acc[m], 0, bad), gelu4(acc[m], 1, bad), gelu4(acc[m], 2, bad), gelu4(acc[m], 3, bad)};
                pack_out(o, (LAS char*)lds + wid * 2048, rs, sb + m * 4096, lane);
            }
            WAIT_VM0();
            if (lane == 0) st_flag(p.ws[bsrc] + OFF_RDY + rank * 8192 + (1024 + fo) * 4, call);
        }
        STAMP(2);
    }
    if (s < 24) {
        const int rt = s >> 2, ct = ((~xcd & 1) * 16) + (xcd >> 1) * 4 + (s & 3);
        Gemm<2> G;
        G.rs = rs; G.ldsw = ldsw; G.wid = wid; G.lane = lane;
        G.sa = G.sa2 = par + REL_A2 + (rt * 8 + wid) * 4096;
        G.sb = par + REL_B + ct * 4 * 4096 + wid * 2048;
        const int asrc = g * 4 + ((j + 1 + (rt >> 1)) & 3);
        const int bsrc = ct < 16 ? rank : og + ((ct - 16) >> 2);
        u32 cvo = OFF_CNT + 1024 + bsrc * 128, need = (ct < 16 ? 8u : 16u) * call;
        if (lane == 0) { cvo = OFF_CNT + 2048 + asrc * 128; need = F2 * call; }
        f32x4 acc[1][4] = {};
        const bool bad = call == (wid == 0 ? G.template run<0>(acc, lds, ws, cvo, need, call)
                                  : wid < 4 ? G.template run<1>(acc, lds, ws, cvo, need, call)
                                            : G.template run<2>(acc, lds, ws, cvo, need, call));
        STAMP(3);
        if (ct < 16) {
            const u32 ob = (u32)(asrc * NS + (FI1 + (rt & 1) * 8 + wid) * 16) * 4096 + ct * 256;
#pragma unroll
            for (int r = 0; r < 4; ++r) __builtin_amdgcn_raw_buffer_store_b128(gelu4(acc[0], r, bad), ro, vout + ob + r * 4096, 0, OUT_AUX);
        } else {
            const int z = (rt * 4 + (ct & 3)) * 8 + wid;
            const u32 sb = par + REL_STG + (u32)(((ct - 16) >> 2) * 1024 + 832 + z) * 4096;
            const u32x4 o[4] = {gelu4(acc[0], 0, bad), gelu4(acc[0], 1, bad), gelu4(acc[0], 2, bad), gelu4(acc[0], 3, bad)};
            pack_out(o, (LAS char*)lds + wid * 2048, rs, sb, lane);
            WAIT_VM0();
            if (lane == 0) st_flag(p.ws[bsrc] + OFF_RDY + rank * 8192 + (1024 + 208 + z) * 4, call);
        }
        STAMP(4);
    }
}

__global__ __launch_bounds__(BLOCK, 3) void ag_gemm_kernel(Params p) {
    __shared__ __attribute__((aligned(1024))) char smem[LDS_BYTES];
    const int rank = rfl(p.rank);
    const int lane = threadIdx.x & 63;
    const int wid = rfl(threadIdx.x >> 6);
    const int b = blockIdx.x;
    if (wid >= (b < NCOMM ? NLW + NDEC : 8)) return;
    STAMP(0);
    char* seq = p.ws[rank] + OFF_SEQ + (b * 16 + wid) * 4;
    const u32 prev = ld_agent(seq);
    char* wsl = p.ws[lane & 7];
    auto next_call = [&] {
        const u32 call = rfl(prev) + 1;
        if (lane == 0) __hip_atomic_store((GAS u32*)seq, call, __ATOMIC_RELAXED, __HIP_MEMORY_SCOPE_AGENT);
        STAMP(7);
        return call;
    };
    if (b >= NCOMM) {
        constexpr int B0 = PB > 0 ? PB - 1 : 0;
        if ((u32)(b - NCOMM - B0 * 16) >= EARLY * 16) __builtin_amdgcn_s_sleep(LATE_SLEEP);
        f32x4 v[8];
        load_item(v, p, b - NCOMM, wid, lane);
        const u32 call = next_call();
        convert_local(p, (LAS char*)smem, v, wsl, b - NCOMM, wid, lane, call);
        STAMP(5);
        gemm_role(p, (const LAS char*)smem, b & 7, (b - NCOMM) >> 3, wid, lane, call);
        return;
    }
    const int nld = 2 * NCOMM + b < 7 * WL ? 3 : 2;
    const int sw = wid < NLW + NDEC - 5 ? NPUSH : (NLW + NDEC - 1 - wid) * NCOMM + b;
    f32x4 v[8];
    if (sw < NPUSH) {
        int blk, it;
        push_decode(sw, blk, it);
        load_item(v, p, blk * 16 + (it >> 3), it & 7, lane);
    }
    if (wid < 4) *(LAS u32*)(smem + LDS_CTL + (wid * 64 + lane) * 4) = 0;
    for (int k = 0; k < nld; ++k)
        build_table((LAS char*)smem + LDS_TAB + k * TAB_BYTES, rank, (k * NCOMM + b) % 7, threadIdx.x, (NLW + NDEC) * 64);
    asm volatile("s_waitcnt lgkmcnt(0)\n\ts_barrier" ::: "memory");
    const u32 call = next_call();
    if (sw < NPUSH) push_role(p, v, wsl, sw, lane, call);
    if (wid >= NLW) decode_role(p, (LAS char*)smem, wid - NLW, lane, call);
    else if (wid < nld) {
        const int ps = rfl((wid * NCOMM + b) % 7), wv = rfl((wid * NCOMM + b) / 7);
        if (ps < 3) load_role<false>(p, (LAS char*)smem, wid, nld, ps, wv, lane, call);
        else        load_role<true>(p, (LAS char*)smem, wid, nld, ps, wv, lane, call);
    }
}

}

constexpr int g_mode = 0;

extern "C" void kernel_launch(int rank, int nranks, ncclComm_t comm, hipStream_t stream,
                              void* const* d_in, const int* in_sizes, int n_in,
                              void* d_out, int out_size, const int* in_dtypes, int out_dtype,
                              void* const* d_out_all, void* const* d_ws_all, size_t ws_bytes) {
    (void)comm; (void)in_sizes; (void)n_in; (void)d_out; (void)out_size; (void)in_dtypes; (void)out_dtype;
    if (nranks != NR || (unsigned)rank >= (unsigned)NR || ws_bytes < WS_NEED) {
        fprintf(stderr, "unsupported configuration\n");
        return;
    }
    Params p;
    p.x = (const float*)d_in[0];
    p.w = (const float*)d_in[1];
    for (int i = 0; i < NR; ++i) { p.ws[i] = (char*)d_ws_all[i]; p.out[i] = (char*)d_out_all[i]; }
    p.rank = rank;
    p.pad = g_mode;
    ag_gemm_kernel<<<GRID, BLOCK, 0, stream>>>(p);
    hipError_t e = hipGetLastError();
    if (e != hipSuccess) fprintf(stderr, "launch failed: %s\n", hipGetErrorString(e));
}
```

```cpp
#include <hip/hip_runtime.h>
#include <rccl/rccl.h>
#include <cstdio>
#include <cstdint>
#include <type_traits>

typedef unsigned u32;
typedef unsigned long long u64;
typedef u32 u32x4 __attribute__((ext_vector_type(4)));
typedef u32 u32x2v __attribute__((ext_vector_type(2)));
typedef float f32x4 __attribute__((ext_vector_type(4)));
typedef _Float16 f16x8 __attribute__((ext_vector_type(8)));
typedef __amdgpu_buffer_rsrc_t rsrc_t;
#define LAS __attribute__((address_space(3)))
#define GAS __attribute__((address_space(1)))

namespace {

constexpr int NR = 8;
constexpr int KD = 8192;
constexpr int NS = 1024;
constexpr int NBLK = 64;
#define P_FI 45
constexpr int FI = P_FI;
constexpr int F2 = 16;
constexpr int FI1 = FI - F2;
constexpr int FX = 64 - FI;
constexpr int FL = 8;
constexpr int RF1 = 208, RF2 = 48, CF = 128;
static_assert(FI + 3 * FI1 + 4 * FX == RF1, "stage-1 rows");

#define P_W 19
#define P_XA 14
#define P_EARLY 4
#define P_SLEEP 100
#define P_PF 1
#define P_PB 1
constexpr int PB = P_PB;
constexpr int EARLY = P_EARLY, LATE_SLEEP = P_SLEEP, PF_FROM = P_PF;
constexpr int NCOMM = 48;
constexpr int WL = P_W;
constexpr int XA = P_XA;
static_assert(WL >= 14 && WL <= 20 && XA <= FX && XA <= 16, "wave roles");
constexpr int GRID = 256, BLOCK = 768, NGEMM = GRID - NCOMM;
constexpr int RS1 = 64 + 3 * FI1 + 4 * FX;

constexpr u32 OFF_SEQ  = 0;
constexpr u32 OFF_TMO  = 16384;
constexpr u32 OFF_QUE  = 16640;
constexpr u32 OFF_RDY  = 32768;
constexpr u32 OFF_CNT  = 131072;
constexpr u32 CNT_BLK  = 24 * 128;
constexpr u32 OFF_DIAG = 524288;
constexpr u32 A1_BYTES = (u32)NBLK * RS1 * 4096;
constexpr u32 A2_BYTES = (u32)NBLK * RF2 * 4096;
constexpr u32 B_BYTES  = (u32)NBLK * CF * 4096;
constexpr u32 STG_BYTES = 4u * 1024 * 4096;
constexpr u32 PK_BYTES = (u32)NBLK * 128 * 4096;
constexpr u32 REL_A2 = A1_BYTES, REL_B = REL_A2 + A2_BYTES, REL_PK = REL_B + B_BYTES, REL_STG = REL_PK + PK_BYTES;
constexpr u32 PAR_BYTES = REL_STG + 2 * STG_BYTES;
constexpr u32 OFF_PAR  = 1u << 20;
constexpr int VAR_XO = 192;
constexpr u32 WS_NEED  = OFF_PAR + 2 * PAR_BYTES;
constexpr u32 OUT_BYTES = (u32)KD * NS * 4;
static_assert(REL_PK < (256u << 20), "table entries hold 18 bits of dst / 1024");
static_assert(256 + VAR_XO == 448, "output items are as long as x items");
static_assert(PK_BYTES == 8192u * 4096u, "staged outputs follow the packed image");
static_assert(OFF_CNT + NBLK * CNT_BLK <= OFF_DIAG && OFF_RDY + 8 * 8192 <= OFF_CNT, "control area");

constexpr u64 DEADLINE = 40000000ull;

constexpr int BUF_BYTES = 49152;
constexpr int LDS_BYTES = 3 * BUF_BYTES;

struct Params {
    const float* x;
    const float* w;
    char* ws[NR];
    char* out[NR];
    int rank;
    int pad;
};

__device__ __forceinline__ u32 rfl(u32 v) { return __builtin_amdgcn_readfirstlane(v); }
__device__ __forceinline__ char* uni(const void* p) {
    const u64 a = (u64)p;
    return (char*)(((u64)rfl((u32)(a >> 32)) << 32) | rfl((u32)a));
}
__device__ __forceinline__ rsrc_t rsrc_u(const void* p, u32 bytes) {
    return __builtin_amdgcn_make_buffer_rsrc((void*)p, (short)0, (int)bytes, 0x00020000);
}
__device__ __forceinline__ rsrc_t mk_rsrc(const void* p, u32 bytes) { return rsrc_u(uni(p), bytes); }
__device__ __forceinline__ u32 ld_sys(const char* p) {
    return __hip_atomic_load((const GAS u32*)p, __ATOMIC_RELAXED, __HIP_MEMORY_SCOPE_SYSTEM);
}
__device__ __forceinline__ u32 ld_agent(const char* p) {
    return __hip_atomic_load((const GAS u32*)p, __ATOMIC_RELAXED, __HIP_MEMORY_SCOPE_AGENT);
}
__device__ __forceinline__ void st_flag(char* p, u32 v) {
    __hip_atomic_store((GAS u32*)p, v, __ATOMIC_RELAXED, __HIP_MEMORY_SCOPE_SYSTEM);
}
__device__ __forceinline__ void add_sys(char* p) {
    __hip_atomic_fetch_add((GAS u32*)p, 1u, __ATOMIC_RELAXED, __HIP_MEMORY_SCOPE_SYSTEM);
}
#define WAIT_VM0() do { asm volatile("s_waitcnt vmcnt(0)" ::: "memory"); __builtin_amdgcn_s_waitcnt(0x0F70); } while (0)

template <int SLEEP = 2>
__device__ __forceinline__ bool spin_ge(const char* p, u32 need, char* tmo, u32 call) {
    const u64 t0 = __builtin_amdgcn_s_memrealtime();
    bool gave_up = false;
    for (u32 polls = 0;; ++polls) {
        const u32 v = ld_sys(p);
        if (__builtin_amdgcn_ballot_w64(v < need) == 0) break;
        if ((polls & 7) == 0 && (rfl(ld_agent(tmo)) == call || __builtin_amdgcn_s_memrealtime() - t0 > DEADLINE)) {
            __hip_atomic_store((GAS u32*)tmo, call, __ATOMIC_RELAXED, __HIP_MEMORY_SCOPE_AGENT);
            gave_up = true;
            break;
        }
        __builtin_amdgcn_s_sleep(SLEEP);
    }
    asm volatile("" ::: "memory");
    return gave_up;
}

__device__ __forceinline__ int dwave() { int w = threadIdx.x >> 6; return w < 8 ? w : (blockIdx.x < NCOMM ? w - 5 : 7); }
#define TICK(v) do {} while (0)
#define ACC(a, x) do {} while (0)
#define STAMP_IF(c, slot) do {} while (0)
#define STAMP(slot) do {} while (0)
#define STAMPB_IF(c, slot) do {} while (0)

__device__ __forceinline__ u32x4 cvt8(f32x4 a, f32x4 b) {
    f16x8 h;
    h[0] = (_Float16)a[0]; h[1] = (_Float16)a[1]; h[2] = (_Float16)a[2]; h[3] = (_Float16)a[3];
    h[4] = (_Float16)b[0]; h[5] = (_Float16)b[1]; h[6] = (_Float16)b[2]; h[7] = (_Float16)b[3];
    return __builtin_bit_cast(u32x4, h);
}
__device__ __forceinline__ void st_sys(u32x4 v, rsrc_t r, u32 voff, u32 soff) {
    __builtin_amdgcn_raw_buffer_store_b128(v, r, voff + soff, 0, 17);
}
__device__ __forceinline__ float gelu(float y) {
    const float u = y * (0.7978845608028654f + 0.7978845608028654f * 0.044715f * y * y);
    const float e = __builtin_amdgcn_exp2f(u * -2.8853900817779268f);
    return y * __builtin_amdgcn_rcpf(1.0f + e);
}

__device__ __forceinline__ int row_of_rf1(int rf, int g, int j) {
    if (rf < FI) return (g * 4 + j) * NS + rf * 16;
    rf -= FI;
    if (rf < 3 * FI1) { const int i = rf / FI1; return (g * 4 + ((j + 1 + i) & 3)) * NS + (rf - i * FI1) * 16; }
    rf -= 3 * FI1;
    const int q = rf / FX;
    return ((g ^ 1) * 4 + q) * NS + (FI + rf - q * FX) * 16;
}
__device__ __forceinline__ int src_of_rf1(int rf, int g, int j) { return row_of_rf1(rf, g, j) / NS; }

constexpr int IN_AUX = 2;
#define P_OUT_AUX 17
constexpr int OUT_AUX = P_OUT_AUX;
__device__ __forceinline__ u32x4 cvt_w(const f32x4* v, int c) {
    return cvt8((f32x4){v[0][c], v[1][c], v[2][c], v[3][c]}, (f32x4){v[4][c], v[5][c], v[6][c], v[7][c]});
}

#define DPP_STEP(op, v, ctrl, rmask) v = op(v, (u32)__builtin_amdgcn_update_dpp(0, (int)v, ctrl, rmask, 0xf, false))
__device__ __forceinline__ u32 umax2(u32 a, u32 b) { return a > b ? a : b; }
__device__ __forceinline__ u32 uadd2(u32 a, u32 b) { return a + b; }
__device__ __forceinline__ u32 wave_max(u32 v) {
    DPP_STEP(umax2, v, 0x111, 0xf); DPP_STEP(umax2, v, 0x112, 0xf); DPP_STEP(umax2, v, 0x114, 0xf); DPP_STEP(umax2, v, 0x118, 0xf);
    DPP_STEP(umax2, v, 0x142, 0xa); DPP_STEP(umax2, v, 0x143, 0xc);
    return __builtin_amdgcn_readlane(v, 63);
}
__device__ __forceinline__ u32 wave_sum(u32 v) {
    DPP_STEP(uadd2, v, 0x111, 0xf); DPP_STEP(uadd2, v, 0x112, 0xf); DPP_STEP(uadd2, v, 0x114, 0xf); DPP_STEP(uadd2, v, 0x118, 0xf);
    DPP_STEP(uadd2, v, 0x142, 0xa); DPP_STEP(uadd2, v, 0x143, 0xc);
    return __builtin_amdgcn_readlane(v, 63);
}

__device__ __forceinline__ u32 wave_scan(u32 v) {
    DPP_STEP(uadd2, v, 0x111, 0xf); DPP_STEP(uadd2, v, 0x112, 0xf); DPP_STEP(uadd2, v, 0x114, 0xf); DPP_STEP(uadd2, v, 0x118, 0xf);
    DPP_STEP(uadd2, v, 0x142, 0xa); DPP_STEP(uadd2, v, 0x143, 0xc);
    return v;
}
__device__ __forceinline__ u32 byte_scan(u32 m) {
    u32 a, r;
    asm("v_lshl_add_u32 %0, %1, 8, %1" : "=v"(a) : "v"(m));
    asm("v_lshl_add_u32 %0, %1, 16, %1" : "=v"(r) : "v"(a));
    return r;
}
__device__ __forceinline__ u32 times7(u32 m) {
    u32 a, r;
    asm("v_lshl_add_u32 %0, %1, 1, %1" : "=v"(a) : "v"(m));
    asm("v_lshl_add_u32 %0, %1, 2, %2" : "=v"(r) : "v"(m), "v"(a));
    return r;
}
__device__ __forceinline__ u32 rank_in(u64 m) { return __builtin_amdgcn_mbcnt_hi((u32)(m >> 32), __builtin_amdgcn_mbcnt_lo((u32)m, 0)); }

constexpr int VAR_X = VAR_XO, VAR_W = 256;
__device__ __forceinline__ u32 h7_of(float a) { return (u32)__builtin_bit_cast(unsigned short, (_Float16)a) >> 8; }
__device__ __forceinline__ void lds_or(LAS char* a, u32 v) {
    __hip_atomic_fetch_or((LAS u32*)a, v, __ATOMIC_RELAXED, __HIP_MEMORY_SCOPE_WORKGROUP);
}
__device__ __forceinline__ void put_bits(LAS char* area, u32 bo, u64 v) {
    const u32 sh = bo & 31;
    const u64 w = v << sh;
    LAS char* a = area + (bo >> 5) * 4;
    lds_or(a, (u32)w); lds_or(a + 4, (u32)(w >> 32)); lds_or(a + 8, (u32)((v >> 1) >> (63 - sh)));
}
__device__ __forceinline__ u64 get_bits(const LAS char* area, u32 bo) {
    const LAS u32* w = (const LAS u32*)(area + (bo >> 5) * 4);
    const u32 w0 = w[0], w1 = w[1], w2 = w[2];
    return (u64)__builtin_amdgcn_alignbit(w2, w1, bo) << 32 | __builtin_amdgcn_alignbit(w1, w0, bo);
}
template <class F>
__device__ __forceinline__ void pack_item(const u32x4 (&h)[4], float amax, u32 var, LAS char* scr, rsrc_t r0, u32 dst, int lane, F before_stores) {
    u32 L[8], H[8];
#pragma unroll
    for (int t = 0; t < 8; ++t) {
        const u32 d0 = h[t >> 1][(t & 1) * 2], d1 = h[t >> 1][(t & 1) * 2 + 1];
        L[t] = __builtin_amdgcn_perm(d1, d0, 0x06040200u);
        H[t] = __builtin_amdgcn_perm(d1, d0, 0x07050301u);
    }
    *(LAS u32x4*)(scr + lane * 16) = (u32x4){H[0], H[1], H[2], H[3]};
    *(LAS u32x4*)(scr + lane * 16 + 1024) = (u32x4){H[4], H[5], H[6], H[7]};
    LAS char* area = scr + 2048;
    *(LAS u32x2v*)(area + lane * 8) = (u32x2v){0u, 0u};
    u32 E1 = (wave_sum(h7_of(amax) & 127) + 122) >> 8;
    E1 = E1 < 31 ? E1 : 31;
    const u32 eb = E1 * 0x01010101u | 0x80808080u;
    u32 K[2] = {0, 0}, e4 = 0, pos = 0;
    u64 b2 = 0, b3 = 0;
#pragma unroll
    for (int t = 0; t < 8; ++t) {
        const u32 o = (eb - ((H[t] >> 2) & 0x1f1f1f1fu)) ^ 0x80808080u, o7 = o & 0x7f7f7f7fu;
        const u32 big = (o | (o7 + 0x7a7a7a7au)) & 0x80808080u, huge = (o | (o7 + 0x73737373u)) & 0x80808080u;
        const u32 M2 = big >> 7, m7 = (big >> 4) - M2;
        const u32 om = (o & ~m7) | (m7 & 0x06060606u);
        const u32 sel = om & 0x07070707u;
        const u32 c = __builtin_amdgcn_perm(0x00030303u, 0x02010003u, sel);
        const u32 q = __builtin_amdgcn_perm(0x00030100u, 0x00000002u, sel);
        const u32 M = __builtin_amdgcn_perm(0x00010101u, 0x00000001u, sel);
        const u32 v = ((o7 + 0x02020202u) | ((huge >> 4) - (huge >> 7))) & m7;
        const u32 P = byte_scan(M), P2 = byte_scan(M2), X2 = P2 - M2;
        const u32 bits2 = __builtin_amdgcn_udot4(q, __builtin_amdgcn_perm(0u, 0x40100401u, P - M), 0u, false);
        const u32 bits3 = __builtin_amdgcn_udot4(v, __builtin_amdgcn_perm(0u, 0x00400801u, X2), (X2 >> 24) == 3 ? (v >> 24) << 9 : 0u, false);
        b2 |= (u64)bits2 << (pos & 63);
        b3 |= (u64)bits3 << ((pos >> 16) & 63);
        pos += (P >> 23) + ((P2 >> 24) * 3 << 16);
        K[t >> 2] |= c << (2 * (t & 3));
        e4 = (e4 >> 1) | huge;
        if (t & 1) __builtin_amdgcn_sched_barrier(0);
    }
    const u32 incl = wave_scan(pos), tot = __builtin_amdgcn_readlane(incl, 63), ex = incl - pos;
    const u32 used = 8 + (tot & 0xffff) + (tot >> 16);
    u32 n4 = 0;
    bool fits = used <= 8 * var && __builtin_amdgcn_ballot_w64(pos >= (65u << 16)) == 0;
    if (fits) {
        put_bits(area, 8 + (ex & 0xffff), b2);
        put_bits(area, 8 + (tot & 0xffff) + (ex >> 16), b3);
        while (__builtin_amdgcn_ballot_w64(e4 != 0) != 0) {
            const u64 act = __builtin_amdgcn_ballot_w64(e4 != 0);
            const u32 any = (e4 | (e4 >> 8) | (e4 >> 16) | (e4 >> 24)) & 255, t = __builtin_ctz(any | 128);
            const u32 b = __builtin_ctz(((e4 >> t) & 0x01010101u) | 0x80000000u) + t;
            const u32 e = (*(const LAS unsigned char*)(scr + (t & 4) * 256 + lane * 16 + (t & 3) * 4 + ((b >> 3) & 3)) >> 2) & 31;
            const u32 k = var - 1 - n4 - rank_in(act);
            if (e4 != 0 && k < var) lds_or(area + (k & ~3u), e << ((k & 3) * 8));
            e4 &= ~(1u << (b & 31));
            n4 += __builtin_popcountll(act);
            if (used + 8 * n4 > 8 * var) { fits = false; break; }
        }
    }
    asm volatile("" ::: "memory");
    u32 vw = *(const LAS u32*)(area + lane * 4);
    if (lane == 0) vw |= fits ? E1 : 255;
    const u32 P0 = (H[0] & 0x83838383u) | ((H[1] & 0x03030303u) << 2) | ((H[1] >> 1) & 0x40404040u) | ((H[6] & 0x03030303u) << 4);
    const u32 P1 = (H[2] & 0x83838383u) | ((H[3] & 0x03030303u) << 2) | ((H[3] >> 1) & 0x40404040u) | ((H[7] & 0x03030303u) << 4);
    const u32 P2 = (H[4] & 0x83838383u) | ((H[5] & 0x03030303u) << 2) | ((H[5] >> 1) & 0x40404040u)
                 | ((H[6] >> 2) & 0x20202020u) | ((H[7] >> 3) & 0x10101010u);
    before_stores(fits);
    st_sys((u32x4){L[0], L[1], L[2], L[3]}, r0, lane * 16, dst);
    st_sys((u32x4){L[4], L[5], L[6], L[7]}, r0, lane * 16, dst + 1024);
    st_sys((u32x4){P0, P1, P2, K[0]}, r0, lane * 16, dst + 2048);
    __builtin_amdgcn_raw_buffer_store_b32(K[1], r0, lane * 4 + dst + 3072, 0, 17);
    __builtin_amdgcn_raw_buffer_store_b32(vw, r0, lane * 4 + dst + 3328, 0, 17);
}
__device__ __forceinline__ void unpack_item(u32x4 (&h)[4], const u32x4& c0, const u32x4& c1, const u32x4& c2, u32 k1,
                                            const LAS char* area, u32 E1, u32 var, LAS char* scr, int lane) {
    const u32 K[2] = {c2[3], k1};
    const u32 X[2] = {K[0] & (K[0] >> 1) & 0x55555555u, K[1] & (K[1] >> 1) & 0x55555555u};
    const u32 n2 = 2 * (__builtin_popcount(X[0]) + __builtin_popcount(X[1])), i2 = wave_scan(n2);
    u64 b2 = get_bits(area, 8 + i2 - n2);
    b2 &= n2 < 64 ? ~(~0ull << n2) : ~0ull;
    const u64 y2 = b2 & (b2 >> 1) & 0x5555555555555555ull;
    const u32 n3 = 3 * __builtin_popcountll(y2), i3 = wave_scan(n3);
    u64 b3 = get_bits(area, 8 + __builtin_amdgcn_readlane(i2, 63) + i3 - n3);
    const u32 e4x = E1 * 0x04040404u | 0x80808080u;
    const u32 t1 = (e4x - 0x000c0804u) & 0x007c7c7cu, t2 = (e4x - 0x00001410u) & 0x007c7c7cu;
    const u32 t3lo = (e4x - 0x24201c18u) & 0x7c7c7c7cu, t3hi = (e4x - 0x00302c28u) & 0x007c7c7cu;
    LAS char* row = scr + lane * 36;
    u32 e4 = 0;
    const u32 nch = (wave_max(n2) + 7) >> 3;
#pragma unroll
    for (int k = 0; k < 8; ++k) {
        if (k >= 4 && k >= nch) break;
        const u32 b8 = (u32)(b2 >> (8 * k)) & 255, y = b8 | (b8 << 12), q = (y | (y << 6)) & 0x03030303u;
        const u32 M2 = q & (q >> 1), P2 = byte_scan(M2), m7 = times7(M2);
        const u32 lo = (u32)b3, z = (lo & 0x3f) | ((lo & 0xfc0) << 10), f = (z | (z << 5)) & 0x07070707u;
        const u32 v = __builtin_amdgcn_perm(0u, f, P2 - M2) & m7;
        b3 >>= (P2 >> 24) * 3;
        *(LAS u32*)(row + 4 * k) = __builtin_amdgcn_perm(0u, t2, q) | __builtin_amdgcn_perm(t3hi, t3lo, v | (0x0c0c0c0cu - (M2 << 2) - (M2 << 3)));
        e4 |= (((v + M2) >> 3) & 0x01010101u) << k;
    }
    u32 n4 = 0;
    while (__builtin_amdgcn_ballot_w64(e4 != 0) != 0) {
        const u64 act = __builtin_amdgcn_ballot_w64(e4 != 0);
        const u32 any = (e4 | (e4 >> 8) | (e4 >> 16) | (e4 >> 24)) & 255, t = __builtin_ctz(any | 128);
        const u32 b = __builtin_ctz(((e4 >> t) & 0x01010101u) | 0x80000000u) + t;
        const u32 x = *(const LAS unsigned char*)(area + ((var - 1 - n4 - rank_in(act)) & 255));
        if (e4 != 0) *(LAS unsigned char*)(row + t * 4 + (b >> 3)) = x << 2;
        e4 &= ~(1u << (b & 31));
        n4 += __builtin_popcountll(act);
    }
    const u32 SM[8] = {c2[0] & 0x83838383u, ((c2[0] << 1) & 0x80808080u) | ((c2[0] >> 2) & 0x03030303u),
                       c2[1] & 0x83838383u, ((c2[1] << 1) & 0x80808080u) | ((c2[1] >> 2) & 0x03030303u),
                       c2[2] & 0x83838383u, ((c2[2] << 1) & 0x80808080u) | ((c2[2] >> 2) & 0x03030303u),
                       ((c2[2] << 2) & 0x80808080u) | ((c2[0] >> 4) & 0x03030303u),
                       ((c2[2] << 3) & 0x80808080u) | ((c2[1] >> 4) & 0x03030303u)};
    u32 M[8], P[8], g[8], at = 0;
#pragma unroll
    for (int t = 0; t < 8; ++t) {
        M[t] = (X[t >> 2] >> (2 * (t & 3))) & 0x01010101u;
        P[t] = byte_scan(M[t]);
        const LAS u32* w = (const LAS u32*)(row + (at & ~3u));
        g[t] = __builtin_amdgcn_alignbyte(w[1], w[0], at);
        at += P[t] >> 24;
    }
#pragma unroll
    for (int t = 0; t < 8; ++t) {
        const u32 c = (K[t >> 2] >> (2 * (t & 3))) & 0x03030303u;
        const u32 m = times7(M[t]);
        const u32 H = SM[t] | __builtin_amdgcn_perm(t1, g[t], ((P[t] - M[t]) & m) | ((c | 0x04040404u) & ~m));
        const u32 L = t < 4 ? c0[t & 3] : c1[t & 3];
        h[t >> 1][(t & 1) * 2] = __builtin_amdgcn_perm(H, L, 0x05010400u);
        h[t >> 1][(t & 1) * 2 + 1] = __builtin_amdgcn_perm(H, L, 0x07030602u);
    }
}

__device__ __forceinline__ u32 sel_lo(u32 sg, u32 lop, u32 lon) {
    u32 m = sg << 8;
    asm("" : "+v"(m));
    m -= sg;
    return (lon * 0x01010101u & m) | (lop * 0x01010101u & ~m);
}
template <int N>
__device__ __forceinline__ void out_windows(const u32x4 (&o)[N][4], u32& lop, u32& lon) {
    u32 mn = 0, ntop = 0;
    int mp = 0;
#pragma unroll
    for (int i = 0; i < 16 * N; ++i) { mp = max(mp, (int)o[i >> 4][(i >> 2) & 3][i & 3]); mn = umax2(mn, o[i >> 4][(i >> 2) & 3][i & 3]); }
    u32 ep = wave_max((u32)mp) >> 23;
    const u32 en = (wave_max(mn) >> 23) & 255;
    ntop = __builtin_popcountll(__builtin_amdgcn_ballot_w64(mp >= (int)(ep << 23)));
    ep -= ntop <= 8 && ep > 0;
    lop = ep > 7 ? ep - 7 : 0; lon = en > 7 ? en - 7 : 0;
}
__device__ __forceinline__ void pack_out(const u32x4 (&o)[4], u32 lop, u32 lon, LAS char* scr, rsrc_t r0, u32 dst, int lane) {
    u32 M[12], S[4];
    u32x4 E;
#pragma unroll
    for (int r = 0; r < 4; ++r) {
        u32 R[4];
#pragma unroll
        for (int c = 0; c < 4; ++c) R[c] = __builtin_amdgcn_alignbit(o[r][c], o[r][c], 31);
#pragma unroll
        for (int i = 0; i < 3; ++i) M[r * 3 + i] = __builtin_amdgcn_perm(R[3], R[i], 0x04020100u + (i << 24));
        E[r] = __builtin_amdgcn_perm(R[1], R[0], 0x0c0c0703u) | __builtin_amdgcn_perm(R[3], R[2], 0x07030c0cu);
        S[r] = (__builtin_amdgcn_perm(R[1], R[0], 0x0c0c0400u) | __builtin_amdgcn_perm(R[3], R[2], 0x04000c0cu)) & 0x01010101u;
    }
    *(LAS u32x4*)(scr + lane * 16) = E;
    u32 G2 = 0, G1 = 0, em = 0;
#pragma unroll
    for (int r = 0; r < 4; ++r) {
        const u32 lo = sel_lo(S[r], lop, lon);
        const u32 d = ((E[r] | 0x80808080u) - (lo & 0x7f7f7f7fu)) ^ ((E[r] ^ ~lo) & 0x80808080u);
        const u32 nz = (d | ((d & 0x78787878u) + 0x78787878u)) & 0x80808080u;
        const u32 g = d & 0x07070707u & ~((nz >> 4) - (nz >> 7));
        G2 |= (g & 0x03030303u) << (2 * r);
        G1 |= (g & 0x04040404u) >> 2 << r;
        em = (em >> 1) | (~(g + 0x7f7f7f7fu) & 0x80808080u);
    }
    LAS unsigned char* list = (LAS unsigned char*)(scr + 1024);
    u32 used = 2;
    for (u64 act = __builtin_amdgcn_ballot_w64(em != 0); act != 0; act = __builtin_amdgcn_ballot_w64(em != 0)) {
        const u32 k = used + rank_in(act), b = __builtin_ctz(em | 0x80000000u);
        list[em != 0 && k < 64 ? k : 64] = *(const LAS unsigned char*)(scr + lane * 16 + (b & 3) * 4 + (b >> 3));
        em &= em - 1;
        used += __builtin_popcountll(act);
        if (used > 64) break;
    }
    *(LAS unsigned short*)list = (used > 64 ? 255 : lop) | lon << 8;
    asm volatile("" ::: "memory");
    const u32 ent = *(const LAS u32*)(list + (lane & 15) * 4);
    if (used > 64) {
#pragma unroll
        for (int r = 0; r < 4; ++r) st_sys(o[r], r0, lane * 16 + r * 1024, dst + STG_BYTES);
    }
#pragma unroll
    for (int i = 0; i < 3; ++i) st_sys((u32x4){M[4 * i], M[4 * i + 1], M[4 * i + 2], M[4 * i + 3]}, r0, lane * 16 + i * 1024, dst);
    __builtin_amdgcn_raw_buffer_store_b32(G2, r0, lane * 4 + dst + 3072, 0, 17);
    __builtin_amdgcn_raw_buffer_store_b16((unsigned short)__builtin_amdgcn_perm(0u, G1 | (G1 >> 4), 0x0c0c0200u), r0, lane * 2 + dst + 3328, 0, 17);
    __builtin_amdgcn_raw_buffer_store_b32(ent, r0, (lane < 16 ? lane * 4 : 0x80000000u) + dst + 3456, 0, 17);
}
__device__ __forceinline__ void unpack_out(u32x4 (&o)[4], const u32x4& c0, const u32x4& c1, const u32x4& c2, u32 G2, u32 g1,
                                           const LAS unsigned char* list, u32 lop, u32 lon, LAS char* scr, int lane) {
    const u32 M[12] = {c0[0], c0[1], c0[2], c0[3], c1[0], c1[1], c1[2], c1[3], c2[0], c2[1], c2[2], c2[3]};
    const u32 t = __builtin_amdgcn_perm(0u, g1, 0x0c010c00u), G1 = t | (t << 4);
    u32x4 E;
    u32 em = 0;
#pragma unroll
    for (int r = 0; r < 4; ++r) {
        const u32 sg = (__builtin_amdgcn_perm(M[r * 3 + 1], M[r * 3], 0x030c0400u) | __builtin_amdgcn_perm(0u, M[r * 3 + 2], 0x0c000c0cu)) & 0x01010101u;
        const u32 g = ((G2 >> (2 * r)) & 0x03030303u) | ((G1 >> r << 2) & 0x04040404u);
        const u32 nz = (g + 0x7f7f7f7fu) & 0x80808080u;
        E[r] = (sel_lo(sg, lop, lon) + g) & ((nz << 1) - (nz >> 7));
        em = (em >> 1) | (nz ^ 0x80808080u);
    }
    *(LAS u32x4*)(scr + lane * 16) = E;
    u32 used = 2;
    auto step = [&](u32& a, u32& val) {
        const u64 act = __builtin_amdgcn_ballot_w64(em != 0);
        const u32 b = __builtin_ctz(em | 0x80000000u);
        a = em != 0 ? lane * 16 + (b & 3) * 4 + (b >> 3) : 1024;
        val = list[(used + rank_in(act)) & 63];
        em &= em - 1;
        used += __builtin_popcountll(act);
    };
    do {
        u32 a0, a1, a2, x0, x1, x2;
        step(a0, x0); step(a1, x1); step(a2, x2);
        *(LAS unsigned char*)(scr + a0) = x0; *(LAS unsigned char*)(scr + a1) = x1; *(LAS unsigned char*)(scr + a2) = x2;
    } while (__builtin_amdgcn_ballot_w64(em != 0) != 0);
    E = *(const LAS u32x4*)(scr + lane * 16);
#pragma unroll
    for (int r = 0; r < 4; ++r) {
        u32 R[4];
#pragma unroll
        for (int i = 0; i < 3; ++i) R[i] = __builtin_amdgcn_perm(E[r], M[r * 3 + i], 0x04020100u + (i << 24));
        R[3] = __builtin_amdgcn_perm(M[r * 3 + 1], M[r * 3], 0x0c0c0703u) | __builtin_amdgcn_perm(E[r], M[r * 3 + 2], 0x07030c0cu);
#pragma unroll
        for (int c = 0; c < 4; ++c) o[r][c] = __builtin_amdgcn_alignbit(R[c], R[c], 1);
    }
}

__device__ __forceinline__ void load_item(f32x4 (&v)[8], const Params& p, int u, int wid, int lane) {
    const int blk = u >> 4, it = (u & 15) * 8 + wid;
    const bool isx = it < 64;
    const rsrc_t r = mk_rsrc(isx ? p.x : p.w, OUT_BYTES);
    const u32 vf = isx ? (lane & 15) * 32768 + (lane >> 4) * 16 : (lane >> 4) * 16384 + (lane & 15) * 16;
    const u32 base = isx ? it * 524288 + blk * 512 : (blk * 128 + (wid & 3) * 32) * 4096 + ((it - 64) >> 2) * 256;
    const u32 sa = isx ? 64 : 4096, sb = isx ? 256 : 65536;
#pragma unroll
    for (int i = 0; i < 8; ++i)
        v[i] = __builtin_bit_cast(f32x4, __builtin_amdgcn_raw_buffer_load_b128(r, vf, base + (i & 3) * sa + (i >> 2) * sb, IN_AUX));
}
__device__ __forceinline__ void convert_local(const Params& p, LAS char* lds, f32x4 (&v)[8], char* wsl, int wg, int wid, int lane, u32 call) {
    const int rank = rfl(p.rank);
    constexpr int NU = NBLK * 16;
    const u32 par = OFF_PAR + (call & 1) * PAR_BYTES;
    const rsrc_t r0 = mk_rsrc(p.ws[rank], WS_NEED);
    LAS char* scr = lds + wid * 4096;
    auto publish = [&](int u) {
        if (wid == 0) {
            if (lane < 8 && lane != rank) st_flag(wsl + OFF_RDY + rank * 8192 + u * 4, call);
            if (lane == 8) add_sys(p.ws[rank] + OFF_CNT + (u >> 4) * CNT_BLK + ((u & 8) ? 1024 : 0) + rank * 128);
        }
    };
    int u = wg, un = wg + NGEMM, up = -1;
    WAIT_VM0();
    for (int r = 0; u < NU; ++r) {
        const int blk = u >> 4, grp = u & 15, it = grp * 8 + wid;
        u32x4 h[4];
        u32 base, step;
        if (grp < 8) {
            base = par + (u32)blk * (RS1 * 4096) + it * 4096; step = 1024;
#pragma unroll
            for (int i = 0; i < 4; ++i) h[i] = cvt8(v[2 * i], v[2 * i + 1]);
        } else {
            base = par + REL_B + (u32)blk * (CF * 4096) + ((it - 64) & 60) * 4096 + (wid & 3) * 1024; step = 4096;
#pragma unroll
            for (int c = 0; c < 4; ++c) h[c] = cvt_w(v, c);
        }
        float amax = 0.0f;
#pragma unroll
        for (int i = 0; i < 8; ++i) amax = fmaxf(fmaxf(fmaxf(amax, fabsf(v[i][0])), fmaxf(fabsf(v[i][1]), fabsf(v[i][2]))), fabsf(v[i][3]));
        STAMPB_IF(r == 0, 0);
        if (r > 0) load_item(v, p, (u32)un < (u32)NU ? un : NU - 1, wid, lane);
        pack_item(h, amax, grp < 8 ? VAR_X : VAR_W, scr, r0, par + REL_PK + (u32)(blk * 128 + it) * 4096, lane, [&](bool fits) {
            WAIT_VM0();
            if (r > 0) {
                __builtin_amdgcn_s_barrier();
                if (up >= 0) publish(up);
            }
            if (!fits || it < FI + FL || it >= 64) {
#pragma unroll
                for (int i = 0; i < 4; ++i) st_sys(h[i], r0, lane * 16, base + i * step);
            }
        });
        up = u;
        if (r == 0) {
            STAMPB_IF(true, 1);
            WAIT_VM0();
            STAMPB_IF(true, 2);
            __builtin_amdgcn_s_barrier();
            STAMP(6);
            publish(u);
            up = -1;
            load_item(v, p, un, wid, lane);
            WAIT_VM0();
        }
        u = un;
        un += NGEMM;
    }
    WAIT_VM0();
    __builtin_amdgcn_s_barrier();
    if (up >= 0) publish(up);
}

constexpr int PUSH_PER = FI1 + 64 + XA, NPUSH = PB * PUSH_PER, NSPARE = NCOMM * 5;
__device__ __forceinline__ void push_decode(int n, int& blk, int& it) {
    blk = n / PUSH_PER; it = n - blk * PUSH_PER;
    if (it >= FI1 + 64) { const int xi = blk * XA + it - FI1 - 64; blk = xi / FX; it = FI + xi - blk * FX; }
    else if (it >= FI1) it += 64 - FI1;
}
__device__ __forceinline__ void push_role(const Params& p, f32x4 (&v)[8], char* wsl, int sw, int lane, u32 call) {
    const int rank = rfl(p.rank);
    const int g = rank >> 2, j = rank & 3, og = (g ^ 1) * 4;
    const u32 par = OFF_PAR + (call & 1) * PAR_BYTES;
    for (int n = sw; n < NPUSH; n += NSPARE) {
        int blk, it;
        push_decode(n, blk, it);
        u32x4 h[4];
        int d0 = 0, nd = 4, dstep = 1; u32 base, step = 1024, ostep = 0;
        if (it < 64) {
#pragma unroll
            for (int i = 0; i < 4; ++i) h[i] = cvt8(v[2 * i], v[2 * i + 1]);
            if (it < FI1) { d0 = j + 3; nd = 3; dstep = 3; ostep = FI1 * 4096; base = par + (blk * RS1 + 64 + it) * 4096; }
            else base = par + (blk * RS1 + 64 + 3 * FI1 + j * FX + it - FI) * 4096;
        } else {
#pragma unroll
            for (int c = 0; c < 4; ++c) h[c] = cvt_w(v, c);
            d0 = (it - 64) >> 4; nd = 1; step = 4096;
            base = par + REL_B + (blk * CF + 64 + 16 * j + (it & 12)) * 4096 + (it & 3) * 1024;
        }
        if (n + NSPARE < NPUSH) { int b2, i2; push_decode(n + NSPARE, b2, i2); load_item(v, p, b2 * 16 + (i2 >> 3), i2 & 7, lane); }
        u32 dmask = 0;
        for (int d = 0; d < nd; ++d) {
            const int dr = (it < FI1 ? g * 4 : og) + ((d0 + d * dstep) & 3);
            const rsrc_t rd = mk_rsrc(p.ws[dr], WS_NEED);
#pragma unroll
            for (int i = 0; i < 4; ++i) st_sys(h[i], rd, lane * 16, base + d * ostep + i * step);
            dmask |= 1u << dr;
        }
        WAIT_VM0();
        if (lane < 8 && ((dmask >> lane) & 1)) add_sys(wsl + OFF_CNT + blk * CNT_BLK + (it < 64 ? 0 : 1024) + rank * 128);
    }
}

#define P_NDEC 9
#define P_DSLEEP 2
constexpr int SCR = 2560;
constexpr int NLW = 3, RING = 6, QLEN = 32, NDEC = P_NDEC;
constexpr int LDS_CTL = NLW * RING * 4096;
constexpr int LDS_DONE = LDS_CTL + QLEN * 16, LDS_TAIL = LDS_DONE + NLW * RING * 4, LDS_SCR = LDS_CTL + 1024;
static_assert(7 * WL <= NLW * NCOMM, "loaders");
constexpr u32 IDX_END = 0xffffffffu, IDX_BAD = 0x80000000u, IDX_CROSS = 0x40000000u;
__device__ __forceinline__ u32 lds_get(const LAS char* a) {
    u32 v;
    asm volatile("ds_read_b32 %0, %1\n\ts_waitcnt lgkmcnt(0)" : "=v"(v) : "v"((u32)(size_t)a) : "memory");
    return rfl(v);
}
__device__ __forceinline__ void lds_put(LAS char* a, u32 v) {
    asm volatile("" ::: "memory");
    *(volatile LAS u32*)a = v;
    asm volatile("" ::: "memory");
}

struct Item { u32 idx, dst; };
constexpr int N1 = NBLK * FI1, NT_I = N1 + NBLK * F2;
constexpr int PER = 16 + XA, NP = NBLK * PER, NX = NBLK * (16 + FX), NO1 = (RF1 - FL) * 4, NT_C = NX + NO1 + 192;
constexpr int TAB_BYTES = (NT_I > NT_C ? NT_I : NT_C) * 4, LDS_TAB = LDS_SCR + NDEC * SCR;
static_assert(LDS_TAB + NLW * TAB_BYTES <= LDS_BYTES, "communication LDS");
__device__ __forceinline__ void build_table(LAS char* tab, int rank, int slot, int first, int stride) {
    const int g = rank >> 2, j = rank & 3, q = slot - 3;
    for (int it = first; it < (slot < 3 ? NT_I : NT_C); it += stride) {
        Item m;
        if (slot < 3) {
            const bool s2 = it >= N1;
            const int a = it - N1, b1 = it / FI1;
            const int blk = s2 ? a >> 4 : b1, s = s2 ? FI1 + (a & 15) : it - b1 * FI1;
            m.idx = blk * 128 + s;
            m.dst = (s2 ? REL_A2 + (slot * F2 - FI1) * 4096 : (64 + slot * FI1) * 4096) + (blk * (s2 ? RF2 : RS1) + s) * 4096;
        } else if (it < NX) {
            const int blk = it / PER, c = it - blk * PER;
            const bool tail = it >= NP, isx = tail || (c < 2 * XA && (c & 1));
            const int cw = c < 2 * XA ? c >> 1 : c - XA;
            const int xi = tail ? it - NP + NBLK * XA : blk * XA + (c >> 1), xb = xi / FX, xs = FI + xi - xb * FX;
            m.idx = isx ? xb * 128 + xs : blk * 128 + 64 + 16 * j + cw;
            m.dst = isx ? (xb * RS1 + xs + 64 + 3 * FI1 + q * FX - FI) * 4096
                        : REL_B + (blk * CF + 64 + 16 * q + (cw & 12)) * 4096 + (cw & 3) * 1024;
        } else {
            const int k = it - NX;
            int row, col;
            if (k < NO1) {
                const int rf = (k >> 2) + ((k >> 2) >= FI + 3 * FI1 + j * FX ? FL : 0);
                m.idx = 8192 + (rf >> 4) * 64 + (k & 2) * 16 + ((rf >> 2) & 3) * 8 + (k & 1) * 4 + (rf & 3);
                row = row_of_rf1(rf, g ^ 1, q);
                col = (k & 3) * 64;
            } else {
                const int z = k - NO1, rt = z >> 5;
                m.idx = 8192 + 832 + z;
                row = ((g ^ 1) * 4 + ((q + 1 + (rt >> 1)) & 3)) * NS + (FI1 + (rt & 1) * 8 + (z & 7)) * 16;
                col = ((z >> 3) & 3) * 64;
            }
            m.dst = (u32)row * 4096 + (q * 256 + col) * 4;
        }
        *(LAS u32*)(tab + it * 4) = m.idx | m.dst << (m.idx < 8192 ? 4 : 6);
    }
}

template <bool CROSS>
__device__ __forceinline__ void load_role(const Params& p, LAS char* lds, int ld, int nld, int slot, int wv, int lane, u32 call) {
    const int rank = rfl(p.rank);
    const int g = rank >> 2, j = rank & 3;
    const int peer = CROSS ? (g ^ 1) * 4 + slot - 3 : g * 4 + ((j + 1 + slot) & 3);
    constexpr int NT = CROSS ? NT_C : NT_I;
    const u32 par = OFF_PAR + (call & 1) * PAR_BYTES;
    char* ws = uni(p.ws[rank]);
    const rsrc_t r0 = rsrc_u(ws, WS_NEED), rp = mk_rsrc(p.ws[peer], WS_NEED);
    const char* rdy = ws + OFF_RDY + peer * 8192;
    const u32 vo = lane * 16;
    constexpr int SKIP = PB * (CROSS ? PER : FI1);
    const u32 head = lane == 0 ? OFF_QUE + slot * 256 : WS_NEED, tbase = SKIP + 4 * WL - (call - 1) * (NT - SKIP);
    const bool nowait = (p.pad & 4) != 0;
    const LAS char* tab = lds + LDS_TAB + ld * TAB_BYTES;
    const LAS char* done = lds + LDS_DONE + ld * RING * 4;
    const u32 ldsb = rfl((u32)(size_t)lds) + ld * RING * 4096;
    auto table = [&](int it) {
        asm volatile("ds_read_b32 v160, %0" :: "v"((u32)(size_t)tab + (it < NT ? it : NT - 1) * 4) : "memory", "v160");
    };
    auto entry = [&] {
        u32 t;
        asm volatile("s_waitcnt lgkmcnt(0)\n\tv_readfirstlane_b32 %0, v160" : "=s"(t) :: "memory");
        Item m;
        m.idx = t & 16383;
        m.dst = (t - m.idx) >> (CROSS && m.idx >= 8192 ? 6 : 4);
        return m;
    };
    auto decode = [&](int it) { table(it); return entry(); };
    auto flag_of = [&](u32 idx) { return idx < 8192 ? idx >> 3 : (idx < 8192 + 832 ? 1024 + ((idx - 8192) >> 2) : idx - (8192 + 832 - 1024 - 208)); };
    u32 okupto = 0, wbase = 0; u64 wmask = 0;
    auto ready = [&](u32 fi) {
        if (fi < okupto) return true;
        if (fi - wbase < 64u && ((wmask >> (fi - wbase)) & 1)) return true;
        for (;;) {
            wbase = okupto;
            wmask = __builtin_amdgcn_ballot_w64(ld_sys(rdy + (wbase + lane) * 4) >= call);
            if (wmask != ~0ull) break;
            okupto += 64;
            if (fi < okupto) return true;
        }
        okupto += __builtin_ctzll(~wmask);
        if (fi - wbase < 64u) return ((wmask >> (fi - wbase)) & 1) != 0;
        return rfl(ld_sys(rdy + fi * 4)) >= call;
    };
    struct Prep { u32 src, m0v, vl; };
    auto prep = [&](const Item& m, u32 n) {
        Prep r;
        r.src = par + REL_PK + (m.idx + (CROSS && m.idx >= 8192 ? j * 1024 : 0)) * 4096;
        r.m0v = ldsb + (n % RING) * 4096;
        const u32 rest = CROSS && m.idx < 8192 && (m.idx & 64) ? 256 + VAR_W : 256 + VAR_X;
        r.vl = vo < rest ? vo : 0x80000000u;
        asm volatile("" : "+s"(r.src), "+s"(r.m0v), "+v"(r.vl));
        return r;
    };
    auto issue = [&](const Prep& r) {
        asm volatile("s_mov_b32 m0, %0\n\ts_nop 0\n\t"
                     "buffer_load_dwordx4 %1, %3, %4 offen sc0 sc1 lds\n\t"
                     "buffer_load_dwordx4 %1, %3, %4 offen offset:1024 sc0 sc1 lds\n\t"
                     "buffer_load_dwordx4 %1, %3, %4 offen offset:2048 sc0 sc1 lds\n\t"
                     "buffer_load_dwordx4 %2, %3, %4 offen offset:3072 sc0 sc1 lds"
                     :: "s"(r.m0v), "v"(vo), "v"(r.vl), "s"(rp), "s"(r.src) : "memory");
    };
    u32 vtail = (u32)(size_t)lds + LDS_TAIL, vone = 1;
    asm volatile("" : "+v"(vtail), "+v"(vone));
    auto enqueue = [&](u32 idx, u32 dst, u32 n) {
        asm volatile("s_mov_b64 exec, 1\n\tds_add_rtn_u32 v162, %0, %1\n\tv_mov_b32 v159, %2\n\tv_mov_b32 v160, %3\n\tv_mov_b32 v161, %4\n\t"
                     "s_waitcnt lgkmcnt(0)\n\tv_add_u32 v158, 1, v162\n\tv_and_b32 v162, %6, v162\n\tv_lshl_add_u32 v162, v162, 4, %5\n\t"
                     "ds_write_b128 v162, v[158:161]\n\ts_mov_b64 exec, -1"
                     :: "v"(vtail), "v"(vone), "s"(idx), "s"(dst), "s"(n << 8 | ld << 4 | peer), "s"(rfl((u32)(size_t)lds) + LDS_CTL), "n"(QLEN - 1)
                     : "memory", "v158", "v159", "v160", "v161", "v162");
    };
    __builtin_amdgcn_s_setprio(3);
    int e = SKIP + wv, e1 = e + WL, e2 = e + 2 * WL;
    const int e3 = e + 3 * WL;
    Item nx = decode(e1), ia = decode(e);
    bool bad = spin_ge<2>(rdy + flag_of(ia.idx) * 4, call, ws + OFF_TMO, call);
    STAMP(5);
    issue(prep(ia, 0));
    bool pre = ready(flag_of(nx.idx));
    u32 n = 0;
    const u32 cls = CROSS ? IDX_CROSS : 0;
    auto wait_free = [&](u32 m) { while (lds_get(done + (m % RING) * 4) + RING <= m) __builtin_amdgcn_s_sleep(1); };
    auto body = [&](auto odd) {
        if constexpr (odd.value) asm volatile("v_mov_b32 v164, 1\n\ts_nop 0\n\tbuffer_atomic_add v164, %0, %1, 0 offen sc0" :: "v"(head), "s"(r0) : "memory", "v164");
        else                     asm volatile("v_mov_b32 v165, 1\n\ts_nop 0\n\tbuffer_atomic_add v165, %0, %1, 0 offen sc0" :: "v"(head), "s"(r0) : "memory", "v165");
        table(e2);
        u32 dn;
        asm volatile("ds_read_b32 v162, %1\n\ts_waitcnt lgkmcnt(0)\n\tv_readfirstlane_b32 %0, v162"
                     : "=s"(dn) : "v"((u32)(size_t)done + ((n + 1) % RING) * 4) : "memory", "v162");
        const Item nn = entry();
        bool pre2 = e2 < NT;
        if (pre2 && !(nowait && nn.idx >= 8192)) pre2 = ready(flag_of(nn.idx));
        TICK(t1);
        if (dn + RING <= n + 1) wait_free(n + 1);
        TICK(t2);
        const Prep px = prep(nx, n + 1);
        u32 told;
        if constexpr (odd.value) asm volatile("s_waitcnt vmcnt(1)\n\tv_readfirstlane_b32 %0, v165" : "=s"(told) :: "memory");
        else                     asm volatile("s_waitcnt vmcnt(1)\n\tv_readfirstlane_b32 %0, v164" : "=s"(told) :: "memory");
        TICK(t3);
        if (pre) issue(px);
        enqueue(ia.idx | cls | (bad ? IDX_BAD : 0), ia.dst, n);
        if (!pre && e1 < NT) {
            bad |= spin_ge<2>(rdy + flag_of(nx.idx) * 4, call, ws + OFF_TMO, call);
            issue(px);
        }
        STAMP_IF(n == 0, 6);
        STAMP_IF(CROSS ? (e < NP && e1 >= NP) : (e < N1 && e1 >= N1), 1);
        STAMP_IF(CROSS && e < NX && e1 >= NX, 2);
        STAMP_IF(CROSS && e < NX + NO1 && e1 >= NX + NO1, 3);
        ia = nx; nx = nn; pre = pre2; ++n;
        e = e1; e1 = e2; e2 = n == 1 ? e3 : (int)(told + tbase);
    };
    for (;;) {
        body(std::false_type{});
        if (e >= NT) break;
        body(std::true_type{});
        if (e >= NT) break;
    }
    u32 fin;
    asm volatile("ds_add_rtn_u32 v162, %1, %2\n\ts_waitcnt lgkmcnt(0)\n\tv_readfirstlane_b32 %0, v162"
                 : "=s"(fin) : "v"((u32)(size_t)lds + LDS_TAIL + 4), "v"(lane == 0 ? 1u : 0u) : "memory", "v162");
    if (fin + 1 == (u32)nld)
        for (int k = 0; k < NDEC; ++k) enqueue(IDX_END, 0, 0);
}

__device__ __forceinline__ void decode_role(const Params& p, LAS char* lds, u32 me, int lane, u32 call) {
    const int rank = rfl(p.rank);
    const int j = rank & 3;
    const u32 par = OFF_PAR + (call & 1) * PAR_BYTES;
    const rsrc_t r0 = mk_rsrc(p.ws[rank], WS_NEED), ro = mk_rsrc(p.out[rank], OUT_BYTES);
    const u32 vo = lane * 16, vout = (lane >> 4) * 4 * 4096 + (lane & 15) * 16;
    LAS char* scr = lds + LDS_SCR + me * SCR;
    u32 q0, q1, q2, q3;
    auto peek = [&](u32 n) {
        asm volatile("ds_read_b128 v[158:161], %4\n\ts_waitcnt lgkmcnt(0)\n\tv_readfirstlane_b32 %0, v158\n\tv_readfirstlane_b32 %1, v159\n\t"
                     "v_readfirstlane_b32 %2, v160\n\tv_readfirstlane_b32 %3, v161"
                     : "=s"(q0), "=s"(q1), "=s"(q2), "=s"(q3) : "v"((u32)(size_t)lds + LDS_CTL + (n % QLEN) * 16) : "memory", "v158", "v159", "v160", "v161");
        return q0 == n + 1;
    };
    const u32 vsig = lane == 0 ? 0u : 0x80000000u;
    u32 one = 1;
    asm volatile("" : "+v"(one));
    auto signal = [&](u32 c) {
        asm volatile("s_nop 4\n\tbuffer_atomic_add %0, %1, %2, %3 offen sc1" :: "v"(one), "v"(vsig), "s"(r0), "s"(rfl(c)) : "memory");
    };
    const u32x4 nan4 = {0x7fc07fc0u, 0x7fc07fc0u, 0x7fc07fc0u, 0x7fc07fc0u};
    u32 vhead = (u32)(size_t)lds + LDS_TAIL + 8, vone = 1;
    asm volatile("" : "+v"(vhead), "+v"(vone));
    for (;;) {
        TICK(t0);
        u32 n;
        asm volatile("s_mov_b64 exec, 1\n\tds_add_rtn_u32 v158, %1, %2\n\ts_waitcnt lgkmcnt(0)\n\tv_readfirstlane_b32 %0, v158\n\ts_mov_b64 exec, -1"
                     : "=s"(n) : "v"(vhead), "v"(vone) : "memory", "v158");
        while (!peek(n)) __builtin_amdgcn_s_sleep(P_DSLEEP);
        TICK(t1);
        if (q1 == IDX_END) break;
        const u32 dst = q2, item = q3 >> 8, ring = (((q3 >> 4) & 3) * RING + item % RING), peer = q3 & 7;
        const LAS char* sl = lds + ring * 4096;
        const u32x4 c0 = *(const LAS u32x4*)(sl + vo), c1 = *(const LAS u32x4*)(sl + 1024 + vo), c2 = *(const LAS u32x4*)(sl + 2048 + vo);
        const u32 k1 = *(const LAS u32*)(sl + 3072 + lane * 4);
        const u32 hx = *(const LAS unsigned char*)(sl + 3328);
        const bool bad = (q1 & IDX_BAD) != 0;
        const u32 idx = q1 & 0xffff;
        const bool out = idx >= 8192, isw = !out && (idx & 64);
        const u32 step = (out || isw) ? 4096 : 1024;
        u32x4 h[4];
        bool done = false;
        if (!bad && !out) {
            const u32 hd = rfl(hx);
            if (hd != 255) { unpack_item(h, c0, c1, c2, k1, sl + 3328, hd, isw ? VAR_W : VAR_X, scr, lane); done = true; }
        } else if (!bad) {
            const u32 g1 = *(const LAS unsigned short*)(sl + 3328 + lane * 2), hd = rfl(*(const LAS unsigned short*)(sl + 3456));
            if ((hd & 255) != 255) { unpack_out(h, c0, c1, c2, k1, g1, (const LAS unsigned char*)(sl + 3456), hd & 255, hd >> 8, scr, lane); done = true; }
        }
        if (!done) {
            const u32 blk = idx >> 7, s = idx & 127;
            const u32 raw = par + (out ? REL_PK + STG_BYTES + (idx + j * 1024) * 4096
                                       : isw ? REL_B + (blk * CF + (s & 60)) * 4096 + (s & 3) * 1024 : (blk * RS1 + s) * 4096);
            const u32 rs = isw ? 4096 : 1024;
            const rsrc_t rp = mk_rsrc(p.ws[peer], WS_NEED);
            asm volatile("s_nop 4\n\tbuffer_load_dwordx4 %0, %4, %5, %6 offen sc0 sc1\n\tbuffer_load_dwordx4 %1, %4, %5, %7 offen sc0 sc1\n\t"
                         "buffer_load_dwordx4 %2, %4, %5, %8 offen sc0 sc1\n\tbuffer_load_dwordx4 %3, %4, %5, %9 offen sc0 sc1\n\ts_waitcnt vmcnt(0)"
                         : "=&v"(h[0]), "=&v"(h[1]), "=&v"(h[2]), "=&v"(h[3])
                         : "v"(vo), "s"(rp), "s"(rfl(raw)), "s"(rfl(raw + rs)), "s"(rfl(raw + 2 * rs)), "s"(rfl(raw + 3 * rs)) : "memory");
            if (bad) { h[0] = nan4; h[1] = nan4; h[2] = nan4; h[3] = nan4; }
        }
        lds_put(lds + LDS_DONE + ring * 4, item + 1);
        const rsrc_t rd = out ? ro : r0;
        const u32 vd = (out ? vout : vo + par) + dst;
#pragma unroll
        for (int i = 0; i < 4; ++i) st_sys(h[i], rd, vd, i * step);
        TICK(t2);
        if (!out) {
            asm volatile("s_waitcnt vmcnt(0)" ::: "memory");
            signal(OFF_CNT + (idx >> 7) * CNT_BLK + peer * 128 + (isw ? 1024 : (!(q1 & IDX_CROSS) && (idx & 127) >= FI1) ? 2048 : 0));
        }
        TICK(t3);
    }
    STAMP(4);
}

#define LDPOL " sc0 sc1"
#define GLDS2(m0v, voff, rs, soff, imm0, imm1) \
    asm volatile("s_mov_b32 m0, %0\n\ts_nop 0\n\tbuffer_load_dwordx4 %1, %2, %3 offen offset:" #imm0 LDPOL " lds\n\t" \
                 "buffer_load_dwordx4 %1, %2, %3 offen offset:" #imm1 LDPOL " lds" \
                 :: "s"(m0v), "v"(voff), "s"(rs), "s"(soff) : "memory")
#define BARRIER() do { __builtin_amdgcn_sched_barrier(0); __builtin_amdgcn_s_barrier(); __builtin_amdgcn_sched_barrier(0); } while (0)

#define SCHED() __builtin_amdgcn_sched_barrier(0)
#define WAIT_LGKM0() __builtin_amdgcn_s_waitcnt(0xC07F)

template <int STAGE>
struct Gemm {
    static constexpr int NT = STAGE == 1 ? 128 : 64;
    static constexpr int NA = STAGE == 1 ? 4 : 2;
    static constexpr int NB = STAGE == 1 ? 4 : 8;
    static constexpr int NM = STAGE == 1 ? 4 : 1;
    static constexpr int NMM = STAGE == 1 ? 16 : 8;
    static constexpr int NRD = NA + NB;
    struct Frags { f16x8 a[NA], b[NB]; };

    rsrc_t rs;
    u32 sa, sa2, sb;
    u32 ablk;
    u32 ldsw;
    int wid, lane;

    template <int I> __device__ __forceinline__ void piece(int t, int buf) const {
        const u32 vo = lane * 16;
        const u32 ma = ldsw + buf * BUF_BYTES + wid * 4096, mb = ldsw + buf * BUF_BYTES + 32768 + wid * 2048;
        const u32 ka = STAGE == 1 ? (u32)(t >> 1) * (RS1 * 4096) + (t & 1) * 2048 : (u32)t * ablk;
        const u32 a = sa + ka, a2 = sa2 + ka;
        const u32 b = STAGE == 1 ? sb + (u32)(t >> 1) * (CF * 4096) + (t & 1) * 2048 : sb + (u32)t * (CF * 4096);
        if (I == 0) GLDS2(ma, vo, rs, a, 0, 1024);
        if (I == 1) GLDS2(ma, vo, rs, a2, 2048, 3072);
        if (I == 2) GLDS2(mb, vo, rs, b, 0, 1024);
    }
    __device__ __forceinline__ void stage(int t, int buf) const { piece<0>(t, buf); piece<1>(t, buf); piece<2>(t, buf); }

    template <int I> __device__ __forceinline__ void rd(Frags& f, const LAS char* lds, int buf, int half) const {
        const LAS char* base = lds + buf * BUF_BYTES + lane * 16;
        if constexpr (STAGE == 1) {
            const LAS char* pa = base + (wid >> 1) * 8192 + half * 1024;
            const LAS char* pb = base + 32768 + (wid & 1) * 8192 + half * 1024;
            if constexpr (I == 0) f.a[0] = *(const LAS f16x8*)pa;
            else if constexpr (I <= 4) f.b[I - 1] = *(const LAS f16x8*)(pb + (I - 1) * 2048);
            else f.a[I - 4] = *(const LAS f16x8*)(pa + (I - 4) * 2048);
        } else {
            const LAS char* pa = base + wid * 4096 + half * 2048;
            const LAS char* pb = base + 32768 + half * 2048;
            constexpr int h = I / 5, n = I % 5;
            if constexpr (n == 0) f.a[h] = *(const LAS f16x8*)(pa + h * 1024);
            else f.b[h * 4 + n - 1] = *(const LAS f16x8*)(pb + (n - 1) * 4096 + h * 1024);
        }
    }
    template <int I> __device__ __forceinline__ void mm(f32x4 (&acc)[NM][4], const Frags& f) const {
        if constexpr (STAGE == 1) acc[I >> 2][I & 3] = __builtin_amdgcn_mfma_f32_16x16x32_f16(f.a[I >> 2], f.b[I & 3], acc[I >> 2][I & 3], 0, 0, 0);
        else            acc[0][I & 3] = __builtin_amdgcn_mfma_f32_16x16x32_f16(f.a[I >> 2], f.b[I], acc[0][I & 3], 0, 0, 0);
    }
    template <int I, int N> __device__ __forceinline__ void rds(Frags& f, const LAS char* lds, int buf, int half) const {
        if constexpr (I < N) { rd<I>(f, lds, buf, half); rds<I + 1, N>(f, lds, buf, half); }
    }
    static __device__ __forceinline__ int blk_of(int t) { return STAGE == 1 ? (t >> 1) : t; }

    template <bool RD, int ST>
    __device__ __forceinline__ void half_step(f32x4 (&acc)[NM][4], const Frags& cur, Frags& nxt, const LAS char* lds, int rbuf, int rhalf,
                                              int ts, int bs, u32 cvo, u32 fso) const {
        auto first = [&] {
            if (ST == 1) asm volatile("buffer_load_dword v166, %0, %1, %2 offen sc0 sc1" :: "v"(cvo), "s"(rs), "s"(fso) : "memory", "v166");
            if (ST) { piece<0>(ts, bs); SCHED(); }
        };
        if constexpr (STAGE == 1) {
            mm<0>(acc, cur);  mm<1>(acc, cur);  if (RD) rds<0, 2>(nxt, lds, rbuf, rhalf); SCHED();
            mm<2>(acc, cur);  mm<3>(acc, cur);  if (RD) rds<2, 4>(nxt, lds, rbuf, rhalf); SCHED();
            mm<4>(acc, cur);  mm<5>(acc, cur);  if (RD) rds<4, 6>(nxt, lds, rbuf, rhalf); SCHED();
            mm<6>(acc, cur);  mm<7>(acc, cur);  if (RD) rds<6, 8>(nxt, lds, rbuf, rhalf); SCHED();
            mm<8>(acc, cur);  mm<9>(acc, cur);  SCHED();
            first();
            mm<10>(acc, cur); mm<11>(acc, cur); SCHED();
            if (ST) { piece<1>(ts, bs); SCHED(); }
            mm<12>(acc, cur); mm<13>(acc, cur); SCHED();
            if (ST) { piece<2>(ts, bs); SCHED(); }
            mm<14>(acc, cur); mm<15>(acc, cur); SCHED();
        } else {
            first();
            mm<0>(acc, cur); SCHED();
            if (ST) { piece<1>(ts, bs); SCHED(); }
            mm<1>(acc, cur); SCHED();
            if (ST) { piece<2>(ts, bs); SCHED(); }
            mm<2>(acc, cur); if (RD) rds<0, 3>(nxt, lds, rbuf, rhalf); SCHED();
            mm<3>(acc, cur); if (RD) rds<3, 5>(nxt, lds, rbuf, rhalf); SCHED();
            mm<4>(acc, cur); if (RD) rds<5, 8>(nxt, lds, rbuf, rhalf); SCHED();
            mm<5>(acc, cur); if (RD) rds<8, 10>(nxt, lds, rbuf, rhalf); SCHED();
            mm<6>(acc, cur); mm<7>(acc, cur); SCHED();
        }
    }

    template <int ROLE>
    __device__ __forceinline__ u32 run(f32x4 (&acc)[NM][4], const LAS char* lds, char* ws, u32 cvo, u32 need, u32 call) const {
        constexpr bool LATE = ROLE == 2 && STAGE == 1;
        char* tmo = ws + OFF_TMO;
        Frags f0, f1;
        if (ROLE == 0) spin_ge(ws + cvo + blk_of(0) * CNT_BLK, need, tmo, call);
        BARRIER();
        stage(0, 0);
        if (ROLE == 0) { spin_ge(ws + cvo + blk_of(1) * CNT_BLK, need, tmo, call); spin_ge(ws + cvo + blk_of(2) * CNT_BLK, need, tmo, call); }
        BARRIER();
        stage(1, 1);
        if (!LATE) { stage(2, 2); asm volatile("s_waitcnt vmcnt(12)" ::: "memory"); }
        else asm volatile("s_waitcnt vmcnt(6)" ::: "memory");
        BARRIER();
        rds<0, NRD>(f0, lds, 0, 0);
        WAIT_LGKM0();
        if (ROLE == 0) asm volatile("v_mov_b32 v166, 0" ::: "v166");
        int buf = 0;
        for (int t = 0; t < NT - 3; ++t) {
            const int nb = buf == 2 ? 0 : buf + 1, pb = buf == 0 ? 2 : buf - 1;
            half_step<true, LATE ? 2 : 0>(acc, f0, f1, lds, buf, 1, t + 2, pb, 0, 0);
            if (ROLE == 0) {
                u64 miss;
                asm volatile("s_waitcnt vmcnt(6)\n\tv_cmp_lt_u32_e64 %0, v166, %1\n\ts_nop 0" : "=s"(miss) : "v"(need) : "memory");
                WAIT_LGKM0();
                if (miss != 0) spin_ge(ws + cvo + blk_of(t + 3) * CNT_BLK, need, tmo, call);
            } else {
                asm volatile("s_waitcnt vmcnt(6)" ::: "memory");
                WAIT_LGKM0();
            }
            BARRIER();
            half_step<true, LATE ? 0 : (ROLE == 0 ? 1 : 2)>(acc, f1, f0, lds, nb, 0, t + 3, buf, cvo, blk_of(t + 4 < NT ? t + 4 : t + 3) * CNT_BLK);
            buf = nb;
        }
#pragma unroll
        for (int t = NT - 3; t < NT; ++t) {
            const int nb = buf == 2 ? 0 : buf + 1, pb = buf == 0 ? 2 : buf - 1;
            if (t == NT - 3) {
                half_step<true, LATE ? 2 : 0>(acc, f0, f1, lds, buf, 1, t + 2, pb, 0, 0);
                asm volatile("s_waitcnt vmcnt(6)" ::: "memory");
            } else {
                if (t == NT - 2) {
                    const u32 to = OFF_TMO;
                    asm volatile("buffer_load_dword v167, %0, %1, 0 offen sc1" :: "v"(to), "s"(rs) : "memory", "v167");
                }
                half_step<true, 0>(acc, f0, f1, lds, buf, 1, 0, 0, 0, 0);
                asm volatile("s_waitcnt vmcnt(0)" ::: "memory");
            }
            WAIT_LGKM0();
            BARRIER();
            if (t < NT - 1) half_step<true, 0>(acc, f1, f0, lds, nb, 0, 0, 0, 0, 0);
            else            half_step<false, 0>(acc, f1, f0, lds, nb, 0, 0, 0, 0, 0);
            buf = nb;
        }
        u32 tw;
        asm volatile("v_readfirstlane_b32 %0, v167\n\ts_nop 1" : "=s"(tw) :: "memory");
        return tw;
    }
};

__device__ __forceinline__ u32x4 gelu4(const f32x4 (&c)[4], int r, bool bad) {
    const float nanv = __builtin_nanf("");
    f32x4 o = {gelu(c[0][r]), gelu(c[1][r]), gelu(c[2][r]), gelu(c[3][r])};
    if (bad) o = (f32x4){nanv, nanv, nanv, nanv};
    return __builtin_bit_cast(u32x4, o);
}

__device__ __forceinline__ void gemm_role(const Params& p, const LAS char* lds, int xcd, int s, int wid, int lane, u32 call) {
    const int rank = rfl(p.rank);
    const int g = rank >> 2, j = rank & 3, og = (g ^ 1) * 4;
    char* ws = p.ws[rank];
    const u32 par = OFF_PAR + (call & 1) * PAR_BYTES;
    const rsrc_t rs = mk_rsrc(ws, WS_NEED), ro = mk_rsrc(p.out[rank], OUT_BYTES);
    const u32 ldsw = rfl((u32)(size_t)lds);
    const u32 vout = (lane >> 4) * 4 * 4096 + (lane & 15) * 16;
    {
        const int u = (xcd & 1) * 26 + s, rt = u >> 2, ct = (xcd >> 1) * 4 + (u & 3);
        Gemm<1> G;
        G.rs = rs; G.ldsw = ldsw; G.wid = wid; G.lane = lane;
        const int rf = rt * 16 + wid * 2;
        G.sa = par + (rf < FI ? rf : rf + FX) * 4096;
        G.sa2 = par + (rf + 1 < FI ? rf + 1 : rf + 1 + FX) * 4096 - 2048;
        G.sb = par + REL_B + (ct * 8 + wid) * 4096;
        const int bsrc = ct < 8 ? rank : og + ((ct - 8) >> 1);
        u32 cvo = OFF_CNT + 1024 + bsrc * 128, need = (ct < 8 ? 8u : 16u) * call;
        u32 mask = 0;
        for (int i = 0; i < 16; ++i) mask |= 1u << src_of_rf1(rt * 16 + i, g, j);
        if (lane < 8 && ((mask >> lane) & 1)) {
            cvo = OFF_CNT + lane * 128;
            need = (lane == rank ? 8 : ((lane >> 2) == g ? FI1 : FX)) * call;
        }
        f32x4 acc[4][4] = {};
        const bool bad = call == (wid == 0 ? G.template run<0>(acc, lds, ws, cvo, need, call)
                                  : wid < 4 ? G.template run<1>(acc, lds, ws, cvo, need, call)
                                            : G.template run<2>(acc, lds, ws, cvo, need, call));
        STAMP(1);
        const int wm = wid >> 1, wn = wid & 1;
        if (ct < 8) {
#pragma unroll
            for (int m = 0; m < 4; ++m) {
                const u32 ob = (u32)row_of_rf1(rt * 16 + wm * 4 + m, g, j) * 4096 + (ct * 128 + wn * 64) * 4;
#pragma unroll
                for (int r = 0; r < 4; ++r) __builtin_amdgcn_raw_buffer_store_b128(gelu4(acc[m], r, bad), ro, vout + ob + r * 4096, 0, OUT_AUX);
            }
        } else {
            const int fo = (rt * 2 + (ct & 1)) * 8 + wid;
            const u32 sb = par + REL_STG + (u32)(((ct - 8) >> 1) * 1024 + fo * 4) * 4096;
            u32x4 o[4][4];
#pragma unroll
            for (int i = 0; i < 16; ++i) o[i >> 2][i & 3] = gelu4(acc[i >> 2], i & 3, bad);
            u32 lop, lon;
            out_windows<4>(o, lop, lon);
#pragma unroll
            for (int m = 0; m < 4; ++m) pack_out(o[m], lop, lon, (LAS char*)lds + wid * 2048, rs, sb + m * 4096, lane);
            WAIT_VM0();
            if (lane == 0) st_flag(p.ws[bsrc] + OFF_RDY + rank * 8192 + (1024 + fo) * 4, call);
        }
        STAMP(2);
    }
    {
        const bool loc = s >= 24;
        const int rt = s >> 2, ct = loc ? xcd * 2 + (s & 1) : ((~xcd & 1) * 16) + (xcd >> 1) * 4 + (s & 3);
        Gemm<2> G;
        G.rs = rs; G.ldsw = ldsw; G.wid = wid; G.lane = lane;
        G.sa = G.sa2 = par + (loc ? (FI + wid) * 4096 : REL_A2 + (rt * 8 + wid) * 4096);
        G.ablk = (loc ? RS1 : RF2) * 4096;
        G.sb = par + REL_B + ct * 4 * 4096 + wid * 2048;
        const int asrc = loc ? rank : g * 4 + ((j + 1 + (rt >> 1)) & 3);
        const int bsrc = ct < 16 ? rank : og + ((ct - 16) >> 2);
        u32 cvo = OFF_CNT + 1024 + bsrc * 128, need = (ct < 16 ? 8u : 16u) * call;
        if (lane == 0) { cvo = OFF_CNT + (loc ? 0 : 2048) + asrc * 128; need = (loc ? 8 : F2) * call; }
        f32x4 acc[1][4] = {};
        const bool bad = call == (wid == 0 ? G.template run<0>(acc, lds, ws, cvo, need, call)
                                  : wid < 4 ? G.template run<1>(acc, lds, ws, cvo, need, call)
                                            : G.template run<2>(acc, lds, ws, cvo, need, call));
        STAMP(3);
        if (ct < 16) {
            const u32 ob = (u32)(asrc * NS + ((loc ? FI : FI1 + (rt & 1) * 8) + wid) * 16) * 4096 + ct * 256;
#pragma unroll
            for (int r = 0; r < 4; ++r) __builtin_amdgcn_raw_buffer_store_b128(gelu4(acc[0], r, bad), ro, vout + ob + r * 4096, 0, OUT_AUX);
        } else {
            const int z = (rt * 4 + (ct & 3)) * 8 + wid;
            const u32 sb = par + REL_STG + (u32)(((ct - 16) >> 2) * 1024 + 832 + z) * 4096;
            const u32x4 o[1][4] = {{gelu4(acc[0], 0, bad), gelu4(acc[0], 1, bad), gelu4(acc[0], 2, bad), gelu4(acc[0], 3, bad)}};
            u32 lop, lon;
            out_windows<1>(o, lop, lon);
            pack_out(o[0], lop, lon, (LAS char*)lds + wid * 2048, rs, sb, lane);
            WAIT_VM0();
            if (lane == 0) st_flag(p.ws[bsrc] + OFF_RDY + rank * 8192 + (1024 + 208 + z) * 4, call);
        }
        STAMP(4);
    }
}

__global__ __launch_bounds__(BLOCK, 3) void ag_gemm_kernel(Params p) {
    __shared__ __attribute__((aligned(1024))) char smem[LDS_BYTES];
    const int rank = rfl(p.rank);
    const int lane = threadIdx.x & 63;
    const int wid = rfl(threadIdx.x >> 6);
    const int b = blockIdx.x;
    if (wid >= (b < NCOMM ? NLW + NDEC : 8)) return;
    STAMP(0);
    char* seq = p.ws[rank] + OFF_SEQ + (b * 16 + wid) * 4;
    const u32 prev = ld_agent(seq);
    char* wsl = p.ws[lane & 7];
    auto next_call = [&] {
        const u32 call = rfl(prev) + 1;
        if (lane == 0) __hip_atomic_store((GAS u32*)seq, call, __ATOMIC_RELAXED, __HIP_MEMORY_SCOPE_AGENT);
        STAMP(7);
        return call;
    };
    if (b >= NCOMM) {
        constexpr int B0 = PB > 0 ? PB - 1 : 0;
        if ((u32)(b - NCOMM - B0 * 16) >= EARLY * 16) __builtin_amdgcn_s_sleep(LATE_SLEEP);
        f32x4 v[8];
        load_item(v, p, b - NCOMM, wid, lane);
        const u32 call = next_call();
        convert_local(p, (LAS char*)smem, v, wsl, b - NCOMM, wid, lane, call);
        STAMP(5);
        gemm_role(p, (const LAS char*)smem, b & 7, (b - NCOMM) >> 3, wid, lane, call);
        return;
    }
    const int nld = 2 * NCOMM + b < 7 * WL ? 3 : 2;
    const int sw = wid < NLW + NDEC - 5 ? NPUSH : (NLW + NDEC - 1 - wid) * NCOMM + b;
    f32x4 v[8];
    if (sw < NPUSH) {
        int blk, it;
        push_decode(sw, blk, it);
        load_item(v, p, blk * 16 + (it >> 3), it & 7, lane);
    }
    if (wid < 4) *(LAS u32*)(smem + LDS_CTL + (wid * 64 + lane) * 4) = 0;
    for (int k = 0; k < nld; ++k)
        build_table((LAS char*)smem + LDS_TAB + k * TAB_BYTES, rank, (k * NCOMM + b) % 7, threadIdx.x, (NLW + NDEC) * 64);
    asm volatile("s_waitcnt lgkmcnt(0)\n\ts_barrier" ::: "memory");
    const u32 call = next_call();
    if (sw < NPUSH) push_role(p, v, wsl, sw, lane, call);
    if (wid >= NLW) decode_role(p, (LAS char*)smem, wid - NLW, lane, call);
    else if (wid < nld) {
        const int ps = rfl((wid * NCOMM + b) % 7), wv = rfl((wid * NCOMM + b) / 7);
        if (ps < 3) load_role<false>(p, (LAS char*)smem, wid, nld, ps, wv, lane, call);
        else        load_role<true>(p, (LAS char*)smem, wid, nld, ps, wv, lane, call);
    }
}

}

constexpr int g_mode = 0;

extern "C" void kernel_launch(int rank, int nranks, ncclComm_t comm, hipStream_t stream,
                              void* const* d_in, const int* in_sizes, int n_in,
                              void* d_out, int out_size, const int* in_dtypes, int out_dtype,
                              void* const* d_out_all, void* const* d_ws_all, size_t ws_bytes) {
    (void)comm; (void)in_sizes; (void)n_in; (void)d_out; (void)out_size; (void)in_dtypes; (void)out_dtype;
    if (nranks != NR || (unsigned)rank >= (unsigned)NR || ws_bytes < WS_NEED) {
        fprintf(stderr, "unsupported configuration\n");
        return;
    }
    Params p;
    p.x = (const float*)d_in[0];
    p.w = (const float*)d_in[1];
    for (int i = 0; i < NR; ++i) { p.ws[i] = (char*)d_ws_all[i]; p.out[i] = (char*)d_out_all[i]; }
    p.rank = rank;
    p.pad = g_mode;
    ag_gemm_kernel<<<GRID, BLOCK, 0, stream>>>(p);
    hipError_t e = hipGetLastError();
    if (e != hipSuccess) fprintf(stderr, "launch failed: %s\n", hipGetErrorString(e));
}
```

```cpp
#include <hip/hip_runtime.h>
#include <rccl/rccl.h>
#include <cstdio>
#include <cstdint>
#include <type_traits>

typedef unsigned u32;
typedef unsigned long long u64;
typedef u32 u32x4 __attribute__((ext_vector_type(4)));
typedef u32 u32x2v __attribute__((ext_vector_type(2)));
typedef float f32x4 __attribute__((ext_vector_type(4)));
typedef _Float16 f16x8 __attribute__((ext_vector_type(8)));
typedef __amdgpu_buffer_rsrc_t rsrc_t;
#define LAS __attribute__((address_space(3)))
#define GAS __attribute__((address_space(1)))

namespace {

constexpr int NR = 8;
constexpr int KD = 8192;
constexpr int NS = 1024;
constexpr int NBLK = 64;
#define P_FI 45
constexpr int FI = P_FI;
constexpr int F2 = 16;
constexpr int FI1 = FI - F2;
constexpr int FX = 64 - FI;
constexpr int FL = 8;
constexpr int RF1 = 208, RF2 = 48, CF = 128;
static_assert(FI + 3 * FI1 + 4 * FX == RF1, "stage-1 rows");

#define P_W 19
#define P_XA 14
#define P_PB 1
#define P_GRADE 20
constexpr int PB = P_PB;
constexpr int NCOMM = 48;
constexpr int WL = P_W;
constexpr int XA = P_XA;
static_assert(WL >= 14 && WL <= 20 && XA <= FX && XA <= 16, "wave roles");
constexpr int GRID = 256, BLOCK = 768, NGEMM = GRID - NCOMM;
constexpr int RS1 = 64 + 3 * FI1 + 4 * FX;

constexpr u32 OFF_SEQ  = 0;
constexpr u32 OFF_TMO  = 16384;
constexpr u32 OFF_QUE  = 16640;
constexpr u32 OFF_RDY  = 32768;
constexpr u32 OFF_CNT  = 131072;
constexpr u32 CNT_BLK  = 24 * 128;
constexpr u32 OFF_DIAG = 524288;
constexpr u32 A1_BYTES = (u32)NBLK * RS1 * 4096;
constexpr u32 A2_BYTES = (u32)NBLK * RF2 * 4096;
constexpr u32 B_BYTES  = (u32)NBLK * CF * 4096;
constexpr u32 STG_BYTES = 4u * 1024 * 4096;
constexpr u32 PK_BYTES = (u32)NBLK * 128 * 4096;
constexpr u32 REL_A2 = A1_BYTES, REL_B = REL_A2 + A2_BYTES, REL_PK = REL_B + B_BYTES, REL_STG = REL_PK + PK_BYTES;
constexpr u32 PAR_BYTES = REL_STG + 2 * STG_BYTES;
constexpr u32 OFF_PAR  = 1u << 20;
constexpr int VAR_XO = 192;
constexpr u32 WS_NEED  = OFF_PAR + 2 * PAR_BYTES;
constexpr u32 OUT_BYTES = (u32)KD * NS * 4;
static_assert(REL_PK < (256u << 20), "table entries hold 18 bits of dst / 1024");
static_assert(256 + VAR_XO == 448, "output items are as long as x items");
static_assert(PK_BYTES == 8192u * 4096u, "staged outputs follow the packed image");
static_assert(OFF_CNT + NBLK * CNT_BLK <= OFF_DIAG && OFF_RDY + 8 * 8192 <= OFF_CNT, "control area");

constexpr u64 DEADLINE = 40000000ull;

constexpr int BUF_BYTES = 49152;
constexpr int LDS_BYTES = 3 * BUF_BYTES;

struct Params {
    const float* x;
    const float* w;
    char* ws[NR];
    char* out[NR];
    int rank;
    int pad;
};

__device__ __forceinline__ u32 rfl(u32 v) { return __builtin_amdgcn_readfirstlane(v); }
__device__ __forceinline__ char* uni(const void* p) {
    const u64 a = (u64)p;
    return (char*)(((u64)rfl((u32)(a >> 32)) << 32) | rfl((u32)a));
}
__device__ __forceinline__ rsrc_t rsrc_u(const void* p, u32 bytes) {
    return __builtin_amdgcn_make_buffer_rsrc((void*)p, (short)0, (int)bytes, 0x00020000);
}
__device__ __forceinline__ rsrc_t mk_rsrc(const void* p, u32 bytes) { return rsrc_u(uni(p), bytes); }
__device__ __forceinline__ u32 ld_sys(const char* p) {
    return __hip_atomic_load((const GAS u32*)p, __ATOMIC_RELAXED, __HIP_MEMORY_SCOPE_SYSTEM);
}
__device__ __forceinline__ u32 ld_agent(const char* p) {
    return __hip_atomic_load((const GAS u32*)p, __ATOMIC_RELAXED, __HIP_MEMORY_SCOPE_AGENT);
}
__device__ __forceinline__ void st_flag(char* p, u32 v) {
    __hip_atomic_store((GAS u32*)p, v, __ATOMIC_RELAXED, __HIP_MEMORY_SCOPE_SYSTEM);
}
__device__ __forceinline__ void add_sys(char* p) {
    __hip_atomic_fetch_add((GAS u32*)p, 1u, __ATOMIC_RELAXED, __HIP_MEMORY_SCOPE_SYSTEM);
}
#define WAIT_VM0() do { asm volatile("s_waitcnt vmcnt(0)" ::: "memory"); __builtin_amdgcn_s_waitcnt(0x0F70); } while (0)

template <int SLEEP = 2>
__device__ __forceinline__ bool spin_ge(const char* p, u32 need, char* tmo, u32 call) {
    const u64 t0 = __builtin_amdgcn_s_memrealtime();
    bool gave_up = false;
    for (u32 polls = 0;; ++polls) {
        const u32 v = ld_sys(p);
        if (__builtin_amdgcn_ballot_w64(v < need) == 0) break;
        if ((polls & 7) == 0 && (rfl(ld_agent(tmo)) == call || __builtin_amdgcn_s_memrealtime() - t0 > DEADLINE)) {
            __hip_atomic_store((GAS u32*)tmo, call, __ATOMIC_RELAXED, __HIP_MEMORY_SCOPE_AGENT);
            gave_up = true;
            break;
        }
        __builtin_amdgcn_s_sleep(SLEEP);
    }
    asm volatile("" ::: "memory");
    return gave_up;
}

#define STAMP_IF(c, slot) do {} while (0)
#define STAMP(slot) do {} while (0)
#define STAMPB_IF(c, slot) do {} while (0)

__device__ __forceinline__ u32x4 cvt8(f32x4 a, f32x4 b) {
    f16x8 h;
    h[0] = (_Float16)a[0]; h[1] = (_Float16)a[1]; h[2] = (_Float16)a[2]; h[3] = (_Float16)a[3];
    h[4] = (_Float16)b[0]; h[5] = (_Float16)b[1]; h[6] = (_Float16)b[2]; h[7] = (_Float16)b[3];
    return __builtin_bit_cast(u32x4, h);
}
__device__ __forceinline__ void st_sys(u32x4 v, rsrc_t r, u32 voff, u32 soff) {
    __builtin_amdgcn_raw_buffer_store_b128(v, r, voff + soff, 0, 17);
}
__device__ __forceinline__ float gelu(float y) {
    const float u = y * (0.7978845608028654f + 0.7978845608028654f * 0.044715f * y * y);
    const float e = __builtin_amdgcn_exp2f(u * -2.8853900817779268f);
    return y * __builtin_amdgcn_rcpf(1.0f + e);
}

__device__ __forceinline__ int row_of_rf1(int rf, int g, int j) {
    if (rf < FI) return (g * 4 + j) * NS + rf * 16;
    rf -= FI;
    if (rf < 3 * FI1) { const int i = rf / FI1; return (g * 4 + ((j + 1 + i) & 3)) * NS + (rf - i * FI1) * 16; }
    rf -= 3 * FI1;
    const int q = rf / FX;
    return ((g ^ 1) * 4 + q) * NS + (FI + rf - q * FX) * 16;
}
__device__ __forceinline__ int src_of_rf1(int rf, int g, int j) { return row_of_rf1(rf, g, j) / NS; }

constexpr int IN_AUX = 2;
constexpr int OUT_AUX = 17;
__device__ __forceinline__ u32x4 cvt_w(const f32x4* v, int c) {
    return cvt8((f32x4){v[0][c], v[1][c], v[2][c], v[3][c]}, (f32x4){v[4][c], v[5][c], v[6][c], v[7][c]});
}

#define DPP_STEP(op, v, ctrl, rmask) v = op(v, (u32)__builtin_amdgcn_update_dpp(0, (int)v, ctrl, rmask, 0xf, false))
__device__ __forceinline__ u32 umax2(u32 a, u32 b) { return a > b ? a : b; }
__device__ __forceinline__ u32 uadd2(u32 a, u32 b) { return a + b; }
__device__ __forceinline__ u32 wave_max(u32 v) {
    DPP_STEP(umax2, v, 0x111, 0xf); DPP_STEP(umax2, v, 0x112, 0xf); DPP_STEP(umax2, v, 0x114, 0xf); DPP_STEP(umax2, v, 0x118, 0xf);
    DPP_STEP(umax2, v, 0x142, 0xa); DPP_STEP(umax2, v, 0x143, 0xc);
    return __builtin_amdgcn_readlane(v, 63);
}
__device__ __forceinline__ u32 wave_sum(u32 v) {
    DPP_STEP(uadd2, v, 0x111, 0xf); DPP_STEP(uadd2, v, 0x112, 0xf); DPP_STEP(uadd2, v, 0x114, 0xf); DPP_STEP(uadd2, v, 0x118, 0xf);
    DPP_STEP(uadd2, v, 0x142, 0xa); DPP_STEP(uadd2, v, 0x143, 0xc);
    return __builtin_amdgcn_readlane(v, 63);
}

__device__ __forceinline__ u32 wave_scan(u32 v) {
    DPP_STEP(uadd2, v, 0x111, 0xf); DPP_STEP(uadd2, v, 0x112, 0xf); DPP_STEP(uadd2, v, 0x114, 0xf); DPP_STEP(uadd2, v, 0x118, 0xf);
    DPP_STEP(uadd2, v, 0x142, 0xa); DPP_STEP(uadd2, v, 0x143, 0xc);
    return v;
}
__device__ __forceinline__ u32 byte_scan(u32 m) {
    u32 a, r;
    asm("v_lshl_add_u32 %0, %1, 8, %1" : "=v"(a) : "v"(m));
    asm("v_lshl_add_u32 %0, %1, 16, %1" : "=v"(r) : "v"(a));
    return r;
}
__device__ __forceinline__ u32 times7(u32 m) {
    u32 a, r;
    asm("v_lshl_add_u32 %0, %1, 1, %1" : "=v"(a) : "v"(m));
    asm("v_lshl_add_u32 %0, %1, 2, %2" : "=v"(r) : "v"(m), "v"(a));
    return r;
}
__device__ __forceinline__ u32 rank_in(u64 m) { return __builtin_amdgcn_mbcnt_hi((u32)(m >> 32), __builtin_amdgcn_mbcnt_lo((u32)m, 0)); }

constexpr int VAR_X = VAR_XO, VAR_W = 256;
__device__ __forceinline__ u32 h7_of(float a) { return (u32)__builtin_bit_cast(unsigned short, (_Float16)a) >> 8; }
__device__ __forceinline__ void lds_or(LAS char* a, u32 v) {
    __hip_atomic_fetch_or((LAS u32*)a, v, __ATOMIC_RELAXED, __HIP_MEMORY_SCOPE_WORKGROUP);
}
__device__ __forceinline__ void put_bits(LAS char* area, u32 bo, u64 v) {
    const u32 sh = bo & 31;
    const u64 w = v << sh;
    LAS char* a = area + (bo >> 5) * 4;
    lds_or(a, (u32)w); lds_or(a + 4, (u32)(w >> 32)); lds_or(a + 8, (u32)((v >> 1) >> (63 - sh)));
}
__device__ __forceinline__ u64 get_bits(const LAS char* area, u32 bo) {
    const LAS u32* w = (const LAS u32*)(area + (bo >> 5) * 4);
    const u32 w0 = w[0], w1 = w[1], w2 = w[2];
    return (u64)__builtin_amdgcn_alignbit(w2, w1, bo) << 32 | __builtin_amdgcn_alignbit(w1, w0, bo);
}
template <class F>
__device__ __forceinline__ void pack_item(const u32x4 (&h)[4], float amax, LAS char* scr, rsrc_t r0, u32 dst, int lane, F before_stores) {
    u32 L[8], H[8];
#pragma unroll
    for (int t = 0; t < 8; ++t) {
        const u32 d0 = h[t >> 1][(t & 1) * 2], d1 = h[t >> 1][(t & 1) * 2 + 1];
        L[t] = __builtin_amdgcn_perm(d1, d0, 0x06040200u);
        H[t] = __builtin_amdgcn_perm(d1, d0, 0x07050301u);
    }
    *(LAS u32x4*)(scr + lane * 16) = (u32x4){H[0], H[1], H[2], H[3]};
    *(LAS u32x4*)(scr + lane * 16 + 1024) = (u32x4){H[4], H[5], H[6], H[7]};
    LAS char* area = scr + 2048;
    *(LAS u32x2v*)(area + lane * 8) = (u32x2v){0u, 0u};
    const u32 lm = wave_sum(h7_of(amax) & 127) + 132, rare = (lm & 255) < 115 ? 1 : 0;
    const u32 E1 = lm < 256 ? 1 : lm < 31 * 256 ? lm >> 8 : 31;
    const u32 eb = E1 * 0x01010101u | 0x80808080u;
    const u32 tbig = rare ? 0x79797979u : 0x7a7a7a7au, vadd = rare ? 0x01010101u : 0x02020202u, top6 = rare ? 0x06060606u : 0u;
    const u32 clo = 0x02010003u, qlo = rare ? 3u : 2u, qhi = rare ? 0x03020100u : 0x03030100u;
    u32 K[2] = {0, 0}, e4 = 0, pos = 0;
    u64 b2 = 0, b3 = 0;
#pragma unroll
    for (int t = 0; t < 8; ++t) {
        const u32 o = (eb - ((H[t] >> 2) & 0x1f1f1f1fu)) ^ 0x80808080u, o7 = o & 0x7f7f7f7fu;
        const u32 far = (o | (o7 + tbig)) & 0x80808080u, huge = (o | (o7 + 0x73737373u)) & 0x80808080u;
        const u32 f7 = (far >> 4) - (far >> 7), sel = (o | f7) & 0x07070707u;
        const u32 c = __builtin_amdgcn_perm(0x03030303u, clo, sel);
        const u32 q = __builtin_amdgcn_perm(qhi, qlo, sel);
        const u32 M = __builtin_amdgcn_perm(0x01010101u, 0x00000001u, sel);
        const u32 M2 = q & (q >> 1);
        const u32 v = (((o7 + vadd) | ((huge >> 4) - (huge >> 7))) & f7) | __builtin_amdgcn_perm(0u, top6 & 255, sel);
        const u32 P = byte_scan(M), P2 = byte_scan(M2), X2 = P2 - M2;
        const u32 bits2 = __builtin_amdgcn_udot4(q, __builtin_amdgcn_perm(0u, 0x40100401u, P - M), 0u, false);
        const u32 bits3 = __builtin_amdgcn_udot4(v, __builtin_amdgcn_perm(0u, 0x00400801u, X2), (X2 >> 24) == 3 ? (v >> 24) << 9 : 0u, false);
        b2 |= (u64)bits2 << (pos & 63);
        b3 |= (u64)bits3 << ((pos >> 16) & 63);
        pos += (P >> 23) + ((P2 >> 24) * 3 << 16);
        K[t >> 2] |= c << (2 * (t & 3));
        e4 = (e4 >> 1) | huge;
        if (t & 1) { asm volatile("" : "+v"(b2), "+v"(b3), "+v"(e4)); __builtin_amdgcn_sched_barrier(0); }
    }
    const u32 incl = wave_scan(pos), tot = __builtin_amdgcn_readlane(incl, 63), ex = incl - pos;
    const u32 used = 8 + (tot & 0xffff) + (tot >> 16);
    u32 need = used;
    if (__builtin_amdgcn_ballot_w64(e4 != 0) != 0) need += 8 * wave_sum(__builtin_popcount(e4));
    const u32 var = need <= 8 * VAR_X ? VAR_X : VAR_W;
    const bool fits = need <= 8 * VAR_W && __builtin_amdgcn_ballot_w64(pos >= (65u << 16)) == 0;
    if (fits) {
        u32 n4 = 0;
        put_bits(area, 8 + (ex & 0xffff), b2);
        put_bits(area, 8 + (tot & 0xffff) + (ex >> 16), b3);
        while (__builtin_amdgcn_ballot_w64(e4 != 0) != 0) {
            const u64 act = __builtin_amdgcn_ballot_w64(e4 != 0);
            const u32 any = (e4 | (e4 >> 8) | (e4 >> 16) | (e4 >> 24)) & 255, t = __builtin_ctz(any | 128);
            const u32 b = __builtin_ctz(((e4 >> t) & 0x01010101u) | 0x80000000u) + t;
            const u32 e = (*(const LAS unsigned char*)(scr + (t & 4) * 256 + lane * 16 + (t & 3) * 4 + ((b >> 3) & 3)) >> 2) & 31;
            const u32 k = var - 1 - n4 - rank_in(act);
            if (e4 != 0) lds_or(area + (k & ~3u), e << ((k & 3) * 8));
            e4 &= ~(1u << (b & 31));
            n4 += __builtin_popcountll(act);
        }
    }
    asm volatile("" ::: "memory");
    u32 vw = *(const LAS u32*)(area + lane * 4);
    if (lane == 0) vw |= fits ? E1 | rare << 5 | (var == VAR_W ? 64 : 0) : 255;
    const u32 P0 = (H[0] & 0x83838383u) | ((H[1] & 0x03030303u) << 2) | ((H[1] >> 1) & 0x40404040u) | ((H[6] & 0x03030303u) << 4);
    const u32 P1 = (H[2] & 0x83838383u) | ((H[3] & 0x03030303u) << 2) | ((H[3] >> 1) & 0x40404040u) | ((H[7] & 0x03030303u) << 4);
    const u32 P2 = (H[4] & 0x83838383u) | ((H[5] & 0x03030303u) << 2) | ((H[5] >> 1) & 0x40404040u)
                 | ((H[6] >> 2) & 0x20202020u) | ((H[7] >> 3) & 0x10101010u);
    before_stores(fits);
    st_sys((u32x4){L[0], L[1], L[2], L[3]}, r0, lane * 16, dst);
    st_sys((u32x4){L[4], L[5], L[6], L[7]}, r0, lane * 16, dst + 1024);
    st_sys((u32x4){P0, P1, P2, K[0]}, r0, lane * 16, dst + 2048);
    __builtin_amdgcn_raw_buffer_store_b32(K[1], r0, lane * 4 + dst + 3072, 0, 17);
    __builtin_amdgcn_raw_buffer_store_b32(vw, r0, lane * 4 + dst + 3328, 0, 17);
}
__device__ __forceinline__ void unpack_item(u32x4 (&h)[4], const u32x4& c0, const u32x4& c1, const u32x4& c2, u32 k1,
                                            const LAS char* area, u32 hd, LAS char* scr, int lane) {
    const u32 E1 = hd & 31, var = (hd & 64) ? VAR_W : VAR_X;
    const bool rare = hd & 32;
    const u32 K[2] = {c2[3], k1};
    const u32 X[2] = {K[0] & (K[0] >> 1) & 0x55555555u, K[1] & (K[1] >> 1) & 0x55555555u};
    const u32 n2 = 2 * (__builtin_popcount(X[0]) + __builtin_popcount(X[1])), i2 = wave_scan(n2);
    u64 b2 = get_bits(area, 8 + i2 - n2);
    b2 &= n2 < 64 ? ~(~0ull << n2) : ~0ull;
    const u64 y2 = b2 & (b2 >> 1) & 0x5555555555555555ull;
    const u32 n3 = 3 * __builtin_popcountll(y2), i3 = wave_scan(n3);
    u64 b3 = get_bits(area, 8 + __builtin_amdgcn_readlane(i2, 63) + i3 - n3);
    const u32 e4x = E1 * 0x04040404u | 0x80808080u;
    const u32 t1 = (e4x - 0x000c0804u) & 0x007c7c7cu, t2 = (e4x - (rare ? 0x00181410u : 0x00001410u)) & 0x007c7c7cu;
    const u32 t3lo = (e4x - (rare ? 0x2824201cu : 0x24201c18u)) & 0x7c7c7c7cu, t3hi = (e4x - (rare ? 0x0000302cu : 0x00302c28u)) & 0x007c7c7cu;
    LAS char* row = scr + lane * 36;
    u32 e4 = 0;
    const u32 nch = (wave_max(n2) + 7) >> 3;
#pragma unroll
    for (int k = 0; k < 8; ++k) {
        if (k >= 4 && k >= nch) break;
        const u32 b8 = (u32)(b2 >> (8 * k)) & 255, y = b8 | (b8 << 12), q = (y | (y << 6)) & 0x03030303u;
        const u32 M2 = q & (q >> 1), P2 = byte_scan(M2), m7 = times7(M2);
        const u32 lo = (u32)b3, z = (lo & 0x3f) | ((lo & 0xfc0) << 10), f = (z | (z << 5)) & 0x07070707u;
        const u32 v = __builtin_amdgcn_perm(0u, f, P2 - M2) & m7;
        b3 >>= (P2 >> 24) * 3;
        *(LAS u32*)(row + 4 * k) = __builtin_amdgcn_perm(0u, t2, q) | __builtin_amdgcn_perm(t3hi, t3lo, v | (0x0c0c0c0cu - (M2 << 2) - (M2 << 3)));
        e4 |= (((v + M2) >> 3) & 0x01010101u) << k;
    }
    u32 n4 = 0;
    while (__builtin_amdgcn_ballot_w64(e4 != 0) != 0) {
        const u64 act = __builtin_amdgcn_ballot_w64(e4 != 0);
        const u32 any = (e4 | (e4 >> 8) | (e4 >> 16) | (e4 >> 24)) & 255, t = __builtin_ctz(any | 128);
        const u32 b = __builtin_ctz(((e4 >> t) & 0x01010101u) | 0x80000000u) + t;
        const u32 x = *(const LAS unsigned char*)(area + ((var - 1 - n4 - rank_in(act)) & 255));
        if (e4 != 0) *(LAS unsigned char*)(row + t * 4 + (b >> 3)) = x << 2;
        e4 &= ~(1u << (b & 31));
        n4 += __builtin_popcountll(act);
    }
    const u32 SM[8] = {c2[0] & 0x83838383u, ((c2[0] << 1) & 0x80808080u) | ((c2[0] >> 2) & 0x03030303u),
                       c2[1] & 0x83838383u, ((c2[1] << 1) & 0x80808080u) | ((c2[1] >> 2) & 0x03030303u),
                       c2[2] & 0x83838383u, ((c2[2] << 1) & 0x80808080u) | ((c2[2] >> 2) & 0x03030303u),
                       ((c2[2] << 2) & 0x80808080u) | ((c2[0] >> 4) & 0x03030303u),
                       ((c2[2] << 3) & 0x80808080u) | ((c2[1] >> 4) & 0x03030303u)};
    u32 M[8], P[8], g[8], at = 0;
#pragma unroll
    for (int t = 0; t < 8; ++t) {
        M[t] = (X[t >> 2] >> (2 * (t & 3))) & 0x01010101u;
        P[t] = byte_scan(M[t]);
        const LAS u32* w = (const LAS u32*)(row + (at & ~3u));
        g[t] = __builtin_amdgcn_alignbyte(w[1], w[0], at);
        at += P[t] >> 24;
    }
#pragma unroll
    for (int t = 0; t < 8; ++t) {
        const u32 c = (K[t >> 2] >> (2 * (t & 3))) & 0x03030303u;
        const u32 m = times7(M[t]);
        const u32 H = SM[t] | __builtin_amdgcn_perm(t1, g[t], ((P[t] - M[t]) & m) | ((c | 0x04040404u) & ~m));
        const u32 L = t < 4 ? c0[t & 3] : c1[t & 3];
        h[t >> 1][(t & 1) * 2] = __builtin_amdgcn_perm(H, L, 0x05010400u);
        h[t >> 1][(t & 1) * 2 + 1] = __builtin_amdgcn_perm(H, L, 0x07030602u);
    }
}

__device__ __forceinline__ u32 sel_lo(u32 sg, u32 lop, u32 lon) {
    u32 m = sg << 8;
    asm("" : "+v"(m));
    m -= sg;
    return (lon * 0x01010101u & m) | (lop * 0x01010101u & ~m);
}
template <int N>
__device__ __forceinline__ void out_windows(const u32x4 (&o)[N][4], u32& lop, u32& lon) {
    u32 mn = 0, ntop = 0;
    int mp = 0;
#pragma unroll
    for (int i = 0; i < 16 * N; ++i) { mp = max(mp, (int)o[i >> 4][(i >> 2) & 3][i & 3]); mn = umax2(mn, o[i >> 4][(i >> 2) & 3][i & 3]); }
    u32 ep = wave_max((u32)mp) >> 23;
    const u32 en = (wave_max(mn) >> 23) & 255;
    ntop = __builtin_popcountll(__builtin_amdgcn_ballot_w64(mp >= (int)(ep << 23)));
    ep -= ntop <= 8 && ep > 0;
    lop = ep > 7 ? ep - 7 : 0; lon = en > 7 ? en - 7 : 0;
}
__device__ __forceinline__ void pack_out(const u32x4 (&o)[4], u32 lop, u32 lon, LAS char* scr, rsrc_t r0, u32 dst, int lane) {
    u32 M[12], S[4];
    u32x4 E;
#pragma unroll
    for (int r = 0; r < 4; ++r) {
        u32 R[4];
#pragma unroll
        for (int c = 0; c < 4; ++c) R[c] = __builtin_amdgcn_alignbit(o[r][c], o[r][c], 31);
#pragma unroll
        for (int i = 0; i < 3; ++i) M[r * 3 + i] = __builtin_amdgcn_perm(R[3], R[i], 0x04020100u + (i << 24));
        E[r] = __builtin_amdgcn_perm(R[1], R[0], 0x0c0c0703u) | __builtin_amdgcn_perm(R[3], R[2], 0x07030c0cu);
        S[r] = (__builtin_amdgcn_perm(R[1], R[0], 0x0c0c0400u) | __builtin_amdgcn_perm(R[3], R[2], 0x04000c0cu)) & 0x01010101u;
    }
    *(LAS u32x4*)(scr + lane * 16) = E;
    u32 G2 = 0, G1 = 0, em = 0;
#pragma unroll
    for (int r = 0; r < 4; ++r) {
        const u32 lo = sel_lo(S[r], lop, lon);
        const u32 d = ((E[r] | 0x80808080u) - (lo & 0x7f7f7f7fu)) ^ ((E[r] ^ ~lo) & 0x80808080u);
        const u32 nz = (d | ((d & 0x78787878u) + 0x78787878u)) & 0x80808080u;
        const u32 g = d & 0x07070707u & ~((nz >> 4) - (nz >> 7));
        G2 |= (g & 0x03030303u) << (2 * r);
        G1 |= (g & 0x04040404u) >> 2 << r;
        em = (em >> 1) | (~(g + 0x7f7f7f7fu) & 0x80808080u);
    }
    LAS unsigned char* list = (LAS unsigned char*)(scr + 1024);
    u32 used = 2;
    for (u64 act = __builtin_amdgcn_ballot_w64(em != 0); act != 0; act = __builtin_amdgcn_ballot_w64(em != 0)) {
        const u32 k = used + rank_in(act), b = __builtin_ctz(em | 0x80000000u);
        list[em != 0 && k < 64 ? k : 64] = *(const LAS unsigned char*)(scr + lane * 16 + (b & 3) * 4 + (b >> 3));
        em &= em - 1;
        used += __builtin_popcountll(act);
        if (used > 64) break;
    }
    *(LAS unsigned short*)list = (used > 64 ? 255 : lop) | lon << 8;
    asm volatile("" ::: "memory");
    const u32 ent = *(const LAS u32*)(list + (lane & 15) * 4);
    if (used > 64) {
#pragma unroll
        for (int r = 0; r < 4; ++r) st_sys(o[r], r0, lane * 16 + r * 1024, dst + STG_BYTES);
    }
#pragma unroll
    for (int i = 0; i < 3; ++i) st_sys((u32x4){M[4 * i], M[4 * i + 1], M[4 * i + 2], M[4 * i + 3]}, r0, lane * 16 + i * 1024, dst);
    __builtin_amdgcn_raw_buffer_store_b32(G2, r0, lane * 4 + dst + 3072, 0, 17);
    __builtin_amdgcn_raw_buffer_store_b16((unsigned short)__builtin_amdgcn_perm(0u, G1 | (G1 >> 4), 0x0c0c0200u), r0, lane * 2 + dst + 3328, 0, 17);
    __builtin_amdgcn_raw_buffer_store_b32(ent, r0, (lane < 16 ? lane * 4 : 0x80000000u) + dst + 3456, 0, 17);
}
__device__ __forceinline__ void unpack_out(u32x4 (&o)[4], const u32x4& c0, const u32x4& c1, const u32x4& c2, u32 G2, u32 g1,
                                           const LAS unsigned char* list, u32 lop, u32 lon, LAS char* scr, int lane) {
    const u32 M[12] = {c0[0], c0[1], c0[2], c0[3], c1[0], c1[1], c1[2], c1[3], c2[0], c2[1], c2[2], c2[3]};
    const u32 t = __builtin_amdgcn_perm(0u, g1, 0x0c010c00u), G1 = t | (t << 4);
    u32x4 E;
    u32 em = 0;
#pragma unroll
    for (int r = 0; r < 4; ++r) {
        const u32 sg = (__builtin_amdgcn_perm(M[r * 3 + 1], M[r * 3], 0x030c0400u) | __builtin_amdgcn_perm(0u, M[r * 3 + 2], 0x0c000c0cu)) & 0x01010101u;
        const u32 g = ((G2 >> (2 * r)) & 0x03030303u) | ((G1 >> r << 2) & 0x04040404u);
        const u32 nz = (g + 0x7f7f7f7fu) & 0x80808080u;
        E[r] = (sel_lo(sg, lop, lon) + g) & ((nz << 1) - (nz >> 7));
        em = (em >> 1) | (nz ^ 0x80808080u);
    }
    *(LAS u32x4*)(scr + lane * 16) = E;
    u32 used = 2;
    auto step = [&](u32& a, u32& val) {
        const u64 act = __builtin_amdgcn_ballot_w64(em != 0);
        const u32 b = __builtin_ctz(em | 0x80000000u);
        a = em != 0 ? lane * 16 + (b & 3) * 4 + (b >> 3) : 1024;
        val = list[(used + rank_in(act)) & 63];
        em &= em - 1;
        used += __builtin_popcountll(act);
    };
    do {
        u32 a0, a1, a2, x0, x1, x2;
        step(a0, x0); step(a1, x1); step(a2, x2);
        *(LAS unsigned char*)(scr + a0) = x0; *(LAS unsigned char*)(scr + a1) = x1; *(LAS unsigned char*)(scr + a2) = x2;
    } while (__builtin_amdgcn_ballot_w64(em != 0) != 0);
    E = *(const LAS u32x4*)(scr + lane * 16);
#pragma unroll
    for (int r = 0; r < 4; ++r) {
        u32 R[4];
#pragma unroll
        for (int i = 0; i < 3; ++i) R[i] = __builtin_amdgcn_perm(E[r], M[r * 3 + i], 0x04020100u + (i << 24));
        R[3] = __builtin_amdgcn_perm(M[r * 3 + 1], M[r * 3], 0x0c0c0703u) | __builtin_amdgcn_perm(E[r], M[r * 3 + 2], 0x07030c0cu);
#pragma unroll
        for (int c = 0; c < 4; ++c) o[r][c] = __builtin_amdgcn_alignbit(R[c], R[c], 1);
    }
}

__device__ __forceinline__ void load_item(f32x4 (&v)[8], const Params& p, int u, int wid, int lane) {
    const int blk = u >> 4, it = (u & 15) * 8 + wid;
    const bool isx = it < 64;
    const rsrc_t r = mk_rsrc(isx ? p.x : p.w, OUT_BYTES);
    const u32 vf = isx ? (lane & 15) * 32768 + (lane >> 4) * 16 : (lane >> 4) * 16384 + (lane & 15) * 16;
    const u32 base = isx ? it * 524288 + blk * 512 : (blk * 128 + (wid & 3) * 32) * 4096 + ((it - 64) >> 2) * 256;
    const u32 sa = isx ? 64 : 4096, sb = isx ? 256 : 65536;
#pragma unroll
    for (int i = 0; i < 8; ++i)
        v[i] = __builtin_bit_cast(f32x4, __builtin_amdgcn_raw_buffer_load_b128(r, vf, base + (i & 3) * sa + (i >> 2) * sb, IN_AUX));
}
__device__ __forceinline__ void convert_local(const Params& p, LAS char* lds, f32x4 (&v)[8], char* wsl, int wg, int wid, int lane, u32 call) {
    const int rank = rfl(p.rank);
    constexpr int NU = NBLK * 16;
    const u32 par = OFF_PAR + (call & 1) * PAR_BYTES;
    const rsrc_t r0 = mk_rsrc(p.ws[rank], WS_NEED);
    LAS char* scr = lds + wid * 4096;
    auto publish = [&](int u) {
        if (wid == 0) {
            if (lane < 8 && lane != rank) st_flag(wsl + OFF_RDY + rank * 8192 + u * 4, call);
            if (lane == 8) add_sys(p.ws[rank] + OFF_CNT + (u >> 4) * CNT_BLK + ((u & 8) ? 1024 : 0) + rank * 128);
        }
    };
    int u = wg, un = wg + NGEMM, up = -1;
    WAIT_VM0();
    for (int r = 0; u < NU; ++r) {
        const int blk = u >> 4, grp = u & 15, it = grp * 8 + wid;
        u32x4 h[4];
        u32 base, step;
        if (grp < 8) {
            base = par + (u32)blk * (RS1 * 4096) + it * 4096; step = 1024;
#pragma unroll
            for (int i = 0; i < 4; ++i) h[i] = cvt8(v[2 * i], v[2 * i + 1]);
        } else {
            base = par + REL_B + (u32)blk * (CF * 4096) + ((it - 64) & 60) * 4096 + (wid & 3) * 1024; step = 4096;
#pragma unroll
            for (int c = 0; c < 4; ++c) h[c] = cvt_w(v, c);
        }
        float amax = 0.0f;
#pragma unroll
        for (int i = 0; i < 8; ++i) amax = fmaxf(fmaxf(fmaxf(amax, fabsf(v[i][0])), fmaxf(fabsf(v[i][1]), fabsf(v[i][2]))), fabsf(v[i][3]));
        STAMPB_IF(r == 0, 0);
        if (r > 0) load_item(v, p, (u32)un < (u32)NU ? un : NU - 1, wid, lane);
        pack_item(h, amax, scr, r0, par + REL_PK + (u32)(blk * 128 + it) * 4096, lane, [&](bool fits) {
            WAIT_VM0();
            if (r > 0) {
                __builtin_amdgcn_s_barrier();
                if (up >= 0) publish(up);
            }
            if (!fits || it < FI + FL || it >= 64) {
#pragma unroll
                for (int i = 0; i < 4; ++i) st_sys(h[i], r0, lane * 16, base + i * step);
            }
        });
        up = u;
        if (r == 0) {
            STAMPB_IF(true, 1);
            WAIT_VM0();
            STAMPB_IF(true, 2);
            __builtin_amdgcn_s_barrier();
            STAMP(6);
            publish(u);
            up = -1;
            load_item(v, p, un, wid, lane);
            WAIT_VM0();
        }
        u = un;
        un += NGEMM;
    }
    WAIT_VM0();
    __builtin_amdgcn_s_barrier();
    if (up >= 0) publish(up);
}

constexpr int PUSH_PER = FI1 + 64 + XA, NPUSH = PB * PUSH_PER, NSPARE = NCOMM * 5;
__device__ __forceinline__ void push_decode(int n, int& blk, int& it) {
    blk = n / PUSH_PER; it = n - blk * PUSH_PER;
    if (it >= FI1 + 64) { const int xi = blk * XA + it - FI1 - 64; blk = xi / FX; it = FI + xi - blk * FX; }
    else if (it >= FI1) it += 64 - FI1;
}
__device__ __forceinline__ void push_role(const Params& p, f32x4 (&v)[8], char* wsl, int sw, int lane, u32 call) {
    const int rank = rfl(p.rank);
    const int g = rank >> 2, j = rank & 3, og = (g ^ 1) * 4;
    const u32 par = OFF_PAR + (call & 1) * PAR_BYTES;
    for (int n = sw; n < NPUSH; n += NSPARE) {
        int blk, it;
        push_decode(n, blk, it);
        u32x4 h[4];
        int d0 = 0, nd = 4, dstep = 1; u32 base, step = 1024, ostep = 0;
        if (it < 64) {
#pragma unroll
            for (int i = 0; i < 4; ++i) h[i] = cvt8(v[2 * i], v[2 * i + 1]);
            if (it < FI1) { d0 = j + 3; nd = 3; dstep = 3; ostep = FI1 * 4096; base = par + (blk * RS1 + 64 + it) * 4096; }
            else base = par + (blk * RS1 + 64 + 3 * FI1 + j * FX + it - FI) * 4096;
        } else {
#pragma unroll
            for (int c = 0; c < 4; ++c) h[c] = cvt_w(v, c);
            d0 = (it - 64) >> 4; nd = 1; step = 4096;
            base = par + REL_B + (blk * CF + 64 + 16 * j + (it & 12)) * 4096 + (it & 3) * 1024;
        }
        if (n + NSPARE < NPUSH) { int b2, i2; push_decode(n + NSPARE, b2, i2); load_item(v, p, b2 * 16 + (i2 >> 3), i2 & 7, lane); }
        u32 dmask = 0;
        for (int d = 0; d < nd; ++d) {
            const int dr = (it < FI1 ? g * 4 : og) + ((d0 + d * dstep) & 3);
            const rsrc_t rd = mk_rsrc(p.ws[dr], WS_NEED);
#pragma unroll
            for (int i = 0; i < 4; ++i) st_sys(h[i], rd, lane * 16, base + d * ostep + i * step);
            dmask |= 1u << dr;
        }
        WAIT_VM0();
        if (lane < 8 && ((dmask >> lane) & 1)) add_sys(wsl + OFF_CNT + blk * CNT_BLK + (it < 64 ? 0 : 1024) + rank * 128);
    }
}

#define P_NDEC 9
#define P_DSLEEP 2
constexpr int SCR = 2560;
constexpr int NLW = 3, RING = 6, QLEN = 32, NDEC = P_NDEC;
constexpr int LDS_CTL = NLW * RING * 4096;
constexpr int LDS_DONE = LDS_CTL + QLEN * 16, LDS_TAIL = LDS_DONE + NLW * RING * 4, LDS_SCR = LDS_CTL + 1024;
static_assert(7 * WL <= NLW * NCOMM, "loaders");
constexpr u32 IDX_END = 0xffffffffu, IDX_BAD = 0x80000000u, IDX_CROSS = 0x40000000u;
__device__ __forceinline__ u32 lds_get(const LAS char* a) {
    u32 v;
    asm volatile("ds_read_b32 %0, %1\n\ts_waitcnt lgkmcnt(0)" : "=v"(v) : "v"((u32)(size_t)a) : "memory");
    return rfl(v);
}
__device__ __forceinline__ void lds_put(LAS char* a, u32 v) {
    asm volatile("" ::: "memory");
    *(volatile LAS u32*)a = v;
    asm volatile("" ::: "memory");
}

struct Item { u32 idx, dst; };
constexpr int N1 = NBLK * FI1, NT_I = N1 + NBLK * F2;
constexpr int PER = 16 + XA, NP = NBLK * PER, NX = NBLK * (16 + FX), NO1 = (RF1 - FL) * 4, NT_C = NX + NO1 + 192;
constexpr int TAB_BYTES = (NT_I > NT_C ? NT_I : NT_C) * 4, LDS_TAB = LDS_SCR + NDEC * SCR;
static_assert(LDS_TAB + NLW * TAB_BYTES <= LDS_BYTES, "communication LDS");
__device__ __forceinline__ void build_table(LAS char* tab, int rank, int slot, int first, int stride) {
    const int g = rank >> 2, j = rank & 3, q = slot - 3;
    for (int it = first; it < (slot < 3 ? NT_I : NT_C); it += stride) {
        Item m;
        if (slot < 3) {
            const bool s2 = it >= N1;
            const int a = it - N1, b1 = it / FI1;
            const int blk = s2 ? a >> 4 : b1, s = s2 ? FI1 + (a & 15) : it - b1 * FI1;
            m.idx = blk * 128 + s;
            m.dst = (s2 ? REL_A2 + (slot * F2 - FI1) * 4096 : (64 + slot * FI1) * 4096) + (blk * (s2 ? RF2 : RS1) + s) * 4096;
        } else if (it < NX) {
            const int blk = it / PER, c = it - blk * PER;
            const bool tail = it >= NP, isx = tail || (c < 2 * XA && (c & 1));
            const int cw = c < 2 * XA ? c >> 1 : c - XA;
            const int xi = tail ? it - NP + NBLK * XA : blk * XA + (c >> 1), xb = xi / FX, xs = FI + xi - xb * FX;
            m.idx = isx ? xb * 128 + xs : blk * 128 + 64 + 16 * j + cw;
            m.dst = isx ? (xb * RS1 + xs + 64 + 3 * FI1 + q * FX - FI) * 4096
                        : REL_B + (blk * CF + 64 + 16 * q + (cw & 12)) * 4096 + (cw & 3) * 1024;
        } else {
            const int k = it - NX;
            int row, col;
            if (k < NO1) {
                const int rf = (k >> 2) + ((k >> 2) >= FI + 3 * FI1 + j * FX ? FL : 0);
                m.idx = 8192 + (rf >> 4) * 64 + (k & 2) * 16 + ((rf >> 2) & 3) * 8 + (k & 1) * 4 + (rf & 3);
                row = row_of_rf1(rf, g ^ 1, q);
                col = (k & 3) * 64;
            } else {
                const int z = k - NO1, rt = z >> 5;
                m.idx = 8192 + 832 + z;
                row = ((g ^ 1) * 4 + ((q + 1 + (rt >> 1)) & 3)) * NS + (FI1 + (rt & 1) * 8 + (z & 7)) * 16;
                col = ((z >> 3) & 3) * 64;
            }
            m.dst = (u32)row * 4096 + (q * 256 + col) * 4;
        }
        *(LAS u32*)(tab + it * 4) = m.idx | m.dst << (m.idx < 8192 ? 4 : 6);
    }
}

template <bool CROSS>
__device__ __forceinline__ void load_role(const Params& p, LAS char* lds, int ld, int nld, int slot, int wv, int lane, u32 call) {
    const int rank = rfl(p.rank);
    const int g = rank >> 2, j = rank & 3;
    const int peer = CROSS ? (g ^ 1) * 4 + slot - 3 : g * 4 + ((j + 1 + slot) & 3);
    constexpr int NT = CROSS ? NT_C : NT_I;
    const u32 par = OFF_PAR + (call & 1) * PAR_BYTES;
    char* ws = uni(p.ws[rank]);
    const rsrc_t r0 = rsrc_u(ws, WS_NEED), rp = mk_rsrc(p.ws[peer], WS_NEED);
    const char* rdy = ws + OFF_RDY + peer * 8192;
    const u32 vo = lane * 16;
    constexpr int SKIP = PB * (CROSS ? PER : FI1);
    const u32 head = lane == 0 ? OFF_QUE + slot * 256 : WS_NEED, tbase = SKIP + 4 * WL - (call - 1) * (NT - SKIP);
    const bool nowait = (p.pad & 4) != 0;
    const LAS char* tab = lds + LDS_TAB + ld * TAB_BYTES;
    const LAS char* done = lds + LDS_DONE + ld * RING * 4;
    const u32 ldsb = rfl((u32)(size_t)lds) + ld * RING * 4096;
    auto table = [&](int it) {
        asm volatile("ds_read_b32 v160, %0" :: "v"((u32)(size_t)tab + (it < NT ? it : NT - 1) * 4) : "memory", "v160");
    };
    auto entry = [&] {
        u32 t;
        asm volatile("s_waitcnt lgkmcnt(0)\n\tv_readfirstlane_b32 %0, v160" : "=s"(t) :: "memory");
        Item m;
        m.idx = t & 16383;
        m.dst = (t - m.idx) >> (CROSS && m.idx >= 8192 ? 6 : 4);
        return m;
    };
    auto decode = [&](int it) { table(it); return entry(); };
    auto flag_of = [&](u32 idx) { return idx < 8192 ? idx >> 3 : (idx < 8192 + 832 ? 1024 + ((idx - 8192) >> 2) : idx - (8192 + 832 - 1024 - 208)); };
    u32 okupto = 0, wbase = 0; u64 wmask = 0;
    auto ready = [&](u32 fi) {
        if (fi < okupto) return true;
        if (fi - wbase < 64u && ((wmask >> (fi - wbase)) & 1)) return true;
        for (;;) {
            wbase = okupto;
            wmask = __builtin_amdgcn_ballot_w64(ld_sys(rdy + (wbase + lane) * 4) >= call);
            if (wmask != ~0ull) break;
            okupto += 64;
            if (fi < okupto) return true;
        }
        okupto += __builtin_ctzll(~wmask);
        if (fi - wbase < 64u) return ((wmask >> (fi - wbase)) & 1) != 0;
        return rfl(ld_sys(rdy + fi * 4)) >= call;
    };
    const u32 vl = vo < 256 + VAR_X ? vo : 0x80000000u;
    struct Prep { u32 src, m0v; };
    auto prep = [&](const Item& m, u32 n) {
        Prep r;
        r.src = par + REL_PK + (m.idx + (CROSS && m.idx >= 8192 ? j * 1024 : 0)) * 4096;
        r.m0v = ldsb + (n % RING) * 4096;
        asm volatile("" : "+s"(r.src), "+s"(r.m0v));
        return r;
    };
    auto issue = [&](const Prep& r) {
        asm volatile("s_mov_b32 m0, %0\n\ts_nop 0\n\t"
                     "buffer_load_dwordx4 %1, %3, %4 offen sc0 sc1 lds\n\t"
                     "buffer_load_dwordx4 %1, %3, %4 offen offset:1024 sc0 sc1 lds\n\t"
                     "buffer_load_dwordx4 %1, %3, %4 offen offset:2048 sc0 sc1 lds\n\t"
                     "buffer_load_dwordx4 %2, %3, %4 offen offset:3072 sc0 sc1 lds"
                     :: "s"(r.m0v), "v"(vo), "v"(vl), "s"(rp), "s"(r.src) : "memory");
    };
    u32 vtail = (u32)(size_t)lds + LDS_TAIL, vone = 1;
    asm volatile("" : "+v"(vtail), "+v"(vone));
    auto enqueue = [&](u32 idx, u32 dst, u32 n) {
        asm volatile("s_mov_b64 exec, 1\n\tds_add_rtn_u32 v162, %0, %1\n\tv_mov_b32 v159, %2\n\tv_mov_b32 v160, %3\n\tv_mov_b32 v161, %4\n\t"
                     "s_waitcnt lgkmcnt(0)\n\tv_add_u32 v158, 1, v162\n\tv_and_b32 v162, %6, v162\n\tv_lshl_add_u32 v162, v162, 4, %5\n\t"
                     "ds_write_b128 v162, v[158:161]\n\ts_mov_b64 exec, -1"
                     :: "v"(vtail), "v"(vone), "s"(idx), "s"(dst), "s"(n << 8 | ld << 4 | peer), "s"(rfl((u32)(size_t)lds) + LDS_CTL), "n"(QLEN - 1)
                     : "memory", "v158", "v159", "v160", "v161", "v162");
    };
    __builtin_amdgcn_s_setprio(3);
    int e = SKIP + wv, e1 = e + WL, e2 = e + 2 * WL;
    const int e3 = e + 3 * WL;
    Item nx = decode(e1), ia = decode(e);
    bool bad = spin_ge<2>(rdy + flag_of(ia.idx) * 4, call, ws + OFF_TMO, call);
    STAMP(5);
    issue(prep(ia, 0));
    bool pre = ready(flag_of(nx.idx));
    u32 n = 0;
    const u32 cls = CROSS ? IDX_CROSS : 0;
    auto wait_free = [&](u32 m) { while (lds_get(done + (m % RING) * 4) + RING <= m) __builtin_amdgcn_s_sleep(1); };
    auto body = [&](auto odd) {
        if constexpr (odd.value) asm volatile("v_mov_b32 v164, 1\n\ts_nop 0\n\tbuffer_atomic_add v164, %0, %1, 0 offen sc0" :: "v"(head), "s"(r0) : "memory", "v164");
        else                     asm volatile("v_mov_b32 v165, 1\n\ts_nop 0\n\tbuffer_atomic_add v165, %0, %1, 0 offen sc0" :: "v"(head), "s"(r0) : "memory", "v165");
        table(e2);
        u32 dn;
        asm volatile("ds_read_b32 v162, %1\n\ts_waitcnt lgkmcnt(0)\n\tv_readfirstlane_b32 %0, v162"
                     : "=s"(dn) : "v"((u32)(size_t)done + ((n + 1) % RING) * 4) : "memory", "v162");
        const Item nn = entry();
        bool pre2 = e2 < NT;
        if (pre2 && !(nowait && nn.idx >= 8192)) pre2 = ready(flag_of(nn.idx));
        if (dn + RING <= n + 1) wait_free(n + 1);
        const Prep px = prep(nx, n + 1);
        u32 told;
        if constexpr (odd.value) asm volatile("s_waitcnt vmcnt(1)\n\tv_readfirstlane_b32 %0, v165" : "=s"(told) :: "memory");
        else                     asm volatile("s_waitcnt vmcnt(1)\n\tv_readfirstlane_b32 %0, v164" : "=s"(told) :: "memory");
        if (pre) issue(px);
        enqueue(ia.idx | cls | (bad ? IDX_BAD : 0), ia.dst, n);
        if (!pre && e1 < NT) {
            bad |= spin_ge<2>(rdy + flag_of(nx.idx) * 4, call, ws + OFF_TMO, call);
            issue(px);
        }
        STAMP_IF(n == 0, 6);
        STAMP_IF(CROSS ? (e < NP && e1 >= NP) : (e < N1 && e1 >= N1), 1);
        STAMP_IF(CROSS && e < NX && e1 >= NX, 2);
        STAMP_IF(CROSS && e < NX + NO1 && e1 >= NX + NO1, 3);
        ia = nx; nx = nn; pre = pre2; ++n;
        e = e1; e1 = e2; e2 = n == 1 ? e3 : (int)(told + tbase);
    };
    for (;;) {
        body(std::false_type{});
        if (e >= NT) break;
        body(std::true_type{});
        if (e >= NT) break;
    }
    u32 fin;
    asm volatile("ds_add_rtn_u32 v162, %1, %2\n\ts_waitcnt lgkmcnt(0)\n\tv_readfirstlane_b32 %0, v162"
                 : "=s"(fin) : "v"((u32)(size_t)lds + LDS_TAIL + 4), "v"(lane == 0 ? 1u : 0u) : "memory", "v162");
    if (fin + 1 == (u32)nld)
        for (int k = 0; k < NDEC; ++k) enqueue(IDX_END, 0, 0);
}

__device__ __forceinline__ void decode_role(const Params& p, LAS char* lds, u32 me, int lane, u32 call) {
    const int rank = rfl(p.rank);
    const int j = rank & 3;
    const u32 par = OFF_PAR + (call & 1) * PAR_BYTES;
    const rsrc_t r0 = mk_rsrc(p.ws[rank], WS_NEED), ro = mk_rsrc(p.out[rank], OUT_BYTES);
    const u32 vo = lane * 16, vout = (lane >> 4) * 4 * 4096 + (lane & 15) * 16;
    LAS char* scr = lds + LDS_SCR + me * SCR;
    u32 q0, q1, q2, q3;
    auto peek = [&](u32 n) {
        asm volatile("ds_read_b128 v[158:161], %4\n\ts_waitcnt lgkmcnt(0)\n\tv_readfirstlane_b32 %0, v158\n\tv_readfirstlane_b32 %1, v159\n\t"
                     "v_readfirstlane_b32 %2, v160\n\tv_readfirstlane_b32 %3, v161"
                     : "=s"(q0), "=s"(q1), "=s"(q2), "=s"(q3) : "v"((u32)(size_t)lds + LDS_CTL + (n % QLEN) * 16) : "memory", "v158", "v159", "v160", "v161");
        return q0 == n + 1;
    };
    const u32 vsig = lane == 0 ? 0u : 0x80000000u;
    u32 one = 1;
    asm volatile("" : "+v"(one));
    auto signal = [&](u32 c) {
        asm volatile("s_nop 4\n\tbuffer_atomic_add %0, %1, %2, %3 offen sc1" :: "v"(one), "v"(vsig), "s"(r0), "s"(rfl(c)) : "memory");
    };
    const u32x4 nan4 = {0x7fc07fc0u, 0x7fc07fc0u, 0x7fc07fc0u, 0x7fc07fc0u};
    u32 vhead = (u32)(size_t)lds + LDS_TAIL + 8, vone = 1;
    asm volatile("" : "+v"(vhead), "+v"(vone));
    for (;;) {
        u32 n;
        asm volatile("s_mov_b64 exec, 1\n\tds_add_rtn_u32 v158, %1, %2\n\ts_waitcnt lgkmcnt(0)\n\tv_readfirstlane_b32 %0, v158\n\ts_mov_b64 exec, -1"
                     : "=s"(n) : "v"(vhead), "v"(vone) : "memory", "v158");
        while (!peek(n)) __builtin_amdgcn_s_sleep(P_DSLEEP);
        if (q1 == IDX_END) break;
        const u32 dst = q2, item = q3 >> 8, ring = (((q3 >> 4) & 3) * RING + item % RING), peer = q3 & 7;
        const LAS char* sl = lds + ring * 4096;
        const u32x4 c0 = *(const LAS u32x4*)(sl + vo), c1 = *(const LAS u32x4*)(sl + 1024 + vo), c2 = *(const LAS u32x4*)(sl + 2048 + vo);
        const u32 k1 = *(const LAS u32*)(sl + 3072 + lane * 4);
        const u32 hx = *(const LAS unsigned char*)(sl + 3328);
        const bool bad = (q1 & IDX_BAD) != 0;
        const u32 idx = q1 & 0xffff;
        const bool out = idx >= 8192, isw = !out && (idx & 64);
        const u32 step = (out || isw) ? 4096 : 1024;
        u32x4 h[4];
        bool done = false;
        if (!bad && !out) {
            const u32 hd = rfl(hx);
            if (hd != 255) {
                if (hd & 64) {
                    u32 t;
                    asm volatile("s_nop 4\n\tbuffer_load_dword %0, %1, %2, %3 offen sc0 sc1\n\ts_waitcnt vmcnt(0)"
                                 : "=&v"(t) : "v"(lane < 16 ? lane * 4 : 0x80000000u), "s"(mk_rsrc(p.ws[peer], WS_NEED)),
                                   "s"(rfl(par + REL_PK + idx * 4096 + 3328 + VAR_X)) : "memory");
                    if (lane < 16) *(LAS u32*)(lds + ring * 4096 + 3328 + VAR_X + lane * 4) = t;
                    asm volatile("" ::: "memory");
                }
                unpack_item(h, c0, c1, c2, k1, sl + 3328, hd, scr, lane);
                done = true;
            }
        } else if (!bad) {
            const u32 g1 = *(const LAS unsigned short*)(sl + 3328 + lane * 2), hd = rfl(*(const LAS unsigned short*)(sl + 3456));
            if ((hd & 255) != 255) { unpack_out(h, c0, c1, c2, k1, g1, (const LAS unsigned char*)(sl + 3456), hd & 255, hd >> 8, scr, lane); done = true; }
        }
        if (!done) {
            const u32 blk = idx >> 7, s = idx & 127;
            const u32 raw = par + (out ? REL_PK + STG_BYTES + (idx + j * 1024) * 4096
                                       : isw ? REL_B + (blk * CF + (s & 60)) * 4096 + (s & 3) * 1024 : (blk * RS1 + s) * 4096);
            const u32 rs = isw ? 4096 : 1024;
            const rsrc_t rp = mk_rsrc(p.ws[peer], WS_NEED);
            asm volatile("s_nop 4\n\tbuffer_load_dwordx4 %0, %4, %5, %6 offen sc0 sc1\n\tbuffer_load_dwordx4 %1, %4, %5, %7 offen sc0 sc1\n\t"
                         "buffer_load_dwordx4 %2, %4, %5, %8 offen sc0 sc1\n\tbuffer_load_dwordx4 %3, %4, %5, %9 offen sc0 sc1\n\ts_waitcnt vmcnt(0)"
                         : "=&v"(h[0]), "=&v"(h[1]), "=&v"(h[2]), "=&v"(h[3])
                         : "v"(vo), "s"(rp), "s"(rfl(raw)), "s"(rfl(raw + rs)), "s"(rfl(raw + 2 * rs)), "s"(rfl(raw + 3 * rs)) : "memory");
            if (bad) { h[0] = nan4; h[1] = nan4; h[2] = nan4; h[3] = nan4; }
        }
        lds_put(lds + LDS_DONE + ring * 4, item + 1);
        const rsrc_t rd = out ? ro : r0;
        const u32 vd = (out ? vout : vo + par) + dst;
#pragma unroll
        for (int i = 0; i < 4; ++i) st_sys(h[i], rd, vd, i * step);
        if (!out) {
            asm volatile("s_waitcnt vmcnt(0)" ::: "memory");
            signal(OFF_CNT + (idx >> 7) * CNT_BLK + peer * 128 + (isw ? 1024 : (!(q1 & IDX_CROSS) && (idx & 127) >= FI1) ? 2048 : 0));
        }
    }
    STAMP(4);
}

#define GLDS2(m0v, voff, rs, soff, imm0, imm1) \
    asm volatile("s_mov_b32 m0, %0\n\ts_nop 0\n\tbuffer_load_dwordx4 %1, %2, %3 offen offset:" #imm0 " sc0 sc1 lds\n\t" \
                 "buffer_load_dwordx4 %1, %2, %3 offen offset:" #imm1 " sc0 sc1 lds" \
                 :: "s"(m0v), "v"(voff), "s"(rs), "s"(soff) : "memory")
#define BARRIER() do { __builtin_amdgcn_sched_barrier(0); __builtin_amdgcn_s_barrier(); __builtin_amdgcn_sched_barrier(0); } while (0)

#define SCHED() __builtin_amdgcn_sched_barrier(0)
#define WAIT_LGKM0() __builtin_amdgcn_s_waitcnt(0xC07F)

template <int STAGE>
struct Gemm {
    static constexpr int NT = STAGE == 1 ? 128 : 64;
    static constexpr int NA = STAGE == 1 ? 4 : 2;
    static constexpr int NB = STAGE == 1 ? 4 : 8;
    static constexpr int NM = STAGE == 1 ? 4 : 1;
    static constexpr int NMM = STAGE == 1 ? 16 : 8;
    static constexpr int NRD = NA + NB;
    struct Frags { f16x8 a[NA], b[NB]; };

    rsrc_t rs;
    u32 sa, sa2, sb;
    u32 ablk;
    u32 ldsw;
    int wid, lane;

    template <int I> __device__ __forceinline__ void piece(int t, int buf) const {
        const u32 vo = lane * 16;
        const u32 ma = ldsw + buf * BUF_BYTES + wid * 4096, mb = ldsw + buf * BUF_BYTES + 32768 + wid * 2048;
        const u32 ka = STAGE == 1 ? (u32)(t >> 1) * (RS1 * 4096) + (t & 1) * 2048 : (u32)t * ablk;
        const u32 a = sa + ka, a2 = sa2 + ka;
        const u32 b = STAGE == 1 ? sb + (u32)(t >> 1) * (CF * 4096) + (t & 1) * 2048 : sb + (u32)t * (CF * 4096);
        if (I == 0) GLDS2(ma, vo, rs, a, 0, 1024);
        if (I == 1) GLDS2(ma, vo, rs, a2, 2048, 3072);
        if (I == 2) GLDS2(mb, vo, rs, b, 0, 1024);
    }
    __device__ __forceinline__ void stage(int t, int buf) const { piece<0>(t, buf); piece<1>(t, buf); piece<2>(t, buf); }

    template <int I> __device__ __forceinline__ void rd(Frags& f, const LAS char* lds, int buf, int half) const {
        const LAS char* base = lds + buf * BUF_BYTES + lane * 16;
        if constexpr (STAGE == 1) {
            const LAS char* pa = base + (wid >> 1) * 8192 + half * 1024;
            const LAS char* pb = base + 32768 + (wid & 1) * 8192 + half * 1024;
            if constexpr (I == 0) f.a[0] = *(const LAS f16x8*)pa;
            else if constexpr (I <= 4) f.b[I - 1] = *(const LAS f16x8*)(pb + (I - 1) * 2048);
            else f.a[I - 4] = *(const LAS f16x8*)(pa + (I - 4) * 2048);
        } else {
            const LAS char* pa = base + wid * 4096 + half * 2048;
            const LAS char* pb = base + 32768 + half * 2048;
            constexpr int h = I / 5, n = I % 5;
            if constexpr (n == 0) f.a[h] = *(const LAS f16x8*)(pa + h * 1024);
            else f.b[h * 4 + n - 1] = *(const LAS f16x8*)(pb + (n - 1) * 4096 + h * 1024);
        }
    }
    template <int I> __device__ __forceinline__ void mm(f32x4 (&acc)[NM][4], const Frags& f) const {
        if constexpr (STAGE == 1) acc[I >> 2][I & 3] = __builtin_amdgcn_mfma_f32_16x16x32_f16(f.a[I >> 2], f.b[I & 3], acc[I >> 2][I & 3], 0, 0, 0);
        else            acc[0][I & 3] = __builtin_amdgcn_mfma_f32_16x16x32_f16(f.a[I >> 2], f.b[I], acc[0][I & 3], 0, 0, 0);
    }
    template <int I, int N> __device__ __forceinline__ void rds(Frags& f, const LAS char* lds, int buf, int half) const {
        if constexpr (I < N) { rd<I>(f, lds, buf, half); rds<I + 1, N>(f, lds, buf, half); }
    }
    static __device__ __forceinline__ int blk_of(int t) { return STAGE == 1 ? (t >> 1) : t; }

    template <bool RD, int ST>
    __device__ __forceinline__ void half_step(f32x4 (&acc)[NM][4], const Frags& cur, Frags& nxt, const LAS char* lds, int rbuf, int rhalf,
                                              int ts, int bs, u32 cvo, u32 fso) const {
        auto first = [&] {
            if (ST == 1) asm volatile("buffer_load_dword v166, %0, %1, %2 offen sc0 sc1" :: "v"(cvo), "s"(rs), "s"(fso) : "memory", "v166");
            if (ST) { piece<0>(ts, bs); SCHED(); }
        };
        if constexpr (STAGE == 1) {
            mm<0>(acc, cur);  mm<1>(acc, cur);  if (RD) rds<0, 2>(nxt, lds, rbuf, rhalf); SCHED();
            mm<2>(acc, cur);  mm<3>(acc, cur);  if (RD) rds<2, 4>(nxt, lds, rbuf, rhalf); SCHED();
            mm<4>(acc, cur);  mm<5>(acc, cur);  if (RD) rds<4, 6>(nxt, lds, rbuf, rhalf); SCHED();
            mm<6>(acc, cur);  mm<7>(acc, cur);  if (RD) rds<6, 8>(nxt, lds, rbuf, rhalf); SCHED();
            mm<8>(acc, cur);  mm<9>(acc, cur);  SCHED();
            first();
            mm<10>(acc, cur); mm<11>(acc, cur); SCHED();
            if (ST) { piece<1>(ts, bs); SCHED(); }
            mm<12>(acc, cur); mm<13>(acc, cur); SCHED();
            if (ST) { piece<2>(ts, bs); SCHED(); }
            mm<14>(acc, cur); mm<15>(acc, cur); SCHED();
        } else {
            first();
            mm<0>(acc, cur); SCHED();
            if (ST) { piece<1>(ts, bs); SCHED(); }
            mm<1>(acc, cur); SCHED();
            if (ST) { piece<2>(ts, bs); SCHED(); }
            mm<2>(acc, cur); if (RD) rds<0, 3>(nxt, lds, rbuf, rhalf); SCHED();
            mm<3>(acc, cur); if (RD) rds<3, 5>(nxt, lds, rbuf, rhalf); SCHED();
            mm<4>(acc, cur); if (RD) rds<5, 8>(nxt, lds, rbuf, rhalf); SCHED();
            mm<5>(acc, cur); if (RD) rds<8, 10>(nxt, lds, rbuf, rhalf); SCHED();
            mm<6>(acc, cur); mm<7>(acc, cur); SCHED();
        }
    }

    template <int ROLE>
    __device__ __forceinline__ u32 run(f32x4 (&acc)[NM][4], const LAS char* lds, char* ws, u32 cvo, u32 need, u32 call) const {
        constexpr bool LATE = ROLE == 2 && STAGE == 1;
        char* tmo = ws + OFF_TMO;
        Frags f0, f1;
        if (ROLE == 0) spin_ge(ws + cvo + blk_of(0) * CNT_BLK, need, tmo, call);
        BARRIER();
        stage(0, 0);
        if (ROLE == 0) { spin_ge(ws + cvo + blk_of(1) * CNT_BLK, need, tmo, call); spin_ge(ws + cvo + blk_of(2) * CNT_BLK, need, tmo, call); }
        BARRIER();
        stage(1, 1);
        if (!LATE) { stage(2, 2); asm volatile("s_waitcnt vmcnt(12)" ::: "memory"); }
        else asm volatile("s_waitcnt vmcnt(6)" ::: "memory");
        BARRIER();
        rds<0, NRD>(f0, lds, 0, 0);
        WAIT_LGKM0();
        if (ROLE == 0) asm volatile("v_mov_b32 v166, 0" ::: "v166");
        int buf = 0;
        for (int t = 0; t < NT - 3; ++t) {
            const int nb = buf == 2 ? 0 : buf + 1, pb = buf == 0 ? 2 : buf - 1;
            half_step<true, LATE ? 2 : 0>(acc, f0, f1, lds, buf, 1, t + 2, pb, 0, 0);
            if (ROLE == 0) {
                u64 miss;
                asm volatile("s_waitcnt vmcnt(6)\n\tv_cmp_lt_u32_e64 %0, v166, %1\n\ts_nop 0" : "=s"(miss) : "v"(need) : "memory");
                WAIT_LGKM0();
                if (miss != 0) spin_ge(ws + cvo + blk_of(t + 3) * CNT_BLK, need, tmo, call);
            } else {
                asm volatile("s_waitcnt vmcnt(6)" ::: "memory");
                WAIT_LGKM0();
            }
            BARRIER();
            half_step<true, LATE ? 0 : (ROLE == 0 ? 1 : 2)>(acc, f1, f0, lds, nb, 0, t + 3, buf, cvo, blk_of(t + 4 < NT ? t + 4 : t + 3) * CNT_BLK);
            buf = nb;
        }
#pragma unroll
        for (int t = NT - 3; t < NT; ++t) {
            const int nb = buf == 2 ? 0 : buf + 1, pb = buf == 0 ? 2 : buf - 1;
            if (t == NT - 3) {
                half_step<true, LATE ? 2 : 0>(acc, f0, f1, lds, buf, 1, t + 2, pb, 0, 0);
                asm volatile("s_waitcnt vmcnt(6)" ::: "memory");
            } else {
                if (t == NT - 2) {
                    const u32 to = OFF_TMO;
                    asm volatile("buffer_load_dword v167, %0, %1, 0 offen sc1" :: "v"(to), "s"(rs) : "memory", "v167");
                }
                half_step<true, 0>(acc, f0, f1, lds, buf, 1, 0, 0, 0, 0);
                asm volatile("s_waitcnt vmcnt(0)" ::: "memory");
            }
            WAIT_LGKM0();
            BARRIER();
            if (t < NT - 1) half_step<true, 0>(acc, f1, f0, lds, nb, 0, 0, 0, 0, 0);
            else            half_step<false, 0>(acc, f1, f0, lds, nb, 0, 0, 0, 0, 0);
            buf = nb;
        }
        u32 tw;
        asm volatile("v_readfirstlane_b32 %0, v167\n\ts_nop 1" : "=s"(tw) :: "memory");
        return tw;
    }
};

__device__ __forceinline__ u32x4 gelu4(const f32x4 (&c)[4], int r, bool bad) {
    const float nanv = __builtin_nanf("");
    f32x4 o = {gelu(c[0][r]), gelu(c[1][r]), gelu(c[2][r]), gelu(c[3][r])};
    if (bad) o = (f32x4){nanv, nanv, nanv, nanv};
    return __builtin_bit_cast(u32x4, o);
}

__device__ __forceinline__ void gemm_role(const Params& p, const LAS char* lds, int xcd, int s, int wid, int lane, u32 call) {
    const int rank = rfl(p.rank);
    const int g = rank >> 2, j = rank & 3, og = (g ^ 1) * 4;
    char* ws = p.ws[rank];
    const u32 par = OFF_PAR + (call & 1) * PAR_BYTES;
    const rsrc_t rs = mk_rsrc(ws, WS_NEED), ro = mk_rsrc(p.out[rank], OUT_BYTES);
    const u32 ldsw = rfl((u32)(size_t)lds);
    const u32 vout = (lane >> 4) * 4 * 4096 + (lane & 15) * 16;
    {
        const int u = (xcd & 1) * 26 + s, rt = u >> 2, ct = (xcd >> 1) * 4 + (u & 3);
        Gemm<1> G;
        G.rs = rs; G.ldsw = ldsw; G.wid = wid; G.lane = lane;
        const int rf = rt * 16 + wid * 2;
        G.sa = par + (rf < FI ? rf : rf + FX) * 4096;
        G.sa2 = par + (rf + 1 < FI ? rf + 1 : rf + 1 + FX) * 4096 - 2048;
        G.sb = par + REL_B + (ct * 8 + wid) * 4096;
        const int bsrc = ct < 8 ? rank : og + ((ct - 8) >> 1);
        u32 cvo = OFF_CNT + 1024 + bsrc * 128, need = (ct < 8 ? 8u : 16u) * call;
        u32 mask = 0;
        for (int i = 0; i < 16; ++i) mask |= 1u << src_of_rf1(rt * 16 + i, g, j);
        if (lane < 8 && ((mask >> lane) & 1)) {
            cvo = OFF_CNT + lane * 128;
            need = (lane == rank ? 8 : ((lane >> 2) == g ? FI1 : FX)) * call;
        }
        f32x4 acc[4][4] = {};
        const bool bad = call == (wid == 0 ? G.template run<0>(acc, lds, ws, cvo, need, call)
                                  : wid < 4 ? G.template run<1>(acc, lds, ws, cvo, need, call)
                                            : G.template run<2>(acc, lds, ws, cvo, need, call));
        STAMP(1);
        const int wm = wid >> 1, wn = wid & 1;
        if (ct < 8) {
#pragma unroll
            for (int m = 0; m < 4; ++m) {
                const u32 ob = (u32)row_of_rf1(rt * 16 + wm * 4 + m, g, j) * 4096 + (ct * 128 + wn * 64) * 4;
#pragma unroll
                for (int r = 0; r < 4; ++r) __builtin_amdgcn_raw_buffer_store_b128(gelu4(acc[m], r, bad), ro, vout + ob + r * 4096, 0, OUT_AUX);
            }
        } else {
            const int fo = (rt * 2 + (ct & 1)) * 8 + wid;
            const u32 sb = par + REL_STG + (u32)(((ct - 8) >> 1) * 1024 + fo * 4) * 4096;
            u32x4 o[4][4];
#pragma unroll
            for (int i = 0; i < 16; ++i) o[i >> 2][i & 3] = gelu4(acc[i >> 2], i & 3, bad);
            u32 lop, lon;
            out_windows<4>(o, lop, lon);
#pragma unroll
            for (int m = 0; m < 4; ++m) pack_out(o[m], lop, lon, (LAS char*)lds + wid * 2048, rs, sb + m * 4096, lane);
            WAIT_VM0();
            if (lane == 0) st_flag(p.ws[bsrc] + OFF_RDY + rank * 8192 + (1024 + fo) * 4, call);
        }
        STAMP(2);
    }
    {
        const bool loc = s >= 24;
        const int rt = s >> 2, ct = loc ? xcd * 2 + (s & 1) : ((~xcd & 1) * 16) + (xcd >> 1) * 4 + (s & 3);
        Gemm<2> G;
        G.rs = rs; G.ldsw = ldsw; G.wid = wid; G.lane = lane;
        G.sa = G.sa2 = par + (loc ? (FI + wid) * 4096 : REL_A2 + (rt * 8 + wid) * 4096);
        G.ablk = (loc ? RS1 : RF2) * 4096;
        G.sb = par + REL_B + ct * 4 * 4096 + wid * 2048;
        const int asrc = loc ? rank : g * 4 + ((j + 1 + (rt >> 1)) & 3);
        const int bsrc = ct < 16 ? rank : og + ((ct - 16) >> 2);
        u32 cvo = OFF_CNT + 1024 + bsrc * 128, need = (ct < 16 ? 8u : 16u) * call;
        if (lane == 0) { cvo = OFF_CNT + (loc ? 0 : 2048) + asrc * 128; need = (loc ? 8 : F2) * call; }
        f32x4 acc[1][4] = {};
        const bool bad = call == (wid == 0 ? G.template run<0>(acc, lds, ws, cvo, need, call)
                                  : wid < 4 ? G.template run<1>(acc, lds, ws, cvo, need, call)
                                            : G.template run<2>(acc, lds, ws, cvo, need, call));
        STAMP(3);
        if (ct < 16) {
            const u32 ob = (u32)(asrc * NS + ((loc ? FI : FI1 + (rt & 1) * 8) + wid) * 16) * 4096 + ct * 256;
#pragma unroll
            for (int r = 0; r < 4; ++r) __builtin_amdgcn_raw_buffer_store_b128(gelu4(acc[0], r, bad), ro, vout + ob + r * 4096, 0, OUT_AUX);
        } else {
            const int z = (rt * 4 + (ct & 3)) * 8 + wid;
            const u32 sb = par + REL_STG + (u32)(((ct - 16) >> 2) * 1024 + 832 + z) * 4096;
            const u32x4 o[1][4] = {{gelu4(acc[0], 0, bad), gelu4(acc[0], 1, bad), gelu4(acc[0], 2, bad), gelu4(acc[0], 3, bad)}};
            u32 lop, lon;
            out_windows<1>(o, lop, lon);
            pack_out(o[0], lop, lon, (LAS char*)lds + wid * 2048, rs, sb, lane);
            WAIT_VM0();
            if (lane == 0) st_flag(p.ws[bsrc] + OFF_RDY + rank * 8192 + (1024 + 208 + z) * 4, call);
        }
        STAMP(4);
    }
}

__global__ __launch_bounds__(BLOCK, 3) void ag_gemm_kernel(Params p) {
    __shared__ __attribute__((aligned(1024))) char smem[LDS_BYTES];
    const int rank = rfl(p.rank);
    const int lane = threadIdx.x & 63;
    const int wid = rfl(threadIdx.x >> 6);
    const int b = blockIdx.x;
    if (wid >= (b < NCOMM ? NLW + NDEC : 8)) return;
    STAMP(0);
    char* seq = p.ws[rank] + OFF_SEQ + (b * 16 + wid) * 4;
    const u32 prev = ld_agent(seq);
    char* wsl = p.ws[lane & 7];
    auto next_call = [&] {
        const u32 call = rfl(prev) + 1;
        if (lane == 0) __hip_atomic_store((GAS u32*)seq, call, __ATOMIC_RELAXED, __HIP_MEMORY_SCOPE_AGENT);
        STAMP(7);
        return call;
    };
    if (b >= NCOMM) {
        for (int i = ((b - NCOMM) >> 4) - 1; i > 0; --i) __builtin_amdgcn_s_sleep(P_GRADE);
        f32x4 v[8];
        load_item(v, p, b - NCOMM, wid, lane);
        const u32 call = next_call();
        convert_local(p, (LAS char*)smem, v, wsl, b - NCOMM, wid, lane, call);
        STAMP(5);
        gemm_role(p, (const LAS char*)smem, b & 7, (b - NCOMM) >> 3, wid, lane, call);
        return;
    }
    const int nld = 2 * NCOMM + b < 7 * WL ? 3 : 2;
    const int sw = wid < NLW + NDEC - 5 ? NPUSH : (NLW + NDEC - 1 - wid) * NCOMM + b;
    f32x4 v[8];
    if (sw < NPUSH) {
        int blk, it;
        push_decode(sw, blk, it);
        load_item(v, p, blk * 16 + (it >> 3), it & 7, lane);
    }
    if (wid < 4) *(LAS u32*)(smem + LDS_CTL + (wid * 64 + lane) * 4) = 0;
    for (int k = 0; k < nld; ++k)
        build_table((LAS char*)smem + LDS_TAB + k * TAB_BYTES, rank, (k * NCOMM + b) % 7, threadIdx.x, (NLW + NDEC) * 64);
    asm volatile("s_waitcnt lgkmcnt(0)\n\ts_barrier" ::: "memory");
    const u32 call = next_call();
    if (sw < NPUSH) push_role(p, v, wsl, sw, lane, call);
    if (wid >= NLW) decode_role(p, (LAS char*)smem, wid - NLW, lane, call);
    else if (wid < nld) {
        const int ps = rfl((wid * NCOMM + b) % 7), wv = rfl((wid * NCOMM + b) / 7);
        if (ps < 3) load_role<false>(p, (LAS char*)smem, wid, nld, ps, wv, lane, call);
        else        load_role<true>(p, (LAS char*)smem, wid, nld, ps, wv, lane, call);
    }
}

}

constexpr int g_mode = 0;

extern "C" void kernel_launch(int rank, int nranks, ncclComm_t comm, hipStream_t stream,
                              void* const* d_in, const int* in_sizes, int n_in,
                              void* d_out, int out_size, const int* in_dtypes, int out_dtype,
                              void* const* d_out_all, void* const* d_ws_all, size_t ws_bytes) {
    (void)comm; (void)in_sizes; (void)n_in; (void)d_out; (void)out_size; (void)in_dtypes; (void)out_dtype;
    if (nranks != NR || (unsigned)rank >= (unsigned)NR || ws_bytes < WS_NEED) {
        fprintf(stderr, "unsupported configuration\n");
        return;
    }
    Params p;
    p.x = (const float*)d_in[0];
    p.w = (const float*)d_in[1];
    for (int i = 0; i < NR; ++i) { p.ws[i] = (char*)d_ws_all[i]; p.out[i] = (char*)d_out_all[i]; }
    p.rank = rank;
    p.pad = g_mode;
    ag_gemm_kernel<<<GRID, BLOCK, 0, stream>>>(p);
    hipError_t e = hipGetLastError();
    if (e != hipSuccess) fprintf(stderr, "launch failed: %s\n", hipGetErrorString(e));
}
```

```cpp
#include <hip/hip_runtime.h>
#include <rccl/rccl.h>
#include <cstdio>
#include <cstdint>
#include <type_traits>

typedef unsigned u32;
typedef unsigned long long u64;
typedef u32 u32x4 __attribute__((ext_vector_type(4)));
typedef u32 u32x2v __attribute__((ext_vector_type(2)));
typedef float f32x4 __attribute__((ext_vector_type(4)));
typedef _Float16 f16x8 __attribute__((ext_vector_type(8)));
typedef __amdgpu_buffer_rsrc_t rsrc_t;
#define LAS __attribute__((address_space(3)))
#define GAS __attribute__((address_space(1)))

namespace {

constexpr int NR = 8;
constexpr int KD = 8192;
constexpr int NS = 1024;
constexpr int NBLK = 64;
#define P_FI 45
constexpr int FI = P_FI;
constexpr int F2 = 16;
constexpr int FI1 = FI - F2;
constexpr int FX = 64 - FI;
constexpr int FL = 8;
constexpr int RF1 = 208, RF2 = 48, CF = 128;
static_assert(FI + 3 * FI1 + 4 * FX == RF1, "stage-1 rows");

#define P_W 19
#define P_XA 14
#define P_PB 1
#define P_GRADE 20
constexpr int PB = P_PB;
constexpr int NCOMM = 48;
constexpr int WL = P_W;
constexpr int XA = P_XA;
static_assert(WL >= 14 && WL <= 20 && XA <= FX && XA <= 16, "wave roles");
constexpr int GRID = 256, BLOCK = 768, NGEMM = GRID - NCOMM;
constexpr int RS1 = 64 + 3 * FI1 + 4 * FX;

constexpr u32 OFF_SEQ  = 0;
constexpr u32 OFF_TMO  = 16384;
constexpr u32 OFF_QUE  = 16640;
constexpr u32 OFF_RDY  = 32768;
constexpr u32 OFF_CNT  = 131072;
constexpr u32 CNT_BLK  = 24 * 128;
constexpr u32 OFF_DIAG = 524288;
constexpr u32 A1_BYTES = (u32)NBLK * RS1 * 4096;
constexpr u32 A2_BYTES = (u32)NBLK * RF2 * 4096;
constexpr u32 B_BYTES  = (u32)NBLK * CF * 4096;
constexpr u32 STG_BYTES = 4u * 1024 * 4096;
constexpr u32 PK_BYTES = (u32)NBLK * 128 * 4096;
constexpr u32 REL_A2 = A1_BYTES, REL_B = REL_A2 + A2_BYTES, REL_PK = REL_B + B_BYTES, REL_STG = REL_PK + PK_BYTES;
constexpr u32 PAR_BYTES = REL_STG + 2 * STG_BYTES;
constexpr u32 OFF_PAR  = 1u << 20;
constexpr int VAR_XO = 192;
constexpr u32 WS_NEED  = OFF_PAR + 2 * PAR_BYTES;
constexpr u32 OUT_BYTES = (u32)KD * NS * 4;
static_assert(REL_PK < (256u << 20), "table entries hold 18 bits of dst / 1024");
static_assert(256 + VAR_XO == 448, "output items are as long as x items");
static_assert(PK_BYTES == 8192u * 4096u, "staged outputs follow the packed image");
static_assert(OFF_CNT + NBLK * CNT_BLK <= OFF_DIAG && OFF_RDY + 8 * 8192 <= OFF_CNT, "control area");

constexpr u64 DEADLINE = 40000000ull;

constexpr int BUF_BYTES = 49152;
constexpr int LDS_BYTES = 3 * BUF_BYTES;

struct Params {
    const float* x;
    const float* w;
    char* ws[NR];
    char* out[NR];
    int rank;
    int pad;
};

__device__ __forceinline__ u32 rfl(u32 v) { return __builtin_amdgcn_readfirstlane(v); }
__device__ __forceinline__ char* uni(const void* p) {
    const u64 a = (u64)p;
    return (char*)(((u64)rfl((u32)(a >> 32)) << 32) | rfl((u32)a));
}
__device__ __forceinline__ rsrc_t rsrc_u(const void* p, u32 bytes) {
    return __builtin_amdgcn_make_buffer_rsrc((void*)p, (short)0, (int)bytes, 0x00020000);
}
__device__ __forceinline__ rsrc_t mk_rsrc(const void* p, u32 bytes) { return rsrc_u(uni(p), bytes); }
__device__ __forceinline__ u32 ld_sys(const char* p) {
    return __hip_atomic_load((const GAS u32*)p, __ATOMIC_RELAXED, __HIP_MEMORY_SCOPE_SYSTEM);
}
__device__ __forceinline__ u32 ld_agent(const char* p) {
    return __hip_atomic_load((const GAS u32*)p, __ATOMIC_RELAXED, __HIP_MEMORY_SCOPE_AGENT);
}
__device__ __forceinline__ void st_flag(char* p, u32 v) {
    __hip_atomic_store((GAS u32*)p, v, __ATOMIC_RELAXED, __HIP_MEMORY_SCOPE_SYSTEM);
}
__device__ __forceinline__ void add_sys(char* p) {
    __hip_atomic_fetch_add((GAS u32*)p, 1u, __ATOMIC_RELAXED, __HIP_MEMORY_SCOPE_SYSTEM);
}
#define WAIT_VM0() do { asm volatile("s_waitcnt vmcnt(0)" ::: "memory"); __builtin_amdgcn_s_waitcnt(0x0F70); } while (0)

template <int SLEEP = 2>
__device__ __forceinline__ bool spin_ge(const char* p, u32 need, char* tmo, u32 call) {
    const u64 t0 = __builtin_amdgcn_s_memrealtime();
    bool gave_up = false;
    for (u32 polls = 0;; ++polls) {
        const u32 v = ld_sys(p);
        if (__builtin_amdgcn_ballot_w64(v < need) == 0) break;
        if ((polls & 7) == 0 && (rfl(ld_agent(tmo)) == call || __builtin_amdgcn_s_memrealtime() - t0 > DEADLINE)) {
            __hip_atomic_store((GAS u32*)tmo, call, __ATOMIC_RELAXED, __HIP_MEMORY_SCOPE_AGENT);
            gave_up = true;
            break;
        }
        __builtin_amdgcn_s_sleep(SLEEP);
    }
    asm volatile("" ::: "memory");
    return gave_up;
}

#define STAMP_IF(c, slot) do {} while (0)
#define STAMP(slot) do {} while (0)
#define STAMPB_IF(c, slot) do {} while (0)

__device__ __forceinline__ u32x4 cvt8(f32x4 a, f32x4 b) {
    f16x8 h;
    h[0] = (_Float16)a[0]; h[1] = (_Float16)a[1]; h[2] = (_Float16)a[2]; h[3] = (_Float16)a[3];
    h[4] = (_Float16)b[0]; h[5] = (_Float16)b[1]; h[6] = (_Float16)b[2]; h[7] = (_Float16)b[3];
    return __builtin_bit_cast(u32x4, h);
}
__device__ __forceinline__ void st_sys(u32x4 v, rsrc_t r, u32 voff, u32 soff) {
    __builtin_amdgcn_raw_buffer_store_b128(v, r, voff + soff, 0, 17);
}
__device__ __forceinline__ float gelu(float y) {
    const float u = y * (0.7978845608028654f + 0.7978845608028654f * 0.044715f * y * y);
    const float e = __builtin_amdgcn_exp2f(u * -2.8853900817779268f);
    return y * __builtin_amdgcn_rcpf(1.0f + e);
}

__device__ __forceinline__ int row_of_rf1(int rf, int g, int j) {
    if (rf < FI) return (g * 4 + j) * NS + rf * 16;
    rf -= FI;
    if (rf < 3 * FI1) { const int i = rf / FI1; return (g * 4 + ((j + 1 + i) & 3)) * NS + (rf - i * FI1) * 16; }
    rf -= 3 * FI1;
    const int q = rf / FX;
    return ((g ^ 1) * 4 + q) * NS + (FI + rf - q * FX) * 16;
}
__device__ __forceinline__ int src_of_rf1(int rf, int g, int j) { return row_of_rf1(rf, g, j) / NS; }

constexpr int IN_AUX = 2;
constexpr int OUT_AUX = 17;
__device__ __forceinline__ u32x4 cvt_w(const f32x4* v, int c) {
    return cvt8((f32x4){v[0][c], v[1][c], v[2][c], v[3][c]}, (f32x4){v[4][c], v[5][c], v[6][c], v[7][c]});
}

#define DPP_STEP(op, v, ctrl, rmask) v = op(v, (u32)__builtin_amdgcn_update_dpp(0, (int)v, ctrl, rmask, 0xf, false))
__device__ __forceinline__ u32 umax2(u32 a, u32 b) { return a > b ? a : b; }
__device__ __forceinline__ u32 uadd2(u32 a, u32 b) { return a + b; }
__device__ __forceinline__ u32 wave_max(u32 v) {
    DPP_STEP(umax2, v, 0x111, 0xf); DPP_STEP(umax2, v, 0x112, 0xf); DPP_STEP(umax2, v, 0x114, 0xf); DPP_STEP(umax2, v, 0x118, 0xf);
    DPP_STEP(umax2, v, 0x142, 0xa); DPP_STEP(umax2, v, 0x143, 0xc);
    return __builtin_amdgcn_readlane(v, 63);
}
__device__ __forceinline__ u32 wave_sum(u32 v) {
    DPP_STEP(uadd2, v, 0x111, 0xf); DPP_STEP(uadd2, v, 0x112, 0xf); DPP_STEP(uadd2, v, 0x114, 0xf); DPP_STEP(uadd2, v, 0x118, 0xf);
    DPP_STEP(uadd2, v, 0x142, 0xa); DPP_STEP(uadd2, v, 0x143, 0xc);
    return __builtin_amdgcn_readlane(v, 63);
}

__device__ __forceinline__ u32 wave_scan(u32 v) {
    DPP_STEP(uadd2, v, 0x111, 0xf); DPP_STEP(uadd2, v, 0x112, 0xf); DPP_STEP(uadd2, v, 0x114, 0xf); DPP_STEP(uadd2, v, 0x118, 0xf);
    DPP_STEP(uadd2, v, 0x142, 0xa); DPP_STEP(uadd2, v, 0x143, 0xc);
    return v;
}
__device__ __forceinline__ u32 byte_scan(u32 m) {
    u32 a, r;
    asm("v_lshl_add_u32 %0, %1, 8, %1" : "=v"(a) : "v"(m));
    asm("v_lshl_add_u32 %0, %1, 16, %1" : "=v"(r) : "v"(a));
    return r;
}
__device__ __forceinline__ u32 times7(u32 m) {
    u32 a, r;
    asm("v_lshl_add_u32 %0, %1, 1, %1" : "=v"(a) : "v"(m));
    asm("v_lshl_add_u32 %0, %1, 2, %2" : "=v"(r) : "v"(m), "v"(a));
    return r;
}
__device__ __forceinline__ u32 rank_in(u64 m) { return __builtin_amdgcn_mbcnt_hi((u32)(m >> 32), __builtin_amdgcn_mbcnt_lo((u32)m, 0)); }

constexpr int VAR_X = VAR_XO, VAR_W = 256;
__device__ __forceinline__ u32 h7_of(float a) { return (u32)__builtin_bit_cast(unsigned short, (_Float16)a) >> 8; }
__device__ __forceinline__ void lds_or(LAS char* a, u32 v) {
    __hip_atomic_fetch_or((LAS u32*)a, v, __ATOMIC_RELAXED, __HIP_MEMORY_SCOPE_WORKGROUP);
}
__device__ __forceinline__ void put_bits(LAS char* area, u32 bo, u64 v) {
    const u32 sh = bo & 31;
    const u64 w = v << sh;
    LAS char* a = area + (bo >> 5) * 4;
    lds_or(a, (u32)w); lds_or(a + 4, (u32)(w >> 32)); lds_or(a + 8, (u32)((v >> 1) >> (63 - sh)));
}
__device__ __forceinline__ u64 get_bits(const LAS char* area, u32 bo) {
    const LAS u32* w = (const LAS u32*)(area + (bo >> 5) * 4);
    const u32 w0 = w[0], w1 = w[1], w2 = w[2];
    return (u64)__builtin_amdgcn_alignbit(w2, w1, bo) << 32 | __builtin_amdgcn_alignbit(w1, w0, bo);
}
template <class F>
__device__ __forceinline__ void pack_item(const u32x4 (&h)[4], float amax, LAS char* scr, rsrc_t r0, u32 dst, int lane, F before_stores) {
    u32 L[8], H[8];
#pragma unroll
    for (int t = 0; t < 8; ++t) {
        const u32 d0 = h[t >> 1][(t & 1) * 2], d1 = h[t >> 1][(t & 1) * 2 + 1];
        L[t] = __builtin_amdgcn_perm(d1, d0, 0x06040200u);
        H[t] = __builtin_amdgcn_perm(d1, d0, 0x07050301u);
    }
    *(LAS u32x4*)(scr + lane * 16) = (u32x4){H[0], H[1], H[2], H[3]};
    *(LAS u32x4*)(scr + lane * 16 + 1024) = (u32x4){H[4], H[5], H[6], H[7]};
    LAS char* area = scr + 2048;
    *(LAS u32x2v*)(area + lane * 8) = (u32x2v){0u, 0u};
    const u32 lm = wave_sum(h7_of(amax) & 127) + 132, rare = (lm & 255) < 115 ? 1 : 0;
    const u32 E1 = lm < 256 ? 1 : lm < 31 * 256 ? lm >> 8 : 31;
    const u32 eb = E1 * 0x01010101u | 0x80808080u;
    const u32 tbig = rare ? 0x79797979u : 0x7a7a7a7au, vadd = rare ? 0x01010101u : 0x02020202u, top6 = rare ? 0x06060606u : 0u;
    const u32 clo = 0x02010003u, qlo = rare ? 3u : 2u, qhi = rare ? 0x03020100u : 0x03030100u;
    u32 K[2] = {0, 0}, e4 = 0, pos = 0;
    u64 b2 = 0, b3 = 0;
#pragma unroll
    for (int t = 0; t < 8; ++t) {
        const u32 o = (eb - ((H[t] >> 2) & 0x1f1f1f1fu)) ^ 0x80808080u, o7 = o & 0x7f7f7f7fu;
        const u32 far = (o | (o7 + tbig)) & 0x80808080u, huge = (o | (o7 + 0x73737373u)) & 0x80808080u;
        const u32 f7 = (far >> 4) - (far >> 7), sel = (o | f7) & 0x07070707u;
        const u32 c = __builtin_amdgcn_perm(0x03030303u, clo, sel);
        const u32 q = __builtin_amdgcn_perm(qhi, qlo, sel);
        const u32 M = __builtin_amdgcn_perm(0x01010101u, 0x00000001u, sel);
        const u32 M2 = q & (q >> 1);
        const u32 v = (((o7 + vadd) | ((huge >> 4) - (huge >> 7))) & f7) | __builtin_amdgcn_perm(0u, top6 & 255, sel);
        const u32 P = byte_scan(M), P2 = byte_scan(M2), X2 = P2 - M2;
        const u32 bits2 = __builtin_amdgcn_udot4(q, __builtin_amdgcn_perm(0u, 0x40100401u, P - M), 0u, false);
        const u32 bits3 = __builtin_amdgcn_udot4(v, __builtin_amdgcn_perm(0u, 0x00400801u, X2), (X2 >> 24) == 3 ? (v >> 24) << 9 : 0u, false);
        b2 |= (u64)bits2 << (pos & 63);
        b3 |= (u64)bits3 << ((pos >> 16) & 63);
        pos += (P >> 23) + ((P2 >> 24) * 3 << 16);
        K[t >> 2] |= c << (2 * (t & 3));
        e4 = (e4 >> 1) | huge;
        if (t & 1) { asm volatile("" : "+v"(b2), "+v"(b3), "+v"(e4)); __builtin_amdgcn_sched_barrier(0); }
    }
    const u32 incl = wave_scan(pos), tot = __builtin_amdgcn_readlane(incl, 63), ex = incl - pos;
    const u32 used = 8 + (tot & 0xffff) + (tot >> 16);
    u32 need = used;
    if (__builtin_amdgcn_ballot_w64(e4 != 0) != 0) need += 8 * wave_sum(__builtin_popcount(e4));
    const u32 var = need <= 8 * VAR_X ? VAR_X : VAR_W;
    const bool fits = need <= 8 * VAR_W && __builtin_amdgcn_ballot_w64(pos >= (65u << 16)) == 0;
    if (fits) {
        u32 n4 = 0;
        put_bits(area, 8 + (ex & 0xffff), b2);
        put_bits(area, 8 + (tot & 0xffff) + (ex >> 16), b3);
        while (__builtin_amdgcn_ballot_w64(e4 != 0) != 0) {
            const u64 act = __builtin_amdgcn_ballot_w64(e4 != 0);
            const u32 any = (e4 | (e4 >> 8) | (e4 >> 16) | (e4 >> 24)) & 255, t = __builtin_ctz(any | 128);
            const u32 b = __builtin_ctz(((e4 >> t) & 0x01010101u) | 0x80000000u) + t;
            const u32 e = (*(const LAS unsigned char*)(scr + (t & 4) * 256 + lane * 16 + (t & 3) * 4 + ((b >> 3) & 3)) >> 2) & 31;
            const u32 k = var - 1 - n4 - rank_in(act);
            if (e4 != 0) lds_or(area + (k & ~3u), e << ((k & 3) * 8));
            e4 &= ~(1u << (b & 31));
            n4 += __builtin_popcountll(act);
        }
    }
    asm volatile("" ::: "memory");
    u32 vw = *(const LAS u32*)(area + lane * 4);
    if (lane == 0) vw |= fits ? E1 | rare << 5 | (var == VAR_W ? 64 : 0) : 255;
    const u32 P0 = (H[0] & 0x83838383u) | ((H[1] & 0x03030303u) << 2) | ((H[1] >> 1) & 0x40404040u) | ((H[6] & 0x03030303u) << 4);
    const u32 P1 = (H[2] & 0x83838383u) | ((H[3] & 0x03030303u) << 2) | ((H[3] >> 1) & 0x40404040u) | ((H[7] & 0x03030303u) << 4);
    const u32 P2 = (H[4] & 0x83838383u) | ((H[5] & 0x03030303u) << 2) | ((H[5] >> 1) & 0x40404040u)
                 | ((H[6] >> 2) & 0x20202020u) | ((H[7] >> 3) & 0x10101010u);
    before_stores(fits);
    st_sys((u32x4){L[0], L[1], L[2], L[3]}, r0, lane * 16, dst);
    st_sys((u32x4){L[4], L[5], L[6], L[7]}, r0, lane * 16, dst + 1024);
    st_sys((u32x4){P0, P1, P2, K[0]}, r0, lane * 16, dst + 2048);
    __builtin_amdgcn_raw_buffer_store_b32(K[1], r0, lane * 4 + dst + 3072, 0, 17);
    __builtin_amdgcn_raw_buffer_store_b32(vw, r0, lane * 4 + dst + 3328, 0, 17);
}
__device__ __forceinline__ void unpack_item(u32x4 (&h)[4], const u32x4& c0, const u32x4& c1, const u32x4& c2, u32 k1,
                                            const LAS char* area, u32 hd, LAS char* scr, int lane) {
    const u32 E1 = hd & 31, var = (hd & 64) ? VAR_W : VAR_X;
    const bool rare = hd & 32;
    const u32 K[2] = {c2[3], k1};
    const u32 X[2] = {K[0] & (K[0] >> 1) & 0x55555555u, K[1] & (K[1] >> 1) & 0x55555555u};
    const u32 n2 = 2 * (__builtin_popcount(X[0]) + __builtin_popcount(X[1])), i2 = wave_scan(n2);
    u64 b2 = get_bits(area, 8 + i2 - n2);
    b2 &= n2 < 64 ? ~(~0ull << n2) : ~0ull;
    const u64 y2 = b2 & (b2 >> 1) & 0x5555555555555555ull;
    const u32 n3 = 3 * __builtin_popcountll(y2), i3 = wave_scan(n3);
    u64 b3 = get_bits(area, 8 + __builtin_amdgcn_readlane(i2, 63) + i3 - n3);
    const u32 e4x = E1 * 0x04040404u | 0x80808080u;
    const u32 t1 = (e4x - 0x000c0804u) & 0x007c7c7cu, t2 = (e4x - (rare ? 0x00181410u : 0x00001410u)) & 0x007c7c7cu;
    const u32 t3lo = (e4x - (rare ? 0x2824201cu : 0x24201c18u)) & 0x7c7c7c7cu, t3hi = (e4x - (rare ? 0x0000302cu : 0x00302c28u)) & 0x007c7c7cu;
    LAS char* row = scr + lane * 36;
    u32 e4 = 0;
    const u32 nch = (wave_max(n2) + 7) >> 3;
#pragma unroll
    for (int k = 0; k < 8; ++k) {
        if (k >= 4 && k >= nch) break;
        const u32 b8 = (u32)(b2 >> (8 * k)) & 255, y = b8 | (b8 << 12), q = (y | (y << 6)) & 0x03030303u;
        const u32 M2 = q & (q >> 1), P2 = byte_scan(M2), m7 = times7(M2);
        const u32 lo = (u32)b3, z = (lo & 0x3f) | ((lo & 0xfc0) << 10), f = (z | (z << 5)) & 0x07070707u;
        const u32 v = __builtin_amdgcn_perm(0u, f, P2 - M2) & m7;
        b3 >>= (P2 >> 24) * 3;
        *(LAS u32*)(row + 4 * k) = __builtin_amdgcn_perm(0u, t2, q) | __builtin_amdgcn_perm(t3hi, t3lo, v | (0x0c0c0c0cu - (M2 << 2) - (M2 << 3)));
        e4 |= (((v + M2) >> 3) & 0x01010101u) << k;
    }
    u32 n4 = 0;
    while (__builtin_amdgcn_ballot_w64(e4 != 0) != 0) {
        const u64 act = __builtin_amdgcn_ballot_w64(e4 != 0);
        const u32 any = (e4 | (e4 >> 8) | (e4 >> 16) | (e4 >> 24)) & 255, t = __builtin_ctz(any | 128);
        const u32 b = __builtin_ctz(((e4 >> t) & 0x01010101u) | 0x80000000u) + t;
        const u32 x = *(const LAS unsigned char*)(area + ((var - 1 - n4 - rank_in(act)) & 255));
        if (e4 != 0) *(LAS unsigned char*)(row + t * 4 + (b >> 3)) = x << 2;
        e4 &= ~(1u << (b & 31));
        n4 += __builtin_popcountll(act);
    }
    const u32 SM[8] = {c2[0] & 0x83838383u, ((c2[0] << 1) & 0x80808080u) | ((c2[0] >> 2) & 0x03030303u),
                       c2[1] & 0x83838383u, ((c2[1] << 1) & 0x80808080u) | ((c2[1] >> 2) & 0x03030303u),
                       c2[2] & 0x83838383u, ((c2[2] << 1) & 0x80808080u) | ((c2[2] >> 2) & 0x03030303u),
                       ((c2[2] << 2) & 0x80808080u) | ((c2[0] >> 4) & 0x03030303u),
                       ((c2[2] << 3) & 0x80808080u) | ((c2[1] >> 4) & 0x03030303u)};
    u32 M[8], P[8], g[8], at = 0;
#pragma unroll
    for (int t = 0; t < 8; ++t) {
        M[t] = (X[t >> 2] >> (2 * (t & 3))) & 0x01010101u;
        P[t] = byte_scan(M[t]);
        const LAS u32* w = (const LAS u32*)(row + (at & ~3u));
        g[t] = __builtin_amdgcn_alignbyte(w[1], w[0], at);
        at += P[t] >> 24;
    }
#pragma unroll
    for (int t = 0; t < 8; ++t) {
        const u32 c = (K[t >> 2] >> (2 * (t & 3))) & 0x03030303u;
        const u32 m = times7(M[t]);
        const u32 H = SM[t] | __builtin_amdgcn_perm(t1, g[t], ((P[t] - M[t]) & m) | ((c | 0x04040404u) & ~m));
        const u32 L = t < 4 ? c0[t & 3] : c1[t & 3];
        h[t >> 1][(t & 1) * 2] = __builtin_amdgcn_perm(H, L, 0x05010400u);
        h[t >> 1][(t & 1) * 2 + 1] = __builtin_amdgcn_perm(H, L, 0x07030602u);
    }
}

__device__ __forceinline__ u32 sel_lo(u32 sg, u32 lop, u32 lon) {
    u32 m = sg << 8;
    asm("" : "+v"(m));
    m -= sg;
    return (lon * 0x01010101u & m) | (lop * 0x01010101u & ~m);
}
template <int N>
__device__ __forceinline__ void out_windows(const u32x4 (&o)[N][4], u32& lop, u32& lon) {
    u32 mn = 0, ntop = 0;
    int mp = 0;
#pragma unroll
    for (int i = 0; i < 16 * N; ++i) { mp = max(mp, (int)o[i >> 4][(i >> 2) & 3][i & 3]); mn = umax2(mn, o[i >> 4][(i >> 2) & 3][i & 3]); }
    u32 ep = wave_max((u32)mp) >> 23;
    const u32 en = (wave_max(mn) >> 23) & 255;
    ntop = __builtin_popcountll(__builtin_amdgcn_ballot_w64(mp >= (int)(ep << 23)));
    ep -= ntop <= 8 && ep > 0;
    lop = ep > 7 ? ep - 7 : 0; lon = en > 7 ? en - 7 : 0;
}
__device__ __forceinline__ void pack_out(const u32x4 (&o)[4], u32 lop, u32 lon, LAS char* scr, rsrc_t r0, u32 dst, int lane) {
    u32 M[12], S[4];
    u32x4 E;
#pragma unroll
    for (int r = 0; r < 4; ++r) {
        u32 R[4];
#pragma unroll
        for (int c = 0; c < 4; ++c) R[c] = __builtin_amdgcn_alignbit(o[r][c], o[r][c], 31);
#pragma unroll
        for (int i = 0; i < 3; ++i) M[r * 3 + i] = __builtin_amdgcn_perm(R[3], R[i], 0x04020100u + (i << 24));
        E[r] = __builtin_amdgcn_perm(R[1], R[0], 0x0c0c0703u) | __builtin_amdgcn_perm(R[3], R[2], 0x07030c0cu);
        S[r] = (__builtin_amdgcn_perm(R[1], R[0], 0x0c0c0400u) | __builtin_amdgcn_perm(R[3], R[2], 0x04000c0cu)) & 0x01010101u;
    }
    *(LAS u32x4*)(scr + lane * 16) = E;
    u32 G2 = 0, G1 = 0, em = 0;
#pragma unroll
    for (int r = 0; r < 4; ++r) {
        const u32 lo = sel_lo(S[r], lop, lon);
        const u32 d = ((E[r] | 0x80808080u) - (lo & 0x7f7f7f7fu)) ^ ((E[r] ^ ~lo) & 0x80808080u);
        const u32 nz = (d | ((d & 0x78787878u) + 0x78787878u)) & 0x80808080u;
        const u32 g = d & 0x07070707u & ~((nz >> 4) - (nz >> 7));
        G2 |= (g & 0x03030303u) << (2 * r);
        G1 |= (g & 0x04040404u) >> 2 << r;
        em = (em >> 1) | (~(g + 0x7f7f7f7fu) & 0x80808080u);
    }
    LAS unsigned char* list = (LAS unsigned char*)(scr + 1024);
    u32 used = 2;
    for (u64 act = __builtin_amdgcn_ballot_w64(em != 0); act != 0; act = __builtin_amdgcn_ballot_w64(em != 0)) {
        const u32 k = used + rank_in(act), b = __builtin_ctz(em | 0x80000000u);
        list[em != 0 && k < 64 ? k : 64] = *(const LAS unsigned char*)(scr + lane * 16 + (b & 3) * 4 + (b >> 3));
        em &= em - 1;
        used += __builtin_popcountll(act);
        if (used > 64) break;
    }
    *(LAS unsigned short*)list = (used > 64 ? 255 : lop) | lon << 8;
    asm volatile("" ::: "memory");
    const u32 ent = *(const LAS u32*)(list + (lane & 15) * 4);
    if (used > 64) {
#pragma unroll
        for (int r = 0; r < 4; ++r) st_sys(o[r], r0, lane * 16 + r * 1024, dst + STG_BYTES);
    }
#pragma unroll
    for (int i = 0; i < 3; ++i) st_sys((u32x4){M[4 * i], M[4 * i + 1], M[4 * i + 2], M[4 * i + 3]}, r0, lane * 16 + i * 1024, dst);
    __builtin_amdgcn_raw_buffer_store_b32(G2, r0, lane * 4 + dst + 3072, 0, 17);
    __builtin_amdgcn_raw_buffer_store_b16((unsigned short)__builtin_amdgcn_perm(0u, G1 | (G1 >> 4), 0x0c0c0200u), r0, lane * 2 + dst + 3328, 0, 17);
    __builtin_amdgcn_raw_buffer_store_b32(ent, r0, (lane < 16 ? lane * 4 : 0x80000000u) + dst + 3456, 0, 17);
}
__device__ __forceinline__ void unpack_out(u32x4 (&o)[4], const u32x4& c0, const u32x4& c1, const u32x4& c2, u32 G2, u32 g1,
                                           const LAS unsigned char* list, u32 lop, u32 lon, LAS char* scr, int lane) {
    const u32 M[12] = {c0[0], c0[1], c0[2], c0[3], c1[0], c1[1], c1[2], c1[3], c2[0], c2[1], c2[2], c2[3]};
    const u32 t = __builtin_amdgcn_perm(0u, g1, 0x0c010c00u), G1 = t | (t << 4);
    u32x4 E;
    u32 em = 0;
#pragma unroll
    for (int r = 0; r < 4; ++r) {
        const u32 sg = (__builtin_amdgcn_perm(M[r * 3 + 1], M[r * 3], 0x030c0400u) | __builtin_amdgcn_perm(0u, M[r * 3 + 2], 0x0c000c0cu)) & 0x01010101u;
        const u32 g = ((G2 >> (2 * r)) & 0x03030303u) | ((G1 >> r << 2) & 0x04040404u);
        const u32 nz = (g + 0x7f7f7f7fu) & 0x80808080u;
        E[r] = (sel_lo(sg, lop, lon) + g) & ((nz << 1) - (nz >> 7));
        em = (em >> 1) | (nz ^ 0x80808080u);
    }
    *(LAS u32x4*)(scr + lane * 16) = E;
    u32 used = 2;
    auto step = [&](u32& a, u32& val) {
        const u64 act = __builtin_amdgcn_ballot_w64(em != 0);
        const u32 b = __builtin_ctz(em | 0x80000000u);
        a = em != 0 ? lane * 16 + (b & 3) * 4 + (b >> 3) : 1024;
        val = list[(used + rank_in(act)) & 63];
        em &= em - 1;
        used += __builtin_popcountll(act);
    };
    do {
        u32 a0, a1, a2, x0, x1, x2;
        step(a0, x0); step(a1, x1); step(a2, x2);
        *(LAS unsigned char*)(scr + a0) = x0; *(LAS unsigned char*)(scr + a1) = x1; *(LAS unsigned char*)(scr + a2) = x2;
    } while (__builtin_amdgcn_ballot_w64(em != 0) != 0);
    E = *(const LAS u32x4*)(scr + lane * 16);
#pragma unroll
    for (int r = 0; r < 4; ++r) {
        u32 R[4];
#pragma unroll
        for (int i = 0; i < 3; ++i) R[i] = __builtin_amdgcn_perm(E[r], M[r * 3 + i], 0x04020100u + (i << 24));
        R[3] = __builtin_amdgcn_perm(M[r * 3 + 1], M[r * 3], 0x0c0c0703u) | __builtin_amdgcn_perm(E[r], M[r * 3 + 2], 0x07030c0cu);
#pragma unroll
        for (int c = 0; c < 4; ++c) o[r][c] = __builtin_amdgcn_alignbit(R[c], R[c], 1);
    }
}

__device__ __forceinline__ void load_item(f32x4 (&v)[8], const Params& p, int u, int wid, int lane) {
    const int blk = u >> 4, it = (u & 15) * 8 + wid;
    const bool isx = it < 64;
    const rsrc_t r = mk_rsrc(isx ? p.x : p.w, OUT_BYTES);
    const u32 vf = isx ? (lane & 15) * 32768 + (lane >> 4) * 16 : (lane >> 4) * 16384 + (lane & 15) * 16;
    const u32 base = isx ? it * 524288 + blk * 512 : (blk * 128 + (wid & 3) * 32) * 4096 + ((it - 64) >> 2) * 256;
    const u32 sa = isx ? 64 : 4096, sb = isx ? 256 : 65536;
#pragma unroll
    for (int i = 0; i < 8; ++i)
        v[i] = __builtin_bit_cast(f32x4, __builtin_amdgcn_raw_buffer_load_b128(r, vf, base + (i & 3) * sa + (i >> 2) * sb, IN_AUX));
}
__device__ __forceinline__ void convert_local(const Params& p, LAS char* lds, f32x4 (&v)[8], char* wsl, int wg, int wid, int lane, u32 call) {
    const int rank = rfl(p.rank);
    constexpr int NU = NBLK * 16;
    const u32 par = OFF_PAR + (call & 1) * PAR_BYTES;
    const rsrc_t r0 = mk_rsrc(p.ws[rank], WS_NEED);
    LAS char* scr = lds + wid * 4096;
    auto publish = [&](int u) {
        if (wid == 0) {
            if (lane < 8 && lane != rank) st_flag(wsl + OFF_RDY + rank * 8192 + u * 4, call);
            if (lane == 8) add_sys(p.ws[rank] + OFF_CNT + (u >> 4) * CNT_BLK + ((u & 8) ? 1024 : 0) + rank * 128);
        }
    };
    int u = wg, un = wg + NGEMM, up = -1;
    WAIT_VM0();
    for (int r = 0; u < NU; ++r) {
        const int blk = u >> 4, grp = u & 15, it = grp * 8 + wid;
        u32x4 h[4];
        u32 base, step;
        if (grp < 8) {
            base = par + (u32)blk * (RS1 * 4096) + it * 4096; step = 1024;
#pragma unroll
            for (int i = 0; i < 4; ++i) h[i] = cvt8(v[2 * i], v[2 * i + 1]);
        } else {
            base = par + REL_B + (u32)blk * (CF * 4096) + ((it - 64) & 60) * 4096 + (wid & 3) * 1024; step = 4096;
#pragma unroll
            for (int c = 0; c < 4; ++c) h[c] = cvt_w(v, c);
        }
        float amax = 0.0f;
#pragma unroll
        for (int i = 0; i < 8; ++i) amax = fmaxf(fmaxf(fmaxf(amax, fabsf(v[i][0])), fmaxf(fabsf(v[i][1]), fabsf(v[i][2]))), fabsf(v[i][3]));
        STAMPB_IF(r == 0, 0);
        if (r > 0) load_item(v, p, (u32)un < (u32)NU ? un : NU - 1, wid, lane);
        pack_item(h, amax, scr, r0, par + REL_PK + (u32)(blk * 128 + it) * 4096, lane, [&](bool fits) {
            WAIT_VM0();
            if (r > 0) {
                __builtin_amdgcn_s_barrier();
                if (up >= 0) publish(up);
            }
            if (!fits || it < FI + FL || it >= 64) {
#pragma unroll
                for (int i = 0; i < 4; ++i) st_sys(h[i], r0, lane * 16, base + i * step);
            }
        });
        up = u;
        if (r == 0) {
            STAMPB_IF(true, 1);
            WAIT_VM0();
            STAMPB_IF(true, 2);
            __builtin_amdgcn_s_barrier();
            STAMP(6);
            publish(u);
            up = -1;
            load_item(v, p, un, wid, lane);
            WAIT_VM0();
        }
        u = un;
        un += NGEMM;
    }
    WAIT_VM0();
    __builtin_amdgcn_s_barrier();
    if (up >= 0) publish(up);
}

constexpr int PUSH_PER = FI1 + 64 + XA, NPUSH = PB * PUSH_PER, NPW = (NPUSH + NCOMM - 1) / NCOMM;
__device__ __forceinline__ void push_decode(int n, int& blk, int& it) {
    blk = n / PUSH_PER; it = n - blk * PUSH_PER;
    if (it >= FI1 + 64) { const int xi = blk * XA + it - FI1 - 64; blk = xi / FX; it = FI + xi - blk * FX; }
    else if (it >= FI1) it += 64 - FI1;
}
__device__ __forceinline__ void push_role(const Params& p, char* wsl, int n, int lane, u32 call) {
    const int rank = rfl(p.rank);
    const int g = rank >> 2, j = rank & 3, og = (g ^ 1) * 4;
    int blk, it;
    push_decode(n, blk, it);
    f32x4 v[8];
    load_item(v, p, blk * 16 + (it >> 3), it & 7, lane);
    const u32 par = OFF_PAR + (call & 1) * PAR_BYTES;
    u32x4 h[4];
    int d0 = 0, nd = 4, dstep = 1; u32 base, step = 1024, ostep = 0;
    if (it < 64) {
#pragma unroll
        for (int i = 0; i < 4; ++i) h[i] = cvt8(v[2 * i], v[2 * i + 1]);
        if (it < FI1) { d0 = j + 3; nd = 3; dstep = 3; ostep = FI1 * 4096; base = par + (blk * RS1 + 64 + it) * 4096; }
        else base = par + (blk * RS1 + 64 + 3 * FI1 + j * FX + it - FI) * 4096;
    } else {
#pragma unroll
        for (int c = 0; c < 4; ++c) h[c] = cvt_w(v, c);
        d0 = (it - 64) >> 4; nd = 1; step = 4096;
        base = par + REL_B + (blk * CF + 64 + 16 * j + (it & 12)) * 4096 + (it & 3) * 1024;
    }
    u32 dmask = 0;
    for (int d = 0; d < nd; ++d) {
        const int dr = (it < FI1 ? g * 4 : og) + ((d0 + d * dstep) & 3);
        const rsrc_t rd = mk_rsrc(p.ws[dr], WS_NEED);
#pragma unroll
        for (int i = 0; i < 4; ++i) st_sys(h[i], rd, lane * 16, base + d * ostep + i * step);
        dmask |= 1u << dr;
    }
    __builtin_amdgcn_s_barrier();
    WAIT_VM0();
    STAMPB_IF(true, 3);
    if (lane < 8 && ((dmask >> lane) & 1)) add_sys(wsl + OFF_CNT + blk * CNT_BLK + (it < 64 ? 0 : 1024) + rank * 128);
}

#define P_NDEC 9
#define P_DSLEEP 2
constexpr int SCR = 2560;
constexpr int NLW = 3, RING = 6, QLEN = 32, NDEC = P_NDEC;
constexpr int LDS_CTL = NLW * RING * 4096;
constexpr int LDS_DONE = LDS_CTL + QLEN * 16, LDS_TAIL = LDS_DONE + NLW * RING * 4, LDS_SCR = LDS_CTL + 1024;
static_assert(7 * WL <= NLW * NCOMM, "loaders");
constexpr u32 IDX_END = 0xffffffffu, IDX_BAD = 0x80000000u, IDX_CROSS = 0x40000000u;
__device__ __forceinline__ u32 lds_get(const LAS char* a) {
    u32 v;
    asm volatile("ds_read_b32 %0, %1\n\ts_waitcnt lgkmcnt(0)" : "=v"(v) : "v"((u32)(size_t)a) : "memory");
    return rfl(v);
}
__device__ __forceinline__ void lds_put(LAS char* a, u32 v) {
    asm volatile("" ::: "memory");
    *(volatile LAS u32*)a = v;
    asm volatile("" ::: "memory");
}

struct Item { u32 idx, dst; };
constexpr int N1 = NBLK * FI1, NT_I = N1 + NBLK * F2;
constexpr int PER = 16 + XA, NP = NBLK * PER, NX = NBLK * (16 + FX), NO1 = (RF1 - FL) * 4, NT_C = NX + NO1 + 192;
constexpr int TAB_BYTES = (NT_I > NT_C ? NT_I : NT_C) * 4, LDS_TAB = LDS_SCR + NDEC * SCR;
static_assert(LDS_TAB + NLW * TAB_BYTES <= LDS_BYTES, "communication LDS");
__device__ __forceinline__ void build_table(LAS char* tab, int rank, int slot, int first, int stride) {
    const int g = rank >> 2, j = rank & 3, q = slot - 3;
    for (int it = first; it < (slot < 3 ? NT_I : NT_C); it += stride) {
        Item m;
        if (slot < 3) {
            const bool s2 = it >= N1;
            const int a = it - N1, b1 = it / FI1;
            const int blk = s2 ? a >> 4 : b1, s = s2 ? FI1 + (a & 15) : it - b1 * FI1;
            m.idx = blk * 128 + s;
            m.dst = (s2 ? REL_A2 + (slot * F2 - FI1) * 4096 : (64 + slot * FI1) * 4096) + (blk * (s2 ? RF2 : RS1) + s) * 4096;
        } else if (it < NX) {
            const int blk = it / PER, c = it - blk * PER;
            const bool tail = it >= NP, isx = tail || (c < 2 * XA && (c & 1));
            const int cw = c < 2 * XA ? c >> 1 : c - XA;
            const int xi = tail ? it - NP + NBLK * XA : blk * XA + (c >> 1), xb = xi / FX, xs = FI + xi - xb * FX;
            m.idx = isx ? xb * 128 + xs : blk * 128 + 64 + 16 * j + cw;
            m.dst = isx ? (xb * RS1 + xs + 64 + 3 * FI1 + q * FX - FI) * 4096
                        : REL_B + (blk * CF + 64 + 16 * q + (cw & 12)) * 4096 + (cw & 3) * 1024;
        } else {
            const int k = it - NX;
            int row, col;
            if (k < NO1) {
                const int rf = (k >> 2) + ((k >> 2) >= FI + 3 * FI1 + j * FX ? FL : 0);
                m.idx = 8192 + (rf >> 4) * 64 + (k & 2) * 16 + ((rf >> 2) & 3) * 8 + (k & 1) * 4 + (rf & 3);
                row = row_of_rf1(rf, g ^ 1, q);
                col = (k & 3) * 64;
            } else {
                const int z = k - NO1, rt = z >> 5;
                m.idx = 8192 + 832 + z;
                row = ((g ^ 1) * 4 + ((q + 1 + (rt >> 1)) & 3)) * NS + (FI1 + (rt & 1) * 8 + (z & 7)) * 16;
                col = ((z >> 3) & 3) * 64;
            }
            m.dst = (u32)row * 4096 + (q * 256 + col) * 4;
        }
        *(LAS u32*)(tab + it * 4) = m.idx | m.dst << (m.idx < 8192 ? 4 : 6);
    }
}

template <bool CROSS>
__device__ __forceinline__ void load_role(const Params& p, LAS char* lds, int ld, int nld, int slot, int wv, int lane, u32 call) {
    const int rank = rfl(p.rank);
    const int g = rank >> 2, j = rank & 3;
    const int peer = CROSS ? (g ^ 1) * 4 + slot - 3 : g * 4 + ((j + 1 + slot) & 3);
    constexpr int NT = CROSS ? NT_C : NT_I;
    const u32 par = OFF_PAR + (call & 1) * PAR_BYTES;
    char* ws = uni(p.ws[rank]);
    const rsrc_t r0 = rsrc_u(ws, WS_NEED), rp = mk_rsrc(p.ws[peer], WS_NEED);
    const char* rdy = ws + OFF_RDY + peer * 8192;
    const u32 vo = lane * 16;
    constexpr int SKIP = PB * (CROSS ? PER : FI1);
    const u32 head = lane == 0 ? OFF_QUE + slot * 256 : WS_NEED, tbase = SKIP + 4 * WL - (call - 1) * (NT - SKIP);
    const bool nowait = (p.pad & 4) != 0;
    const LAS char* tab = lds + LDS_TAB + ld * TAB_BYTES;
    const LAS char* done = lds + LDS_DONE + ld * RING * 4;
    const u32 ldsb = rfl((u32)(size_t)lds) + ld * RING * 4096;
    auto table = [&](int it) {
        asm volatile("ds_read_b32 v160, %0" :: "v"((u32)(size_t)tab + (it < NT ? it : NT - 1) * 4) : "memory", "v160");
    };
    auto entry = [&] {
        u32 t;
        asm volatile("s_waitcnt lgkmcnt(0)\n\tv_readfirstlane_b32 %0, v160" : "=s"(t) :: "memory");
        Item m;
        m.idx = t & 16383;
        m.dst = (t - m.idx) >> (CROSS && m.idx >= 8192 ? 6 : 4);
        return m;
    };
    auto decode = [&](int it) { table(it); return entry(); };
    auto flag_of = [&](u32 idx) { return idx < 8192 ? idx >> 3 : (idx < 8192 + 832 ? 1024 + ((idx - 8192) >> 2) : idx - (8192 + 832 - 1024 - 208)); };
    u32 okupto = 0, wbase = 0; u64 wmask = 0;
    auto ready = [&](u32 fi) {
        if (fi < okupto) return true;
        if (fi - wbase < 64u && ((wmask >> (fi - wbase)) & 1)) return true;
        for (;;) {
            wbase = okupto;
            wmask = __builtin_amdgcn_ballot_w64(ld_sys(rdy + (wbase + lane) * 4) >= call);
            if (wmask != ~0ull) break;
            okupto += 64;
            if (fi < okupto) return true;
        }
        okupto += __builtin_ctzll(~wmask);
        if (fi - wbase < 64u) return ((wmask >> (fi - wbase)) & 1) != 0;
        return rfl(ld_sys(rdy + fi * 4)) >= call;
    };
    const u32 vl = vo < 256 + VAR_X ? vo : 0x80000000u;
    struct Prep { u32 src, m0v; };
    auto prep = [&](const Item& m, u32 n) {
        Prep r;
        r.src = par + REL_PK + (m.idx + (CROSS && m.idx >= 8192 ? j * 1024 : 0)) * 4096;
        r.m0v = ldsb + (n % RING) * 4096;
        asm volatile("" : "+s"(r.src), "+s"(r.m0v));
        return r;
    };
    auto issue = [&](const Prep& r) {
        asm volatile("s_mov_b32 m0, %0\n\ts_nop 0\n\t"
                     "buffer_load_dwordx4 %1, %3, %4 offen sc0 sc1 lds\n\t"
                     "buffer_load_dwordx4 %1, %3, %4 offen offset:1024 sc0 sc1 lds\n\t"
                     "buffer_load_dwordx4 %1, %3, %4 offen offset:2048 sc0 sc1 lds\n\t"
                     "buffer_load_dwordx4 %2, %3, %4 offen offset:3072 sc0 sc1 lds"
                     :: "s"(r.m0v), "v"(vo), "v"(vl), "s"(rp), "s"(r.src) : "memory");
    };
    u32 vtail = (u32)(size_t)lds + LDS_TAIL, vone = 1;
    asm volatile("" : "+v"(vtail), "+v"(vone));
    auto enqueue = [&](u32 idx, u32 dst, u32 n) {
        asm volatile("s_mov_b64 exec, 1\n\tds_add_rtn_u32 v162, %0, %1\n\tv_mov_b32 v159, %2\n\tv_mov_b32 v160, %3\n\tv_mov_b32 v161, %4\n\t"
                     "s_waitcnt lgkmcnt(0)\n\tv_add_u32 v158, 1, v162\n\tv_and_b32 v162, %6, v162\n\tv_lshl_add_u32 v162, v162, 4, %5\n\t"
                     "ds_write_b128 v162, v[158:161]\n\ts_mov_b64 exec, -1"
                     :: "v"(vtail), "v"(vone), "s"(idx), "s"(dst), "s"(n << 8 | ld << 4 | peer), "s"(rfl((u32)(size_t)lds) + LDS_CTL), "n"(QLEN - 1)
                     : "memory", "v158", "v159", "v160", "v161", "v162");
    };
    __builtin_amdgcn_s_setprio(3);
    int e = SKIP + wv, e1 = e + WL, e2 = e + 2 * WL;
    const int e3 = e + 3 * WL;
    Item nx = decode(e1), ia = decode(e);
    bool bad = spin_ge<2>(rdy + flag_of(ia.idx) * 4, call, ws + OFF_TMO, call);
    STAMP(5);
    issue(prep(ia, 0));
    bool pre = ready(flag_of(nx.idx));
    u32 n = 0;
    const u32 cls = CROSS ? IDX_CROSS : 0;
    auto wait_free = [&](u32 m) { while (lds_get(done + (m % RING) * 4) + RING <= m) __builtin_amdgcn_s_sleep(1); };
    auto body = [&](auto odd) {
        if constexpr (odd.value) asm volatile("v_mov_b32 v164, 1\n\ts_nop 0\n\tbuffer_atomic_add v164, %0, %1, 0 offen sc0" :: "v"(head), "s"(r0) : "memory", "v164");
        else                     asm volatile("v_mov_b32 v165, 1\n\ts_nop 0\n\tbuffer_atomic_add v165, %0, %1, 0 offen sc0" :: "v"(head), "s"(r0) : "memory", "v165");
        table(e2);
        u32 dn;
        asm volatile("ds_read_b32 v162, %1\n\ts_waitcnt lgkmcnt(0)\n\tv_readfirstlane_b32 %0, v162"
                     : "=s"(dn) : "v"((u32)(size_t)done + ((n + 1) % RING) * 4) : "memory", "v162");
        const Item nn = entry();
        bool pre2 = e2 < NT;
        if (pre2 && !(nowait && nn.idx >= 8192)) pre2 = ready(flag_of(nn.idx));
        if (dn + RING <= n + 1) wait_free(n + 1);
        const Prep px = prep(nx, n + 1);
        u32 told;
        if constexpr (odd.value) asm volatile("s_waitcnt vmcnt(1)\n\tv_readfirstlane_b32 %0, v165" : "=s"(told) :: "memory");
        else                     asm volatile("s_waitcnt vmcnt(1)\n\tv_readfirstlane_b32 %0, v164" : "=s"(told) :: "memory");
        if (pre) issue(px);
        enqueue(ia.idx | cls | (bad ? IDX_BAD : 0), ia.dst, n);
        if (!pre && e1 < NT) {
            bad |= spin_ge<2>(rdy + flag_of(nx.idx) * 4, call, ws + OFF_TMO, call);
            issue(px);
        }
        STAMP_IF(n == 0, 6);
        STAMP_IF(CROSS ? (e < NP && e1 >= NP) : (e < N1 && e1 >= N1), 1);
        STAMP_IF(CROSS && e < NX && e1 >= NX, 2);
        STAMP_IF(CROSS && e < NX + NO1 && e1 >= NX + NO1, 3);
        ia = nx; nx = nn; pre = pre2; ++n;
        e = e1; e1 = e2; e2 = n == 1 ? e3 : (int)(told + tbase);
    };
    for (;;) {
        body(std::false_type{});
        if (e >= NT) break;
        body(std::true_type{});
        if (e >= NT) break;
    }
    u32 fin;
    asm volatile("ds_add_rtn_u32 v162, %1, %2\n\ts_waitcnt lgkmcnt(0)\n\tv_readfirstlane_b32 %0, v162"
                 : "=s"(fin) : "v"((u32)(size_t)lds + LDS_TAIL + 4), "v"(lane == 0 ? 1u : 0u) : "memory", "v162");
    if (fin + 1 == (u32)nld)
        for (int k = 0; k < NDEC; ++k) enqueue(IDX_END, 0, 0);
}

__device__ __forceinline__ void decode_role(const Params& p, LAS char* lds, u32 me, int lane, u32 call) {
    const int rank = rfl(p.rank);
    const int j = rank & 3;
    const u32 par = OFF_PAR + (call & 1) * PAR_BYTES;
    const rsrc_t r0 = mk_rsrc(p.ws[rank], WS_NEED), ro = mk_rsrc(p.out[rank], OUT_BYTES);
    const u32 vo = lane * 16, vout = (lane >> 4) * 4 * 4096 + (lane & 15) * 16;
    LAS char* scr = lds + LDS_SCR + me * SCR;
    u32 q0, q1, q2, q3;
    auto peek = [&](u32 n) {
        asm volatile("ds_read_b128 v[158:161], %4\n\ts_waitcnt lgkmcnt(0)\n\tv_readfirstlane_b32 %0, v158\n\tv_readfirstlane_b32 %1, v159\n\t"
                     "v_readfirstlane_b32 %2, v160\n\tv_readfirstlane_b32 %3, v161"
                     : "=s"(q0), "=s"(q1), "=s"(q2), "=s"(q3) : "v"((u32)(size_t)lds + LDS_CTL + (n % QLEN) * 16) : "memory", "v158", "v159", "v160", "v161");
        return q0 == n + 1;
    };
    const u32 vsig = lane == 0 ? 0u : 0x80000000u;
    u32 one = 1;
    asm volatile("" : "+v"(one));
    auto signal = [&](u32 c) {
        asm volatile("s_nop 4\n\tbuffer_atomic_add %0, %1, %2, %3 offen sc1" :: "v"(one), "v"(vsig), "s"(r0), "s"(rfl(c)) : "memory");
    };
    const u32x4 nan4 = {0x7fc07fc0u, 0x7fc07fc0u, 0x7fc07fc0u, 0x7fc07fc0u};
    u32 vhead = (u32)(size_t)lds + LDS_TAIL + 8, vone = 1;
    asm volatile("" : "+v"(vhead), "+v"(vone));
    for (;;) {
        u32 n;
        asm volatile("s_mov_b64 exec, 1\n\tds_add_rtn_u32 v158, %1, %2\n\ts_waitcnt lgkmcnt(0)\n\tv_readfirstlane_b32 %0, v158\n\ts_mov_b64 exec, -1"
                     : "=s"(n) : "v"(vhead), "v"(vone) : "memory", "v158");
        while (!peek(n)) __builtin_amdgcn_s_sleep(P_DSLEEP);
        if (q1 == IDX_END) break;
        const u32 dst = q2, item = q3 >> 8, ring = (((q3 >> 4) & 3) * RING + item % RING), peer = q3 & 7;
        const LAS char* sl = lds + ring * 4096;
        const u32x4 c0 = *(const LAS u32x4*)(sl + vo), c1 = *(const LAS u32x4*)(sl + 1024 + vo), c2 = *(const LAS u32x4*)(sl + 2048 + vo);
        const u32 k1 = *(const LAS u32*)(sl + 3072 + lane * 4);
        const u32 hx = *(const LAS unsigned char*)(sl + 3328);
        const bool bad = (q1 & IDX_BAD) != 0;
        const u32 idx = q1 & 0xffff;
        const bool out = idx >= 8192, isw = !out && (idx & 64);
        const u32 step = (out || isw) ? 4096 : 1024;
        u32x4 h[4];
        bool done = false;
        if (!bad && !out) {
            const u32 hd = rfl(hx);
            if (hd != 255) {
                if (hd & 64) {
                    u32 t;
                    asm volatile("s_nop 4\n\tbuffer_load_dword %0, %1, %2, %3 offen sc0 sc1\n\ts_waitcnt vmcnt(0)"
                                 : "=&v"(t) : "v"(lane < 16 ? lane * 4 : 0x80000000u), "s"(mk_rsrc(p.ws[peer], WS_NEED)),
                                   "s"(rfl(par + REL_PK + idx * 4096 + 3328 + VAR_X)) : "memory");
                    if (lane < 16) *(LAS u32*)(lds + ring * 4096 + 3328 + VAR_X + lane * 4) = t;
                    asm volatile("" ::: "memory");
                }
                unpack_item(h, c0, c1, c2, k1, sl + 3328, hd, scr, lane);
                done = true;
            }
        } else if (!bad) {
            const u32 g1 = *(const LAS unsigned short*)(sl + 3328 + lane * 2), hd = rfl(*(const LAS unsigned short*)(sl + 3456));
            if ((hd & 255) != 255) { unpack_out(h, c0, c1, c2, k1, g1, (const LAS unsigned char*)(sl + 3456), hd & 255, hd >> 8, scr, lane); done = true; }
        }
        if (!done) {
            const u32 blk = idx >> 7, s = idx & 127;
            const u32 raw = par + (out ? REL_PK + STG_BYTES + (idx + j * 1024) * 4096
                                       : isw ? REL_B + (blk * CF + (s & 60)) * 4096 + (s & 3) * 1024 : (blk * RS1 + s) * 4096);
            const u32 rs = isw ? 4096 : 1024;
            const rsrc_t rp = mk_rsrc(p.ws[peer], WS_NEED);
            asm volatile("s_nop 4\n\tbuffer_load_dwordx4 %0, %4, %5, %6 offen sc0 sc1\n\tbuffer_load_dwordx4 %1, %4, %5, %7 offen sc0 sc1\n\t"
                         "buffer_load_dwordx4 %2, %4, %5, %8 offen sc0 sc1\n\tbuffer_load_dwordx4 %3, %4, %5, %9 offen sc0 sc1\n\ts_waitcnt vmcnt(0)"
                         : "=&v"(h[0]), "=&v"(h[1]), "=&v"(h[2]), "=&v"(h[3])
                         : "v"(vo), "s"(rp), "s"(rfl(raw)), "s"(rfl(raw + rs)), "s"(rfl(raw + 2 * rs)), "s"(rfl(raw + 3 * rs)) : "memory");
            if (bad) { h[0] = nan4; h[1] = nan4; h[2] = nan4; h[3] = nan4; }
        }
        lds_put(lds + LDS_DONE + ring * 4, item + 1);
        const rsrc_t rd = out ? ro : r0;
        const u32 vd = (out ? vout : vo + par) + dst;
#pragma unroll
        for (int i = 0; i < 4; ++i) st_sys(h[i], rd, vd, i * step);
        if (!out) {
            asm volatile("s_waitcnt vmcnt(0)" ::: "memory");
            signal(OFF_CNT + (idx >> 7) * CNT_BLK + peer * 128 + (isw ? 1024 : (!(q1 & IDX_CROSS) && (idx & 127) >= FI1) ? 2048 : 0));
        }
    }
    STAMP(4);
}

#define GLDS2(m0v, voff, rs, soff, imm0, imm1) \
    asm volatile("s_mov_b32 m0, %0\n\ts_nop 0\n\tbuffer_load_dwordx4 %1, %2, %3 offen offset:" #imm0 " sc0 sc1 lds\n\t" \
                 "buffer_load_dwordx4 %1, %2, %3 offen offset:" #imm1 " sc0 sc1 lds" \
                 :: "s"(m0v), "v"(voff), "s"(rs), "s"(soff) : "memory")
#define BARRIER() do { __builtin_amdgcn_sched_barrier(0); __builtin_amdgcn_s_barrier(); __builtin_amdgcn_sched_barrier(0); } while (0)

#define SCHED() __builtin_amdgcn_sched_barrier(0)
#define WAIT_LGKM0() __builtin_amdgcn_s_waitcnt(0xC07F)

template <int STAGE>
struct Gemm {
    static constexpr int NT = STAGE == 1 ? 128 : 64;
    static constexpr int NA = STAGE == 1 ? 4 : 2;
    static constexpr int NB = STAGE == 1 ? 4 : 8;
    static constexpr int NM = STAGE == 1 ? 4 : 1;
    static constexpr int NMM = STAGE == 1 ? 16 : 8;
    static constexpr int NRD = NA + NB;
    struct Frags { f16x8 a[NA], b[NB]; };

    rsrc_t rs;
    u32 sa, sa2, sb;
    u32 ablk;
    u32 ldsw;
    int wid, lane;

    template <int I> __device__ __forceinline__ void piece(int t, int buf) const {
        const u32 vo = lane * 16;
        const u32 ma = ldsw + buf * BUF_BYTES + wid * 4096, mb = ldsw + buf * BUF_BYTES + 32768 + wid * 2048;
        const u32 ka = STAGE == 1 ? (u32)(t >> 1) * (RS1 * 4096) + (t & 1) * 2048 : (u32)t * ablk;
        const u32 a = sa + ka, a2 = sa2 + ka;
        const u32 b = STAGE == 1 ? sb + (u32)(t >> 1) * (CF * 4096) + (t & 1) * 2048 : sb + (u32)t * (CF * 4096);
        if (I == 0) GLDS2(ma, vo, rs, a, 0, 1024);
        if (I == 1) GLDS2(ma, vo, rs, a2, 2048, 3072);
        if (I == 2) GLDS2(mb, vo, rs, b, 0, 1024);
    }
    __device__ __forceinline__ void stage(int t, int buf) const { piece<0>(t, buf); piece<1>(t, buf); piece<2>(t, buf); }

    template <int I> __device__ __forceinline__ void rd(Frags& f, const LAS char* lds, int buf, int half) const {
        const LAS char* base = lds + buf * BUF_BYTES + lane * 16;
        if constexpr (STAGE == 1) {
            const LAS char* pa = base + (wid >> 1) * 8192 + half * 1024;
            const LAS char* pb = base + 32768 + (wid & 1) * 8192 + half * 1024;
            if constexpr (I == 0) f.a[0] = *(const LAS f16x8*)pa;
            else if constexpr (I <= 4) f.b[I - 1] = *(const LAS f16x8*)(pb + (I - 1) * 2048);
            else f.a[I - 4] = *(const LAS f16x8*)(pa + (I - 4) * 2048);
        } else {
            const LAS char* pa = base + wid * 4096 + half * 2048;
            const LAS char* pb = base + 32768 + half * 2048;
            constexpr int h = I / 5, n = I % 5;
            if constexpr (n == 0) f.a[h] = *(const LAS f16x8*)(pa + h * 1024);
            else f.b[h * 4 + n - 1] = *(const LAS f16x8*)(pb + (n - 1) * 4096 + h * 1024);
        }
    }
    template <int I> __device__ __forceinline__ void mm(f32x4 (&acc)[NM][4], const Frags& f) const {
        if constexpr (STAGE == 1) acc[I >> 2][I & 3] = __builtin_amdgcn_mfma_f32_16x16x32_f16(f.a[I >> 2], f.b[I & 3], acc[I >> 2][I & 3], 0, 0, 0);
        else            acc[0][I & 3] = __builtin_amdgcn_mfma_f32_16x16x32_f16(f.a[I >> 2], f.b[I], acc[0][I & 3], 0, 0, 0);
    }
    template <int I, int N> __device__ __forceinline__ void rds(Frags& f, const LAS char* lds, int buf, int half) const {
        if constexpr (I < N) { rd<I>(f, lds, buf, half); rds<I + 1, N>(f, lds, buf, half); }
    }
    static __device__ __forceinline__ int blk_of(int t) { return STAGE == 1 ? (t >> 1) : t; }

    template <bool RD, int ST>
    __device__ __forceinline__ void half_step(f32x4 (&acc)[NM][4], const Frags& cur, Frags& nxt, const LAS char* lds, int rbuf, int rhalf,
                                              int ts, int bs, u32 cvo, u32 fso) const {
        auto first = [&] {
            if (ST == 1) asm volatile("buffer_load_dword v166, %0, %1, %2 offen sc0 sc1" :: "v"(cvo), "s"(rs), "s"(fso) : "memory", "v166");
            if (ST) { piece<0>(ts, bs); SCHED(); }
        };
        if constexpr (STAGE == 1) {
            mm<0>(acc, cur);  mm<1>(acc, cur);  if (RD) rds<0, 2>(nxt, lds, rbuf, rhalf); SCHED();
            mm<2>(acc, cur);  mm<3>(acc, cur);  if (RD) rds<2, 4>(nxt, lds, rbuf, rhalf); SCHED();
            mm<4>(acc, cur);  mm<5>(acc, cur);  if (RD) rds<4, 6>(nxt, lds, rbuf, rhalf); SCHED();
            mm<6>(acc, cur);  mm<7>(acc, cur);  if (RD) rds<6, 8>(nxt, lds, rbuf, rhalf); SCHED();
            mm<8>(acc, cur);  mm<9>(acc, cur);  SCHED();
            first();
            mm<10>(acc, cur); mm<11>(acc, cur); SCHED();
            if (ST) { piece<1>(ts, bs); SCHED(); }
            mm<12>(acc, cur); mm<13>(acc, cur); SCHED();
            if (ST) { piece<2>(ts, bs); SCHED(); }
            mm<14>(acc, cur); mm<15>(acc, cur); SCHED();
        } else {
            first();
            mm<0>(acc, cur); SCHED();
            if (ST) { piece<1>(ts, bs); SCHED(); }
            mm<1>(acc, cur); SCHED();
            if (ST) { piece<2>(ts, bs); SCHED(); }
            mm<2>(acc, cur); if (RD) rds<0, 3>(nxt, lds, rbuf, rhalf); SCHED();
            mm<3>(acc, cur); if (RD) rds<3, 5>(nxt, lds, rbuf, rhalf); SCHED();
            mm<4>(acc, cur); if (RD) rds<5, 8>(nxt, lds, rbuf, rhalf); SCHED();
            mm<5>(acc, cur); if (RD) rds<8, 10>(nxt, lds, rbuf, rhalf); SCHED();
            mm<6>(acc, cur); mm<7>(acc, cur); SCHED();
        }
    }

    template <int ROLE>
    __device__ __forceinline__ u32 run(f32x4 (&acc)[NM][4], const LAS char* lds, char* ws, u32 cvo, u32 need, u32 call) const {
        constexpr bool LATE = ROLE == 2 && STAGE == 1;
        char* tmo = ws + OFF_TMO;
        Frags f0, f1;
        if (ROLE == 0) spin_ge(ws + cvo + blk_of(0) * CNT_BLK, need, tmo, call);
        BARRIER();
        stage(0, 0);
        if (ROLE == 0) { spin_ge(ws + cvo + blk_of(1) * CNT_BLK, need, tmo, call); spin_ge(ws + cvo + blk_of(2) * CNT_BLK, need, tmo, call); }
        BARRIER();
        stage(1, 1);
        if (!LATE) { stage(2, 2); asm volatile("s_waitcnt vmcnt(12)" ::: "memory"); }
        else asm volatile("s_waitcnt vmcnt(6)" ::: "memory");
        BARRIER();
        rds<0, NRD>(f0, lds, 0, 0);
        WAIT_LGKM0();
        if (ROLE == 0) asm volatile("v_mov_b32 v166, 0" ::: "v166");
        int buf = 0;
        for (int t = 0; t < NT - 3; ++t) {
            const int nb = buf == 2 ? 0 : buf + 1, pb = buf == 0 ? 2 : buf - 1;
            half_step<true, LATE ? 2 : 0>(acc, f0, f1, lds, buf, 1, t + 2, pb, 0, 0);
            if (ROLE == 0) {
                u64 miss;
                asm volatile("s_waitcnt vmcnt(6)\n\tv_cmp_lt_u32_e64 %0, v166, %1\n\ts_nop 0" : "=s"(miss) : "v"(need) : "memory");
                WAIT_LGKM0();
                if (miss != 0) spin_ge(ws + cvo + blk_of(t + 3) * CNT_BLK, need, tmo, call);
            } else {
                asm volatile("s_waitcnt vmcnt(6)" ::: "memory");
                WAIT_LGKM0();
            }
            BARRIER();
            half_step<true, LATE ? 0 : (ROLE == 0 ? 1 : 2)>(acc, f1, f0, lds, nb, 0, t + 3, buf, cvo, blk_of(t + 4 < NT ? t + 4 : t + 3) * CNT_BLK);
            buf = nb;
        }
#pragma unroll
        for (int t = NT - 3; t < NT; ++t) {
            const int nb = buf == 2 ? 0 : buf + 1, pb = buf == 0 ? 2 : buf - 1;
            if (t == NT - 3) {
                half_step<true, LATE ? 2 : 0>(acc, f0, f1, lds, buf, 1, t + 2, pb, 0, 0);
                asm volatile("s_waitcnt vmcnt(6)" ::: "memory");
            } else {
                if (t == NT - 2) {
                    const u32 to = OFF_TMO;
                    asm volatile("buffer_load_dword v167, %0, %1, 0 offen sc1" :: "v"(to), "s"(rs) : "memory", "v167");
                }
                half_step<true, 0>(acc, f0, f1, lds, buf, 1, 0, 0, 0, 0);
                asm volatile("s_waitcnt vmcnt(0)" ::: "memory");
            }
            WAIT_LGKM0();
            BARRIER();
            if (t < NT - 1) half_step<true, 0>(acc, f1, f0, lds, nb, 0, 0, 0, 0, 0);
            else            half_step<false, 0>(acc, f1, f0, lds, nb, 0, 0, 0, 0, 0);
            buf = nb;
        }
        u32 tw;
        asm volatile("v_readfirstlane_b32 %0, v167\n\ts_nop 1" : "=s"(tw) :: "memory");
        return tw;
    }
};

__device__ __forceinline__ u32x4 gelu4(const f32x4 (&c)[4], int r, bool bad) {
    const float nanv = __builtin_nanf("");
    f32x4 o = {gelu(c[0][r]), gelu(c[1][r]), gelu(c[2][r]), gelu(c[3][r])};
    if (bad) o = (f32x4){nanv, nanv, nanv, nanv};
    return __builtin_bit_cast(u32x4, o);
}

__device__ __forceinline__ void gemm_role(const Params& p, const LAS char* lds, int xcd, int s, int wid, int lane, u32 call) {
    const int rank = rfl(p.rank);
    const int g = rank >> 2, j = rank & 3, og = (g ^ 1) * 4;
    char* ws = p.ws[rank];
    const u32 par = OFF_PAR + (call & 1) * PAR_BYTES;
    const rsrc_t rs = mk_rsrc(ws, WS_NEED), ro = mk_rsrc(p.out[rank], OUT_BYTES);
    const u32 ldsw = rfl((u32)(size_t)lds);
    const u32 vout = (lane >> 4) * 4 * 4096 + (lane & 15) * 16;
    {
        const int u = (xcd & 1) * 26 + s, rt = u >> 2, ct = (xcd >> 1) * 4 + (u & 3);
        Gemm<1> G;
        G.rs = rs; G.ldsw = ldsw; G.wid = wid; G.lane = lane;
        const int rf = rt * 16 + wid * 2;
        G.sa = par + (rf < FI ? rf : rf + FX) * 4096;
        G.sa2 = par + (rf + 1 < FI ? rf + 1 : rf + 1 + FX) * 4096 - 2048;
        G.sb = par + REL_B + (ct * 8 + wid) * 4096;
        const int bsrc = ct < 8 ? rank : og + ((ct - 8) >> 1);
        u32 cvo = OFF_CNT + 1024 + bsrc * 128, need = (ct < 8 ? 8u : 16u) * call;
        u32 mask = 0;
        for (int i = 0; i < 16; ++i) mask |= 1u << src_of_rf1(rt * 16 + i, g, j);
        if (lane < 8 && ((mask >> lane) & 1)) {
            cvo = OFF_CNT + lane * 128;
            need = (lane == rank ? 8 : ((lane >> 2) == g ? FI1 : FX)) * call;
        }
        f32x4 acc[4][4] = {};
        const bool bad = call == (wid == 0 ? G.template run<0>(acc, lds, ws, cvo, need, call)
                                  : wid < 4 ? G.template run<1>(acc, lds, ws, cvo, need, call)
                                            : G.template run<2>(acc, lds, ws, cvo, need, call));
        STAMP(1);
        const int wm = wid >> 1, wn = wid & 1;
        if (ct < 8) {
#pragma unroll
            for (int m = 0; m < 4; ++m) {
                const u32 ob = (u32)row_of_rf1(rt * 16 + wm * 4 + m, g, j) * 4096 + (ct * 128 + wn * 64) * 4;
#pragma unroll
                for (int r = 0; r < 4; ++r) __builtin_amdgcn_raw_buffer_store_b128(gelu4(acc[m], r, bad), ro, vout + ob + r * 4096, 0, OUT_AUX);
            }
        } else {
            const int fo = (rt * 2 + (ct & 1)) * 8 + wid;
            const u32 sb = par + REL_STG + (u32)(((ct - 8) >> 1) * 1024 + fo * 4) * 4096;
            u32x4 o[4][4];
#pragma unroll
            for (int i = 0; i < 16; ++i) o[i >> 2][i & 3] = gelu4(acc[i >> 2], i & 3, bad);
            u32 lop, lon;
            out_windows<4>(o, lop, lon);
#pragma unroll
            for (int m = 0; m < 4; ++m) pack_out(o[m], lop, lon, (LAS char*)lds + wid * 2048, rs, sb + m * 4096, lane);
            WAIT_VM0();
            if (lane == 0) st_flag(p.ws[bsrc] + OFF_RDY + rank * 8192 + (1024 + fo) * 4, call);
        }
        STAMP(2);
    }
    {
        const bool loc = s >= 24;
        const int rt = s >> 2, ct = loc ? xcd * 2 + (s & 1) : ((~xcd & 1) * 16) + (xcd >> 1) * 4 + (s & 3);
        Gemm<2> G;
        G.rs = rs; G.ldsw = ldsw; G.wid = wid; G.lane = lane;
        G.sa = G.sa2 = par + (loc ? (FI + wid) * 4096 : REL_A2 + (rt * 8 + wid) * 4096);
        G.ablk = (loc ? RS1 : RF2) * 4096;
        G.sb = par + REL_B + ct * 4 * 4096 + wid * 2048;
        const int asrc = loc ? rank : g * 4 + ((j + 1 + (rt >> 1)) & 3);
        const int bsrc = ct < 16 ? rank : og + ((ct - 16) >> 2);
        u32 cvo = OFF_CNT + 1024 + bsrc * 128, need = (ct < 16 ? 8u : 16u) * call;
        if (lane == 0) { cvo = OFF_CNT + (loc ? 0 : 2048) + asrc * 128; need = (loc ? 8 : F2) * call; }
        f32x4 acc[1][4] = {};
        const bool bad = call == (wid == 0 ? G.template run<0>(acc, lds, ws, cvo, need, call)
                                  : wid < 4 ? G.template run<1>(acc, lds, ws, cvo, need, call)
                                            : G.template run<2>(acc, lds, ws, cvo, need, call));
        STAMP(3);
        if (ct < 16) {
            const u32 ob = (u32)(asrc * NS + ((loc ? FI : FI1 + (rt & 1) * 8) + wid) * 16) * 4096 + ct * 256;
#pragma unroll
            for (int r = 0; r < 4; ++r) __builtin_amdgcn_raw_buffer_store_b128(gelu4(acc[0], r, bad), ro, vout + ob + r * 4096, 0, OUT_AUX);
        } else {
            const int z = (rt * 4 + (ct & 3)) * 8 + wid;
            const u32 sb = par + REL_STG + (u32)(((ct - 16) >> 2) * 1024 + 832 + z) * 4096;
            const u32x4 o[1][4] = {{gelu4(acc[0], 0, bad), gelu4(acc[0], 1, bad), gelu4(acc[0], 2, bad), gelu4(acc[0], 3, bad)}};
            u32 lop, lon;
            out_windows<1>(o, lop, lon);
            pack_out(o[0], lop, lon, (LAS char*)lds + wid * 2048, rs, sb, lane);
            WAIT_VM0();
            if (lane == 0) st_flag(p.ws[bsrc] + OFF_RDY + rank * 8192 + (1024 + 208 + z) * 4, call);
        }
        STAMP(4);
    }
}

__global__ __launch_bounds__(BLOCK, 3) void ag_gemm_kernel(Params p) {
    __shared__ __attribute__((aligned(1024))) char smem[LDS_BYTES];
    const int rank = rfl(p.rank);
    const int lane = threadIdx.x & 63;
    const int wid = rfl(threadIdx.x >> 6);
    const int b = blockIdx.x;
    if (wid >= (b < NCOMM ? NLW + NDEC : 8)) return;
    STAMP(0);
    char* seq = p.ws[rank] + OFF_SEQ + (b * 16 + wid) * 4;
    const u32 prev = ld_agent(seq);
    char* wsl = p.ws[lane & 7];
    auto next_call = [&] {
        const u32 call = rfl(prev) + 1;
        if (lane == 0) __hip_atomic_store((GAS u32*)seq, call, __ATOMIC_RELAXED, __HIP_MEMORY_SCOPE_AGENT);
        STAMP(7);
        return call;
    };
    if (b >= NCOMM) {
        for (int i = ((b - NCOMM) >> 4) - 1; i > 0; --i) __builtin_amdgcn_s_sleep(P_GRADE);
        f32x4 v[8];
        load_item(v, p, b - NCOMM, wid, lane);
        const u32 call = next_call();
        convert_local(p, (LAS char*)smem, v, wsl, b - NCOMM, wid, lane, call);
        STAMP(5);
        gemm_role(p, (const LAS char*)smem, b & 7, (b - NCOMM) >> 3, wid, lane, call);
        return;
    }
    const int nld = 2 * NCOMM + b < 7 * WL ? 3 : 2;
    constexpr int NSET = NLW + NDEC - NPW;
    const int sw = (NLW + NDEC - 1 - wid) * NCOMM + b;
    if (wid < NSET) {
        if (wid < 4) *(LAS u32*)(smem + LDS_CTL + (wid * 64 + lane) * 4) = 0;
        for (int k = 0; k < nld; ++k)
            build_table((LAS char*)smem + LDS_TAB + k * TAB_BYTES, rank, (k * NCOMM + b) % 7, threadIdx.x, NSET * 64);
    }
    const u32 call = next_call();
    if (wid >= NSET && sw < NPUSH) push_role(p, wsl, sw, lane, call);
    else asm volatile("s_waitcnt lgkmcnt(0)\n\ts_barrier" ::: "memory");
    STAMPB_IF(wid == 0, 4);
    if (wid >= NLW) decode_role(p, (LAS char*)smem, wid - NLW, lane, call);
    else if (wid < nld) {
        const int ps = rfl((wid * NCOMM + b) % 7), wv = rfl((wid * NCOMM + b) / 7);
        if (ps < 3) load_role<false>(p, (LAS char*)smem, wid, nld, ps, wv, lane, call);
        else        load_role<true>(p, (LAS char*)smem, wid, nld, ps, wv, lane, call);
    }
}

}

constexpr int g_mode = 0;

extern "C" void kernel_launch(int rank, int nranks, ncclComm_t comm, hipStream_t stream,
                              void* const* d_in, const int* in_sizes, int n_in,
                              void* d_out, int out_size, const int* in_dtypes, int out_dtype,
                              void* const* d_out_all, void* const* d_ws_all, size_t ws_bytes) {
    (void)comm; (void)in_sizes; (void)n_in; (void)d_out; (void)out_size; (void)in_dtypes; (void)out_dtype;
    if (nranks != NR || (unsigned)rank >= (unsigned)NR || ws_bytes < WS_NEED) {
        fprintf(stderr, "unsupported configuration\n");
        return;
    }
    Params p;
    p.x = (const float*)d_in[0];
    p.w = (const float*)d_in[1];
    for (int i = 0; i < NR; ++i) { p.ws[i] = (char*)d_ws_all[i]; p.out[i] = (char*)d_out_all[i]; }
    p.rank = rank;
    p.pad = g_mode;
    ag_gemm_kernel<<<GRID, BLOCK, 0, stream>>>(p);
    hipError_t e = hipGetLastError();
    if (e != hipSuccess) fprintf(stderr, "launch failed: %s\n", hipGetErrorString(e));
}
```

```cpp
#include <hip/hip_runtime.h>
#include <rccl/rccl.h>
#include <cstdio>
#include <cstdint>
#include <type_traits>

typedef unsigned u32;
typedef unsigned long long u64;
typedef u32 u32x4 __attribute__((ext_vector_type(4)));
typedef u32 u32x2v __attribute__((ext_vector_type(2)));
typedef float f32x4 __attribute__((ext_vector_type(4)));
typedef _Float16 f16x8 __attribute__((ext_vector_type(8)));
typedef __amdgpu_buffer_rsrc_t rsrc_t;
#define LAS __attribute__((address_space(3)))
#define GAS __attribute__((address_space(1)))

namespace {

constexpr int NR = 8;
constexpr int KD = 8192;
constexpr int NS = 1024;
constexpr int NBLK = 64;
#define P_FI 45
constexpr int FI = P_FI;
constexpr int FX = 64 - FI;
#define P_FO2 8
constexpr int FO2 = P_FO2, RF2 = 48 - FO2, FO1 = FI - FO2;
__host__ __device__ constexpr int OFF2(int i) { return (RF2 * i + 2) / 3; }
__host__ __device__ constexpr int F2(int i) { return OFF2(i + 1) - OFF2(i); }
__host__ __device__ constexpr int FI1(int i) { return FI - F2(i); }
__host__ __device__ constexpr int OFF1(int i) { return i * FI - OFF2(i); }
constexpr int NI1 = 3 * FI - RF2, FI1MAX = FI1(2);
constexpr int FL = 8;
constexpr int RF1 = 208, CF = 128;
static_assert(FO1 + NI1 + 4 * FX == RF1 && FX >= 16 && FX - FL <= 16 && (FO2 == 0 || FO2 == 8) && F2(0) >= F2(2), "rows");

#define P_W 19
#define P_XA 14
#define P_PB 1
#define P_GRADE 20
constexpr int PB = P_PB;
constexpr int NCOMM = 48;
constexpr int WL = P_W;
constexpr int XA = P_XA;
static_assert(WL >= 14 && WL <= 20 && XA <= FX && XA <= 16, "wave roles");
constexpr int GRID = 256, BLOCK = 768, NGEMM = GRID - NCOMM;
constexpr int RS1 = 64 + NI1 + 4 * FX;

constexpr u32 OFF_SEQ  = 0;
constexpr u32 OFF_TMO  = 16384;
constexpr u32 OFF_QUE  = 16640;
constexpr u32 OFF_RDY  = 32768;
constexpr u32 OFF_CNT  = 131072;
constexpr u32 CNT_BLK  = 24 * 128;
constexpr u32 OFF_DIAG = 524288;
constexpr u32 A1_BYTES = (u32)NBLK * RS1 * 4096;
constexpr u32 A2_BYTES = (u32)NBLK * RF2 * 4096;
constexpr u32 B_BYTES  = (u32)NBLK * CF * 4096;
constexpr u32 STG_BYTES = 4u * 1024 * 4096;
constexpr u32 PK_BYTES = (u32)NBLK * 128 * 4096;
constexpr u32 REL_A2 = A1_BYTES, REL_B = REL_A2 + A2_BYTES, REL_PK = REL_B + B_BYTES, REL_STG = REL_PK + PK_BYTES;
constexpr u32 PAR_BYTES = REL_STG + 2 * STG_BYTES;
constexpr u32 OFF_PAR  = 1u << 20;
constexpr int VAR_XO = 192;
constexpr u32 WS_NEED  = OFF_PAR + 2 * PAR_BYTES;
constexpr u32 OUT_BYTES = (u32)KD * NS * 4;
static_assert(REL_PK < (256u << 20), "table entries hold 18 bits of dst / 1024");
static_assert(256 + VAR_XO == 448, "output items are as long as x items");
static_assert(PK_BYTES == 8192u * 4096u, "staged outputs follow the packed image");
static_assert(OFF_CNT + NBLK * CNT_BLK <= OFF_DIAG && OFF_RDY + 8 * 8192 + 256 <= OFF_CNT, "control area");

constexpr u64 DEADLINE = 40000000ull;

constexpr int BUF_BYTES = 49152;
constexpr int LDS_BYTES = 3 * BUF_BYTES;

struct Params {
    const float* x;
    const float* w;
    char* ws[NR];
    char* out[NR];
    int rank;
    int pad;
};

__device__ __forceinline__ u32 rfl(u32 v) { return __builtin_amdgcn_readfirstlane(v); }
__device__ __forceinline__ char* uni(const void* p) {
    const u64 a = (u64)p;
    return (char*)(((u64)rfl((u32)(a >> 32)) << 32) | rfl((u32)a));
}
__device__ __forceinline__ rsrc_t rsrc_u(const void* p, u32 bytes) {
    return __builtin_amdgcn_make_buffer_rsrc((void*)p, (short)0, (int)bytes, 0x00020000);
}
__device__ __forceinline__ rsrc_t mk_rsrc(const void* p, u32 bytes) { return rsrc_u(uni(p), bytes); }
__device__ __forceinline__ u32 ld_sys(const char* p) {
    return __hip_atomic_load((const GAS u32*)p, __ATOMIC_RELAXED, __HIP_MEMORY_SCOPE_SYSTEM);
}
__device__ __forceinline__ u32 ld_agent(const char* p) {
    return __hip_atomic_load((const GAS u32*)p, __ATOMIC_RELAXED, __HIP_MEMORY_SCOPE_AGENT);
}
__device__ __forceinline__ void st_flag(char* p, u32 v) {
    __hip_atomic_store((GAS u32*)p, v, __ATOMIC_RELAXED, __HIP_MEMORY_SCOPE_SYSTEM);
}
__device__ __forceinline__ void add_sys(char* p) {
    __hip_atomic_fetch_add((GAS u32*)p, 1u, __ATOMIC_RELAXED, __HIP_MEMORY_SCOPE_SYSTEM);
}
#define WAIT_VM0() do { asm volatile("s_waitcnt vmcnt(0)" ::: "memory"); __builtin_amdgcn_s_waitcnt(0x0F70); } while (0)

template <int SLEEP = 2>
__device__ __forceinline__ bool spin_ge(const char* p, u32 need, char* tmo, u32 call) {
    const u64 t0 = __builtin_amdgcn_s_memrealtime();
    bool gave_up = false;
    for (u32 polls = 0;; ++polls) {
        const u32 v = ld_sys(p);
        if (__builtin_amdgcn_ballot_w64(v < need) == 0) break;
        if ((polls & 7) == 0 && (rfl(ld_agent(tmo)) == call || __builtin_amdgcn_s_memrealtime() - t0 > DEADLINE)) {
            __hip_atomic_store((GAS u32*)tmo, call, __ATOMIC_RELAXED, __HIP_MEMORY_SCOPE_AGENT);
            gave_up = true;
            break;
        }
        __builtin_amdgcn_s_sleep(SLEEP);
    }
    asm volatile("" ::: "memory");
    return gave_up;
}

#define STAMP_IF(c, slot) do {} while (0)
#define STAMP(slot) do {} while (0)
#define STAMPB_IF(c, slot) do {} while (0)

__device__ __forceinline__ u32x4 cvt8(f32x4 a, f32x4 b) {
    f16x8 h;
    h[0] = (_Float16)a[0]; h[1] = (_Float16)a[1]; h[2] = (_Float16)a[2]; h[3] = (_Float16)a[3];
    h[4] = (_Float16)b[0]; h[5] = (_Float16)b[1]; h[6] = (_Float16)b[2]; h[7] = (_Float16)b[3];
    return __builtin_bit_cast(u32x4, h);
}
__device__ __forceinline__ void st_sys(u32x4 v, rsrc_t r, u32 voff, u32 soff) {
    __builtin_amdgcn_raw_buffer_store_b128(v, r, voff + soff, 0, 17);
}
typedef float f32x2 __attribute__((ext_vector_type(2)));
__device__ __forceinline__ f32x2 gelu2(f32x2 y) {
    constexpr float A = -2.8853900817779268f * 0.7978845608028654f, B = A * 0.044715f;
    const f32x2 z = y * (y * y * B + A);
    const f32x2 d = (f32x2){__builtin_amdgcn_exp2f(z[0]), __builtin_amdgcn_exp2f(z[1])} + 1.0f;
    return y * (f32x2){__builtin_amdgcn_rcpf(d[0]), __builtin_amdgcn_rcpf(d[1])};
}

__device__ __forceinline__ int row_of_slot(int sl, int g, int j) {
    if (sl < 64) return (g * 4 + j) * NS + sl * 16;
    sl -= 64;
    if (sl < NI1) { const int i = sl >= OFF1(2) ? 2 : sl >= OFF1(1) ? 1 : 0; return (g * 4 + ((j + 1 + i) & 3)) * NS + (sl - OFF1(i)) * 16; }
    sl -= NI1;
    const int q = sl / FX;
    return ((g ^ 1) * 4 + q) * NS + (FI + sl - q * FX) * 16;
}
__device__ __forceinline__ int slot_of(int f, int q) {
    if (q >= 0) {
        if (f >= RF1 - FX) return -1;
        if (f >= RF1 - 4 * FX + q * FX) f += FX;
    } else if (f >= RF1) return f - RF1 < FX - FL ? FI + FL + f - RF1 : -1;
    return f < FO1 ? f : f + 64 - FO1;
}
__device__ __forceinline__ int row_of_f2(int f2, int g, int j) {
    if (f2 >= RF2) return (g * 4 + j) * NS + (FO1 + f2 - RF2) * 16;
    const int i = f2 >= OFF2(2) ? 2 : f2 >= OFF2(1) ? 1 : 0;
    return (g * 4 + ((j + 1 + i) & 3)) * NS + (FI1(i) + f2 - OFF2(i)) * 16;
}

constexpr int IN_AUX = 2;
constexpr int OUT_AUX = 17;
__device__ __forceinline__ u32x4 cvt_w(const f32x4* v, int c) {
    return cvt8((f32x4){v[0][c], v[1][c], v[2][c], v[3][c]}, (f32x4){v[4][c], v[5][c], v[6][c], v[7][c]});
}

#define DPP_STEP(op, v, ctrl, rmask) v = op(v, (u32)__builtin_amdgcn_update_dpp(0, (int)v, ctrl, rmask, 0xf, false))
__device__ __forceinline__ u32 umax2(u32 a, u32 b) { return a > b ? a : b; }
__device__ __forceinline__ u32 uadd2(u32 a, u32 b) { return a + b; }
__device__ __forceinline__ u32 wave_max(u32 v) {
    DPP_STEP(umax2, v, 0x111, 0xf); DPP_STEP(umax2, v, 0x112, 0xf); DPP_STEP(umax2, v, 0x114, 0xf); DPP_STEP(umax2, v, 0x118, 0xf);
    DPP_STEP(umax2, v, 0x142, 0xa); DPP_STEP(umax2, v, 0x143, 0xc);
    return __builtin_amdgcn_readlane(v, 63);
}
__device__ __forceinline__ u32 wave_sum(u32 v) {
    DPP_STEP(uadd2, v, 0x111, 0xf); DPP_STEP(uadd2, v, 0x112, 0xf); DPP_STEP(uadd2, v, 0x114, 0xf); DPP_STEP(uadd2, v, 0x118, 0xf);
    DPP_STEP(uadd2, v, 0x142, 0xa); DPP_STEP(uadd2, v, 0x143, 0xc);
    return __builtin_amdgcn_readlane(v, 63);
}

__device__ __forceinline__ u32 wave_scan(u32 v) {
    DPP_STEP(uadd2, v, 0x111, 0xf); DPP_STEP(uadd2, v, 0x112, 0xf); DPP_STEP(uadd2, v, 0x114, 0xf); DPP_STEP(uadd2, v, 0x118, 0xf);
    DPP_STEP(uadd2, v, 0x142, 0xa); DPP_STEP(uadd2, v, 0x143, 0xc);
    return v;
}
__device__ __forceinline__ u32 byte_scan(u32 m) {
    u32 a, r;
    asm("v_lshl_add_u32 %0, %1, 8, %1" : "=v"(a) : "v"(m));
    asm("v_lshl_add_u32 %0, %1, 16, %1" : "=v"(r) : "v"(a));
    return r;
}
__device__ __forceinline__ u32 times7(u32 m) {
    u32 a, r;
    asm("v_lshl_add_u32 %0, %1, 1, %1" : "=v"(a) : "v"(m));
    asm("v_lshl_add_u32 %0, %1, 2, %2" : "=v"(r) : "v"(m), "v"(a));
    return r;
}
__device__ __forceinline__ u32 rank_in(u64 m) { return __builtin_amdgcn_mbcnt_hi((u32)(m >> 32), __builtin_amdgcn_mbcnt_lo((u32)m, 0)); }

constexpr int VAR_X = VAR_XO, VAR_W = 256;
__device__ __forceinline__ u32 h7_of(float a) { return (u32)__builtin_bit_cast(unsigned short, (_Float16)a) >> 8; }
__device__ __forceinline__ void lds_or(LAS char* a, u32 v) {
    __hip_atomic_fetch_or((LAS u32*)a, v, __ATOMIC_RELAXED, __HIP_MEMORY_SCOPE_WORKGROUP);
}
__device__ __forceinline__ void put_bits(LAS char* area, u32 bo, u64 v) {
    const u32 sh = bo & 31;
    const u64 w = v << sh;
    LAS char* a = area + (bo >> 5) * 4;
    lds_or(a, (u32)w); lds_or(a + 4, (u32)(w >> 32)); lds_or(a + 8, (u32)((v >> 1) >> (63 - sh)));
}
__device__ __forceinline__ u64 get_bits(const LAS char* area, u32 bo) {
    const LAS u32* w = (const LAS u32*)(area + (bo >> 5) * 4);
    const u32 w0 = w[0], w1 = w[1], w2 = w[2];
    return (u64)__builtin_amdgcn_alignbit(w2, w1, bo) << 32 | __builtin_amdgcn_alignbit(w1, w0, bo);
}
template <class F>
__device__ __forceinline__ void pack_item(const u32x4 (&h)[4], float amax, LAS char* scr, rsrc_t r0, u32 dst, int lane, F before_stores) {
    u32 L[8], H[8];
#pragma unroll
    for (int t = 0; t < 8; ++t) {
        const u32 d0 = h[t >> 1][(t & 1) * 2], d1 = h[t >> 1][(t & 1) * 2 + 1];
        L[t] = __builtin_amdgcn_perm(d1, d0, 0x06040200u);
        H[t] = __builtin_amdgcn_perm(d1, d0, 0x07050301u);
    }
    *(LAS u32x4*)(scr + lane * 16) = (u32x4){H[0], H[1], H[2], H[3]};
    *(LAS u32x4*)(scr + lane * 16 + 1024) = (u32x4){H[4], H[5], H[6], H[7]};
    LAS char* area = scr + 2048;
    *(LAS u32x2v*)(area + lane * 8) = (u32x2v){0u, 0u};
    const u32 lm = wave_sum(h7_of(amax) & 127) + 132, rare = (lm & 255) < 115 ? 1 : 0;
    const u32 E1 = lm < 256 ? 1 : lm < 31 * 256 ? lm >> 8 : 31;
    const u32 eb = E1 * 0x01010101u | 0x80808080u;
    const u32 tbig = rare ? 0x79797979u : 0x7a7a7a7au, vadd = rare ? 0x01010101u : 0x02020202u, top6 = rare ? 0x06060606u : 0u;
    const u32 clo = 0x02010003u, qlo = rare ? 3u : 2u, qhi = rare ? 0x03020100u : 0x03030100u;
    u32 K[2] = {0, 0}, e4 = 0, pos = 0;
    u64 b2 = 0, b3 = 0;
#pragma unroll
    for (int t = 0; t < 8; ++t) {
        const u32 o = (eb - ((H[t] >> 2) & 0x1f1f1f1fu)) ^ 0x80808080u, o7 = o & 0x7f7f7f7fu;
        const u32 far = (o | (o7 + tbig)) & 0x80808080u, huge = (o | (o7 + 0x73737373u)) & 0x80808080u;
        const u32 f7 = (far >> 4) - (far >> 7), sel = (o | f7) & 0x07070707u;
        const u32 c = __builtin_amdgcn_perm(0x03030303u, clo, sel);
        const u32 q = __builtin_amdgcn_perm(qhi, qlo, sel);
        const u32 M = __builtin_amdgcn_perm(0x01010101u, 0x00000001u, sel);
        const u32 M2 = q & (q >> 1);
        const u32 v = (((o7 + vadd) | ((huge >> 4) - (huge >> 7))) & f7) | __builtin_amdgcn_perm(0u, top6 & 255, sel);
        const u32 P = byte_scan(M), P2 = byte_scan(M2), X2 = P2 - M2;
        const u32 bits2 = __builtin_amdgcn_udot4(q, __builtin_amdgcn_perm(0u, 0x40100401u, P - M), 0u, false);
        const u32 bits3 = __builtin_amdgcn_udot4(v, __builtin_amdgcn_perm(0u, 0x00400801u, X2), (X2 >> 24) == 3 ? (v >> 24) << 9 : 0u, false);
        b2 |= (u64)bits2 << (pos & 63);
        b3 |= (u64)bits3 << ((pos >> 16) & 63);
        pos += (P >> 23) + ((P2 >> 24) * 3 << 16);
        K[t >> 2] |= c << (2 * (t & 3));
        e4 = (e4 >> 1) | huge;
        if (t & 1) { asm volatile("" : "+v"(b2), "+v"(b3), "+v"(e4)); __builtin_amdgcn_sched_barrier(0); }
    }
    const u32 incl = wave_scan(pos), tot = __builtin_amdgcn_readlane(incl, 63), ex = incl - pos;
    const u32 used = 8 + (tot & 0xffff) + (tot >> 16);
    u32 need = used;
    if (__builtin_amdgcn_ballot_w64(e4 != 0) != 0) need += 8 * wave_sum(__builtin_popcount(e4));
    const u32 var = need <= 8 * VAR_X ? VAR_X : VAR_W;
    const bool fits = need <= 8 * VAR_W && __builtin_amdgcn_ballot_w64(pos >= (65u << 16)) == 0;
    if (fits) {
        u32 n4 = 0;
        put_bits(area, 8 + (ex & 0xffff), b2);
        put_bits(area, 8 + (tot & 0xffff) + (ex >> 16), b3);
        while (__builtin_amdgcn_ballot_w64(e4 != 0) != 0) {
            const u64 act = __builtin_amdgcn_ballot_w64(e4 != 0);
            const u32 any = (e4 | (e4 >> 8) | (e4 >> 16) | (e4 >> 24)) & 255, t = __builtin_ctz(any | 128);
            const u32 b = __builtin_ctz(((e4 >> t) & 0x01010101u) | 0x80000000u) + t;
            const u32 e = (*(const LAS unsigned char*)(scr + (t & 4) * 256 + lane * 16 + (t & 3) * 4 + ((b >> 3) & 3)) >> 2) & 31;
            const u32 k = var - 1 - n4 - rank_in(act);
            if (e4 != 0) lds_or(area + (k & ~3u), e << ((k & 3) * 8));
            e4 &= ~(1u << (b & 31));
            n4 += __builtin_popcountll(act);
        }
    }
    asm volatile("" ::: "memory");
    u32 vw = *(const LAS u32*)(area + lane * 4);
    if (lane == 0) vw |= fits ? E1 | rare << 5 | (var == VAR_W ? 64 : 0) : 255;
    const u32 P0 = (H[0] & 0x83838383u) | ((H[1] & 0x03030303u) << 2) | ((H[1] >> 1) & 0x40404040u) | ((H[6] & 0x03030303u) << 4);
    const u32 P1 = (H[2] & 0x83838383u) | ((H[3] & 0x03030303u) << 2) | ((H[3] >> 1) & 0x40404040u) | ((H[7] & 0x03030303u) << 4);
    const u32 P2 = (H[4] & 0x83838383u) | ((H[5] & 0x03030303u) << 2) | ((H[5] >> 1) & 0x40404040u)
                 | ((H[6] >> 2) & 0x20202020u) | ((H[7] >> 3) & 0x10101010u);
    before_stores();
    st_sys((u32x4){L[0], L[1], L[2], L[3]}, r0, lane * 16, dst);
    st_sys((u32x4){L[4], L[5], L[6], L[7]}, r0, lane * 16, dst + 1024);
    st_sys((u32x4){P0, P1, P2, K[0]}, r0, lane * 16, dst + 2048);
    __builtin_amdgcn_raw_buffer_store_b32(K[1], r0, lane * 4 + dst + 3072, 0, 17);
    __builtin_amdgcn_raw_buffer_store_b32(vw, r0, lane * 4 + dst + 3328, 0, 17);
}
__device__ __forceinline__ void unpack_item(u32x4 (&h)[4], const u32x4& c0, const u32x4& c1, const u32x4& c2, u32 k1,
                                            const LAS char* area, u32 hd, LAS char* scr, int lane) {
    const u32 E1 = hd & 31, var = (hd & 64) ? VAR_W : VAR_X;
    const bool rare = hd & 32;
    const u32 K[2] = {c2[3], k1};
    const u32 X[2] = {K[0] & (K[0] >> 1) & 0x55555555u, K[1] & (K[1] >> 1) & 0x55555555u};
    const u32 n2 = 2 * (__builtin_popcount(X[0]) + __builtin_popcount(X[1])), i2 = wave_scan(n2);
    u64 b2 = get_bits(area, 8 + i2 - n2);
    b2 &= n2 < 64 ? ~(~0ull << n2) : ~0ull;
    const u64 y2 = b2 & (b2 >> 1) & 0x5555555555555555ull;
    const u32 n3 = 3 * __builtin_popcountll(y2), i3 = wave_scan(n3);
    u64 b3 = get_bits(area, 8 + __builtin_amdgcn_readlane(i2, 63) + i3 - n3);
    const u32 e4x = E1 * 0x04040404u | 0x80808080u;
    const u32 t1 = (e4x - 0x000c0804u) & 0x007c7c7cu, t2 = (e4x - (rare ? 0x00181410u : 0x00001410u)) & 0x007c7c7cu;
    const u32 t3lo = (e4x - (rare ? 0x2824201cu : 0x24201c18u)) & 0x7c7c7c7cu, t3hi = (e4x - (rare ? 0x0000302cu : 0x00302c28u)) & 0x007c7c7cu;
    LAS char* row = scr + lane * 36;
    u32 e4 = 0;
    const u32 nch = (wave_max(n2) + 7) >> 3;
#pragma unroll
    for (int k = 0; k < 8; ++k) {
        if (k >= 4 && k >= nch) break;
        const u32 b8 = (u32)(b2 >> (8 * k)) & 255, y = b8 | (b8 << 12), q = (y | (y << 6)) & 0x03030303u;
        const u32 M2 = q & (q >> 1), P2 = byte_scan(M2), m7 = times7(M2);
        const u32 lo = (u32)b3, z = (lo & 0x3f) | ((lo & 0xfc0) << 10), f = (z | (z << 5)) & 0x07070707u;
        const u32 v = __builtin_amdgcn_perm(0u, f, P2 - M2) & m7;
        b3 >>= (P2 >> 24) * 3;
        *(LAS u32*)(row + 4 * k) = __builtin_amdgcn_perm(0u, t2, q) | __builtin_amdgcn_perm(t3hi, t3lo, v | (0x0c0c0c0cu - (M2 << 2) - (M2 << 3)));
        e4 |= (((v + M2) >> 3) & 0x01010101u) << k;
    }
    u32 n4 = 0;
    while (__builtin_amdgcn_ballot_w64(e4 != 0) != 0) {
        const u64 act = __builtin_amdgcn_ballot_w64(e4 != 0);
        const u32 any = (e4 | (e4 >> 8) | (e4 >> 16) | (e4 >> 24)) & 255, t = __builtin_ctz(any | 128);
        const u32 b = __builtin_ctz(((e4 >> t) & 0x01010101u) | 0x80000000u) + t;
        const u32 x = *(const LAS unsigned char*)(area + ((var - 1 - n4 - rank_in(act)) & 255));
        if (e4 != 0) *(LAS unsigned char*)(row + t * 4 + (b >> 3)) = x << 2;
        e4 &= ~(1u << (b & 31));
        n4 += __builtin_popcountll(act);
    }
    const u32 SM[8] = {c2[0] & 0x83838383u, ((c2[0] << 1) & 0x80808080u) | ((c2[0] >> 2) & 0x03030303u),
                       c2[1] & 0x83838383u, ((c2[1] << 1) & 0x80808080u) | ((c2[1] >> 2) & 0x03030303u),
                       c2[2] & 0x83838383u, ((c2[2] << 1) & 0x80808080u) | ((c2[2] >> 2) & 0x03030303u),
                       ((c2[2] << 2) & 0x80808080u) | ((c2[0] >> 4) & 0x03030303u),
                       ((c2[2] << 3) & 0x80808080u) | ((c2[1] >> 4) & 0x03030303u)};
    u32 M[8], P[8], g[8], at = 0;
#pragma unroll
    for (int t = 0; t < 8; ++t) {
        M[t] = (X[t >> 2] >> (2 * (t & 3))) & 0x01010101u;
        P[t] = byte_scan(M[t]);
        const LAS u32* w = (const LAS u32*)(row + (at & ~3u));
        g[t] = __builtin_amdgcn_alignbyte(w[1], w[0], at);
        at += P[t] >> 24;
    }
#pragma unroll
    for (int t = 0; t < 8; ++t) {
        const u32 c = (K[t >> 2] >> (2 * (t & 3))) & 0x03030303u;
        const u32 m = times7(M[t]);
        const u32 H = SM[t] | __builtin_amdgcn_perm(t1, g[t], ((P[t] - M[t]) & m) | ((c | 0x04040404u) & ~m));
        const u32 L = t < 4 ? c0[t & 3] : c1[t & 3];
        h[t >> 1][(t & 1) * 2] = __builtin_amdgcn_perm(H, L, 0x05010400u);
        h[t >> 1][(t & 1) * 2 + 1] = __builtin_amdgcn_perm(H, L, 0x07030602u);
    }
}

__device__ __forceinline__ void out_windows(const u32x4 (&o)[4], u32& tp, u32& tn) {
    u32 mn = 0;
    int mp = 0;
#pragma unroll
    for (int i = 0; i < 16; ++i) { mp = max(mp, (int)o[i >> 2][i & 3]); mn = umax2(mn, o[i >> 2][i & 3]); }
    tp = wave_max((u32)mp) >> 24;
    tp -= __builtin_popcountll(__builtin_amdgcn_ballot_w64(mp >= (int)(tp << 24))) <= 6;
    tp = (int)tp > 3 ? tp : 3;
    tn = wave_max(mn) >> 24;
    tn = tn > 130 ? tn : 130;
}
__device__ __forceinline__ void pack_out(const u32x4 (&o)[4], u32 tp, u32 tn, LAS char* scr, rsrc_t r0, u32 dst, int lane) {
    u32 M[12], G2 = 0, G1 = 0, em = 0;
    u32x4 E;
#pragma unroll
    for (int r = 0; r < 4; ++r) {
#pragma unroll
        for (int i = 0; i < 3; ++i) M[r * 3 + i] = __builtin_amdgcn_perm(o[r][3], o[r][i], 0x04020100u + (i << 24));
        E[r] = __builtin_amdgcn_perm(o[r][1], o[r][0], 0x0c0c0703u) | __builtin_amdgcn_perm(o[r][3], o[r][2], 0x07030c0cu);
    }
    *(LAS u32x4*)(scr + lane * 16) = E;
#pragma unroll
    for (int r = 0; r < 4; ++r) {
        const u32 sg = E[r] & 0x80808080u, sm = sg - (sg >> 7);
        const u32 top = ((tn & 127) * 0x01010101u & sm) | (tp * 0x01010101u & ~sm);
        const u32 d = ((top | 0x80808080u) - (E[r] & 0x7f7f7f7fu)) ^ 0x80808080u;
        const u32 out = (d | ((d & 0x7f7f7f7fu) + 0x7c7c7c7cu + (sg >> 7))) & 0x80808080u;
        const u32 g = ((d & 0x03030303u) + 0x01010101u + (sg >> 5)) & ~((out >> 4) - (out >> 7));
        G2 |= (g & 0x03030303u) << (2 * r);
        G1 |= (g & 0x04040404u) >> 2 << r;
        em = (em >> 1) | out;
    }
    LAS unsigned char* list = (LAS unsigned char*)(scr + 1024);
    u32 used = 2;
    for (u64 act = __builtin_amdgcn_ballot_w64(em != 0); act != 0; act = __builtin_amdgcn_ballot_w64(em != 0)) {
        const u32 k = used + rank_in(act), b = __builtin_ctz(em | 0x80000000u);
        list[em != 0 && k < 64 ? k : 64] = *(const LAS unsigned char*)(scr + lane * 16 + (b & 3) * 4 + (b >> 3));
        em &= em - 1;
        used += __builtin_popcountll(act);
        if (used > 64) break;
    }
    *(LAS unsigned short*)list = (used > 64 ? 255 : tp) | tn << 8;
    asm volatile("" ::: "memory");
    const u32 ent = *(const LAS u32*)(list + (lane & 15) * 4);
    if (used > 64) {
#pragma unroll
        for (int r = 0; r < 4; ++r) st_sys(o[r], r0, lane * 16 + r * 1024, dst + STG_BYTES);
    }
#pragma unroll
    for (int i = 0; i < 3; ++i) st_sys((u32x4){M[4 * i], M[4 * i + 1], M[4 * i + 2], M[4 * i + 3]}, r0, lane * 16 + i * 1024, dst);
    __builtin_amdgcn_raw_buffer_store_b32(G2, r0, lane * 4 + dst + 3072, 0, 17);
    __builtin_amdgcn_raw_buffer_store_b16((unsigned short)__builtin_amdgcn_perm(0u, G1 | (G1 >> 4), 0x0c0c0200u), r0, lane * 2 + dst + 3328, 0, 17);
    __builtin_amdgcn_raw_buffer_store_b32(ent, r0, (lane < 16 ? lane * 4 : 0x80000000u) + dst + 3456, 0, 17);
}
__device__ __forceinline__ void unpack_out(u32x4 (&o)[4], const u32x4& c0, const u32x4& c1, const u32x4& c2, u32 G2, u32 g1,
                                           const LAS unsigned char* list, u32 tp, u32 tn, LAS char* scr, int lane) {
    const u32 M[12] = {c0[0], c0[1], c0[2], c0[3], c1[0], c1[1], c1[2], c1[3], c2[0], c2[1], c2[2], c2[3]};
    const u32 t = __builtin_amdgcn_perm(0u, g1, 0x0c010c00u), G1 = t | (t << 4);
    const u32 tlo = tp * 0x01010100u - 0x02010000u, thi = tn * 0x01010100u - 0x02010000u + tp - 3;
    u32x4 E;
    u32 em = 0;
#pragma unroll
    for (int r = 0; r < 4; ++r) {
        const u32 g = ((G2 >> (2 * r)) & 0x03030303u) | ((G1 >> r << 2) & 0x04040404u);
        E[r] = __builtin_amdgcn_perm(thi, tlo, g);
        em = (em >> 1) | (~(g + 0x7f7f7f7fu) & 0x80808080u);
    }
    *(LAS u32x4*)(scr + lane * 16) = E;
    u32 used = 2;
    auto step = [&](u32& a, u32& val) {
        const u64 act = __builtin_amdgcn_ballot_w64(em != 0);
        const u32 b = __builtin_ctz(em | 0x80000000u);
        a = em != 0 ? lane * 16 + (b & 3) * 4 + (b >> 3) : 1024;
        val = list[(used + rank_in(act)) & 63];
        em &= em - 1;
        used += __builtin_popcountll(act);
    };
    do {
        u32 a0, a1, a2, x0, x1, x2;
        step(a0, x0); step(a1, x1); step(a2, x2);
        *(LAS unsigned char*)(scr + a0) = x0; *(LAS unsigned char*)(scr + a1) = x1; *(LAS unsigned char*)(scr + a2) = x2;
    } while (__builtin_amdgcn_ballot_w64(em != 0) != 0);
    E = *(const LAS u32x4*)(scr + lane * 16);
#pragma unroll
    for (int r = 0; r < 4; ++r) {
#pragma unroll
        for (int i = 0; i < 3; ++i) o[r][i] = __builtin_amdgcn_perm(E[r], M[r * 3 + i], 0x04020100u + (i << 24));
        o[r][3] = __builtin_amdgcn_perm(M[r * 3 + 1], M[r * 3], 0x0c0c0703u) | __builtin_amdgcn_perm(E[r], M[r * 3 + 2], 0x07030c0cu);
    }
}

__device__ __forceinline__ void load_item(f32x4 (&v)[8], const Params& p, int u, int wid, int lane) {
    const int blk = u >> 4, it = (u & 15) * 8 + wid;
    const bool isx = it < 64;
    const rsrc_t r = mk_rsrc(isx ? p.x : p.w, OUT_BYTES);
    const u32 vf = isx ? (lane & 15) * 32768 + (lane >> 4) * 16 : (lane >> 4) * 16384 + (lane & 15) * 16;
    const u32 base = isx ? it * 524288 + blk * 512 : (blk * 128 + (wid & 3) * 32) * 4096 + ((it - 64) >> 2) * 256;
    const u32 sa = isx ? 64 : 4096, sb = isx ? 256 : 65536;
#pragma unroll
    for (int i = 0; i < 8; ++i)
        v[i] = __builtin_bit_cast(f32x4, __builtin_amdgcn_raw_buffer_load_b128(r, vf, base + (i & 3) * sa + (i >> 2) * sb, IN_AUX));
}
__device__ __forceinline__ void convert_local(const Params& p, LAS char* lds, f32x4 (&v)[8], char* wsl, int wg, int wid, int lane, u32 call) {
    const int rank = rfl(p.rank);
    constexpr int NU = NBLK * 16;
    const u32 par = OFF_PAR + (call & 1) * PAR_BYTES;
    const rsrc_t r0 = mk_rsrc(p.ws[rank], WS_NEED);
    LAS char* scr = lds + wid * 4096;
    auto publish = [&](int u) {
        if (wid == 0) {
            if (lane < 8 && lane != rank) st_flag(wsl + OFF_RDY + rank * 8192 + u * 4, call);
            if (lane == 8) add_sys(p.ws[rank] + OFF_CNT + (u >> 4) * CNT_BLK + ((u & 8) ? 1024 : 0) + rank * 128);
        }
    };
    int u = wg, un = (wg + NGEMM) ^ 8, up = -1;
    WAIT_VM0();
    for (int r = 0; u < NU; ++r) {
        const int blk = u >> 4, grp = u & 15, it = grp * 8 + wid;
        u32x4 h[4];
        u32 base, step;
        if (grp < 8) {
            base = par + (u32)blk * (RS1 * 4096) + it * 4096; step = 1024;
#pragma unroll
            for (int i = 0; i < 4; ++i) h[i] = cvt8(v[2 * i], v[2 * i + 1]);
        } else {
            base = par + REL_B + (u32)blk * (CF * 4096) + ((it - 64) & 60) * 4096 + (wid & 3) * 1024; step = 4096;
#pragma unroll
            for (int c = 0; c < 4; ++c) h[c] = cvt_w(v, c);
        }
        float amax = 0.0f;
#pragma unroll
        for (int i = 0; i < 8; ++i) amax = fmaxf(fmaxf(fmaxf(amax, fabsf(v[i][0])), fmaxf(fabsf(v[i][1]), fabsf(v[i][2]))), fabsf(v[i][3]));
        STAMPB_IF(r == 0, 0);
        if (r > 0) load_item(v, p, (u32)un < (u32)NU ? un : NU - 1, wid, lane);
        pack_item(h, amax, scr, r0, par + REL_PK + (u32)(blk * 128 + it) * 4096, lane, [&] {
            WAIT_VM0();
            if (r > 0) {
                __builtin_amdgcn_s_barrier();
                if (up >= 0) publish(up);
            }
#pragma unroll
            for (int i = 0; i < 4; ++i) st_sys(h[i], r0, lane * 16, base + i * step);
        });
        up = u;
        if (r == 0) {
            STAMPB_IF(true, 1);
            WAIT_VM0();
            STAMPB_IF(true, 2);
            __builtin_amdgcn_s_barrier();
            STAMP(6);
            publish(u);
            up = -1;
            load_item(v, p, un, wid, lane);
            WAIT_VM0();
        }
        u = un;
        un = (un + NGEMM) ^ 8;
    }
    WAIT_VM0();
    __builtin_amdgcn_s_barrier();
    if (up >= 0) publish(up);
}

constexpr int PUSH_PER = FI1MAX + 64 + XA, NPUSH = PB * PUSH_PER, NPW = (NPUSH + NCOMM - 1) / NCOMM;
__device__ __forceinline__ void push_decode(int n, int& blk, int& it) {
    blk = n / PUSH_PER; it = n - blk * PUSH_PER;
    if (it >= FI1MAX + 64) { const int xi = blk * XA + it - FI1MAX - 64; blk = xi / FX; it = FI + xi - blk * FX; }
    else if (it >= FI1MAX) it += 64 - FI1MAX;
}
__device__ __forceinline__ void push_role(const Params& p, char* wsl, int n, int lane, u32 call) {
    const int rank = rfl(p.rank);
    const int g = rank >> 2, j = rank & 3, og = (g ^ 1) * 4;
    int blk, it;
    push_decode(n, blk, it);
    f32x4 v[8];
    load_item(v, p, blk * 16 + (it >> 3), it & 7, lane);
    const u32 par = OFF_PAR + (call & 1) * PAR_BYTES;
    u32x4 h[4];
    int d0 = 0, nd = 4, dstep = 1; u32 base, step = 1024;
    if (it < 64) {
#pragma unroll
        for (int i = 0; i < 4; ++i) h[i] = cvt8(v[2 * i], v[2 * i + 1]);
        if (it < FI) { d0 = j + 3; nd = 3; dstep = 3; base = par + (blk * RS1 + 64 + it) * 4096; }
        else base = par + (blk * RS1 + 64 + NI1 + j * FX + it - FI) * 4096;
    } else {
#pragma unroll
        for (int c = 0; c < 4; ++c) h[c] = cvt_w(v, c);
        d0 = (it - 64) >> 4; nd = 1; step = 4096;
        base = par + REL_B + (blk * CF + 64 + 16 * j + (it & 12)) * 4096 + (it & 3) * 1024;
    }
    u32 dmask = 0;
    for (int d = 0; d < nd; ++d) {
        if (it < FI && it >= FI1(d)) continue;
        const int dr = (it < FI ? g * 4 : og) + ((d0 + d * dstep) & 3);
        const rsrc_t rd = mk_rsrc(p.ws[dr], WS_NEED);
#pragma unroll
        for (int i = 0; i < 4; ++i) st_sys(h[i], rd, lane * 16, base + (it < FI ? OFF1(d) * 4096 : 0) + i * step);
        dmask |= 1u << dr;
    }
    __builtin_amdgcn_s_barrier();
    WAIT_VM0();
    STAMPB_IF(true, 3);
    if (lane < 8 && ((dmask >> lane) & 1)) add_sys(wsl + OFF_CNT + blk * CNT_BLK + (it < 64 ? 0 : 1024) + rank * 128);
}

#define P_NDEC 9
#define P_DSLEEP 2
constexpr int SCR = 2560;
constexpr int NLW = 3, RING = 6, QLEN = 32, NDEC = P_NDEC;
constexpr int LDS_CTL = NLW * RING * 4096;
constexpr int LDS_DONE = LDS_CTL + QLEN * 16, LDS_TAIL = LDS_DONE + NLW * RING * 4, LDS_SCR = LDS_CTL + 1024;
static_assert(7 * WL <= NLW * NCOMM, "loaders");
constexpr u32 IDX_END = 0xffffffffu, IDX_BAD = 0x80000000u;
__device__ __forceinline__ u32 lds_get(const LAS char* a) {
    u32 v;
    asm volatile("ds_read_b32 %0, %1\n\ts_waitcnt lgkmcnt(0)" : "=v"(v) : "v"((u32)(size_t)a) : "memory");
    return rfl(v);
}
__device__ __forceinline__ void lds_put(LAS char* a, u32 v) {
    asm volatile("" ::: "memory");
    *(volatile LAS u32*)a = v;
    asm volatile("" ::: "memory");
}

struct Item { u32 idx, dst; };
constexpr int NT_I = NBLK * FI;
constexpr int PER = 16 + XA, NP = NBLK * PER, NX = NBLK * (16 + FX), NO1 = (RF1 - FX) * 4, NT_C = NX + NO1 + 192;
constexpr int TAB_BYTES = (NT_I > NT_C ? NT_I : NT_C) * 4, LDS_TAB = LDS_SCR + NDEC * SCR;
static_assert(LDS_TAB + NLW * TAB_BYTES <= LDS_BYTES, "communication LDS");
__device__ __forceinline__ void build_table(LAS char* tab, int rank, int slot, int first, int stride) {
    const int g = rank >> 2, j = rank & 3, q = slot - 3;
    for (int it = first; it < (slot < 3 ? NT_I : NT_C); it += stride) {
        Item m;
        if (slot < 3) {
            const int f1 = FI1(slot), f2 = F2(slot);
            const bool s2 = it >= NBLK * f1;
            const int a = s2 ? it - NBLK * f1 : it, blk = a / (s2 ? f2 : f1), s = a - blk * (s2 ? f2 : f1) + (s2 ? f1 : 0);
            m.idx = blk * 128 + s;
            m.dst = (s2 ? REL_A2 + (OFF2(slot) - f1) * 4096 : (64 + OFF1(slot)) * 4096) + (blk * (s2 ? RF2 : RS1) + s) * 4096;
        } else if (it < NX) {
            const int blk = it / PER, c = it - blk * PER;
            const bool tail = it >= NP, isx = tail || (c < 2 * XA && (c & 1));
            const int cw = c < 2 * XA ? c >> 1 : c - XA;
            const int xi = tail ? it - NP + NBLK * XA : blk * XA + (c >> 1), xb = xi / FX, xs = FI + xi - xb * FX;
            m.idx = isx ? xb * 128 + xs : blk * 128 + 64 + 16 * j + cw;
            m.dst = isx ? (xb * RS1 + xs + 64 + NI1 + q * FX - FI) * 4096
                        : REL_B + (blk * CF + 64 + 16 * q + (cw & 12)) * 4096 + (cw & 3) * 1024;
        } else {
            const int k = it - NX;
            int row, col;
            if (k < NO1) {
                const int f = k >> 2;
                m.idx = 8192 + (f >> 4) * 64 + (k & 2) * 16 + ((f >> 2) & 3) * 8 + (k & 1) * 4 + (f & 3);
                row = row_of_slot(slot_of(f, j), g ^ 1, q);
                col = (k & 3) * 64;
            } else {
                const int z = (k - NO1 + RF2 * 4) % 192;
                m.idx = 8192 + 768 + k - NO1;
                row = row_of_f2((z >> 5) * 8 + (z & 7), g ^ 1, q);
                col = ((z >> 3) & 3) * 64;
            }
            m.dst = (u32)row * 4096 + (q * 256 + col) * 4;
        }
        *(LAS u32*)(tab + it * 4) = m.idx | m.dst << (m.idx < 8192 ? 4 : 6);
    }
}

template <bool CROSS>
__device__ __forceinline__ void load_role(const Params& p, LAS char* lds, int ld, int nld, int slot, int wv, int lane, u32 call) {
    const int rank = rfl(p.rank);
    const int g = rank >> 2, j = rank & 3;
    const int peer = CROSS ? (g ^ 1) * 4 + slot - 3 : g * 4 + ((j + 1 + slot) & 3);
    constexpr int NT = CROSS ? NT_C : NT_I;
    const u32 par = OFF_PAR + (call & 1) * PAR_BYTES;
    char* ws = uni(p.ws[rank]);
    const rsrc_t r0 = rsrc_u(ws, WS_NEED), rp = mk_rsrc(p.ws[peer], WS_NEED);
    const char* rdy = ws + OFF_RDY + peer * 8192;
    const u32 vo = lane * 16;
    const int SKIP = PB * (CROSS ? PER : FI1(slot));
    const u32 head = lane == 0 ? OFF_QUE + slot * 256 : WS_NEED, tbase = SKIP + 4 * WL - (call - 1) * (NT - SKIP);
    const bool nowait = (p.pad & 4) != 0;
    const LAS char* tab = lds + LDS_TAB + ld * TAB_BYTES;
    const LAS char* done = lds + LDS_DONE + ld * RING * 4;
    const u32 ldsb = rfl((u32)(size_t)lds) + ld * RING * 4096;
    auto table = [&](int it) {
        asm volatile("ds_read_b32 v160, %0" :: "v"((u32)(size_t)tab + (it < NT ? it : NT - 1) * 4) : "memory", "v160");
    };
    auto entry = [&] {
        u32 t;
        asm volatile("s_waitcnt lgkmcnt(0)\n\tv_readfirstlane_b32 %0, v160" : "=s"(t) :: "memory");
        Item m;
        m.idx = t & 16383;
        m.dst = (t - m.idx) >> (CROSS && m.idx >= 8192 ? 6 : 4);
        return m;
    };
    auto decode = [&](int it) { table(it); return entry(); };
    auto flag_of = [&](u32 idx) { return idx < 8192 ? idx >> 3 : (idx < 8192 + 768 ? 1024 + ((idx - 8192) >> 2) : idx - (8192 + 768 - 1024 - 192)); };
    u32 okupto = 0, wbase = 0; u64 wmask = 0;
    auto seen = [&](u32 fi) { return fi < okupto || (fi - wbase < 64u && ((wmask >> (fi - wbase)) & 1)); };
    auto ready = [&](u32 fi) {
        if (seen(fi)) return true;
        for (;;) {
            wbase = okupto;
            wmask = __builtin_amdgcn_ballot_w64(ld_sys(rdy + (wbase + lane) * 4) >= call);
            if (wmask != ~0ull) break;
            okupto += 64;
            if (fi < okupto) return true;
        }
        okupto += __builtin_ctzll(~wmask);
        if (fi - wbase < 64u) return ((wmask >> (fi - wbase)) & 1) != 0;
        return rfl(ld_sys(rdy + fi * 4)) >= call;
    };
    const u32 vl = vo < 256 + VAR_X ? vo : 0x80000000u;
    struct Prep { u32 src, m0v; };
    auto prep = [&](const Item& m, u32 n) {
        Prep r;
        r.src = par + REL_PK + (m.idx + (CROSS && m.idx >= 8192 ? j * 1024 : 0)) * 4096;
        r.m0v = ldsb + (n % RING) * 4096;
        asm volatile("" : "+s"(r.src), "+s"(r.m0v));
        return r;
    };
    auto issue = [&](const Prep& r) {
        asm volatile("s_mov_b32 m0, %0\n\ts_nop 0\n\t"
                     "buffer_load_dwordx4 %1, %3, %4 offen sc0 sc1 lds\n\t"
                     "buffer_load_dwordx4 %1, %3, %4 offen offset:1024 sc0 sc1 lds\n\t"
                     "buffer_load_dwordx4 %1, %3, %4 offen offset:2048 sc0 sc1 lds\n\t"
                     "buffer_load_dwordx4 %2, %3, %4 offen offset:3072 sc0 sc1 lds"
                     :: "s"(r.m0v), "v"(vo), "v"(vl), "s"(rp), "s"(r.src) : "memory");
    };
    u32 vtail = (u32)(size_t)lds + LDS_TAIL, vone = 1;
    asm volatile("" : "+v"(vtail), "+v"(vone));
    auto enqueue = [&](u32 idx, u32 dst, u32 n) {
        asm volatile("s_mov_b64 exec, 1\n\tds_add_rtn_u32 v162, %0, %1\n\tv_mov_b32 v159, %2\n\tv_mov_b32 v160, %3\n\tv_mov_b32 v161, %4\n\t"
                     "s_waitcnt lgkmcnt(0)\n\tv_add_u32 v158, 1, v162\n\tv_and_b32 v162, %6, v162\n\tv_lshl_add_u32 v162, v162, 4, %5\n\t"
                     "ds_write_b128 v162, v[158:161]\n\ts_mov_b64 exec, -1"
                     :: "v"(vtail), "v"(vone), "s"(idx), "s"(dst), "s"(n << 8 | ld << 4 | peer), "s"(rfl((u32)(size_t)lds) + LDS_CTL), "n"(QLEN - 1)
                     : "memory", "v158", "v159", "v160", "v161", "v162");
    };
    __builtin_amdgcn_s_setprio(3);
    int e = SKIP + wv, e1 = e + WL, e2 = e + 2 * WL;
    const int e3 = e + 3 * WL;
    Item nx = decode(e1), ia = decode(e);
    bool bad = spin_ge<2>(rdy + flag_of(ia.idx) * 4, call, ws + OFF_TMO, call);
    STAMP(5);
    issue(prep(ia, 0));
    bool pre = ready(flag_of(nx.idx));
    u32 n = 0;
    auto wait_free = [&](u32 m) { while (lds_get(done + (m % RING) * 4) + RING <= m) __builtin_amdgcn_s_sleep(1); };
    auto body = [&](auto odd) {
        if constexpr (odd.value) asm volatile("v_mov_b32 v164, 1\n\ts_nop 0\n\tbuffer_atomic_add v164, %0, %1, 0 offen sc0" :: "v"(head), "s"(r0) : "memory", "v164");
        else                     asm volatile("v_mov_b32 v165, 1\n\ts_nop 0\n\tbuffer_atomic_add v165, %0, %1, 0 offen sc0" :: "v"(head), "s"(r0) : "memory", "v165");
        table(e2);
        u32 dn;
        asm volatile("ds_read_b32 v162, %1\n\ts_waitcnt lgkmcnt(0)\n\tv_readfirstlane_b32 %0, v162"
                     : "=s"(dn) : "v"((u32)(size_t)done + ((n + 1) % RING) * 4) : "memory", "v162");
        const Item nn = entry();
        bool pre2 = e2 < NT;
        if (pre2 && !(nowait && nn.idx >= 8192)) pre2 = ready(flag_of(nn.idx));
        if (!pre && e1 < NT) pre = seen(flag_of(nx.idx));
        if (dn + RING <= n + 1) wait_free(n + 1);
        const Prep px = prep(nx, n + 1);
        u32 told;
        if constexpr (odd.value) asm volatile("s_waitcnt vmcnt(1)\n\tv_readfirstlane_b32 %0, v165" : "=s"(told) :: "memory");
        else                     asm volatile("s_waitcnt vmcnt(1)\n\tv_readfirstlane_b32 %0, v164" : "=s"(told) :: "memory");
        if (pre) issue(px);
        enqueue(ia.idx | (bad ? IDX_BAD : 0), ia.dst, n);
        if (!pre && e1 < NT) {
            bad |= spin_ge<2>(rdy + flag_of(nx.idx) * 4, call, ws + OFF_TMO, call);
            issue(px);
        }
        STAMP_IF(n == 0, 6);
        STAMP_IF(CROSS ? (e < NP && e1 >= NP) : (e < NBLK * FI1(slot) && e1 >= NBLK * FI1(slot)), 1);
        STAMP_IF(CROSS && e < NX && e1 >= NX, 2);
        STAMP_IF(CROSS && e < NX + NO1 && e1 >= NX + NO1, 3);
        ia = nx; nx = nn; pre = pre2; ++n;
        e = e1; e1 = e2; e2 = n == 1 ? e3 : (int)(told + tbase);
    };
    for (;;) {
        body(std::false_type{});
        if (e >= NT) break;
        body(std::true_type{});
        if (e >= NT) break;
    }
    u32 fin;
    asm volatile("ds_add_rtn_u32 v162, %1, %2\n\ts_waitcnt lgkmcnt(0)\n\tv_readfirstlane_b32 %0, v162"
                 : "=s"(fin) : "v"((u32)(size_t)lds + LDS_TAIL + 4), "v"(lane == 0 ? 1u : 0u) : "memory", "v162");
    if (fin + 1 == (u32)nld)
        for (int k = 0; k < NDEC; ++k) enqueue(IDX_END, 0, 0);
}

__device__ __forceinline__ void decode_role(const Params& p, LAS char* lds, u32 me, int lane, u32 call) {
    const int rank = rfl(p.rank);
    const int j = rank & 3;
    const u32 par = OFF_PAR + (call & 1) * PAR_BYTES;
    const rsrc_t r0 = mk_rsrc(p.ws[rank], WS_NEED), ro = mk_rsrc(p.out[rank], OUT_BYTES);
    const u32 vo = lane * 16, vout = (lane >> 4) * 4 * 4096 + (lane & 15) * 16;
    LAS char* scr = lds + LDS_SCR + me * SCR;
    u32 q0, q1, q2, q3;
    auto peek = [&](u32 n) {
        asm volatile("ds_read_b128 v[158:161], %4\n\ts_waitcnt lgkmcnt(0)\n\tv_readfirstlane_b32 %0, v158\n\tv_readfirstlane_b32 %1, v159\n\t"
                     "v_readfirstlane_b32 %2, v160\n\tv_readfirstlane_b32 %3, v161"
                     : "=s"(q0), "=s"(q1), "=s"(q2), "=s"(q3) : "v"((u32)(size_t)lds + LDS_CTL + (n % QLEN) * 16) : "memory", "v158", "v159", "v160", "v161");
        return q0 == n + 1;
    };
    const u32 vsig = lane == 0 ? 0u : 0x80000000u;
    u32 one = 1;
    asm volatile("" : "+v"(one));
    auto signal = [&](u32 c) {
        asm volatile("s_nop 4\n\tbuffer_atomic_add %0, %1, %2, %3 offen sc1" :: "v"(one), "v"(vsig), "s"(r0), "s"(rfl(c)) : "memory");
    };
    const u32x4 nan4 = {0x7fc07fc0u, 0x7fc07fc0u, 0x7fc07fc0u, 0x7fc07fc0u};
    u32 vhead = (u32)(size_t)lds + LDS_TAIL + 8, vone = 1;
    asm volatile("" : "+v"(vhead), "+v"(vone));
    for (;;) {
        u32 n;
        asm volatile("s_mov_b64 exec, 1\n\tds_add_rtn_u32 v158, %1, %2\n\ts_waitcnt lgkmcnt(0)\n\tv_readfirstlane_b32 %0, v158\n\ts_mov_b64 exec, -1"
                     : "=s"(n) : "v"(vhead), "v"(vone) : "memory", "v158");
        while (!peek(n)) __builtin_amdgcn_s_sleep(P_DSLEEP);
        if (q1 == IDX_END) break;
        const u32 dst = q2, item = q3 >> 8, ring = (((q3 >> 4) & 3) * RING + item % RING), peer = q3 & 7;
        const LAS char* sl = lds + ring * 4096;
        const u32x4 c0 = *(const LAS u32x4*)(sl + vo), c1 = *(const LAS u32x4*)(sl + 1024 + vo), c2 = *(const LAS u32x4*)(sl + 2048 + vo);
        const u32 k1 = *(const LAS u32*)(sl + 3072 + lane * 4);
        const u32 hx = *(const LAS unsigned char*)(sl + 3328);
        const bool bad = (q1 & IDX_BAD) != 0;
        const u32 idx = q1 & 0xffff;
        const bool out = idx >= 8192, isw = !out && (idx & 64);
        const u32 step = (out || isw) ? 4096 : 1024;
        u32x4 h[4];
        bool done = false;
        if (!bad && !out) {
            const u32 hd = rfl(hx);
            if (hd != 255) {
                if (hd & 64) {
                    u32 t;
                    asm volatile("s_nop 4\n\tbuffer_load_dword %0, %1, %2, %3 offen sc0 sc1\n\ts_waitcnt vmcnt(0)"
                                 : "=&v"(t) : "v"(lane < 16 ? lane * 4 : 0x80000000u), "s"(mk_rsrc(p.ws[peer], WS_NEED)),
                                   "s"(rfl(par + REL_PK + idx * 4096 + 3328 + VAR_X)) : "memory");
                    if (lane < 16) *(LAS u32*)(lds + ring * 4096 + 3328 + VAR_X + lane * 4) = t;
                    asm volatile("" ::: "memory");
                }
                unpack_item(h, c0, c1, c2, k1, sl + 3328, hd, scr, lane);
                done = true;
            }
        } else if (!bad) {
            const u32 g1 = *(const LAS unsigned short*)(sl + 3328 + lane * 2), hd = rfl(*(const LAS unsigned short*)(sl + 3456));
            if ((hd & 255) != 255) { unpack_out(h, c0, c1, c2, k1, g1, (const LAS unsigned char*)(sl + 3456), hd & 255, hd >> 8, scr, lane); done = true; }
        }
        if (!done) {
            const u32 blk = idx >> 7, s = idx & 127;
            const u32 raw = par + (out ? REL_PK + STG_BYTES + (idx + j * 1024) * 4096
                                       : isw ? REL_B + (blk * CF + (s & 60)) * 4096 + (s & 3) * 1024 : (blk * RS1 + s) * 4096);
            const u32 rs = isw ? 4096 : 1024;
            const rsrc_t rp = mk_rsrc(p.ws[peer], WS_NEED);
            asm volatile("s_nop 4\n\tbuffer_load_dwordx4 %0, %4, %5, %6 offen sc0 sc1\n\tbuffer_load_dwordx4 %1, %4, %5, %7 offen sc0 sc1\n\t"
                         "buffer_load_dwordx4 %2, %4, %5, %8 offen sc0 sc1\n\tbuffer_load_dwordx4 %3, %4, %5, %9 offen sc0 sc1\n\ts_waitcnt vmcnt(0)"
                         : "=&v"(h[0]), "=&v"(h[1]), "=&v"(h[2]), "=&v"(h[3])
                         : "v"(vo), "s"(rp), "s"(rfl(raw)), "s"(rfl(raw + rs)), "s"(rfl(raw + 2 * rs)), "s"(rfl(raw + 3 * rs)) : "memory");
            if (bad) { h[0] = nan4; h[1] = nan4; h[2] = nan4; h[3] = nan4; }
        }
        lds_put(lds + LDS_DONE + ring * 4, item + 1);
        const rsrc_t rd = out ? ro : r0;
        const u32 vd = (out ? vout : vo + par) + dst;
#pragma unroll
        for (int i = 0; i < 4; ++i) st_sys(h[i], rd, vd, i * step);
        if (!out) {
            asm volatile("s_waitcnt vmcnt(0)" ::: "memory");
            signal(OFF_CNT + (idx >> 7) * CNT_BLK + peer * 128 + (isw ? 1024 : dst >= REL_A2 ? 2048 : 0));
        }
    }
    STAMP(4);
}

#define GLDS2(m0v, voff, rs, soff, imm0, imm1) \
    asm volatile("s_mov_b32 m0, %0\n\ts_nop 0\n\tbuffer_load_dwordx4 %1, %2, %3 offen offset:" #imm0 " sc0 sc1 lds\n\t" \
                 "buffer_load_dwordx4 %1, %2, %3 offen offset:" #imm1 " sc0 sc1 lds" \
                 :: "s"(m0v), "v"(voff), "s"(rs), "s"(soff) : "memory")
#define BARRIER() do { __builtin_amdgcn_sched_barrier(0); __builtin_amdgcn_s_barrier(); __builtin_amdgcn_sched_barrier(0); } while (0)

#define SCHED() __builtin_amdgcn_sched_barrier(0)
#define WAIT_LGKM0() __builtin_amdgcn_s_waitcnt(0xC07F)

template <int STAGE>
struct Gemm {
    static constexpr int NT = STAGE == 1 ? 128 : 64;
    static constexpr int NA = STAGE == 1 ? 4 : 2;
    static constexpr int NB = STAGE == 1 ? 4 : 8;
    static constexpr int NM = STAGE == 1 ? 4 : 1;
    static constexpr int NMM = STAGE == 1 ? 16 : 8;
    static constexpr int NRD = NA + NB;
    struct Frags { f16x8 a[NA], b[NB]; };

    rsrc_t rs;
    u32 sa, sa2, sb;
    u32 ablk;
    u32 ldsw;
    int wid, lane;

    template <int I> __device__ __forceinline__ void piece(int t, int buf) const {
        const u32 vo = lane * 16;
        const u32 ma = ldsw + buf * BUF_BYTES + wid * 4096, mb = ldsw + buf * BUF_BYTES + 32768 + wid * 2048;
        const u32 ka = STAGE == 1 ? (u32)(t >> 1) * (RS1 * 4096) + (t & 1) * 2048 : (u32)t * ablk;
        const u32 a = sa + ka, a2 = sa2 + ka;
        const u32 b = STAGE == 1 ? sb + (u32)(t >> 1) * (CF * 4096) + (t & 1) * 2048 : sb + (u32)t * (CF * 4096);
        if (I == 0) GLDS2(ma, vo, rs, a, 0, 1024);
        if (I == 1) GLDS2(ma, vo, rs, a2, 2048, 3072);
        if (I == 2) GLDS2(mb, vo, rs, b, 0, 1024);
    }
    __device__ __forceinline__ void stage(int t, int buf) const { piece<0>(t, buf); piece<1>(t, buf); piece<2>(t, buf); }

    template <int I> __device__ __forceinline__ void rd(Frags& f, const LAS char* lds, int buf, int half) const {
        const LAS char* base = lds + buf * BUF_BYTES + lane * 16;
        if constexpr (STAGE == 1) {
            const LAS char* pa = base + (wid >> 1) * 8192 + half * 1024;
            const LAS char* pb = base + 32768 + (wid & 1) * 8192 + half * 1024;
            if constexpr (I == 0) f.a[0] = *(const LAS f16x8*)pa;
            else if constexpr (I <= 4) f.b[I - 1] = *(const LAS f16x8*)(pb + (I - 1) * 2048);
            else f.a[I - 4] = *(const LAS f16x8*)(pa + (I - 4) * 2048);
        } else {
            const LAS char* pa = base + wid * 4096 + half * 2048;
            const LAS char* pb = base + 32768 + half * 2048;
            constexpr int h = I / 5, n = I % 5;
            if constexpr (n == 0) f.a[h] = *(const LAS f16x8*)(pa + h * 1024);
            else f.b[h * 4 + n - 1] = *(const LAS f16x8*)(pb + (n - 1) * 4096 + h * 1024);
        }
    }
    template <int I> __device__ __forceinline__ void mm(f32x4 (&acc)[NM][4], const Frags& f) const {
        if constexpr (STAGE == 1) acc[I >> 2][I & 3] = __builtin_amdgcn_mfma_f32_16x16x32_f16(f.a[I >> 2], f.b[I & 3], acc[I >> 2][I & 3], 0, 0, 0);
        else            acc[0][I & 3] = __builtin_amdgcn_mfma_f32_16x16x32_f16(f.a[I >> 2], f.b[I], acc[0][I & 3], 0, 0, 0);
    }
    template <int I, int N> __device__ __forceinline__ void rds(Frags& f, const LAS char* lds, int buf, int half) const {
        if constexpr (I < N) { rd<I>(f, lds, buf, half); rds<I + 1, N>(f, lds, buf, half); }
    }
    static __device__ __forceinline__ int blk_of(int t) { return STAGE == 1 ? (t >> 1) : t; }

    template <bool RD, int ST>
    __device__ __forceinline__ void half_step(f32x4 (&acc)[NM][4], const Frags& cur, Frags& nxt, const LAS char* lds, int rbuf, int rhalf,
                                              int ts, int bs, u32 cvo, u32 fso) const {
        auto first = [&] {
            if (ST == 1) asm volatile("buffer_load_dword v166, %0, %1, %2 offen sc0 sc1" :: "v"(cvo), "s"(rs), "s"(fso) : "memory", "v166");
            if (ST) { piece<0>(ts, bs); SCHED(); }
        };
        if constexpr (STAGE == 1) {
            mm<0>(acc, cur);  mm<1>(acc, cur);  if (RD) rds<0, 2>(nxt, lds, rbuf, rhalf); SCHED();
            mm<2>(acc, cur);  mm<3>(acc, cur);  if (RD) rds<2, 4>(nxt, lds, rbuf, rhalf); SCHED();
            mm<4>(acc, cur);  mm<5>(acc, cur);  if (RD) rds<4, 6>(nxt, lds, rbuf, rhalf); SCHED();
            mm<6>(acc, cur);  mm<7>(acc, cur);  if (RD) rds<6, 8>(nxt, lds, rbuf, rhalf); SCHED();
            mm<8>(acc, cur);  mm<9>(acc, cur);  SCHED();
            first();
            mm<10>(acc, cur); mm<11>(acc, cur); SCHED();
            if (ST) { piece<1>(ts, bs); SCHED(); }
            mm<12>(acc, cur); mm<13>(acc, cur); SCHED();
            if (ST) { piece<2>(ts, bs); SCHED(); }
            mm<14>(acc, cur); mm<15>(acc, cur); SCHED();
        } else {
            first();
            mm<0>(acc, cur); SCHED();
            if (ST) { piece<1>(ts, bs); SCHED(); }
            mm<1>(acc, cur); SCHED();
            if (ST) { piece<2>(ts, bs); SCHED(); }
            mm<2>(acc, cur); if (RD) rds<0, 3>(nxt, lds, rbuf, rhalf); SCHED();
            mm<3>(acc, cur); if (RD) rds<3, 5>(nxt, lds, rbuf, rhalf); SCHED();
            mm<4>(acc, cur); if (RD) rds<5, 8>(nxt, lds, rbuf, rhalf); SCHED();
            mm<5>(acc, cur); if (RD) rds<8, 10>(nxt, lds, rbuf, rhalf); SCHED();
            mm<6>(acc, cur); mm<7>(acc, cur); SCHED();
        }
    }

    template <int ROLE>
    __device__ __forceinline__ u32 run(f32x4 (&acc)[NM][4], const LAS char* lds, char* ws, u32 cvo, u32 need, u32 call) const {
        constexpr bool LATE = ROLE == 2 && STAGE == 1;
        char* tmo = ws + OFF_TMO;
        Frags f0, f1;
        if (ROLE == 0) spin_ge(ws + cvo + blk_of(0) * CNT_BLK, need, tmo, call);
        BARRIER();
        stage(0, 0);
        if (ROLE == 0) { spin_ge(ws + cvo + blk_of(1) * CNT_BLK, need, tmo, call); spin_ge(ws + cvo + blk_of(2) * CNT_BLK, need, tmo, call); }
        BARRIER();
        stage(1, 1);
        if (!LATE) { stage(2, 2); asm volatile("s_waitcnt vmcnt(12)" ::: "memory"); }
        else asm volatile("s_waitcnt vmcnt(6)" ::: "memory");
        BARRIER();
        rds<0, NRD>(f0, lds, 0, 0);
        WAIT_LGKM0();
        if (ROLE == 0) asm volatile("v_mov_b32 v166, 0" ::: "v166");
        int buf = 0;
        for (int t = 0; t < NT - 3; ++t) {
            const int nb = buf == 2 ? 0 : buf + 1, pb = buf == 0 ? 2 : buf - 1;
            half_step<true, LATE ? 2 : 0>(acc, f0, f1, lds, buf, 1, t + 2, pb, 0, 0);
            if (ROLE == 0) {
                u64 miss;
                asm volatile("s_waitcnt vmcnt(6)\n\tv_cmp_lt_u32_e64 %0, v166, %1\n\ts_nop 0" : "=s"(miss) : "v"(need) : "memory");
                WAIT_LGKM0();
                if (miss != 0) spin_ge(ws + cvo + blk_of(t + 3) * CNT_BLK, need, tmo, call);
            } else {
                asm volatile("s_waitcnt vmcnt(6)" ::: "memory");
                WAIT_LGKM0();
            }
            BARRIER();
            half_step<true, LATE ? 0 : (ROLE == 0 ? 1 : 2)>(acc, f1, f0, lds, nb, 0, t + 3, buf, cvo, blk_of(t + 4 < NT ? t + 4 : t + 3) * CNT_BLK);
            buf = nb;
        }
#pragma unroll
        for (int t = NT - 3; t < NT; ++t) {
            const int nb = buf == 2 ? 0 : buf + 1, pb = buf == 0 ? 2 : buf - 1;
            if (t == NT - 3) {
                half_step<true, LATE ? 2 : 0>(acc, f0, f1, lds, buf, 1, t + 2, pb, 0, 0);
                asm volatile("s_waitcnt vmcnt(6)" ::: "memory");
            } else {
                if (t == NT - 2) {
                    const u32 to = OFF_TMO;
                    asm volatile("buffer_load_dword v167, %0, %1, 0 offen sc1" :: "v"(to), "s"(rs) : "memory", "v167");
                }
                half_step<true, 0>(acc, f0, f1, lds, buf, 1, 0, 0, 0, 0);
                asm volatile("s_waitcnt vmcnt(0)" ::: "memory");
            }
            WAIT_LGKM0();
            BARRIER();
            if (t < NT - 1) half_step<true, 0>(acc, f1, f0, lds, nb, 0, 0, 0, 0, 0);
            else            half_step<false, 0>(acc, f1, f0, lds, nb, 0, 0, 0, 0, 0);
            buf = nb;
        }
        u32 tw;
        asm volatile("v_readfirstlane_b32 %0, v167\n\ts_nop 1" : "=s"(tw) :: "memory");
        return tw;
    }
};

__device__ __forceinline__ void activate(u32x4 (&o)[4], f32x4 (&c)[4], bool bad) {
    if (bad) {
        asm volatile("");
#pragma unroll
        for (int n = 0; n < 4; ++n) c[n] = __builtin_bit_cast(f32x4, (u32x4){0x7fc00000u, 0x7fc00000u, 0x7fc00000u, 0x7fc00000u});
    }
#pragma unroll
    for (int n = 0; n < 4; ++n)
#pragma unroll
        for (int r = 0; r < 4; r += 2) {
            const u32x2v y = __builtin_bit_cast(u32x2v, gelu2((f32x2){c[n][r], c[n][r + 1]}));
            o[r][n] = y[0]; o[r + 1][n] = y[1];
        }
}

__device__ __forceinline__ void gemm_role(const Params& p, const LAS char* lds, int xcd, int s, int wid, int lane, u32 call) {
    const int rank = rfl(p.rank);
    const int g = rank >> 2, j = rank & 3, og = (g ^ 1) * 4;
    char* ws = p.ws[rank];
    const u32 par = OFF_PAR + (call & 1) * PAR_BYTES;
    const rsrc_t rs = mk_rsrc(ws, WS_NEED), ro = mk_rsrc(p.out[rank], OUT_BYTES);
    const u32 ldsw = rfl((u32)(size_t)lds);
    const u32 vout = (lane >> 4) * 4 * 4096 + (lane & 15) * 16;
    {
        const int u = (xcd & 1) * (xcd < 4 ? 26 : 24) + s;
        const bool extra = xcd >= 4 && s >= 24;
        const int rt = extra ? 13 : u >> 2, ct = extra ? (xcd - 4) * 2 + (s & 1) : (xcd >> 1) * 4 + (u & 3);
        const int cq = ct < 8 ? -1 : (ct - 8) >> 1;
        Gemm<1> G;
        G.rs = rs; G.ldsw = ldsw; G.wid = wid; G.lane = lane;
        G.sa = par + max(slot_of(rt * 16 + wid * 2, cq), 0) * 4096;
        G.sa2 = par + max(slot_of(rt * 16 + wid * 2 + 1, cq), 0) * 4096 - 2048;
        G.sb = par + REL_B + (ct * 8 + wid) * 4096;
        const int bsrc = ct < 8 ? rank : og + cq;
        u32 cvo = OFF_CNT + 1024 + bsrc * 128, need = (ct < 8 ? 8u : 16u) * call;
        const int sl16 = max(slot_of(rt * 16 + (lane & 15), cq), 0), src16 = row_of_slot(sl16, g, j) / NS;
        u32 mask = 0;
        for (int i = 0; i < 8; ++i) mask |= (__builtin_amdgcn_ballot_w64(src16 == i) != 0) << i;
        if (lane < 8 && ((mask >> lane) & 1)) {
            cvo = OFF_CNT + lane * 128;
            need = (lane == rank ? 8 : ((lane >> 2) == g ? FI1((lane - j - 1) & 3) : FX)) * call;
        }
        f32x4 acc[4][4] = {};
        const bool bad = call == (wid == 0 ? G.template run<0>(acc, lds, ws, cvo, need, call)
                                  : wid < 4 ? G.template run<1>(acc, lds, ws, cvo, need, call)
                                            : G.template run<2>(acc, lds, ws, cvo, need, call));
        STAMP(1);
        const int wm = wid >> 1, wn = wid & 1;
        if (ct < 8) {
#pragma unroll
            for (int m = 0; m < 4; ++m) {
                const int sl = slot_of(rt * 16 + wm * 4 + m, -1);
                if (sl < 0) continue;
                const u32 ob = (u32)row_of_slot(sl, g, j) * 4096 + (ct * 128 + wn * 64) * 4;
                u32x4 o[4];
                activate(o, acc[m], bad);
#pragma unroll
                for (int r = 0; r < 4; ++r) __builtin_amdgcn_raw_buffer_store_b128(o[r], ro, vout + ob + r * 4096, 0, OUT_AUX);
            }
        } else {
            const int fo = (rt * 2 + (ct & 1)) * 8 + wid;
            const u32 sb = par + REL_STG + (u32)(((ct - 8) >> 1) * 1024 + fo * 4) * 4096;
            u32 tp = 0, tn = 0;
#pragma unroll
            for (int m = 0; m < 4; ++m) {
                u32x4 o[4];
                activate(o, acc[m], bad);
                if (m == 0) out_windows(o, tp, tn);
                pack_out(o, tp, tn, (LAS char*)lds + wid * 2048, rs, sb + m * 4096, lane);
            }
            WAIT_VM0();
            if (lane == 0) st_flag(p.ws[bsrc] + OFF_RDY + rank * 8192 + (1024 + fo) * 4, call);
        }
        STAMP(2);
    }
    {
        const bool loc = s >= 24;
        const int rt = ((s >> 2) + 5) % 6, ct = loc ? xcd * 2 + (s & 1) : ((~xcd & 1) * 16) + (xcd >> 1) * 4 + (s & 3);
        Gemm<2> G;
        G.rs = rs; G.ldsw = ldsw; G.wid = wid; G.lane = lane;
        const bool mine = loc || rt * 8 >= RF2;
        G.sa = G.sa2 = par + (mine ? ((loc ? FI : FO1) + wid) * 4096 : REL_A2 + (rt * 8 + wid) * 4096);
        G.ablk = (mine ? RS1 : RF2) * 4096;
        G.sb = par + REL_B + ct * 4 * 4096 + wid * 2048;
        const int bsrc = ct < 16 ? rank : og + ((ct - 16) >> 2);
        u32 cvo = OFF_CNT + 1024 + bsrc * 128, need = (ct < 16 ? 8u : 16u) * call;
        if (mine) { if (lane == 0) { cvo = OFF_CNT + rank * 128; need = 8 * call; } }
        else if (lane < 3 && rt * 8 < OFF2(lane + 1) && rt * 8 + 8 > OFF2(lane)) {
            cvo = OFF_CNT + 2048 + (g * 4 + ((j + 1 + lane) & 3)) * 128; need = F2(lane) * call;
        }
        f32x4 acc[1][4] = {};
        const bool bad = call == (wid == 0 ? G.template run<0>(acc, lds, ws, cvo, need, call)
                                  : wid < 4 ? G.template run<1>(acc, lds, ws, cvo, need, call)
                                            : G.template run<2>(acc, lds, ws, cvo, need, call));
        STAMP(3);
        u32x4 o[4];
        activate(o, acc[0], bad);
        if (ct < 16) {
            const u32 ob = (u32)(loc ? rank * NS + (FI + wid) * 16 : row_of_f2(rt * 8 + wid, g, j)) * 4096 + ct * 256;
#pragma unroll
            for (int r = 0; r < 4; ++r) __builtin_amdgcn_raw_buffer_store_b128(o[r], ro, vout + ob + r * 4096, 0, OUT_AUX);
        } else {
            const int z = ((rt * 4 + (ct & 3)) * 8 + wid + FO2 * 4) % 192;
            const u32 sb = par + REL_STG + (u32)(((ct - 16) >> 2) * 1024 + 768 + z) * 4096;
            u32 tp, tn;
            out_windows(o, tp, tn);
            pack_out(o, tp, tn, (LAS char*)lds + wid * 2048, rs, sb, lane);
            WAIT_VM0();
            if (lane == 0) st_flag(p.ws[bsrc] + OFF_RDY + rank * 8192 + (1024 + 192 + z) * 4, call);
        }
        STAMP(4);
    }
}

__global__ __launch_bounds__(BLOCK, 3) void ag_gemm_kernel(Params p) {
    __shared__ __attribute__((aligned(1024))) char smem[LDS_BYTES];
    const int rank = rfl(p.rank);
    const int lane = threadIdx.x & 63;
    const int wid = rfl(threadIdx.x >> 6);
    const int b = blockIdx.x;
    if (wid >= (b < NCOMM ? NLW + NDEC : 8)) return;
    STAMP(0);
    char* seq = p.ws[rank] + OFF_SEQ + (b * 16 + wid) * 4;
    const u32 prev = ld_agent(seq);
    char* wsl = p.ws[lane & 7];
    auto next_call = [&] {
        const u32 call = rfl(prev) + 1;
        if (lane == 0) __hip_atomic_store((GAS u32*)seq, call, __ATOMIC_RELAXED, __HIP_MEMORY_SCOPE_AGENT);
        STAMP(7);
        return call;
    };
    if (b >= NCOMM) {
        for (int i = ((b - NCOMM) >> 4) - 1; i > 0; --i) __builtin_amdgcn_s_sleep(P_GRADE);
        f32x4 v[8];
        load_item(v, p, b - NCOMM, wid, lane);
        const u32 call = next_call();
        convert_local(p, (LAS char*)smem, v, wsl, b - NCOMM, wid, lane, call);
        STAMP(5);
        gemm_role(p, (const LAS char*)smem, b & 7, (b - NCOMM) >> 3, wid, lane, call);
        return;
    }
    const int nld = 2 * NCOMM + b < 7 * WL ? 3 : 2;
    constexpr int NSET = NLW + NDEC - NPW;
    const int sw = (NLW + NDEC - 1 - wid) * NCOMM + b;
    if (wid < NSET) {
        if (wid < 4) *(LAS u32*)(smem + LDS_CTL + (wid * 64 + lane) * 4) = 0;
        for (int k = 0; k < nld; ++k)
            build_table((LAS char*)smem + LDS_TAB + k * TAB_BYTES, rank, (k * NCOMM + b) % 7, threadIdx.x, NSET * 64);
    }
    const u32 call = next_call();
    if (wid >= NSET && sw < NPUSH) push_role(p, wsl, sw, lane, call);
    else asm volatile("s_waitcnt lgkmcnt(0)\n\ts_barrier" ::: "memory");
    STAMPB_IF(wid == 0, 4);
    if (wid >= NLW) decode_role(p, (LAS char*)smem, wid - NLW, lane, call);
    else if (wid < nld) {
        const int ps = rfl((wid * NCOMM + b) % 7), wv = rfl((wid * NCOMM + b) / 7);
        if (ps < 3) load_role<false>(p, (LAS char*)smem, wid, nld, ps, wv, lane, call);
        else        load_role<true>(p, (LAS char*)smem, wid, nld, ps, wv, lane, call);
    }
}

}

constexpr int g_mode = 0;

extern "C" void kernel_launch(int rank, int nranks, ncclComm_t comm, hipStream_t stream,
                              void* const* d_in, const int* in_sizes, int n_in,
                              void* d_out, int out_size, const int* in_dtypes, int out_dtype,
                              void* const* d_out_all, void* const* d_ws_all, size_t ws_bytes) {
    (void)comm; (void)in_sizes; (void)n_in; (void)d_out; (void)out_size; (void)in_dtypes; (void)out_dtype;
    if (nranks != NR || (unsigned)rank >= (unsigned)NR || ws_bytes < WS_NEED) {
        fprintf(stderr, "unsupported configuration\n");
        return;
    }
    Params p;
    p.x = (const float*)d_in[0];
    p.w = (const float*)d_in[1];
    for (int i = 0; i < NR; ++i) { p.ws[i] = (char*)d_ws_all[i]; p.out[i] = (char*)d_out_all[i]; }
    p.rank = rank;
    p.pad = g_mode;
    ag_gemm_kernel<<<GRID, BLOCK, 0, stream>>>(p);
    hipError_t e = hipGetLastError();
    if (e != hipSuccess) fprintf(stderr, "launch failed: %s\n", hipGetErrorString(e));
}
```

```cpp
#include <hip/hip_runtime.h>
#include <rccl/rccl.h>
#include <cstdio>
#include <cstdint>
#include <type_traits>

typedef unsigned u32;
typedef unsigned long long u64;
typedef u32 u32x4 __attribute__((ext_vector_type(4)));
typedef u32 u32x2v __attribute__((ext_vector_type(2)));
typedef float f32x4 __attribute__((ext_vector_type(4)));
typedef _Float16 f16x8 __attribute__((ext_vector_type(8)));
typedef __amdgpu_buffer_rsrc_t rsrc_t;
#define LAS __attribute__((address_space(3)))
#define GAS __attribute__((address_space(1)))

namespace {

constexpr int NR = 8;
constexpr int KD = 8192;
constexpr int NS = 1024;
constexpr int NBLK = 64;
#define P_FI 45
constexpr int FI = P_FI;
constexpr int FX = 64 - FI;
#define P_FO2 8
constexpr int FO2 = P_FO2, RF2 = 48 - FO2, FO1 = FI - FO2;
__host__ __device__ constexpr int OFF2(int i) { return (RF2 * i + 2) / 3; }
__host__ __device__ constexpr int F2(int i) { return OFF2(i + 1) - OFF2(i); }
__host__ __device__ constexpr int FI1(int i) { return FI - F2(i); }
__host__ __device__ constexpr int OFF1(int i) { return i * FI - OFF2(i); }
constexpr int NI1 = 3 * FI - RF2, FI1MAX = FI1(2);
constexpr int FL = 8;
constexpr int RF1 = 208, CF = 128;
static_assert(FO1 + NI1 + 4 * FX == RF1 && FX >= 16 && FX - FL <= 16 && (FO2 == 0 || FO2 == 8) && F2(0) >= F2(2), "rows");

#define P_W 19
#define P_XA 15
#define P_PB 1
#define P_GRADE 20
constexpr int PB = P_PB;
constexpr int NCOMM = 48;
constexpr int WL = P_W;
constexpr int XA = P_XA;
static_assert(WL >= 14 && WL <= 20 && XA <= FX && XA <= 16, "wave roles");
constexpr int GRID = 256, BLOCK = 768, NGEMM = GRID - NCOMM;
constexpr int RS1 = 64 + NI1 + 4 * FX;

constexpr u32 OFF_SEQ  = 0;
constexpr u32 OFF_TMO  = 16384;
constexpr u32 OFF_QUE  = 16640;
constexpr u32 OFF_RDY  = 32768;
constexpr u32 OFF_CNT  = 131072;
constexpr u32 CNT_BLK  = 24 * 128;
constexpr u32 OFF_DIAG = 524288;
constexpr u32 A1_BYTES = (u32)NBLK * RS1 * 4096;
constexpr u32 A2_BYTES = (u32)NBLK * RF2 * 4096;
constexpr u32 B_BYTES  = (u32)NBLK * CF * 4096;
constexpr u32 STG_BYTES = 4u * 1024 * 4096;
constexpr u32 PK_BYTES = (u32)NBLK * 128 * 4096;
constexpr u32 REL_A2 = A1_BYTES, REL_B = REL_A2 + A2_BYTES, REL_PK = REL_B + B_BYTES, REL_STG = REL_PK + PK_BYTES;
constexpr u32 PAR_BYTES = REL_STG + 2 * STG_BYTES;
constexpr u32 OFF_PAR  = 1u << 20;
constexpr int VAR_XO = 192;
constexpr u32 WS_NEED  = OFF_PAR + 2 * PAR_BYTES;
constexpr u32 OUT_BYTES = (u32)KD * NS * 4;
static_assert(REL_PK < (256u << 20), "table entries hold 18 bits of dst / 1024");
static_assert(256 + VAR_XO == 448, "output items are as long as x items");
static_assert(PK_BYTES == 8192u * 4096u, "staged outputs follow the packed image");
static_assert(OFF_CNT + NBLK * CNT_BLK <= OFF_DIAG && OFF_RDY + 8 * 8192 + 256 <= OFF_CNT, "control area");

constexpr u64 DEADLINE = 40000000ull;

constexpr int BUF_BYTES = 49152;
constexpr int LDS_BYTES = 3 * BUF_BYTES;

struct Params {
    const float* x;
    const float* w;
    char* ws[NR];
    char* out[NR];
    int rank;
    int pad;
};

__device__ __forceinline__ u32 rfl(u32 v) { return __builtin_amdgcn_readfirstlane(v); }
__device__ __forceinline__ char* uni(const void* p) {
    const u64 a = (u64)p;
    return (char*)(((u64)rfl((u32)(a >> 32)) << 32) | rfl((u32)a));
}
__device__ __forceinline__ rsrc_t rsrc_u(const void* p, u32 bytes) {
    return __builtin_amdgcn_make_buffer_rsrc((void*)p, (short)0, (int)bytes, 0x00020000);
}
__device__ __forceinline__ rsrc_t mk_rsrc(const void* p, u32 bytes) { return rsrc_u(uni(p), bytes); }
__device__ __forceinline__ u32 ld_sys(const char* p) {
    return __hip_atomic_load((const GAS u32*)p, __ATOMIC_RELAXED, __HIP_MEMORY_SCOPE_SYSTEM);
}
__device__ __forceinline__ u32 ld_agent(const char* p) {
    return __hip_atomic_load((const GAS u32*)p, __ATOMIC_RELAXED, __HIP_MEMORY_SCOPE_AGENT);
}
__device__ __forceinline__ void st_flag(char* p, u32 v) {
    __hip_atomic_store((GAS u32*)p, v, __ATOMIC_RELAXED, __HIP_MEMORY_SCOPE_SYSTEM);
}
__device__ __forceinline__ void add_sys(char* p) {
    __hip_atomic_fetch_add((GAS u32*)p, 1u, __ATOMIC_RELAXED, __HIP_MEMORY_SCOPE_SYSTEM);
}
#define WAIT_VM0() do { asm volatile("s_waitcnt vmcnt(0)" ::: "memory"); __builtin_amdgcn_s_waitcnt(0x0F70); } while (0)

template <int SLEEP = 2>
__device__ __forceinline__ bool spin_ge(const char* p, u32 need, char* tmo, u32 call) {
    const u64 t0 = __builtin_amdgcn_s_memrealtime();
    bool gave_up = false;
    for (u32 polls = 0;; ++polls) {
        const u32 v = ld_sys(p);
        if (__builtin_amdgcn_ballot_w64(v < need) == 0) break;
        if ((polls & 7) == 0 && (rfl(ld_agent(tmo)) == call || __builtin_amdgcn_s_memrealtime() - t0 > DEADLINE)) {
            __hip_atomic_store((GAS u32*)tmo, call, __ATOMIC_RELAXED, __HIP_MEMORY_SCOPE_AGENT);
            gave_up = true;
            break;
        }
        __builtin_amdgcn_s_sleep(SLEEP);
    }
    asm volatile("" ::: "memory");
    return gave_up;
}

#define STAMP_IF(c, slot) do {} while (0)
#define STAMP(slot) do {} while (0)
#define STAMPB_IF(c, slot) do {} while (0)

__device__ __forceinline__ u32x4 cvt8(f32x4 a, f32x4 b) {
    f16x8 h;
    h[0] = (_Float16)a[0]; h[1] = (_Float16)a[1]; h[2] = (_Float16)a[2]; h[3] = (_Float16)a[3];
    h[4] = (_Float16)b[0]; h[5] = (_Float16)b[1]; h[6] = (_Float16)b[2]; h[7] = (_Float16)b[3];
    return __builtin_bit_cast(u32x4, h);
}
__device__ __forceinline__ void st_sys(u32x4 v, rsrc_t r, u32 voff, u32 soff) {
    __builtin_amdgcn_raw_buffer_store_b128(v, r, voff + soff, 0, 17);
}
typedef float f32x2 __attribute__((ext_vector_type(2)));
__device__ __forceinline__ f32x2 gelu2(f32x2 y) {
    constexpr float A = -2.8853900817779268f * 0.7978845608028654f, B = A * 0.044715f;
    const f32x2 z = y * (y * y * B + A);
    const f32x2 d = (f32x2){__builtin_amdgcn_exp2f(z[0]), __builtin_amdgcn_exp2f(z[1])} + 1.0f;
    return y * (f32x2){__builtin_amdgcn_rcpf(d[0]), __builtin_amdgcn_rcpf(d[1])};
}

__device__ __forceinline__ int row_of_slot(int sl, int g, int j) {
    if (sl < 64) return (g * 4 + j) * NS + sl * 16;
    sl -= 64;
    if (sl < NI1) { const int i = sl >= OFF1(2) ? 2 : sl >= OFF1(1) ? 1 : 0; return (g * 4 + ((j + 1 + i) & 3)) * NS + (sl - OFF1(i)) * 16; }
    sl -= NI1;
    const int q = sl / FX;
    return ((g ^ 1) * 4 + q) * NS + (FI + sl - q * FX) * 16;
}
__device__ __forceinline__ int slot_of(int f, int q) {
    if (q >= 0) {
        if (f >= RF1 - FX) return -1;
        if (f >= RF1 - 4 * FX + q * FX) f += FX;
    } else if (f >= RF1) return f - RF1 < FX - FL ? FI + FL + f - RF1 : -1;
    return f < FO1 ? f : f + 64 - FO1;
}
__device__ __forceinline__ int row_of_f2(int f2, int g, int j) {
    if (f2 >= RF2) return (g * 4 + j) * NS + (FO1 + f2 - RF2) * 16;
    const int i = f2 >= OFF2(2) ? 2 : f2 >= OFF2(1) ? 1 : 0;
    return (g * 4 + ((j + 1 + i) & 3)) * NS + (FI1(i) + f2 - OFF2(i)) * 16;
}

constexpr int IN_AUX = 2;
constexpr int OUT_AUX = 17;
__device__ __forceinline__ u32x4 cvt_w(const f32x4* v, int c) {
    return cvt8((f32x4){v[0][c], v[1][c], v[2][c], v[3][c]}, (f32x4){v[4][c], v[5][c], v[6][c], v[7][c]});
}

#define DPP_STEP(op, v, ctrl, rmask) v = op(v, (u32)__builtin_amdgcn_update_dpp(0, (int)v, ctrl, rmask, 0xf, false))
__device__ __forceinline__ u32 umax2(u32 a, u32 b) { return a > b ? a : b; }
__device__ __forceinline__ u32 uadd2(u32 a, u32 b) { return a + b; }
__device__ __forceinline__ u32 wave_max(u32 v) {
    DPP_STEP(umax2, v, 0x111, 0xf); DPP_STEP(umax2, v, 0x112, 0xf); DPP_STEP(umax2, v, 0x114, 0xf); DPP_STEP(umax2, v, 0x118, 0xf);
    DPP_STEP(umax2, v, 0x142, 0xa); DPP_STEP(umax2, v, 0x143, 0xc);
    return __builtin_amdgcn_readlane(v, 63);
}
__device__ __forceinline__ u32 wave_sum(u32 v) {
    DPP_STEP(uadd2, v, 0x111, 0xf); DPP_STEP(uadd2, v, 0x112, 0xf); DPP_STEP(uadd2, v, 0x114, 0xf); DPP_STEP(uadd2, v, 0x118, 0xf);
    DPP_STEP(uadd2, v, 0x142, 0xa); DPP_STEP(uadd2, v, 0x143, 0xc);
    return __builtin_amdgcn_readlane(v, 63);
}

__device__ __forceinline__ u32 wave_scan(u32 v) {
    DPP_STEP(uadd2, v, 0x111, 0xf); DPP_STEP(uadd2, v, 0x112, 0xf); DPP_STEP(uadd2, v, 0x114, 0xf); DPP_STEP(uadd2, v, 0x118, 0xf);
    DPP_STEP(uadd2, v, 0x142, 0xa); DPP_STEP(uadd2, v, 0x143, 0xc);
    return v;
}
__device__ __forceinline__ u32 byte_scan(u32 m) {
    u32 a, r;
    asm("v_lshl_add_u32 %0, %1, 8, %1" : "=v"(a) : "v"(m));
    asm("v_lshl_add_u32 %0, %1, 16, %1" : "=v"(r) : "v"(a));
    return r;
}
__device__ __forceinline__ u32 times7(u32 m) {
    u32 a, r;
    asm("v_lshl_add_u32 %0, %1, 1, %1" : "=v"(a) : "v"(m));
    asm("v_lshl_add_u32 %0, %1, 2, %2" : "=v"(r) : "v"(m), "v"(a));
    return r;
}
__device__ __forceinline__ u32 rank_in(u64 m) { return __builtin_amdgcn_mbcnt_hi((u32)(m >> 32), __builtin_amdgcn_mbcnt_lo((u32)m, 0)); }

constexpr int VAR_X = VAR_XO, VAR_W = 256;
__device__ __forceinline__ u32 h7_of(float a) { return (u32)__builtin_bit_cast(unsigned short, (_Float16)a) >> 8; }
__device__ __forceinline__ void lds_or(LAS char* a, u32 v) {
    __hip_atomic_fetch_or((LAS u32*)a, v, __ATOMIC_RELAXED, __HIP_MEMORY_SCOPE_WORKGROUP);
}
__device__ __forceinline__ void put_bits(LAS char* area, u32 bo, u64 v) {
    const u32 sh = bo & 31;
    const u64 w = v << sh;
    LAS char* a = area + (bo >> 5) * 4;
    lds_or(a, (u32)w); lds_or(a + 4, (u32)(w >> 32)); lds_or(a + 8, (u32)((v >> 1) >> (63 - sh)));
}
__device__ __forceinline__ u64 get_bits(const LAS char* area, u32 bo) {
    const LAS u32* w = (const LAS u32*)(area + (bo >> 5) * 4);
    const u32 w0 = w[0], w1 = w[1], w2 = w[2];
    return (u64)__builtin_amdgcn_alignbit(w2, w1, bo) << 32 | __builtin_amdgcn_alignbit(w1, w0, bo);
}
template <class F>
__device__ __forceinline__ void pack_item(const u32x4 (&h)[4], float amax, LAS char* scr, rsrc_t r0, u32 dst, int lane, F before_stores) {
    u32 L[8], H[8];
#pragma unroll
    for (int t = 0; t < 8; ++t) {
        const u32 d0 = h[t >> 1][(t & 1) * 2], d1 = h[t >> 1][(t & 1) * 2 + 1];
        L[t] = __builtin_amdgcn_perm(d1, d0, 0x06040200u);
        H[t] = __builtin_amdgcn_perm(d1, d0, 0x07050301u);
    }
    *(LAS u32x4*)(scr + lane * 16) = (u32x4){H[0], H[1], H[2], H[3]};
    *(LAS u32x4*)(scr + lane * 16 + 1024) = (u32x4){H[4], H[5], H[6], H[7]};
    LAS char* area = scr + 2048;
    *(LAS u32x2v*)(area + lane * 8) = (u32x2v){0u, 0u};
    const u32 lm = wave_sum(h7_of(amax) & 127) + 132, rare = (lm & 255) < 115 ? 1 : 0;
    const u32 E1 = lm < 256 ? 1 : lm < 31 * 256 ? lm >> 8 : 31;
    const u32 eb = E1 * 0x01010101u | 0x80808080u;
    const u32 tbig = rare ? 0x79797979u : 0x7a7a7a7au, vadd = rare ? 0x01010101u : 0x02020202u, top6 = rare ? 0x06060606u : 0u;
    const u32 clo = 0x02010003u, qlo = rare ? 3u : 2u, qhi = rare ? 0x03020100u : 0x03030100u;
    u32 K[2] = {0, 0}, e4 = 0, pos = 0;
    u64 b2 = 0, b3 = 0;
#pragma unroll
    for (int t = 0; t < 8; ++t) {
        const u32 o = (eb - ((H[t] >> 2) & 0x1f1f1f1fu)) ^ 0x80808080u, o7 = o & 0x7f7f7f7fu;
        const u32 far = (o | (o7 + tbig)) & 0x80808080u, huge = (o | (o7 + 0x73737373u)) & 0x80808080u;
        const u32 f7 = (far >> 4) - (far >> 7), sel = (o | f7) & 0x07070707u;
        const u32 c = __builtin_amdgcn_perm(0x03030303u, clo, sel);
        const u32 q = __builtin_amdgcn_perm(qhi, qlo, sel);
        const u32 M = __builtin_amdgcn_perm(0x01010101u, 0x00000001u, sel);
        const u32 M2 = q & (q >> 1);
        const u32 v = (((o7 + vadd) | ((huge >> 4) - (huge >> 7))) & f7) | __builtin_amdgcn_perm(0u, top6 & 255, sel);
        const u32 P = byte_scan(M), P2 = byte_scan(M2), X2 = P2 - M2;
        const u32 bits2 = __builtin_amdgcn_udot4(q, __builtin_amdgcn_perm(0u, 0x40100401u, P - M), 0u, false);
        const u32 bits3 = __builtin_amdgcn_udot4(v, __builtin_amdgcn_perm(0u, 0x00400801u, X2), (X2 >> 24) == 3 ? (v >> 24) << 9 : 0u, false);
        b2 |= (u64)bits2 << (pos & 63);
        b3 |= (u64)bits3 << ((pos >> 16) & 63);
        pos += (P >> 23) + ((P2 >> 24) * 3 << 16);
        K[t >> 2] |= c << (2 * (t & 3));
        e4 = (e4 >> 1) | huge;
        if (t & 1) { asm volatile("" : "+v"(b2), "+v"(b3), "+v"(e4)); __builtin_amdgcn_sched_barrier(0); }
    }
    const u32 incl = wave_scan(pos), tot = __builtin_amdgcn_readlane(incl, 63), ex = incl - pos;
    const u32 used = 8 + (tot & 0xffff) + (tot >> 16);
    u32 need = used;
    if (__builtin_amdgcn_ballot_w64(e4 != 0) != 0) need += 8 * wave_sum(__builtin_popcount(e4));
    const u32 var = need <= 8 * VAR_X ? VAR_X : VAR_W;
    const bool fits = need <= 8 * VAR_W && __builtin_amdgcn_ballot_w64(pos >= (65u << 16)) == 0;
    if (fits) {
        u32 n4 = 0;
        put_bits(area, 8 + (ex & 0xffff), b2);
        put_bits(area, 8 + (tot & 0xffff) + (ex >> 16), b3);
        while (__builtin_amdgcn_ballot_w64(e4 != 0) != 0) {
            const u64 act = __builtin_amdgcn_ballot_w64(e4 != 0);
            const u32 any = (e4 | (e4 >> 8) | (e4 >> 16) | (e4 >> 24)) & 255, t = __builtin_ctz(any | 128);
            const u32 b = __builtin_ctz(((e4 >> t) & 0x01010101u) | 0x80000000u) + t;
            const u32 e = (*(const LAS unsigned char*)(scr + (t & 4) * 256 + lane * 16 + (t & 3) * 4 + ((b >> 3) & 3)) >> 2) & 31;
            const u32 k = var - 1 - n4 - rank_in(act);
            if (e4 != 0) lds_or(area + (k & ~3u), e << ((k & 3) * 8));
            e4 &= ~(1u << (b & 31));
            n4 += __builtin_popcountll(act);
        }
    }
    asm volatile("" ::: "memory");
    u32 vw = *(const LAS u32*)(area + lane * 4);
    if (lane == 0) vw |= fits ? E1 | rare << 5 | (var == VAR_W ? 64 : 0) : 255;
    const u32 P0 = (H[0] & 0x83838383u) | ((H[1] & 0x03030303u) << 2) | ((H[1] >> 1) & 0x40404040u) | ((H[6] & 0x03030303u) << 4);
    const u32 P1 = (H[2] & 0x83838383u) | ((H[3] & 0x03030303u) << 2) | ((H[3] >> 1) & 0x40404040u) | ((H[7] & 0x03030303u) << 4);
    const u32 P2 = (H[4] & 0x83838383u) | ((H[5] & 0x03030303u) << 2) | ((H[5] >> 1) & 0x40404040u)
                 | ((H[6] >> 2) & 0x20202020u) | ((H[7] >> 3) & 0x10101010u);
    before_stores();
    st_sys((u32x4){L[0], L[1], L[2], L[3]}, r0, lane * 16, dst);
    st_sys((u32x4){L[4], L[5], L[6], L[7]}, r0, lane * 16, dst + 1024);
    st_sys((u32x4){P0, P1, P2, K[0]}, r0, lane * 16, dst + 2048);
    __builtin_amdgcn_raw_buffer_store_b32(K[1], r0, lane * 4 + dst + 3072, 0, 17);
    __builtin_amdgcn_raw_buffer_store_b32(vw, r0, lane * 4 + dst + 3328, 0, 17);
}
__device__ __forceinline__ void unpack_item(u32x4 (&h)[4], const u32x4& c0, const u32x4& c1, const u32x4& c2, u32 k1,
                                            const LAS char* area, u32 hd, LAS char* scr, int lane) {
    const u32 E1 = hd & 31, var = (hd & 64) ? VAR_W : VAR_X;
    const bool rare = hd & 32;
    const u32 K[2] = {c2[3], k1};
    const u32 X[2] = {K[0] & (K[0] >> 1) & 0x55555555u, K[1] & (K[1] >> 1) & 0x55555555u};
    const u32 n2 = 2 * (__builtin_popcount(X[0]) + __builtin_popcount(X[1])), i2 = wave_scan(n2);
    u64 b2 = get_bits(area, 8 + i2 - n2);
    b2 &= n2 < 64 ? ~(~0ull << n2) : ~0ull;
    const u64 y2 = b2 & (b2 >> 1) & 0x5555555555555555ull;
    const u32 n3 = 3 * __builtin_popcountll(y2), i3 = wave_scan(n3);
    u64 b3 = get_bits(area, 8 + __builtin_amdgcn_readlane(i2, 63) + i3 - n3);
    const u32 e4x = E1 * 0x04040404u | 0x80808080u;
    const u32 t1 = (e4x - 0x000c0804u) & 0x007c7c7cu, t2 = (e4x - (rare ? 0x00181410u : 0x00001410u)) & 0x007c7c7cu;
    const u32 t3lo = (e4x - (rare ? 0x2824201cu : 0x24201c18u)) & 0x7c7c7c7cu, t3hi = (e4x - (rare ? 0x0000302cu : 0x00302c28u)) & 0x007c7c7cu;
    LAS char* row = scr + lane * 36;
    u32 e4 = 0;
    const u32 nch = (wave_max(n2) + 7) >> 3;
#pragma unroll
    for (int k = 0; k < 8; ++k) {
        if (k >= 4 && k >= nch) break;
        const u32 b8 = (u32)(b2 >> (8 * k)) & 255, y = b8 | (b8 << 12), q = (y | (y << 6)) & 0x03030303u;
        const u32 M2 = q & (q >> 1), P2 = byte_scan(M2), m7 = times7(M2);
        const u32 lo = (u32)b3, z = (lo & 0x3f) | ((lo & 0xfc0) << 10), f = (z | (z << 5)) & 0x07070707u;
        const u32 v = __builtin_amdgcn_perm(0u, f, P2 - M2) & m7;
        b3 >>= (P2 >> 24) * 3;
        *(LAS u32*)(row + 4 * k) = __builtin_amdgcn_perm(0u, t2, q) | __builtin_amdgcn_perm(t3hi, t3lo, v | (0x0c0c0c0cu - (M2 << 2) - (M2 << 3)));
        e4 |= (((v + M2) >> 3) & 0x01010101u) << k;
    }
    u32 n4 = 0;
    while (__builtin_amdgcn_ballot_w64(e4 != 0) != 0) {
        const u64 act = __builtin_amdgcn_ballot_w64(e4 != 0);
        const u32 any = (e4 | (e4 >> 8) | (e4 >> 16) | (e4 >> 24)) & 255, t = __builtin_ctz(any | 128);
        const u32 b = __builtin_ctz(((e4 >> t) & 0x01010101u) | 0x80000000u) + t;
        const u32 x = *(const LAS unsigned char*)(area + ((var - 1 - n4 - rank_in(act)) & 255));
        if (e4 != 0) *(LAS unsigned char*)(row + t * 4 + (b >> 3)) = x << 2;
        e4 &= ~(1u << (b & 31));
        n4 += __builtin_popcountll(act);
    }
    const u32 SM[8] = {c2[0] & 0x83838383u, ((c2[0] << 1) & 0x80808080u) | ((c2[0] >> 2) & 0x03030303u),
                       c2[1] & 0x83838383u, ((c2[1] << 1) & 0x80808080u) | ((c2[1] >> 2) & 0x03030303u),
                       c2[2] & 0x83838383u, ((c2[2] << 1) & 0x80808080u) | ((c2[2] >> 2) & 0x03030303u),
                       ((c2[2] << 2) & 0x80808080u) | ((c2[0] >> 4) & 0x03030303u),
                       ((c2[2] << 3) & 0x80808080u) | ((c2[1] >> 4) & 0x03030303u)};
    u32 M[8], P[8], g[8], at = 0;
#pragma unroll
    for (int t = 0; t < 8; ++t) {
        M[t] = (X[t >> 2] >> (2 * (t & 3))) & 0x01010101u;
        P[t] = byte_scan(M[t]);
        const LAS u32* w = (const LAS u32*)(row + (at & ~3u));
        g[t] = __builtin_amdgcn_alignbyte(w[1], w[0], at);
        at += P[t] >> 24;
    }
#pragma unroll
    for (int t = 0; t < 8; ++t) {
        const u32 c = (K[t >> 2] >> (2 * (t & 3))) & 0x03030303u;
        const u32 m = times7(M[t]);
        const u32 H = SM[t] | __builtin_amdgcn_perm(t1, g[t], ((P[t] - M[t]) & m) | ((c | 0x04040404u) & ~m));
        const u32 L = t < 4 ? c0[t & 3] : c1[t & 3];
        h[t >> 1][(t & 1) * 2] = __builtin_amdgcn_perm(H, L, 0x05010400u);
        h[t >> 1][(t & 1) * 2 + 1] = __builtin_amdgcn_perm(H, L, 0x07030602u);
    }
}

__device__ __forceinline__ void out_windows(const u32x4 (&o)[4], u32& tp, u32& tn) {
    u32 mn = 0;
    int mp = 0;
#pragma unroll
    for (int i = 0; i < 16; ++i) { mp = max(mp, (int)o[i >> 2][i & 3]); mn = umax2(mn, o[i >> 2][i & 3]); }
    tp = wave_max((u32)mp) >> 24;
    tp -= __builtin_popcountll(__builtin_amdgcn_ballot_w64(mp >= (int)(tp << 24))) <= 6;
    tp = (int)tp > 3 ? tp : 3;
    tn = wave_max(mn) >> 24;
    tn = tn > 130 ? tn : 130;
}
__device__ __forceinline__ void pack_out(const u32x4 (&o)[4], u32 tp, u32 tn, LAS char* scr, rsrc_t r0, u32 dst, int lane) {
    u32 M[12], G2 = 0, G1 = 0, em = 0;
    u32x4 E;
#pragma unroll
    for (int r = 0; r < 4; ++r) {
#pragma unroll
        for (int i = 0; i < 3; ++i) M[r * 3 + i] = __builtin_amdgcn_perm(o[r][3], o[r][i], 0x04020100u + (i << 24));
        E[r] = __builtin_amdgcn_perm(o[r][1], o[r][0], 0x0c0c0703u) | __builtin_amdgcn_perm(o[r][3], o[r][2], 0x07030c0cu);
    }
    *(LAS u32x4*)(scr + lane * 16) = E;
#pragma unroll
    for (int r = 0; r < 4; ++r) {
        const u32 sg = E[r] & 0x80808080u, sm = sg - (sg >> 7);
        const u32 top = ((tn & 127) * 0x01010101u & sm) | (tp * 0x01010101u & ~sm);
        const u32 d = ((top | 0x80808080u) - (E[r] & 0x7f7f7f7fu)) ^ 0x80808080u;
        const u32 out = (d | ((d & 0x7f7f7f7fu) + 0x7c7c7c7cu + (sg >> 7))) & 0x80808080u;
        const u32 g = ((d & 0x03030303u) + 0x01010101u + (sg >> 5)) & ~((out >> 4) - (out >> 7));
        G2 |= (g & 0x03030303u) << (2 * r);
        G1 |= (g & 0x04040404u) >> 2 << r;
        em = (em >> 1) | out;
    }
    LAS unsigned char* list = (LAS unsigned char*)(scr + 1024);
    u32 used = 2;
    for (u64 act = __builtin_amdgcn_ballot_w64(em != 0); act != 0; act = __builtin_amdgcn_ballot_w64(em != 0)) {
        const u32 k = used + rank_in(act), b = __builtin_ctz(em | 0x80000000u);
        list[em != 0 && k < 64 ? k : 64] = *(const LAS unsigned char*)(scr + lane * 16 + (b & 3) * 4 + (b >> 3));
        em &= em - 1;
        used += __builtin_popcountll(act);
        if (used > 64) break;
    }
    *(LAS unsigned short*)list = (used > 64 ? 255 : tp) | tn << 8;
    asm volatile("" ::: "memory");
    const u32 ent = *(const LAS u32*)(list + (lane & 15) * 4);
    if (used > 64) {
#pragma unroll
        for (int r = 0; r < 4; ++r) st_sys(o[r], r0, lane * 16 + r * 1024, dst + STG_BYTES);
    }
#pragma unroll
    for (int i = 0; i < 3; ++i) st_sys((u32x4){M[4 * i], M[4 * i + 1], M[4 * i + 2], M[4 * i + 3]}, r0, lane * 16 + i * 1024, dst);
    __builtin_amdgcn_raw_buffer_store_b32(G2, r0, lane * 4 + dst + 3072, 0, 17);
    __builtin_amdgcn_raw_buffer_store_b16((unsigned short)__builtin_amdgcn_perm(0u, G1 | (G1 >> 4), 0x0c0c0200u), r0, lane * 2 + dst + 3328, 0, 17);
    __builtin_amdgcn_raw_buffer_store_b32(ent, r0, (lane < 16 ? lane * 4 : 0x80000000u) + dst + 3456, 0, 17);
}
__device__ __forceinline__ void unpack_out(u32x4 (&o)[4], const u32x4& c0, const u32x4& c1, const u32x4& c2, u32 G2, u32 g1,
                                           const LAS unsigned char* list, u32 tp, u32 tn, LAS char* scr, int lane) {
    const u32 M[12] = {c0[0], c0[1], c0[2], c0[3], c1[0], c1[1], c1[2], c1[3], c2[0], c2[1], c2[2], c2[3]};
    const u32 t = __builtin_amdgcn_perm(0u, g1, 0x0c010c00u), G1 = t | (t << 4);
    const u32 tlo = tp * 0x01010100u - 0x02010000u, thi = tn * 0x01010100u - 0x02010000u + tp - 3;
    u32x4 E;
    u32 em = 0;
#pragma unroll
    for (int r = 0; r < 4; ++r) {
        const u32 g = ((G2 >> (2 * r)) & 0x03030303u) | ((G1 >> r << 2) & 0x04040404u);
        E[r] = __builtin_amdgcn_perm(thi, tlo, g);
        em = (em >> 1) | (~(g + 0x7f7f7f7fu) & 0x80808080u);
    }
    *(LAS u32x4*)(scr + lane * 16) = E;
    u32 used = 2;
    auto step = [&](u32& a, u32& val) {
        const u64 act = __builtin_amdgcn_ballot_w64(em != 0);
        const u32 b = __builtin_ctz(em | 0x80000000u);
        a = em != 0 ? lane * 16 + (b & 3) * 4 + (b >> 3) : 1024;
        val = list[(used + rank_in(act)) & 63];
        em &= em - 1;
        used += __builtin_popcountll(act);
    };
    do {
        u32 a0, a1, a2, x0, x1, x2;
        step(a0, x0); step(a1, x1); step(a2, x2);
        *(LAS unsigned char*)(scr + a0) = x0; *(LAS unsigned char*)(scr + a1) = x1; *(LAS unsigned char*)(scr + a2) = x2;
    } while (__builtin_amdgcn_ballot_w64(em != 0) != 0);
    E = *(const LAS u32x4*)(scr + lane * 16);
#pragma unroll
    for (int r = 0; r < 4; ++r) {
#pragma unroll
        for (int i = 0; i < 3; ++i) o[r][i] = __builtin_amdgcn_perm(E[r], M[r * 3 + i], 0x04020100u + (i << 24));
        o[r][3] = __builtin_amdgcn_perm(M[r * 3 + 1], M[r * 3], 0x0c0c0703u) | __builtin_amdgcn_perm(E[r], M[r * 3 + 2], 0x07030c0cu);
    }
}

__device__ __forceinline__ void load_item(f32x4 (&v)[8], const Params& p, int u, int wid, int lane) {
    const int blk = u >> 4, it = (u & 15) * 8 + wid;
    const bool isx = it < 64;
    const rsrc_t r = mk_rsrc(isx ? p.x : p.w, OUT_BYTES);
    const u32 vf = isx ? (lane & 15) * 32768 + (lane >> 4) * 16 : (lane >> 4) * 16384 + (lane & 15) * 16;
    const u32 base = isx ? it * 524288 + blk * 512 : (blk * 128 + (wid & 3) * 32) * 4096 + ((it - 64) >> 2) * 256;
    const u32 sa = isx ? 64 : 4096, sb = isx ? 256 : 65536;
#pragma unroll
    for (int i = 0; i < 8; ++i)
        v[i] = __builtin_bit_cast(f32x4, __builtin_amdgcn_raw_buffer_load_b128(r, vf, base + (i & 3) * sa + (i >> 2) * sb, IN_AUX));
}
__device__ __forceinline__ void convert_local(const Params& p, LAS char* lds, f32x4 (&v)[8], char* wsl, int wg, int wid, int lane, u32 call) {
    const int rank = rfl(p.rank);
    constexpr int NU = NBLK * 16;
    const u32 par = OFF_PAR + (call & 1) * PAR_BYTES;
    const rsrc_t r0 = mk_rsrc(p.ws[rank], WS_NEED);
    LAS char* scr = lds + wid * 4096;
    auto publish = [&](int u) {
        if (wid == 0) {
            if (lane < 8 && lane != rank) st_flag(wsl + OFF_RDY + rank * 8192 + u * 4, call);
            if (lane == 8) add_sys(p.ws[rank] + OFF_CNT + (u >> 4) * CNT_BLK + ((u & 8) ? 1024 : 0) + rank * 128);
        }
    };
    int u = wg, un = (wg + NGEMM) ^ 8, up = -1;
    WAIT_VM0();
    for (int r = 0; u < NU; ++r) {
        const int blk = u >> 4, grp = u & 15, it = grp * 8 + wid;
        u32x4 h[4];
        u32 base, step;
        if (grp < 8) {
            base = par + (u32)blk * (RS1 * 4096) + it * 4096; step = 1024;
#pragma unroll
            for (int i = 0; i < 4; ++i) h[i] = cvt8(v[2 * i], v[2 * i + 1]);
        } else {
            base = par + REL_B + (u32)blk * (CF * 4096) + ((it - 64) & 60) * 4096 + (wid & 3) * 1024; step = 4096;
#pragma unroll
            for (int c = 0; c < 4; ++c) h[c] = cvt_w(v, c);
        }
        float amax = 0.0f;
#pragma unroll
        for (int i = 0; i < 8; ++i) amax = fmaxf(fmaxf(fmaxf(amax, fabsf(v[i][0])), fmaxf(fabsf(v[i][1]), fabsf(v[i][2]))), fabsf(v[i][3]));
        STAMPB_IF(r == 0, 0);
        if (r > 0) load_item(v, p, (u32)un < (u32)NU ? un : NU - 1, wid, lane);
        pack_item(h, amax, scr, r0, par + REL_PK + (u32)(blk * 128 + it) * 4096, lane, [&] {
            WAIT_VM0();
            if (r > 0) {
                __builtin_amdgcn_s_barrier();
                if (up >= 0) publish(up);
            }
#pragma unroll
            for (int i = 0; i < 4; ++i) st_sys(h[i], r0, lane * 16, base + i * step);
        });
        up = u;
        if (r == 0) {
            STAMPB_IF(true, 1);
            WAIT_VM0();
            STAMPB_IF(true, 2);
            __builtin_amdgcn_s_barrier();
            STAMP(6);
            publish(u);
            up = -1;
            load_item(v, p, un, wid, lane);
            WAIT_VM0();
        }
        u = un;
        un = (un + NGEMM) ^ 8;
    }
    WAIT_VM0();
    __builtin_amdgcn_s_barrier();
    if (up >= 0) publish(up);
}

constexpr int PUSH_PER = FI1MAX + 64 + XA, NPUSH = PB * PUSH_PER, NPW = (NPUSH + NCOMM - 1) / NCOMM;
__device__ __forceinline__ void push_decode(int n, int& blk, int& it) {
    blk = n / PUSH_PER; it = n - blk * PUSH_PER;
    if (it >= FI1MAX + 64) { const int xi = blk * XA + it - FI1MAX - 64; blk = xi / FX; it = FI + xi - blk * FX; }
    else if (it >= FI1MAX) it += 64 - FI1MAX;
}
__device__ __forceinline__ void push_role(const Params& p, char* wsl, int n, int lane, u32 call) {
    const int rank = rfl(p.rank);
    const int g = rank >> 2, j = rank & 3, og = (g ^ 1) * 4;
    int blk, it;
    push_decode(n, blk, it);
    f32x4 v[8];
    load_item(v, p, blk * 16 + (it >> 3), it & 7, lane);
    const u32 par = OFF_PAR + (call & 1) * PAR_BYTES;
    u32x4 h[4];
    int d0 = 0, nd = 4, dstep = 1; u32 base, step = 1024;
    if (it < 64) {
#pragma unroll
        for (int i = 0; i < 4; ++i) h[i] = cvt8(v[2 * i], v[2 * i + 1]);
        if (it < FI) { d0 = j + 3; nd = 3; dstep = 3; base = par + (blk * RS1 + 64 + it) * 4096; }
        else base = par + (blk * RS1 + 64 + NI1 + j * FX + it - FI) * 4096;
    } else {
#pragma unroll
        for (int c = 0; c < 4; ++c) h[c] = cvt_w(v, c);
        d0 = (it - 64) >> 4; nd = 1; step = 4096;
        base = par + REL_B + (blk * CF + 64 + 16 * j + (it & 12)) * 4096 + (it & 3) * 1024;
    }
    u32 dmask = 0;
    for (int d = 0; d < nd; ++d) {
        if (it < FI && it >= FI1(d)) continue;
        const int dr = (it < FI ? g * 4 : og) + ((d0 + d * dstep) & 3);
        const rsrc_t rd = mk_rsrc(p.ws[dr], WS_NEED);
#pragma unroll
        for (int i = 0; i < 4; ++i) st_sys(h[i], rd, lane * 16, base + (it < FI ? OFF1(d) * 4096 : 0) + i * step);
        dmask |= 1u << dr;
    }
    __builtin_amdgcn_s_barrier();
    WAIT_VM0();
    STAMPB_IF(true, 3);
    if (lane < 8 && ((dmask >> lane) & 1)) add_sys(wsl + OFF_CNT + blk * CNT_BLK + (it < 64 ? 0 : 1024) + rank * 128);
}

#define P_NDEC 9
#define P_DSLEEP 2
constexpr int SCR = 2560;
constexpr int NLW = 3, RING = 6, QLEN = 32, NDEC = P_NDEC;
constexpr int LDS_CTL = NLW * RING * 4096;
constexpr int LDS_DONE = LDS_CTL + QLEN * 16, LDS_TAIL = LDS_DONE + NLW * RING * 4, LDS_SCR = LDS_CTL + 1024;
static_assert(7 * WL <= NLW * NCOMM, "loaders");
constexpr u32 IDX_END = 0xffffffffu, IDX_BAD = 0x80000000u;
__device__ __forceinline__ u32 lds_get(const LAS char* a) {
    u32 v;
    asm volatile("ds_read_b32 %0, %1\n\ts_waitcnt lgkmcnt(0)" : "=v"(v) : "v"((u32)(size_t)a) : "memory");
    return rfl(v);
}
__device__ __forceinline__ void lds_put(LAS char* a, u32 v) {
    asm volatile("" ::: "memory");
    *(volatile LAS u32*)a = v;
    asm volatile("" ::: "memory");
}

struct Item { u32 idx, dst; };
constexpr int NT_I = NBLK * FI;
constexpr int PER = 16 + XA, NP = NBLK * PER, NX = NBLK * (16 + FX), NO1 = (RF1 - FX) * 4, NT_C = NX + NO1 + 192;
constexpr int TAB_BYTES = (NT_I > NT_C ? NT_I : NT_C) * 4, LDS_TAB = LDS_SCR + NDEC * SCR;
static_assert(LDS_TAB + NLW * TAB_BYTES <= LDS_BYTES, "communication LDS");
__device__ __forceinline__ void build_table(LAS char* tab, int rank, int slot, int first, int stride) {
    const int g = rank >> 2, j = rank & 3, q = slot - 3;
    for (int it = first; it < (slot < 3 ? NT_I : NT_C); it += stride) {
        Item m;
        if (slot < 3) {
            const int f1 = FI1(slot), f2 = F2(slot);
            const bool s2 = it >= NBLK * f1;
            const int a = s2 ? it - NBLK * f1 : it, blk = a / (s2 ? f2 : f1), s = a - blk * (s2 ? f2 : f1) + (s2 ? f1 : 0);
            m.idx = blk * 128 + s;
            m.dst = (s2 ? REL_A2 + (OFF2(slot) - f1) * 4096 : (64 + OFF1(slot)) * 4096) + (blk * (s2 ? RF2 : RS1) + s) * 4096;
        } else if (it < NX) {
            const int blk = it / PER, c = it - blk * PER;
            const bool tail = it >= NP, isx = tail || (c < 2 * XA && (c & 1));
            const int cw = c < 2 * XA ? c >> 1 : c - XA;
            const int xi = tail ? it - NP + NBLK * XA : blk * XA + (c >> 1), xb = xi / FX, xs = FI + xi - xb * FX;
            m.idx = isx ? xb * 128 + xs : blk * 128 + 64 + 16 * j + cw;
            m.dst = isx ? (xb * RS1 + xs + 64 + NI1 + q * FX - FI) * 4096
                        : REL_B + (blk * CF + 64 + 16 * q + (cw & 12)) * 4096 + (cw & 3) * 1024;
        } else {
            const int k = it - NX;
            int row, col;
            if (k < NO1) {
                const int f = k >> 2;
                m.idx = 8192 + (f >> 4) * 64 + (k & 2) * 16 + ((f >> 2) & 3) * 8 + (k & 1) * 4 + (f & 3);
                row = row_of_slot(slot_of(f, j), g ^ 1, q);
                col = (k & 3) * 64;
            } else {
                const int z = (k - NO1 + RF2 * 4) % 192;
                m.idx = 8192 + 768 + k - NO1;
                row = row_of_f2((z >> 5) * 8 + (z & 7), g ^ 1, q);
                col = ((z >> 3) & 3) * 64;
            }
            m.dst = (u32)row * 4096 + (q * 256 + col) * 4;
        }
        *(LAS u32*)(tab + it * 4) = m.idx | m.dst << (m.idx < 8192 ? 4 : 6);
    }
}

template <bool CROSS>
__device__ __forceinline__ void load_role(const Params& p, LAS char* lds, int ld, int nld, int slot, int wv, int lane, u32 call) {
    const int rank = rfl(p.rank);
    const int g = rank >> 2, j = rank & 3;
    const int peer = CROSS ? (g ^ 1) * 4 + slot - 3 : g * 4 + ((j + 1 + slot) & 3);
    constexpr int NT = CROSS ? NT_C : NT_I;
    const u32 par = OFF_PAR + (call & 1) * PAR_BYTES;
    char* ws = uni(p.ws[rank]);
    const rsrc_t r0 = rsrc_u(ws, WS_NEED), rp = mk_rsrc(p.ws[peer], WS_NEED);
    const char* rdy = ws + OFF_RDY + peer * 8192;
    const u32 vo = lane * 16;
    const int SKIP = PB * (CROSS ? PER : FI1(slot));
    const u32 head = lane == 0 ? OFF_QUE + slot * 256 : WS_NEED, tbase = SKIP + 4 * WL - (call - 1) * (NT - SKIP);
    const bool nowait = (p.pad & 4) != 0;
    const LAS char* tab = lds + LDS_TAB + ld * TAB_BYTES;
    const LAS char* done = lds + LDS_DONE + ld * RING * 4;
    const u32 ldsb = rfl((u32)(size_t)lds) + ld * RING * 4096;
    auto table = [&](int it) {
        asm volatile("ds_read_b32 v160, %0" :: "v"((u32)(size_t)tab + (it < NT ? it : NT - 1) * 4) : "memory", "v160");
    };
    auto entry = [&] {
        u32 t;
        asm volatile("s_waitcnt lgkmcnt(0)\n\tv_readfirstlane_b32 %0, v160" : "=s"(t) :: "memory");
        Item m;
        m.idx = t & 16383;
        m.dst = (t - m.idx) >> (CROSS && m.idx >= 8192 ? 6 : 4);
        return m;
    };
    auto decode = [&](int it) { table(it); return entry(); };
    auto flag_of = [&](u32 idx) { return idx < 8192 ? idx >> 3 : (idx < 8192 + 768 ? 1024 + ((idx - 8192) >> 2) : idx - (8192 + 768 - 1024 - 192)); };
    u32 okupto = 0, wbase = 0; u64 wmask = 0;
    auto seen = [&](u32 fi) { return fi < okupto || (fi - wbase < 64u && ((wmask >> (fi - wbase)) & 1)); };
    auto ready = [&](u32 fi) {
        if (seen(fi)) return true;
        for (;;) {
            wbase = okupto;
            wmask = __builtin_amdgcn_ballot_w64(ld_sys(rdy + (wbase + lane) * 4) >= call);
            if (wmask != ~0ull) break;
            okupto += 64;
            if (fi < okupto) return true;
        }
        okupto += __builtin_ctzll(~wmask);
        if (fi - wbase < 64u) return ((wmask >> (fi - wbase)) & 1) != 0;
        return rfl(ld_sys(rdy + fi * 4)) >= call;
    };
    const u32 vl = vo < 256 + VAR_X ? vo : 0x80000000u;
    struct Prep { u32 src, m0v; };
    auto prep = [&](const Item& m, u32 n) {
        Prep r;
        r.src = par + REL_PK + (m.idx + (CROSS && m.idx >= 8192 ? j * 1024 : 0)) * 4096;
        r.m0v = ldsb + (n % RING) * 4096;
        asm volatile("" : "+s"(r.src), "+s"(r.m0v));
        return r;
    };
    auto issue = [&](const Prep& r) {
        asm volatile("s_mov_b32 m0, %0\n\ts_nop 0\n\t"
                     "buffer_load_dwordx4 %1, %3, %4 offen sc0 sc1 lds\n\t"
                     "buffer_load_dwordx4 %1, %3, %4 offen offset:1024 sc0 sc1 lds\n\t"
                     "buffer_load_dwordx4 %1, %3, %4 offen offset:2048 sc0 sc1 lds\n\t"
                     "buffer_load_dwordx4 %2, %3, %4 offen offset:3072 sc0 sc1 lds"
                     :: "s"(r.m0v), "v"(vo), "v"(vl), "s"(rp), "s"(r.src) : "memory");
    };
    u32 vtail = (u32)(size_t)lds + LDS_TAIL, vone = 1;
    asm volatile("" : "+v"(vtail), "+v"(vone));
    auto enqueue = [&](u32 idx, u32 dst, u32 n) {
        asm volatile("s_mov_b64 exec, 1\n\tds_add_rtn_u32 v162, %0, %1\n\tv_mov_b32 v159, %2\n\tv_mov_b32 v160, %3\n\tv_mov_b32 v161, %4\n\t"
                     "s_waitcnt lgkmcnt(0)\n\tv_add_u32 v158, 1, v162\n\tv_and_b32 v162, %6, v162\n\tv_lshl_add_u32 v162, v162, 4, %5\n\t"
                     "ds_write_b128 v162, v[158:161]\n\ts_mov_b64 exec, -1"
                     :: "v"(vtail), "v"(vone), "s"(idx), "s"(dst), "s"(n << 8 | ld << 4 | peer), "s"(rfl((u32)(size_t)lds) + LDS_CTL), "n"(QLEN - 1)
                     : "memory", "v158", "v159", "v160", "v161", "v162");
    };
    __builtin_amdgcn_s_setprio(3);
    int e = SKIP + wv, e1 = e + WL, e2 = e + 2 * WL;
    const int e3 = e + 3 * WL;
    Item nx = decode(e1), ia = decode(e);
    bool bad = spin_ge<2>(rdy + flag_of(ia.idx) * 4, call, ws + OFF_TMO, call);
    STAMP(5);
    issue(prep(ia, 0));
    bool pre = ready(flag_of(nx.idx));
    u32 n = 0;
    auto wait_free = [&](u32 m) { while (lds_get(done + (m % RING) * 4) + RING <= m) __builtin_amdgcn_s_sleep(1); };
    auto body = [&](auto odd) {
        if constexpr (odd.value) asm volatile("v_mov_b32 v164, 1\n\ts_nop 0\n\tbuffer_atomic_add v164, %0, %1, 0 offen sc0" :: "v"(head), "s"(r0) : "memory", "v164");
        else                     asm volatile("v_mov_b32 v165, 1\n\ts_nop 0\n\tbuffer_atomic_add v165, %0, %1, 0 offen sc0" :: "v"(head), "s"(r0) : "memory", "v165");
        table(e2);
        u32 dn;
        asm volatile("ds_read_b32 v162, %1\n\ts_waitcnt lgkmcnt(0)\n\tv_readfirstlane_b32 %0, v162"
                     : "=s"(dn) : "v"((u32)(size_t)done + ((n + 1) % RING) * 4) : "memory", "v162");
        const Item nn = entry();
        bool pre2 = e2 < NT;
        if (pre2 && !(nowait && nn.idx >= 8192)) pre2 = ready(flag_of(nn.idx));
        if (!pre && e1 < NT) pre = seen(flag_of(nx.idx));
        if (dn + RING <= n + 1) wait_free(n + 1);
        const Prep px = prep(nx, n + 1);
        u32 told;
        if constexpr (odd.value) asm volatile("s_waitcnt vmcnt(1)\n\tv_readfirstlane_b32 %0, v165" : "=s"(told) :: "memory");
        else                     asm volatile("s_waitcnt vmcnt(1)\n\tv_readfirstlane_b32 %0, v164" : "=s"(told) :: "memory");
        if (pre) issue(px);
        enqueue(ia.idx | (bad ? IDX_BAD : 0), ia.dst, n);
        if (!pre && e1 < NT) {
            bad |= spin_ge<2>(rdy + flag_of(nx.idx) * 4, call, ws + OFF_TMO, call);
            issue(px);
        }
        STAMP_IF(n == 0, 6);
        STAMP_IF(CROSS ? (e < NP && e1 >= NP) : (e < NBLK * FI1(slot) && e1 >= NBLK * FI1(slot)), 1);
        STAMP_IF(CROSS && e < NX && e1 >= NX, 2);
        STAMP_IF(CROSS && e < NX + NO1 && e1 >= NX + NO1, 3);
        ia = nx; nx = nn; pre = pre2; ++n;
        e = e1; e1 = e2; e2 = n == 1 ? e3 : (int)(told + tbase);
    };
    for (;;) {
        body(std::false_type{});
        if (e >= NT) break;
        body(std::true_type{});
        if (e >= NT) break;
    }
    u32 fin;
    asm volatile("ds_add_rtn_u32 v162, %1, %2\n\ts_waitcnt lgkmcnt(0)\n\tv_readfirstlane_b32 %0, v162"
                 : "=s"(fin) : "v"((u32)(size_t)lds + LDS_TAIL + 4), "v"(lane == 0 ? 1u : 0u) : "memory", "v162");
    if (fin + 1 == (u32)nld)
        for (int k = 0; k < NDEC; ++k) enqueue(IDX_END, 0, 0);
}

__device__ __forceinline__ void decode_role(const Params& p, LAS char* lds, u32 me, int lane, u32 call) {
    const int rank = rfl(p.rank);
    const int j = rank & 3;
    const u32 par = OFF_PAR + (call & 1) * PAR_BYTES;
    const rsrc_t r0 = mk_rsrc(p.ws[rank], WS_NEED), ro = mk_rsrc(p.out[rank], OUT_BYTES);
    const u32 vo = lane * 16, vout = (lane >> 4) * 4 * 4096 + (lane & 15) * 16;
    LAS char* scr = lds + LDS_SCR + me * SCR;
    u32 q0, q1, q2, q3;
    auto peek = [&](u32 n) {
        asm volatile("ds_read_b128 v[158:161], %4\n\ts_waitcnt lgkmcnt(0)\n\tv_readfirstlane_b32 %0, v158\n\tv_readfirstlane_b32 %1, v159\n\t"
                     "v_readfirstlane_b32 %2, v160\n\tv_readfirstlane_b32 %3, v161"
                     : "=s"(q0), "=s"(q1), "=s"(q2), "=s"(q3) : "v"((u32)(size_t)lds + LDS_CTL + (n % QLEN) * 16) : "memory", "v158", "v159", "v160", "v161");
        return q0 == n + 1;
    };
    const u32 vsig = lane == 0 ? 0u : 0x80000000u;
    u32 one = 1;
    asm volatile("" : "+v"(one));
    auto signal = [&](u32 c) {
        asm volatile("s_nop 4\n\tbuffer_atomic_add %0, %1, %2, %3 offen sc1" :: "v"(one), "v"(vsig), "s"(r0), "s"(rfl(c)) : "memory");
    };
    const u32x4 nan4 = {0x7fc07fc0u, 0x7fc07fc0u, 0x7fc07fc0u, 0x7fc07fc0u};
    u32 vhead = (u32)(size_t)lds + LDS_TAIL + 8, vone = 1;
    asm volatile("" : "+v"(vhead), "+v"(vone));
    for (;;) {
        u32 n;
        asm volatile("s_mov_b64 exec, 1\n\tds_add_rtn_u32 v158, %1, %2\n\ts_waitcnt lgkmcnt(0)\n\tv_readfirstlane_b32 %0, v158\n\ts_mov_b64 exec, -1"
                     : "=s"(n) : "v"(vhead), "v"(vone) : "memory", "v158");
        while (!peek(n)) __builtin_amdgcn_s_sleep(P_DSLEEP);
        if (q1 == IDX_END) break;
        const u32 dst = q2, item = q3 >> 8, ring = (((q3 >> 4) & 3) * RING + item % RING), peer = q3 & 7;
        const LAS char* sl = lds + ring * 4096;
        const u32x4 c0 = *(const LAS u32x4*)(sl + vo), c1 = *(const LAS u32x4*)(sl + 1024 + vo), c2 = *(const LAS u32x4*)(sl + 2048 + vo);
        const u32 k1 = *(const LAS u32*)(sl + 3072 + lane * 4);
        const u32 hx = *(const LAS unsigned char*)(sl + 3328);
        const bool bad = (q1 & IDX_BAD) != 0;
        const u32 idx = q1 & 0xffff;
        const bool out = idx >= 8192, isw = !out && (idx & 64);
        const u32 step = (out || isw) ? 4096 : 1024;
        u32x4 h[4];
        bool done = false;
        if (!bad && !out) {
            const u32 hd = rfl(hx);
            if (hd != 255) {
                if (hd & 64) {
                    u32 t;
                    asm volatile("s_nop 4\n\tbuffer_load_dword %0, %1, %2, %3 offen sc0 sc1\n\ts_waitcnt vmcnt(0)"
                                 : "=&v"(t) : "v"(lane < 16 ? lane * 4 : 0x80000000u), "s"(mk_rsrc(p.ws[peer], WS_NEED)),
                                   "s"(rfl(par + REL_PK + idx * 4096 + 3328 + VAR_X)) : "memory");
                    if (lane < 16) *(LAS u32*)(lds + ring * 4096 + 3328 + VAR_X + lane * 4) = t;
                    asm volatile("" ::: "memory");
                }
                unpack_item(h, c0, c1, c2, k1, sl + 3328, hd, scr, lane);
                done = true;
            }
        } else if (!bad) {
            const u32 g1 = *(const LAS unsigned short*)(sl + 3328 + lane * 2), hd = rfl(*(const LAS unsigned short*)(sl + 3456));
            if ((hd & 255) != 255) { unpack_out(h, c0, c1, c2, k1, g1, (const LAS unsigned char*)(sl + 3456), hd & 255, hd >> 8, scr, lane); done = true; }
        }
        if (!done) {
            const u32 blk = idx >> 7, s = idx & 127;
            const u32 raw = par + (out ? REL_PK + STG_BYTES + (idx + j * 1024) * 4096
                                       : isw ? REL_B + (blk * CF + (s & 60)) * 4096 + (s & 3) * 1024 : (blk * RS1 + s) * 4096);
            const u32 rs = isw ? 4096 : 1024;
            const rsrc_t rp = mk_rsrc(p.ws[peer], WS_NEED);
            asm volatile("s_nop 4\n\tbuffer_load_dwordx4 %0, %4, %5, %6 offen sc0 sc1\n\tbuffer_load_dwordx4 %1, %4, %5, %7 offen sc0 sc1\n\t"
                         "buffer_load_dwordx4 %2, %4, %5, %8 offen sc0 sc1\n\tbuffer_load_dwordx4 %3, %4, %5, %9 offen sc0 sc1\n\ts_waitcnt vmcnt(0)"
                         : "=&v"(h[0]), "=&v"(h[1]), "=&v"(h[2]), "=&v"(h[3])
                         : "v"(vo), "s"(rp), "s"(rfl(raw)), "s"(rfl(raw + rs)), "s"(rfl(raw + 2 * rs)), "s"(rfl(raw + 3 * rs)) : "memory");
            if (bad) { h[0] = nan4; h[1] = nan4; h[2] = nan4; h[3] = nan4; }
        }
        lds_put(lds + LDS_DONE + ring * 4, item + 1);
        const rsrc_t rd = out ? ro : r0;
        const u32 vd = (out ? vout : vo + par) + dst;
#pragma unroll
        for (int i = 0; i < 4; ++i) st_sys(h[i], rd, vd, i * step);
        if (!out) {
            asm volatile("s_waitcnt vmcnt(0)" ::: "memory");
            signal(OFF_CNT + (idx >> 7) * CNT_BLK + peer * 128 + (isw ? 1024 : dst >= REL_A2 ? 2048 : 0));
        }
    }
    STAMP(4);
}

#define GLDS2(m0v, voff, rs, soff, imm0, imm1) \
    asm volatile("s_mov_b32 m0, %0\n\ts_nop 0\n\tbuffer_load_dwordx4 %1, %2, %3 offen offset:" #imm0 " sc0 sc1 lds\n\t" \
                 "buffer_load_dwordx4 %1, %2, %3 offen offset:" #imm1 " sc0 sc1 lds" \
                 :: "s"(m0v), "v"(voff), "s"(rs), "s"(soff) : "memory")
#define BARRIER() do { __builtin_amdgcn_sched_barrier(0); __builtin_amdgcn_s_barrier(); __builtin_amdgcn_sched_barrier(0); } while (0)

#define SCHED() __builtin_amdgcn_sched_barrier(0)
#define WAIT_LGKM0() __builtin_amdgcn_s_waitcnt(0xC07F)

template <int STAGE>
struct Gemm {
    static constexpr int NT = STAGE == 1 ? 128 : 64;
    static constexpr int NA = STAGE == 1 ? 4 : 2;
    static constexpr int NB = STAGE == 1 ? 4 : 8;
    static constexpr int NM = STAGE == 1 ? 4 : 1;
    static constexpr int NRD = NA + NB;
    struct Frags { f16x8 a[NA], b[NB]; };

    rsrc_t rs;
    u32 sa, sa2, sb;
    u32 ablk;
    u32 ldsw;
    int wid, lane;

    template <int I> __device__ __forceinline__ void piece(int t, int buf) const {
        const u32 vo = lane * 16;
        const u32 ma = ldsw + buf * BUF_BYTES + wid * 4096, mb = ldsw + buf * BUF_BYTES + 32768 + wid * 2048;
        const u32 ka = STAGE == 1 ? (u32)(t >> 1) * (RS1 * 4096) + (t & 1) * 2048 : (u32)t * ablk;
        const u32 a = sa + ka, a2 = sa2 + ka;
        const u32 b = STAGE == 1 ? sb + (u32)(t >> 1) * (CF * 4096) + (t & 1) * 2048 : sb + (u32)t * (CF * 4096);
        if (I == 0) GLDS2(ma, vo, rs, a, 0, 1024);
        if (I == 1) GLDS2(ma, vo, rs, a2, 2048, 3072);
        if (I == 2) GLDS2(mb, vo, rs, b, 0, 1024);
    }
    __device__ __forceinline__ void stage(int t, int buf) const { piece<0>(t, buf); piece<1>(t, buf); piece<2>(t, buf); }

    template <int I> __device__ __forceinline__ void rd(Frags& f, const LAS char* lds, int buf, int half) const {
        const LAS char* base = lds + buf * BUF_BYTES + lane * 16;
        if constexpr (STAGE == 1) {
            const LAS char* pa = base + (wid >> 1) * 8192 + half * 1024;
            const LAS char* pb = base + 32768 + (wid & 1) * 8192 + half * 1024;
            if constexpr (I == 0) f.a[0] = *(const LAS f16x8*)pa;
            else if constexpr (I <= 4) f.b[I - 1] = *(const LAS f16x8*)(pb + (I - 1) * 2048);
            else f.a[I - 4] = *(const LAS f16x8*)(pa + (I - 4) * 2048);
        } else {
            const LAS char* pa = base + wid * 4096 + half * 2048;
            const LAS char* pb = base + 32768 + half * 2048;
            constexpr int h = I / 5, n = I % 5;
            if constexpr (n == 0) f.a[h] = *(const LAS f16x8*)(pa + h * 1024);
            else f.b[h * 4 + n - 1] = *(const LAS f16x8*)(pb + (n - 1) * 4096 + h * 1024);
        }
    }
    template <int I> __device__ __forceinline__ void mm(f32x4 (&acc)[NM][4], const Frags& f) const {
        if constexpr (STAGE == 1) acc[I >> 2][I & 3] = __builtin_amdgcn_mfma_f32_16x16x32_f16(f.a[I >> 2], f.b[I & 3], acc[I >> 2][I & 3], 0, 0, 0);
        else            acc[0][I & 3] = __builtin_amdgcn_mfma_f32_16x16x32_f16(f.a[I >> 2], f.b[I], acc[0][I & 3], 0, 0, 0);
    }
    template <int I, int N> __device__ __forceinline__ void rds(Frags& f, const LAS char* lds, int buf, int half) const {
        if constexpr (I < N) { rd<I>(f, lds, buf, half); rds<I + 1, N>(f, lds, buf, half); }
    }
    static __device__ __forceinline__ int blk_of(int t) { return STAGE == 1 ? (t >> 1) : t; }

    template <bool RD, int ST>
    __device__ __forceinline__ void half_step(f32x4 (&acc)[NM][4], const Frags& cur, Frags& nxt, const LAS char* lds, int rbuf, int rhalf,
                                              int ts, int bs, u32 cvo, u32 fso) const {
        auto first = [&] {
            if (ST == 1) asm volatile("buffer_load_dword v166, %0, %1, %2 offen sc0 sc1" :: "v"(cvo), "s"(rs), "s"(fso) : "memory", "v166");
            if (ST) { piece<0>(ts, bs); SCHED(); }
        };
        if constexpr (STAGE == 1) {
            mm<0>(acc, cur);  mm<1>(acc, cur);  if (RD) rds<0, 2>(nxt, lds, rbuf, rhalf); SCHED();
            mm<2>(acc, cur);  mm<3>(acc, cur);  if (RD) rds<2, 4>(nxt, lds, rbuf, rhalf); SCHED();
            mm<4>(acc, cur);  mm<5>(acc, cur);  if (RD) rds<4, 6>(nxt, lds, rbuf, rhalf); SCHED();
            mm<6>(acc, cur);  mm<7>(acc, cur);  if (RD) rds<6, 8>(nxt, lds, rbuf, rhalf); SCHED();
            mm<8>(acc, cur);  mm<9>(acc, cur);  SCHED();
            first();
            mm<10>(acc, cur); mm<11>(acc, cur); SCHED();
            if (ST) { piece<1>(ts, bs); SCHED(); }
            mm<12>(acc, cur); mm<13>(acc, cur); SCHED();
            if (ST) { piece<2>(ts, bs); SCHED(); }
            mm<14>(acc, cur); mm<15>(acc, cur); SCHED();
        } else {
            first();
            mm<0>(acc, cur); SCHED();
            if (ST) { piece<1>(ts, bs); SCHED(); }
            mm<1>(acc, cur); SCHED();
            if (ST) { piece<2>(ts, bs); SCHED(); }
            mm<2>(acc, cur); if (RD) rds<0, 3>(nxt, lds, rbuf, rhalf); SCHED();
            mm<3>(acc, cur); if (RD) rds<3, 5>(nxt, lds, rbuf, rhalf); SCHED();
            mm<4>(acc, cur); if (RD) rds<5, 8>(nxt, lds, rbuf, rhalf); SCHED();
            mm<5>(acc, cur); if (RD) rds<8, 10>(nxt, lds, rbuf, rhalf); SCHED();
            mm<6>(acc, cur); mm<7>(acc, cur); SCHED();
        }
    }

    template <int ROLE>
    __device__ __forceinline__ u32 run(f32x4 (&acc)[NM][4], const LAS char* lds, char* ws, u32 cvo, u32 need, u32 call) const {
        constexpr bool LATE = ROLE == 2 && STAGE == 1;
        char* tmo = ws + OFF_TMO;
        Frags f0, f1;
        if (ROLE == 0) spin_ge(ws + __shfl(cvo, lane & 15) + blk_of(lane < 48 ? lane >> 4 : 0) * CNT_BLK, __shfl(need, lane & 15), tmo, call);
        BARRIER();
        stage(0, 0);
        stage(1, 1);
        if (!LATE) { stage(2, 2); asm volatile("s_waitcnt vmcnt(12)" ::: "memory"); }
        else asm volatile("s_waitcnt vmcnt(6)" ::: "memory");
        BARRIER();
        rds<0, NRD>(f0, lds, 0, 0);
        WAIT_LGKM0();
        if (ROLE == 0) asm volatile("v_mov_b32 v166, 0" ::: "v166");
        int buf = 0;
        for (int t = 0; t < NT - 3; ++t) {
            const int nb = buf == 2 ? 0 : buf + 1, pb = buf == 0 ? 2 : buf - 1;
            half_step<true, LATE ? 2 : 0>(acc, f0, f1, lds, buf, 1, t + 2, pb, 0, 0);
            if (ROLE == 0) {
                u64 miss;
                asm volatile("s_waitcnt vmcnt(6)\n\tv_cmp_lt_u32_e64 %0, v166, %1\n\ts_nop 0" : "=s"(miss) : "v"(need) : "memory");
                WAIT_LGKM0();
                if (miss != 0) spin_ge(ws + cvo + blk_of(t + 3) * CNT_BLK, need, tmo, call);
            } else {
                asm volatile("s_waitcnt vmcnt(6)" ::: "memory");
                WAIT_LGKM0();
            }
            BARRIER();
            half_step<true, LATE ? 0 : (ROLE == 0 ? 1 : 2)>(acc, f1, f0, lds, nb, 0, t + 3, buf, cvo, blk_of(t + 4 < NT ? t + 4 : t + 3) * CNT_BLK);
            buf = nb;
        }
#pragma unroll
        for (int t = NT - 3; t < NT; ++t) {
            const int nb = buf == 2 ? 0 : buf + 1, pb = buf == 0 ? 2 : buf - 1;
            if (t == NT - 3) {
                half_step<true, LATE ? 2 : 0>(acc, f0, f1, lds, buf, 1, t + 2, pb, 0, 0);
                asm volatile("s_waitcnt vmcnt(6)" ::: "memory");
            } else {
                if (t == NT - 2) {
                    const u32 to = OFF_TMO;
                    asm volatile("buffer_load_dword v167, %0, %1, 0 offen sc1" :: "v"(to), "s"(rs) : "memory", "v167");
                }
                half_step<true, 0>(acc, f0, f1, lds, buf, 1, 0, 0, 0, 0);
                asm volatile("s_waitcnt vmcnt(0)" ::: "memory");
            }
            WAIT_LGKM0();
            BARRIER();
            if (t < NT - 1) half_step<true, 0>(acc, f1, f0, lds, nb, 0, 0, 0, 0, 0);
            else            half_step<false, 0>(acc, f1, f0, lds, nb, 0, 0, 0, 0, 0);
            buf = nb;
        }
        u32 tw;
        asm volatile("v_readfirstlane_b32 %0, v167\n\ts_nop 1" : "=s"(tw) :: "memory");
        return tw;
    }
};

__device__ __forceinline__ void activate(u32x4 (&o)[4], f32x4 (&c)[4], bool bad) {
    if (bad) {
        asm volatile("");
#pragma unroll
        for (int n = 0; n < 4; ++n) c[n] = __builtin_bit_cast(f32x4, (u32x4){0x7fc00000u, 0x7fc00000u, 0x7fc00000u, 0x7fc00000u});
    }
#pragma unroll
    for (int n = 0; n < 4; ++n)
#pragma unroll
        for (int r = 0; r < 4; r += 2) {
            const u32x2v y = __builtin_bit_cast(u32x2v, gelu2((f32x2){c[n][r], c[n][r + 1]}));
            o[r][n] = y[0]; o[r + 1][n] = y[1];
        }
}

__device__ __forceinline__ void gemm_role(const Params& p, const LAS char* lds, int xcd, int s, int wid, int lane, u32 call) {
    const int rank = rfl(p.rank);
    const int g = rank >> 2, j = rank & 3, og = (g ^ 1) * 4;
    char* ws = p.ws[rank];
    const u32 par = OFF_PAR + (call & 1) * PAR_BYTES;
    const rsrc_t rs = mk_rsrc(ws, WS_NEED), ro = mk_rsrc(p.out[rank], OUT_BYTES);
    const u32 ldsw = rfl((u32)(size_t)lds);
    const u32 vout = (lane >> 4) * 4 * 4096 + (lane & 15) * 16;
    {
        const int u = (xcd & 1) * (xcd < 4 ? 26 : 24) + s;
        const bool extra = xcd >= 4 && s >= 24;
        const int rt = extra ? 13 : u >> 2, ct = extra ? (xcd - 4) * 2 + (s & 1) : (xcd >> 1) * 4 + (u & 3);
        const int cq = ct < 8 ? -1 : (ct - 8) >> 1;
        Gemm<1> G;
        G.rs = rs; G.ldsw = ldsw; G.wid = wid; G.lane = lane;
        G.sa = par + max(slot_of(rt * 16 + wid * 2, cq), 0) * 4096;
        G.sa2 = par + max(slot_of(rt * 16 + wid * 2 + 1, cq), 0) * 4096 - 2048;
        G.sb = par + REL_B + (ct * 8 + wid) * 4096;
        const int bsrc = ct < 8 ? rank : og + cq;
        u32 cvo = OFF_CNT + 1024 + bsrc * 128, need = (ct < 8 ? 8u : 16u) * call;
        const int sl16 = max(slot_of(rt * 16 + (lane & 15), cq), 0), src16 = row_of_slot(sl16, g, j) / NS;
        u32 mask = 0;
        for (int i = 0; i < 8; ++i) mask |= (__builtin_amdgcn_ballot_w64(src16 == i) != 0) << i;
        if (lane < 8 && ((mask >> lane) & 1)) {
            cvo = OFF_CNT + lane * 128;
            need = (lane == rank ? 8 : ((lane >> 2) == g ? FI1((lane - j - 1) & 3) : FX)) * call;
        }
        f32x4 acc[4][4] = {};
        const bool bad = call == (wid == 0 ? G.template run<0>(acc, lds, ws, cvo, need, call)
                                  : wid < 4 ? G.template run<1>(acc, lds, ws, cvo, need, call)
                                            : G.template run<2>(acc, lds, ws, cvo, need, call));
        STAMP(1);
        const int wm = wid >> 1, wn = wid & 1;
        if (ct < 8) {
#pragma unroll
            for (int m = 0; m < 4; ++m) {
                const int sl = slot_of(rt * 16 + wm * 4 + m, -1);
                if (sl < 0) continue;
                const u32 ob = (u32)row_of_slot(sl, g, j) * 4096 + (ct * 128 + wn * 64) * 4;
                u32x4 o[4];
                activate(o, acc[m], bad);
#pragma unroll
                for (int r = 0; r < 4; ++r) __builtin_amdgcn_raw_buffer_store_b128(o[r], ro, vout + ob + r * 4096, 0, OUT_AUX);
            }
        } else {
            const int fo = (rt * 2 + (ct & 1)) * 8 + wid;
            const u32 sb = par + REL_STG + (u32)(((ct - 8) >> 1) * 1024 + fo * 4) * 4096;
            u32 tp = 0, tn = 0;
#pragma unroll
            for (int m = 0; m < 4; ++m) {
                u32x4 o[4];
                activate(o, acc[m], bad);
                if (m == 0) out_windows(o, tp, tn);
                pack_out(o, tp, tn, (LAS char*)lds + wid * 2048, rs, sb + m * 4096, lane);
            }
            WAIT_VM0();
            if (lane == 0) st_flag(p.ws[bsrc] + OFF_RDY + rank * 8192 + (1024 + fo) * 4, call);
        }
        STAMP(2);
    }
    {
        const bool loc = s >= 24;
        const int rt = ((s >> 2) + 5) % 6, ct = loc ? xcd * 2 + (s & 1) : ((~xcd & 1) * 16) + (xcd >> 1) * 4 + (s & 3);
        Gemm<2> G;
        G.rs = rs; G.ldsw = ldsw; G.wid = wid; G.lane = lane;
        const bool mine = loc || rt * 8 >= RF2;
        G.sa = G.sa2 = par + (mine ? ((loc ? FI : FO1) + wid) * 4096 : REL_A2 + (rt * 8 + wid) * 4096);
        G.ablk = (mine ? RS1 : RF2) * 4096;
        G.sb = par + REL_B + ct * 4 * 4096 + wid * 2048;
        const int bsrc = ct < 16 ? rank : og + ((ct - 16) >> 2);
        u32 cvo = OFF_CNT + 1024 + bsrc * 128, need = (ct < 16 ? 8u : 16u) * call;
        if (mine) { if (lane == 0) { cvo = OFF_CNT + rank * 128; need = 8 * call; } }
        else if (lane < 3 && rt * 8 < OFF2(lane + 1) && rt * 8 + 8 > OFF2(lane)) {
            cvo = OFF_CNT + 2048 + (g * 4 + ((j + 1 + lane) & 3)) * 128; need = F2(lane) * call;
        }
        f32x4 acc[1][4] = {};
        const bool bad = call == (wid == 0 ? G.template run<0>(acc, lds, ws, cvo, need, call)
                                  : wid < 4 ? G.template run<1>(acc, lds, ws, cvo, need, call)
                                            : G.template run<2>(acc, lds, ws, cvo, need, call));
        STAMP(3);
        u32x4 o[4];
        activate(o, acc[0], bad);
        if (ct < 16) {
            const u32 ob = (u32)(loc ? rank * NS + (FI + wid) * 16 : row_of_f2(rt * 8 + wid, g, j)) * 4096 + ct * 256;
#pragma unroll
            for (int r = 0; r < 4; ++r) __builtin_amdgcn_raw_buffer_store_b128(o[r], ro, vout + ob + r * 4096, 0, OUT_AUX);
        } else {
            const int z = ((rt * 4 + (ct & 3)) * 8 + wid + FO2 * 4) % 192;
            const u32 sb = par + REL_STG + (u32)(((ct - 16) >> 2) * 1024 + 768 + z) * 4096;
            u32 tp, tn;
            out_windows(o, tp, tn);
            pack_out(o, tp, tn, (LAS char*)lds + wid * 2048, rs, sb, lane);
            WAIT_VM0();
            if (lane == 0) st_flag(p.ws[bsrc] + OFF_RDY + rank * 8192 + (1024 + 192 + z) * 4, call);
        }
        STAMP(4);
    }
}

__global__ __launch_bounds__(BLOCK, 3) void ag_gemm_kernel(Params p) {
    __shared__ __attribute__((aligned(1024))) char smem[LDS_BYTES];
    const int rank = rfl(p.rank);
    const int lane = threadIdx.x & 63;
    const int wid = rfl(threadIdx.x >> 6);
    const int b = blockIdx.x;
    if (wid >= (b < NCOMM ? NLW + NDEC : 8)) return;
    STAMP(0);
    char* seq = p.ws[rank] + OFF_SEQ + (b * 16 + wid) * 4;
    const u32 prev = ld_agent(seq);
    char* wsl = p.ws[lane & 7];
    auto next_call = [&] {
        const u32 call = rfl(prev) + 1;
        if (lane == 0) __hip_atomic_store((GAS u32*)seq, call, __ATOMIC_RELAXED, __HIP_MEMORY_SCOPE_AGENT);
        STAMP(7);
        return call;
    };
    if (b >= NCOMM) {
        for (int i = ((b - NCOMM) >> 4) - 1; i > 0; --i) __builtin_amdgcn_s_sleep(P_GRADE);
        f32x4 v[8];
        load_item(v, p, b - NCOMM, wid, lane);
        const u32 call = next_call();
        convert_local(p, (LAS char*)smem, v, wsl, b - NCOMM, wid, lane, call);
        STAMP(5);
        gemm_role(p, (const LAS char*)smem, b & 7, (b - NCOMM) >> 3, wid, lane, call);
        return;
    }
    const int nld = 2 * NCOMM + b < 7 * WL ? 3 : 2;
    constexpr int NSET = NLW + NDEC - NPW;
    const int sw = (NLW + NDEC - 1 - wid) * NCOMM + b;
    if (wid < NSET) {
        if (wid < 4) *(LAS u32*)(smem + LDS_CTL + (wid * 64 + lane) * 4) = 0;
        for (int k = 0; k < nld; ++k)
            build_table((LAS char*)smem + LDS_TAB + k * TAB_BYTES, rank, (k * NCOMM + b) % 7, threadIdx.x, NSET * 64);
    }
    const u32 call = next_call();
    if (wid >= NSET && sw < NPUSH) push_role(p, wsl, sw, lane, call);
    else asm volatile("s_waitcnt lgkmcnt(0)\n\ts_barrier" ::: "memory");
    STAMPB_IF(wid == 0, 4);
    if (wid >= NLW) decode_role(p, (LAS char*)smem, wid - NLW, lane, call);
    else if (wid < nld) {
        const int ps = rfl((wid * NCOMM + b) % 7), wv = rfl((wid * NCOMM + b) / 7);
        if (ps < 3) load_role<false>(p, (LAS char*)smem, wid, nld, ps, wv, lane, call);
        else        load_role<true>(p, (LAS char*)smem, wid, nld, ps, wv, lane, call);
    }
}

}

constexpr int g_mode = 0;

extern "C" void kernel_launch(int rank, int nranks, ncclComm_t comm, hipStream_t stream,
                              void* const* d_in, const int* in_sizes, int n_in,
                              void* d_out, int out_size, const int* in_dtypes, int out_dtype,
                              void* const* d_out_all, void* const* d_ws_all, size_t ws_bytes) {
    (void)comm; (void)in_sizes; (void)n_in; (void)d_out; (void)out_size; (void)in_dtypes; (void)out_dtype;
    if (nranks != NR || (unsigned)rank >= (unsigned)NR || ws_bytes < WS_NEED) {
        fprintf(stderr, "unsupported configuration\n");
        return;
    }
    Params p;
    p.x = (const float*)d_in[0];
    p.w = (const float*)d_in[1];
    for (int i = 0; i < NR; ++i) { p.ws[i] = (char*)d_ws_all[i]; p.out[i] = (char*)d_out_all[i]; }
    p.rank = rank;
    p.pad = g_mode;
    ag_gemm_kernel<<<GRID, BLOCK, 0, stream>>>(p);
    hipError_t e = hipGetLastError();
    if (e != hipSuccess) fprintf(stderr, "launch failed: %s\n", hipGetErrorString(e));
}
```

```cpp
#include <hip/hip_runtime.h>
#include <rccl/rccl.h>
#include <cstdio>
#include <cstdint>
#include <type_traits>

typedef unsigned u32;
typedef unsigned long long u64;
typedef u32 u32x4 __attribute__((ext_vector_type(4)));
typedef u32 u32x2v __attribute__((ext_vector_type(2)));
typedef float f32x4 __attribute__((ext_vector_type(4)));
typedef _Float16 f16x8 __attribute__((ext_vector_type(8)));
typedef __amdgpu_buffer_rsrc_t rsrc_t;
#define LAS __attribute__((address_space(3)))
#define GAS __attribute__((address_space(1)))

namespace {

constexpr int NR = 8;
constexpr int KD = 8192;
constexpr int NS = 1024;
constexpr int NBLK = 64;
#define P_FI 45
constexpr int FI = P_FI;
constexpr int FX = 64 - FI;
#define P_FO2 8
constexpr int FO2 = P_FO2, RF2 = 48 - FO2, FO1 = FI - FO2;
__host__ __device__ constexpr int OFF2(int i) { return (RF2 * i + 2) / 3; }
__host__ __device__ constexpr int F2(int i) { return OFF2(i + 1) - OFF2(i); }
__host__ __device__ constexpr int FI1(int i) { return FI - F2(i); }
__host__ __device__ constexpr int OFF1(int i) { return i * FI - OFF2(i); }
constexpr int NI1 = 3 * FI - RF2, FI1MAX = FI1(2);
constexpr int FL = 8;
constexpr int RF1 = 208, CF = 128;
static_assert(FO1 + NI1 + 4 * FX == RF1 && FX >= 16 && FX - FL <= 16 && (FO2 == 0 || FO2 == 8) && F2(0) >= F2(2), "rows");

#define P_W 19
#define P_XA 15
#define P_PB 1
#define P_G0 1
#define P_GRADE 20
#define P_RB 2
constexpr int PB = P_PB, RB = P_RB;
static_assert(PB + RB <= 4 && RB * 32 <= 4 * P_W, "early flags; early items are among the statically dealt ones");
constexpr int NCOMM = 48;
constexpr int WL = P_W;
constexpr int XA = P_XA;
static_assert(WL >= 14 && WL <= 20 && XA <= FX && XA <= 16, "wave roles");
constexpr int GRID = 256, BLOCK = 768, NGEMM = GRID - NCOMM;
constexpr int RS1 = 64 + NI1 + 4 * FX;

constexpr u32 OFF_SEQ  = 0;
constexpr u32 OFF_TMO  = 16384;
constexpr u32 OFF_QUE  = 16640;
constexpr u32 OFF_RDY  = 32768;
constexpr u32 OFF_CNT  = 131072;
constexpr u32 CNT_BLK  = 24 * 128;
constexpr u32 OFF_DIAG = 524288;
constexpr u32 A1_BYTES = (u32)NBLK * RS1 * 4096;
constexpr u32 A2_BYTES = (u32)NBLK * RF2 * 4096;
constexpr u32 B_BYTES  = (u32)NBLK * CF * 4096;
constexpr u32 STG_BYTES = 4u * 1024 * 4096;
constexpr u32 PK_BYTES = (u32)NBLK * 128 * 4096;
constexpr u32 REL_A2 = A1_BYTES, REL_B = REL_A2 + A2_BYTES, REL_PK = REL_B + B_BYTES, REL_STG = REL_PK + PK_BYTES;
constexpr u32 PAR_BYTES = REL_STG + 2 * STG_BYTES;
constexpr u32 OFF_PAR  = 1u << 20;
constexpr int VAR_XO = 192;
constexpr u32 WS_NEED  = OFF_PAR + 2 * PAR_BYTES;
constexpr u32 OUT_BYTES = (u32)KD * NS * 4;
static_assert(REL_PK < (256u << 20), "table entries hold 18 bits of dst / 1024");
static_assert(256 + VAR_XO == 448, "output items are as long as x items");
static_assert(PK_BYTES == 8192u * 4096u, "staged outputs follow the packed image");
static_assert(OFF_CNT + NBLK * CNT_BLK <= OFF_DIAG && OFF_RDY + 8 * 8192 + 256 <= OFF_CNT, "control area");

constexpr u64 DEADLINE = 40000000ull;

constexpr int BUF_BYTES = 49152;
constexpr int LDS_BYTES = 3 * BUF_BYTES;

struct Params {
    const float* x;
    const float* w;
    char* ws[NR];
    char* out[NR];
    int rank;
    int pad;
};

__device__ __forceinline__ u32 rfl(u32 v) { return __builtin_amdgcn_readfirstlane(v); }
__device__ __forceinline__ char* uni(const void* p) {
    const u64 a = (u64)p;
    return (char*)(((u64)rfl((u32)(a >> 32)) << 32) | rfl((u32)a));
}
__device__ __forceinline__ rsrc_t rsrc_u(const void* p, u32 bytes) {
    return __builtin_amdgcn_make_buffer_rsrc((void*)p, (short)0, (int)bytes, 0x00020000);
}
__device__ __forceinline__ rsrc_t mk_rsrc(const void* p, u32 bytes) { return rsrc_u(uni(p), bytes); }
__device__ __forceinline__ u32 ld_sys(const char* p) {
    return __hip_atomic_load((const GAS u32*)p, __ATOMIC_RELAXED, __HIP_MEMORY_SCOPE_SYSTEM);
}
__device__ __forceinline__ u32 ld_agent(const char* p) {
    return __hip_atomic_load((const GAS u32*)p, __ATOMIC_RELAXED, __HIP_MEMORY_SCOPE_AGENT);
}
__device__ __forceinline__ void st_flag(char* p, u32 v) {
    __hip_atomic_store((GAS u32*)p, v, __ATOMIC_RELAXED, __HIP_MEMORY_SCOPE_SYSTEM);
}
__device__ __forceinline__ void add_sys(char* p) {
    __hip_atomic_fetch_add((GAS u32*)p, 1u, __ATOMIC_RELAXED, __HIP_MEMORY_SCOPE_SYSTEM);
}
#define WAIT_VM0() do { asm volatile("s_waitcnt vmcnt(0)" ::: "memory"); __builtin_amdgcn_s_waitcnt(0x0F70); } while (0)

template <int SLEEP = 2>
__device__ __forceinline__ bool spin_ge(const char* p, u32 need, char* tmo, u32 call) {
    const u64 t0 = __builtin_amdgcn_s_memrealtime();
    bool gave_up = false;
    for (u32 polls = 0;; ++polls) {
        const u32 v = ld_sys(p);
        if (__builtin_amdgcn_ballot_w64(v < need) == 0) break;
        if ((polls & 7) == 0 && (rfl(ld_agent(tmo)) == call || __builtin_amdgcn_s_memrealtime() - t0 > DEADLINE)) {
            __hip_atomic_store((GAS u32*)tmo, call, __ATOMIC_RELAXED, __HIP_MEMORY_SCOPE_AGENT);
            gave_up = true;
            break;
        }
        __builtin_amdgcn_s_sleep(SLEEP);
    }
    asm volatile("" ::: "memory");
    return gave_up;
}

#define STAMP_IF(c, slot) do {} while (0)
#define STAMP(slot) do {} while (0)
#define STAMPB_IF(c, slot) do {} while (0)

__device__ __forceinline__ u32x4 cvt8(f32x4 a, f32x4 b) {
    f16x8 h;
    h[0] = (_Float16)a[0]; h[1] = (_Float16)a[1]; h[2] = (_Float16)a[2]; h[3] = (_Float16)a[3];
    h[4] = (_Float16)b[0]; h[5] = (_Float16)b[1]; h[6] = (_Float16)b[2]; h[7] = (_Float16)b[3];
    return __builtin_bit_cast(u32x4, h);
}
__device__ __forceinline__ void st_sys(u32x4 v, rsrc_t r, u32 voff, u32 soff) {
    __builtin_amdgcn_raw_buffer_store_b128(v, r, voff + soff, 0, 17);
}
typedef float f32x2 __attribute__((ext_vector_type(2)));
__device__ __forceinline__ f32x2 gelu2(f32x2 y) {
    constexpr float A = -2.8853900817779268f * 0.7978845608028654f, B = A * 0.044715f;
    const f32x2 z = y * (y * y * B + A);
    const f32x2 d = (f32x2){__builtin_amdgcn_exp2f(z[0]), __builtin_amdgcn_exp2f(z[1])} + 1.0f;
    return y * (f32x2){__builtin_amdgcn_rcpf(d[0]), __builtin_amdgcn_rcpf(d[1])};
}

__device__ __forceinline__ int row_of_slot(int sl, int g, int j) {
    if (sl < 64) return (g * 4 + j) * NS + sl * 16;
    sl -= 64;
    if (sl < NI1) { const int i = sl >= OFF1(2) ? 2 : sl >= OFF1(1) ? 1 : 0; return (g * 4 + ((j + 1 + i) & 3)) * NS + (sl - OFF1(i)) * 16; }
    sl -= NI1;
    const int q = sl / FX;
    return ((g ^ 1) * 4 + q) * NS + (FI + sl - q * FX) * 16;
}
__device__ __forceinline__ int slot_of(int f, int q) {
    if (q >= 0) {
        if (f >= RF1 - FX) return -1;
        if (f >= RF1 - 4 * FX + q * FX) f += FX;
    } else if (f >= RF1) return f - RF1 < FX - FL ? FI + FL + f - RF1 : -1;
    return f < FO1 ? f : f + 64 - FO1;
}
__device__ __forceinline__ int row_of_f2(int f2, int g, int j) {
    if (f2 >= RF2) return (g * 4 + j) * NS + (FO1 + f2 - RF2) * 16;
    const int i = f2 >= OFF2(2) ? 2 : f2 >= OFF2(1) ? 1 : 0;
    return (g * 4 + ((j + 1 + i) & 3)) * NS + (FI1(i) + f2 - OFF2(i)) * 16;
}

constexpr int IN_AUX = 2;
constexpr int OUT_AUX = 17;
__device__ __forceinline__ u32x4 cvt_w(const f32x4* v, int c) {
    return cvt8((f32x4){v[0][c], v[1][c], v[2][c], v[3][c]}, (f32x4){v[4][c], v[5][c], v[6][c], v[7][c]});
}

#define DPP_STEP(op, v, ctrl, rmask) v = op(v, (u32)__builtin_amdgcn_update_dpp(0, (int)v, ctrl, rmask, 0xf, false))
__device__ __forceinline__ u32 umax2(u32 a, u32 b) { return a > b ? a : b; }
__device__ __forceinline__ u32 uadd2(u32 a, u32 b) { return a + b; }
__device__ __forceinline__ u32 wave_max(u32 v) {
    DPP_STEP(umax2, v, 0x111, 0xf); DPP_STEP(umax2, v, 0x112, 0xf); DPP_STEP(umax2, v, 0x114, 0xf); DPP_STEP(umax2, v, 0x118, 0xf);
    DPP_STEP(umax2, v, 0x142, 0xa); DPP_STEP(umax2, v, 0x143, 0xc);
    return __builtin_amdgcn_readlane(v, 63);
}
__device__ __forceinline__ u32 wave_sum(u32 v) {
    DPP_STEP(uadd2, v, 0x111, 0xf); DPP_STEP(uadd2, v, 0x112, 0xf); DPP_STEP(uadd2, v, 0x114, 0xf); DPP_STEP(uadd2, v, 0x118, 0xf);
    DPP_STEP(uadd2, v, 0x142, 0xa); DPP_STEP(uadd2, v, 0x143, 0xc);
    return __builtin_amdgcn_readlane(v, 63);
}

__device__ __forceinline__ u32 wave_scan(u32 v) {
    DPP_STEP(uadd2, v, 0x111, 0xf); DPP_STEP(uadd2, v, 0x112, 0xf); DPP_STEP(uadd2, v, 0x114, 0xf); DPP_STEP(uadd2, v, 0x118, 0xf);
    DPP_STEP(uadd2, v, 0x142, 0xa); DPP_STEP(uadd2, v, 0x143, 0xc);
    return v;
}
__device__ __forceinline__ u32 byte_scan(u32 m) {
    u32 a, r;
    asm("v_lshl_add_u32 %0, %1, 8, %1" : "=v"(a) : "v"(m));
    asm("v_lshl_add_u32 %0, %1, 16, %1" : "=v"(r) : "v"(a));
    return r;
}
__device__ __forceinline__ u32 times7(u32 m) {
    u32 a, r;
    asm("v_lshl_add_u32 %0, %1, 1, %1" : "=v"(a) : "v"(m));
    asm("v_lshl_add_u32 %0, %1, 2, %2" : "=v"(r) : "v"(m), "v"(a));
    return r;
}
__device__ __forceinline__ u32 rank_in(u64 m) { return __builtin_amdgcn_mbcnt_hi((u32)(m >> 32), __builtin_amdgcn_mbcnt_lo((u32)m, 0)); }

constexpr int VAR_X = VAR_XO, VAR_W = 256;
__device__ __forceinline__ u32 h7_of(float a) { return (u32)__builtin_bit_cast(unsigned short, (_Float16)a) >> 8; }
__device__ __forceinline__ void lds_or(LAS char* a, u32 v) {
    __hip_atomic_fetch_or((LAS u32*)a, v, __ATOMIC_RELAXED, __HIP_MEMORY_SCOPE_WORKGROUP);
}
__device__ __forceinline__ void put_bits(LAS char* area, u32 bo, u64 v) {
    const u32 sh = bo & 31;
    const u64 w = v << sh;
    LAS char* a = area + (bo >> 5) * 4;
    lds_or(a, (u32)w); lds_or(a + 4, (u32)(w >> 32)); lds_or(a + 8, (u32)((v >> 1) >> (63 - sh)));
}
__device__ __forceinline__ u64 get_bits(const LAS char* area, u32 bo) {
    const LAS u32* w = (const LAS u32*)(area + (bo >> 5) * 4);
    const u32 w0 = w[0], w1 = w[1], w2 = w[2];
    return (u64)__builtin_amdgcn_alignbit(w2, w1, bo) << 32 | __builtin_amdgcn_alignbit(w1, w0, bo);
}
template <class F>
__device__ __forceinline__ void pack_item(const u32x4 (&h)[4], float amax, LAS char* scr, rsrc_t r0, u32 dst, int lane, F before_stores) {
    u32 L[8], H[8];
#pragma unroll
    for (int t = 0; t < 8; ++t) {
        const u32 d0 = h[t >> 1][(t & 1) * 2], d1 = h[t >> 1][(t & 1) * 2 + 1];
        L[t] = __builtin_amdgcn_perm(d1, d0, 0x06040200u);
        H[t] = __builtin_amdgcn_perm(d1, d0, 0x07050301u);
    }
    *(LAS u32x4*)(scr + lane * 16) = (u32x4){H[0], H[1], H[2], H[3]};
    *(LAS u32x4*)(scr + lane * 16 + 1024) = (u32x4){H[4], H[5], H[6], H[7]};
    LAS char* area = scr + 2048;
    *(LAS u32x2v*)(area + lane * 8) = (u32x2v){0u, 0u};
    const u32 lm = wave_sum(h7_of(amax) & 127) + 132, rare = (lm & 255) < 115 ? 1 : 0;
    const u32 E1 = lm < 256 ? 1 : lm < 31 * 256 ? lm >> 8 : 31;
    const u32 eb = E1 * 0x01010101u | 0x80808080u;
    const u32 tbig = rare ? 0x79797979u : 0x7a7a7a7au, vadd = rare ? 0x01010101u : 0x02020202u, top6 = rare ? 0x06060606u : 0u;
    const u32 clo = 0x02010003u, qlo = rare ? 3u : 2u, qhi = rare ? 0x03020100u : 0x03030100u;
    u32 K[2] = {0, 0}, e4 = 0, pos = 0;
    u64 b2 = 0, b3 = 0;
#pragma unroll
    for (int t = 0; t < 8; ++t) {
        const u32 o = (eb - ((H[t] >> 2) & 0x1f1f1f1fu)) ^ 0x80808080u, o7 = o & 0x7f7f7f7fu;
        const u32 far = (o | (o7 + tbig)) & 0x80808080u, huge = (o | (o7 + 0x73737373u)) & 0x80808080u;
        const u32 f7 = (far >> 4) - (far >> 7), sel = (o | f7) & 0x07070707u;
        const u32 c = __builtin_amdgcn_perm(0x03030303u, clo, sel);
        const u32 q = __builtin_amdgcn_perm(qhi, qlo, sel);
        const u32 M = __builtin_amdgcn_perm(0x01010101u, 0x00000001u, sel);
        const u32 M2 = q & (q >> 1);
        const u32 v = (((o7 + vadd) | ((huge >> 4) - (huge >> 7))) & f7) | __builtin_amdgcn_perm(0u, top6 & 255, sel);
        const u32 P = byte_scan(M), P2 = byte_scan(M2), X2 = P2 - M2;
        const u32 bits2 = __builtin_amdgcn_udot4(q, __builtin_amdgcn_perm(0u, 0x40100401u, P - M), 0u, false);
        const u32 bits3 = __builtin_amdgcn_udot4(v, __builtin_amdgcn_perm(0u, 0x00400801u, X2), (X2 >> 24) == 3 ? (v >> 24) << 9 : 0u, false);
        b2 |= (u64)bits2 << (pos & 63);
        b3 |= (u64)bits3 << ((pos >> 16) & 63);
        pos += (P >> 23) + ((P2 >> 24) * 3 << 16);
        K[t >> 2] |= c << (2 * (t & 3));
        e4 = (e4 >> 1) | huge;
        if (t & 1) { asm volatile("" : "+v"(b2), "+v"(b3), "+v"(e4)); __builtin_amdgcn_sched_barrier(0); }
    }
    const u32 incl = wave_scan(pos), tot = __builtin_amdgcn_readlane(incl, 63), ex = incl - pos;
    const u32 used = 8 + (tot & 0xffff) + (tot >> 16);
    u32 need = used;
    if (__builtin_amdgcn_ballot_w64(e4 != 0) != 0) need += 8 * wave_sum(__builtin_popcount(e4));
    const u32 var = need <= 8 * VAR_X ? VAR_X : VAR_W;
    const bool fits = need <= 8 * VAR_W && __builtin_amdgcn_ballot_w64(pos >= (65u << 16)) == 0;
    if (fits) {
        u32 n4 = 0;
        put_bits(area, 8 + (ex & 0xffff), b2);
        put_bits(area, 8 + (tot & 0xffff) + (ex >> 16), b3);
        while (__builtin_amdgcn_ballot_w64(e4 != 0) != 0) {
            const u64 act = __builtin_amdgcn_ballot_w64(e4 != 0);
            const u32 any = (e4 | (e4 >> 8) | (e4 >> 16) | (e4 >> 24)) & 255, t = __builtin_ctz(any | 128);
            const u32 b = __builtin_ctz(((e4 >> t) & 0x01010101u) | 0x80000000u) + t;
            const u32 e = (*(const LAS unsigned char*)(scr + (t & 4) * 256 + lane * 16 + (t & 3) * 4 + ((b >> 3) & 3)) >> 2) & 31;
            const u32 k = var - 1 - n4 - rank_in(act);
            if (e4 != 0) lds_or(area + (k & ~3u), e << ((k & 3) * 8));
            e4 &= ~(1u << (b & 31));
            n4 += __builtin_popcountll(act);
        }
    }
    asm volatile("" ::: "memory");
    u32 vw = *(const LAS u32*)(area + lane * 4);
    if (lane == 0) vw |= fits ? E1 | rare << 5 | (var == VAR_W ? 64 : 0) : 255;
    const u32 P0 = (H[0] & 0x83838383u) | ((H[1] & 0x03030303u) << 2) | ((H[1] >> 1) & 0x40404040u) | ((H[6] & 0x03030303u) << 4);
    const u32 P1 = (H[2] & 0x83838383u) | ((H[3] & 0x03030303u) << 2) | ((H[3] >> 1) & 0x40404040u) | ((H[7] & 0x03030303u) << 4);
    const u32 P2 = (H[4] & 0x83838383u) | ((H[5] & 0x03030303u) << 2) | ((H[5] >> 1) & 0x40404040u)
                 | ((H[6] >> 2) & 0x20202020u) | ((H[7] >> 3) & 0x10101010u);
    before_stores();
    st_sys((u32x4){L[0], L[1], L[2], L[3]}, r0, lane * 16, dst);
    st_sys((u32x4){L[4], L[5], L[6], L[7]}, r0, lane * 16, dst + 1024);
    st_sys((u32x4){P0, P1, P2, K[0]}, r0, lane * 16, dst + 2048);
    __builtin_amdgcn_raw_buffer_store_b32(K[1], r0, lane * 4 + dst + 3072, 0, 17);
    __builtin_amdgcn_raw_buffer_store_b32(vw, r0, lane * 4 + dst + 3328, 0, 17);
}
__device__ __forceinline__ void unpack_item(u32x4 (&h)[4], const u32x4& c0, const u32x4& c1, const u32x4& c2, u32 k1,
                                            const LAS char* area, u32 hd, LAS char* scr, int lane) {
    const u32 E1 = hd & 31, var = (hd & 64) ? VAR_W : VAR_X;
    const bool rare = hd & 32;
    const u32 K[2] = {c2[3], k1};
    const u32 X[2] = {K[0] & (K[0] >> 1) & 0x55555555u, K[1] & (K[1] >> 1) & 0x55555555u};
    const u32 n2 = 2 * (__builtin_popcount(X[0]) + __builtin_popcount(X[1])), i2 = wave_scan(n2);
    u64 b2 = get_bits(area, 8 + i2 - n2);
    b2 &= n2 < 64 ? ~(~0ull << n2) : ~0ull;
    const u64 y2 = b2 & (b2 >> 1) & 0x5555555555555555ull;
    const u32 n3 = 3 * __builtin_popcountll(y2), i3 = wave_scan(n3);
    u64 b3 = get_bits(area, 8 + __builtin_amdgcn_readlane(i2, 63) + i3 - n3);
    const u32 e4x = E1 * 0x04040404u | 0x80808080u;
    const u32 t1 = (e4x - 0x000c0804u) & 0x007c7c7cu, t2 = (e4x - (rare ? 0x00181410u : 0x00001410u)) & 0x007c7c7cu;
    const u32 t3lo = (e4x - (rare ? 0x2824201cu : 0x24201c18u)) & 0x7c7c7c7cu, t3hi = (e4x - (rare ? 0x0000302cu : 0x00302c28u)) & 0x007c7c7cu;
    LAS char* row = scr + lane * 36;
    u32 e4 = 0;
    const u32 nch = (wave_max(n2) + 7) >> 3;
#pragma unroll
    for (int k = 0; k < 8; ++k) {
        if (k >= 4 && k >= nch) break;
        const u32 b8 = (u32)(b2 >> (8 * k)) & 255, y = b8 | (b8 << 12), q = (y | (y << 6)) & 0x03030303u;
        const u32 M2 = q & (q >> 1), P2 = byte_scan(M2), m7 = times7(M2);
        const u32 lo = (u32)b3, z = (lo & 0x3f) | ((lo & 0xfc0) << 10), f = (z | (z << 5)) & 0x07070707u;
        const u32 v = __builtin_amdgcn_perm(0u, f, P2 - M2) & m7;
        b3 >>= (P2 >> 24) * 3;
        *(LAS u32*)(row + 4 * k) = __builtin_amdgcn_perm(0u, t2, q) | __builtin_amdgcn_perm(t3hi, t3lo, v | (0x0c0c0c0cu - (M2 << 2) - (M2 << 3)));
        e4 |= (((v + M2) >> 3) & 0x01010101u) << k;
    }
    u32 n4 = 0;
    while (__builtin_amdgcn_ballot_w64(e4 != 0) != 0) {
        const u64 act = __builtin_amdgcn_ballot_w64(e4 != 0);
        const u32 any = (e4 | (e4 >> 8) | (e4 >> 16) | (e4 >> 24)) & 255, t = __builtin_ctz(any | 128);
        const u32 b = __builtin_ctz(((e4 >> t) & 0x01010101u) | 0x80000000u) + t;
        const u32 x = *(const LAS unsigned char*)(area + ((var - 1 - n4 - rank_in(act)) & 255));
        if (e4 != 0) *(LAS unsigned char*)(row + t * 4 + (b >> 3)) = x << 2;
        e4 &= ~(1u << (b & 31));
        n4 += __builtin_popcountll(act);
    }
    const u32 SM[8] = {c2[0] & 0x83838383u, ((c2[0] << 1) & 0x80808080u) | ((c2[0] >> 2) & 0x03030303u),
                       c2[1] & 0x83838383u, ((c2[1] << 1) & 0x80808080u) | ((c2[1] >> 2) & 0x03030303u),
                       c2[2] & 0x83838383u, ((c2[2] << 1) & 0x80808080u) | ((c2[2] >> 2) & 0x03030303u),
                       ((c2[2] << 2) & 0x80808080u) | ((c2[0] >> 4) & 0x03030303u),
                       ((c2[2] << 3) & 0x80808080u) | ((c2[1] >> 4) & 0x03030303u)};
    u32 M[8], P[8], g[8], at = 0;
#pragma unroll
    for (int t = 0; t < 8; ++t) {
        M[t] = (X[t >> 2] >> (2 * (t & 3))) & 0x01010101u;
        P[t] = byte_scan(M[t]);
        const LAS u32* w = (const LAS u32*)(row + (at & ~3u));
        g[t] = __builtin_amdgcn_alignbyte(w[1], w[0], at);
        at += P[t] >> 24;
    }
#pragma unroll
    for (int t = 0; t < 8; ++t) {
        const u32 c = (K[t >> 2] >> (2 * (t & 3))) & 0x03030303u;
        const u32 m = times7(M[t]);
        const u32 H = SM[t] | __builtin_amdgcn_perm(t1, g[t], ((P[t] - M[t]) & m) | ((c | 0x04040404u) & ~m));
        const u32 L = t < 4 ? c0[t & 3] : c1[t & 3];
        h[t >> 1][(t & 1) * 2] = __builtin_amdgcn_perm(H, L, 0x05010400u);
        h[t >> 1][(t & 1) * 2 + 1] = __builtin_amdgcn_perm(H, L, 0x07030602u);
    }
}

__device__ __forceinline__ void out_windows(const u32x4 (&o)[4], u32& tp, u32& tn) {
    u32 mn = 0;
    int mp = 0;
#pragma unroll
    for (int i = 0; i < 16; ++i) { mp = max(mp, (int)o[i >> 2][i & 3]); mn = umax2(mn, o[i >> 2][i & 3]); }
    tp = wave_max((u32)mp) >> 24;
    tp -= __builtin_popcountll(__builtin_amdgcn_ballot_w64(mp >= (int)(tp << 24))) <= 6;
    tp = (int)tp > 3 ? tp : 3;
    tn = wave_max(mn) >> 24;
    tn = tn > 130 ? tn : 130;
}
__device__ __forceinline__ void pack_out(const u32x4 (&o)[4], u32 tp, u32 tn, LAS char* scr, rsrc_t r0, u32 dst, int lane) {
    u32 M[12], G2 = 0, G1 = 0, em = 0;
    u32x4 E;
#pragma unroll
    for (int r = 0; r < 4; ++r) {
#pragma unroll
        for (int i = 0; i < 3; ++i) M[r * 3 + i] = __builtin_amdgcn_perm(o[r][3], o[r][i], 0x04020100u + (i << 24));
        E[r] = __builtin_amdgcn_perm(o[r][1], o[r][0], 0x0c0c0703u) | __builtin_amdgcn_perm(o[r][3], o[r][2], 0x07030c0cu);
    }
    *(LAS u32x4*)(scr + lane * 16) = E;
#pragma unroll
    for (int r = 0; r < 4; ++r) {
        const u32 sg = E[r] & 0x80808080u, sm = sg - (sg >> 7);
        const u32 top = ((tn & 127) * 0x01010101u & sm) | (tp * 0x01010101u & ~sm);
        const u32 d = ((top | 0x80808080u) - (E[r] & 0x7f7f7f7fu)) ^ 0x80808080u;
        const u32 out = (d | ((d & 0x7f7f7f7fu) + 0x7c7c7c7cu + (sg >> 7))) & 0x80808080u;
        const u32 g = ((d & 0x03030303u) + 0x01010101u + (sg >> 5)) & ~((out >> 4) - (out >> 7));
        G2 |= (g & 0x03030303u) << (2 * r);
        G1 |= (g & 0x04040404u) >> 2 << r;
        em = (em >> 1) | out;
    }
    LAS unsigned char* list = (LAS unsigned char*)(scr + 1024);
    u32 used = 2;
    for (u64 act = __builtin_amdgcn_ballot_w64(em != 0); act != 0; act = __builtin_amdgcn_ballot_w64(em != 0)) {
        const u32 k = used + rank_in(act), b = __builtin_ctz(em | 0x80000000u);
        list[em != 0 && k < 64 ? k : 64] = *(const LAS unsigned char*)(scr + lane * 16 + (b & 3) * 4 + (b >> 3));
        em &= em - 1;
        used += __builtin_popcountll(act);
        if (used > 64) break;
    }
    *(LAS unsigned short*)list = (used > 64 ? 255 : tp) | tn << 8;
    asm volatile("" ::: "memory");
    const u32 ent = *(const LAS u32*)(list + (lane & 15) * 4);
    if (used > 64) {
#pragma unroll
        for (int r = 0; r < 4; ++r) st_sys(o[r], r0, lane * 16 + r * 1024, dst + STG_BYTES);
    }
#pragma unroll
    for (int i = 0; i < 3; ++i) st_sys((u32x4){M[4 * i], M[4 * i + 1], M[4 * i + 2], M[4 * i + 3]}, r0, lane * 16 + i * 1024, dst);
    __builtin_amdgcn_raw_buffer_store_b32(G2, r0, lane * 4 + dst + 3072, 0, 17);
    __builtin_amdgcn_raw_buffer_store_b16((unsigned short)__builtin_amdgcn_perm(0u, G1 | (G1 >> 4), 0x0c0c0200u), r0, lane * 2 + dst + 3328, 0, 17);
    __builtin_amdgcn_raw_buffer_store_b32(ent, r0, (lane < 16 ? lane * 4 : 0x80000000u) + dst + 3456, 0, 17);
}
__device__ __forceinline__ void unpack_out(u32x4 (&o)[4], const u32x4& c0, const u32x4& c1, const u32x4& c2, u32 G2, u32 g1,
                                           const LAS unsigned char* list, u32 tp, u32 tn, LAS char* scr, int lane) {
    const u32 M[12] = {c0[0], c0[1], c0[2], c0[3], c1[0], c1[1], c1[2], c1[3], c2[0], c2[1], c2[2], c2[3]};
    const u32 t = __builtin_amdgcn_perm(0u, g1, 0x0c010c00u), G1 = t | (t << 4);
    const u32 tlo = tp * 0x01010100u - 0x02010000u, thi = tn * 0x01010100u - 0x02010000u + tp - 3;
    u32x4 E;
    u32 em = 0;
#pragma unroll
    for (int r = 0; r < 4; ++r) {
        const u32 g = ((G2 >> (2 * r)) & 0x03030303u) | ((G1 >> r << 2) & 0x04040404u);
        E[r] = __builtin_amdgcn_perm(thi, tlo, g);
        em = (em >> 1) | (~(g + 0x7f7f7f7fu) & 0x80808080u);
    }
    *(LAS u32x4*)(scr + lane * 16) = E;
    u32 used = 2;
    auto step = [&](u32& a, u32& val) {
        const u64 act = __builtin_amdgcn_ballot_w64(em != 0);
        const u32 b = __builtin_ctz(em | 0x80000000u);
        a = em != 0 ? lane * 16 + (b & 3) * 4 + (b >> 3) : 1024;
        val = list[(used + rank_in(act)) & 63];
        em &= em - 1;
        used += __builtin_popcountll(act);
    };
    do {
        u32 a0, a1, a2, x0, x1, x2;
        step(a0, x0); step(a1, x1); step(a2, x2);
        *(LAS unsigned char*)(scr + a0) = x0; *(LAS unsigned char*)(scr + a1) = x1; *(LAS unsigned char*)(scr + a2) = x2;
    } while (__builtin_amdgcn_ballot_w64(em != 0) != 0);
    E = *(const LAS u32x4*)(scr + lane * 16);
#pragma unroll
    for (int r = 0; r < 4; ++r) {
#pragma unroll
        for (int i = 0; i < 3; ++i) o[r][i] = __builtin_amdgcn_perm(E[r], M[r * 3 + i], 0x04020100u + (i << 24));
        o[r][3] = __builtin_amdgcn_perm(M[r * 3 + 1], M[r * 3], 0x0c0c0703u) | __builtin_amdgcn_perm(E[r], M[r * 3 + 2], 0x07030c0cu);
    }
}

__device__ __forceinline__ void load_item(f32x4 (&v)[8], const Params& p, int u, int wid, int lane) {
    const int blk = u >> 4, it = (u & 15) * 8 + wid;
    const bool isx = it < 64;
    const rsrc_t r = mk_rsrc(isx ? p.x : p.w, OUT_BYTES);
    const u32 vf = isx ? (lane & 15) * 32768 + (lane >> 4) * 16 : (lane >> 4) * 16384 + (lane & 15) * 16;
    const u32 base = isx ? it * 524288 + blk * 512 : (blk * 128 + (wid & 3) * 32) * 4096 + ((it - 64) >> 2) * 256;
    const u32 sa = isx ? 64 : 4096, sb = isx ? 256 : 65536;
#pragma unroll
    for (int i = 0; i < 8; ++i)
        v[i] = __builtin_bit_cast(f32x4, __builtin_amdgcn_raw_buffer_load_b128(r, vf, base + (i & 3) * sa + (i >> 2) * sb, IN_AUX));
}
__device__ __forceinline__ void convert_local(const Params& p, LAS char* lds, f32x4 (&v)[8], char* wsl, int wg, int wid, int lane, u32 call) {
    const int rank = rfl(p.rank);
    constexpr int NU = NBLK * 16;
    const u32 par = OFF_PAR + (call & 1) * PAR_BYTES;
    const rsrc_t r0 = mk_rsrc(p.ws[rank], WS_NEED);
    LAS char* scr = lds + wid * 4096;
    auto publish = [&](int u) {
        if (wid == 0) {
            if (lane < 8 && lane != rank) st_flag(wsl + OFF_RDY + rank * 8192 + u * 4, call);
            if (lane == 8) add_sys(p.ws[rank] + OFF_CNT + (u >> 4) * CNT_BLK + ((u & 8) ? 1024 : 0) + rank * 128);
        }
    };
    int u = wg, un = (wg + NGEMM) ^ 8, up = -1;
    WAIT_VM0();
    for (int r = 0; u < NU; ++r) {
        const int blk = u >> 4, grp = u & 15, it = grp * 8 + wid;
        u32x4 h[4];
        u32 base, step;
        if (grp < 8) {
            base = par + (u32)blk * (RS1 * 4096) + it * 4096; step = 1024;
#pragma unroll
            for (int i = 0; i < 4; ++i) h[i] = cvt8(v[2 * i], v[2 * i + 1]);
        } else {
            base = par + REL_B + (u32)blk * (CF * 4096) + ((it - 64) & 60) * 4096 + (wid & 3) * 1024; step = 4096;
#pragma unroll
            for (int c = 0; c < 4; ++c) h[c] = cvt_w(v, c);
        }
        float amax = 0.0f;
#pragma unroll
        for (int i = 0; i < 8; ++i) amax = fmaxf(fmaxf(fmaxf(amax, fabsf(v[i][0])), fmaxf(fabsf(v[i][1]), fabsf(v[i][2]))), fabsf(v[i][3]));
        STAMPB_IF(r == 0, 0);
        if (r > 0) load_item(v, p, (u32)un < (u32)NU ? un : NU - 1, wid, lane);
        u32 early = 0;
        if (blk < PB + RB) {
            const int xi = blk * FX + it - FI;
            if (it < FI1MAX) early = blk >= PB ? 15u << (rank & 4) : 0;
            else if (it >= 64) early = blk >= PB ? 1u << ((~rank & 4) + ((it - 64) >> 4)) : 0;
            else if (it >= FI && xi >= PB * XA && xi < (PB + RB) * XA) early = 15u << (~rank & 4);
        }
        if (early) {
#pragma unroll
            for (int i = 0; i < 4; ++i) st_sys(h[i], r0, lane * 16, base + i * step);
            WAIT_VM0();
            if (lane < 8 && lane != rank && ((early >> lane) & 1)) st_flag(wsl + OFF_RDY + rank * 8192 + (1536 + blk * 128 + it) * 4, call);
        }
        if (!early || (it >= FI1(0) && it < FI))
        pack_item(h, amax, scr, r0, par + REL_PK + (u32)(blk * 128 + it) * 4096, lane, [&] {
            WAIT_VM0();
            if (r > 0) {
                __builtin_amdgcn_s_barrier();
                if (up >= 0) publish(up);
            }
#pragma unroll
            for (int i = 0; i < 4; ++i) st_sys(h[i], r0, lane * 16, base + i * step);
        });
        up = u;
        if (r == 0) {
            STAMPB_IF(true, 1);
            WAIT_VM0();
            STAMPB_IF(true, 2);
            __builtin_amdgcn_s_barrier();
            STAMP(6);
            publish(u);
            up = -1;
            load_item(v, p, un, wid, lane);
            WAIT_VM0();
        }
        u = un;
        un = (un + NGEMM) ^ 8;
    }
    WAIT_VM0();
    __builtin_amdgcn_s_barrier();
    if (up >= 0) publish(up);
}

constexpr int PUSH_PER = FI1MAX + 64 + XA, NPUSH = PB * PUSH_PER, NPW = (NPUSH + NCOMM - 1) / NCOMM;
__device__ __forceinline__ void push_decode(int n, int& blk, int& it) {
    blk = n / PUSH_PER; it = n - blk * PUSH_PER;
    if (it >= FI1MAX + 64) { const int xi = blk * XA + it - FI1MAX - 64; blk = xi / FX; it = FI + xi - blk * FX; }
    else if (it >= FI1MAX) it += 64 - FI1MAX;
}
__device__ __forceinline__ void push_role(const Params& p, char* wsl, int n, int lane, u32 call) {
    const int rank = rfl(p.rank);
    const int g = rank >> 2, j = rank & 3, og = (g ^ 1) * 4;
    int blk, it;
    push_decode(n, blk, it);
    f32x4 v[8];
    load_item(v, p, blk * 16 + (it >> 3), it & 7, lane);
    const u32 par = OFF_PAR + (call & 1) * PAR_BYTES;
    u32x4 h[4];
    int d0 = 0, nd = 4, dstep = 1; u32 base, step = 1024;
    if (it < 64) {
#pragma unroll
        for (int i = 0; i < 4; ++i) h[i] = cvt8(v[2 * i], v[2 * i + 1]);
        if (it < FI) { d0 = j + 3; nd = 3; dstep = 3; base = par + (blk * RS1 + 64 + it) * 4096; }
        else base = par + (blk * RS1 + 64 + NI1 + j * FX + it - FI) * 4096;
    } else {
#pragma unroll
        for (int c = 0; c < 4; ++c) h[c] = cvt_w(v, c);
        d0 = (it - 64) >> 4; nd = 1; step = 4096;
        base = par + REL_B + (blk * CF + 64 + 16 * j + (it & 12)) * 4096 + (it & 3) * 1024;
    }
    u32 dmask = 0;
    for (int d = 0; d < nd; ++d) {
        if (it < FI && it >= FI1(d)) continue;
        const int dr = (it < FI ? g * 4 : og) + ((d0 + d * dstep) & 3);
        const rsrc_t rd = mk_rsrc(p.ws[dr], WS_NEED);
#pragma unroll
        for (int i = 0; i < 4; ++i) st_sys(h[i], rd, lane * 16, base + (it < FI ? OFF1(d) * 4096 : 0) + i * step);
        dmask |= 1u << dr;
    }
    WAIT_VM0();
    STAMPB_IF(true, 3);
    if (lane < 8 && ((dmask >> lane) & 1)) add_sys(wsl + OFF_CNT + blk * CNT_BLK + (it < 64 ? 0 : 1024) + rank * 128);
}

#define P_NDEC 9
#define P_DSLEEP 2
constexpr int SCR = 2560;
constexpr int NLW = 3, RING = 6, QLEN = 32, NDEC = P_NDEC;
constexpr int LDS_CTL = NLW * RING * 4096;
constexpr int LDS_DONE = LDS_CTL + QLEN * 16, LDS_TAIL = LDS_DONE + NLW * RING * 4, LDS_SCR = LDS_CTL + 1024;
static_assert(7 * WL <= NLW * NCOMM, "loaders");
constexpr u32 IDX_END = 0xffffffffu, IDX_BAD = 0x80000000u, IDX_RAW = 0x20000000u;
__device__ __forceinline__ u32 lds_get(const LAS char* a) {
    u32 v;
    asm volatile("ds_read_b32 %0, %1\n\ts_waitcnt lgkmcnt(0)" : "=v"(v) : "v"((u32)(size_t)a) : "memory");
    return rfl(v);
}
__device__ __forceinline__ void lds_put(LAS char* a, u32 v) {
    asm volatile("" ::: "memory");
    *(volatile LAS u32*)a = v;
    asm volatile("" ::: "memory");
}

struct Item { u32 idx, dst; };
constexpr int NT_I = NBLK * FI;
constexpr int PER = 16 + XA, NP = NBLK * PER, NX = NBLK * (16 + FX), NO1 = (RF1 - FX) * 4, NT_C = NX + NO1 + 192;
constexpr int TAB_BYTES = (NT_I > NT_C ? NT_I : NT_C) * 4, LDS_TAB = LDS_SCR + NDEC * SCR;
static_assert(LDS_TAB + NLW * TAB_BYTES <= LDS_BYTES, "communication LDS");
constexpr int TAB_HEAD = 128;
static_assert(PB * (PER > FI1MAX ? PER : FI1MAX) + 4 * WL <= TAB_HEAD, "table head");
__device__ __forceinline__ void build_table(LAS char* tab, int rank, int slot, int first, int stride, int end) {
    const int g = rank >> 2, j = rank & 3, q = slot - 3;
    const u32 rc1 = (1u << 20) / FI1(slot < 3 ? slot : 0) + 1, rc2 = (1u << 20) / F2(slot < 3 ? slot : 0) + 1;
    for (int it = first; it < min(end, slot < 3 ? NT_I : NT_C); it += stride) {
        Item m;
        if (slot < 3) {
            const int f1 = FI1(slot), f2 = F2(slot);
            const bool s2 = it >= NBLK * f1;
            const int a = s2 ? it - NBLK * f1 : it, blk = (a * (s2 ? rc2 : rc1)) >> 20, s = a - blk * (s2 ? f2 : f1) + (s2 ? f1 : 0);
            m.idx = blk * 128 + s;
            m.dst = (s2 ? REL_A2 + (OFF2(slot) - f1) * 4096 : (64 + OFF1(slot)) * 4096) + (blk * (s2 ? RF2 : RS1) + s) * 4096;
        } else if (it < NX) {
            const int blk = it / PER, c = it - blk * PER;
            const bool tail = it >= NP, isx = tail || (c < 2 * XA && (c & 1));
            const int cw = c < 2 * XA ? c >> 1 : c - XA;
            const int xi = tail ? it - NP + NBLK * XA : blk * XA + (c >> 1), xb = xi / FX, xs = FI + xi - xb * FX;
            m.idx = isx ? xb * 128 + xs : blk * 128 + 64 + 16 * j + cw;
            m.dst = isx ? (xb * RS1 + xs + 64 + NI1 + q * FX - FI) * 4096
                        : REL_B + (blk * CF + 64 + 16 * q + (cw & 12)) * 4096 + (cw & 3) * 1024;
        } else {
            const int k = it - NX;
            int row, col;
            if (k < NO1) {
                const int f = k >> 2;
                m.idx = 8192 + (f >> 4) * 64 + (k & 2) * 16 + ((f >> 2) & 3) * 8 + (k & 1) * 4 + (f & 3);
                row = row_of_slot(slot_of(f, j), g ^ 1, q);
                col = (k & 3) * 64;
            } else {
                const int z = (k - NO1 + RF2 * 4) % 192;
                m.idx = 8192 + 768 + k - NO1;
                row = row_of_f2((z >> 5) * 8 + (z & 7), g ^ 1, q);
                col = ((z >> 3) & 3) * 64;
            }
            m.dst = (u32)row * 4096 + (q * 256 + col) * 4;
        }
        *(LAS u32*)(tab + it * 4) = m.idx | m.dst << (m.idx < 8192 ? 4 : 6);
    }
}

template <bool CROSS>
__device__ __forceinline__ void load_role(const Params& p, LAS char* lds, int ld, int nld, int slot, int wv, int lane, u32 call) {
    const int rank = rfl(p.rank);
    const int g = rank >> 2, j = rank & 3;
    const int peer = CROSS ? (g ^ 1) * 4 + slot - 3 : g * 4 + ((j + 1 + slot) & 3);
    constexpr int NT = CROSS ? NT_C : NT_I;
    const u32 par = OFF_PAR + (call & 1) * PAR_BYTES;
    char* ws = uni(p.ws[rank]);
    const rsrc_t r0 = rsrc_u(ws, WS_NEED), rp = mk_rsrc(p.ws[peer], WS_NEED);
    const char* rdy = ws + OFF_RDY + peer * 8192;
    const u32 vo = lane * 16;
    const int SKIP = PB * (CROSS ? PER : FI1(slot));
    const u32 head = lane == 0 ? OFF_QUE + slot * 256 : WS_NEED, tbase = SKIP + 4 * WL - (call - 1) * (NT - SKIP);
    const bool nowait = (p.pad & 4) != 0;
    const LAS char* tab = lds + LDS_TAB + ld * TAB_BYTES;
    const LAS char* done = lds + LDS_DONE + ld * RING * 4;
    const u32 ldsb = rfl((u32)(size_t)lds) + ld * RING * 4096;
    auto table = [&](int it) {
        asm volatile("ds_read_b32 v160, %0" :: "v"((u32)(size_t)tab + (it < NT ? it : NT - 1) * 4) : "memory", "v160");
    };
    auto entry = [&] {
        u32 t;
        asm volatile("s_waitcnt lgkmcnt(0)\n\tv_readfirstlane_b32 %0, v160" : "=s"(t) :: "memory");
        Item m;
        m.idx = t & 16383;
        m.dst = (t - m.idx) >> (CROSS && m.idx >= 8192 ? 6 : 4);
        return m;
    };
    const int RAWEND = SKIP + RB * (CROSS ? PER : FI1(slot));
    auto mark = [&](Item m, int it) { if (RB > 0 && it < RAWEND) m.idx |= IDX_RAW; return m; };
    auto decode = [&](int it) { table(it); return mark(entry(), it); };
    auto flag_lean = [&](u32 idx) { return idx < 8192 ? idx >> 3 : (idx < 8192 + 768 ? 1024 + ((idx - 8192) >> 2) : idx - (8192 + 768 - 1024 - 192)); };
    auto flag_of = [&](u32 idx) { return (idx & IDX_RAW) ? 1536 + (idx & 511) : flag_lean(idx); };
    u32 okupto = 0, wbase = 0; u64 wmask = 0;
    auto seen = [&](u32 fi) { return fi < okupto || (fi - wbase < 64u && ((wmask >> (fi - wbase)) & 1)); };
    auto ready = [&](u32 fi) {
        if (seen(fi)) return true;
        for (;;) {
            wbase = okupto;
            wmask = __builtin_amdgcn_ballot_w64(ld_sys(rdy + (wbase + lane) * 4) >= call);
            if (wmask != ~0ull) break;
            okupto += 64;
            if (fi < okupto) return true;
        }
        okupto += __builtin_ctzll(~wmask);
        if (fi - wbase < 64u) return ((wmask >> (fi - wbase)) & 1) != 0;
        return rfl(ld_sys(rdy + fi * 4)) >= call;
    };
    auto ready_early = [&](u32 fi) { return fi >= 1536 ? rfl(ld_sys(rdy + fi * 4)) >= call : ready(fi); };
    const u32 vlean = vo < 256 + VAR_X ? vo : 0x80000000u;
    struct Prep { u32 s0, s1, s2, s3, m0v, vl; };
    auto prep = [&](const Item& m, u32 n) {
        Prep r;
        const bool out = CROSS && m.idx >= 8192 && m.idx < IDX_RAW;
        u32 d = 0, rest = 256 + VAR_X;
        r.s0 = par + REL_PK + (m.idx + (out ? j * 1024 : 0)) * 4096;
        if (m.idx & IDX_RAW) {
            const bool isw = CROSS && (m.idx & 64);
            const u32 blk = (m.idx >> 7) & 3, s = m.idx & 127;
            r.s0 = par + (isw ? REL_B + (blk * CF + (s & 60)) * 4096 + (s & 3) * 1024 : (blk * RS1 + s) * 4096);
            d = isw ? 3072 : 0; rest = 1024;
        }
        r.s1 = r.s0 + d; r.s2 = r.s1 + d; r.s3 = r.s2 + d;
        r.m0v = ldsb + (n % RING) * 4096;
        r.vl = vo < rest ? vo : 0x80000000u;
        asm volatile("" : "+s"(r.s0), "+s"(r.s1), "+s"(r.s2), "+s"(r.s3), "+s"(r.m0v), "+v"(r.vl));
        return r;
    };
    auto prep_lean = [&](const Item& m, u32 n) {
        Prep r;
        r.s0 = par + REL_PK + (m.idx + (CROSS && m.idx >= 8192 ? j * 1024 : 0)) * 4096;
        r.m0v = ldsb + (n % RING) * 4096;
        asm volatile("" : "+s"(r.s0), "+s"(r.m0v));
        r.s1 = r.s2 = r.s3 = r.s0; r.vl = vlean;
        return r;
    };
    auto issue = [&](const Prep& r) {
        asm volatile("s_mov_b32 m0, %0\n\ts_nop 0\n\t"
                     "buffer_load_dwordx4 %1, %3, %4 offen sc0 sc1 lds\n\t"
                     "buffer_load_dwordx4 %1, %3, %5 offen offset:1024 sc0 sc1 lds\n\t"
                     "buffer_load_dwordx4 %1, %3, %6 offen offset:2048 sc0 sc1 lds\n\t"
                     "buffer_load_dwordx4 %2, %3, %7 offen offset:3072 sc0 sc1 lds"
                     :: "s"(r.m0v), "v"(vo), "v"(r.vl), "s"(rp), "s"(r.s0), "s"(r.s1), "s"(r.s2), "s"(r.s3) : "memory");
    };
    u32 vtail = (u32)(size_t)lds + LDS_TAIL, vone = 1;
    asm volatile("" : "+v"(vtail), "+v"(vone));
    auto enqueue = [&](u32 idx, u32 dst, u32 n) {
        asm volatile("s_mov_b64 exec, 1\n\tds_add_rtn_u32 v162, %0, %1\n\tv_mov_b32 v159, %2\n\tv_mov_b32 v160, %3\n\tv_mov_b32 v161, %4\n\t"
                     "s_waitcnt lgkmcnt(0)\n\tv_add_u32 v158, 1, v162\n\tv_and_b32 v162, %6, v162\n\tv_lshl_add_u32 v162, v162, 4, %5\n\t"
                     "ds_write_b128 v162, v[158:161]\n\ts_mov_b64 exec, -1"
                     :: "v"(vtail), "v"(vone), "s"(idx), "s"(dst), "s"(n << 8 | ld << 4 | peer), "s"(rfl((u32)(size_t)lds) + LDS_CTL), "n"(QLEN - 1)
                     : "memory", "v158", "v159", "v160", "v161", "v162");
    };
    __builtin_amdgcn_s_setprio(3);
    int e = SKIP + wv, e1 = e + WL, e2 = e + 2 * WL;
    const int e3 = e + 3 * WL;
    Item nx = decode(e1), ia = decode(e);
    bool bad = spin_ge<2>(rdy + flag_of(ia.idx) * 4, call, ws + OFF_TMO, call);
    STAMP(5);
    issue(prep(ia, 0));
    bool pre = ready_early(flag_of(nx.idx));
    u32 n = 0;
    auto wait_free = [&](u32 m) { while (lds_get(done + (m % RING) * 4) + RING <= m) __builtin_amdgcn_s_sleep(1); };
    auto body = [&](auto odd, auto early) {
        if constexpr (odd.value) asm volatile("v_mov_b32 v164, 1\n\ts_nop 0\n\tbuffer_atomic_add v164, %0, %1, 0 offen sc0" :: "v"(head), "s"(r0) : "memory", "v164");
        else                     asm volatile("v_mov_b32 v165, 1\n\ts_nop 0\n\tbuffer_atomic_add v165, %0, %1, 0 offen sc0" :: "v"(head), "s"(r0) : "memory", "v165");
        if (early.value && n == 2) while (lds_get(lds + LDS_TAIL + 12) < NDEC - NPW) __builtin_amdgcn_s_sleep(1);
        table(e2);
        u32 dn;
        asm volatile("ds_read_b32 v162, %1\n\ts_waitcnt lgkmcnt(0)\n\tv_readfirstlane_b32 %0, v162"
                     : "=s"(dn) : "v"((u32)(size_t)done + ((n + 1) % RING) * 4) : "memory", "v162");
        const Item nn = early.value ? mark(entry(), e2) : entry();
        bool pre2 = e2 < NT;
        if (pre2 && !(nowait && nn.idx >= 8192 && nn.idx < IDX_RAW)) pre2 = early.value ? ready_early(flag_of(nn.idx)) : ready(flag_lean(nn.idx));
        if (!pre && e1 < NT) pre = seen(early.value ? flag_of(nx.idx) : flag_lean(nx.idx));
        if (dn + RING <= n + 1) wait_free(n + 1);
        const Prep px = early.value ? prep(nx, n + 1) : prep_lean(nx, n + 1);
        u32 told;
        if constexpr (odd.value) asm volatile("s_waitcnt vmcnt(1)\n\tv_readfirstlane_b32 %0, v165" : "=s"(told) :: "memory");
        else                     asm volatile("s_waitcnt vmcnt(1)\n\tv_readfirstlane_b32 %0, v164" : "=s"(told) :: "memory");
        if (pre) issue(px);
        enqueue(ia.idx | (bad ? IDX_BAD : 0), ia.dst, n);
        if (!pre && e1 < NT) {
            bad |= spin_ge<2>(rdy + flag_of(nx.idx) * 4, call, ws + OFF_TMO, call);
            issue(px);
        }
        STAMP_IF(n == 0, 6);
        STAMP_IF(CROSS ? (e < NP && e1 >= NP) : (e < NBLK * FI1(slot) && e1 >= NBLK * FI1(slot)), 1);
        STAMP_IF(CROSS && e < NX && e1 >= NX, 2);
        STAMP_IF(CROSS && e < NX + NO1 && e1 >= NX + NO1, 3);
        ia = nx; nx = nn; pre = pre2; ++n;
        e = e1; e1 = e2; e2 = early.value && n == 1 ? e3 : (int)(told + tbase);
    };
    body(std::false_type{}, std::true_type{});
    body(std::true_type{}, std::true_type{});
    body(std::false_type{}, std::true_type{});
    for (;;) {
        body(std::true_type{}, std::false_type{});
        if (e >= NT) break;
        body(std::false_type{}, std::false_type{});
        if (e >= NT) break;
    }
    u32 fin;
    asm volatile("ds_add_rtn_u32 v162, %1, %2\n\ts_waitcnt lgkmcnt(0)\n\tv_readfirstlane_b32 %0, v162"
                 : "=s"(fin) : "v"((u32)(size_t)lds + LDS_TAIL + 4), "v"(lane == 0 ? 1u : 0u) : "memory", "v162");
    if (fin + 1 == (u32)nld)
        for (int k = 0; k < NDEC; ++k) enqueue(IDX_END, 0, 0);
}

__device__ __forceinline__ void decode_role(const Params& p, LAS char* lds, u32 me, int lane, u32 call) {
    const int rank = rfl(p.rank);
    const int j = rank & 3;
    const u32 par = OFF_PAR + (call & 1) * PAR_BYTES;
    const rsrc_t r0 = mk_rsrc(p.ws[rank], WS_NEED), ro = mk_rsrc(p.out[rank], OUT_BYTES);
    const u32 vo = lane * 16, vout = (lane >> 4) * 4 * 4096 + (lane & 15) * 16;
    LAS char* scr = lds + LDS_SCR + me * SCR;
    u32 q0, q1, q2, q3;
    auto peek = [&](u32 n) {
        asm volatile("ds_read_b128 v[158:161], %4\n\ts_waitcnt lgkmcnt(0)\n\tv_readfirstlane_b32 %0, v158\n\tv_readfirstlane_b32 %1, v159\n\t"
                     "v_readfirstlane_b32 %2, v160\n\tv_readfirstlane_b32 %3, v161"
                     : "=s"(q0), "=s"(q1), "=s"(q2), "=s"(q3) : "v"((u32)(size_t)lds + LDS_CTL + (n % QLEN) * 16) : "memory", "v158", "v159", "v160", "v161");
        return q0 == n + 1;
    };
    const u32 vsig = lane == 0 ? 0u : 0x80000000u;
    u32 one = 1;
    asm volatile("" : "+v"(one));
    auto signal = [&](u32 c) {
        asm volatile("s_nop 4\n\tbuffer_atomic_add %0, %1, %2, %3 offen sc1" :: "v"(one), "v"(vsig), "s"(r0), "s"(rfl(c)) : "memory");
    };
    const u32x4 nan4 = {0x7fc07fc0u, 0x7fc07fc0u, 0x7fc07fc0u, 0x7fc07fc0u};
    u32 vhead = (u32)(size_t)lds + LDS_TAIL + 8, vone = 1;
    asm volatile("" : "+v"(vhead), "+v"(vone));
    for (;;) {
        u32 n;
        asm volatile("s_mov_b64 exec, 1\n\tds_add_rtn_u32 v158, %1, %2\n\ts_waitcnt lgkmcnt(0)\n\tv_readfirstlane_b32 %0, v158\n\ts_mov_b64 exec, -1"
                     : "=s"(n) : "v"(vhead), "v"(vone) : "memory", "v158");
        while (!peek(n)) __builtin_amdgcn_s_sleep(P_DSLEEP);
        if (q1 == IDX_END) break;
        const u32 dst = q2, item = q3 >> 8, ring = (((q3 >> 4) & 3) * RING + item % RING), peer = q3 & 7;
        const LAS char* sl = lds + ring * 4096;
        const u32x4 c0 = *(const LAS u32x4*)(sl + vo), c1 = *(const LAS u32x4*)(sl + 1024 + vo), c2 = *(const LAS u32x4*)(sl + 2048 + vo);
        const u32 k1 = *(const LAS u32*)(sl + 3072 + lane * 4);
        const u32 hx = *(const LAS unsigned char*)(sl + 3328);
        const bool bad = (q1 & IDX_BAD) != 0;
        const u32 idx = q1 & 0xffff;
        const bool out = idx >= 8192, isw = !out && (idx & 64);
        const u32 step = (out || isw) ? 4096 : 1024;
        u32x4 h[4];
        bool done = false;
        if (!bad && (q1 & IDX_RAW)) {
            h[0] = c0; h[1] = c1; h[2] = c2; h[3] = *(const LAS u32x4*)(sl + 3072 + vo); done = true;
        } else if (!bad && !out) {
            const u32 hd = rfl(hx);
            if (hd != 255) {
                if (hd & 64) {
                    u32 t;
                    asm volatile("s_nop 4\n\tbuffer_load_dword %0, %1, %2, %3 offen sc0 sc1\n\ts_waitcnt vmcnt(0)"
                                 : "=&v"(t) : "v"(lane < 16 ? lane * 4 : 0x80000000u), "s"(mk_rsrc(p.ws[peer], WS_NEED)),
                                   "s"(rfl(par + REL_PK + idx * 4096 + 3328 + VAR_X)) : "memory");
                    if (lane < 16) *(LAS u32*)(lds + ring * 4096 + 3328 + VAR_X + lane * 4) = t;
                    asm volatile("" ::: "memory");
                }
                unpack_item(h, c0, c1, c2, k1, sl + 3328, hd, scr, lane);
                done = true;
            }
        } else if (!bad) {
            const u32 g1 = *(const LAS unsigned short*)(sl + 3328 + lane * 2), hd = rfl(*(const LAS unsigned short*)(sl + 3456));
            if ((hd & 255) != 255) { unpack_out(h, c0, c1, c2, k1, g1, (const LAS unsigned char*)(sl + 3456), hd & 255, hd >> 8, scr, lane); done = true; }
        }
        if (!done) {
            const u32 blk = idx >> 7, s = idx & 127;
            const u32 raw = par + (out ? REL_PK + STG_BYTES + (idx + j * 1024) * 4096
                                       : isw ? REL_B + (blk * CF + (s & 60)) * 4096 + (s & 3) * 1024 : (blk * RS1 + s) * 4096);
            const u32 rs = isw ? 4096 : 1024;
            const rsrc_t rp = mk_rsrc(p.ws[peer], WS_NEED);
            asm volatile("s_nop 4\n\tbuffer_load_dwordx4 %0, %4, %5, %6 offen sc0 sc1\n\tbuffer_load_dwordx4 %1, %4, %5, %7 offen sc0 sc1\n\t"
                         "buffer_load_dwordx4 %2, %4, %5, %8 offen sc0 sc1\n\tbuffer_load_dwordx4 %3, %4, %5, %9 offen sc0 sc1\n\ts_waitcnt vmcnt(0)"
                         : "=&v"(h[0]), "=&v"(h[1]), "=&v"(h[2]), "=&v"(h[3])
                         : "v"(vo), "s"(rp), "s"(rfl(raw)), "s"(rfl(raw + rs)), "s"(rfl(raw + 2 * rs)), "s"(rfl(raw + 3 * rs)) : "memory");
            if (bad) { h[0] = nan4; h[1] = nan4; h[2] = nan4; h[3] = nan4; }
        }
        lds_put(lds + LDS_DONE + ring * 4, item + 1);
        const rsrc_t rd = out ? ro : r0;
        const u32 vd = (out ? vout : vo + par) + dst;
#pragma unroll
        for (int i = 0; i < 4; ++i) st_sys(h[i], rd, vd, i * step);
        if (!out) {
            asm volatile("s_waitcnt vmcnt(0)" ::: "memory");
            signal(OFF_CNT + (idx >> 7) * CNT_BLK + peer * 128 + (isw ? 1024 : dst >= REL_A2 ? 2048 : 0));
        }
    }
    STAMP(4);
}

#define GLDS2(m0v, voff, rs, soff, imm0, imm1) \
    asm volatile("s_mov_b32 m0, %0\n\ts_nop 0\n\tbuffer_load_dwordx4 %1, %2, %3 offen offset:" #imm0 " sc0 sc1 lds\n\t" \
                 "buffer_load_dwordx4 %1, %2, %3 offen offset:" #imm1 " sc0 sc1 lds" \
                 :: "s"(m0v), "v"(voff), "s"(rs), "s"(soff) : "memory")
#define BARRIER() do { __builtin_amdgcn_sched_barrier(0); __builtin_amdgcn_s_barrier(); __builtin_amdgcn_sched_barrier(0); } while (0)

#define SCHED() __builtin_amdgcn_sched_barrier(0)
#define WAIT_LGKM0() __builtin_amdgcn_s_waitcnt(0xC07F)

template <int STAGE>
struct Gemm {
    static constexpr int NT = STAGE == 1 ? 128 : 64;
    static constexpr int NA = STAGE == 1 ? 4 : 2;
    static constexpr int NB = STAGE == 1 ? 4 : 8;
    static constexpr int NM = STAGE == 1 ? 4 : 1;
    static constexpr int NRD = NA + NB;
    struct Frags { f16x8 a[NA], b[NB]; };

    rsrc_t rs;
    u32 sa, sa2, sb;
    u32 ablk;
    u32 ldsw;
    int wid, lane;

    template <int I> __device__ __forceinline__ void piece(int t, int buf) const {
        const u32 vo = lane * 16;
        const u32 ma = ldsw + buf * BUF_BYTES + wid * 4096, mb = ldsw + buf * BUF_BYTES + 32768 + wid * 2048;
        const u32 ka = STAGE == 1 ? (u32)(t >> 1) * (RS1 * 4096) + (t & 1) * 2048 : (u32)t * ablk;
        const u32 a = sa + ka, a2 = sa2 + ka;
        const u32 b = STAGE == 1 ? sb + (u32)(t >> 1) * (CF * 4096) + (t & 1) * 2048 : sb + (u32)t * (CF * 4096);
        if (I == 0) GLDS2(ma, vo, rs, a, 0, 1024);
        if (I == 1) GLDS2(ma, vo, rs, a2, 2048, 3072);
        if (I == 2) GLDS2(mb, vo, rs, b, 0, 1024);
    }
    __device__ __forceinline__ void stage(int t, int buf) const { piece<0>(t, buf); piece<1>(t, buf); piece<2>(t, buf); }

    template <int I> __device__ __forceinline__ void rd(Frags& f, const LAS char* lds, int buf, int half) const {
        const LAS char* base = lds + buf * BUF_BYTES + lane * 16;
        if constexpr (STAGE == 1) {
            const LAS char* pa = base + (wid >> 1) * 8192 + half * 1024;
            const LAS char* pb = base + 32768 + (wid & 1) * 8192 + half * 1024;
            if constexpr (I == 0) f.a[0] = *(const LAS f16x8*)pa;
            else if constexpr (I <= 4) f.b[I - 1] = *(const LAS f16x8*)(pb + (I - 1) * 2048);
            else f.a[I - 4] = *(const LAS f16x8*)(pa + (I - 4) * 2048);
        } else {
            const LAS char* pa = base + wid * 4096 + half * 2048;
            const LAS char* pb = base + 32768 + half * 2048;
            constexpr int h = I / 5, n = I % 5;
            if constexpr (n == 0) f.a[h] = *(const LAS f16x8*)(pa + h * 1024);
            else f.b[h * 4 + n - 1] = *(const LAS f16x8*)(pb + (n - 1) * 4096 + h * 1024);
        }
    }
    template <int I> __device__ __forceinline__ void mm(f32x4 (&acc)[NM][4], const Frags& f) const {
        if constexpr (STAGE == 1) acc[I >> 2][I & 3] = __builtin_amdgcn_mfma_f32_16x16x32_f16(f.a[I >> 2], f.b[I & 3], acc[I >> 2][I & 3], 0, 0, 0);
        else            acc[0][I & 3] = __builtin_amdgcn_mfma_f32_16x16x32_f16(f.a[I >> 2], f.b[I], acc[0][I & 3], 0, 0, 0);
    }
    template <int I, int N> __device__ __forceinline__ void rds(Frags& f, const LAS char* lds, int buf, int half) const {
        if constexpr (I < N) { rd<I>(f, lds, buf, half); rds<I + 1, N>(f, lds, buf, half); }
    }
    static __device__ __forceinline__ int blk_of(int t) { return STAGE == 1 ? (t >> 1) : t; }

    template <bool RD, int ST>
    __device__ __forceinline__ void half_step(f32x4 (&acc)[NM][4], const Frags& cur, Frags& nxt, const LAS char* lds, int rbuf, int rhalf,
                                              int ts, int bs, u32 cvo, u32 fso) const {
        auto first = [&] {
            if (ST == 1) asm volatile("buffer_load_dword v166, %0, %1, %2 offen sc0 sc1" :: "v"(cvo), "s"(rs), "s"(fso) : "memory", "v166");
            if (ST) { piece<0>(ts, bs); SCHED(); }
        };
        if constexpr (STAGE == 1) {
            mm<0>(acc, cur);  mm<1>(acc, cur);  if (RD) rds<0, 2>(nxt, lds, rbuf, rhalf); SCHED();
            mm<2>(acc, cur);  mm<3>(acc, cur);  if (RD) rds<2, 4>(nxt, lds, rbuf, rhalf); SCHED();
            mm<4>(acc, cur);  mm<5>(acc, cur);  if (RD) rds<4, 6>(nxt, lds, rbuf, rhalf); SCHED();
            mm<6>(acc, cur);  mm<7>(acc, cur);  if (RD) rds<6, 8>(nxt, lds, rbuf, rhalf); SCHED();
            mm<8>(acc, cur);  mm<9>(acc, cur);  SCHED();
            first();
            mm<10>(acc, cur); mm<11>(acc, cur); SCHED();
            if (ST) { piece<1>(ts, bs); SCHED(); }
            mm<12>(acc, cur); mm<13>(acc, cur); SCHED();
            if (ST) { piece<2>(ts, bs); SCHED(); }
            mm<14>(acc, cur); mm<15>(acc, cur); SCHED();
        } else {
            first();
            mm<0>(acc, cur); SCHED();
            if (ST) { piece<1>(ts, bs); SCHED(); }
            mm<1>(acc, cur); SCHED();
            if (ST) { piece<2>(ts, bs); SCHED(); }
            mm<2>(acc, cur); if (RD) rds<0, 3>(nxt, lds, rbuf, rhalf); SCHED();
            mm<3>(acc, cur); if (RD) rds<3, 5>(nxt, lds, rbuf, rhalf); SCHED();
            mm<4>(acc, cur); if (RD) rds<5, 8>(nxt, lds, rbuf, rhalf); SCHED();
            mm<5>(acc, cur); if (RD) rds<8, 10>(nxt, lds, rbuf, rhalf); SCHED();
            mm<6>(acc, cur); mm<7>(acc, cur); SCHED();
        }
    }

    template <int ROLE>
    __device__ __forceinline__ u32 run(f32x4 (&acc)[NM][4], const LAS char* lds, char* ws, u32 cvo, u32 need, u32 call) const {
        constexpr bool LATE = ROLE == 2 && STAGE == 1;
        char* tmo = ws + OFF_TMO;
        Frags f0, f1;
        if (ROLE == 0) spin_ge(ws + __shfl(cvo, lane & 15) + blk_of(lane < 48 ? lane >> 4 : 0) * CNT_BLK, __shfl(need, lane & 15), tmo, call);
        BARRIER();
        stage(0, 0);
        stage(1, 1);
        if (!LATE) { stage(2, 2); asm volatile("s_waitcnt vmcnt(12)" ::: "memory"); }
        else asm volatile("s_waitcnt vmcnt(6)" ::: "memory");
        BARRIER();
        rds<0, NRD>(f0, lds, 0, 0);
        WAIT_LGKM0();
        if (ROLE == 0) asm volatile("v_mov_b32 v166, 0" ::: "v166");
        int buf = 0;
        for (int t = 0; t < NT - 3; ++t) {
            const int nb = buf == 2 ? 0 : buf + 1, pb = buf == 0 ? 2 : buf - 1;
            half_step<true, LATE ? 2 : 0>(acc, f0, f1, lds, buf, 1, t + 2, pb, 0, 0);
            if (ROLE == 0) {
                u64 miss;
                asm volatile("s_waitcnt vmcnt(6)\n\tv_cmp_lt_u32_e64 %0, v166, %1\n\ts_nop 0" : "=s"(miss) : "v"(need) : "memory");
                WAIT_LGKM0();
                if (miss != 0) spin_ge(ws + cvo + blk_of(t + 3) * CNT_BLK, need, tmo, call);
            } else {
                asm volatile("s_waitcnt vmcnt(6)" ::: "memory");
                WAIT_LGKM0();
            }
            BARRIER();
            half_step<true, LATE ? 0 : (ROLE == 0 ? 1 : 2)>(acc, f1, f0, lds, nb, 0, t + 3, buf, cvo, blk_of(t + 4 < NT ? t + 4 : t + 3) * CNT_BLK);
            buf = nb;
        }
#pragma unroll
        for (int t = NT - 3; t < NT; ++t) {
            const int nb = buf == 2 ? 0 : buf + 1, pb = buf == 0 ? 2 : buf - 1;
            if (t == NT - 3) {
                half_step<true, LATE ? 2 : 0>(acc, f0, f1, lds, buf, 1, t + 2, pb, 0, 0);
                asm volatile("s_waitcnt vmcnt(6)" ::: "memory");
            } else {
                if (t == NT - 2) {
                    const u32 to = OFF_TMO;
                    asm volatile("buffer_load_dword v167, %0, %1, 0 offen sc1" :: "v"(to), "s"(rs) : "memory", "v167");
                }
                half_step<true, 0>(acc, f0, f1, lds, buf, 1, 0, 0, 0, 0);
                asm volatile("s_waitcnt vmcnt(0)" ::: "memory");
            }
            WAIT_LGKM0();
            BARRIER();
            if (t < NT - 1) half_step<true, 0>(acc, f1, f0, lds, nb, 0, 0, 0, 0, 0);
            else            half_step<false, 0>(acc, f1, f0, lds, nb, 0, 0, 0, 0, 0);
            buf = nb;
        }
        u32 tw;
        asm volatile("v_readfirstlane_b32 %0, v167\n\ts_nop 1" : "=s"(tw) :: "memory");
        return tw;
    }
};

__device__ __forceinline__ void activate(u32x4 (&o)[4], f32x4 (&c)[4], bool bad) {
    if (bad) {
        asm volatile("");
#pragma unroll
        for (int n = 0; n < 4; ++n) c[n] = __builtin_bit_cast(f32x4, (u32x4){0x7fc00000u, 0x7fc00000u, 0x7fc00000u, 0x7fc00000u});
    }
#pragma unroll
    for (int n = 0; n < 4; ++n)
#pragma unroll
        for (int r = 0; r < 4; r += 2) {
            const u32x2v y = __builtin_bit_cast(u32x2v, gelu2((f32x2){c[n][r], c[n][r + 1]}));
            o[r][n] = y[0]; o[r + 1][n] = y[1];
        }
}

__device__ __forceinline__ void gemm_role(const Params& p, const LAS char* lds, int xcd, int s, int wid, int lane, u32 call) {
    const int rank = rfl(p.rank);
    const int g = rank >> 2, j = rank & 3, og = (g ^ 1) * 4;
    char* ws = p.ws[rank];
    const u32 par = OFF_PAR + (call & 1) * PAR_BYTES;
    const rsrc_t rs = mk_rsrc(ws, WS_NEED), ro = mk_rsrc(p.out[rank], OUT_BYTES);
    const u32 ldsw = rfl((u32)(size_t)lds);
    const u32 vout = (lane >> 4) * 4 * 4096 + (lane & 15) * 16;
    {
        const int u = (xcd & 1) * (xcd < 4 ? 26 : 24) + s;
        const bool extra = xcd >= 4 && s >= 24;
        const int rt = extra ? 13 : u >> 2, ct = extra ? (xcd - 4) * 2 + (s & 1) : (xcd >> 1) * 4 + (u & 3);
        const int cq = ct < 8 ? -1 : (ct - 8) >> 1;
        Gemm<1> G;
        G.rs = rs; G.ldsw = ldsw; G.wid = wid; G.lane = lane;
        G.sa = par + max(slot_of(rt * 16 + wid * 2, cq), 0) * 4096;
        G.sa2 = par + max(slot_of(rt * 16 + wid * 2 + 1, cq), 0) * 4096 - 2048;
        G.sb = par + REL_B + (ct * 8 + wid) * 4096;
        const int bsrc = ct < 8 ? rank : og + cq;
        u32 cvo = OFF_CNT + 1024 + bsrc * 128, need = (ct < 8 ? 8u : 16u) * call;
        const int sl16 = max(slot_of(rt * 16 + (lane & 15), cq), 0), src16 = row_of_slot(sl16, g, j) / NS;
        u32 mask = 0;
        for (int i = 0; i < 8; ++i) mask |= (__builtin_amdgcn_ballot_w64(src16 == i) != 0) << i;
        if (lane < 8 && ((mask >> lane) & 1)) {
            cvo = OFF_CNT + lane * 128;
            need = (lane == rank ? 8 : ((lane >> 2) == g ? FI1((lane - j - 1) & 3) : FX)) * call;
        }
        f32x4 acc[4][4] = {};
        const bool bad = call == (wid == 0 ? G.template run<0>(acc, lds, ws, cvo, need, call)
                                  : wid < 4 ? G.template run<1>(acc, lds, ws, cvo, need, call)
                                            : G.template run<2>(acc, lds, ws, cvo, need, call));
        STAMP(1);
        const int wm = wid >> 1, wn = wid & 1;
        if (ct < 8) {
#pragma unroll
            for (int m = 0; m < 4; ++m) {
                const int sl = slot_of(rt * 16 + wm * 4 + m, -1);
                if (sl < 0) continue;
                const u32 ob = (u32)row_of_slot(sl, g, j) * 4096 + (ct * 128 + wn * 64) * 4;
                u32x4 o[4];
                activate(o, acc[m], bad);
#pragma unroll
                for (int r = 0; r < 4; ++r) __builtin_amdgcn_raw_buffer_store_b128(o[r], ro, vout + ob + r * 4096, 0, OUT_AUX);
            }
        } else {
            const int fo = (rt * 2 + (ct & 1)) * 8 + wid;
            const u32 sb = par + REL_STG + (u32)(((ct - 8) >> 1) * 1024 + fo * 4) * 4096;
            u32 tp = 0, tn = 0;
#pragma unroll
            for (int m = 0; m < 4; ++m) {
                u32x4 o[4];
                activate(o, acc[m], bad);
                if (m == 0) out_windows(o, tp, tn);
                pack_out(o, tp, tn, (LAS char*)lds + wid * 2048, rs, sb + m * 4096, lane);
            }
            WAIT_VM0();
            if (lane == 0) st_flag(p.ws[bsrc] + OFF_RDY + rank * 8192 + (1024 + fo) * 4, call);
        }
        STAMP(2);
    }
    {
        const bool loc = s >= 24;
        const int rt = ((s >> 2) + 5) % 6, ct = loc ? xcd * 2 + (s & 1) : ((~xcd & 1) * 16) + (xcd >> 1) * 4 + (s & 3);
        Gemm<2> G;
        G.rs = rs; G.ldsw = ldsw; G.wid = wid; G.lane = lane;
        const bool mine = loc || rt * 8 >= RF2;
        G.sa = G.sa2 = par + (mine ? ((loc ? FI : FO1) + wid) * 4096 : REL_A2 + (rt * 8 + wid) * 4096);
        G.ablk = (mine ? RS1 : RF2) * 4096;
        G.sb = par + REL_B + ct * 4 * 4096 + wid * 2048;
        const int bsrc = ct < 16 ? rank : og + ((ct - 16) >> 2);
        u32 cvo = OFF_CNT + 1024 + bsrc * 128, need = (ct < 16 ? 8u : 16u) * call;
        if (mine) { if (lane == 0) { cvo = OFF_CNT + rank * 128; need = 8 * call; } }
        else if (lane < 3 && rt * 8 < OFF2(lane + 1) && rt * 8 + 8 > OFF2(lane)) {
            cvo = OFF_CNT + 2048 + (g * 4 + ((j + 1 + lane) & 3)) * 128; need = F2(lane) * call;
        }
        f32x4 acc[1][4] = {};
        const bool bad = call == (wid == 0 ? G.template run<0>(acc, lds, ws, cvo, need, call)
                                  : wid < 4 ? G.template run<1>(acc, lds, ws, cvo, need, call)
                                            : G.template run<2>(acc, lds, ws, cvo, need, call));
        STAMP(3);
        u32x4 o[4];
        activate(o, acc[0], bad);
        if (ct < 16) {
            const u32 ob = (u32)(loc ? rank * NS + (FI + wid) * 16 : row_of_f2(rt * 8 + wid, g, j)) * 4096 + ct * 256;
#pragma unroll
            for (int r = 0; r < 4; ++r) __builtin_amdgcn_raw_buffer_store_b128(o[r], ro, vout + ob + r * 4096, 0, OUT_AUX);
        } else {
            const int z = ((rt * 4 + (ct & 3)) * 8 + wid + FO2 * 4) % 192;
            const u32 sb = par + REL_STG + (u32)(((ct - 16) >> 2) * 1024 + 768 + z) * 4096;
            u32 tp, tn;
            out_windows(o, tp, tn);
            pack_out(o, tp, tn, (LAS char*)lds + wid * 2048, rs, sb, lane);
            WAIT_VM0();
            if (lane == 0) st_flag(p.ws[bsrc] + OFF_RDY + rank * 8192 + (1024 + 192 + z) * 4, call);
        }
        STAMP(4);
    }
}

__global__ __launch_bounds__(BLOCK, 3) void ag_gemm_kernel(Params p) {
    __shared__ __attribute__((aligned(1024))) char smem[LDS_BYTES];
    const int rank = rfl(p.rank);
    const int lane = threadIdx.x & 63;
    const int wid = rfl(threadIdx.x >> 6);
    const int b = blockIdx.x;
    if (wid >= (b < NCOMM ? NLW + NDEC : 8)) return;
    STAMP(0);
    char* seq = p.ws[rank] + OFF_SEQ + (b * 16 + wid) * 4;
    const u32 prev = ld_agent(seq);
    char* wsl = p.ws[lane & 7];
    auto next_call = [&] {
        const u32 call = rfl(prev) + 1;
        if (lane == 0) __hip_atomic_store((GAS u32*)seq, call, __ATOMIC_RELAXED, __HIP_MEMORY_SCOPE_AGENT);
        STAMP(7);
        return call;
    };
    if (b >= NCOMM) {
        for (int i = ((b - NCOMM) >> 4) - P_G0; i > 0; --i) __builtin_amdgcn_s_sleep(P_GRADE);
        f32x4 v[8];
        load_item(v, p, b - NCOMM, wid, lane);
        const u32 call = next_call();
        convert_local(p, (LAS char*)smem, v, wsl, b - NCOMM, wid, lane, call);
        STAMP(5);
        gemm_role(p, (const LAS char*)smem, b & 7, (b - NCOMM) >> 3, wid, lane, call);
        return;
    }
    const int nld = 2 * NCOMM + b < 7 * WL ? 3 : 2;
    constexpr int NSET = NLW + NDEC - NPW;
    const int sw = (NLW + NDEC - 1 - wid) * NCOMM + b;
    if (wid < 4) *(LAS u32*)(smem + LDS_CTL + (wid * 64 + lane) * 4) = 0;
    asm volatile("s_waitcnt lgkmcnt(0)\n\ts_barrier" ::: "memory");
    if (wid < nld) build_table((LAS char*)smem + LDS_TAB + wid * TAB_BYTES, rank, (wid * NCOMM + b) % 7, lane, 64, TAB_HEAD);
    else if (wid >= NLW && wid < NSET) {
        for (int k = 0; k < nld; ++k)
            build_table((LAS char*)smem + LDS_TAB + k * TAB_BYTES, rank, (k * NCOMM + b) % 7, TAB_HEAD + (wid - NLW) * 64 + lane,
                        (NSET - NLW) * 64, 1 << 30);
        asm volatile("" ::: "memory");
        if (lane == 0) __hip_atomic_fetch_add((LAS u32*)(smem + LDS_TAIL + 12), 1u, __ATOMIC_RELAXED, __HIP_MEMORY_SCOPE_WORKGROUP);
        STAMPB_IF(wid == NLW, 4);
    }
    const u32 call = next_call();
    if (wid >= NSET && sw < NPUSH) push_role(p, wsl, sw, lane, call);
    if (wid >= NLW) decode_role(p, (LAS char*)smem, wid - NLW, lane, call);
    else if (wid < nld) {
        const int ps = rfl((wid * NCOMM + b) % 7), wv = rfl((wid * NCOMM + b) / 7);
        if (ps < 3) load_role<false>(p, (LAS char*)smem, wid, nld, ps, wv, lane, call);
        else        load_role<true>(p, (LAS char*)smem, wid, nld, ps, wv, lane, call);
    }
}

}

constexpr int g_mode = 0;

extern "C" void kernel_launch(int rank, int nranks, ncclComm_t comm, hipStream_t stream,
                              void* const* d_in, const int* in_sizes, int n_in,
                              void* d_out, int out_size, const int* in_dtypes, int out_dtype,
                              void* const* d_out_all, void* const* d_ws_all, size_t ws_bytes) {
    (void)comm; (void)in_sizes; (void)n_in; (void)d_out; (void)out_size; (void)in_dtypes; (void)out_dtype;
    if (nranks != NR || (unsigned)rank >= (unsigned)NR || ws_bytes < WS_NEED) {
        fprintf(stderr, "unsupported configuration\n");
        return;
    }
    Params p;
    p.x = (const float*)d_in[0];
    p.w = (const float*)d_in[1];
    for (int i = 0; i < NR; ++i) { p.ws[i] = (char*)d_ws_all[i]; p.out[i] = (char*)d_out_all[i]; }
    p.rank = rank;
    p.pad = g_mode;
    ag_gemm_kernel<<<GRID, BLOCK, 0, stream>>>(p);
    hipError_t e = hipGetLastError();
    if (e != hipSuccess) fprintf(stderr, "launch failed: %s\n", hipGetErrorString(e));
}
```

```cpp
#include <hip/hip_runtime.h>
#include <rccl/rccl.h>
#include <cstdio>
#include <cstdint>
#include <type_traits>

typedef unsigned u32;
typedef unsigned long long u64;
typedef u32 u32x4 __attribute__((ext_vector_type(4)));
typedef u32 u32x2v __attribute__((ext_vector_type(2)));
typedef float f32x4 __attribute__((ext_vector_type(4)));
typedef _Float16 f16x8 __attribute__((ext_vector_type(8)));
typedef __amdgpu_buffer_rsrc_t rsrc_t;
#define LAS __attribute__((address_space(3)))
#define GAS __attribute__((address_space(1)))

namespace {

constexpr int NR = 8;
constexpr int KD = 8192;
constexpr int NS = 1024;
constexpr int NBLK = 64;
#define P_FI 45
constexpr int FI = P_FI;
constexpr int FX = 64 - FI;
#define P_FO2 8
constexpr int FO2 = P_FO2, RF2 = 48 - FO2, FO1 = FI - FO2;
__host__ __device__ constexpr int OFF2(int i) { return (RF2 * i + 2) / 3; }
__host__ __device__ constexpr int F2(int i) { return OFF2(i + 1) - OFF2(i); }
__host__ __device__ constexpr int FI1(int i) { return FI - F2(i); }
__host__ __device__ constexpr int OFF1(int i) { return i * FI - OFF2(i); }
constexpr int NI1 = 3 * FI - RF2, FI1MAX = FI1(2);
constexpr int FL = 8;
constexpr int RF1 = 208, CF = 128;
static_assert(FO1 + NI1 + 4 * FX == RF1 && FX >= 16 && FX - FL <= 16 && (FO2 == 0 || FO2 == 8) && F2(0) >= F2(2), "rows");

#define P_W 19
#define P_XA 15
#define P_PB 1
#define P_G0 1
#define P_GRADE 20
#define P_RB 2
constexpr int PB = P_PB, RB = P_RB;
static_assert(PB + RB <= 4 && RB * 32 <= 4 * P_W, "early flags; early items are among the statically dealt ones");
constexpr int NCOMM = 48;
constexpr int WL = P_W;
constexpr int XA = P_XA;
static_assert(WL >= 14 && WL <= 20 && XA <= FX && XA <= 16, "wave roles");
constexpr int GRID = 256, BLOCK = 768, NGEMM = GRID - NCOMM;
constexpr int RS1 = 64 + NI1 + 4 * FX;

constexpr u32 OFF_SEQ  = 0;
constexpr u32 OFF_TMO  = 16384;
constexpr u32 OFF_QUE  = 16640;
constexpr u32 OFF_RDY  = 32768;
constexpr u32 OFF_CNT  = 131072;
constexpr u32 CNT_BLK  = 24 * 128;
constexpr u32 OFF_DIAG = 524288;
constexpr u32 A1_BYTES = (u32)NBLK * RS1 * 4096;
constexpr u32 A2_BYTES = (u32)NBLK * RF2 * 4096;
constexpr u32 B_BYTES  = (u32)NBLK * CF * 4096;
constexpr u32 STG_BYTES = 4u * 1024 * 4096;
constexpr u32 PK_BYTES = (u32)NBLK * 128 * 4096;
constexpr u32 REL_A2 = A1_BYTES, REL_B = REL_A2 + A2_BYTES, REL_PK = REL_B + B_BYTES, REL_STG = REL_PK + PK_BYTES;
constexpr u32 PAR_BYTES = REL_STG + 2 * STG_BYTES;
constexpr u32 OFF_PAR  = 1u << 20;
constexpr int VAR_XO = 192;
constexpr u32 WS_NEED  = OFF_PAR + 2 * PAR_BYTES;
constexpr u32 OUT_BYTES = (u32)KD * NS * 4;
static_assert(REL_PK < (256u << 20), "table entries hold 18 bits of dst / 1024");
static_assert(256 + VAR_XO == 448, "output items are as long as x items");
static_assert(PK_BYTES == 8192u * 4096u, "staged outputs follow the packed image");
static_assert(OFF_CNT + NBLK * CNT_BLK <= OFF_DIAG && OFF_RDY + 8 * 8192 + 256 <= OFF_CNT, "control area");

constexpr u64 DEADLINE = 40000000ull;

constexpr int BUF_BYTES = 49152;
constexpr int LDS_BYTES = 3 * BUF_BYTES;

struct Params {
    const float* x;
    const float* w;
    char* ws[NR];
    char* out[NR];
    int rank;
    int pad;
};

__device__ __forceinline__ u32 rfl(u32 v) { return __builtin_amdgcn_readfirstlane(v); }
__device__ __forceinline__ char* uni(const void* p) {
    const u64 a = (u64)p;
    return (char*)(((u64)rfl((u32)(a >> 32)) << 32) | rfl((u32)a));
}
__device__ __forceinline__ rsrc_t rsrc_u(const void* p, u32 bytes) {
    return __builtin_amdgcn_make_buffer_rsrc((void*)p, (short)0, (int)bytes, 0x00020000);
}
__device__ __forceinline__ rsrc_t mk_rsrc(const void* p, u32 bytes) { return rsrc_u(uni(p), bytes); }
__device__ __forceinline__ u32 ld_sys(const char* p) {
    return __hip_atomic_load((const GAS u32*)p, __ATOMIC_RELAXED, __HIP_MEMORY_SCOPE_SYSTEM);
}
__device__ __forceinline__ u32 ld_agent(const char* p) {
    return __hip_atomic_load((const GAS u32*)p, __ATOMIC_RELAXED, __HIP_MEMORY_SCOPE_AGENT);
}
__device__ __forceinline__ void st_flag(char* p, u32 v) {
    __hip_atomic_store((GAS u32*)p, v, __ATOMIC_RELAXED, __HIP_MEMORY_SCOPE_SYSTEM);
}
__device__ __forceinline__ void add_sys(char* p) {
    __hip_atomic_fetch_add((GAS u32*)p, 1u, __ATOMIC_RELAXED, __HIP_MEMORY_SCOPE_SYSTEM);
}
#define WAIT_VM0() do { asm volatile("s_waitcnt vmcnt(0)" ::: "memory"); __builtin_amdgcn_s_waitcnt(0x0F70); } while (0)

template <int SLEEP = 2>
__device__ __forceinline__ bool spin_ge(const char* p, u32 need, char* tmo, u32 call) {
    const u64 t0 = __builtin_amdgcn_s_memrealtime();
    bool gave_up = false;
    for (u32 polls = 0;; ++polls) {
        const u32 v = ld_sys(p);
        if (__builtin_amdgcn_ballot_w64(v < need) == 0) break;
        if ((polls & 7) == 0 && (rfl(ld_agent(tmo)) == call || __builtin_amdgcn_s_memrealtime() - t0 > DEADLINE)) {
            __hip_atomic_store((GAS u32*)tmo, call, __ATOMIC_RELAXED, __HIP_MEMORY_SCOPE_AGENT);
            gave_up = true;
            break;
        }
        __builtin_amdgcn_s_sleep(SLEEP);
    }
    asm volatile("" ::: "memory");
    return gave_up;
}

#define STAMP_IF(c, slot) do {} while (0)
#define STAMP(slot) do {} while (0)
#define STAMPB_IF(c, slot) do {} while (0)

__device__ __forceinline__ u32x4 cvt8(f32x4 a, f32x4 b) {
    f16x8 h;
    h[0] = (_Float16)a[0]; h[1] = (_Float16)a[1]; h[2] = (_Float16)a[2]; h[3] = (_Float16)a[3];
    h[4] = (_Float16)b[0]; h[5] = (_Float16)b[1]; h[6] = (_Float16)b[2]; h[7] = (_Float16)b[3];
    return __builtin_bit_cast(u32x4, h);
}
__device__ __forceinline__ void st_sys(u32x4 v, rsrc_t r, u32 voff, u32 soff) {
    __builtin_amdgcn_raw_buffer_store_b128(v, r, voff + soff, 0, 17);
}
typedef float f32x2 __attribute__((ext_vector_type(2)));
__device__ __forceinline__ f32x2 gelu2(f32x2 y) {
    constexpr float A = -2.8853900817779268f * 0.7978845608028654f, B = A * 0.044715f;
    const f32x2 z = y * (y * y * B + A);
    const f32x2 d = (f32x2){__builtin_amdgcn_exp2f(z[0]), __builtin_amdgcn_exp2f(z[1])} + 1.0f;
    return y * (f32x2){__builtin_amdgcn_rcpf(d[0]), __builtin_amdgcn_rcpf(d[1])};
}

__device__ __forceinline__ int row_of_slot(int sl, int g, int j) {
    if (sl < 64) return (g * 4 + j) * NS + sl * 16;
    sl -= 64;
    if (sl < NI1) { const int i = sl >= OFF1(2) ? 2 : sl >= OFF1(1) ? 1 : 0; return (g * 4 + ((j + 1 + i) & 3)) * NS + (sl - OFF1(i)) * 16; }
    sl -= NI1;
    const int q = sl / FX;
    return ((g ^ 1) * 4 + q) * NS + (FI + sl - q * FX) * 16;
}
__device__ __forceinline__ int slot_of(int f, int q) {
    if (q >= 0) {
        if (f >= RF1 - FX) return -1;
        if (f >= RF1 - 4 * FX + q * FX) f += FX;
    } else if (f >= RF1) return f - RF1 < FX - FL ? FI + FL + f - RF1 : -1;
    return f < FO1 ? f : f + 64 - FO1;
}
__device__ __forceinline__ int row_of_f2(int f2, int g, int j) {
    if (f2 >= RF2) return (g * 4 + j) * NS + (FO1 + f2 - RF2) * 16;
    const int i = f2 >= OFF2(2) ? 2 : f2 >= OFF2(1) ? 1 : 0;
    return (g * 4 + ((j + 1 + i) & 3)) * NS + (FI1(i) + f2 - OFF2(i)) * 16;
}

constexpr int IN_AUX = 2;
constexpr int OUT_AUX = 17;
__device__ __forceinline__ u32x4 cvt_w(const f32x4* v, int c) {
    return cvt8((f32x4){v[0][c], v[1][c], v[2][c], v[3][c]}, (f32x4){v[4][c], v[5][c], v[6][c], v[7][c]});
}

#define DPP_STEP(op, v, ctrl, rmask) v = op(v, (u32)__builtin_amdgcn_update_dpp(0, (int)v, ctrl, rmask, 0xf, false))
__device__ __forceinline__ u32 umax2(u32 a, u32 b) { return a > b ? a : b; }
__device__ __forceinline__ u32 uadd2(u32 a, u32 b) { return a + b; }
__device__ __forceinline__ u32 wave_max(u32 v) {
    DPP_STEP(umax2, v, 0x111, 0xf); DPP_STEP(umax2, v, 0x112, 0xf); DPP_STEP(umax2, v, 0x114, 0xf); DPP_STEP(umax2, v, 0x118, 0xf);
    DPP_STEP(umax2, v, 0x142, 0xa); DPP_STEP(umax2, v, 0x143, 0xc);
    return __builtin_amdgcn_readlane(v, 63);
}
__device__ __forceinline__ u32 wave_sum(u32 v) {
    DPP_STEP(uadd2, v, 0x111, 0xf); DPP_STEP(uadd2, v, 0x112, 0xf); DPP_STEP(uadd2, v, 0x114, 0xf); DPP_STEP(uadd2, v, 0x118, 0xf);
    DPP_STEP(uadd2, v, 0x142, 0xa); DPP_STEP(uadd2, v, 0x143, 0xc);
    return __builtin_amdgcn_readlane(v, 63);
}

__device__ __forceinline__ u32 wave_scan(u32 v) {
    DPP_STEP(uadd2, v, 0x111, 0xf); DPP_STEP(uadd2, v, 0x112, 0xf); DPP_STEP(uadd2, v, 0x114, 0xf); DPP_STEP(uadd2, v, 0x118, 0xf);
    DPP_STEP(uadd2, v, 0x142, 0xa); DPP_STEP(uadd2, v, 0x143, 0xc);
    return v;
}
__device__ __forceinline__ u32 byte_scan(u32 m) {
    u32 a, r;
    asm("v_lshl_add_u32 %0, %1, 8, %1" : "=v"(a) : "v"(m));
    asm("v_lshl_add_u32 %0, %1, 16, %1" : "=v"(r) : "v"(a));
    return r;
}
__device__ __forceinline__ u32 times7(u32 m) {
    u32 a, r;
    asm("v_lshl_add_u32 %0, %1, 1, %1" : "=v"(a) : "v"(m));
    asm("v_lshl_add_u32 %0, %1, 2, %2" : "=v"(r) : "v"(m), "v"(a));
    return r;
}
__device__ __forceinline__ u32 rank_in(u64 m) { return __builtin_amdgcn_mbcnt_hi((u32)(m >> 32), __builtin_amdgcn_mbcnt_lo((u32)m, 0)); }

constexpr int VAR_X = VAR_XO, VAR_W = 256;
__device__ __forceinline__ u32 h7_of(float a) { return (u32)__builtin_bit_cast(unsigned short, (_Float16)a) >> 8; }
__device__ __forceinline__ void lds_or(LAS char* a, u32 v) {
    __hip_atomic_fetch_or((LAS u32*)a, v, __ATOMIC_RELAXED, __HIP_MEMORY_SCOPE_WORKGROUP);
}
__device__ __forceinline__ void put_bits(LAS char* area, u32 bo, u64 v) {
    const u32 sh = bo & 31;
    const u64 w = v << sh;
    LAS char* a = area + (bo >> 5) * 4;
    lds_or(a, (u32)w); lds_or(a + 4, (u32)(w >> 32)); lds_or(a + 8, (u32)((v >> 1) >> (63 - sh)));
}
__device__ __forceinline__ u64 get_bits(const LAS char* area, u32 bo) {
    const LAS u32* w = (const LAS u32*)(area + (bo >> 5) * 4);
    const u32 w0 = w[0], w1 = w[1], w2 = w[2];
    return (u64)__builtin_amdgcn_alignbit(w2, w1, bo) << 32 | __builtin_amdgcn_alignbit(w1, w0, bo);
}
template <class F>
__device__ __forceinline__ void pack_item(const u32x4 (&h)[4], float amax, LAS char* scr, rsrc_t r0, u32 dst, int lane, F before_stores) {
    u32 L[8], H[8];
#pragma unroll
    for (int t = 0; t < 8; ++t) {
        const u32 d0 = h[t >> 1][(t & 1) * 2], d1 = h[t >> 1][(t & 1) * 2 + 1];
        L[t] = __builtin_amdgcn_perm(d1, d0, 0x06040200u);
        H[t] = __builtin_amdgcn_perm(d1, d0, 0x07050301u);
    }
    *(LAS u32x4*)(scr + lane * 16) = (u32x4){H[0], H[1], H[2], H[3]};
    *(LAS u32x4*)(scr + lane * 16 + 1024) = (u32x4){H[4], H[5], H[6], H[7]};
    LAS char* area = scr + 2048;
    *(LAS u32x2v*)(area + lane * 8) = (u32x2v){0u, 0u};
    const u32 lm = wave_sum(h7_of(amax) & 127) + 132, rare = (lm & 255) < 115 ? 1 : 0;
    const u32 E1 = lm < 256 ? 1 : lm < 31 * 256 ? lm >> 8 : 31;
    const u32 eb = E1 * 0x01010101u | 0x80808080u;
    const u32 tbig = rare ? 0x79797979u : 0x7a7a7a7au, vadd = rare ? 0x01010101u : 0x02020202u, top6 = rare ? 0x06060606u : 0u;
    const u32 clo = 0x02010003u, qlo = rare ? 3u : 2u, qhi = rare ? 0x03020100u : 0x03030100u;
    u32 K[2] = {0, 0}, e4 = 0, pos = 0;
    u64 b2 = 0, b3 = 0;
#pragma unroll
    for (int t = 0; t < 8; ++t) {
        const u32 o = (eb - ((H[t] >> 2) & 0x1f1f1f1fu)) ^ 0x80808080u, o7 = o & 0x7f7f7f7fu;
        const u32 far = (o | (o7 + tbig)) & 0x80808080u, huge = (o | (o7 + 0x73737373u)) & 0x80808080u;
        const u32 f7 = (far >> 4) - (far >> 7), sel = (o | f7) & 0x07070707u;
        const u32 c = __builtin_amdgcn_perm(0x03030303u, clo, sel);
        const u32 q = __builtin_amdgcn_perm(qhi, qlo, sel);
        const u32 M = __builtin_amdgcn_perm(0x01010101u, 0x00000001u, sel);
        const u32 M2 = q & (q >> 1);
        const u32 v = (((o7 + vadd) | ((huge >> 4) - (huge >> 7))) & f7) | __builtin_amdgcn_perm(0u, top6 & 255, sel);
        const u32 P = byte_scan(M), P2 = byte_scan(M2), X2 = P2 - M2;
        const u32 bits2 = __builtin_amdgcn_udot4(q, __builtin_amdgcn_perm(0u, 0x40100401u, P - M), 0u, false);
        const u32 bits3 = __builtin_amdgcn_udot4(v, __builtin_amdgcn_perm(0u, 0x00400801u, X2), (X2 >> 24) == 3 ? (v >> 24) << 9 : 0u, false);
        b2 |= (u64)bits2 << (pos & 63);
        b3 |= (u64)bits3 << ((pos >> 16) & 63);
        pos += (P >> 23) + ((P2 >> 24) * 3 << 16);
        K[t >> 2] |= c << (2 * (t & 3));
        e4 = (e4 >> 1) | huge;
        if (t & 1) { asm volatile("" : "+v"(b2), "+v"(b3), "+v"(e4)); __builtin_amdgcn_sched_barrier(0); }
    }
    const u32 incl = wave_scan(pos), tot = __builtin_amdgcn_readlane(incl, 63), ex = incl - pos;
    const u32 used = 8 + (tot & 0xffff) + (tot >> 16);
    u32 need = used;
    if (__builtin_amdgcn_ballot_w64(e4 != 0) != 0) need += 8 * wave_sum(__builtin_popcount(e4));
    const u32 var = need <= 8 * VAR_X ? VAR_X : VAR_W;
    const bool fits = need <= 8 * VAR_W && __builtin_amdgcn_ballot_w64(pos >= (65u << 16)) == 0;
    if (fits) {
        u32 n4 = 0;
        put_bits(area, 8 + (ex & 0xffff), b2);
        put_bits(area, 8 + (tot & 0xffff) + (ex >> 16), b3);
        while (__builtin_amdgcn_ballot_w64(e4 != 0) != 0) {
            const u64 act = __builtin_amdgcn_ballot_w64(e4 != 0);
            const u32 any = (e4 | (e4 >> 8) | (e4 >> 16) | (e4 >> 24)) & 255, t = __builtin_ctz(any | 128);
            const u32 b = __builtin_ctz(((e4 >> t) & 0x01010101u) | 0x80000000u) + t;
            const u32 e = (*(const LAS unsigned char*)(scr + (t & 4) * 256 + lane * 16 + (t & 3) * 4 + ((b >> 3) & 3)) >> 2) & 31;
            const u32 k = var - 1 - n4 - rank_in(act);
            if (e4 != 0) lds_or(area + (k & ~3u), e << ((k & 3) * 8));
            e4 &= ~(1u << (b & 31));
            n4 += __builtin_popcountll(act);
        }
    }
    asm volatile("" ::: "memory");
    u32 vw = *(const LAS u32*)(area + lane * 4);
    if (lane == 0) vw |= fits ? E1 | rare << 5 | (var == VAR_W ? 64 : 0) : 255;
    const u32 P0 = (H[0] & 0x83838383u) | ((H[1] & 0x03030303u) << 2) | ((H[1] >> 1) & 0x40404040u) | ((H[6] & 0x03030303u) << 4);
    const u32 P1 = (H[2] & 0x83838383u) | ((H[3] & 0x03030303u) << 2) | ((H[3] >> 1) & 0x40404040u) | ((H[7] & 0x03030303u) << 4);
    const u32 P2 = (H[4] & 0x83838383u) | ((H[5] & 0x03030303u) << 2) | ((H[5] >> 1) & 0x40404040u)
                 | ((H[6] >> 2) & 0x20202020u) | ((H[7] >> 3) & 0x10101010u);
    before_stores();
    st_sys((u32x4){L[0], L[1], L[2], L[3]}, r0, lane * 16, dst);
    st_sys((u32x4){L[4], L[5], L[6], L[7]}, r0, lane * 16, dst + 1024);
    st_sys((u32x4){P0, P1, P2, K[0]}, r0, lane * 16, dst + 2048);
    __builtin_amdgcn_raw_buffer_store_b32(K[1], r0, lane * 4 + dst + 3072, 0, 17);
    __builtin_amdgcn_raw_buffer_store_b32(vw, r0, lane * 4 + dst + 3328, 0, 17);
}
__device__ __forceinline__ void unpack_item(u32x4 (&h)[4], const u32x4& c0, const u32x4& c1, const u32x4& c2, u32 k1,
                                            const LAS char* area, u32 hd, LAS char* scr, int lane) {
    const u32 E1 = hd & 31, var = (hd & 64) ? VAR_W : VAR_X;
    const bool rare = hd & 32;
    const u32 K[2] = {c2[3], k1};
    const u32 X[2] = {K[0] & (K[0] >> 1) & 0x55555555u, K[1] & (K[1] >> 1) & 0x55555555u};
    const u32 n2 = 2 * (__builtin_popcount(X[0]) + __builtin_popcount(X[1])), i2 = wave_scan(n2);
    u64 b2 = get_bits(area, 8 + i2 - n2);
    b2 &= n2 < 64 ? ~(~0ull << n2) : ~0ull;
    const u64 y2 = b2 & (b2 >> 1) & 0x5555555555555555ull;
    const u32 n3 = 3 * __builtin_popcountll(y2), i3 = wave_scan(n3);
    u64 b3 = get_bits(area, 8 + __builtin_amdgcn_readlane(i2, 63) + i3 - n3);
    const u32 e4x = E1 * 0x04040404u | 0x80808080u;
    const u32 t1 = (e4x - 0x000c0804u) & 0x007c7c7cu, t2 = (e4x - (rare ? 0x00181410u : 0x00001410u)) & 0x007c7c7cu;
    const u32 t3lo = (e4x - (rare ? 0x2824201cu : 0x24201c18u)) & 0x7c7c7c7cu, t3hi = (e4x - (rare ? 0x0000302cu : 0x00302c28u)) & 0x007c7c7cu;
    LAS char* row = scr + lane * 36;
    u32 e4 = 0;
    const u32 nch = (wave_max(n2) + 7) >> 3;
#pragma unroll
    for (int k = 0; k < 8; ++k) {
        if (k >= 4 && k >= nch) break;
        const u32 b8 = (u32)(b2 >> (8 * k)) & 255, y = b8 | (b8 << 12), q = (y | (y << 6)) & 0x03030303u;
        const u32 M2 = q & (q >> 1), P2 = byte_scan(M2), m7 = times7(M2);
        const u32 lo = (u32)b3, z = (lo & 0x3f) | ((lo & 0xfc0) << 10), f = (z | (z << 5)) & 0x07070707u;
        const u32 v = __builtin_amdgcn_perm(0u, f, P2 - M2) & m7;
        b3 >>= (P2 >> 24) * 3;
        *(LAS u32*)(row + 4 * k) = __builtin_amdgcn_perm(0u, t2, q) | __builtin_amdgcn_perm(t3hi, t3lo, v | (0x0c0c0c0cu - (M2 << 2) - (M2 << 3)));
        e4 |= (((v + M2) >> 3) & 0x01010101u) << k;
    }
    u32 n4 = 0;
    while (__builtin_amdgcn_ballot_w64(e4 != 0) != 0) {
        const u64 act = __builtin_amdgcn_ballot_w64(e4 != 0);
        const u32 any = (e4 | (e4 >> 8) | (e4 >> 16) | (e4 >> 24)) & 255, t = __builtin_ctz(any | 128);
        const u32 b = __builtin_ctz(((e4 >> t) & 0x01010101u) | 0x80000000u) + t;
        const u32 x = *(const LAS unsigned char*)(area + ((var - 1 - n4 - rank_in(act)) & 255));
        if (e4 != 0) *(LAS unsigned char*)(row + t * 4 + (b >> 3)) = x << 2;
        e4 &= ~(1u << (b & 31));
        n4 += __builtin_popcountll(act);
    }
    const u32 SM[8] = {c2[0] & 0x83838383u, ((c2[0] << 1) & 0x80808080u) | ((c2[0] >> 2) & 0x03030303u),
                       c2[1] & 0x83838383u, ((c2[1] << 1) & 0x80808080u) | ((c2[1] >> 2) & 0x03030303u),
                       c2[2] & 0x83838383u, ((c2[2] << 1) & 0x80808080u) | ((c2[2] >> 2) & 0x03030303u),
                       ((c2[2] << 2) & 0x80808080u) | ((c2[0] >> 4) & 0x03030303u),
                       ((c2[2] << 3) & 0x80808080u) | ((c2[1] >> 4) & 0x03030303u)};
    u32 M[8], P[8], g[8], at = 0;
#pragma unroll
    for (int t = 0; t < 8; ++t) {
        M[t] = (X[t >> 2] >> (2 * (t & 3))) & 0x01010101u;
        P[t] = byte_scan(M[t]);
        const LAS u32* w = (const LAS u32*)(row + (at & ~3u));
        g[t] = __builtin_amdgcn_alignbyte(w[1], w[0], at);
        at += P[t] >> 24;
    }
#pragma unroll
    for (int t = 0; t < 8; ++t) {
        const u32 c = (K[t >> 2] >> (2 * (t & 3))) & 0x03030303u;
        const u32 m = times7(M[t]);
        const u32 H = SM[t] | __builtin_amdgcn_perm(t1, g[t], ((P[t] - M[t]) & m) | ((c | 0x04040404u) & ~m));
        const u32 L = t < 4 ? c0[t & 3] : c1[t & 3];
        h[t >> 1][(t & 1) * 2] = __builtin_amdgcn_perm(H, L, 0x05010400u);
        h[t >> 1][(t & 1) * 2 + 1] = __builtin_amdgcn_perm(H, L, 0x07030602u);
    }
}

__device__ __forceinline__ void out_windows(const u32x4 (&o)[4], u32& tp, u32& tn) {
    u32 mn = 0;
    int mp = 0;
#pragma unroll
    for (int i = 0; i < 16; ++i) { mp = max(mp, (int)o[i >> 2][i & 3]); mn = umax2(mn, o[i >> 2][i & 3]); }
    tp = wave_max((u32)mp) >> 24;
    tp -= __builtin_popcountll(__builtin_amdgcn_ballot_w64(mp >= (int)(tp << 24))) <= 6;
    tp = (int)tp > 3 ? tp : 3;
    tn = wave_max(mn) >> 24;
    tn = tn > 130 ? tn : 130;
}
__device__ __forceinline__ void pack_out(const u32x4 (&o)[4], u32 tp, u32 tn, LAS char* scr, rsrc_t r0, u32 dst, int lane) {
    u32 M[12], G2 = 0, G1 = 0, em = 0;
    u32x4 E;
#pragma unroll
    for (int r = 0; r < 4; ++r) {
#pragma unroll
        for (int i = 0; i < 3; ++i) M[r * 3 + i] = __builtin_amdgcn_perm(o[r][3], o[r][i], 0x04020100u + (i << 24));
        E[r] = __builtin_amdgcn_perm(o[r][1], o[r][0], 0x0c0c0703u) | __builtin_amdgcn_perm(o[r][3], o[r][2], 0x07030c0cu);
    }
    *(LAS u32x4*)(scr + lane * 16) = E;
#pragma unroll
    for (int r = 0; r < 4; ++r) {
        const u32 sg = E[r] & 0x80808080u, sm = sg - (sg >> 7);
        const u32 top = ((tn & 127) * 0x01010101u & sm) | (tp * 0x01010101u & ~sm);
        const u32 d = ((top | 0x80808080u) - (E[r] & 0x7f7f7f7fu)) ^ 0x80808080u;
        const u32 out = (d | ((d & 0x7f7f7f7fu) + 0x7c7c7c7cu + (sg >> 7))) & 0x80808080u;
        const u32 g = ((d & 0x03030303u) + 0x01010101u + (sg >> 5)) & ~((out >> 4) - (out >> 7));
        G2 |= (g & 0x03030303u) << (2 * r);
        G1 |= (g & 0x04040404u) >> 2 << r;
        em = (em >> 1) | out;
    }
    LAS unsigned char* list = (LAS unsigned char*)(scr + 1024);
    u32 used = 2;
    for (u64 act = __builtin_amdgcn_ballot_w64(em != 0); act != 0; act = __builtin_amdgcn_ballot_w64(em != 0)) {
        const u32 k = used + rank_in(act), b = __builtin_ctz(em | 0x80000000u);
        list[em != 0 && k < 64 ? k : 64] = *(const LAS unsigned char*)(scr + lane * 16 + (b & 3) * 4 + (b >> 3));
        em &= em - 1;
        used += __builtin_popcountll(act);
        if (used > 64) break;
    }
    *(LAS unsigned short*)list = (used > 64 ? 255 : tp) | tn << 8;
    asm volatile("" ::: "memory");
    const u32 ent = *(const LAS u32*)(list + (lane & 15) * 4);
    if (used > 64) {
#pragma unroll
        for (int r = 0; r < 4; ++r) st_sys(o[r], r0, lane * 16 + r * 1024, dst + STG_BYTES);
    }
#pragma unroll
    for (int i = 0; i < 3; ++i) st_sys((u32x4){M[4 * i], M[4 * i + 1], M[4 * i + 2], M[4 * i + 3]}, r0, lane * 16 + i * 1024, dst);
    __builtin_amdgcn_raw_buffer_store_b32(G2, r0, lane * 4 + dst + 3072, 0, 17);
    __builtin_amdgcn_raw_buffer_store_b16((unsigned short)__builtin_amdgcn_perm(0u, G1 | (G1 >> 4), 0x0c0c0200u), r0, lane * 2 + dst + 3328, 0, 17);
    __builtin_amdgcn_raw_buffer_store_b32(ent, r0, (lane < 16 ? lane * 4 : 0x80000000u) + dst + 3456, 0, 17);
}
__device__ __forceinline__ void unpack_out(u32x4 (&o)[4], const u32x4& c0, const u32x4& c1, const u32x4& c2, u32 G2, u32 g1,
                                           const LAS unsigned char* list, u32 tp, u32 tn, LAS char* scr, int lane) {
    const u32 M[12] = {c0[0], c0[1], c0[2], c0[3], c1[0], c1[1], c1[2], c1[3], c2[0], c2[1], c2[2], c2[3]};
    const u32 t = __builtin_amdgcn_perm(0u, g1, 0x0c010c00u), G1 = t | (t << 4);
    const u32 tlo = tp * 0x01010100u - 0x02010000u, thi = tn * 0x01010100u - 0x02010000u + tp - 3;
    u32x4 E;
    u32 em = 0;
#pragma unroll
    for (int r = 0; r < 4; ++r) {
        const u32 g = ((G2 >> (2 * r)) & 0x03030303u) | ((G1 >> r << 2) & 0x04040404u);
        E[r] = __builtin_amdgcn_perm(thi, tlo, g);
        em = (em >> 1) | (~(g + 0x7f7f7f7fu) & 0x80808080u);
    }
    *(LAS u32x4*)(scr + lane * 16) = E;
    u32 used = 2;
    auto step = [&](u32& a, u32& val) {
        const u64 act = __builtin_amdgcn_ballot_w64(em != 0);
        const u32 b = __builtin_ctz(em | 0x80000000u);
        a = em != 0 ? lane * 16 + (b & 3) * 4 + (b >> 3) : 1024;
        val = list[(used + rank_in(act)) & 63];
        em &= em - 1;
        used += __builtin_popcountll(act);
    };
    do {
        u32 a0, a1, a2, x0, x1, x2;
        step(a0, x0); step(a1, x1); step(a2, x2);
        *(LAS unsigned char*)(scr + a0) = x0; *(LAS unsigned char*)(scr + a1) = x1; *(LAS unsigned char*)(scr + a2) = x2;
    } while (__builtin_amdgcn_ballot_w64(em != 0) != 0);
    E = *(const LAS u32x4*)(scr + lane * 16);
#pragma unroll
    for (int r = 0; r < 4; ++r) {
#pragma unroll
        for (int i = 0; i < 3; ++i) o[r][i] = __builtin_amdgcn_perm(E[r], M[r * 3 + i], 0x04020100u + (i << 24));
        o[r][3] = __builtin_amdgcn_perm(M[r * 3 + 1], M[r * 3], 0x0c0c0703u) | __builtin_amdgcn_perm(E[r], M[r * 3 + 2], 0x07030c0cu);
    }
}

__device__ __forceinline__ void load_item(f32x4 (&v)[8], const Params& p, int u, int wid, int lane) {
    const int blk = u >> 4, it = (u & 15) * 8 + wid;
    const bool isx = it < 64;
    const rsrc_t r = mk_rsrc(isx ? p.x : p.w, OUT_BYTES);
    const u32 vf = isx ? (lane & 15) * 32768 + (lane >> 4) * 16 : (lane >> 4) * 16384 + (lane & 15) * 16;
    const u32 base = isx ? it * 524288 + blk * 512 : (blk * 128 + (wid & 3) * 32) * 4096 + ((it - 64) >> 2) * 256;
    const u32 sa = isx ? 64 : 4096, sb = isx ? 256 : 65536;
#pragma unroll
    for (int i = 0; i < 8; ++i)
        v[i] = __builtin_bit_cast(f32x4, __builtin_amdgcn_raw_buffer_load_b128(r, vf, base + (i & 3) * sa + (i >> 2) * sb, IN_AUX));
}
__device__ __forceinline__ void convert_local(const Params& p, LAS char* lds, f32x4 (&v)[8], char* wsl, int wg, int wid, int lane, u32 call) {
    const int rank = rfl(p.rank);
    constexpr int NU = NBLK * 16;
    const u32 par = OFF_PAR + (call & 1) * PAR_BYTES;
    const rsrc_t r0 = mk_rsrc(p.ws[rank], WS_NEED);
    LAS char* scr = lds + wid * 4096;
    auto publish = [&](int u) {
        if (wid == 0) {
            if (lane < 8 && lane != rank) st_flag(wsl + OFF_RDY + rank * 8192 + u * 4, call);
            if (lane == 8) add_sys(p.ws[rank] + OFF_CNT + (u >> 4) * CNT_BLK + ((u & 8) ? 1024 : 0) + rank * 128);
        }
    };
    int u = wg, un = (wg + NGEMM) ^ 8, up = -1;
    WAIT_VM0();
    for (int r = 0; u < NU; ++r) {
        const int blk = u >> 4, grp = u & 15, it = grp * 8 + wid;
        u32x4 h[4];
        u32 base, step;
        if (grp < 8) {
            base = par + (u32)blk * (RS1 * 4096) + it * 4096; step = 1024;
#pragma unroll
            for (int i = 0; i < 4; ++i) h[i] = cvt8(v[2 * i], v[2 * i + 1]);
        } else {
            base = par + REL_B + (u32)blk * (CF * 4096) + ((it - 64) & 60) * 4096 + (wid & 3) * 1024; step = 4096;
#pragma unroll
            for (int c = 0; c < 4; ++c) h[c] = cvt_w(v, c);
        }
        float amax = 0.0f;
#pragma unroll
        for (int i = 0; i < 8; ++i) amax = fmaxf(fmaxf(fmaxf(amax, fabsf(v[i][0])), fmaxf(fabsf(v[i][1]), fabsf(v[i][2]))), fabsf(v[i][3]));
        STAMPB_IF(r == 0, 0);
        if (r > 0) load_item(v, p, (u32)un < (u32)NU ? un : NU - 1, wid, lane);
        u32 early = 0;
        if (blk < PB + RB) {
            const int xi = blk * FX + it - FI;
            if (it < FI1MAX) early = blk >= PB ? 15u << (rank & 4) : 0;
            else if (it >= 64) early = blk >= PB ? 1u << ((~rank & 4) + ((it - 64) >> 4)) : 0;
            else if (it >= FI && xi >= PB * FX && xi < PB * FX + RB * XA) early = 15u << (~rank & 4);
        }
        if (early) {
#pragma unroll
            for (int i = 0; i < 4; ++i) st_sys(h[i], r0, lane * 16, base + i * step);
            WAIT_VM0();
            if (lane < 8 && lane != rank && ((early >> lane) & 1)) st_flag(wsl + OFF_RDY + rank * 8192 + (1536 + blk * 128 + it) * 4, call);
        }
        if (!early || (it >= FI1(0) && it < FI))
        pack_item(h, amax, scr, r0, par + REL_PK + (u32)(blk * 128 + it) * 4096, lane, [&] {
            WAIT_VM0();
            if (r > 0) {
                __builtin_amdgcn_s_barrier();
                if (up >= 0) publish(up);
            }
#pragma unroll
            for (int i = 0; i < 4; ++i) st_sys(h[i], r0, lane * 16, base + i * step);
        });
        up = u;
        if (r == 0) {
            STAMPB_IF(true, 1);
            WAIT_VM0();
            STAMPB_IF(true, 2);
            __builtin_amdgcn_s_barrier();
            STAMP(6);
            publish(u);
            up = -1;
            load_item(v, p, un, wid, lane);
            WAIT_VM0();
        }
        u = un;
        un = (un + NGEMM) ^ 8;
    }
    WAIT_VM0();
    __builtin_amdgcn_s_barrier();
    if (up >= 0) publish(up);
}

constexpr int PUSH_PER = FI1MAX + FX + 64, NPUSH = PB * PUSH_PER, NPW = (NPUSH + NCOMM - 1) / NCOMM;
__device__ __forceinline__ void push_decode(int n, int& blk, int& it) {
    blk = n / PUSH_PER; it = n - blk * PUSH_PER;
    if (it >= FI1MAX) it += FI - FI1MAX;
}
__device__ __forceinline__ void push_role(const Params& p, char* wsl, int n, int lane, u32 call) {
    const int rank = rfl(p.rank);
    const int g = rank >> 2, j = rank & 3, og = (g ^ 1) * 4;
    int blk, it;
    push_decode(n, blk, it);
    f32x4 v[8];
    load_item(v, p, blk * 16 + (it >> 3), it & 7, lane);
    const u32 par = OFF_PAR + (call & 1) * PAR_BYTES;
    u32x4 h[4];
    int d0 = 0, nd = 4, dstep = 1; u32 base, step = 1024;
    if (it < 64) {
#pragma unroll
        for (int i = 0; i < 4; ++i) h[i] = cvt8(v[2 * i], v[2 * i + 1]);
        if (it < FI) { d0 = j + 3; nd = 3; dstep = 3; base = par + (blk * RS1 + 64 + it) * 4096; }
        else base = par + (blk * RS1 + 64 + NI1 + j * FX + it - FI) * 4096;
    } else {
#pragma unroll
        for (int c = 0; c < 4; ++c) h[c] = cvt_w(v, c);
        d0 = (it - 64) >> 4; nd = 1; step = 4096;
        base = par + REL_B + (blk * CF + 64 + 16 * j + (it & 12)) * 4096 + (it & 3) * 1024;
    }
    u32 dmask = 0;
    for (int d = 0; d < nd; ++d) {
        if (it < FI && it >= FI1(d)) continue;
        const int dr = (it < FI ? g * 4 : og) + ((d0 + d * dstep) & 3);
        const rsrc_t rd = mk_rsrc(p.ws[dr], WS_NEED);
#pragma unroll
        for (int i = 0; i < 4; ++i) st_sys(h[i], rd, lane * 16, base + (it < FI ? OFF1(d) * 4096 : 0) + i * step);
        dmask |= 1u << dr;
    }
    WAIT_VM0();
    STAMPB_IF(true, 3);
    if (lane < 8 && ((dmask >> lane) & 1)) add_sys(wsl + OFF_CNT + blk * CNT_BLK + (it < 64 ? 0 : 1024) + rank * 128);
}

#define P_NDEC 9
#define P_DSLEEP 2
constexpr int SCR = 2560;
constexpr int NLW = 3, RING = 6, QLEN = 32, NDEC = P_NDEC;
constexpr int LDS_CTL = NLW * RING * 4096;
constexpr int LDS_DONE = LDS_CTL + QLEN * 16, LDS_TAIL = LDS_DONE + NLW * RING * 4, LDS_SCR = LDS_CTL + 1024;
static_assert(7 * WL <= NLW * NCOMM, "loaders");
constexpr u32 IDX_END = 0xffffffffu, IDX_BAD = 0x80000000u, IDX_RAW = 0x20000000u;
__device__ __forceinline__ u32 lds_get(const LAS char* a) {
    u32 v;
    asm volatile("ds_read_b32 %0, %1\n\ts_waitcnt lgkmcnt(0)" : "=v"(v) : "v"((u32)(size_t)a) : "memory");
    return rfl(v);
}
__device__ __forceinline__ void lds_put(LAS char* a, u32 v) {
    asm volatile("" ::: "memory");
    *(volatile LAS u32*)a = v;
    asm volatile("" ::: "memory");
}

struct Item { u32 idx, dst; };
constexpr int NT_I = NBLK * FI;
constexpr int PER = 16 + XA, NP = NBLK * PER, XOFF = PB * (FX - XA), NX = NBLK * (16 + FX) - XOFF, NO1 = (RF1 - FX) * 4, NT_C = NX + NO1 + 192;
constexpr int TAB_BYTES = (NT_I > NT_C ? NT_I : NT_C) * 4, LDS_TAB = LDS_SCR + NDEC * SCR;
static_assert(LDS_TAB + NLW * TAB_BYTES <= LDS_BYTES, "communication LDS");
constexpr int TAB_HEAD = 128;
static_assert(PB * (PER > FI1MAX ? PER : FI1MAX) + 4 * WL <= TAB_HEAD, "table head");
__device__ __forceinline__ void build_table(LAS char* tab, int rank, int slot, int first, int stride, int end) {
    const int g = rank >> 2, j = rank & 3, q = slot - 3;
    const u32 rc1 = (1u << 20) / FI1(slot < 3 ? slot : 0) + 1, rc2 = (1u << 20) / F2(slot < 3 ? slot : 0) + 1;
    for (int it = first; it < min(end, slot < 3 ? NT_I : NT_C); it += stride) {
        Item m;
        if (slot < 3) {
            const int f1 = FI1(slot), f2 = F2(slot);
            const bool s2 = it >= NBLK * f1;
            const int a = s2 ? it - NBLK * f1 : it, blk = (a * (s2 ? rc2 : rc1)) >> 20, s = a - blk * (s2 ? f2 : f1) + (s2 ? f1 : 0);
            m.idx = blk * 128 + s;
            m.dst = (s2 ? REL_A2 + (OFF2(slot) - f1) * 4096 : (64 + OFF1(slot)) * 4096) + (blk * (s2 ? RF2 : RS1) + s) * 4096;
        } else if (it < NX) {
            const int blk = it / PER, c = it - blk * PER;
            const bool tail = it >= NP, isx = tail || (c < 2 * XA && (c & 1));
            const int cw = c < 2 * XA ? c >> 1 : c - XA;
            const int xi = XOFF + (tail ? it - NP + NBLK * XA : blk * XA + (c >> 1)), xb = xi / FX, xs = FI + xi - xb * FX;
            m.idx = isx ? xb * 128 + xs : blk * 128 + 64 + 16 * j + cw;
            m.dst = isx ? (xb * RS1 + xs + 64 + NI1 + q * FX - FI) * 4096
                        : REL_B + (blk * CF + 64 + 16 * q + (cw & 12)) * 4096 + (cw & 3) * 1024;
        } else {
            const int k = it - NX;
            int row, col;
            if (k < NO1) {
                const int f = k >> 2;
                m.idx = 8192 + (f >> 4) * 64 + (k & 2) * 16 + ((f >> 2) & 3) * 8 + (k & 1) * 4 + (f & 3);
                row = row_of_slot(slot_of(f, j), g ^ 1, q);
                col = (k & 3) * 64;
            } else {
                const int z = (k - NO1 + RF2 * 4) % 192;
                m.idx = 8192 + 768 + k - NO1;
                row = row_of_f2((z >> 5) * 8 + (z & 7), g ^ 1, q);
                col = ((z >> 3) & 3) * 64;
            }
            m.dst = (u32)row * 4096 + (q * 256 + col) * 4;
        }
        *(LAS u32*)(tab + it * 4) = m.idx | m.dst << (m.idx < 8192 ? 4 : 6);
    }
}

template <bool CROSS>
__device__ __forceinline__ void load_role(const Params& p, LAS char* lds, int ld, int nld, int slot, int wv, int lane, u32 call) {
    const int rank = rfl(p.rank);
    const int g = rank >> 2, j = rank & 3;
    const int peer = CROSS ? (g ^ 1) * 4 + slot - 3 : g * 4 + ((j + 1 + slot) & 3);
    constexpr int NT = CROSS ? NT_C : NT_I;
    const u32 par = OFF_PAR + (call & 1) * PAR_BYTES;
    char* ws = uni(p.ws[rank]);
    const rsrc_t r0 = rsrc_u(ws, WS_NEED), rp = mk_rsrc(p.ws[peer], WS_NEED);
    const char* rdy = ws + OFF_RDY + peer * 8192;
    const u32 vo = lane * 16;
    const int SKIP = PB * (CROSS ? PER : FI1(slot));
    const u32 head = lane == 0 ? OFF_QUE + slot * 256 : WS_NEED, tbase = SKIP + 4 * WL - (call - 1) * (NT - SKIP);
    const bool nowait = (p.pad & 4) != 0;
    const LAS char* tab = lds + LDS_TAB + ld * TAB_BYTES;
    const LAS char* done = lds + LDS_DONE + ld * RING * 4;
    const u32 ldsb = rfl((u32)(size_t)lds) + ld * RING * 4096;
    auto table = [&](int it) {
        asm volatile("ds_read_b32 v160, %0" :: "v"((u32)(size_t)tab + (it < NT ? it : NT - 1) * 4) : "memory", "v160");
    };
    auto entry = [&] {
        u32 t;
        asm volatile("s_waitcnt lgkmcnt(0)\n\tv_readfirstlane_b32 %0, v160" : "=s"(t) :: "memory");
        Item m;
        m.idx = t & 16383;
        m.dst = (t - m.idx) >> (CROSS && m.idx >= 8192 ? 6 : 4);
        return m;
    };
    const int RAWEND = SKIP + RB * (CROSS ? PER : FI1(slot));
    auto mark = [&](Item m, int it) { if (RB > 0 && it < RAWEND) m.idx |= IDX_RAW; return m; };
    auto decode = [&](int it) { table(it); return mark(entry(), it); };
    auto flag_lean = [&](u32 idx) { return idx < 8192 ? idx >> 3 : (idx < 8192 + 768 ? 1024 + ((idx - 8192) >> 2) : idx - (8192 + 768 - 1024 - 192)); };
    auto flag_of = [&](u32 idx) { return (idx & IDX_RAW) ? 1536 + (idx & 511) : flag_lean(idx); };
    u32 okupto = 0, wbase = 0; u64 wmask = 0;
    auto seen = [&](u32 fi) { return fi < okupto || (fi - wbase < 64u && ((wmask >> (fi - wbase)) & 1)); };
    auto ready = [&](u32 fi) {
        if (seen(fi)) return true;
        for (;;) {
            wbase = okupto;
            wmask = __builtin_amdgcn_ballot_w64(ld_sys(rdy + (wbase + lane) * 4) >= call);
            if (wmask != ~0ull) break;
            okupto += 64;
            if (fi < okupto) return true;
        }
        okupto += __builtin_ctzll(~wmask);
        if (fi - wbase < 64u) return ((wmask >> (fi - wbase)) & 1) != 0;
        return rfl(ld_sys(rdy + fi * 4)) >= call;
    };
    auto ready_early = [&](u32 fi) { return fi >= 1536 ? rfl(ld_sys(rdy + fi * 4)) >= call : ready(fi); };
    const u32 vlean = vo < 256 + VAR_X ? vo : 0x80000000u;
    struct Prep { u32 s0, s1, s2, s3, m0v, vl; };
    auto prep = [&](const Item& m, u32 n) {
        Prep r;
        const bool out = CROSS && m.idx >= 8192 && m.idx < IDX_RAW;
        u32 d = 0, rest = 256 + VAR_X;
        r.s0 = par + REL_PK + (m.idx + (out ? j * 1024 : 0)) * 4096;
        if (m.idx & IDX_RAW) {
            const bool isw = CROSS && (m.idx & 64);
            const u32 blk = (m.idx >> 7) & 3, s = m.idx & 127;
            r.s0 = par + (isw ? REL_B + (blk * CF + (s & 60)) * 4096 + (s & 3) * 1024 : (blk * RS1 + s) * 4096);
            d = isw ? 3072 : 0; rest = 1024;
        }
        r.s1 = r.s0 + d; r.s2 = r.s1 + d; r.s3 = r.s2 + d;
        r.m0v = ldsb + (n % RING) * 4096;
        r.vl = vo < rest ? vo : 0x80000000u;
        asm volatile("" : "+s"(r.s0), "+s"(r.s1), "+s"(r.s2), "+s"(r.s3), "+s"(r.m0v), "+v"(r.vl));
        return r;
    };
    auto prep_lean = [&](const Item& m, u32 n) {
        Prep r;
        r.s0 = par + REL_PK + (m.idx + (CROSS && m.idx >= 8192 ? j * 1024 : 0)) * 4096;
        r.m0v = ldsb + (n % RING) * 4096;
        asm volatile("" : "+s"(r.s0), "+s"(r.m0v));
        r.s1 = r.s2 = r.s3 = r.s0; r.vl = vlean;
        return r;
    };
    auto issue = [&](const Prep& r) {
        asm volatile("s_mov_b32 m0, %0\n\ts_nop 0\n\t"
                     "buffer_load_dwordx4 %1, %3, %4 offen sc0 sc1 lds\n\t"
                     "buffer_load_dwordx4 %1, %3, %5 offen offset:1024 sc0 sc1 lds\n\t"
                     "buffer_load_dwordx4 %1, %3, %6 offen offset:2048 sc0 sc1 lds\n\t"
                     "buffer_load_dwordx4 %2, %3, %7 offen offset:3072 sc0 sc1 lds"
                     :: "s"(r.m0v), "v"(vo), "v"(r.vl), "s"(rp), "s"(r.s0), "s"(r.s1), "s"(r.s2), "s"(r.s3) : "memory");
    };
    u32 vtail = (u32)(size_t)lds + LDS_TAIL, vone = 1;
    asm volatile("" : "+v"(vtail), "+v"(vone));
    auto enqueue = [&](u32 idx, u32 dst, u32 n) {
        asm volatile("s_mov_b64 exec, 1\n\tds_add_rtn_u32 v162, %0, %1\n\tv_mov_b32 v159, %2\n\tv_mov_b32 v160, %3\n\tv_mov_b32 v161, %4\n\t"
                     "s_waitcnt lgkmcnt(0)\n\tv_add_u32 v158, 1, v162\n\tv_and_b32 v162, %6, v162\n\tv_lshl_add_u32 v162, v162, 4, %5\n\t"
                     "ds_write_b128 v162, v[158:161]\n\ts_mov_b64 exec, -1"
                     :: "v"(vtail), "v"(vone), "s"(idx), "s"(dst), "s"(n << 8 | ld << 4 | peer), "s"(rfl((u32)(size_t)lds) + LDS_CTL), "n"(QLEN - 1)
                     : "memory", "v158", "v159", "v160", "v161", "v162");
    };
    __builtin_amdgcn_s_setprio(3);
    int e = SKIP + wv, e1 = e + WL, e2 = e + 2 * WL;
    const int e3 = e + 3 * WL;
    Item nx = decode(e1), ia = decode(e);
    bool bad = spin_ge<2>(rdy + flag_of(ia.idx) * 4, call, ws + OFF_TMO, call);
    STAMP(5);
    issue(prep(ia, 0));
    bool pre = ready_early(flag_of(nx.idx));
    u32 n = 0;
    auto wait_free = [&](u32 m) { while (lds_get(done + (m % RING) * 4) + RING <= m) __builtin_amdgcn_s_sleep(1); };
    auto body = [&](auto odd, auto early) {
        if constexpr (odd.value) asm volatile("v_mov_b32 v164, 1\n\ts_nop 0\n\tbuffer_atomic_add v164, %0, %1, 0 offen sc0" :: "v"(head), "s"(r0) : "memory", "v164");
        else                     asm volatile("v_mov_b32 v165, 1\n\ts_nop 0\n\tbuffer_atomic_add v165, %0, %1, 0 offen sc0" :: "v"(head), "s"(r0) : "memory", "v165");
        if (early.value && n == 2) while (lds_get(lds + LDS_TAIL + 12) < NDEC - NPW) __builtin_amdgcn_s_sleep(1);
        table(e2);
        u32 dn;
        asm volatile("ds_read_b32 v162, %1\n\ts_waitcnt lgkmcnt(0)\n\tv_readfirstlane_b32 %0, v162"
                     : "=s"(dn) : "v"((u32)(size_t)done + ((n + 1) % RING) * 4) : "memory", "v162");
        const Item nn = early.value ? mark(entry(), e2) : entry();
        bool pre2 = e2 < NT;
        if (pre2 && !(nowait && nn.idx >= 8192 && nn.idx < IDX_RAW)) pre2 = early.value ? ready_early(flag_of(nn.idx)) : ready(flag_lean(nn.idx));
        if (!pre && e1 < NT) pre = seen(early.value ? flag_of(nx.idx) : flag_lean(nx.idx));
        if (dn + RING <= n + 1) wait_free(n + 1);
        const Prep px = early.value ? prep(nx, n + 1) : prep_lean(nx, n + 1);
        u32 told;
        if constexpr (odd.value) asm volatile("s_waitcnt vmcnt(1)\n\tv_readfirstlane_b32 %0, v165" : "=s"(told) :: "memory");
        else                     asm volatile("s_waitcnt vmcnt(1)\n\tv_readfirstlane_b32 %0, v164" : "=s"(told) :: "memory");
        if (pre) issue(px);
        enqueue(ia.idx | (bad ? IDX_BAD : 0), ia.dst, n);
        if (!pre && e1 < NT) {
            bad |= spin_ge<2>(rdy + flag_of(nx.idx) * 4, call, ws + OFF_TMO, call);
            issue(px);
        }
        STAMP_IF(n == 0, 6);
        STAMP_IF(CROSS ? (e < NP && e1 >= NP) : (e < NBLK * FI1(slot) && e1 >= NBLK * FI1(slot)), 1);
        STAMP_IF(CROSS && e < NX && e1 >= NX, 2);
        STAMP_IF(CROSS && e < NX + NO1 && e1 >= NX + NO1, 3);
        ia = nx; nx = nn; pre = pre2; ++n;
        e = e1; e1 = e2; e2 = early.value && n == 1 ? e3 : (int)(told + tbase);
    };
    body(std::false_type{}, std::true_type{});
    body(std::true_type{}, std::true_type{});
    body(std::false_type{}, std::true_type{});
    for (;;) {
        body(std::true_type{}, std::false_type{});
        if (e >= NT) break;
        body(std::false_type{}, std::false_type{});
        if (e >= NT) break;
    }
    u32 fin;
    asm volatile("ds_add_rtn_u32 v162, %1, %2\n\ts_waitcnt lgkmcnt(0)\n\tv_readfirstlane_b32 %0, v162"
                 : "=s"(fin) : "v"((u32)(size_t)lds + LDS_TAIL + 4), "v"(lane == 0 ? 1u : 0u) : "memory", "v162");
    if (fin + 1 == (u32)nld)
        for (int k = 0; k < NDEC; ++k) enqueue(IDX_END, 0, 0);
}

__device__ __forceinline__ void decode_role(const Params& p, LAS char* lds, u32 me, int lane, u32 call) {
    const int rank = rfl(p.rank);
    const int j = rank & 3;
    const u32 par = OFF_PAR + (call & 1) * PAR_BYTES;
    const rsrc_t r0 = mk_rsrc(p.ws[rank], WS_NEED), ro = mk_rsrc(p.out[rank], OUT_BYTES);
    const u32 vo = lane * 16, vout = (lane >> 4) * 4 * 4096 + (lane & 15) * 16;
    LAS char* scr = lds + LDS_SCR + me * SCR;
    u32 q0, q1, q2, q3;
    auto peek = [&](u32 n) {
        asm volatile("ds_read_b128 v[158:161], %4\n\ts_waitcnt lgkmcnt(0)\n\tv_readfirstlane_b32 %0, v158\n\tv_readfirstlane_b32 %1, v159\n\t"
                     "v_readfirstlane_b32 %2, v160\n\tv_readfirstlane_b32 %3, v161"
                     : "=s"(q0), "=s"(q1), "=s"(q2), "=s"(q3) : "v"((u32)(size_t)lds + LDS_CTL + (n % QLEN) * 16) : "memory", "v158", "v159", "v160", "v161");
        return q0 == n + 1;
    };
    const u32 vsig = lane == 0 ? 0u : 0x80000000u;
    u32 one = 1;
    asm volatile("" : "+v"(one));
    auto signal = [&](u32 c) {
        asm volatile("s_nop 4\n\tbuffer_atomic_add %0, %1, %2, %3 offen sc1" :: "v"(one), "v"(vsig), "s"(r0), "s"(rfl(c)) : "memory");
    };
    const u32x4 nan4 = {0x7fc07fc0u, 0x7fc07fc0u, 0x7fc07fc0u, 0x7fc07fc0u};
    u32 vhead = (u32)(size_t)lds + LDS_TAIL + 8, vone = 1;
    asm volatile("" : "+v"(vhead), "+v"(vone));
    for (;;) {
        u32 n;
        asm volatile("s_mov_b64 exec, 1\n\tds_add_rtn_u32 v158, %1, %2\n\ts_waitcnt lgkmcnt(0)\n\tv_readfirstlane_b32 %0, v158\n\ts_mov_b64 exec, -1"
                     : "=s"(n) : "v"(vhead), "v"(vone) : "memory", "v158");
        while (!peek(n)) __builtin_amdgcn_s_sleep(P_DSLEEP);
        if (q1 == IDX_END) break;
        const u32 dst = q2, item = q3 >> 8, ring = (((q3 >> 4) & 3) * RING + item % RING), peer = q3 & 7;
        const LAS char* sl = lds + ring * 4096;
        const u32x4 c0 = *(const LAS u32x4*)(sl + vo), c1 = *(const LAS u32x4*)(sl + 1024 + vo), c2 = *(const LAS u32x4*)(sl + 2048 + vo);
        const u32 k1 = *(const LAS u32*)(sl + 3072 + lane * 4);
        const u32 hx = *(const LAS unsigned char*)(sl + 3328);
        const bool bad = (q1 & IDX_BAD) != 0;
        const u32 idx = q1 & 0xffff;
        const bool out = idx >= 8192, isw = !out && (idx & 64);
        const u32 step = (out || isw) ? 4096 : 1024;
        u32x4 h[4];
        bool done = false;
        if (!bad && (q1 & IDX_RAW)) {
            h[0] = c0; h[1] = c1; h[2] = c2; h[3] = *(const LAS u32x4*)(sl + 3072 + vo); done = true;
        } else if (!bad && !out) {
            const u32 hd = rfl(hx);
            if (hd != 255) {
                if (hd & 64) {
                    u32 t;
                    asm volatile("s_nop 4\n\tbuffer_load_dword %0, %1, %2, %3 offen sc0 sc1\n\ts_waitcnt vmcnt(0)"
                                 : "=&v"(t) : "v"(lane < 16 ? lane * 4 : 0x80000000u), "s"(mk_rsrc(p.ws[peer], WS_NEED)),
                                   "s"(rfl(par + REL_PK + idx * 4096 + 3328 + VAR_X)) : "memory");
                    if (lane < 16) *(LAS u32*)(lds + ring * 4096 + 3328 + VAR_X + lane * 4) = t;
                    asm volatile("" ::: "memory");
                }
                unpack_item(h, c0, c1, c2, k1, sl + 3328, hd, scr, lane);
                done = true;
            }
        } else if (!bad) {
            const u32 g1 = *(const LAS unsigned short*)(sl + 3328 + lane * 2), hd = rfl(*(const LAS unsigned short*)(sl + 3456));
            if ((hd & 255) != 255) { unpack_out(h, c0, c1, c2, k1, g1, (const LAS unsigned char*)(sl + 3456), hd & 255, hd >> 8, scr, lane); done = true; }
        }
        if (!done) {
            const u32 blk = idx >> 7, s = idx & 127;
            const u32 raw = par + (out ? REL_PK + STG_BYTES + (idx + j * 1024) * 4096
                                       : isw ? REL_B + (blk * CF + (s & 60)) * 4096 + (s & 3) * 1024 : (blk * RS1 + s) * 4096);
            const u32 rs = isw ? 4096 : 1024;
            const rsrc_t rp = mk_rsrc(p.ws[peer], WS_NEED);
            asm volatile("s_nop 4\n\tbuffer_load_dwordx4 %0, %4, %5, %6 offen sc0 sc1\n\tbuffer_load_dwordx4 %1, %4, %5, %7 offen sc0 sc1\n\t"
                         "buffer_load_dwordx4 %2, %4, %5, %8 offen sc0 sc1\n\tbuffer_load_dwordx4 %3, %4, %5, %9 offen sc0 sc1\n\ts_waitcnt vmcnt(0)"
                         : "=&v"(h[0]), "=&v"(h[1]), "=&v"(h[2]), "=&v"(h[3])
                         : "v"(vo), "s"(rp), "s"(rfl(raw)), "s"(rfl(raw + rs)), "s"(rfl(raw + 2 * rs)), "s"(rfl(raw + 3 * rs)) : "memory");
            if (bad) { h[0] = nan4; h[1] = nan4; h[2] = nan4; h[3] = nan4; }
        }
        lds_put(lds + LDS_DONE + ring * 4, item + 1);
        const rsrc_t rd = out ? ro : r0;
        const u32 vd = (out ? vout : vo + par) + dst;
#pragma unroll
        for (int i = 0; i < 4; ++i) st_sys(h[i], rd, vd, i * step);
        if (!out) {
            asm volatile("s_waitcnt vmcnt(0)" ::: "memory");
            signal(OFF_CNT + (idx >> 7) * CNT_BLK + peer * 128 + (isw ? 1024 : dst >= REL_A2 ? 2048 : 0));
        }
    }
    STAMP(4);
}

#define GLDS2(m0v, voff, rs, soff, imm0, imm1) \
    asm volatile("s_mov_b32 m0, %0\n\ts_nop 0\n\tbuffer_load_dwordx4 %1, %2, %3 offen offset:" #imm0 " sc0 sc1 lds\n\t" \
                 "buffer_load_dwordx4 %1, %2, %3 offen offset:" #imm1 " sc0 sc1 lds" \
                 :: "s"(m0v), "v"(voff), "s"(rs), "s"(soff) : "memory")
#define BARRIER() do { __builtin_amdgcn_sched_barrier(0); __builtin_amdgcn_s_barrier(); __builtin_amdgcn_sched_barrier(0); } while (0)

#define SCHED() __builtin_amdgcn_sched_barrier(0)
#define WAIT_LGKM0() __builtin_amdgcn_s_waitcnt(0xC07F)

template <int STAGE>
struct Gemm {
    static constexpr int NT = STAGE == 1 ? 128 : 64;
    static constexpr int NA = STAGE == 1 ? 4 : 2;
    static constexpr int NB = STAGE == 1 ? 4 : 8;
    static constexpr int NM = STAGE == 1 ? 4 : 1;
    static constexpr int NRD = NA + NB;
    struct Frags { f16x8 a[NA], b[NB]; };

    rsrc_t rs;
    u32 sa, sa2, sb;
    u32 ablk;
    u32 ldsw;
    int wid, lane;

    template <int I> __device__ __forceinline__ void piece(int t, int buf) const {
        const u32 vo = lane * 16;
        const u32 ma = ldsw + buf * BUF_BYTES + wid * 4096, mb = ldsw + buf * BUF_BYTES + 32768 + wid * 2048;
        const u32 ka = STAGE == 1 ? (u32)(t >> 1) * (RS1 * 4096) + (t & 1) * 2048 : (u32)t * ablk;
        const u32 a = sa + ka, a2 = sa2 + ka;
        const u32 b = STAGE == 1 ? sb + (u32)(t >> 1) * (CF * 4096) + (t & 1) * 2048 : sb + (u32)t * (CF * 4096);
        if (I == 0) GLDS2(ma, vo, rs, a, 0, 1024);
        if (I == 1) GLDS2(ma, vo, rs, a2, 2048, 3072);
        if (I == 2) GLDS2(mb, vo, rs, b, 0, 1024);
    }
    __device__ __forceinline__ void stage(int t, int buf) const { piece<0>(t, buf); piece<1>(t, buf); piece<2>(t, buf); }

    template <int I> __device__ __forceinline__ void rd(Frags& f, const LAS char* lds, int buf, int half) const {
        const LAS char* base = lds + buf * BUF_BYTES + lane * 16;
        if constexpr (STAGE == 1) {
            const LAS char* pa = base + (wid >> 1) * 8192 + half * 1024;
            const LAS char* pb = base + 32768 + (wid & 1) * 8192 + half * 1024;
            if constexpr (I == 0) f.a[0] = *(const LAS f16x8*)pa;
            else if constexpr (I <= 4) f.b[I - 1] = *(const LAS f16x8*)(pb + (I - 1) * 2048);
            else f.a[I - 4] = *(const LAS f16x8*)(pa + (I - 4) * 2048);
        } else {
            const LAS char* pa = base + wid * 4096 + half * 2048;
            const LAS char* pb = base + 32768 + half * 2048;
            constexpr int h = I / 5, n = I % 5;
            if constexpr (n == 0) f.a[h] = *(const LAS f16x8*)(pa + h * 1024);
            else f.b[h * 4 + n - 1] = *(const LAS f16x8*)(pb + (n - 1) * 4096 + h * 1024);
        }
    }
    template <int I> __device__ __forceinline__ void mm(f32x4 (&acc)[NM][4], const Frags& f) const {
        if constexpr (STAGE == 1) acc[I >> 2][I & 3] = __builtin_amdgcn_mfma_f32_16x16x32_f16(f.a[I >> 2], f.b[I & 3], acc[I >> 2][I & 3], 0, 0, 0);
        else            acc[0][I & 3] = __builtin_amdgcn_mfma_f32_16x16x32_f16(f.a[I >> 2], f.b[I], acc[0][I & 3], 0, 0, 0);
    }
    template <int I, int N> __device__ __forceinline__ void rds(Frags& f, const LAS char* lds, int buf, int half) const {
        if constexpr (I < N) { rd<I>(f, lds, buf, half); rds<I + 1, N>(f, lds, buf, half); }
    }
    static __device__ __forceinline__ int blk_of(int t) { return STAGE == 1 ? (t >> 1) : t; }

    template <bool RD, int ST>
    __device__ __forceinline__ void half_step(f32x4 (&acc)[NM][4], const Frags& cur, Frags& nxt, const LAS char* lds, int rbuf, int rhalf,
                                              int ts, int bs, u32 cvo, u32 fso) const {
        auto first = [&] {
            if (ST == 1) asm volatile("buffer_load_dword v166, %0, %1, %2 offen sc0 sc1" :: "v"(cvo), "s"(rs), "s"(fso) : "memory", "v166");
            if (ST) { piece<0>(ts, bs); SCHED(); }
        };
        if constexpr (STAGE == 1) {
            mm<0>(acc, cur);  mm<1>(acc, cur);  if (RD) rds<0, 2>(nxt, lds, rbuf, rhalf); SCHED();
            mm<2>(acc, cur);  mm<3>(acc, cur);  if (RD) rds<2, 4>(nxt, lds, rbuf, rhalf); SCHED();
            mm<4>(acc, cur);  mm<5>(acc, cur);  if (RD) rds<4, 6>(nxt, lds, rbuf, rhalf); SCHED();
            mm<6>(acc, cur);  mm<7>(acc, cur);  if (RD) rds<6, 8>(nxt, lds, rbuf, rhalf); SCHED();
            mm<8>(acc, cur);  mm<9>(acc, cur);  SCHED();
            first();
            mm<10>(acc, cur); mm<11>(acc, cur); SCHED();
            if (ST) { piece<1>(ts, bs); SCHED(); }
            mm<12>(acc, cur); mm<13>(acc, cur); SCHED();
            if (ST) { piece<2>(ts, bs); SCHED(); }
            mm<14>(acc, cur); mm<15>(acc, cur); SCHED();
        } else {
            first();
            mm<0>(acc, cur); SCHED();
            if (ST) { piece<1>(ts, bs); SCHED(); }
            mm<1>(acc, cur); SCHED();
            if (ST) { piece<2>(ts, bs); SCHED(); }
            mm<2>(acc, cur); if (RD) rds<0, 3>(nxt, lds, rbuf, rhalf); SCHED();
            mm<3>(acc, cur); if (RD) rds<3, 5>(nxt, lds, rbuf, rhalf); SCHED();
            mm<4>(acc, cur); if (RD) rds<5, 8>(nxt, lds, rbuf, rhalf); SCHED();
            mm<5>(acc, cur); if (RD) rds<8, 10>(nxt, lds, rbuf, rhalf); SCHED();
            mm<6>(acc, cur); mm<7>(acc, cur); SCHED();
        }
    }

    template <int ROLE>
    __device__ __forceinline__ u32 run(f32x4 (&acc)[NM][4], const LAS char* lds, char* ws, u32 cvo, u32 need, u32 call) const {
        constexpr bool LATE = ROLE == 2 && STAGE == 1;
        char* tmo = ws + OFF_TMO;
        Frags f0, f1;
        if (ROLE == 0) spin_ge(ws + __shfl(cvo, lane & 15) + blk_of(lane < 48 ? lane >> 4 : 0) * CNT_BLK, __shfl(need, lane & 15), tmo, call);
        BARRIER();
        stage(0, 0);
        stage(1, 1);
        if (!LATE) { stage(2, 2); asm volatile("s_waitcnt vmcnt(12)" ::: "memory"); }
        else asm volatile("s_waitcnt vmcnt(6)" ::: "memory");
        BARRIER();
        rds<0, NRD>(f0, lds, 0, 0);
        WAIT_LGKM0();
        if (ROLE == 0) asm volatile("v_mov_b32 v166, 0" ::: "v166");
        int buf = 0;
        for (int t = 0; t < NT - 3; ++t) {
            const int nb = buf == 2 ? 0 : buf + 1, pb = buf == 0 ? 2 : buf - 1;
            half_step<true, LATE ? 2 : 0>(acc, f0, f1, lds, buf, 1, t + 2, pb, 0, 0);
            if (ROLE == 0) {
                u64 miss;
                asm volatile("s_waitcnt vmcnt(6)\n\tv_cmp_lt_u32_e64 %0, v166, %1\n\ts_nop 0" : "=s"(miss) : "v"(need) : "memory");
                WAIT_LGKM0();
                if (miss != 0) spin_ge(ws + cvo + blk_of(t + 3) * CNT_BLK, need, tmo, call);
            } else {
                asm volatile("s_waitcnt vmcnt(6)" ::: "memory");
                WAIT_LGKM0();
            }
            BARRIER();
            half_step<true, LATE ? 0 : (ROLE == 0 ? 1 : 2)>(acc, f1, f0, lds, nb, 0, t + 3, buf, cvo, blk_of(t + 4 < NT ? t + 4 : t + 3) * CNT_BLK);
            buf = nb;
        }
#pragma unroll
        for (int t = NT - 3; t < NT; ++t) {
            const int nb = buf == 2 ? 0 : buf + 1, pb = buf == 0 ? 2 : buf - 1;
            if (t == NT - 3) {
                half_step<true, LATE ? 2 : 0>(acc, f0, f1, lds, buf, 1, t + 2, pb, 0, 0);
                asm volatile("s_waitcnt vmcnt(6)" ::: "memory");
            } else {
                if (t == NT - 2) {
                    const u32 to = OFF_TMO;
                    asm volatile("buffer_load_dword v167, %0, %1, 0 offen sc1" :: "v"(to), "s"(rs) : "memory", "v167");
                }
                half_step<true, 0>(acc, f0, f1, lds, buf, 1, 0, 0, 0, 0);
                asm volatile("s_waitcnt vmcnt(0)" ::: "memory");
            }
            WAIT_LGKM0();
            BARRIER();
            if (t < NT - 1) half_step<true, 0>(acc, f1, f0, lds, nb, 0, 0, 0, 0, 0);
            else            half_step<false, 0>(acc, f1, f0, lds, nb, 0, 0, 0, 0, 0);
            buf = nb;
        }
        u32 tw;
        asm volatile("v_readfirstlane_b32 %0, v167\n\ts_nop 1" : "=s"(tw) :: "memory");
        return tw;
    }
};

__device__ __forceinline__ void activate(u32x4 (&o)[4], f32x4 (&c)[4], bool bad) {
    if (bad) {
        asm volatile("");
#pragma unroll
        for (int n = 0; n < 4; ++n) c[n] = __builtin_bit_cast(f32x4, (u32x4){0x7fc00000u, 0x7fc00000u, 0x7fc00000u, 0x7fc00000u});
    }
#pragma unroll
    for (int n = 0; n < 4; ++n)
#pragma unroll
        for (int r = 0; r < 4; r += 2) {
            const u32x2v y = __builtin_bit_cast(u32x2v, gelu2((f32x2){c[n][r], c[n][r + 1]}));
            o[r][n] = y[0]; o[r + 1][n] = y[1];
        }
}

__device__ __forceinline__ void gemm_role(const Params& p, const LAS char* lds, int xcd, int s, int wid, int lane, u32 call) {
    const int rank = rfl(p.rank);
    const int g = rank >> 2, j = rank & 3, og = (g ^ 1) * 4;
    char* ws = p.ws[rank];
    const u32 par = OFF_PAR + (call & 1) * PAR_BYTES;
    const rsrc_t rs = mk_rsrc(ws, WS_NEED), ro = mk_rsrc(p.out[rank], OUT_BYTES);
    const u32 ldsw = rfl((u32)(size_t)lds);
    const u32 vout = (lane >> 4) * 4 * 4096 + (lane & 15) * 16;
    {
        const int u = (xcd & 1) * (xcd < 4 ? 26 : 24) + s;
        const bool extra = xcd >= 4 && s >= 24;
        const int rt = extra ? 13 : u >> 2, ct = extra ? (xcd - 4) * 2 + (s & 1) : (xcd >> 1) * 4 + (u & 3);
        const int cq = ct < 8 ? -1 : (ct - 8) >> 1;
        Gemm<1> G;
        G.rs = rs; G.ldsw = ldsw; G.wid = wid; G.lane = lane;
        G.sa = par + max(slot_of(rt * 16 + wid * 2, cq), 0) * 4096;
        G.sa2 = par + max(slot_of(rt * 16 + wid * 2 + 1, cq), 0) * 4096 - 2048;
        G.sb = par + REL_B + (ct * 8 + wid) * 4096;
        const int bsrc = ct < 8 ? rank : og + cq;
        u32 cvo = OFF_CNT + 1024 + bsrc * 128, need = (ct < 8 ? 8u : 16u) * call;
        const int sl16 = max(slot_of(rt * 16 + (lane & 15), cq), 0), src16 = row_of_slot(sl16, g, j) / NS;
        u32 mask = 0;
        for (int i = 0; i < 8; ++i) mask |= (__builtin_amdgcn_ballot_w64(src16 == i) != 0) << i;
        if (lane < 8 && ((mask >> lane) & 1)) {
            cvo = OFF_CNT + lane * 128;
            need = (lane == rank ? 8 : ((lane >> 2) == g ? FI1((lane - j - 1) & 3) : FX)) * call;
        }
        f32x4 acc[4][4] = {};
        const bool bad = call == (wid == 0 ? G.template run<0>(acc, lds, ws, cvo, need, call)
                                  : wid < 4 ? G.template run<1>(acc, lds, ws, cvo, need, call)
                                            : G.template run<2>(acc, lds, ws, cvo, need, call));
        STAMP(1);
        const int wm = wid >> 1, wn = wid & 1;
        if (ct < 8) {
#pragma unroll
            for (int m = 0; m < 4; ++m) {
                const int sl = slot_of(rt * 16 + wm * 4 + m, -1);
                if (sl < 0) continue;
                const u32 ob = (u32)row_of_slot(sl, g, j) * 4096 + (ct * 128 + wn * 64) * 4;
                u32x4 o[4];
                activate(o, acc[m], bad);
#pragma unroll
                for (int r = 0; r < 4; ++r) __builtin_amdgcn_raw_buffer_store_b128(o[r], ro, vout + ob + r * 4096, 0, OUT_AUX);
            }
        } else {
            const int fo = (rt * 2 + (ct & 1)) * 8 + wid;
            const u32 sb = par + REL_STG + (u32)(((ct - 8) >> 1) * 1024 + fo * 4) * 4096;
            u32 tp = 0, tn = 0;
#pragma unroll
            for (int m = 0; m < 4; ++m) {
                u32x4 o[4];
                activate(o, acc[m], bad);
                if (m == 0) out_windows(o, tp, tn);
                pack_out(o, tp, tn, (LAS char*)lds + wid * 2048, rs, sb + m * 4096, lane);
            }
            WAIT_VM0();
            if (lane == 0) st_flag(p.ws[bsrc] + OFF_RDY + rank * 8192 + (1024 + fo) * 4, call);
        }
        STAMP(2);
    }
    {
        const bool loc = s >= 24;
        const int rt = ((s >> 2) + 5) % 6, ct = loc ? xcd * 2 + (s & 1) : ((~xcd & 1) * 16) + (xcd >> 1) * 4 + (s & 3);
        Gemm<2> G;
        G.rs = rs; G.ldsw = ldsw; G.wid = wid; G.lane = lane;
        const bool mine = loc || rt * 8 >= RF2;
        G.sa = G.sa2 = par + (mine ? ((loc ? FI : FO1) + wid) * 4096 : REL_A2 + (rt * 8 + wid) * 4096);
        G.ablk = (mine ? RS1 : RF2) * 4096;
        G.sb = par + REL_B + ct * 4 * 4096 + wid * 2048;
        const int bsrc = ct < 16 ? rank : og + ((ct - 16) >> 2);
        u32 cvo = OFF_CNT + 1024 + bsrc * 128, need = (ct < 16 ? 8u : 16u) * call;
        if (mine) { if (lane == 0) { cvo = OFF_CNT + rank * 128; need = 8 * call; } }
        else if (lane < 3 && rt * 8 < OFF2(lane + 1) && rt * 8 + 8 > OFF2(lane)) {
            cvo = OFF_CNT + 2048 + (g * 4 + ((j + 1 + lane) & 3)) * 128; need = F2(lane) * call;
        }
        f32x4 acc[1][4] = {};
        const bool bad = call == (wid == 0 ? G.template run<0>(acc, lds, ws, cvo, need, call)
                                  : wid < 4 ? G.template run<1>(acc, lds, ws, cvo, need, call)
                                            : G.template run<2>(acc, lds, ws, cvo, need, call));
        STAMP(3);
        u32x4 o[4];
        activate(o, acc[0], bad);
        if (ct < 16) {
            const u32 ob = (u32)(loc ? rank * NS + (FI + wid) * 16 : row_of_f2(rt * 8 + wid, g, j)) * 4096 + ct * 256;
#pragma unroll
            for (int r = 0; r < 4; ++r) __builtin_amdgcn_raw_buffer_store_b128(o[r], ro, vout + ob + r * 4096, 0, OUT_AUX);
        } else {
            const int z = ((rt * 4 + (ct & 3)) * 8 + wid + FO2 * 4) % 192;
            const u32 sb = par + REL_STG + (u32)(((ct - 16) >> 2) * 1024 + 768 + z) * 4096;
            u32 tp, tn;
            out_windows(o, tp, tn);
            pack_out(o, tp, tn, (LAS char*)lds + wid * 2048, rs, sb, lane);
            WAIT_VM0();
            if (lane == 0) st_flag(p.ws[bsrc] + OFF_RDY + rank * 8192 + (1024 + 192 + z) * 4, call);
        }
        STAMP(4);
    }
}

__global__ __launch_bounds__(BLOCK, 3) void ag_gemm_kernel(Params p) {
    __shared__ __attribute__((aligned(1024))) char smem[LDS_BYTES];
    const int rank = rfl(p.rank);
    const int lane = threadIdx.x & 63;
    const int wid = rfl(threadIdx.x >> 6);
    const int b = blockIdx.x;
    if (wid >= (b < NCOMM ? NLW + NDEC : 8)) return;
    STAMP(0);
    char* seq = p.ws[rank] + OFF_SEQ + (b * 16 + wid) * 4;
    const u32 prev = ld_agent(seq);
    char* wsl = p.ws[lane & 7];
    auto next_call = [&] {
        const u32 call = rfl(prev) + 1;
        if (lane == 0) __hip_atomic_store((GAS u32*)seq, call, __ATOMIC_RELAXED, __HIP_MEMORY_SCOPE_AGENT);
        STAMP(7);
        return call;
    };
    if (b >= NCOMM) {
        for (int i = ((b - NCOMM) >> 4) - P_G0; i > 0; --i) __builtin_amdgcn_s_sleep(P_GRADE);
        f32x4 v[8];
        load_item(v, p, b - NCOMM, wid, lane);
        const u32 call = next_call();
        convert_local(p, (LAS char*)smem, v, wsl, b - NCOMM, wid, lane, call);
        STAMP(5);
        gemm_role(p, (const LAS char*)smem, b & 7, (b - NCOMM) >> 3, wid, lane, call);
        return;
    }
    const int nld = 2 * NCOMM + b < 7 * WL ? 3 : 2;
    constexpr int NSET = NLW + NDEC - NPW;
    const int sw = (NLW + NDEC - 1 - wid) * NCOMM + b;
    if (wid < 4) *(LAS u32*)(smem + LDS_CTL + (wid * 64 + lane) * 4) = 0;
    asm volatile("s_waitcnt lgkmcnt(0)\n\ts_barrier" ::: "memory");
    if (wid < nld) build_table((LAS char*)smem + LDS_TAB + wid * TAB_BYTES, rank, (wid * NCOMM + b) % 7, lane, 64, TAB_HEAD);
    else if (wid >= NLW && wid < NSET) {
        for (int k = 0; k < nld; ++k)
            build_table((LAS char*)smem + LDS_TAB + k * TAB_BYTES, rank, (k * NCOMM + b) % 7, TAB_HEAD + (wid - NLW) * 64 + lane,
                        (NSET - NLW) * 64, 1 << 30);
        asm volatile("" ::: "memory");
        if (lane == 0) __hip_atomic_fetch_add((LAS u32*)(smem + LDS_TAIL + 12), 1u, __ATOMIC_RELAXED, __HIP_MEMORY_SCOPE_WORKGROUP);
        STAMPB_IF(wid == NLW, 4);
    }
    const u32 call = next_call();
    if (wid >= NSET && sw < NPUSH) push_role(p, wsl, sw, lane, call);
    if (wid >= NLW) decode_role(p, (LAS char*)smem, wid - NLW, lane, call);
    else if (wid < nld) {
        const int ps = rfl((wid * NCOMM + b) % 7), wv = rfl((wid * NCOMM + b) / 7);
        if (ps < 3) load_role<false>(p, (LAS char*)smem, wid, nld, ps, wv, lane, call);
        else        load_role<true>(p, (LAS char*)smem, wid, nld, ps, wv, lane, call);
    }
}

}

constexpr int g_mode = 0;

extern "C" void kernel_launch(int rank, int nranks, ncclComm_t comm, hipStream_t stream,
                              void* const* d_in, const int* in_sizes, int n_in,
                              void* d_out, int out_size, const int* in_dtypes, int out_dtype,
                              void* const* d_out_all, void* const* d_ws_all, size_t ws_bytes) {
    (void)comm; (void)in_sizes; (void)n_in; (void)d_out; (void)out_size; (void)in_dtypes; (void)out_dtype;
    if (nranks != NR || (unsigned)rank >= (unsigned)NR || ws_bytes < WS_NEED) {
        fprintf(stderr, "unsupported configuration\n");
        return;
    }
    Params p;
    p.x = (const float*)d_in[0];
    p.w = (const float*)d_in[1];
    for (int i = 0; i < NR; ++i) { p.ws[i] = (char*)d_ws_all[i]; p.out[i] = (char*)d_out_all[i]; }
    p.rank = rank;
    p.pad = g_mode;
    ag_gemm_kernel<<<GRID, BLOCK, 0, stream>>>(p);
    hipError_t e = hipGetLastError();
    if (e != hipSuccess) fprintf(stderr, "launch failed: %s\n", hipGetErrorString(e));
}
```
